```python
import numpy as np

kernel_name = "gpu_gemm_a2a_m8192_k8192_n8192_bf16_silu"


ABSMAX_RTOL = 0.008

COMPUTE_DTYPE = "bfloat16"
EPILOGUE = "silu"
IN_QUANT = None
OUT_QUANT = None
IN_DTYPES = {"x": "bfloat16", "w_mat": "bfloat16"}
WORLD_SIZE = 8

def _to_bf16(x):
    u = np.ascontiguousarray(x, dtype=np.float32).view(np.uint32)
    return ((u + 0x7FFF + ((u >> 16) & 1)) & 0xFFFF0000).view(np.float32)

def setup_inputs():

    x = _to_bf16(np.random.randn(8192, 8192).astype(np.float32))
    w_mat = _to_bf16((np.random.randn(8192, 8192) / np.sqrt(8192)).astype(np.float32))
    return {"x": x, "w_mat": w_mat}

def _epilogue(y):
    return (y / (1.0 + np.exp(-np.clip(y, -60.0, 60.0)))).astype(np.float32)

def reference(x, w_mat):
    y = (x @ w_mat).astype(np.float32)
    return _epilogue(y)
```

```cpp
#include <hip/hip_runtime.h>
#include <rccl/rccl.h>
#include <cstdio>

#define HIP_CHECK(expr) do { hipError_t _e = (expr); if (_e != hipSuccess) { \
    fprintf(stderr, "HIP error %s at %s:%d\n", hipGetErrorString(_e), __FILE__, __LINE__); \
    return; } } while (0)

#define LAS __attribute__((address_space(3)))
typedef unsigned short bf16_t;
typedef short bf16x8 __attribute__((ext_vector_type(8)));
typedef short bf16x4 __attribute__((ext_vector_type(4)));
typedef float f32x4 __attribute__((ext_vector_type(4)));
typedef float f32x2 __attribute__((ext_vector_type(2)));
typedef unsigned u32x4 __attribute__((ext_vector_type(4)));
typedef __bf16 bf16x2_t __attribute__((ext_vector_type(2)));
typedef __bf16 bfv8 __attribute__((ext_vector_type(8)));

constexpr int NR = 8;
constexpr int M = 1024, N = 8192, K = 8192;
constexpr int NS = N / NR;
constexpr int BK = 64, HTB = 16384;
constexpr int GRID = 128;
constexpr unsigned ARRIVALS = GRID / NR;
constexpr unsigned long long DEADLINE_TICKS = 100000000ull;
constexpr int WS_CALL = 0, WS_TMO = 32, WS_ARR = 64;

struct Args { const bf16_t* X; const bf16_t* W; bf16_t* out[NR]; unsigned* ws[NR]; int rank; };

__device__ __forceinline__ unsigned cvtpk(float lo, float hi) { f32x2 v = {lo, hi}; bf16x2_t b = __builtin_convertvector(v, bf16x2_t); return __builtin_bit_cast(unsigned, b); }
__device__ __forceinline__ float silu(float y) {
  const float c = fminf(fmaxf(y, -60.f), 60.f);
  return y * __builtin_amdgcn_rcpf(1.f + __builtin_amdgcn_exp2f(c * -1.44269504088896340736f));
}

__global__ __launch_bounds__(512, 2) void gemm_a2a(Args a) {
  extern __shared__ __attribute__((aligned(16))) char shm_[];
  LAS char* lds = (LAS char*)shm_;
  const int tid = threadIdx.x, wid = __builtin_amdgcn_readfirstlane(tid >> 6), lane = tid & 63, wr = wid >> 2, wc = wid & 3;
  const int bid = blockIdx.x, xcd = bid & 7, j = bid >> 3;
  const int pm = j & 3, pn = xcd * 4 + (j >> 2);
  const int nt = K / BK;
  __amdgpu_buffer_rsrc_t rA = __builtin_amdgcn_make_buffer_rsrc((void*)a.X, 0, M * K * 2, 0x00020000);
  __amdgpu_buffer_rsrc_t rB = __builtin_amdgcn_make_buffer_rsrc((void*)a.W, 0, K * N * 2, 0x00020000);
  const unsigned voffA = (unsigned)(wid * 8 + (lane >> 3)) * (K * 2) + (unsigned)(((lane & 7) ^ (((wid & 1) << 2) | (lane >> 4))) * 16);
  const unsigned voffB = (unsigned)(wid * 4 + (lane >> 4)) * (N * 2) + (unsigned)(((lane & 15) ^ (2 * ((lane >> 4) | (((wid >> 1) & 1) << 2)))) * 16);
  const unsigned sA0 = (unsigned)(pm * 256) * (K * 2);
  const unsigned sB0 = (unsigned)(pn * 256) * 2;
  const int fr = lane & 15, fq = lane >> 4;
  int aoff[2], boff[2];
#pragma unroll
  for (int k = 0; k < 2; ++k) aoff[k] = (wr * 64 + fr) * 128 + (((4 * k + fq) ^ (fr >> 1)) * 16);
  { const int q = (lane & 15) >> 2, p = lane & 3, g = lane >> 4;
#pragma unroll
    for (int n = 0; n < 2; ++n) boff[n] = (int)(unsigned)(size_t)lds + 4 * HTB + (8 * g + q) * 256 + (((4 * wc + 2 * n + (p >> 1)) ^ (2 * (q | ((g & 1) << 2)))) * 16) + 8 * (p & 1); }
#define SA(b, h) (((b) * 2 + (h)) * HTB)
#define SB(b, h) (((b) * 2 + (h)) * HTB)
#define STAGE_A(b, h, kt) do { _Pragma("unroll") for (int _i = 0; _i < 2; ++_i) \
    __builtin_amdgcn_raw_ptr_buffer_load_lds(rA, (LAS void*)(lds + SA(b, h) + wid * 1024 + _i * 8192), 16, voffA, sA0 + (unsigned)((h) * 128 + _i * 64) * (K * 2) + (unsigned)(kt) * (BK * 2), 0, 0); } while (0)
#define STAGE_B(b, h, kt) do { _Pragma("unroll") for (int _i = 0; _i < 2; ++_i) \
    __builtin_amdgcn_raw_ptr_buffer_load_lds(rB, (LAS void*)(lds + 4 * HTB + SB(b, h) + wid * 1024 + _i * 8192), 16, voffB, sB0 + (unsigned)(h) * 256 + (unsigned)((kt) * BK + _i * 32) * (N * 2), 0, 0); } while (0)
#define LDA(dst, b, h) do { _Pragma("unroll") for (int m = 0; m < 4; ++m) _Pragma("unroll") for (int k = 0; k < 2; ++k) \
    dst[m][k] = *(const LAS bf16x8*)(lds + SA(b, h) + aoff[k] + m * 2048); } while (0)
#define TRRD(dst, base, off) asm volatile("ds_read_b64_tr_b16 %0, %1 offset:%2" : "=v"(dst) : "v"(base), "i"(off))
#define LDB(dst, b, h) do { _Pragma("unroll") for (int n = 0; n < 2; ++n) _Pragma("unroll") for (int k = 0; k < 2; ++k) { \
    TRRD(dst[n][k][0], boff[n], SB(b, h) + k * 8192); TRRD(dst[n][k][1], boff[n], SB(b, h) + k * 8192 + 1024); } } while (0)
#define WAIT_LB(B) asm volatile("s_waitcnt lgkmcnt(0)" : "+v"(B[0][0][0]), "+v"(B[0][0][1]), "+v"(B[0][1][0]), "+v"(B[0][1][1]), "+v"(B[1][0][0]), "+v"(B[1][0][1]), "+v"(B[1][1][0]), "+v"(B[1][1][1]) :: "memory")
#define CAT(x) __builtin_shufflevector(x[0], x[1], 0, 1, 2, 3, 4, 5, 6, 7)
#define MSB __builtin_amdgcn_sched_barrier(0)
#define MMA(ai, bj, At, Bt) do { __builtin_amdgcn_s_setprio(1); _Pragma("unroll") for (int m = 0; m < 4; ++m) _Pragma("unroll") for (int n = 0; n < 2; ++n) _Pragma("unroll") for (int k = 0; k < 2; ++k) { \
    acc[ai][bj][m][n] = __builtin_amdgcn_mfma_f32_16x16x32_bf16(__builtin_bit_cast(bfv8, CAT(Bt[n][k])), __builtin_bit_cast(bfv8, At[m][k]), acc[ai][bj][m][n], 0, 0, 0); MSB; } __builtin_amdgcn_s_setprio(0); } while (0)
#define WAIT_V(n) asm volatile("s_waitcnt vmcnt(" #n ")" ::: "memory")
#define WAIT_L(n) asm volatile("s_waitcnt lgkmcnt(" #n ")" ::: "memory")
#define BAR __builtin_amdgcn_s_barrier()
#define SCHED __builtin_amdgcn_sched_barrier(0)
  f32x4 acc[2][2][4][2];
#pragma unroll
  for (int x = 0; x < 2; ++x)
#pragma unroll
    for (int y = 0; y < 2; ++y)
#pragma unroll
      for (int m = 0; m < 4; ++m)
#pragma unroll
        for (int n = 0; n < 2; ++n) acc[x][y][m][n] = (f32x4){0.f, 0.f, 0.f, 0.f};
  bf16x8 At[4][2]; bf16x4 B0[2][2][2], B1[2][2][2];
  STAGE_B(0, 0, 0); STAGE_A(0, 0, 0); STAGE_B(0, 1, 0); STAGE_A(0, 1, 0);
  if (wr == 1) BAR;
  WAIT_V(4); BAR;
  STAGE_B(1, 0, 1); STAGE_A(1, 0, 1); STAGE_B(1, 1, 1);
  WAIT_V(6); BAR;
  for (int t = 0; t < nt; t += 2) {
    const int t2 = (t + 2 < nt) ? t + 2 : 0, t3 = t2 + 1;
    LDB(B0, 0, 0); SCHED; LDA(At, 0, 0); STAGE_A(1, 1, t + 1);
    WAIT_L(8); BAR; WAIT_LB(B0); MMA(0, 0, At, B0); BAR; SCHED;
    LDB(B1, 0, 1); STAGE_B(0, 0, t2);
    BAR; WAIT_LB(B1); MMA(0, 1, At, B1); BAR;
    LDA(At, 0, 1); STAGE_A(0, 0, t2);
    BAR; WAIT_L(0); MMA(1, 0, At, B0); BAR; SCHED;
    STAGE_B(0, 1, t2);
    WAIT_V(6); BAR; MMA(1, 1, At, B1); BAR;
    LDB(B0, 1, 0); SCHED; LDA(At, 1, 0); STAGE_A(0, 1, t2);
    WAIT_L(8); BAR; WAIT_LB(B0); MMA(0, 0, At, B0); BAR; SCHED;
    LDB(B1, 1, 1); STAGE_B(1, 0, t3);
    BAR; WAIT_LB(B1); MMA(0, 1, At, B1); BAR;
    LDA(At, 1, 1); STAGE_A(1, 0, t3);
    BAR; WAIT_L(0); MMA(1, 0, At, B0); BAR; SCHED;
    STAGE_B(1, 1, t3);
    WAIT_V(6); BAR; MMA(1, 1, At, B1); BAR;
  }
  const unsigned dst = (unsigned)pn >> 2;
  { int tz = threadIdx.x; asm volatile("" : "+v"(tz)); const int ln = tz & 63, fr2 = ln & 15, fq2 = ln >> 4;
    bf16_t* Cb = a.out[dst & (NR - 1)] + (size_t)(a.rank * M + pm * 256 + wr * 64 + fr2) * NS + (pn & 3) * 256 + wc * 32 + ((fq2 & 1) * 16 + (fq2 >> 1) * 8);
#pragma unroll
    for (int ai = 0; ai < 2; ++ai)
#pragma unroll
      for (int m = 0; m < 4; ++m)
#pragma unroll
        for (int bj = 0; bj < 2; ++bj) {
          f32x4 v0 = acc[ai][bj][m][0], v1 = acc[ai][bj][m][1];
          unsigned d0 = cvtpk(silu(v0[0]), silu(v0[1])), d1 = cvtpk(silu(v0[2]), silu(v0[3])), s0 = cvtpk(silu(v1[0]), silu(v1[1])), s1 = cvtpk(silu(v1[2]), silu(v1[3]));
          auto r0 = __builtin_amdgcn_permlane16_swap(d0, s0, false, false);
          auto r1 = __builtin_amdgcn_permlane16_swap(d1, s1, false, false);
          u32x4 w = {r0[0], r1[0], r0[1], r1[1]};
          asm volatile("global_store_dwordx4 %0, %1, off sc0 sc1\n\ts_nop 1" :: "v"(Cb + (size_t)(ai * 128 + m * 16) * NS + bj * 128), "v"(w) : "memory");
        }
  }
  WAIT_V(0);
  if (wr == 0) BAR;
  __syncthreads();
  if (tid == 0 && (int)dst != a.rank) {
    __builtin_amdgcn_fence(__ATOMIC_RELEASE, "");
    asm volatile("s_waitcnt vmcnt(0)" ::: "memory");
    __hip_atomic_fetch_add(a.ws[dst & (NR - 1)] + WS_ARR + 32 * a.rank, 1u, __ATOMIC_RELAXED, __HIP_MEMORY_SCOPE_SYSTEM);
  }
  if (bid == 0 && wid == 0) {
    unsigned* my = a.ws[a.rank];
    unsigned call = 0;
    if (lane == 0) call = __hip_atomic_fetch_add(my + WS_CALL, 1u, __ATOMIC_RELAXED, __HIP_MEMORY_SCOPE_AGENT) + 1u;
    call = __builtin_amdgcn_readfirstlane(call);
    const unsigned want = call * ARRIVALS;
    const unsigned long long t0 = __builtin_amdgcn_s_memrealtime();
    for (;;) {
      unsigned got = want;
      if (lane < NR && lane != a.rank) got = __hip_atomic_load(my + WS_ARR + 32 * lane, __ATOMIC_RELAXED, __HIP_MEMORY_SCOPE_SYSTEM);
      if (__all((int)(got - want) >= 0)) break;
      if (__builtin_amdgcn_s_memrealtime() - t0 > DEADLINE_TICKS) {
        if (lane == 0) { __hip_atomic_store(my + WS_TMO, call, __ATOMIC_RELAXED, __HIP_MEMORY_SCOPE_AGENT); a.out[a.rank][(size_t)a.rank * M * NS] = (bf16_t)0x7fc0; }
        break;
      }
      __builtin_amdgcn_s_sleep(2);
    }
    __builtin_amdgcn_fence(__ATOMIC_ACQUIRE, "");
  }
}

extern "C" void kernel_launch(int rank, int nranks, ncclComm_t comm, hipStream_t stream,
                              void* const* d_in, const int* in_sizes, int n_in,
                              void* d_out, int out_size, const int* in_dtypes, int out_dtype,
                              void* const* d_out_all, void* const* d_ws_all, size_t ws_bytes) {
  if (nranks != NR || (unsigned)rank >= (unsigned)NR || n_in < 2 || in_sizes[0] != M * K || in_sizes[1] != K * N || out_size != NR * M * NS || ws_bytes < 4096) {
    fprintf(stderr, "kernel_launch: unexpected problem shape\n"); return; }
  Args a;
  a.X = (const bf16_t*)d_in[0]; a.W = (const bf16_t*)d_in[1]; a.rank = rank;
  for (int p = 0; p < NR; ++p) { a.out[p] = (bf16_t*)d_out_all[p]; a.ws[p] = (unsigned*)d_ws_all[p]; }
  HIP_CHECK(hipFuncSetAttribute((const void*)gemm_a2a, hipFuncAttributeMaxDynamicSharedMemorySize, 8 * HTB));
  gemm_a2a<<<GRID, 512, 8 * HTB, stream>>>(a);
  HIP_CHECK(hipGetLastError());
}
```

```cpp
#include <hip/hip_runtime.h>
#include <rccl/rccl.h>
#include <cstdio>
#include <cstddef>

#define HIP_CHECK(expr) do { hipError_t _e = (expr); if (_e != hipSuccess) { \
    fprintf(stderr, "HIP error %s at %s:%d\n", hipGetErrorString(_e), __FILE__, __LINE__); \
    return; } } while (0)

#define LAS __attribute__((address_space(3)))
#define GAS __attribute__((address_space(1)))
typedef unsigned short bf16_t;
typedef unsigned u32x4 __attribute__((ext_vector_type(4)));
typedef unsigned u32x8 __attribute__((ext_vector_type(8)));
typedef unsigned u32x16 __attribute__((ext_vector_type(16)));

constexpr int NR = 8;
constexpr int M = 1024, N = 8192, K = 8192;
constexpr int NS = N / NR;
constexpr int BK = 64, HTB = 16384;
constexpr int SLOTS = 10, NSTAGE = 4, REGIONS = 2;
constexpr int GRID = 256;
constexpr unsigned ARRIVALS = 256;
constexpr unsigned long long DEADLINE_TICKS = 50000000ull;
constexpr unsigned WS_TMO = 0;
constexpr unsigned WS_EPOCH = 4096;
constexpr unsigned WS_ARR = 8192;
constexpr unsigned WS_XCC = 20480;
constexpr unsigned WS_FLAG = 262144;
constexpr unsigned WS_X = 1u << 20;
constexpr unsigned XREG = 256u * 65536u;
constexpr unsigned WS_FLAG2 = WS_X + REGIONS * XREG;
constexpr unsigned WS_END = WS_FLAG2 + (8u << 20);

struct Args { const bf16_t* X; const bf16_t* W; char* wsl; int rank, pad; bf16_t* out[NR]; char* ws[NR]; unsigned long long tab[64]; };

#define K_SPLIT {4, 4, 4, 4}
#define K_BASE {0, 16, 32, 48}


__global__ __launch_bounds__(512, 2) void gemm_a2a(Args a) {
  extern __shared__ __attribute__((aligned(16))) char shm_[];
  const int tid = threadIdx.x, wid = __builtin_amdgcn_readfirstlane(tid >> 6);
  int lane = tid & 63;
  const int grp = wid >> 2, w4 = wid & 3, wr = wid >> 1, wc = wid & 1;
  const int bid = blockIdx.x, x = bid & 7, j = bid >> 3, pm = j & 3;
  const int qa = x >> 2, qm = (x >> 1) & 1, qn = x & 1;
  const unsigned long long karg = (unsigned long long)__builtin_amdgcn_kernarg_segment_ptr();
  unsigned long long pX, pW, pS, tabw; unsigned rk;
  asm volatile("s_load_dwordx2 s[28:29], %5, %6\n\ts_load_dwordx2 s[36:37], %5, 0x0\n\ts_load_dwordx2 s[40:41], %5, 0x8\n\ts_load_dwordx2 s[44:45], %5, 0x10\n\ts_load_dword s30, %5, 0x18"
    : "=&{s[28:29]}"(tabw), "=&{s[36:37]}"(pX), "=&{s[40:41]}"(pW), "=&{s[44:45]}"(pS), "=&{s30}"(rk) : "s"(karg), "s"((unsigned)offsetof(Args, tab) + (unsigned)((x & 1) * 32 + j) * 8u) : "memory");
  unsigned call;
  const unsigned xcc = WS_XCC + (__builtin_amdgcn_s_getreg((3 << 11) | 20) & 15u) * 256u;
  const unsigned ldsb = (unsigned)(size_t)(LAS char*)shm_;
  const unsigned ldsA = ldsb + 4 * HTB + grp * 3 * HTB;
  const unsigned long long otab = karg + offsetof(Args, out);
  u32x16 sp;
  const unsigned soffO = (unsigned)(pm * 256 + qa * 128 + wr * 32 + qm * 16) * (NS * 2) + (unsigned)(wc * 128 + (1 ? 0 : qn * 64));
  sp[0] = (unsigned)(pm * 256 + qa * 128 + grp * 64) * (K * 2) + (unsigned)((x & 7) * 16) * (BK * 2);
  sp[1] = WS_EPOCH + bid * 4;
  sp[2] = ldsA + w4 * 1024; sp[3] = ldsb + wid * 1024; sp[4] = wid ^ ((j & 1) << 2); sp[5] = (unsigned)(pm * 16) * (N * 2);
  sp[6] = (unsigned)bid * 65536u + (unsigned)wid * 1024u;
  sp[7] = xcc; sp[8] = 1 ? WS_FLAG2 + (unsigned)(bid * 8 + wid) * 512 : WS_FLAG + (unsigned)(bid * 8 + wid) * 32; sp[9] = soffO; sp[10] = wid; sp[11] = grp;
  sp[12] = (unsigned)otab; sp[13] = (unsigned)(otab >> 32); sp[14] = __float_as_uint(60.f); sp[15] = __float_as_uint(-1.44269504088896340736f);
  const int fr = lane & 15, fq = lane >> 4, q_ = fr >> 2, p_ = lane & 3;
  u32x4 v4; u32x8 v8;
  v4[0] = ldsA + ((wr & 1) * 32 + fr) * 128 + ((fq ^ (fr >> 1)) * 16); v4[1] = ldsA + ((wr & 1) * 32 + fr) * 128 + (((4 + fq) ^ (fr >> 1)) * 16);
#define BOFF(n) (ldsb + (8 * fq + q_) * 256 + (((8 * wc + 2 * (n) + (p_ >> 1)) ^ (2 * (q_ | ((fq & 1) << 2)))) * 16) + 8 * (p_ & 1))
  v4[2] = BOFF(0); v4[3] = BOFF(1); v8[0] = BOFF(2); v8[1] = BOFF(3);
  v8[2] = (unsigned)((w4 ^ ((1 ^ qm) << 1)) * 8 + (lane >> 3)) * (K * 2) + (unsigned)(((lane & 7) ^ (((w4 & 1) << 2) | (lane >> 4))) * 16);
  v8[3] = (unsigned)(wid * 4 + (lane >> 4)) * (N * 2) + (unsigned)((((lane & 15) ^ (2 * ((lane >> 4) | (((wid >> 1) & 1) << 2)))) ^ ((1 ^ qn) << 2)) * 16);
  v8[4] = (unsigned)(lane * 16);
  v8[5] = 1 ? (fq ? (unsigned)(((bid ^ (2 * fq)) * 8 + wid) * 512 + fq * 128 + fr * 8) : 0x40000000u) : lane < 8 ? (unsigned)((((bid ^ lane) * 8 + wid) * 8 + lane) * 4) : 0x40000000u;
  v8[6] = lane < 32 ? (unsigned)(lane >> 1) * (N * 2) + (unsigned)(lane & 1) * 128 : 0x40000000u;
  v8[7] = (unsigned)((1 ? (fr & 7) * (NS * 2) + ((fr >> 3) ^ qn) * 64 : fr * (NS * 2)) + (fq & 1) * 32 + (fq >> 1) * 16);
  asm volatile(
    "s_mov_b32 s38, 16777216\n"
    "s_mov_b32 s39, 0x20000\n"
    "s_mov_b32 s42, 134217728\n"
    "s_mov_b32 s43, 0x20000\n"
    "s_mov_b32 s46, 0x2900000\n"
    "s_mov_b32 s47, 0x20000\n"
    "s_and_b32 s83, s31, 7\n"
    "s_lshl_b32 s83, s83, 24\n"
    "s_waitcnt lgkmcnt(0)\n"
    "s_load_dword s90, s[44:45], s49\n"
    "s_bfe_u32 s82, s28, 0x60000\n"
    "s_lshl_b32 s82, s82, 8\n"
    "s_or_b32 s64, s82, s83\n"
    "s_bfe_u32 s82, s28, 0x60008\n"
    "s_lshl_b32 s82, s82, 8\n"
    "s_or_b32 s65, s82, s83\n"
    "s_bfe_u32 s82, s28, 0x60010\n"
    "s_lshl_b32 s82, s82, 8\n"
    "s_or_b32 s66, s82, s83\n"
    "s_bfe_u32 s82, s28, 0x60018\n"
    "s_lshl_b32 s82, s82, 8\n"
    "s_or_b32 s67, s82, s83\n"
    "s_bfe_u32 s82, s29, 0x60000\n"
    "s_lshl_b32 s82, s82, 8\n"
    "s_or_b32 s68, s82, s83\n"
    "s_add_u32 m0, s51, 0x0\n"
    "s_xor_b32 s82, s64, 0x0\n"
    "buffer_load_dwordx4 v251, s[40:43], s82 offen  lds\n"
    "s_add_u32 m0, s51, 0x2000\n"
    "s_xor_b32 s82, s64, 0x80000\n"
    "buffer_load_dwordx4 v251, s[40:43], s82 offen  lds\n"
    "s_add_u32 m0, s50, 0x0\n"
    "s_xor_b32 s82, s48, 0x200000\n"
    "buffer_load_dwordx4 v250, s[36:39], s82 offen  lds\n"
    "s_add_u32 m0, s50, 0x1000\n"
    "s_xor_b32 s82, s48, 0x280000\n"
    "buffer_load_dwordx4 v250, s[36:39], s82 offen  lds\n"
    "s_add_u32 m0, s50, 0x2000\n"
    "s_xor_b32 s82, s48, 0x0\n"
    "buffer_load_dwordx4 v250, s[36:39], s82 offen  lds\n"
    "s_add_u32 m0, s50, 0x3000\n"
    "s_xor_b32 s82, s48, 0x80000\n"
    "buffer_load_dwordx4 v250, s[36:39], s82 offen  lds\n"
    "s_add_u32 m0, s51, 0x4000\n"
    "s_xor_b32 s82, s64, 0x100000\n"
    "buffer_load_dwordx4 v251, s[40:43], s82 offen  lds\n"
    "s_add_u32 m0, s51, 0x6000\n"
    "s_xor_b32 s82, s64, 0x180000\n"
    "buffer_load_dwordx4 v251, s[40:43], s82 offen  lds\n"
    "s_add_u32 m0, s50, 0x4000\n"
    "s_xor_b32 s82, s48, 0x200080\n"
    "buffer_load_dwordx4 v250, s[36:39], s82 offen  lds\n"
    "s_add_u32 m0, s50, 0x5000\n"
    "s_xor_b32 s82, s48, 0x280080\n"
    "buffer_load_dwordx4 v250, s[36:39], s82 offen  lds\n"
    "s_add_u32 m0, s50, 0x6000\n"
    "s_xor_b32 s82, s48, 0x80\n"
    "buffer_load_dwordx4 v250, s[36:39], s82 offen  lds\n"
    "s_add_u32 m0, s50, 0x7000\n"
    "s_xor_b32 s82, s48, 0x80080\n"
    "buffer_load_dwordx4 v250, s[36:39], s82 offen  lds\n"
    "s_and_b32 s30, s30, 7\n"
    "s_lshl_b32 s82, s30, 21\n"
    "s_add_u32 s57, s57, s82\n"
    "s_lshl_b32 s82, s30, 3\n"
    "s_add_u32 s82, s82, s31\n"
    "s_lshl_b32 s82, s82, 7\n"
    "s_add_u32 s71, s82, 0x2000\n"
    "s_mov_b32 s74, 0x1000000\n"
    "s_mov_b32 s75, 0x20000\n"
    "s_mov_b32 s78, 0x1000000\n"
    "s_mov_b32 s79, 0x20000\n"
    "v_mov_b32 v8, v254\n"
    "v_bfe_u32 v254, v252, 4, 2\n"
    "v_max_u32 v254, 1, v254\n"
    "v_lshlrev_b32 v254, 7, v254\n"
    "v_bfe_u32 v6, v252, 4, 4\n"
    "v_lshrrev_b32 v7, 8, v252\n"
    "v_lshlrev_b32 v7, 2, v7\n"
    "v_sub_u32 v6, v6, v7\n"
    "v_mov_b32 v0, 0\n"
    "v_mov_b32 v1, 0\n"
    "v_mov_b32 v2, 0\n"
    "v_mov_b32 v3, 0\n"
    "v_mov_b32 v4, 0\n"
    "v_mov_b32 v5, 0\n"
    "v_mov_b32 v7, 0x3f80\n"
    "v_cmp_eq_u32 vcc, 0, v6\n"
    "s_nop 1\n"
    "v_cndmask_b32 v2, v2, v7, vcc\n"
    "v_mov_b32 v7, 0x3f800000\n"
    "v_cmp_eq_u32 vcc, 1, v6\n"
    "s_nop 1\n"
    "v_cndmask_b32 v2, v2, v7, vcc\n"
    "v_mov_b32 v7, 0x3f80\n"
    "v_cmp_eq_u32 vcc, 2, v6\n"
    "s_nop 1\n"
    "v_cndmask_b32 v3, v3, v7, vcc\n"
    "v_mov_b32 v7, 0x3f800000\n"
    "v_cmp_eq_u32 vcc, 3, v6\n"
    "s_nop 1\n"
    "v_cndmask_b32 v3, v3, v7, vcc\n"
    "s_waitcnt vmcnt(6) lgkmcnt(0)\n"
    "s_add_u32 s90, s90, 1\n"
    "v_mov_b32 v10, s90\n"
    "v_mov_b32 v11, s90\n"
    "s_barrier\n"
    "s_cmp_eq_u32 s59, 0\n"
    "s_cbranch_scc1 .La1_%=\n"
    "s_barrier\n"
    ".La1_%=:\n"
    "; phase 0: step 0 slot 0 stage 0\n"
    "ds_read_b128 v[180:183], v244 offset:0\n"
    "ds_read_b128 v[184:187], v245 offset:0\n"
    "s_and_b32 s82, s54, 0x71c00\n"
    "s_cbranch_scc1 .La2_%=\n"
    "s_bfe_u32 s83, s55, 0x10008\n"
    "s_lshl1_add_u32 s83, s90, s83\n"
    "v_mov_b32 v7, s83\n"
    "s_lshr_b32 s82, s54, 12\n"
    "s_and_b32 s82, s82, 0xf80\n"
    "s_add_u32 s82, s82, 0x40000\n"
    "s_mov_b64 exec, 1\n"
    "buffer_store_dword v7, off, s[44:47], s82 sc0 sc1\n"
    "s_mov_b64 exec, -1\n"
    ".La2_%=:\n"
    "ds_read_b128 v[188:191], v244 offset:2048\n"
    "ds_read_b128 v[192:195], v245 offset:2048\n"
    "s_add_u32 m0, s51, 0x8000\n"
    "s_xor_b32 s82, s64, 0x200000\n"
    "buffer_load_dwordx4 v251, s[40:43], s82 offen  lds\n"
    "ds_read_b128 v[196:199], v244 offset:8192\n"
    "ds_read_b128 v[200:203], v245 offset:8192\n"
    "s_add_u32 m0, s51, 0xa000\n"
    "s_xor_b32 s82, s64, 0x280000\n"
    "buffer_load_dwordx4 v251, s[40:43], s82 offen  lds\n"
    "ds_read_b128 v[204:207], v244 offset:10240\n"
    "ds_read_b128 v[208:211], v245 offset:10240\n"
    "s_add_u32 m0, s50, 0x8000\n"
    "s_xor_b32 s82, s48, 0x200100\n"
    "buffer_load_dwordx4 v250, s[36:39], s82 offen  lds\n"
    "ds_read_b64_tr_b16 v[212:213], v246 offset:0\n"
    "ds_read_b64_tr_b16 v[214:215], v246 offset:1024\n"
    "s_add_u32 m0, s50, 0x9000\n"
    "s_xor_b32 s82, s48, 0x280100\n"
    "buffer_load_dwordx4 v250, s[36:39], s82 offen  lds\n"
    "ds_read_b64_tr_b16 v[216:217], v246 offset:8192\n"
    "ds_read_b64_tr_b16 v[218:219], v246 offset:9216\n"
    "s_add_u32 m0, s50, 0xa000\n"
    "s_xor_b32 s82, s48, 0x100\n"
    "buffer_load_dwordx4 v250, s[36:39], s82 offen  lds\n"
    "ds_read_b64_tr_b16 v[220:221], v247 offset:0\n"
    "ds_read_b64_tr_b16 v[222:223], v247 offset:1024\n"
    "s_add_u32 m0, s50, 0xb000\n"
    "s_xor_b32 s82, s48, 0x80100\n"
    "buffer_load_dwordx4 v250, s[36:39], s82 offen  lds\n"
    "ds_read_b64_tr_b16 v[224:225], v247 offset:8192\n"
    "ds_read_b64_tr_b16 v[226:227], v247 offset:9216\n"
    "s_add_u32 m0, s51, 0xc000\n"
    "s_xor_b32 s82, s64, 0x300000\n"
    "buffer_load_dwordx4 v251, s[40:43], s82 offen  lds\n"
    "ds_read_b64_tr_b16 v[228:229], v248 offset:0\n"
    "ds_read_b64_tr_b16 v[230:231], v248 offset:1024\n"
    "s_add_u32 m0, s51, 0xe000\n"
    "s_xor_b32 s82, s64, 0x380000\n"
    "buffer_load_dwordx4 v251, s[40:43], s82 offen  lds\n"
    "ds_read_b64_tr_b16 v[232:233], v248 offset:8192\n"
    "ds_read_b64_tr_b16 v[234:235], v248 offset:9216\n"
    "ds_read_b64_tr_b16 v[236:237], v249 offset:0\n"
    "ds_read_b64_tr_b16 v[238:239], v249 offset:1024\n"
    "ds_read_b64_tr_b16 v[240:241], v249 offset:8192\n"
    "ds_read_b64_tr_b16 v[242:243], v249 offset:9216\n"
    "s_waitcnt vmcnt(12) lgkmcnt(0)\n"
    "s_barrier\n"
    "s_setprio 1\n"
    "v_mfma_f32_16x16x32_bf16 v[12:15], v[212:215], v[180:183], 0\n"
    "v_mfma_f32_16x16x32_bf16 v[12:15], v[216:219], v[184:187], v[12:15]\n"
    "v_mfma_f32_16x16x32_bf16 v[16:19], v[220:223], v[180:183], 0\n"
    "v_mfma_f32_16x16x32_bf16 v[16:19], v[224:227], v[184:187], v[16:19]\n"
    "v_mfma_f32_16x16x32_bf16 v[20:23], v[228:231], v[180:183], 0\n"
    "v_mfma_f32_16x16x32_bf16 v[20:23], v[232:235], v[184:187], v[20:23]\n"
    "v_mfma_f32_16x16x32_bf16 v[24:27], v[236:239], v[180:183], 0\n"
    "v_mfma_f32_16x16x32_bf16 v[24:27], v[240:243], v[184:187], v[24:27]\n"
    "v_mfma_f32_16x16x32_bf16 v[28:31], v[212:215], v[188:191], 0\n"
    "v_mfma_f32_16x16x32_bf16 v[28:31], v[216:219], v[192:195], v[28:31]\n"
    "v_mfma_f32_16x16x32_bf16 v[32:35], v[220:223], v[188:191], 0\n"
    "v_mfma_f32_16x16x32_bf16 v[32:35], v[224:227], v[192:195], v[32:35]\n"
    "v_mfma_f32_16x16x32_bf16 v[36:39], v[228:231], v[188:191], 0\n"
    "v_mfma_f32_16x16x32_bf16 v[36:39], v[232:235], v[192:195], v[36:39]\n"
    "v_mfma_f32_16x16x32_bf16 v[40:43], v[236:239], v[188:191], 0\n"
    "v_mfma_f32_16x16x32_bf16 v[40:43], v[240:243], v[192:195], v[40:43]\n"
    "v_mfma_f32_16x16x32_bf16 v[44:47], v[212:215], v[196:199], 0\n"
    "v_mfma_f32_16x16x32_bf16 v[44:47], v[216:219], v[200:203], v[44:47]\n"
    "v_mfma_f32_16x16x32_bf16 v[48:51], v[220:223], v[196:199], 0\n"
    "v_mfma_f32_16x16x32_bf16 v[48:51], v[224:227], v[200:203], v[48:51]\n"
    "v_mfma_f32_16x16x32_bf16 v[52:55], v[228:231], v[196:199], 0\n"
    "v_mfma_f32_16x16x32_bf16 v[52:55], v[232:235], v[200:203], v[52:55]\n"
    "v_mfma_f32_16x16x32_bf16 v[56:59], v[236:239], v[196:199], 0\n"
    "v_mfma_f32_16x16x32_bf16 v[56:59], v[240:243], v[200:203], v[56:59]\n"
    "v_mfma_f32_16x16x32_bf16 v[60:63], v[212:215], v[204:207], 0\n"
    "v_mfma_f32_16x16x32_bf16 v[60:63], v[216:219], v[208:211], v[60:63]\n"
    "v_mfma_f32_16x16x32_bf16 v[64:67], v[220:223], v[204:207], 0\n"
    "v_mfma_f32_16x16x32_bf16 v[64:67], v[224:227], v[208:211], v[64:67]\n"
    "v_mfma_f32_16x16x32_bf16 v[68:71], v[228:231], v[204:207], 0\n"
    "v_mfma_f32_16x16x32_bf16 v[68:71], v[232:235], v[208:211], v[68:71]\n"
    "v_mfma_f32_16x16x32_bf16 v[72:75], v[236:239], v[204:207], 0\n"
    "v_mfma_f32_16x16x32_bf16 v[72:75], v[240:243], v[208:211], v[72:75]\n"
    "s_setprio 0\n"
    "s_waitcnt vmcnt(8)\n"
    "s_barrier\n"
    "; phase 1: step 1 slot 0 stage 0\n"
    "ds_read_b128 v[180:183], v244 offset:16384\n"
    "ds_read_b128 v[184:187], v245 offset:16384\n"
    "ds_read_b128 v[188:191], v244 offset:18432\n"
    "ds_read_b128 v[192:195], v245 offset:18432\n"
    "s_add_u32 m0, s50, 0x0\n"
    "s_xor_b32 s82, s48, 0x200180\n"
    "buffer_load_dwordx4 v250, s[36:39], s82 offen  lds\n"
    "ds_read_b128 v[196:199], v244 offset:24576\n"
    "ds_read_b128 v[200:203], v245 offset:24576\n"
    "ds_read_b128 v[204:207], v244 offset:26624\n"
    "ds_read_b128 v[208:211], v245 offset:26624\n"
    "s_add_u32 m0, s50, 0x1000\n"
    "s_xor_b32 s82, s48, 0x280180\n"
    "buffer_load_dwordx4 v250, s[36:39], s82 offen  lds\n"
    "ds_read_b64_tr_b16 v[212:213], v246 offset:16384\n"
    "ds_read_b64_tr_b16 v[214:215], v246 offset:17408\n"
    "ds_read_b64_tr_b16 v[216:217], v246 offset:24576\n"
    "ds_read_b64_tr_b16 v[218:219], v246 offset:25600\n"
    "s_add_u32 m0, s50, 0x2000\n"
    "s_xor_b32 s82, s48, 0x180\n"
    "buffer_load_dwordx4 v250, s[36:39], s82 offen  lds\n"
    "ds_read_b64_tr_b16 v[220:221], v247 offset:16384\n"
    "ds_read_b64_tr_b16 v[222:223], v247 offset:17408\n"
    "ds_read_b64_tr_b16 v[224:225], v247 offset:24576\n"
    "ds_read_b64_tr_b16 v[226:227], v247 offset:25600\n"
    "s_add_u32 m0, s50, 0x3000\n"
    "s_xor_b32 s82, s48, 0x80180\n"
    "buffer_load_dwordx4 v250, s[36:39], s82 offen  lds\n"
    "ds_read_b64_tr_b16 v[228:229], v248 offset:16384\n"
    "ds_read_b64_tr_b16 v[230:231], v248 offset:17408\n"
    "ds_read_b64_tr_b16 v[232:233], v248 offset:24576\n"
    "ds_read_b64_tr_b16 v[234:235], v248 offset:25600\n"
    "s_add_u32 m0, s51, 0x0\n"
    "s_xor_b32 s82, s64, 0x400000\n"
    "buffer_load_dwordx4 v251, s[40:43], s82 offen  lds\n"
    "ds_read_b64_tr_b16 v[236:237], v249 offset:16384\n"
    "ds_read_b64_tr_b16 v[238:239], v249 offset:17408\n"
    "ds_read_b64_tr_b16 v[240:241], v249 offset:24576\n"
    "ds_read_b64_tr_b16 v[242:243], v249 offset:25600\n"
    "s_add_u32 m0, s51, 0x2000\n"
    "s_xor_b32 s82, s64, 0x480000\n"
    "buffer_load_dwordx4 v251, s[40:43], s82 offen  lds\n"
    "s_waitcnt vmcnt(12) lgkmcnt(0)\n"
    "s_barrier\n"
    "s_setprio 1\n"
    "v_mfma_f32_16x16x32_bf16 v[12:15], v[212:215], v[180:183], v[12:15]\n"
    "v_mfma_f32_16x16x32_bf16 v[12:15], v[216:219], v[184:187], v[12:15]\n"
    "v_mfma_f32_16x16x32_bf16 v[16:19], v[220:223], v[180:183], v[16:19]\n"
    "v_mfma_f32_16x16x32_bf16 v[16:19], v[224:227], v[184:187], v[16:19]\n"
    "v_mfma_f32_16x16x32_bf16 v[20:23], v[228:231], v[180:183], v[20:23]\n"
    "v_mfma_f32_16x16x32_bf16 v[20:23], v[232:235], v[184:187], v[20:23]\n"
    "v_mfma_f32_16x16x32_bf16 v[24:27], v[236:239], v[180:183], v[24:27]\n"
    "v_mfma_f32_16x16x32_bf16 v[24:27], v[240:243], v[184:187], v[24:27]\n"
    "v_mfma_f32_16x16x32_bf16 v[28:31], v[212:215], v[188:191], v[28:31]\n"
    "v_mfma_f32_16x16x32_bf16 v[28:31], v[216:219], v[192:195], v[28:31]\n"
    "v_mfma_f32_16x16x32_bf16 v[32:35], v[220:223], v[188:191], v[32:35]\n"
    "v_mfma_f32_16x16x32_bf16 v[32:35], v[224:227], v[192:195], v[32:35]\n"
    "v_mfma_f32_16x16x32_bf16 v[36:39], v[228:231], v[188:191], v[36:39]\n"
    "v_mfma_f32_16x16x32_bf16 v[36:39], v[232:235], v[192:195], v[36:39]\n"
    "v_mfma_f32_16x16x32_bf16 v[40:43], v[236:239], v[188:191], v[40:43]\n"
    "v_mfma_f32_16x16x32_bf16 v[40:43], v[240:243], v[192:195], v[40:43]\n"
    "v_mfma_f32_16x16x32_bf16 v[44:47], v[212:215], v[196:199], v[44:47]\n"
    "v_mfma_f32_16x16x32_bf16 v[44:47], v[216:219], v[200:203], v[44:47]\n"
    "v_mfma_f32_16x16x32_bf16 v[48:51], v[220:223], v[196:199], v[48:51]\n"
    "v_mfma_f32_16x16x32_bf16 v[48:51], v[224:227], v[200:203], v[48:51]\n"
    "v_mfma_f32_16x16x32_bf16 v[52:55], v[228:231], v[196:199], v[52:55]\n"
    "v_mfma_f32_16x16x32_bf16 v[52:55], v[232:235], v[200:203], v[52:55]\n"
    "v_mfma_f32_16x16x32_bf16 v[56:59], v[236:239], v[196:199], v[56:59]\n"
    "v_mfma_f32_16x16x32_bf16 v[56:59], v[240:243], v[200:203], v[56:59]\n"
    "v_mfma_f32_16x16x32_bf16 v[60:63], v[212:215], v[204:207], v[60:63]\n"
    "v_mfma_f32_16x16x32_bf16 v[60:63], v[216:219], v[208:211], v[60:63]\n"
    "v_mfma_f32_16x16x32_bf16 v[64:67], v[220:223], v[204:207], v[64:67]\n"
    "v_mfma_f32_16x16x32_bf16 v[64:67], v[224:227], v[208:211], v[64:67]\n"
    "v_mfma_f32_16x16x32_bf16 v[68:71], v[228:231], v[204:207], v[68:71]\n"
    "v_mfma_f32_16x16x32_bf16 v[68:71], v[232:235], v[208:211], v[68:71]\n"
    "v_mfma_f32_16x16x32_bf16 v[72:75], v[236:239], v[204:207], v[72:75]\n"
    "v_mfma_f32_16x16x32_bf16 v[72:75], v[240:243], v[208:211], v[72:75]\n"
    "s_setprio 0\n"
    "s_waitcnt vmcnt(8)\n"
    "s_barrier\n"
    "; phase 2: step 2 slot 0 stage 0\n"
    "ds_read_b128 v[180:183], v244 offset:32768\n"
    "ds_read_b128 v[184:187], v245 offset:32768\n"
    "ds_read_b128 v[188:191], v244 offset:34816\n"
    "ds_read_b128 v[192:195], v245 offset:34816\n"
    "s_add_u32 m0, s50, 0x4000\n"
    "s_xor_b32 s82, s48, 0x200200\n"
    "buffer_load_dwordx4 v250, s[36:39], s82 offen  lds\n"
    "ds_read_b128 v[196:199], v244 offset:40960\n"
    "ds_read_b128 v[200:203], v245 offset:40960\n"
    "ds_read_b128 v[204:207], v244 offset:43008\n"
    "ds_read_b128 v[208:211], v245 offset:43008\n"
    "s_add_u32 m0, s50, 0x5000\n"
    "s_xor_b32 s82, s48, 0x280200\n"
    "buffer_load_dwordx4 v250, s[36:39], s82 offen  lds\n"
    "ds_read_b64_tr_b16 v[212:213], v246 offset:32768\n"
    "ds_read_b64_tr_b16 v[214:215], v246 offset:33792\n"
    "ds_read_b64_tr_b16 v[216:217], v246 offset:40960\n"
    "ds_read_b64_tr_b16 v[218:219], v246 offset:41984\n"
    "s_add_u32 m0, s50, 0x6000\n"
    "s_xor_b32 s82, s48, 0x200\n"
    "buffer_load_dwordx4 v250, s[36:39], s82 offen  lds\n"
    "ds_read_b64_tr_b16 v[220:221], v247 offset:32768\n"
    "ds_read_b64_tr_b16 v[222:223], v247 offset:33792\n"
    "ds_read_b64_tr_b16 v[224:225], v247 offset:40960\n"
    "ds_read_b64_tr_b16 v[226:227], v247 offset:41984\n"
    "s_add_u32 m0, s50, 0x7000\n"
    "s_xor_b32 s82, s48, 0x80200\n"
    "buffer_load_dwordx4 v250, s[36:39], s82 offen  lds\n"
    "ds_read_b64_tr_b16 v[228:229], v248 offset:32768\n"
    "ds_read_b64_tr_b16 v[230:231], v248 offset:33792\n"
    "ds_read_b64_tr_b16 v[232:233], v248 offset:40960\n"
    "ds_read_b64_tr_b16 v[234:235], v248 offset:41984\n"
    "s_add_u32 m0, s51, 0x4000\n"
    "s_xor_b32 s82, s64, 0x500000\n"
    "buffer_load_dwordx4 v251, s[40:43], s82 offen  lds\n"
    "ds_read_b64_tr_b16 v[236:237], v249 offset:32768\n"
    "ds_read_b64_tr_b16 v[238:239], v249 offset:33792\n"
    "ds_read_b64_tr_b16 v[240:241], v249 offset:40960\n"
    "ds_read_b64_tr_b16 v[242:243], v249 offset:41984\n"
    "s_add_u32 m0, s51, 0x6000\n"
    "s_xor_b32 s82, s64, 0x580000\n"
    "buffer_load_dwordx4 v251, s[40:43], s82 offen  lds\n"
    "s_waitcnt vmcnt(12) lgkmcnt(0)\n"
    "s_barrier\n"
    "s_setprio 1\n"
    "v_mfma_f32_16x16x32_bf16 v[12:15], v[212:215], v[180:183], v[12:15]\n"
    "v_mfma_f32_16x16x32_bf16 v[12:15], v[216:219], v[184:187], v[12:15]\n"
    "v_mfma_f32_16x16x32_bf16 v[16:19], v[220:223], v[180:183], v[16:19]\n"
    "v_mfma_f32_16x16x32_bf16 v[16:19], v[224:227], v[184:187], v[16:19]\n"
    "v_mfma_f32_16x16x32_bf16 v[20:23], v[228:231], v[180:183], v[20:23]\n"
    "v_mfma_f32_16x16x32_bf16 v[20:23], v[232:235], v[184:187], v[20:23]\n"
    "v_mfma_f32_16x16x32_bf16 v[24:27], v[236:239], v[180:183], v[24:27]\n"
    "v_mfma_f32_16x16x32_bf16 v[24:27], v[240:243], v[184:187], v[24:27]\n"
    "v_mfma_f32_16x16x32_bf16 v[28:31], v[212:215], v[188:191], v[28:31]\n"
    "v_mfma_f32_16x16x32_bf16 v[28:31], v[216:219], v[192:195], v[28:31]\n"
    "v_mfma_f32_16x16x32_bf16 v[32:35], v[220:223], v[188:191], v[32:35]\n"
    "v_mfma_f32_16x16x32_bf16 v[32:35], v[224:227], v[192:195], v[32:35]\n"
    "v_mfma_f32_16x16x32_bf16 v[36:39], v[228:231], v[188:191], v[36:39]\n"
    "v_mfma_f32_16x16x32_bf16 v[36:39], v[232:235], v[192:195], v[36:39]\n"
    "v_mfma_f32_16x16x32_bf16 v[40:43], v[236:239], v[188:191], v[40:43]\n"
    "v_mfma_f32_16x16x32_bf16 v[40:43], v[240:243], v[192:195], v[40:43]\n"
    "v_mfma_f32_16x16x32_bf16 v[44:47], v[212:215], v[196:199], v[44:47]\n"
    "v_mfma_f32_16x16x32_bf16 v[44:47], v[216:219], v[200:203], v[44:47]\n"
    "v_mfma_f32_16x16x32_bf16 v[48:51], v[220:223], v[196:199], v[48:51]\n"
    "v_mfma_f32_16x16x32_bf16 v[48:51], v[224:227], v[200:203], v[48:51]\n"
    "v_mfma_f32_16x16x32_bf16 v[52:55], v[228:231], v[196:199], v[52:55]\n"
    "v_mfma_f32_16x16x32_bf16 v[52:55], v[232:235], v[200:203], v[52:55]\n"
    "v_mfma_f32_16x16x32_bf16 v[56:59], v[236:239], v[196:199], v[56:59]\n"
    "v_mfma_f32_16x16x32_bf16 v[56:59], v[240:243], v[200:203], v[56:59]\n"
    "v_mfma_f32_16x16x32_bf16 v[60:63], v[212:215], v[204:207], v[60:63]\n"
    "v_mfma_f32_16x16x32_bf16 v[60:63], v[216:219], v[208:211], v[60:63]\n"
    "v_mfma_f32_16x16x32_bf16 v[64:67], v[220:223], v[204:207], v[64:67]\n"
    "v_mfma_f32_16x16x32_bf16 v[64:67], v[224:227], v[208:211], v[64:67]\n"
    "v_mfma_f32_16x16x32_bf16 v[68:71], v[228:231], v[204:207], v[68:71]\n"
    "v_mfma_f32_16x16x32_bf16 v[68:71], v[232:235], v[208:211], v[68:71]\n"
    "v_mfma_f32_16x16x32_bf16 v[72:75], v[236:239], v[204:207], v[72:75]\n"
    "v_mfma_f32_16x16x32_bf16 v[72:75], v[240:243], v[208:211], v[72:75]\n"
    "s_setprio 0\n"
    "s_waitcnt vmcnt(8)\n"
    "s_barrier\n"
    "; phase 3: step 3 slot 0 stage 0\n"
    "ds_read_b128 v[180:183], v244 offset:0\n"
    "ds_read_b128 v[184:187], v245 offset:0\n"
    "ds_read_b128 v[188:191], v244 offset:2048\n"
    "ds_read_b128 v[192:195], v245 offset:2048\n"
    "s_add_u32 m0, s50, 0x8000\n"
    "s_xor_b32 s82, s48, 0x200280\n"
    "buffer_load_dwordx4 v250, s[36:39], s82 offen  lds\n"
    "ds_read_b128 v[196:199], v244 offset:8192\n"
    "ds_read_b128 v[200:203], v245 offset:8192\n"
    "ds_read_b128 v[204:207], v244 offset:10240\n"
    "ds_read_b128 v[208:211], v245 offset:10240\n"
    "s_add_u32 m0, s50, 0x9000\n"
    "s_xor_b32 s82, s48, 0x280280\n"
    "buffer_load_dwordx4 v250, s[36:39], s82 offen  lds\n"
    "ds_read_b64_tr_b16 v[212:213], v246 offset:49152\n"
    "ds_read_b64_tr_b16 v[214:215], v246 offset:50176\n"
    "ds_read_b64_tr_b16 v[216:217], v246 offset:57344\n"
    "ds_read_b64_tr_b16 v[218:219], v246 offset:58368\n"
    "s_add_u32 m0, s50, 0xa000\n"
    "s_xor_b32 s82, s48, 0x280\n"
    "buffer_load_dwordx4 v250, s[36:39], s82 offen  lds\n"
    "ds_read_b64_tr_b16 v[220:221], v247 offset:49152\n"
    "ds_read_b64_tr_b16 v[222:223], v247 offset:50176\n"
    "ds_read_b64_tr_b16 v[224:225], v247 offset:57344\n"
    "ds_read_b64_tr_b16 v[226:227], v247 offset:58368\n"
    "s_add_u32 m0, s50, 0xb000\n"
    "s_xor_b32 s82, s48, 0x80280\n"
    "buffer_load_dwordx4 v250, s[36:39], s82 offen  lds\n"
    "ds_read_b64_tr_b16 v[228:229], v248 offset:49152\n"
    "ds_read_b64_tr_b16 v[230:231], v248 offset:50176\n"
    "ds_read_b64_tr_b16 v[232:233], v248 offset:57344\n"
    "ds_read_b64_tr_b16 v[234:235], v248 offset:58368\n"
    "s_add_u32 m0, s51, 0x8000\n"
    "s_xor_b32 s82, s64, 0x600000\n"
    "buffer_load_dwordx4 v251, s[40:43], s82 offen  lds\n"
    "ds_read_b64_tr_b16 v[236:237], v249 offset:49152\n"
    "ds_read_b64_tr_b16 v[238:239], v249 offset:50176\n"
    "ds_read_b64_tr_b16 v[240:241], v249 offset:57344\n"
    "ds_read_b64_tr_b16 v[242:243], v249 offset:58368\n"
    "s_add_u32 m0, s51, 0xa000\n"
    "s_xor_b32 s82, s64, 0x680000\n"
    "buffer_load_dwordx4 v251, s[40:43], s82 offen  lds\n"
    "s_waitcnt vmcnt(12) lgkmcnt(0)\n"
    "s_barrier\n"
    "s_setprio 1\n"
    "v_mfma_f32_16x16x32_bf16 v[12:15], v[212:215], v[180:183], v[12:15]\n"
    "v_mfma_f32_16x16x32_bf16 v[12:15], v[216:219], v[184:187], v[12:15]\n"
    "v_mfma_f32_16x16x32_bf16 v[16:19], v[220:223], v[180:183], v[16:19]\n"
    "v_mfma_f32_16x16x32_bf16 v[16:19], v[224:227], v[184:187], v[16:19]\n"
    "v_mfma_f32_16x16x32_bf16 v[20:23], v[228:231], v[180:183], v[20:23]\n"
    "v_mfma_f32_16x16x32_bf16 v[20:23], v[232:235], v[184:187], v[20:23]\n"
    "v_mfma_f32_16x16x32_bf16 v[24:27], v[236:239], v[180:183], v[24:27]\n"
    "v_mfma_f32_16x16x32_bf16 v[24:27], v[240:243], v[184:187], v[24:27]\n"
    "v_mfma_f32_16x16x32_bf16 v[28:31], v[212:215], v[188:191], v[28:31]\n"
    "v_mfma_f32_16x16x32_bf16 v[28:31], v[216:219], v[192:195], v[28:31]\n"
    "v_mfma_f32_16x16x32_bf16 v[32:35], v[220:223], v[188:191], v[32:35]\n"
    "v_mfma_f32_16x16x32_bf16 v[32:35], v[224:227], v[192:195], v[32:35]\n"
    "v_mfma_f32_16x16x32_bf16 v[36:39], v[228:231], v[188:191], v[36:39]\n"
    "v_mfma_f32_16x16x32_bf16 v[36:39], v[232:235], v[192:195], v[36:39]\n"
    "v_mfma_f32_16x16x32_bf16 v[40:43], v[236:239], v[188:191], v[40:43]\n"
    "v_mfma_f32_16x16x32_bf16 v[40:43], v[240:243], v[192:195], v[40:43]\n"
    "v_mfma_f32_16x16x32_bf16 v[44:47], v[212:215], v[196:199], v[44:47]\n"
    "v_mfma_f32_16x16x32_bf16 v[44:47], v[216:219], v[200:203], v[44:47]\n"
    "v_mfma_f32_16x16x32_bf16 v[48:51], v[220:223], v[196:199], v[48:51]\n"
    "v_mfma_f32_16x16x32_bf16 v[48:51], v[224:227], v[200:203], v[48:51]\n"
    "v_mfma_f32_16x16x32_bf16 v[52:55], v[228:231], v[196:199], v[52:55]\n"
    "v_mfma_f32_16x16x32_bf16 v[52:55], v[232:235], v[200:203], v[52:55]\n"
    "v_mfma_f32_16x16x32_bf16 v[56:59], v[236:239], v[196:199], v[56:59]\n"
    "v_mfma_f32_16x16x32_bf16 v[56:59], v[240:243], v[200:203], v[56:59]\n"
    "v_mfma_f32_16x16x32_bf16 v[60:63], v[212:215], v[204:207], v[60:63]\n"
    "v_mfma_f32_16x16x32_bf16 v[60:63], v[216:219], v[208:211], v[60:63]\n"
    "v_mfma_f32_16x16x32_bf16 v[64:67], v[220:223], v[204:207], v[64:67]\n"
    "v_mfma_f32_16x16x32_bf16 v[64:67], v[224:227], v[208:211], v[64:67]\n"
    "v_mfma_f32_16x16x32_bf16 v[68:71], v[228:231], v[204:207], v[68:71]\n"
    "v_mfma_f32_16x16x32_bf16 v[68:71], v[232:235], v[208:211], v[68:71]\n"
    "v_mfma_f32_16x16x32_bf16 v[72:75], v[236:239], v[204:207], v[72:75]\n"
    "v_mfma_f32_16x16x32_bf16 v[72:75], v[240:243], v[208:211], v[72:75]\n"
    "s_setprio 0\n"
    "s_waitcnt vmcnt(8)\n"
    "s_barrier\n"
    "; phase 4: step 4 slot 0 stage 0\n"
    "ds_read_b128 v[180:183], v244 offset:16384\n"
    "ds_read_b128 v[184:187], v245 offset:16384\n"
    "ds_read_b128 v[188:191], v244 offset:18432\n"
    "ds_read_b128 v[192:195], v245 offset:18432\n"
    "s_add_u32 m0, s50, 0x0\n"
    "s_xor_b32 s82, s48, 0x200300\n"
    "buffer_load_dwordx4 v250, s[36:39], s82 offen  lds\n"
    "ds_read_b128 v[196:199], v244 offset:24576\n"
    "ds_read_b128 v[200:203], v245 offset:24576\n"
    "ds_read_b128 v[204:207], v244 offset:26624\n"
    "ds_read_b128 v[208:211], v245 offset:26624\n"
    "s_add_u32 m0, s50, 0x1000\n"
    "s_xor_b32 s82, s48, 0x280300\n"
    "buffer_load_dwordx4 v250, s[36:39], s82 offen  lds\n"
    "ds_read_b64_tr_b16 v[212:213], v246 offset:0\n"
    "ds_read_b64_tr_b16 v[214:215], v246 offset:1024\n"
    "ds_read_b64_tr_b16 v[216:217], v246 offset:8192\n"
    "ds_read_b64_tr_b16 v[218:219], v246 offset:9216\n"
    "s_add_u32 m0, s50, 0x2000\n"
    "s_xor_b32 s82, s48, 0x300\n"
    "buffer_load_dwordx4 v250, s[36:39], s82 offen  lds\n"
    "ds_read_b64_tr_b16 v[220:221], v247 offset:0\n"
    "ds_read_b64_tr_b16 v[222:223], v247 offset:1024\n"
    "ds_read_b64_tr_b16 v[224:225], v247 offset:8192\n"
    "ds_read_b64_tr_b16 v[226:227], v247 offset:9216\n"
    "s_add_u32 m0, s50, 0x3000\n"
    "s_xor_b32 s82, s48, 0x80300\n"
    "buffer_load_dwordx4 v250, s[36:39], s82 offen  lds\n"
    "ds_read_b64_tr_b16 v[228:229], v248 offset:0\n"
    "ds_read_b64_tr_b16 v[230:231], v248 offset:1024\n"
    "ds_read_b64_tr_b16 v[232:233], v248 offset:8192\n"
    "ds_read_b64_tr_b16 v[234:235], v248 offset:9216\n"
    "s_add_u32 m0, s51, 0xc000\n"
    "s_xor_b32 s82, s64, 0x700000\n"
    "buffer_load_dwordx4 v251, s[40:43], s82 offen  lds\n"
    "ds_read_b64_tr_b16 v[236:237], v249 offset:0\n"
    "ds_read_b64_tr_b16 v[238:239], v249 offset:1024\n"
    "ds_read_b64_tr_b16 v[240:241], v249 offset:8192\n"
    "ds_read_b64_tr_b16 v[242:243], v249 offset:9216\n"
    "s_add_u32 m0, s51, 0xe000\n"
    "s_xor_b32 s82, s64, 0x780000\n"
    "buffer_load_dwordx4 v251, s[40:43], s82 offen  lds\n"
    "s_waitcnt vmcnt(12) lgkmcnt(0)\n"
    "s_barrier\n"
    "s_setprio 1\n"
    "v_mfma_f32_16x16x32_bf16 v[12:15], v[212:215], v[180:183], v[12:15]\n"
    "v_mfma_f32_16x16x32_bf16 v[12:15], v[216:219], v[184:187], v[12:15]\n"
    "v_mfma_f32_16x16x32_bf16 v[16:19], v[220:223], v[180:183], v[16:19]\n"
    "v_mfma_f32_16x16x32_bf16 v[16:19], v[224:227], v[184:187], v[16:19]\n"
    "v_mfma_f32_16x16x32_bf16 v[20:23], v[228:231], v[180:183], v[20:23]\n"
    "v_mfma_f32_16x16x32_bf16 v[20:23], v[232:235], v[184:187], v[20:23]\n"
    "v_mfma_f32_16x16x32_bf16 v[24:27], v[236:239], v[180:183], v[24:27]\n"
    "v_mfma_f32_16x16x32_bf16 v[24:27], v[240:243], v[184:187], v[24:27]\n"
    "v_mfma_f32_16x16x32_bf16 v[28:31], v[212:215], v[188:191], v[28:31]\n"
    "v_mfma_f32_16x16x32_bf16 v[28:31], v[216:219], v[192:195], v[28:31]\n"
    "v_mfma_f32_16x16x32_bf16 v[32:35], v[220:223], v[188:191], v[32:35]\n"
    "v_mfma_f32_16x16x32_bf16 v[32:35], v[224:227], v[192:195], v[32:35]\n"
    "v_mfma_f32_16x16x32_bf16 v[36:39], v[228:231], v[188:191], v[36:39]\n"
    "v_mfma_f32_16x16x32_bf16 v[36:39], v[232:235], v[192:195], v[36:39]\n"
    "v_mfma_f32_16x16x32_bf16 v[40:43], v[236:239], v[188:191], v[40:43]\n"
    "v_mfma_f32_16x16x32_bf16 v[40:43], v[240:243], v[192:195], v[40:43]\n"
    "v_mfma_f32_16x16x32_bf16 v[44:47], v[212:215], v[196:199], v[44:47]\n"
    "v_mfma_f32_16x16x32_bf16 v[44:47], v[216:219], v[200:203], v[44:47]\n"
    "v_mfma_f32_16x16x32_bf16 v[48:51], v[220:223], v[196:199], v[48:51]\n"
    "v_mfma_f32_16x16x32_bf16 v[48:51], v[224:227], v[200:203], v[48:51]\n"
    "v_mfma_f32_16x16x32_bf16 v[52:55], v[228:231], v[196:199], v[52:55]\n"
    "v_mfma_f32_16x16x32_bf16 v[52:55], v[232:235], v[200:203], v[52:55]\n"
    "v_mfma_f32_16x16x32_bf16 v[56:59], v[236:239], v[196:199], v[56:59]\n"
    "v_mfma_f32_16x16x32_bf16 v[56:59], v[240:243], v[200:203], v[56:59]\n"
    "v_mfma_f32_16x16x32_bf16 v[60:63], v[212:215], v[204:207], v[60:63]\n"
    "v_mfma_f32_16x16x32_bf16 v[60:63], v[216:219], v[208:211], v[60:63]\n"
    "v_mfma_f32_16x16x32_bf16 v[64:67], v[220:223], v[204:207], v[64:67]\n"
    "v_mfma_f32_16x16x32_bf16 v[64:67], v[224:227], v[208:211], v[64:67]\n"
    "v_mfma_f32_16x16x32_bf16 v[68:71], v[228:231], v[204:207], v[68:71]\n"
    "v_mfma_f32_16x16x32_bf16 v[68:71], v[232:235], v[208:211], v[68:71]\n"
    "v_mfma_f32_16x16x32_bf16 v[72:75], v[236:239], v[204:207], v[72:75]\n"
    "v_mfma_f32_16x16x32_bf16 v[72:75], v[240:243], v[208:211], v[72:75]\n"
    "s_setprio 0\n"
    "s_waitcnt vmcnt(8)\n"
    "s_barrier\n"
    "; phase 5: step 5 slot 0 stage 0\n"
    "ds_read_b128 v[180:183], v244 offset:32768\n"
    "ds_read_b128 v[184:187], v245 offset:32768\n"
    "ds_read_b128 v[188:191], v244 offset:34816\n"
    "ds_read_b128 v[192:195], v245 offset:34816\n"
    "s_add_u32 m0, s50, 0x4000\n"
    "s_xor_b32 s82, s48, 0x200380\n"
    "buffer_load_dwordx4 v250, s[36:39], s82 offen  lds\n"
    "ds_read_b128 v[196:199], v244 offset:40960\n"
    "ds_read_b128 v[200:203], v245 offset:40960\n"
    "ds_read_b128 v[204:207], v244 offset:43008\n"
    "ds_read_b128 v[208:211], v245 offset:43008\n"
    "s_add_u32 m0, s50, 0x5000\n"
    "s_xor_b32 s82, s48, 0x280380\n"
    "buffer_load_dwordx4 v250, s[36:39], s82 offen  lds\n"
    "ds_read_b64_tr_b16 v[212:213], v246 offset:16384\n"
    "ds_read_b64_tr_b16 v[214:215], v246 offset:17408\n"
    "ds_read_b64_tr_b16 v[216:217], v246 offset:24576\n"
    "ds_read_b64_tr_b16 v[218:219], v246 offset:25600\n"
    "s_add_u32 m0, s50, 0x6000\n"
    "s_xor_b32 s82, s48, 0x380\n"
    "buffer_load_dwordx4 v250, s[36:39], s82 offen  lds\n"
    "ds_read_b64_tr_b16 v[220:221], v247 offset:16384\n"
    "ds_read_b64_tr_b16 v[222:223], v247 offset:17408\n"
    "ds_read_b64_tr_b16 v[224:225], v247 offset:24576\n"
    "ds_read_b64_tr_b16 v[226:227], v247 offset:25600\n"
    "s_add_u32 m0, s50, 0x7000\n"
    "s_xor_b32 s82, s48, 0x80380\n"
    "buffer_load_dwordx4 v250, s[36:39], s82 offen  lds\n"
    "ds_read_b64_tr_b16 v[228:229], v248 offset:16384\n"
    "ds_read_b64_tr_b16 v[230:231], v248 offset:17408\n"
    "ds_read_b64_tr_b16 v[232:233], v248 offset:24576\n"
    "ds_read_b64_tr_b16 v[234:235], v248 offset:25600\n"
    "s_add_u32 m0, s51, 0x0\n"
    "s_xor_b32 s82, s64, 0x800000\n"
    "buffer_load_dwordx4 v251, s[40:43], s82 offen  lds\n"
    "ds_read_b64_tr_b16 v[236:237], v249 offset:16384\n"
    "ds_read_b64_tr_b16 v[238:239], v249 offset:17408\n"
    "ds_read_b64_tr_b16 v[240:241], v249 offset:24576\n"
    "ds_read_b64_tr_b16 v[242:243], v249 offset:25600\n"
    "s_add_u32 m0, s51, 0x2000\n"
    "s_xor_b32 s82, s64, 0x880000\n"
    "buffer_load_dwordx4 v251, s[40:43], s82 offen  lds\n"
    "s_waitcnt vmcnt(12) lgkmcnt(0)\n"
    "s_barrier\n"
    "s_setprio 1\n"
    "v_mfma_f32_16x16x32_bf16 v[12:15], v[212:215], v[180:183], v[12:15]\n"
    "v_mfma_f32_16x16x32_bf16 v[12:15], v[216:219], v[184:187], v[12:15]\n"
    "v_mfma_f32_16x16x32_bf16 v[16:19], v[220:223], v[180:183], v[16:19]\n"
    "v_mfma_f32_16x16x32_bf16 v[16:19], v[224:227], v[184:187], v[16:19]\n"
    "v_mfma_f32_16x16x32_bf16 v[20:23], v[228:231], v[180:183], v[20:23]\n"
    "v_mfma_f32_16x16x32_bf16 v[20:23], v[232:235], v[184:187], v[20:23]\n"
    "v_mfma_f32_16x16x32_bf16 v[24:27], v[236:239], v[180:183], v[24:27]\n"
    "v_mfma_f32_16x16x32_bf16 v[24:27], v[240:243], v[184:187], v[24:27]\n"
    "v_mfma_f32_16x16x32_bf16 v[28:31], v[212:215], v[188:191], v[28:31]\n"
    "v_mfma_f32_16x16x32_bf16 v[28:31], v[216:219], v[192:195], v[28:31]\n"
    "v_mfma_f32_16x16x32_bf16 v[32:35], v[220:223], v[188:191], v[32:35]\n"
    "v_mfma_f32_16x16x32_bf16 v[32:35], v[224:227], v[192:195], v[32:35]\n"
    "v_mfma_f32_16x16x32_bf16 v[36:39], v[228:231], v[188:191], v[36:39]\n"
    "v_mfma_f32_16x16x32_bf16 v[36:39], v[232:235], v[192:195], v[36:39]\n"
    "v_mfma_f32_16x16x32_bf16 v[40:43], v[236:239], v[188:191], v[40:43]\n"
    "v_mfma_f32_16x16x32_bf16 v[40:43], v[240:243], v[192:195], v[40:43]\n"
    "v_mfma_f32_16x16x32_bf16 v[44:47], v[212:215], v[196:199], v[44:47]\n"
    "v_mfma_f32_16x16x32_bf16 v[44:47], v[216:219], v[200:203], v[44:47]\n"
    "v_mfma_f32_16x16x32_bf16 v[48:51], v[220:223], v[196:199], v[48:51]\n"
    "v_mfma_f32_16x16x32_bf16 v[48:51], v[224:227], v[200:203], v[48:51]\n"
    "v_mfma_f32_16x16x32_bf16 v[52:55], v[228:231], v[196:199], v[52:55]\n"
    "v_mfma_f32_16x16x32_bf16 v[52:55], v[232:235], v[200:203], v[52:55]\n"
    "v_mfma_f32_16x16x32_bf16 v[56:59], v[236:239], v[196:199], v[56:59]\n"
    "v_mfma_f32_16x16x32_bf16 v[56:59], v[240:243], v[200:203], v[56:59]\n"
    "v_mfma_f32_16x16x32_bf16 v[60:63], v[212:215], v[204:207], v[60:63]\n"
    "v_mfma_f32_16x16x32_bf16 v[60:63], v[216:219], v[208:211], v[60:63]\n"
    "v_mfma_f32_16x16x32_bf16 v[64:67], v[220:223], v[204:207], v[64:67]\n"
    "v_mfma_f32_16x16x32_bf16 v[64:67], v[224:227], v[208:211], v[64:67]\n"
    "v_mfma_f32_16x16x32_bf16 v[68:71], v[228:231], v[204:207], v[68:71]\n"
    "v_mfma_f32_16x16x32_bf16 v[68:71], v[232:235], v[208:211], v[68:71]\n"
    "v_mfma_f32_16x16x32_bf16 v[72:75], v[236:239], v[204:207], v[72:75]\n"
    "v_mfma_f32_16x16x32_bf16 v[72:75], v[240:243], v[208:211], v[72:75]\n"
    "s_setprio 0\n"
    "s_waitcnt vmcnt(8)\n"
    "s_barrier\n"
    "; phase 6: step 6 slot 0 stage 0\n"
    "ds_read_b128 v[180:183], v244 offset:0\n"
    "ds_read_b128 v[184:187], v245 offset:0\n"
    "ds_read_b128 v[188:191], v244 offset:2048\n"
    "ds_read_b128 v[192:195], v245 offset:2048\n"
    "s_add_u32 m0, s50, 0x8000\n"
    "s_xor_b32 s82, s48, 0x200400\n"
    "buffer_load_dwordx4 v250, s[36:39], s82 offen  lds\n"
    "ds_read_b128 v[196:199], v244 offset:8192\n"
    "ds_read_b128 v[200:203], v245 offset:8192\n"
    "ds_read_b128 v[204:207], v244 offset:10240\n"
    "ds_read_b128 v[208:211], v245 offset:10240\n"
    "s_add_u32 m0, s50, 0x9000\n"
    "s_xor_b32 s82, s48, 0x280400\n"
    "buffer_load_dwordx4 v250, s[36:39], s82 offen  lds\n"
    "ds_read_b64_tr_b16 v[212:213], v246 offset:32768\n"
    "ds_read_b64_tr_b16 v[214:215], v246 offset:33792\n"
    "ds_read_b64_tr_b16 v[216:217], v246 offset:40960\n"
    "ds_read_b64_tr_b16 v[218:219], v246 offset:41984\n"
    "s_add_u32 m0, s50, 0xa000\n"
    "s_xor_b32 s82, s48, 0x400\n"
    "buffer_load_dwordx4 v250, s[36:39], s82 offen  lds\n"
    "ds_read_b64_tr_b16 v[220:221], v247 offset:32768\n"
    "ds_read_b64_tr_b16 v[222:223], v247 offset:33792\n"
    "ds_read_b64_tr_b16 v[224:225], v247 offset:40960\n"
    "ds_read_b64_tr_b16 v[226:227], v247 offset:41984\n"
    "s_add_u32 m0, s50, 0xb000\n"
    "s_xor_b32 s82, s48, 0x80400\n"
    "buffer_load_dwordx4 v250, s[36:39], s82 offen  lds\n"
    "ds_read_b64_tr_b16 v[228:229], v248 offset:32768\n"
    "ds_read_b64_tr_b16 v[230:231], v248 offset:33792\n"
    "ds_read_b64_tr_b16 v[232:233], v248 offset:40960\n"
    "ds_read_b64_tr_b16 v[234:235], v248 offset:41984\n"
    "s_add_u32 m0, s51, 0x4000\n"
    "s_xor_b32 s82, s64, 0x900000\n"
    "buffer_load_dwordx4 v251, s[40:43], s82 offen  lds\n"
    "ds_read_b64_tr_b16 v[236:237], v249 offset:32768\n"
    "ds_read_b64_tr_b16 v[238:239], v249 offset:33792\n"
    "ds_read_b64_tr_b16 v[240:241], v249 offset:40960\n"
    "ds_read_b64_tr_b16 v[242:243], v249 offset:41984\n"
    "s_add_u32 m0, s51, 0x6000\n"
    "s_xor_b32 s82, s64, 0x980000\n"
    "buffer_load_dwordx4 v251, s[40:43], s82 offen  lds\n"
    "s_waitcnt vmcnt(12) lgkmcnt(0)\n"
    "s_barrier\n"
    "s_setprio 1\n"
    "v_mfma_f32_16x16x32_bf16 v[12:15], v[212:215], v[180:183], v[12:15]\n"
    "v_mfma_f32_16x16x32_bf16 v[12:15], v[216:219], v[184:187], v[12:15]\n"
    "v_mfma_f32_16x16x32_bf16 v[16:19], v[220:223], v[180:183], v[16:19]\n"
    "v_mfma_f32_16x16x32_bf16 v[16:19], v[224:227], v[184:187], v[16:19]\n"
    "v_mfma_f32_16x16x32_bf16 v[20:23], v[228:231], v[180:183], v[20:23]\n"
    "v_mfma_f32_16x16x32_bf16 v[20:23], v[232:235], v[184:187], v[20:23]\n"
    "v_mfma_f32_16x16x32_bf16 v[24:27], v[236:239], v[180:183], v[24:27]\n"
    "v_mfma_f32_16x16x32_bf16 v[24:27], v[240:243], v[184:187], v[24:27]\n"
    "v_mfma_f32_16x16x32_bf16 v[28:31], v[212:215], v[188:191], v[28:31]\n"
    "v_mfma_f32_16x16x32_bf16 v[28:31], v[216:219], v[192:195], v[28:31]\n"
    "v_mfma_f32_16x16x32_bf16 v[32:35], v[220:223], v[188:191], v[32:35]\n"
    "v_mfma_f32_16x16x32_bf16 v[32:35], v[224:227], v[192:195], v[32:35]\n"
    "v_mfma_f32_16x16x32_bf16 v[36:39], v[228:231], v[188:191], v[36:39]\n"
    "v_mfma_f32_16x16x32_bf16 v[36:39], v[232:235], v[192:195], v[36:39]\n"
    "v_mfma_f32_16x16x32_bf16 v[40:43], v[236:239], v[188:191], v[40:43]\n"
    "v_mfma_f32_16x16x32_bf16 v[40:43], v[240:243], v[192:195], v[40:43]\n"
    "v_mfma_f32_16x16x32_bf16 v[44:47], v[212:215], v[196:199], v[44:47]\n"
    "v_mfma_f32_16x16x32_bf16 v[44:47], v[216:219], v[200:203], v[44:47]\n"
    "v_mfma_f32_16x16x32_bf16 v[48:51], v[220:223], v[196:199], v[48:51]\n"
    "v_mfma_f32_16x16x32_bf16 v[48:51], v[224:227], v[200:203], v[48:51]\n"
    "v_mfma_f32_16x16x32_bf16 v[52:55], v[228:231], v[196:199], v[52:55]\n"
    "v_mfma_f32_16x16x32_bf16 v[52:55], v[232:235], v[200:203], v[52:55]\n"
    "v_mfma_f32_16x16x32_bf16 v[56:59], v[236:239], v[196:199], v[56:59]\n"
    "v_mfma_f32_16x16x32_bf16 v[56:59], v[240:243], v[200:203], v[56:59]\n"
    "v_mfma_f32_16x16x32_bf16 v[60:63], v[212:215], v[204:207], v[60:63]\n"
    "v_mfma_f32_16x16x32_bf16 v[60:63], v[216:219], v[208:211], v[60:63]\n"
    "v_mfma_f32_16x16x32_bf16 v[64:67], v[220:223], v[204:207], v[64:67]\n"
    "v_mfma_f32_16x16x32_bf16 v[64:67], v[224:227], v[208:211], v[64:67]\n"
    "v_mfma_f32_16x16x32_bf16 v[68:71], v[228:231], v[204:207], v[68:71]\n"
    "v_mfma_f32_16x16x32_bf16 v[68:71], v[232:235], v[208:211], v[68:71]\n"
    "v_mfma_f32_16x16x32_bf16 v[72:75], v[236:239], v[204:207], v[72:75]\n"
    "v_mfma_f32_16x16x32_bf16 v[72:75], v[240:243], v[208:211], v[72:75]\n"
    "s_setprio 0\n"
    "s_waitcnt vmcnt(8)\n"
    "s_barrier\n"
    "; phase 7: step 7 slot 0 stage 0\n"
    "ds_read_b128 v[180:183], v244 offset:16384\n"
    "ds_read_b128 v[184:187], v245 offset:16384\n"
    "ds_read_b128 v[188:191], v244 offset:18432\n"
    "ds_read_b128 v[192:195], v245 offset:18432\n"
    "s_add_u32 m0, s50, 0x0\n"
    "s_xor_b32 s82, s48, 0x200480\n"
    "buffer_load_dwordx4 v250, s[36:39], s82 offen  lds\n"
    "ds_read_b128 v[196:199], v244 offset:24576\n"
    "ds_read_b128 v[200:203], v245 offset:24576\n"
    "ds_read_b128 v[204:207], v244 offset:26624\n"
    "ds_read_b128 v[208:211], v245 offset:26624\n"
    "s_add_u32 m0, s50, 0x1000\n"
    "s_xor_b32 s82, s48, 0x280480\n"
    "buffer_load_dwordx4 v250, s[36:39], s82 offen  lds\n"
    "ds_read_b64_tr_b16 v[212:213], v246 offset:49152\n"
    "ds_read_b64_tr_b16 v[214:215], v246 offset:50176\n"
    "ds_read_b64_tr_b16 v[216:217], v246 offset:57344\n"
    "ds_read_b64_tr_b16 v[218:219], v246 offset:58368\n"
    "s_add_u32 m0, s50, 0x2000\n"
    "s_xor_b32 s82, s48, 0x480\n"
    "buffer_load_dwordx4 v250, s[36:39], s82 offen  lds\n"
    "ds_read_b64_tr_b16 v[220:221], v247 offset:49152\n"
    "ds_read_b64_tr_b16 v[222:223], v247 offset:50176\n"
    "ds_read_b64_tr_b16 v[224:225], v247 offset:57344\n"
    "ds_read_b64_tr_b16 v[226:227], v247 offset:58368\n"
    "s_add_u32 m0, s50, 0x3000\n"
    "s_xor_b32 s82, s48, 0x80480\n"
    "buffer_load_dwordx4 v250, s[36:39], s82 offen  lds\n"
    "ds_read_b64_tr_b16 v[228:229], v248 offset:49152\n"
    "ds_read_b64_tr_b16 v[230:231], v248 offset:50176\n"
    "ds_read_b64_tr_b16 v[232:233], v248 offset:57344\n"
    "ds_read_b64_tr_b16 v[234:235], v248 offset:58368\n"
    "s_add_u32 m0, s51, 0x8000\n"
    "s_xor_b32 s82, s64, 0xa00000\n"
    "buffer_load_dwordx4 v251, s[40:43], s82 offen  lds\n"
    "ds_read_b64_tr_b16 v[236:237], v249 offset:49152\n"
    "ds_read_b64_tr_b16 v[238:239], v249 offset:50176\n"
    "ds_read_b64_tr_b16 v[240:241], v249 offset:57344\n"
    "ds_read_b64_tr_b16 v[242:243], v249 offset:58368\n"
    "s_add_u32 m0, s51, 0xa000\n"
    "s_xor_b32 s82, s64, 0xa80000\n"
    "buffer_load_dwordx4 v251, s[40:43], s82 offen  lds\n"
    "s_waitcnt vmcnt(12) lgkmcnt(0)\n"
    "s_barrier\n"
    "s_setprio 1\n"
    "v_mfma_f32_16x16x32_bf16 v[12:15], v[212:215], v[180:183], v[12:15]\n"
    "v_mfma_f32_16x16x32_bf16 v[12:15], v[216:219], v[184:187], v[12:15]\n"
    "v_mfma_f32_16x16x32_bf16 v[16:19], v[220:223], v[180:183], v[16:19]\n"
    "v_mfma_f32_16x16x32_bf16 v[16:19], v[224:227], v[184:187], v[16:19]\n"
    "v_mfma_f32_16x16x32_bf16 v[20:23], v[228:231], v[180:183], v[20:23]\n"
    "v_mfma_f32_16x16x32_bf16 v[20:23], v[232:235], v[184:187], v[20:23]\n"
    "v_mfma_f32_16x16x32_bf16 v[24:27], v[236:239], v[180:183], v[24:27]\n"
    "v_mfma_f32_16x16x32_bf16 v[24:27], v[240:243], v[184:187], v[24:27]\n"
    "v_mfma_f32_16x16x32_bf16 v[28:31], v[212:215], v[188:191], v[28:31]\n"
    "v_mfma_f32_16x16x32_bf16 v[28:31], v[216:219], v[192:195], v[28:31]\n"
    "v_mfma_f32_16x16x32_bf16 v[32:35], v[220:223], v[188:191], v[32:35]\n"
    "v_mfma_f32_16x16x32_bf16 v[32:35], v[224:227], v[192:195], v[32:35]\n"
    "v_mfma_f32_16x16x32_bf16 v[36:39], v[228:231], v[188:191], v[36:39]\n"
    "v_mfma_f32_16x16x32_bf16 v[36:39], v[232:235], v[192:195], v[36:39]\n"
    "v_mfma_f32_16x16x32_bf16 v[40:43], v[236:239], v[188:191], v[40:43]\n"
    "v_mfma_f32_16x16x32_bf16 v[40:43], v[240:243], v[192:195], v[40:43]\n"
    "v_mfma_f32_16x16x32_bf16 v[44:47], v[212:215], v[196:199], v[44:47]\n"
    "v_mfma_f32_16x16x32_bf16 v[44:47], v[216:219], v[200:203], v[44:47]\n"
    "v_mfma_f32_16x16x32_bf16 v[48:51], v[220:223], v[196:199], v[48:51]\n"
    "v_mfma_f32_16x16x32_bf16 v[48:51], v[224:227], v[200:203], v[48:51]\n"
    "v_mfma_f32_16x16x32_bf16 v[52:55], v[228:231], v[196:199], v[52:55]\n"
    "v_mfma_f32_16x16x32_bf16 v[52:55], v[232:235], v[200:203], v[52:55]\n"
    "v_mfma_f32_16x16x32_bf16 v[56:59], v[236:239], v[196:199], v[56:59]\n"
    "v_mfma_f32_16x16x32_bf16 v[56:59], v[240:243], v[200:203], v[56:59]\n"
    "v_mfma_f32_16x16x32_bf16 v[60:63], v[212:215], v[204:207], v[60:63]\n"
    "v_mfma_f32_16x16x32_bf16 v[60:63], v[216:219], v[208:211], v[60:63]\n"
    "v_mfma_f32_16x16x32_bf16 v[64:67], v[220:223], v[204:207], v[64:67]\n"
    "v_mfma_f32_16x16x32_bf16 v[64:67], v[224:227], v[208:211], v[64:67]\n"
    "v_mfma_f32_16x16x32_bf16 v[68:71], v[228:231], v[204:207], v[68:71]\n"
    "v_mfma_f32_16x16x32_bf16 v[68:71], v[232:235], v[208:211], v[68:71]\n"
    "v_mfma_f32_16x16x32_bf16 v[72:75], v[236:239], v[204:207], v[72:75]\n"
    "v_mfma_f32_16x16x32_bf16 v[72:75], v[240:243], v[208:211], v[72:75]\n"
    "s_setprio 0\n"
    "s_waitcnt vmcnt(8)\n"
    "s_barrier\n"
    "; phase 8: step 8 slot 0 stage 0\n"
    "ds_read_b128 v[180:183], v244 offset:32768\n"
    "ds_read_b128 v[184:187], v245 offset:32768\n"
    "ds_read_b128 v[188:191], v244 offset:34816\n"
    "ds_read_b128 v[192:195], v245 offset:34816\n"
    "s_add_u32 m0, s50, 0x4000\n"
    "s_xor_b32 s82, s48, 0x200500\n"
    "buffer_load_dwordx4 v250, s[36:39], s82 offen  lds\n"
    "ds_read_b128 v[196:199], v244 offset:40960\n"
    "ds_read_b128 v[200:203], v245 offset:40960\n"
    "ds_read_b128 v[204:207], v244 offset:43008\n"
    "ds_read_b128 v[208:211], v245 offset:43008\n"
    "s_add_u32 m0, s50, 0x5000\n"
    "s_xor_b32 s82, s48, 0x280500\n"
    "buffer_load_dwordx4 v250, s[36:39], s82 offen  lds\n"
    "ds_read_b64_tr_b16 v[212:213], v246 offset:0\n"
    "ds_read_b64_tr_b16 v[214:215], v246 offset:1024\n"
    "ds_read_b64_tr_b16 v[216:217], v246 offset:8192\n"
    "ds_read_b64_tr_b16 v[218:219], v246 offset:9216\n"
    "s_add_u32 m0, s50, 0x6000\n"
    "s_xor_b32 s82, s48, 0x500\n"
    "buffer_load_dwordx4 v250, s[36:39], s82 offen  lds\n"
    "ds_read_b64_tr_b16 v[220:221], v247 offset:0\n"
    "ds_read_b64_tr_b16 v[222:223], v247 offset:1024\n"
    "ds_read_b64_tr_b16 v[224:225], v247 offset:8192\n"
    "ds_read_b64_tr_b16 v[226:227], v247 offset:9216\n"
    "s_add_u32 m0, s50, 0x7000\n"
    "s_xor_b32 s82, s48, 0x80500\n"
    "buffer_load_dwordx4 v250, s[36:39], s82 offen  lds\n"
    "ds_read_b64_tr_b16 v[228:229], v248 offset:0\n"
    "ds_read_b64_tr_b16 v[230:231], v248 offset:1024\n"
    "ds_read_b64_tr_b16 v[232:233], v248 offset:8192\n"
    "ds_read_b64_tr_b16 v[234:235], v248 offset:9216\n"
    "s_add_u32 m0, s51, 0xc000\n"
    "s_xor_b32 s82, s64, 0xb00000\n"
    "buffer_load_dwordx4 v251, s[40:43], s82 offen  lds\n"
    "ds_read_b64_tr_b16 v[236:237], v249 offset:0\n"
    "ds_read_b64_tr_b16 v[238:239], v249 offset:1024\n"
    "ds_read_b64_tr_b16 v[240:241], v249 offset:8192\n"
    "ds_read_b64_tr_b16 v[242:243], v249 offset:9216\n"
    "s_add_u32 m0, s51, 0xe000\n"
    "s_xor_b32 s82, s64, 0xb80000\n"
    "buffer_load_dwordx4 v251, s[40:43], s82 offen  lds\n"
    "s_waitcnt vmcnt(12) lgkmcnt(0)\n"
    "s_barrier\n"
    "s_setprio 1\n"
    "v_mfma_f32_16x16x32_bf16 v[12:15], v[212:215], v[180:183], v[12:15]\n"
    "v_mfma_f32_16x16x32_bf16 v[12:15], v[216:219], v[184:187], v[12:15]\n"
    "v_mfma_f32_16x16x32_bf16 v[16:19], v[220:223], v[180:183], v[16:19]\n"
    "v_mfma_f32_16x16x32_bf16 v[16:19], v[224:227], v[184:187], v[16:19]\n"
    "v_mfma_f32_16x16x32_bf16 v[20:23], v[228:231], v[180:183], v[20:23]\n"
    "v_mfma_f32_16x16x32_bf16 v[20:23], v[232:235], v[184:187], v[20:23]\n"
    "v_mfma_f32_16x16x32_bf16 v[24:27], v[236:239], v[180:183], v[24:27]\n"
    "v_mfma_f32_16x16x32_bf16 v[24:27], v[240:243], v[184:187], v[24:27]\n"
    "v_mfma_f32_16x16x32_bf16 v[28:31], v[212:215], v[188:191], v[28:31]\n"
    "v_mfma_f32_16x16x32_bf16 v[28:31], v[216:219], v[192:195], v[28:31]\n"
    "v_mfma_f32_16x16x32_bf16 v[32:35], v[220:223], v[188:191], v[32:35]\n"
    "v_mfma_f32_16x16x32_bf16 v[32:35], v[224:227], v[192:195], v[32:35]\n"
    "v_mfma_f32_16x16x32_bf16 v[36:39], v[228:231], v[188:191], v[36:39]\n"
    "v_mfma_f32_16x16x32_bf16 v[36:39], v[232:235], v[192:195], v[36:39]\n"
    "v_mfma_f32_16x16x32_bf16 v[40:43], v[236:239], v[188:191], v[40:43]\n"
    "v_mfma_f32_16x16x32_bf16 v[40:43], v[240:243], v[192:195], v[40:43]\n"
    "v_mfma_f32_16x16x32_bf16 v[44:47], v[212:215], v[196:199], v[44:47]\n"
    "v_mfma_f32_16x16x32_bf16 v[44:47], v[216:219], v[200:203], v[44:47]\n"
    "v_mfma_f32_16x16x32_bf16 v[48:51], v[220:223], v[196:199], v[48:51]\n"
    "v_mfma_f32_16x16x32_bf16 v[48:51], v[224:227], v[200:203], v[48:51]\n"
    "v_mfma_f32_16x16x32_bf16 v[52:55], v[228:231], v[196:199], v[52:55]\n"
    "v_mfma_f32_16x16x32_bf16 v[52:55], v[232:235], v[200:203], v[52:55]\n"
    "v_mfma_f32_16x16x32_bf16 v[56:59], v[236:239], v[196:199], v[56:59]\n"
    "v_mfma_f32_16x16x32_bf16 v[56:59], v[240:243], v[200:203], v[56:59]\n"
    "v_mfma_f32_16x16x32_bf16 v[60:63], v[212:215], v[204:207], v[60:63]\n"
    "v_mfma_f32_16x16x32_bf16 v[60:63], v[216:219], v[208:211], v[60:63]\n"
    "v_mfma_f32_16x16x32_bf16 v[64:67], v[220:223], v[204:207], v[64:67]\n"
    "v_mfma_f32_16x16x32_bf16 v[64:67], v[224:227], v[208:211], v[64:67]\n"
    "v_mfma_f32_16x16x32_bf16 v[68:71], v[228:231], v[204:207], v[68:71]\n"
    "v_mfma_f32_16x16x32_bf16 v[68:71], v[232:235], v[208:211], v[68:71]\n"
    "v_mfma_f32_16x16x32_bf16 v[72:75], v[236:239], v[204:207], v[72:75]\n"
    "v_mfma_f32_16x16x32_bf16 v[72:75], v[240:243], v[208:211], v[72:75]\n"
    "s_setprio 0\n"
    "s_waitcnt vmcnt(8)\n"
    "s_barrier\n"
    "; phase 9: step 9 slot 0 stage 0\n"
    "ds_read_b128 v[180:183], v244 offset:0\n"
    "ds_read_b128 v[184:187], v245 offset:0\n"
    "ds_read_b128 v[188:191], v244 offset:2048\n"
    "ds_read_b128 v[192:195], v245 offset:2048\n"
    "s_add_u32 m0, s50, 0x8000\n"
    "s_xor_b32 s82, s48, 0x200580\n"
    "buffer_load_dwordx4 v250, s[36:39], s82 offen  lds\n"
    "ds_read_b128 v[196:199], v244 offset:8192\n"
    "ds_read_b128 v[200:203], v245 offset:8192\n"
    "ds_read_b128 v[204:207], v244 offset:10240\n"
    "ds_read_b128 v[208:211], v245 offset:10240\n"
    "s_add_u32 m0, s50, 0x9000\n"
    "s_xor_b32 s82, s48, 0x280580\n"
    "buffer_load_dwordx4 v250, s[36:39], s82 offen  lds\n"
    "ds_read_b64_tr_b16 v[212:213], v246 offset:16384\n"
    "ds_read_b64_tr_b16 v[214:215], v246 offset:17408\n"
    "ds_read_b64_tr_b16 v[216:217], v246 offset:24576\n"
    "ds_read_b64_tr_b16 v[218:219], v246 offset:25600\n"
    "s_add_u32 m0, s50, 0xa000\n"
    "s_xor_b32 s82, s48, 0x580\n"
    "buffer_load_dwordx4 v250, s[36:39], s82 offen  lds\n"
    "ds_read_b64_tr_b16 v[220:221], v247 offset:16384\n"
    "ds_read_b64_tr_b16 v[222:223], v247 offset:17408\n"
    "ds_read_b64_tr_b16 v[224:225], v247 offset:24576\n"
    "ds_read_b64_tr_b16 v[226:227], v247 offset:25600\n"
    "s_add_u32 m0, s50, 0xb000\n"
    "s_xor_b32 s82, s48, 0x80580\n"
    "buffer_load_dwordx4 v250, s[36:39], s82 offen  lds\n"
    "ds_read_b64_tr_b16 v[228:229], v248 offset:16384\n"
    "ds_read_b64_tr_b16 v[230:231], v248 offset:17408\n"
    "ds_read_b64_tr_b16 v[232:233], v248 offset:24576\n"
    "ds_read_b64_tr_b16 v[234:235], v248 offset:25600\n"
    "s_add_u32 m0, s51, 0x0\n"
    "s_xor_b32 s82, s64, 0xc00000\n"
    "buffer_load_dwordx4 v251, s[40:43], s82 offen  lds\n"
    "ds_read_b64_tr_b16 v[236:237], v249 offset:16384\n"
    "ds_read_b64_tr_b16 v[238:239], v249 offset:17408\n"
    "ds_read_b64_tr_b16 v[240:241], v249 offset:24576\n"
    "ds_read_b64_tr_b16 v[242:243], v249 offset:25600\n"
    "s_add_u32 m0, s51, 0x2000\n"
    "s_xor_b32 s82, s64, 0xc80000\n"
    "buffer_load_dwordx4 v251, s[40:43], s82 offen  lds\n"
    "s_waitcnt vmcnt(12) lgkmcnt(0)\n"
    "s_barrier\n"
    "s_setprio 1\n"
    "v_mfma_f32_16x16x32_bf16 v[12:15], v[212:215], v[180:183], v[12:15]\n"
    "v_mfma_f32_16x16x32_bf16 v[12:15], v[216:219], v[184:187], v[12:15]\n"
    "v_mfma_f32_16x16x32_bf16 v[16:19], v[220:223], v[180:183], v[16:19]\n"
    "v_mfma_f32_16x16x32_bf16 v[16:19], v[224:227], v[184:187], v[16:19]\n"
    "v_mfma_f32_16x16x32_bf16 v[20:23], v[228:231], v[180:183], v[20:23]\n"
    "v_mfma_f32_16x16x32_bf16 v[20:23], v[232:235], v[184:187], v[20:23]\n"
    "v_mfma_f32_16x16x32_bf16 v[24:27], v[236:239], v[180:183], v[24:27]\n"
    "v_mfma_f32_16x16x32_bf16 v[24:27], v[240:243], v[184:187], v[24:27]\n"
    "v_mfma_f32_16x16x32_bf16 v[28:31], v[212:215], v[188:191], v[28:31]\n"
    "v_mfma_f32_16x16x32_bf16 v[28:31], v[216:219], v[192:195], v[28:31]\n"
    "v_mfma_f32_16x16x32_bf16 v[32:35], v[220:223], v[188:191], v[32:35]\n"
    "v_mfma_f32_16x16x32_bf16 v[32:35], v[224:227], v[192:195], v[32:35]\n"
    "v_mfma_f32_16x16x32_bf16 v[36:39], v[228:231], v[188:191], v[36:39]\n"
    "v_mfma_f32_16x16x32_bf16 v[36:39], v[232:235], v[192:195], v[36:39]\n"
    "v_mfma_f32_16x16x32_bf16 v[40:43], v[236:239], v[188:191], v[40:43]\n"
    "v_mfma_f32_16x16x32_bf16 v[40:43], v[240:243], v[192:195], v[40:43]\n"
    "v_mfma_f32_16x16x32_bf16 v[44:47], v[212:215], v[196:199], v[44:47]\n"
    "v_mfma_f32_16x16x32_bf16 v[44:47], v[216:219], v[200:203], v[44:47]\n"
    "v_mfma_f32_16x16x32_bf16 v[48:51], v[220:223], v[196:199], v[48:51]\n"
    "v_mfma_f32_16x16x32_bf16 v[48:51], v[224:227], v[200:203], v[48:51]\n"
    "v_mfma_f32_16x16x32_bf16 v[52:55], v[228:231], v[196:199], v[52:55]\n"
    "v_mfma_f32_16x16x32_bf16 v[52:55], v[232:235], v[200:203], v[52:55]\n"
    "v_mfma_f32_16x16x32_bf16 v[56:59], v[236:239], v[196:199], v[56:59]\n"
    "v_mfma_f32_16x16x32_bf16 v[56:59], v[240:243], v[200:203], v[56:59]\n"
    "v_mfma_f32_16x16x32_bf16 v[60:63], v[212:215], v[204:207], v[60:63]\n"
    "v_mfma_f32_16x16x32_bf16 v[60:63], v[216:219], v[208:211], v[60:63]\n"
    "v_mfma_f32_16x16x32_bf16 v[64:67], v[220:223], v[204:207], v[64:67]\n"
    "v_mfma_f32_16x16x32_bf16 v[64:67], v[224:227], v[208:211], v[64:67]\n"
    "v_mfma_f32_16x16x32_bf16 v[68:71], v[228:231], v[204:207], v[68:71]\n"
    "v_mfma_f32_16x16x32_bf16 v[68:71], v[232:235], v[208:211], v[68:71]\n"
    "v_mfma_f32_16x16x32_bf16 v[72:75], v[236:239], v[204:207], v[72:75]\n"
    "v_mfma_f32_16x16x32_bf16 v[72:75], v[240:243], v[208:211], v[72:75]\n"
    "s_setprio 0\n"
    "s_waitcnt vmcnt(8)\n"
    "s_barrier\n"
    "; phase 10: step 10 slot 0 stage 0\n"
    "ds_read_b128 v[180:183], v244 offset:16384\n"
    "ds_read_b128 v[184:187], v245 offset:16384\n"
    "ds_read_b128 v[188:191], v244 offset:18432\n"
    "s_lshr_b32 s82, s54, 12\n"
    "s_and_b32 s82, s82, 0xf80\n"
    "s_add_u32 s82, s82, 0x40000\n"
    "buffer_load_dword v6, off, s[44:47], s82 sc0 sc1\n"
    "ds_read_b128 v[192:195], v245 offset:18432\n"
    "ds_read_b128 v[196:199], v244 offset:24576\n"
    "ds_read_b128 v[200:203], v245 offset:24576\n"
    "s_add_u32 m0, s50, 0x0\n"
    "s_xor_b32 s82, s48, 0x200600\n"
    "buffer_load_dwordx4 v250, s[36:39], s82 offen  lds\n"
    "ds_read_b128 v[204:207], v244 offset:26624\n"
    "ds_read_b128 v[208:211], v245 offset:26624\n"
    "ds_read_b64_tr_b16 v[212:213], v246 offset:32768\n"
    "s_add_u32 m0, s50, 0x1000\n"
    "s_xor_b32 s82, s48, 0x280600\n"
    "buffer_load_dwordx4 v250, s[36:39], s82 offen  lds\n"
    "ds_read_b64_tr_b16 v[214:215], v246 offset:33792\n"
    "ds_read_b64_tr_b16 v[216:217], v246 offset:40960\n"
    "ds_read_b64_tr_b16 v[218:219], v246 offset:41984\n"
    "s_add_u32 m0, s50, 0x2000\n"
    "s_xor_b32 s82, s48, 0x600\n"
    "buffer_load_dwordx4 v250, s[36:39], s82 offen  lds\n"
    "ds_read_b64_tr_b16 v[220:221], v247 offset:32768\n"
    "ds_read_b64_tr_b16 v[222:223], v247 offset:33792\n"
    "ds_read_b64_tr_b16 v[224:225], v247 offset:40960\n"
    "s_add_u32 m0, s50, 0x3000\n"
    "s_xor_b32 s82, s48, 0x80600\n"
    "buffer_load_dwordx4 v250, s[36:39], s82 offen  lds\n"
    "ds_read_b64_tr_b16 v[226:227], v247 offset:41984\n"
    "ds_read_b64_tr_b16 v[228:229], v248 offset:32768\n"
    "ds_read_b64_tr_b16 v[230:231], v248 offset:33792\n"
    "s_add_u32 m0, s51, 0x4000\n"
    "s_xor_b32 s82, s65, 0xc00000\n"
    "buffer_load_dwordx4 v251, s[40:43], s82 offen  lds\n"
    "ds_read_b64_tr_b16 v[232:233], v248 offset:40960\n"
    "ds_read_b64_tr_b16 v[234:235], v248 offset:41984\n"
    "ds_read_b64_tr_b16 v[236:237], v249 offset:32768\n"
    "s_add_u32 m0, s51, 0x6000\n"
    "s_xor_b32 s82, s65, 0xc80000\n"
    "buffer_load_dwordx4 v251, s[40:43], s82 offen  lds\n"
    "ds_read_b64_tr_b16 v[238:239], v249 offset:33792\n"
    "ds_read_b64_tr_b16 v[240:241], v249 offset:40960\n"
    "ds_read_b64_tr_b16 v[242:243], v249 offset:41984\n"
    "s_waitcnt vmcnt(13) lgkmcnt(0)\n"
    "s_barrier\n"
    "s_setprio 1\n"
    "v_mfma_f32_16x16x32_bf16 v[12:15], v[212:215], v[180:183], v[12:15]\n"
    "v_mfma_f32_16x16x32_bf16 v[12:15], v[216:219], v[184:187], v[12:15]\n"
    "v_mfma_f32_16x16x32_bf16 v[16:19], v[220:223], v[180:183], v[16:19]\n"
    "v_mfma_f32_16x16x32_bf16 v[16:19], v[224:227], v[184:187], v[16:19]\n"
    "v_mfma_f32_16x16x32_bf16 v[20:23], v[228:231], v[180:183], v[20:23]\n"
    "v_mfma_f32_16x16x32_bf16 v[20:23], v[232:235], v[184:187], v[20:23]\n"
    "v_mfma_f32_16x16x32_bf16 v[24:27], v[236:239], v[180:183], v[24:27]\n"
    "v_mfma_f32_16x16x32_bf16 v[24:27], v[240:243], v[184:187], v[24:27]\n"
    "v_mfma_f32_16x16x32_bf16 v[28:31], v[212:215], v[188:191], v[28:31]\n"
    "v_mfma_f32_16x16x32_bf16 v[28:31], v[216:219], v[192:195], v[28:31]\n"
    "v_mfma_f32_16x16x32_bf16 v[32:35], v[220:223], v[188:191], v[32:35]\n"
    "v_mfma_f32_16x16x32_bf16 v[32:35], v[224:227], v[192:195], v[32:35]\n"
    "v_mfma_f32_16x16x32_bf16 v[36:39], v[228:231], v[188:191], v[36:39]\n"
    "v_mfma_f32_16x16x32_bf16 v[36:39], v[232:235], v[192:195], v[36:39]\n"
    "v_mfma_f32_16x16x32_bf16 v[40:43], v[236:239], v[188:191], v[40:43]\n"
    "v_mfma_f32_16x16x32_bf16 v[40:43], v[240:243], v[192:195], v[40:43]\n"
    "v_mfma_f32_16x16x32_bf16 v[44:47], v[212:215], v[196:199], v[44:47]\n"
    "v_mfma_f32_16x16x32_bf16 v[44:47], v[216:219], v[200:203], v[44:47]\n"
    "v_mfma_f32_16x16x32_bf16 v[48:51], v[220:223], v[196:199], v[48:51]\n"
    "v_mfma_f32_16x16x32_bf16 v[48:51], v[224:227], v[200:203], v[48:51]\n"
    "v_mfma_f32_16x16x32_bf16 v[52:55], v[228:231], v[196:199], v[52:55]\n"
    "v_mfma_f32_16x16x32_bf16 v[52:55], v[232:235], v[200:203], v[52:55]\n"
    "v_mfma_f32_16x16x32_bf16 v[56:59], v[236:239], v[196:199], v[56:59]\n"
    "v_mfma_f32_16x16x32_bf16 v[56:59], v[240:243], v[200:203], v[56:59]\n"
    "v_mfma_f32_16x16x32_bf16 v[60:63], v[212:215], v[204:207], v[60:63]\n"
    "v_mfma_f32_16x16x32_bf16 v[60:63], v[216:219], v[208:211], v[60:63]\n"
    "v_mfma_f32_16x16x32_bf16 v[64:67], v[220:223], v[204:207], v[64:67]\n"
    "v_mfma_f32_16x16x32_bf16 v[64:67], v[224:227], v[208:211], v[64:67]\n"
    "v_mfma_f32_16x16x32_bf16 v[68:71], v[228:231], v[204:207], v[68:71]\n"
    "v_mfma_f32_16x16x32_bf16 v[68:71], v[232:235], v[208:211], v[68:71]\n"
    "v_mfma_f32_16x16x32_bf16 v[72:75], v[236:239], v[204:207], v[72:75]\n"
    "v_mfma_f32_16x16x32_bf16 v[72:75], v[240:243], v[208:211], v[72:75]\n"
    "s_setprio 0\n"
    "s_waitcnt vmcnt(9)\n"
    "s_barrier\n"
    "; phase 11: step 11 slot 0 stage 0\n"
    "ds_read_b128 v[180:183], v244 offset:32768\n"
    "ds_read_b128 v[184:187], v245 offset:32768\n"
    "ds_read_b128 v[188:191], v244 offset:34816\n"
    "ds_read_b128 v[192:195], v245 offset:34816\n"
    "s_add_u32 m0, s51, 0x8000\n"
    "s_xor_b32 s82, s64, 0xd00000\n"
    "buffer_load_dwordx4 v251, s[40:43], s82 offen  lds\n"
    "ds_read_b128 v[196:199], v244 offset:40960\n"
    "ds_read_b128 v[200:203], v245 offset:40960\n"
    "ds_read_b128 v[204:207], v244 offset:43008\n"
    "ds_read_b128 v[208:211], v245 offset:43008\n"
    "s_add_u32 m0, s51, 0xa000\n"
    "s_xor_b32 s82, s64, 0xd80000\n"
    "buffer_load_dwordx4 v251, s[40:43], s82 offen  lds\n"
    "ds_read_b64_tr_b16 v[212:213], v246 offset:49152\n"
    "ds_read_b64_tr_b16 v[214:215], v246 offset:50176\n"
    "ds_read_b64_tr_b16 v[216:217], v246 offset:57344\n"
    "ds_read_b64_tr_b16 v[218:219], v246 offset:58368\n"
    "s_add_u32 m0, s50, 0x4000\n"
    "s_xor_b32 s82, s48, 0x200680\n"
    "buffer_load_dwordx4 v250, s[36:39], s82 offen  lds\n"
    "ds_read_b64_tr_b16 v[220:221], v247 offset:49152\n"
    "ds_read_b64_tr_b16 v[222:223], v247 offset:50176\n"
    "ds_read_b64_tr_b16 v[224:225], v247 offset:57344\n"
    "ds_read_b64_tr_b16 v[226:227], v247 offset:58368\n"
    "s_add_u32 m0, s50, 0x5000\n"
    "s_xor_b32 s82, s48, 0x280680\n"
    "buffer_load_dwordx4 v250, s[36:39], s82 offen  lds\n"
    "ds_read_b64_tr_b16 v[228:229], v248 offset:49152\n"
    "ds_read_b64_tr_b16 v[230:231], v248 offset:50176\n"
    "ds_read_b64_tr_b16 v[232:233], v248 offset:57344\n"
    "ds_read_b64_tr_b16 v[234:235], v248 offset:58368\n"
    "s_add_u32 m0, s50, 0x6000\n"
    "s_xor_b32 s82, s48, 0x680\n"
    "buffer_load_dwordx4 v250, s[36:39], s82 offen  lds\n"
    "ds_read_b64_tr_b16 v[236:237], v249 offset:49152\n"
    "ds_read_b64_tr_b16 v[238:239], v249 offset:50176\n"
    "ds_read_b64_tr_b16 v[240:241], v249 offset:57344\n"
    "ds_read_b64_tr_b16 v[242:243], v249 offset:58368\n"
    "s_add_u32 m0, s50, 0x7000\n"
    "s_xor_b32 s82, s48, 0x80680\n"
    "buffer_load_dwordx4 v250, s[36:39], s82 offen  lds\n"
    "s_waitcnt vmcnt(13) lgkmcnt(0)\n"
    "s_barrier\n"
    "s_setprio 1\n"
    "v_mfma_f32_16x16x32_bf16 v[12:15], v[212:215], v[180:183], v[12:15]\n"
    "v_mfma_f32_16x16x32_bf16 v[12:15], v[216:219], v[184:187], v[12:15]\n"
    "v_mfma_f32_16x16x32_bf16 v[16:19], v[220:223], v[180:183], v[16:19]\n"
    "v_mfma_f32_16x16x32_bf16 v[16:19], v[224:227], v[184:187], v[16:19]\n"
    "v_mfma_f32_16x16x32_bf16 v[20:23], v[228:231], v[180:183], v[20:23]\n"
    "v_mfma_f32_16x16x32_bf16 v[20:23], v[232:235], v[184:187], v[20:23]\n"
    "v_mfma_f32_16x16x32_bf16 v[24:27], v[236:239], v[180:183], v[24:27]\n"
    "v_mfma_f32_16x16x32_bf16 v[24:27], v[240:243], v[184:187], v[24:27]\n"
    "v_mfma_f32_16x16x32_bf16 v[28:31], v[212:215], v[188:191], v[28:31]\n"
    "v_mfma_f32_16x16x32_bf16 v[28:31], v[216:219], v[192:195], v[28:31]\n"
    "v_mfma_f32_16x16x32_bf16 v[32:35], v[220:223], v[188:191], v[32:35]\n"
    "v_mfma_f32_16x16x32_bf16 v[32:35], v[224:227], v[192:195], v[32:35]\n"
    "v_mfma_f32_16x16x32_bf16 v[36:39], v[228:231], v[188:191], v[36:39]\n"
    "v_mfma_f32_16x16x32_bf16 v[36:39], v[232:235], v[192:195], v[36:39]\n"
    "v_mfma_f32_16x16x32_bf16 v[40:43], v[236:239], v[188:191], v[40:43]\n"
    "v_mfma_f32_16x16x32_bf16 v[40:43], v[240:243], v[192:195], v[40:43]\n"
    "v_mfma_f32_16x16x32_bf16 v[44:47], v[212:215], v[196:199], v[44:47]\n"
    "v_mfma_f32_16x16x32_bf16 v[44:47], v[216:219], v[200:203], v[44:47]\n"
    "v_mfma_f32_16x16x32_bf16 v[48:51], v[220:223], v[196:199], v[48:51]\n"
    "v_mfma_f32_16x16x32_bf16 v[48:51], v[224:227], v[200:203], v[48:51]\n"
    "v_mfma_f32_16x16x32_bf16 v[52:55], v[228:231], v[196:199], v[52:55]\n"
    "v_mfma_f32_16x16x32_bf16 v[52:55], v[232:235], v[200:203], v[52:55]\n"
    "v_mfma_f32_16x16x32_bf16 v[56:59], v[236:239], v[196:199], v[56:59]\n"
    "v_mfma_f32_16x16x32_bf16 v[56:59], v[240:243], v[200:203], v[56:59]\n"
    "v_mfma_f32_16x16x32_bf16 v[60:63], v[212:215], v[204:207], v[60:63]\n"
    "v_mfma_f32_16x16x32_bf16 v[60:63], v[216:219], v[208:211], v[60:63]\n"
    "v_mfma_f32_16x16x32_bf16 v[64:67], v[220:223], v[204:207], v[64:67]\n"
    "v_mfma_f32_16x16x32_bf16 v[64:67], v[224:227], v[208:211], v[64:67]\n"
    "v_mfma_f32_16x16x32_bf16 v[68:71], v[228:231], v[204:207], v[68:71]\n"
    "v_mfma_f32_16x16x32_bf16 v[68:71], v[232:235], v[208:211], v[68:71]\n"
    "v_mfma_f32_16x16x32_bf16 v[72:75], v[236:239], v[204:207], v[72:75]\n"
    "v_mfma_f32_16x16x32_bf16 v[72:75], v[240:243], v[208:211], v[72:75]\n"
    "s_setprio 0\n"
    "s_waitcnt vmcnt(8)\n"
    "s_barrier\n"
    "; phase 12: step 12 slot 0 stage 0\n"
    "v_readfirstlane_b32 s83, v6\n"
    "s_lshr_b32 s84, s83, 1\n"
    "s_lshl_b32 s85, s90, 1\n"
    "s_lshr_b32 s85, s85, 1\n"
    "s_cmp_eq_u32 s84, s85\n"
    "s_cbranch_scc1 .La3_%=\n"
    "s_memrealtime s[86:87]\n"
    "s_lshr_b32 s82, s54, 12\n"
    "s_and_b32 s82, s82, 0xf80\n"
    "s_add_u32 s82, s82, 0x40000\n"
    ".La4_%=:\n"
    "s_sleep 8\n"
    "buffer_load_dword v6, off, s[44:47], s82 sc0 sc1\n"
    "s_memrealtime s[88:89]\n"
    "s_waitcnt vmcnt(0) lgkmcnt(0)\n"
    "v_readfirstlane_b32 s83, v6\n"
    "s_lshr_b32 s84, s83, 1\n"
    "s_lshl_b32 s85, s90, 1\n"
    "s_lshr_b32 s85, s85, 1\n"
    "s_cmp_eq_u32 s84, s85\n"
    "s_cbranch_scc1 .La3_%=\n"
    "s_sub_u32 s84, s88, s86\n"
    "s_cmp_lt_u32 s84, 0x2faf080\n"
    "s_cbranch_scc1 .La4_%=\n"
    "v_mov_b32 v7, s90\n"
    "buffer_store_dword v7, off, s[44:47], 0 sc0 sc1\n"
    "v_mov_b32 v60, 0x7fc00000\n"
    "v_mov_b32 v61, 0x7fc00000\n"
    "v_mov_b32 v62, 0x7fc00000\n"
    "v_mov_b32 v63, 0x7fc00000\n"
    "v_mov_b32 v64, 0x7fc00000\n"
    "v_mov_b32 v65, 0x7fc00000\n"
    "v_mov_b32 v66, 0x7fc00000\n"
    "v_mov_b32 v67, 0x7fc00000\n"
    "v_mov_b32 v68, 0x7fc00000\n"
    "v_mov_b32 v69, 0x7fc00000\n"
    "v_mov_b32 v70, 0x7fc00000\n"
    "v_mov_b32 v71, 0x7fc00000\n"
    "v_mov_b32 v72, 0x7fc00000\n"
    "v_mov_b32 v73, 0x7fc00000\n"
    "v_mov_b32 v74, 0x7fc00000\n"
    "v_mov_b32 v75, 0x7fc00000\n"
    "s_waitcnt vmcnt(0)\n"
    ".La3_%=:\n"
    "s_bitcmp1_b32 s83, 0\n"
    "s_cselect_b32 s67, s68, s67\n"
    "ds_read_b128 v[180:183], v244 offset:0\n"
    "ds_read_b128 v[184:187], v245 offset:0\n"
    "ds_read_b128 v[188:191], v244 offset:2048\n"
    "ds_read_b128 v[192:195], v245 offset:2048\n"
    "s_add_u32 m0, s51, 0xc000\n"
    "s_xor_b32 s82, s65, 0xd00000\n"
    "buffer_load_dwordx4 v251, s[40:43], s82 offen  lds\n"
    "ds_read_b128 v[196:199], v244 offset:8192\n"
    "ds_read_b128 v[200:203], v245 offset:8192\n"
    "ds_read_b128 v[204:207], v244 offset:10240\n"
    "ds_read_b128 v[208:211], v245 offset:10240\n"
    "s_add_u32 m0, s51, 0xe000\n"
    "s_xor_b32 s82, s65, 0xd80000\n"
    "buffer_load_dwordx4 v251, s[40:43], s82 offen  lds\n"
    "ds_read_b64_tr_b16 v[212:213], v246 offset:0\n"
    "ds_read_b64_tr_b16 v[214:215], v246 offset:1024\n"
    "ds_read_b64_tr_b16 v[216:217], v246 offset:8192\n"
    "ds_read_b64_tr_b16 v[218:219], v246 offset:9216\n"
    "s_add_u32 m0, s50, 0x8000\n"
    "s_xor_b32 s82, s48, 0x200700\n"
    "buffer_load_dwordx4 v250, s[36:39], s82 offen  lds\n"
    "ds_read_b64_tr_b16 v[220:221], v247 offset:0\n"
    "ds_read_b64_tr_b16 v[222:223], v247 offset:1024\n"
    "ds_read_b64_tr_b16 v[224:225], v247 offset:8192\n"
    "ds_read_b64_tr_b16 v[226:227], v247 offset:9216\n"
    "s_add_u32 m0, s50, 0x9000\n"
    "s_xor_b32 s82, s48, 0x280700\n"
    "buffer_load_dwordx4 v250, s[36:39], s82 offen  lds\n"
    "ds_read_b64_tr_b16 v[228:229], v248 offset:0\n"
    "ds_read_b64_tr_b16 v[230:231], v248 offset:1024\n"
    "ds_read_b64_tr_b16 v[232:233], v248 offset:8192\n"
    "ds_read_b64_tr_b16 v[234:235], v248 offset:9216\n"
    "s_add_u32 m0, s50, 0xa000\n"
    "s_xor_b32 s82, s48, 0x700\n"
    "buffer_load_dwordx4 v250, s[36:39], s82 offen  lds\n"
    "ds_read_b64_tr_b16 v[236:237], v249 offset:0\n"
    "ds_read_b64_tr_b16 v[238:239], v249 offset:1024\n"
    "ds_read_b64_tr_b16 v[240:241], v249 offset:8192\n"
    "ds_read_b64_tr_b16 v[242:243], v249 offset:9216\n"
    "s_add_u32 m0, s50, 0xb000\n"
    "s_xor_b32 s82, s48, 0x80700\n"
    "buffer_load_dwordx4 v250, s[36:39], s82 offen  lds\n"
    "s_waitcnt vmcnt(12) lgkmcnt(0)\n"
    "s_barrier\n"
    "s_setprio 1\n"
    "v_mfma_f32_16x16x32_bf16 v[12:15], v[212:215], v[180:183], v[12:15]\n"
    "v_mfma_f32_16x16x32_bf16 v[12:15], v[216:219], v[184:187], v[12:15]\n"
    "v_mfma_f32_16x16x32_bf16 v[16:19], v[220:223], v[180:183], v[16:19]\n"
    "v_mfma_f32_16x16x32_bf16 v[16:19], v[224:227], v[184:187], v[16:19]\n"
    "v_mfma_f32_16x16x32_bf16 v[20:23], v[228:231], v[180:183], v[20:23]\n"
    "v_mfma_f32_16x16x32_bf16 v[20:23], v[232:235], v[184:187], v[20:23]\n"
    "v_mfma_f32_16x16x32_bf16 v[24:27], v[236:239], v[180:183], v[24:27]\n"
    "v_mfma_f32_16x16x32_bf16 v[24:27], v[240:243], v[184:187], v[24:27]\n"
    "v_mfma_f32_16x16x32_bf16 v[28:31], v[212:215], v[188:191], v[28:31]\n"
    "v_mfma_f32_16x16x32_bf16 v[28:31], v[216:219], v[192:195], v[28:31]\n"
    "v_mfma_f32_16x16x32_bf16 v[32:35], v[220:223], v[188:191], v[32:35]\n"
    "v_mfma_f32_16x16x32_bf16 v[32:35], v[224:227], v[192:195], v[32:35]\n"
    "v_mfma_f32_16x16x32_bf16 v[36:39], v[228:231], v[188:191], v[36:39]\n"
    "v_mfma_f32_16x16x32_bf16 v[36:39], v[232:235], v[192:195], v[36:39]\n"
    "v_mfma_f32_16x16x32_bf16 v[40:43], v[236:239], v[188:191], v[40:43]\n"
    "v_mfma_f32_16x16x32_bf16 v[40:43], v[240:243], v[192:195], v[40:43]\n"
    "v_mfma_f32_16x16x32_bf16 v[44:47], v[212:215], v[196:199], v[44:47]\n"
    "v_mfma_f32_16x16x32_bf16 v[44:47], v[216:219], v[200:203], v[44:47]\n"
    "v_mfma_f32_16x16x32_bf16 v[48:51], v[220:223], v[196:199], v[48:51]\n"
    "v_mfma_f32_16x16x32_bf16 v[48:51], v[224:227], v[200:203], v[48:51]\n"
    "v_mfma_f32_16x16x32_bf16 v[52:55], v[228:231], v[196:199], v[52:55]\n"
    "v_mfma_f32_16x16x32_bf16 v[52:55], v[232:235], v[200:203], v[52:55]\n"
    "v_mfma_f32_16x16x32_bf16 v[56:59], v[236:239], v[196:199], v[56:59]\n"
    "v_mfma_f32_16x16x32_bf16 v[56:59], v[240:243], v[200:203], v[56:59]\n"
    "v_mfma_f32_16x16x32_bf16 v[60:63], v[212:215], v[204:207], v[60:63]\n"
    "v_mfma_f32_16x16x32_bf16 v[60:63], v[216:219], v[208:211], v[60:63]\n"
    "v_mfma_f32_16x16x32_bf16 v[64:67], v[220:223], v[204:207], v[64:67]\n"
    "v_mfma_f32_16x16x32_bf16 v[64:67], v[224:227], v[208:211], v[64:67]\n"
    "v_mfma_f32_16x16x32_bf16 v[68:71], v[228:231], v[204:207], v[68:71]\n"
    "v_mfma_f32_16x16x32_bf16 v[68:71], v[232:235], v[208:211], v[68:71]\n"
    "v_mfma_f32_16x16x32_bf16 v[72:75], v[236:239], v[204:207], v[72:75]\n"
    "v_mfma_f32_16x16x32_bf16 v[72:75], v[240:243], v[208:211], v[72:75]\n"
    "s_setprio 0\n"
    "s_barrier\n"
    "; phase 13: step 12 slot 1 stage 1\n"
    "ds_read_b64_tr_b16 v[212:213], v246 offset:16384\n"
    "ds_read_b64_tr_b16 v[214:215], v246 offset:17408\n"
    "s_add_u32 m0, s51, 0x0\n"
    "s_xor_b32 s82, s64, 0xe00000\n"
    "buffer_load_dwordx4 v251, s[40:43], s82 offen  lds\n"
    "ds_read_b64_tr_b16 v[216:217], v246 offset:24576\n"
    "ds_read_b64_tr_b16 v[218:219], v246 offset:25600\n"
    "s_add_u32 m0, s51, 0x2000\n"
    "s_xor_b32 s82, s64, 0xe80000\n"
    "buffer_load_dwordx4 v251, s[40:43], s82 offen  lds\n"
    "ds_read_b64_tr_b16 v[220:221], v247 offset:16384\n"
    "ds_read_b64_tr_b16 v[222:223], v247 offset:17408\n"
    "s_add_u32 m0, s50, 0x0\n"
    "s_xor_b32 s82, s48, 0x200780\n"
    "buffer_load_dwordx4 v250, s[36:39], s82 offen  lds\n"
    "ds_read_b64_tr_b16 v[224:225], v247 offset:24576\n"
    "ds_read_b64_tr_b16 v[226:227], v247 offset:25600\n"
    "s_add_u32 m0, s50, 0x1000\n"
    "s_xor_b32 s82, s48, 0x280780\n"
    "buffer_load_dwordx4 v250, s[36:39], s82 offen  lds\n"
    "ds_read_b64_tr_b16 v[228:229], v248 offset:16384\n"
    "ds_read_b64_tr_b16 v[230:231], v248 offset:17408\n"
    "s_add_u32 m0, s50, 0x2000\n"
    "s_xor_b32 s82, s48, 0x780\n"
    "buffer_load_dwordx4 v250, s[36:39], s82 offen  lds\n"
    "ds_read_b64_tr_b16 v[232:233], v248 offset:24576\n"
    "ds_read_b64_tr_b16 v[234:235], v248 offset:25600\n"
    "s_add_u32 m0, s50, 0x3000\n"
    "s_xor_b32 s82, s48, 0x80780\n"
    "buffer_load_dwordx4 v250, s[36:39], s82 offen  lds\n"
    "ds_read_b64_tr_b16 v[236:237], v249 offset:16384\n"
    "ds_read_b64_tr_b16 v[238:239], v249 offset:17408\n"
    "ds_read_b64_tr_b16 v[240:241], v249 offset:24576\n"
    "ds_read_b64_tr_b16 v[242:243], v249 offset:25600\n"
    "s_waitcnt vmcnt(16) lgkmcnt(0)\n"
    "s_barrier\n"
    "s_setprio 1\n"
    "v_mfma_f32_16x16x32_bf16 v[76:79], v[212:215], v[180:183], 0\n"
    "v_mfma_f32_16x16x32_bf16 v[76:79], v[216:219], v[184:187], v[76:79]\n"
    "v_mfma_f32_16x16x32_bf16 v[80:83], v[220:223], v[180:183], 0\n"
    "v_mfma_f32_16x16x32_bf16 v[80:83], v[224:227], v[184:187], v[80:83]\n"
    "v_mfma_f32_16x16x32_bf16 v[84:87], v[228:231], v[180:183], 0\n"
    "v_mfma_f32_16x16x32_bf16 v[84:87], v[232:235], v[184:187], v[84:87]\n"
    "v_mfma_f32_16x16x32_bf16 v[88:91], v[236:239], v[180:183], 0\n"
    "v_mfma_f32_16x16x32_bf16 v[88:91], v[240:243], v[184:187], v[88:91]\n"
    "v_mfma_f32_16x16x32_bf16 v[92:95], v[212:215], v[188:191], 0\n"
    "v_mfma_f32_16x16x32_bf16 v[92:95], v[216:219], v[192:195], v[92:95]\n"
    "v_mfma_f32_16x16x32_bf16 v[96:99], v[220:223], v[188:191], 0\n"
    "v_mfma_f32_16x16x32_bf16 v[96:99], v[224:227], v[192:195], v[96:99]\n"
    "v_mfma_f32_16x16x32_bf16 v[100:103], v[228:231], v[188:191], 0\n"
    "v_mfma_f32_16x16x32_bf16 v[100:103], v[232:235], v[192:195], v[100:103]\n"
    "v_mfma_f32_16x16x32_bf16 v[104:107], v[236:239], v[188:191], 0\n"
    "v_mfma_f32_16x16x32_bf16 v[104:107], v[240:243], v[192:195], v[104:107]\n"
    "v_mfma_f32_16x16x32_bf16 v[108:111], v[212:215], v[196:199], 0\n"
    "v_mfma_f32_16x16x32_bf16 v[108:111], v[216:219], v[200:203], v[108:111]\n"
    "v_mfma_f32_16x16x32_bf16 v[112:115], v[220:223], v[196:199], 0\n"
    "v_mfma_f32_16x16x32_bf16 v[112:115], v[224:227], v[200:203], v[112:115]\n"
    "v_mfma_f32_16x16x32_bf16 v[116:119], v[228:231], v[196:199], 0\n"
    "v_mfma_f32_16x16x32_bf16 v[116:119], v[232:235], v[200:203], v[116:119]\n"
    "v_mfma_f32_16x16x32_bf16 v[120:123], v[236:239], v[196:199], 0\n"
    "v_mfma_f32_16x16x32_bf16 v[120:123], v[240:243], v[200:203], v[120:123]\n"
    "v_mfma_f32_16x16x32_bf16 v[124:127], v[212:215], v[204:207], 0\n"
    "v_mfma_f32_16x16x32_bf16 v[124:127], v[216:219], v[208:211], v[124:127]\n"
    "v_mfma_f32_16x16x32_bf16 v[128:131], v[220:223], v[204:207], 0\n"
    "v_mfma_f32_16x16x32_bf16 v[128:131], v[224:227], v[208:211], v[128:131]\n"
    "v_mfma_f32_16x16x32_bf16 v[132:135], v[228:231], v[204:207], 0\n"
    "v_mfma_f32_16x16x32_bf16 v[132:135], v[232:235], v[208:211], v[132:135]\n"
    "v_mfma_f32_16x16x32_bf16 v[136:139], v[236:239], v[204:207], 0\n"
    "v_mfma_f32_16x16x32_bf16 v[136:139], v[240:243], v[208:211], v[136:139]\n"
    "s_setprio 0\n"
    "s_waitcnt vmcnt(12)\n"
    "s_barrier\n"
    "; phase 14: step 13 slot 0 stage 0\n"
    "s_bitcmp1_b32 s55, 8\n"
    "s_cbranch_scc1 .La5_%=\n"
    "s_sleep 2\n"
    "s_branch .La6_%=\n"
    ".La5_%=:\n"
    "s_sleep 4\n"
    ".La6_%=:\n"
    "ds_read_b128 v[180:183], v244 offset:16384\n"
    "ds_read_b128 v[184:187], v245 offset:16384\n"
    "ds_read_b128 v[188:191], v244 offset:18432\n"
    "ds_read_b128 v[192:195], v245 offset:18432\n"
    "s_add_u32 m0, s51, 0x4000\n"
    "s_xor_b32 s82, s65, 0xe00000\n"
    "buffer_load_dwordx4 v251, s[40:43], s82 offen  lds\n"
    "ds_read_b128 v[196:199], v244 offset:24576\n"
    "ds_read_b128 v[200:203], v245 offset:24576\n"
    "ds_read_b128 v[204:207], v244 offset:26624\n"
    "ds_read_b128 v[208:211], v245 offset:26624\n"
    "s_add_u32 m0, s51, 0x6000\n"
    "s_xor_b32 s82, s65, 0xe80000\n"
    "buffer_load_dwordx4 v251, s[40:43], s82 offen  lds\n"
    "ds_read_b64_tr_b16 v[212:213], v246 offset:32768\n"
    "ds_read_b64_tr_b16 v[214:215], v246 offset:33792\n"
    "ds_read_b64_tr_b16 v[216:217], v246 offset:40960\n"
    "ds_read_b64_tr_b16 v[218:219], v246 offset:41984\n"
    "ds_read_b64_tr_b16 v[220:221], v247 offset:32768\n"
    "ds_read_b64_tr_b16 v[222:223], v247 offset:33792\n"
    "ds_read_b64_tr_b16 v[224:225], v247 offset:40960\n"
    "ds_read_b64_tr_b16 v[226:227], v247 offset:41984\n"
    "ds_read_b64_tr_b16 v[228:229], v248 offset:32768\n"
    "ds_read_b64_tr_b16 v[230:231], v248 offset:33792\n"
    "ds_read_b64_tr_b16 v[232:233], v248 offset:40960\n"
    "ds_read_b64_tr_b16 v[234:235], v248 offset:41984\n"
    "ds_read_b64_tr_b16 v[236:237], v249 offset:32768\n"
    "ds_read_b64_tr_b16 v[238:239], v249 offset:33792\n"
    "ds_read_b64_tr_b16 v[240:241], v249 offset:40960\n"
    "ds_read_b64_tr_b16 v[242:243], v249 offset:41984\n"
    "s_waitcnt vmcnt(12) lgkmcnt(0)\n"
    "s_barrier\n"
    "s_setprio 1\n"
    "v_mfma_f32_16x16x32_bf16 v[12:15], v[212:215], v[180:183], v[12:15]\n"
    "v_mfma_f32_16x16x32_bf16 v[12:15], v[216:219], v[184:187], v[12:15]\n"
    "v_mfma_f32_16x16x32_bf16 v[16:19], v[220:223], v[180:183], v[16:19]\n"
    "v_mfma_f32_16x16x32_bf16 v[16:19], v[224:227], v[184:187], v[16:19]\n"
    "v_mfma_f32_16x16x32_bf16 v[20:23], v[228:231], v[180:183], v[20:23]\n"
    "v_mfma_f32_16x16x32_bf16 v[20:23], v[232:235], v[184:187], v[20:23]\n"
    "v_mfma_f32_16x16x32_bf16 v[24:27], v[236:239], v[180:183], v[24:27]\n"
    "v_mfma_f32_16x16x32_bf16 v[24:27], v[240:243], v[184:187], v[24:27]\n"
    "v_mfma_f32_16x16x32_bf16 v[28:31], v[212:215], v[188:191], v[28:31]\n"
    "v_mfma_f32_16x16x32_bf16 v[28:31], v[216:219], v[192:195], v[28:31]\n"
    "v_mfma_f32_16x16x32_bf16 v[32:35], v[220:223], v[188:191], v[32:35]\n"
    "v_mfma_f32_16x16x32_bf16 v[32:35], v[224:227], v[192:195], v[32:35]\n"
    "v_mfma_f32_16x16x32_bf16 v[36:39], v[228:231], v[188:191], v[36:39]\n"
    "v_mfma_f32_16x16x32_bf16 v[36:39], v[232:235], v[192:195], v[36:39]\n"
    "v_mfma_f32_16x16x32_bf16 v[40:43], v[236:239], v[188:191], v[40:43]\n"
    "v_mfma_f32_16x16x32_bf16 v[40:43], v[240:243], v[192:195], v[40:43]\n"
    "v_mfma_f32_16x16x32_bf16 v[44:47], v[212:215], v[196:199], v[44:47]\n"
    "v_mfma_f32_16x16x32_bf16 v[44:47], v[216:219], v[200:203], v[44:47]\n"
    "v_mfma_f32_16x16x32_bf16 v[48:51], v[220:223], v[196:199], v[48:51]\n"
    "v_mfma_f32_16x16x32_bf16 v[48:51], v[224:227], v[200:203], v[48:51]\n"
    "v_mfma_f32_16x16x32_bf16 v[52:55], v[228:231], v[196:199], v[52:55]\n"
    "v_mfma_f32_16x16x32_bf16 v[52:55], v[232:235], v[200:203], v[52:55]\n"
    "v_mfma_f32_16x16x32_bf16 v[56:59], v[236:239], v[196:199], v[56:59]\n"
    "v_mfma_f32_16x16x32_bf16 v[56:59], v[240:243], v[200:203], v[56:59]\n"
    "v_mfma_f32_16x16x32_bf16 v[60:63], v[212:215], v[204:207], v[60:63]\n"
    "v_mfma_f32_16x16x32_bf16 v[60:63], v[216:219], v[208:211], v[60:63]\n"
    "v_mfma_f32_16x16x32_bf16 v[64:67], v[220:223], v[204:207], v[64:67]\n"
    "v_mfma_f32_16x16x32_bf16 v[64:67], v[224:227], v[208:211], v[64:67]\n"
    "v_mfma_f32_16x16x32_bf16 v[68:71], v[228:231], v[204:207], v[68:71]\n"
    "v_mfma_f32_16x16x32_bf16 v[68:71], v[232:235], v[208:211], v[68:71]\n"
    "v_mfma_f32_16x16x32_bf16 v[72:75], v[236:239], v[204:207], v[72:75]\n"
    "v_mfma_f32_16x16x32_bf16 v[72:75], v[240:243], v[208:211], v[72:75]\n"
    "s_setprio 0\n"
    "s_barrier\n"
    "; phase 15: step 13 slot 1 stage 1\n"
    "s_bitcmp1_b32 s55, 8\n"
    "s_cbranch_scc1 .La7_%=\n"
    "s_sleep 2\n"
    "s_branch .La8_%=\n"
    ".La7_%=:\n"
    "s_sleep 4\n"
    ".La8_%=:\n"
    "ds_read_b64_tr_b16 v[212:213], v246 offset:49152\n"
    "ds_read_b64_tr_b16 v[214:215], v246 offset:50176\n"
    "s_add_u32 m0, s51, 0x8000\n"
    "s_xor_b32 s82, s64, 0xf00000\n"
    "buffer_load_dwordx4 v251, s[40:43], s82 offen  lds\n"
    "ds_read_b64_tr_b16 v[216:217], v246 offset:57344\n"
    "ds_read_b64_tr_b16 v[218:219], v246 offset:58368\n"
    "s_add_u32 m0, s51, 0xa000\n"
    "s_xor_b32 s82, s64, 0xf80000\n"
    "buffer_load_dwordx4 v251, s[40:43], s82 offen  lds\n"
    "ds_read_b64_tr_b16 v[220:221], v247 offset:49152\n"
    "ds_read_b64_tr_b16 v[222:223], v247 offset:50176\n"
    "s_add_u32 m0, s50, 0x4000\n"
    "s_xor_b32 s82, s48, 0x200800\n"
    "buffer_load_dwordx4 v250, s[36:39], s82 offen  lds\n"
    "ds_read_b64_tr_b16 v[224:225], v247 offset:57344\n"
    "ds_read_b64_tr_b16 v[226:227], v247 offset:58368\n"
    "s_add_u32 m0, s50, 0x5000\n"
    "s_xor_b32 s82, s48, 0x280800\n"
    "buffer_load_dwordx4 v250, s[36:39], s82 offen  lds\n"
    "ds_read_b64_tr_b16 v[228:229], v248 offset:49152\n"
    "ds_read_b64_tr_b16 v[230:231], v248 offset:50176\n"
    "s_add_u32 m0, s50, 0x6000\n"
    "s_xor_b32 s82, s48, 0x800\n"
    "buffer_load_dwordx4 v250, s[36:39], s82 offen  lds\n"
    "ds_read_b64_tr_b16 v[232:233], v248 offset:57344\n"
    "ds_read_b64_tr_b16 v[234:235], v248 offset:58368\n"
    "s_add_u32 m0, s50, 0x7000\n"
    "s_xor_b32 s82, s48, 0x80800\n"
    "buffer_load_dwordx4 v250, s[36:39], s82 offen  lds\n"
    "ds_read_b64_tr_b16 v[236:237], v249 offset:49152\n"
    "ds_read_b64_tr_b16 v[238:239], v249 offset:50176\n"
    "ds_read_b64_tr_b16 v[240:241], v249 offset:57344\n"
    "ds_read_b64_tr_b16 v[242:243], v249 offset:58368\n"
    "s_waitcnt vmcnt(12) lgkmcnt(0)\n"
    "s_barrier\n"
    "s_setprio 1\n"
    "v_mfma_f32_16x16x32_bf16 v[76:79], v[212:215], v[180:183], v[76:79]\n"
    "v_mfma_f32_16x16x32_bf16 v[76:79], v[216:219], v[184:187], v[76:79]\n"
    "v_mfma_f32_16x16x32_bf16 v[80:83], v[220:223], v[180:183], v[80:83]\n"
    "v_mfma_f32_16x16x32_bf16 v[80:83], v[224:227], v[184:187], v[80:83]\n"
    "v_mfma_f32_16x16x32_bf16 v[84:87], v[228:231], v[180:183], v[84:87]\n"
    "v_mfma_f32_16x16x32_bf16 v[84:87], v[232:235], v[184:187], v[84:87]\n"
    "v_mfma_f32_16x16x32_bf16 v[88:91], v[236:239], v[180:183], v[88:91]\n"
    "v_mfma_f32_16x16x32_bf16 v[88:91], v[240:243], v[184:187], v[88:91]\n"
    "v_mfma_f32_16x16x32_bf16 v[92:95], v[212:215], v[188:191], v[92:95]\n"
    "v_mfma_f32_16x16x32_bf16 v[92:95], v[216:219], v[192:195], v[92:95]\n"
    "v_mfma_f32_16x16x32_bf16 v[96:99], v[220:223], v[188:191], v[96:99]\n"
    "v_mfma_f32_16x16x32_bf16 v[96:99], v[224:227], v[192:195], v[96:99]\n"
    "v_mfma_f32_16x16x32_bf16 v[100:103], v[228:231], v[188:191], v[100:103]\n"
    "v_mfma_f32_16x16x32_bf16 v[100:103], v[232:235], v[192:195], v[100:103]\n"
    "v_mfma_f32_16x16x32_bf16 v[104:107], v[236:239], v[188:191], v[104:107]\n"
    "v_mfma_f32_16x16x32_bf16 v[104:107], v[240:243], v[192:195], v[104:107]\n"
    "v_mfma_f32_16x16x32_bf16 v[108:111], v[212:215], v[196:199], v[108:111]\n"
    "v_mfma_f32_16x16x32_bf16 v[108:111], v[216:219], v[200:203], v[108:111]\n"
    "v_mfma_f32_16x16x32_bf16 v[112:115], v[220:223], v[196:199], v[112:115]\n"
    "v_mfma_f32_16x16x32_bf16 v[112:115], v[224:227], v[200:203], v[112:115]\n"
    "v_mfma_f32_16x16x32_bf16 v[116:119], v[228:231], v[196:199], v[116:119]\n"
    "v_mfma_f32_16x16x32_bf16 v[116:119], v[232:235], v[200:203], v[116:119]\n"
    "v_mfma_f32_16x16x32_bf16 v[120:123], v[236:239], v[196:199], v[120:123]\n"
    "v_mfma_f32_16x16x32_bf16 v[120:123], v[240:243], v[200:203], v[120:123]\n"
    "v_mfma_f32_16x16x32_bf16 v[124:127], v[212:215], v[204:207], v[124:127]\n"
    "v_mfma_f32_16x16x32_bf16 v[124:127], v[216:219], v[208:211], v[124:127]\n"
    "v_mfma_f32_16x16x32_bf16 v[128:131], v[220:223], v[204:207], v[128:131]\n"
    "v_mfma_f32_16x16x32_bf16 v[128:131], v[224:227], v[208:211], v[128:131]\n"
    "v_mfma_f32_16x16x32_bf16 v[132:135], v[228:231], v[204:207], v[132:135]\n"
    "v_mfma_f32_16x16x32_bf16 v[132:135], v[232:235], v[208:211], v[132:135]\n"
    "v_mfma_f32_16x16x32_bf16 v[136:139], v[236:239], v[204:207], v[136:139]\n"
    "v_mfma_f32_16x16x32_bf16 v[136:139], v[240:243], v[208:211], v[136:139]\n"
    "s_setprio 0\n"
    "s_barrier\n"
    "; phase 16: step 14 slot 0 stage 0\n"
    "s_bitcmp1_b32 s55, 8\n"
    "s_cbranch_scc1 .La9_%=\n"
    "s_sleep 2\n"
    "s_branch .La10_%=\n"
    ".La9_%=:\n"
    "s_sleep 4\n"
    ".La10_%=:\n"
    "ds_read_b128 v[180:183], v244 offset:32768\n"
    "ds_read_b128 v[184:187], v245 offset:32768\n"
    "ds_read_b128 v[188:191], v244 offset:34816\n"
    "ds_read_b128 v[192:195], v245 offset:34816\n"
    "s_add_u32 m0, s51, 0xc000\n"
    "s_xor_b32 s82, s65, 0xf00000\n"
    "buffer_load_dwordx4 v251, s[40:43], s82 offen  lds\n"
    "ds_read_b128 v[196:199], v244 offset:40960\n"
    "ds_read_b128 v[200:203], v245 offset:40960\n"
    "ds_read_b128 v[204:207], v244 offset:43008\n"
    "ds_read_b128 v[208:211], v245 offset:43008\n"
    "s_add_u32 m0, s51, 0xe000\n"
    "s_xor_b32 s82, s65, 0xf80000\n"
    "buffer_load_dwordx4 v251, s[40:43], s82 offen  lds\n"
    "ds_read_b64_tr_b16 v[212:213], v246 offset:0\n"
    "ds_read_b64_tr_b16 v[214:215], v246 offset:1024\n"
    "ds_read_b64_tr_b16 v[216:217], v246 offset:8192\n"
    "ds_read_b64_tr_b16 v[218:219], v246 offset:9216\n"
    "ds_read_b64_tr_b16 v[220:221], v247 offset:0\n"
    "ds_read_b64_tr_b16 v[222:223], v247 offset:1024\n"
    "ds_read_b64_tr_b16 v[224:225], v247 offset:8192\n"
    "ds_read_b64_tr_b16 v[226:227], v247 offset:9216\n"
    "ds_read_b64_tr_b16 v[228:229], v248 offset:0\n"
    "ds_read_b64_tr_b16 v[230:231], v248 offset:1024\n"
    "ds_read_b64_tr_b16 v[232:233], v248 offset:8192\n"
    "ds_read_b64_tr_b16 v[234:235], v248 offset:9216\n"
    "ds_read_b64_tr_b16 v[236:237], v249 offset:0\n"
    "ds_read_b64_tr_b16 v[238:239], v249 offset:1024\n"
    "ds_read_b64_tr_b16 v[240:241], v249 offset:8192\n"
    "ds_read_b64_tr_b16 v[242:243], v249 offset:9216\n"
    "s_waitcnt vmcnt(8) lgkmcnt(0)\n"
    "s_barrier\n"
    "s_setprio 1\n"
    "v_mfma_f32_16x16x32_bf16 v[12:15], v[212:215], v[180:183], v[12:15]\n"
    "v_mfma_f32_16x16x32_bf16 v[12:15], v[216:219], v[184:187], v[12:15]\n"
    "v_mfma_f32_16x16x32_bf16 v[16:19], v[220:223], v[180:183], v[16:19]\n"
    "v_mfma_f32_16x16x32_bf16 v[16:19], v[224:227], v[184:187], v[16:19]\n"
    "v_mfma_f32_16x16x32_bf16 v[20:23], v[228:231], v[180:183], v[20:23]\n"
    "v_mfma_f32_16x16x32_bf16 v[20:23], v[232:235], v[184:187], v[20:23]\n"
    "v_mfma_f32_16x16x32_bf16 v[24:27], v[236:239], v[180:183], v[24:27]\n"
    "v_mfma_f32_16x16x32_bf16 v[24:27], v[240:243], v[184:187], v[24:27]\n"
    "v_mfma_f32_16x16x32_bf16 v[28:31], v[212:215], v[188:191], v[28:31]\n"
    "v_mfma_f32_16x16x32_bf16 v[28:31], v[216:219], v[192:195], v[28:31]\n"
    "v_mfma_f32_16x16x32_bf16 v[32:35], v[220:223], v[188:191], v[32:35]\n"
    "v_mfma_f32_16x16x32_bf16 v[32:35], v[224:227], v[192:195], v[32:35]\n"
    "v_mfma_f32_16x16x32_bf16 v[36:39], v[228:231], v[188:191], v[36:39]\n"
    "v_mfma_f32_16x16x32_bf16 v[36:39], v[232:235], v[192:195], v[36:39]\n"
    "v_mfma_f32_16x16x32_bf16 v[40:43], v[236:239], v[188:191], v[40:43]\n"
    "v_mfma_f32_16x16x32_bf16 v[40:43], v[240:243], v[192:195], v[40:43]\n"
    "v_mfma_f32_16x16x32_bf16 v[44:47], v[212:215], v[196:199], v[44:47]\n"
    "v_mfma_f32_16x16x32_bf16 v[44:47], v[216:219], v[200:203], v[44:47]\n"
    "v_mfma_f32_16x16x32_bf16 v[48:51], v[220:223], v[196:199], v[48:51]\n"
    "v_mfma_f32_16x16x32_bf16 v[48:51], v[224:227], v[200:203], v[48:51]\n"
    "v_mfma_f32_16x16x32_bf16 v[52:55], v[228:231], v[196:199], v[52:55]\n"
    "v_mfma_f32_16x16x32_bf16 v[52:55], v[232:235], v[200:203], v[52:55]\n"
    "v_mfma_f32_16x16x32_bf16 v[56:59], v[236:239], v[196:199], v[56:59]\n"
    "v_mfma_f32_16x16x32_bf16 v[56:59], v[240:243], v[200:203], v[56:59]\n"
    "v_mfma_f32_16x16x32_bf16 v[60:63], v[212:215], v[204:207], v[60:63]\n"
    "v_mfma_f32_16x16x32_bf16 v[60:63], v[216:219], v[208:211], v[60:63]\n"
    "v_mfma_f32_16x16x32_bf16 v[64:67], v[220:223], v[204:207], v[64:67]\n"
    "v_mfma_f32_16x16x32_bf16 v[64:67], v[224:227], v[208:211], v[64:67]\n"
    "v_mfma_f32_16x16x32_bf16 v[68:71], v[228:231], v[204:207], v[68:71]\n"
    "v_mfma_f32_16x16x32_bf16 v[68:71], v[232:235], v[208:211], v[68:71]\n"
    "v_mfma_f32_16x16x32_bf16 v[72:75], v[236:239], v[204:207], v[72:75]\n"
    "v_mfma_f32_16x16x32_bf16 v[72:75], v[240:243], v[208:211], v[72:75]\n"
    "s_setprio 0\n"
    "s_barrier\n"
    "; phase 17: step 14 slot 1 stage 1\n"
    "s_bitcmp1_b32 s55, 8\n"
    "s_cbranch_scc1 .La11_%=\n"
    "s_sleep 2\n"
    "s_branch .La12_%=\n"
    ".La11_%=:\n"
    "s_sleep 4\n"
    ".La12_%=:\n"
    "ds_read_b64_tr_b16 v[212:213], v246 offset:16384\n"
    "ds_read_b64_tr_b16 v[214:215], v246 offset:17408\n"
    "s_add_u32 m0, s51, 0x0\n"
    "s_xor_b32 s82, s64, 0x1000000\n"
    "buffer_load_dwordx4 v251, s[40:43], s82 offen  lds\n"
    "ds_read_b64_tr_b16 v[216:217], v246 offset:24576\n"
    "ds_read_b64_tr_b16 v[218:219], v246 offset:25600\n"
    "s_add_u32 m0, s51, 0x2000\n"
    "s_xor_b32 s82, s64, 0x1080000\n"
    "buffer_load_dwordx4 v251, s[40:43], s82 offen  lds\n"
    "ds_read_b64_tr_b16 v[220:221], v247 offset:16384\n"
    "ds_read_b64_tr_b16 v[222:223], v247 offset:17408\n"
    "s_add_u32 m0, s50, 0x8000\n"
    "s_xor_b32 s82, s48, 0x200880\n"
    "buffer_load_dwordx4 v250, s[36:39], s82 offen  lds\n"
    "ds_read_b64_tr_b16 v[224:225], v247 offset:24576\n"
    "ds_read_b64_tr_b16 v[226:227], v247 offset:25600\n"
    "s_add_u32 m0, s50, 0x9000\n"
    "s_xor_b32 s82, s48, 0x280880\n"
    "buffer_load_dwordx4 v250, s[36:39], s82 offen  lds\n"
    "ds_read_b64_tr_b16 v[228:229], v248 offset:16384\n"
    "ds_read_b64_tr_b16 v[230:231], v248 offset:17408\n"
    "s_add_u32 m0, s50, 0xa000\n"
    "s_xor_b32 s82, s48, 0x880\n"
    "buffer_load_dwordx4 v250, s[36:39], s82 offen  lds\n"
    "ds_read_b64_tr_b16 v[232:233], v248 offset:24576\n"
    "ds_read_b64_tr_b16 v[234:235], v248 offset:25600\n"
    "s_add_u32 m0, s50, 0xb000\n"
    "s_xor_b32 s82, s48, 0x80880\n"
    "buffer_load_dwordx4 v250, s[36:39], s82 offen  lds\n"
    "ds_read_b64_tr_b16 v[236:237], v249 offset:16384\n"
    "ds_read_b64_tr_b16 v[238:239], v249 offset:17408\n"
    "ds_read_b64_tr_b16 v[240:241], v249 offset:24576\n"
    "ds_read_b64_tr_b16 v[242:243], v249 offset:25600\n"
    "s_waitcnt vmcnt(12) lgkmcnt(0)\n"
    "s_barrier\n"
    "s_setprio 1\n"
    "v_mfma_f32_16x16x32_bf16 v[76:79], v[212:215], v[180:183], v[76:79]\n"
    "v_mfma_f32_16x16x32_bf16 v[76:79], v[216:219], v[184:187], v[76:79]\n"
    "v_mfma_f32_16x16x32_bf16 v[80:83], v[220:223], v[180:183], v[80:83]\n"
    "v_mfma_f32_16x16x32_bf16 v[80:83], v[224:227], v[184:187], v[80:83]\n"
    "v_mfma_f32_16x16x32_bf16 v[84:87], v[228:231], v[180:183], v[84:87]\n"
    "v_mfma_f32_16x16x32_bf16 v[84:87], v[232:235], v[184:187], v[84:87]\n"
    "v_mfma_f32_16x16x32_bf16 v[88:91], v[236:239], v[180:183], v[88:91]\n"
    "v_mfma_f32_16x16x32_bf16 v[88:91], v[240:243], v[184:187], v[88:91]\n"
    "v_mfma_f32_16x16x32_bf16 v[92:95], v[212:215], v[188:191], v[92:95]\n"
    "v_mfma_f32_16x16x32_bf16 v[92:95], v[216:219], v[192:195], v[92:95]\n"
    "v_mfma_f32_16x16x32_bf16 v[96:99], v[220:223], v[188:191], v[96:99]\n"
    "v_mfma_f32_16x16x32_bf16 v[96:99], v[224:227], v[192:195], v[96:99]\n"
    "v_mfma_f32_16x16x32_bf16 v[100:103], v[228:231], v[188:191], v[100:103]\n"
    "v_mfma_f32_16x16x32_bf16 v[100:103], v[232:235], v[192:195], v[100:103]\n"
    "v_mfma_f32_16x16x32_bf16 v[104:107], v[236:239], v[188:191], v[104:107]\n"
    "v_mfma_f32_16x16x32_bf16 v[104:107], v[240:243], v[192:195], v[104:107]\n"
    "v_mfma_f32_16x16x32_bf16 v[108:111], v[212:215], v[196:199], v[108:111]\n"
    "v_mfma_f32_16x16x32_bf16 v[108:111], v[216:219], v[200:203], v[108:111]\n"
    "v_mfma_f32_16x16x32_bf16 v[112:115], v[220:223], v[196:199], v[112:115]\n"
    "v_mfma_f32_16x16x32_bf16 v[112:115], v[224:227], v[200:203], v[112:115]\n"
    "v_mfma_f32_16x16x32_bf16 v[116:119], v[228:231], v[196:199], v[116:119]\n"
    "v_mfma_f32_16x16x32_bf16 v[116:119], v[232:235], v[200:203], v[116:119]\n"
    "v_mfma_f32_16x16x32_bf16 v[120:123], v[236:239], v[196:199], v[120:123]\n"
    "v_mfma_f32_16x16x32_bf16 v[120:123], v[240:243], v[200:203], v[120:123]\n"
    "v_mfma_f32_16x16x32_bf16 v[124:127], v[212:215], v[204:207], v[124:127]\n"
    "v_mfma_f32_16x16x32_bf16 v[124:127], v[216:219], v[208:211], v[124:127]\n"
    "v_mfma_f32_16x16x32_bf16 v[128:131], v[220:223], v[204:207], v[128:131]\n"
    "v_mfma_f32_16x16x32_bf16 v[128:131], v[224:227], v[208:211], v[128:131]\n"
    "v_mfma_f32_16x16x32_bf16 v[132:135], v[228:231], v[204:207], v[132:135]\n"
    "v_mfma_f32_16x16x32_bf16 v[132:135], v[232:235], v[208:211], v[132:135]\n"
    "v_mfma_f32_16x16x32_bf16 v[136:139], v[236:239], v[204:207], v[136:139]\n"
    "v_mfma_f32_16x16x32_bf16 v[136:139], v[240:243], v[208:211], v[136:139]\n"
    "s_setprio 0\n"
    "s_barrier\n"
    "; phase 18: step 15 slot 0 stage 0\n"
    "s_bitcmp1_b32 s55, 8\n"
    "s_cbranch_scc1 .La13_%=\n"
    "s_sleep 2\n"
    "s_branch .La14_%=\n"
    ".La13_%=:\n"
    "s_sleep 4\n"
    ".La14_%=:\n"
    "ds_read_b128 v[180:183], v244 offset:0\n"
    "ds_read_b128 v[184:187], v245 offset:0\n"
    "ds_read_b128 v[188:191], v244 offset:2048\n"
    "ds_read_b128 v[192:195], v245 offset:2048\n"
    "s_add_u32 m0, s51, 0x4000\n"
    "s_xor_b32 s82, s65, 0x1000000\n"
    "buffer_load_dwordx4 v251, s[40:43], s82 offen  lds\n"
    "ds_read_b128 v[196:199], v244 offset:8192\n"
    "ds_read_b128 v[200:203], v245 offset:8192\n"
    "ds_read_b128 v[204:207], v244 offset:10240\n"
    "ds_read_b128 v[208:211], v245 offset:10240\n"
    "s_add_u32 m0, s51, 0x6000\n"
    "s_xor_b32 s82, s65, 0x1080000\n"
    "buffer_load_dwordx4 v251, s[40:43], s82 offen  lds\n"
    "ds_read_b64_tr_b16 v[212:213], v246 offset:32768\n"
    "ds_read_b64_tr_b16 v[214:215], v246 offset:33792\n"
    "ds_read_b64_tr_b16 v[216:217], v246 offset:40960\n"
    "ds_read_b64_tr_b16 v[218:219], v246 offset:41984\n"
    "ds_read_b64_tr_b16 v[220:221], v247 offset:32768\n"
    "ds_read_b64_tr_b16 v[222:223], v247 offset:33792\n"
    "ds_read_b64_tr_b16 v[224:225], v247 offset:40960\n"
    "ds_read_b64_tr_b16 v[226:227], v247 offset:41984\n"
    "ds_read_b64_tr_b16 v[228:229], v248 offset:32768\n"
    "ds_read_b64_tr_b16 v[230:231], v248 offset:33792\n"
    "ds_read_b64_tr_b16 v[232:233], v248 offset:40960\n"
    "ds_read_b64_tr_b16 v[234:235], v248 offset:41984\n"
    "ds_read_b64_tr_b16 v[236:237], v249 offset:32768\n"
    "ds_read_b64_tr_b16 v[238:239], v249 offset:33792\n"
    "ds_read_b64_tr_b16 v[240:241], v249 offset:40960\n"
    "ds_read_b64_tr_b16 v[242:243], v249 offset:41984\n"
    "s_waitcnt vmcnt(8) lgkmcnt(0)\n"
    "s_barrier\n"
    "s_setprio 1\n"
    "v_mfma_f32_16x16x32_bf16 v[12:15], v[212:215], v[180:183], v[12:15]\n"
    "v_mfma_f32_16x16x32_bf16 v[12:15], v[216:219], v[184:187], v[12:15]\n"
    "v_mfma_f32_16x16x32_bf16 v[16:19], v[220:223], v[180:183], v[16:19]\n"
    "v_mfma_f32_16x16x32_bf16 v[16:19], v[224:227], v[184:187], v[16:19]\n"
    "v_mfma_f32_16x16x32_bf16 v[20:23], v[228:231], v[180:183], v[20:23]\n"
    "v_mfma_f32_16x16x32_bf16 v[20:23], v[232:235], v[184:187], v[20:23]\n"
    "v_mfma_f32_16x16x32_bf16 v[24:27], v[236:239], v[180:183], v[24:27]\n"
    "v_mfma_f32_16x16x32_bf16 v[24:27], v[240:243], v[184:187], v[24:27]\n"
    "v_mfma_f32_16x16x32_bf16 v[28:31], v[212:215], v[188:191], v[28:31]\n"
    "v_mfma_f32_16x16x32_bf16 v[28:31], v[216:219], v[192:195], v[28:31]\n"
    "v_mfma_f32_16x16x32_bf16 v[32:35], v[220:223], v[188:191], v[32:35]\n"
    "v_mfma_f32_16x16x32_bf16 v[32:35], v[224:227], v[192:195], v[32:35]\n"
    "v_mfma_f32_16x16x32_bf16 v[36:39], v[228:231], v[188:191], v[36:39]\n"
    "v_mfma_f32_16x16x32_bf16 v[36:39], v[232:235], v[192:195], v[36:39]\n"
    "v_mfma_f32_16x16x32_bf16 v[40:43], v[236:239], v[188:191], v[40:43]\n"
    "v_mfma_f32_16x16x32_bf16 v[40:43], v[240:243], v[192:195], v[40:43]\n"
    "v_mfma_f32_16x16x32_bf16 v[44:47], v[212:215], v[196:199], v[44:47]\n"
    "v_mfma_f32_16x16x32_bf16 v[44:47], v[216:219], v[200:203], v[44:47]\n"
    "v_mfma_f32_16x16x32_bf16 v[48:51], v[220:223], v[196:199], v[48:51]\n"
    "v_mfma_f32_16x16x32_bf16 v[48:51], v[224:227], v[200:203], v[48:51]\n"
    "v_mfma_f32_16x16x32_bf16 v[52:55], v[228:231], v[196:199], v[52:55]\n"
    "v_mfma_f32_16x16x32_bf16 v[52:55], v[232:235], v[200:203], v[52:55]\n"
    "v_mfma_f32_16x16x32_bf16 v[56:59], v[236:239], v[196:199], v[56:59]\n"
    "v_mfma_f32_16x16x32_bf16 v[56:59], v[240:243], v[200:203], v[56:59]\n"
    "v_mfma_f32_16x16x32_bf16 v[60:63], v[212:215], v[204:207], v[60:63]\n"
    "v_mfma_f32_16x16x32_bf16 v[60:63], v[216:219], v[208:211], v[60:63]\n"
    "v_mfma_f32_16x16x32_bf16 v[64:67], v[220:223], v[204:207], v[64:67]\n"
    "v_mfma_f32_16x16x32_bf16 v[64:67], v[224:227], v[208:211], v[64:67]\n"
    "v_mfma_f32_16x16x32_bf16 v[68:71], v[228:231], v[204:207], v[68:71]\n"
    "v_mfma_f32_16x16x32_bf16 v[68:71], v[232:235], v[208:211], v[68:71]\n"
    "v_mfma_f32_16x16x32_bf16 v[72:75], v[236:239], v[204:207], v[72:75]\n"
    "v_mfma_f32_16x16x32_bf16 v[72:75], v[240:243], v[208:211], v[72:75]\n"
    "s_setprio 0\n"
    "s_barrier\n"
    "; phase 19: step 15 slot 1 stage 1\n"
    "s_bitcmp1_b32 s55, 8\n"
    "s_cbranch_scc1 .La15_%=\n"
    "s_sleep 2\n"
    "s_branch .La16_%=\n"
    ".La15_%=:\n"
    "s_sleep 4\n"
    ".La16_%=:\n"
    "ds_read_b64_tr_b16 v[212:213], v246 offset:49152\n"
    "ds_read_b64_tr_b16 v[214:215], v246 offset:50176\n"
    "s_add_u32 m0, s51, 0x8000\n"
    "s_xor_b32 s82, s64, 0x1100000\n"
    "buffer_load_dwordx4 v251, s[40:43], s82 offen  lds\n"
    "ds_read_b64_tr_b16 v[216:217], v246 offset:57344\n"
    "ds_read_b64_tr_b16 v[218:219], v246 offset:58368\n"
    "s_add_u32 m0, s51, 0xa000\n"
    "s_xor_b32 s82, s64, 0x1180000\n"
    "buffer_load_dwordx4 v251, s[40:43], s82 offen  lds\n"
    "ds_read_b64_tr_b16 v[220:221], v247 offset:49152\n"
    "ds_read_b64_tr_b16 v[222:223], v247 offset:50176\n"
    "s_add_u32 m0, s50, 0x0\n"
    "s_xor_b32 s82, s48, 0x200900\n"
    "buffer_load_dwordx4 v250, s[36:39], s82 offen  lds\n"
    "ds_read_b64_tr_b16 v[224:225], v247 offset:57344\n"
    "ds_read_b64_tr_b16 v[226:227], v247 offset:58368\n"
    "s_add_u32 m0, s50, 0x1000\n"
    "s_xor_b32 s82, s48, 0x280900\n"
    "buffer_load_dwordx4 v250, s[36:39], s82 offen  lds\n"
    "ds_read_b64_tr_b16 v[228:229], v248 offset:49152\n"
    "ds_read_b64_tr_b16 v[230:231], v248 offset:50176\n"
    "s_add_u32 m0, s50, 0x2000\n"
    "s_xor_b32 s82, s48, 0x900\n"
    "buffer_load_dwordx4 v250, s[36:39], s82 offen  lds\n"
    "ds_read_b64_tr_b16 v[232:233], v248 offset:57344\n"
    "ds_read_b64_tr_b16 v[234:235], v248 offset:58368\n"
    "s_add_u32 m0, s50, 0x3000\n"
    "s_xor_b32 s82, s48, 0x80900\n"
    "buffer_load_dwordx4 v250, s[36:39], s82 offen  lds\n"
    "ds_read_b64_tr_b16 v[236:237], v249 offset:49152\n"
    "ds_read_b64_tr_b16 v[238:239], v249 offset:50176\n"
    "ds_read_b64_tr_b16 v[240:241], v249 offset:57344\n"
    "ds_read_b64_tr_b16 v[242:243], v249 offset:58368\n"
    "s_waitcnt vmcnt(12) lgkmcnt(0)\n"
    "s_barrier\n"
    "s_setprio 1\n"
    "v_mfma_f32_16x16x32_bf16 v[76:79], v[212:215], v[180:183], v[76:79]\n"
    "v_mfma_f32_16x16x32_bf16 v[76:79], v[216:219], v[184:187], v[76:79]\n"
    "v_mfma_f32_16x16x32_bf16 v[80:83], v[220:223], v[180:183], v[80:83]\n"
    "v_mfma_f32_16x16x32_bf16 v[80:83], v[224:227], v[184:187], v[80:83]\n"
    "v_mfma_f32_16x16x32_bf16 v[84:87], v[228:231], v[180:183], v[84:87]\n"
    "v_mfma_f32_16x16x32_bf16 v[84:87], v[232:235], v[184:187], v[84:87]\n"
    "v_mfma_f32_16x16x32_bf16 v[88:91], v[236:239], v[180:183], v[88:91]\n"
    "v_mfma_f32_16x16x32_bf16 v[88:91], v[240:243], v[184:187], v[88:91]\n"
    "v_mfma_f32_16x16x32_bf16 v[92:95], v[212:215], v[188:191], v[92:95]\n"
    "v_mfma_f32_16x16x32_bf16 v[92:95], v[216:219], v[192:195], v[92:95]\n"
    "v_mfma_f32_16x16x32_bf16 v[96:99], v[220:223], v[188:191], v[96:99]\n"
    "v_mfma_f32_16x16x32_bf16 v[96:99], v[224:227], v[192:195], v[96:99]\n"
    "v_mfma_f32_16x16x32_bf16 v[100:103], v[228:231], v[188:191], v[100:103]\n"
    "v_mfma_f32_16x16x32_bf16 v[100:103], v[232:235], v[192:195], v[100:103]\n"
    "v_mfma_f32_16x16x32_bf16 v[104:107], v[236:239], v[188:191], v[104:107]\n"
    "v_mfma_f32_16x16x32_bf16 v[104:107], v[240:243], v[192:195], v[104:107]\n"
    "v_mfma_f32_16x16x32_bf16 v[108:111], v[212:215], v[196:199], v[108:111]\n"
    "v_mfma_f32_16x16x32_bf16 v[108:111], v[216:219], v[200:203], v[108:111]\n"
    "v_mfma_f32_16x16x32_bf16 v[112:115], v[220:223], v[196:199], v[112:115]\n"
    "v_mfma_f32_16x16x32_bf16 v[112:115], v[224:227], v[200:203], v[112:115]\n"
    "v_mfma_f32_16x16x32_bf16 v[116:119], v[228:231], v[196:199], v[116:119]\n"
    "v_mfma_f32_16x16x32_bf16 v[116:119], v[232:235], v[200:203], v[116:119]\n"
    "v_mfma_f32_16x16x32_bf16 v[120:123], v[236:239], v[196:199], v[120:123]\n"
    "v_mfma_f32_16x16x32_bf16 v[120:123], v[240:243], v[200:203], v[120:123]\n"
    "v_mfma_f32_16x16x32_bf16 v[124:127], v[212:215], v[204:207], v[124:127]\n"
    "v_mfma_f32_16x16x32_bf16 v[124:127], v[216:219], v[208:211], v[124:127]\n"
    "v_mfma_f32_16x16x32_bf16 v[128:131], v[220:223], v[204:207], v[128:131]\n"
    "v_mfma_f32_16x16x32_bf16 v[128:131], v[224:227], v[208:211], v[128:131]\n"
    "v_mfma_f32_16x16x32_bf16 v[132:135], v[228:231], v[204:207], v[132:135]\n"
    "v_mfma_f32_16x16x32_bf16 v[132:135], v[232:235], v[208:211], v[132:135]\n"
    "v_mfma_f32_16x16x32_bf16 v[136:139], v[236:239], v[204:207], v[136:139]\n"
    "v_mfma_f32_16x16x32_bf16 v[136:139], v[240:243], v[208:211], v[136:139]\n"
    "s_setprio 0\n"
    "s_barrier\n"
    "; phase 20: step 16 slot 0 stage 0\n"
    "s_bitcmp1_b32 s55, 8\n"
    "s_cbranch_scc1 .La17_%=\n"
    "s_sleep 2\n"
    "s_branch .La18_%=\n"
    ".La17_%=:\n"
    "s_sleep 4\n"
    ".La18_%=:\n"
    "ds_read_b128 v[180:183], v244 offset:16384\n"
    "ds_read_b128 v[184:187], v245 offset:16384\n"
    "ds_read_b128 v[188:191], v244 offset:18432\n"
    "ds_read_b128 v[192:195], v245 offset:18432\n"
    "s_add_u32 m0, s51, 0xc000\n"
    "s_xor_b32 s82, s65, 0x1100000\n"
    "buffer_load_dwordx4 v251, s[40:43], s82 offen  lds\n"
    "ds_read_b128 v[196:199], v244 offset:24576\n"
    "ds_read_b128 v[200:203], v245 offset:24576\n"
    "ds_read_b128 v[204:207], v244 offset:26624\n"
    "ds_read_b128 v[208:211], v245 offset:26624\n"
    "s_add_u32 m0, s51, 0xe000\n"
    "s_xor_b32 s82, s65, 0x1180000\n"
    "buffer_load_dwordx4 v251, s[40:43], s82 offen  lds\n"
    "ds_read_b64_tr_b16 v[212:213], v246 offset:0\n"
    "ds_read_b64_tr_b16 v[214:215], v246 offset:1024\n"
    "ds_read_b64_tr_b16 v[216:217], v246 offset:8192\n"
    "ds_read_b64_tr_b16 v[218:219], v246 offset:9216\n"
    "ds_read_b64_tr_b16 v[220:221], v247 offset:0\n"
    "ds_read_b64_tr_b16 v[222:223], v247 offset:1024\n"
    "ds_read_b64_tr_b16 v[224:225], v247 offset:8192\n"
    "ds_read_b64_tr_b16 v[226:227], v247 offset:9216\n"
    "ds_read_b64_tr_b16 v[228:229], v248 offset:0\n"
    "ds_read_b64_tr_b16 v[230:231], v248 offset:1024\n"
    "ds_read_b64_tr_b16 v[232:233], v248 offset:8192\n"
    "ds_read_b64_tr_b16 v[234:235], v248 offset:9216\n"
    "ds_read_b64_tr_b16 v[236:237], v249 offset:0\n"
    "ds_read_b64_tr_b16 v[238:239], v249 offset:1024\n"
    "ds_read_b64_tr_b16 v[240:241], v249 offset:8192\n"
    "ds_read_b64_tr_b16 v[242:243], v249 offset:9216\n"
    "s_waitcnt vmcnt(8) lgkmcnt(0)\n"
    "s_barrier\n"
    "s_setprio 1\n"
    "v_mfma_f32_16x16x32_bf16 v[12:15], v[212:215], v[180:183], v[12:15]\n"
    "v_mfma_f32_16x16x32_bf16 v[12:15], v[216:219], v[184:187], v[12:15]\n"
    "v_mfma_f32_16x16x32_bf16 v[16:19], v[220:223], v[180:183], v[16:19]\n"
    "v_mfma_f32_16x16x32_bf16 v[16:19], v[224:227], v[184:187], v[16:19]\n"
    "v_mfma_f32_16x16x32_bf16 v[20:23], v[228:231], v[180:183], v[20:23]\n"
    "v_mfma_f32_16x16x32_bf16 v[20:23], v[232:235], v[184:187], v[20:23]\n"
    "v_mfma_f32_16x16x32_bf16 v[24:27], v[236:239], v[180:183], v[24:27]\n"
    "v_mfma_f32_16x16x32_bf16 v[24:27], v[240:243], v[184:187], v[24:27]\n"
    "v_mfma_f32_16x16x32_bf16 v[28:31], v[212:215], v[188:191], v[28:31]\n"
    "v_mfma_f32_16x16x32_bf16 v[28:31], v[216:219], v[192:195], v[28:31]\n"
    "v_mfma_f32_16x16x32_bf16 v[32:35], v[220:223], v[188:191], v[32:35]\n"
    "v_mfma_f32_16x16x32_bf16 v[32:35], v[224:227], v[192:195], v[32:35]\n"
    "v_mfma_f32_16x16x32_bf16 v[36:39], v[228:231], v[188:191], v[36:39]\n"
    "v_mfma_f32_16x16x32_bf16 v[36:39], v[232:235], v[192:195], v[36:39]\n"
    "v_mfma_f32_16x16x32_bf16 v[40:43], v[236:239], v[188:191], v[40:43]\n"
    "v_mfma_f32_16x16x32_bf16 v[40:43], v[240:243], v[192:195], v[40:43]\n"
    "v_mfma_f32_16x16x32_bf16 v[44:47], v[212:215], v[196:199], v[44:47]\n"
    "v_mfma_f32_16x16x32_bf16 v[44:47], v[216:219], v[200:203], v[44:47]\n"
    "v_mfma_f32_16x16x32_bf16 v[48:51], v[220:223], v[196:199], v[48:51]\n"
    "v_mfma_f32_16x16x32_bf16 v[48:51], v[224:227], v[200:203], v[48:51]\n"
    "v_mfma_f32_16x16x32_bf16 v[52:55], v[228:231], v[196:199], v[52:55]\n"
    "v_mfma_f32_16x16x32_bf16 v[52:55], v[232:235], v[200:203], v[52:55]\n"
    "v_mfma_f32_16x16x32_bf16 v[56:59], v[236:239], v[196:199], v[56:59]\n"
    "v_mfma_f32_16x16x32_bf16 v[56:59], v[240:243], v[200:203], v[56:59]\n"
    "v_mfma_f32_16x16x32_bf16 v[60:63], v[212:215], v[204:207], v[60:63]\n"
    "v_mfma_f32_16x16x32_bf16 v[60:63], v[216:219], v[208:211], v[60:63]\n"
    "v_mfma_f32_16x16x32_bf16 v[64:67], v[220:223], v[204:207], v[64:67]\n"
    "v_mfma_f32_16x16x32_bf16 v[64:67], v[224:227], v[208:211], v[64:67]\n"
    "v_mfma_f32_16x16x32_bf16 v[68:71], v[228:231], v[204:207], v[68:71]\n"
    "v_mfma_f32_16x16x32_bf16 v[68:71], v[232:235], v[208:211], v[68:71]\n"
    "v_mfma_f32_16x16x32_bf16 v[72:75], v[236:239], v[204:207], v[72:75]\n"
    "v_mfma_f32_16x16x32_bf16 v[72:75], v[240:243], v[208:211], v[72:75]\n"
    "s_setprio 0\n"
    "s_barrier\n"
    "; phase 21: step 16 slot 1 stage 1\n"
    "s_bitcmp1_b32 s55, 8\n"
    "s_cbranch_scc1 .La19_%=\n"
    "s_sleep 2\n"
    "s_branch .La20_%=\n"
    ".La19_%=:\n"
    "s_sleep 4\n"
    ".La20_%=:\n"
    "ds_read_b64_tr_b16 v[212:213], v246 offset:16384\n"
    "ds_read_b64_tr_b16 v[214:215], v246 offset:17408\n"
    "s_add_u32 m0, s51, 0x0\n"
    "s_xor_b32 s82, s64, 0x1200000\n"
    "buffer_load_dwordx4 v251, s[40:43], s82 offen  lds\n"
    "ds_read_b64_tr_b16 v[216:217], v246 offset:24576\n"
    "ds_read_b64_tr_b16 v[218:219], v246 offset:25600\n"
    "s_add_u32 m0, s51, 0x2000\n"
    "s_xor_b32 s82, s64, 0x1280000\n"
    "buffer_load_dwordx4 v251, s[40:43], s82 offen  lds\n"
    "ds_read_b64_tr_b16 v[220:221], v247 offset:16384\n"
    "ds_read_b64_tr_b16 v[222:223], v247 offset:17408\n"
    "s_add_u32 m0, s50, 0x4000\n"
    "s_xor_b32 s82, s48, 0x200980\n"
    "buffer_load_dwordx4 v250, s[36:39], s82 offen  lds\n"
    "ds_read_b64_tr_b16 v[224:225], v247 offset:24576\n"
    "ds_read_b64_tr_b16 v[226:227], v247 offset:25600\n"
    "s_add_u32 m0, s50, 0x5000\n"
    "s_xor_b32 s82, s48, 0x280980\n"
    "buffer_load_dwordx4 v250, s[36:39], s82 offen  lds\n"
    "ds_read_b64_tr_b16 v[228:229], v248 offset:16384\n"
    "ds_read_b64_tr_b16 v[230:231], v248 offset:17408\n"
    "s_add_u32 m0, s50, 0x6000\n"
    "s_xor_b32 s82, s48, 0x980\n"
    "buffer_load_dwordx4 v250, s[36:39], s82 offen  lds\n"
    "ds_read_b64_tr_b16 v[232:233], v248 offset:24576\n"
    "ds_read_b64_tr_b16 v[234:235], v248 offset:25600\n"
    "s_add_u32 m0, s50, 0x7000\n"
    "s_xor_b32 s82, s48, 0x80980\n"
    "buffer_load_dwordx4 v250, s[36:39], s82 offen  lds\n"
    "ds_read_b64_tr_b16 v[236:237], v249 offset:16384\n"
    "ds_read_b64_tr_b16 v[238:239], v249 offset:17408\n"
    "ds_read_b64_tr_b16 v[240:241], v249 offset:24576\n"
    "ds_read_b64_tr_b16 v[242:243], v249 offset:25600\n"
    "s_waitcnt vmcnt(12) lgkmcnt(0)\n"
    "s_barrier\n"
    "s_setprio 1\n"
    "v_mfma_f32_16x16x32_bf16 v[76:79], v[212:215], v[180:183], v[76:79]\n"
    "v_mfma_f32_16x16x32_bf16 v[76:79], v[216:219], v[184:187], v[76:79]\n"
    "v_mfma_f32_16x16x32_bf16 v[80:83], v[220:223], v[180:183], v[80:83]\n"
    "v_mfma_f32_16x16x32_bf16 v[80:83], v[224:227], v[184:187], v[80:83]\n"
    "v_mfma_f32_16x16x32_bf16 v[84:87], v[228:231], v[180:183], v[84:87]\n"
    "v_mfma_f32_16x16x32_bf16 v[84:87], v[232:235], v[184:187], v[84:87]\n"
    "v_mfma_f32_16x16x32_bf16 v[88:91], v[236:239], v[180:183], v[88:91]\n"
    "v_mfma_f32_16x16x32_bf16 v[88:91], v[240:243], v[184:187], v[88:91]\n"
    "v_mfma_f32_16x16x32_bf16 v[92:95], v[212:215], v[188:191], v[92:95]\n"
    "v_mfma_f32_16x16x32_bf16 v[92:95], v[216:219], v[192:195], v[92:95]\n"
    "v_mfma_f32_16x16x32_bf16 v[96:99], v[220:223], v[188:191], v[96:99]\n"
    "v_mfma_f32_16x16x32_bf16 v[96:99], v[224:227], v[192:195], v[96:99]\n"
    "v_mfma_f32_16x16x32_bf16 v[100:103], v[228:231], v[188:191], v[100:103]\n"
    "v_mfma_f32_16x16x32_bf16 v[100:103], v[232:235], v[192:195], v[100:103]\n"
    "v_mfma_f32_16x16x32_bf16 v[104:107], v[236:239], v[188:191], v[104:107]\n"
    "v_mfma_f32_16x16x32_bf16 v[104:107], v[240:243], v[192:195], v[104:107]\n"
    "v_mfma_f32_16x16x32_bf16 v[108:111], v[212:215], v[196:199], v[108:111]\n"
    "v_mfma_f32_16x16x32_bf16 v[108:111], v[216:219], v[200:203], v[108:111]\n"
    "v_mfma_f32_16x16x32_bf16 v[112:115], v[220:223], v[196:199], v[112:115]\n"
    "v_mfma_f32_16x16x32_bf16 v[112:115], v[224:227], v[200:203], v[112:115]\n"
    "v_mfma_f32_16x16x32_bf16 v[116:119], v[228:231], v[196:199], v[116:119]\n"
    "v_mfma_f32_16x16x32_bf16 v[116:119], v[232:235], v[200:203], v[116:119]\n"
    "v_mfma_f32_16x16x32_bf16 v[120:123], v[236:239], v[196:199], v[120:123]\n"
    "v_mfma_f32_16x16x32_bf16 v[120:123], v[240:243], v[200:203], v[120:123]\n"
    "v_mfma_f32_16x16x32_bf16 v[124:127], v[212:215], v[204:207], v[124:127]\n"
    "v_mfma_f32_16x16x32_bf16 v[124:127], v[216:219], v[208:211], v[124:127]\n"
    "v_mfma_f32_16x16x32_bf16 v[128:131], v[220:223], v[204:207], v[128:131]\n"
    "v_mfma_f32_16x16x32_bf16 v[128:131], v[224:227], v[208:211], v[128:131]\n"
    "v_mfma_f32_16x16x32_bf16 v[132:135], v[228:231], v[204:207], v[132:135]\n"
    "v_mfma_f32_16x16x32_bf16 v[132:135], v[232:235], v[208:211], v[132:135]\n"
    "v_mfma_f32_16x16x32_bf16 v[136:139], v[236:239], v[204:207], v[136:139]\n"
    "v_mfma_f32_16x16x32_bf16 v[136:139], v[240:243], v[208:211], v[136:139]\n"
    "s_setprio 0\n"
    "s_barrier\n"
    "; phase 22: step 17 slot 0 stage 0\n"
    "s_bitcmp1_b32 s55, 8\n"
    "s_cbranch_scc1 .La21_%=\n"
    "s_sleep 2\n"
    "s_branch .La22_%=\n"
    ".La21_%=:\n"
    "s_sleep 4\n"
    ".La22_%=:\n"
    "ds_read_b128 v[180:183], v244 offset:32768\n"
    "ds_read_b128 v[184:187], v245 offset:32768\n"
    "ds_read_b128 v[188:191], v244 offset:34816\n"
    "ds_read_b128 v[192:195], v245 offset:34816\n"
    "s_add_u32 m0, s51, 0x4000\n"
    "s_xor_b32 s82, s65, 0x1200000\n"
    "buffer_load_dwordx4 v251, s[40:43], s82 offen  lds\n"
    "ds_read_b128 v[196:199], v244 offset:40960\n"
    "ds_read_b128 v[200:203], v245 offset:40960\n"
    "ds_read_b128 v[204:207], v244 offset:43008\n"
    "ds_read_b128 v[208:211], v245 offset:43008\n"
    "s_add_u32 m0, s51, 0x6000\n"
    "s_xor_b32 s82, s65, 0x1280000\n"
    "buffer_load_dwordx4 v251, s[40:43], s82 offen  lds\n"
    "ds_read_b64_tr_b16 v[212:213], v246 offset:32768\n"
    "ds_read_b64_tr_b16 v[214:215], v246 offset:33792\n"
    "ds_read_b64_tr_b16 v[216:217], v246 offset:40960\n"
    "ds_read_b64_tr_b16 v[218:219], v246 offset:41984\n"
    "ds_read_b64_tr_b16 v[220:221], v247 offset:32768\n"
    "ds_read_b64_tr_b16 v[222:223], v247 offset:33792\n"
    "ds_read_b64_tr_b16 v[224:225], v247 offset:40960\n"
    "ds_read_b64_tr_b16 v[226:227], v247 offset:41984\n"
    "ds_read_b64_tr_b16 v[228:229], v248 offset:32768\n"
    "ds_read_b64_tr_b16 v[230:231], v248 offset:33792\n"
    "ds_read_b64_tr_b16 v[232:233], v248 offset:40960\n"
    "ds_read_b64_tr_b16 v[234:235], v248 offset:41984\n"
    "ds_read_b64_tr_b16 v[236:237], v249 offset:32768\n"
    "ds_read_b64_tr_b16 v[238:239], v249 offset:33792\n"
    "ds_read_b64_tr_b16 v[240:241], v249 offset:40960\n"
    "ds_read_b64_tr_b16 v[242:243], v249 offset:41984\n"
    "s_waitcnt vmcnt(8) lgkmcnt(0)\n"
    "s_barrier\n"
    "s_setprio 1\n"
    "v_mfma_f32_16x16x32_bf16 v[12:15], v[212:215], v[180:183], v[12:15]\n"
    "v_mfma_f32_16x16x32_bf16 v[12:15], v[216:219], v[184:187], v[12:15]\n"
    "v_mfma_f32_16x16x32_bf16 v[16:19], v[220:223], v[180:183], v[16:19]\n"
    "v_mfma_f32_16x16x32_bf16 v[16:19], v[224:227], v[184:187], v[16:19]\n"
    "v_mfma_f32_16x16x32_bf16 v[20:23], v[228:231], v[180:183], v[20:23]\n"
    "v_mfma_f32_16x16x32_bf16 v[20:23], v[232:235], v[184:187], v[20:23]\n"
    "v_mfma_f32_16x16x32_bf16 v[24:27], v[236:239], v[180:183], v[24:27]\n"
    "v_mfma_f32_16x16x32_bf16 v[24:27], v[240:243], v[184:187], v[24:27]\n"
    "v_mfma_f32_16x16x32_bf16 v[28:31], v[212:215], v[188:191], v[28:31]\n"
    "v_mfma_f32_16x16x32_bf16 v[28:31], v[216:219], v[192:195], v[28:31]\n"
    "v_mfma_f32_16x16x32_bf16 v[32:35], v[220:223], v[188:191], v[32:35]\n"
    "v_mfma_f32_16x16x32_bf16 v[32:35], v[224:227], v[192:195], v[32:35]\n"
    "v_mfma_f32_16x16x32_bf16 v[36:39], v[228:231], v[188:191], v[36:39]\n"
    "v_mfma_f32_16x16x32_bf16 v[36:39], v[232:235], v[192:195], v[36:39]\n"
    "v_mfma_f32_16x16x32_bf16 v[40:43], v[236:239], v[188:191], v[40:43]\n"
    "v_mfma_f32_16x16x32_bf16 v[40:43], v[240:243], v[192:195], v[40:43]\n"
    "v_mfma_f32_16x16x32_bf16 v[44:47], v[212:215], v[196:199], v[44:47]\n"
    "v_mfma_f32_16x16x32_bf16 v[44:47], v[216:219], v[200:203], v[44:47]\n"
    "v_mfma_f32_16x16x32_bf16 v[48:51], v[220:223], v[196:199], v[48:51]\n"
    "v_mfma_f32_16x16x32_bf16 v[48:51], v[224:227], v[200:203], v[48:51]\n"
    "v_mfma_f32_16x16x32_bf16 v[52:55], v[228:231], v[196:199], v[52:55]\n"
    "v_mfma_f32_16x16x32_bf16 v[52:55], v[232:235], v[200:203], v[52:55]\n"
    "v_mfma_f32_16x16x32_bf16 v[56:59], v[236:239], v[196:199], v[56:59]\n"
    "v_mfma_f32_16x16x32_bf16 v[56:59], v[240:243], v[200:203], v[56:59]\n"
    "v_mfma_f32_16x16x32_bf16 v[60:63], v[212:215], v[204:207], v[60:63]\n"
    "v_mfma_f32_16x16x32_bf16 v[60:63], v[216:219], v[208:211], v[60:63]\n"
    "v_mfma_f32_16x16x32_bf16 v[64:67], v[220:223], v[204:207], v[64:67]\n"
    "v_mfma_f32_16x16x32_bf16 v[64:67], v[224:227], v[208:211], v[64:67]\n"
    "v_mfma_f32_16x16x32_bf16 v[68:71], v[228:231], v[204:207], v[68:71]\n"
    "v_mfma_f32_16x16x32_bf16 v[68:71], v[232:235], v[208:211], v[68:71]\n"
    "v_mfma_f32_16x16x32_bf16 v[72:75], v[236:239], v[204:207], v[72:75]\n"
    "v_mfma_f32_16x16x32_bf16 v[72:75], v[240:243], v[208:211], v[72:75]\n"
    "s_setprio 0\n"
    "s_barrier\n"
    "; phase 23: step 17 slot 1 stage 1\n"
    "s_bitcmp1_b32 s55, 8\n"
    "s_cbranch_scc1 .La23_%=\n"
    "s_sleep 2\n"
    "s_branch .La24_%=\n"
    ".La23_%=:\n"
    "s_sleep 4\n"
    ".La24_%=:\n"
    "ds_read_b64_tr_b16 v[212:213], v246 offset:49152\n"
    "ds_read_b64_tr_b16 v[214:215], v246 offset:50176\n"
    "s_add_u32 m0, s51, 0x8000\n"
    "s_xor_b32 s82, s64, 0x1300000\n"
    "buffer_load_dwordx4 v251, s[40:43], s82 offen  lds\n"
    "ds_read_b64_tr_b16 v[216:217], v246 offset:57344\n"
    "ds_read_b64_tr_b16 v[218:219], v246 offset:58368\n"
    "s_add_u32 m0, s51, 0xa000\n"
    "s_xor_b32 s82, s64, 0x1380000\n"
    "buffer_load_dwordx4 v251, s[40:43], s82 offen  lds\n"
    "ds_read_b64_tr_b16 v[220:221], v247 offset:49152\n"
    "ds_read_b64_tr_b16 v[222:223], v247 offset:50176\n"
    "s_add_u32 m0, s50, 0x8000\n"
    "s_xor_b32 s82, s48, 0x200a00\n"
    "buffer_load_dwordx4 v250, s[36:39], s82 offen  lds\n"
    "ds_read_b64_tr_b16 v[224:225], v247 offset:57344\n"
    "ds_read_b64_tr_b16 v[226:227], v247 offset:58368\n"
    "s_add_u32 m0, s50, 0x9000\n"
    "s_xor_b32 s82, s48, 0x280a00\n"
    "buffer_load_dwordx4 v250, s[36:39], s82 offen  lds\n"
    "ds_read_b64_tr_b16 v[228:229], v248 offset:49152\n"
    "ds_read_b64_tr_b16 v[230:231], v248 offset:50176\n"
    "s_add_u32 m0, s50, 0xa000\n"
    "s_xor_b32 s82, s48, 0xa00\n"
    "buffer_load_dwordx4 v250, s[36:39], s82 offen  lds\n"
    "ds_read_b64_tr_b16 v[232:233], v248 offset:57344\n"
    "ds_read_b64_tr_b16 v[234:235], v248 offset:58368\n"
    "s_add_u32 m0, s50, 0xb000\n"
    "s_xor_b32 s82, s48, 0x80a00\n"
    "buffer_load_dwordx4 v250, s[36:39], s82 offen  lds\n"
    "ds_read_b64_tr_b16 v[236:237], v249 offset:49152\n"
    "ds_read_b64_tr_b16 v[238:239], v249 offset:50176\n"
    "ds_read_b64_tr_b16 v[240:241], v249 offset:57344\n"
    "ds_read_b64_tr_b16 v[242:243], v249 offset:58368\n"
    "s_waitcnt vmcnt(12) lgkmcnt(0)\n"
    "s_barrier\n"
    "s_setprio 1\n"
    "v_mfma_f32_16x16x32_bf16 v[76:79], v[212:215], v[180:183], v[76:79]\n"
    "v_mfma_f32_16x16x32_bf16 v[76:79], v[216:219], v[184:187], v[76:79]\n"
    "v_mfma_f32_16x16x32_bf16 v[80:83], v[220:223], v[180:183], v[80:83]\n"
    "v_mfma_f32_16x16x32_bf16 v[80:83], v[224:227], v[184:187], v[80:83]\n"
    "v_mfma_f32_16x16x32_bf16 v[84:87], v[228:231], v[180:183], v[84:87]\n"
    "v_mfma_f32_16x16x32_bf16 v[84:87], v[232:235], v[184:187], v[84:87]\n"
    "v_mfma_f32_16x16x32_bf16 v[88:91], v[236:239], v[180:183], v[88:91]\n"
    "v_mfma_f32_16x16x32_bf16 v[88:91], v[240:243], v[184:187], v[88:91]\n"
    "v_mfma_f32_16x16x32_bf16 v[92:95], v[212:215], v[188:191], v[92:95]\n"
    "v_mfma_f32_16x16x32_bf16 v[92:95], v[216:219], v[192:195], v[92:95]\n"
    "v_mfma_f32_16x16x32_bf16 v[96:99], v[220:223], v[188:191], v[96:99]\n"
    "v_mfma_f32_16x16x32_bf16 v[96:99], v[224:227], v[192:195], v[96:99]\n"
    "v_mfma_f32_16x16x32_bf16 v[100:103], v[228:231], v[188:191], v[100:103]\n"
    "v_mfma_f32_16x16x32_bf16 v[100:103], v[232:235], v[192:195], v[100:103]\n"
    "v_mfma_f32_16x16x32_bf16 v[104:107], v[236:239], v[188:191], v[104:107]\n"
    "v_mfma_f32_16x16x32_bf16 v[104:107], v[240:243], v[192:195], v[104:107]\n"
    "v_mfma_f32_16x16x32_bf16 v[108:111], v[212:215], v[196:199], v[108:111]\n"
    "v_mfma_f32_16x16x32_bf16 v[108:111], v[216:219], v[200:203], v[108:111]\n"
    "v_mfma_f32_16x16x32_bf16 v[112:115], v[220:223], v[196:199], v[112:115]\n"
    "v_mfma_f32_16x16x32_bf16 v[112:115], v[224:227], v[200:203], v[112:115]\n"
    "v_mfma_f32_16x16x32_bf16 v[116:119], v[228:231], v[196:199], v[116:119]\n"
    "v_mfma_f32_16x16x32_bf16 v[116:119], v[232:235], v[200:203], v[116:119]\n"
    "v_mfma_f32_16x16x32_bf16 v[120:123], v[236:239], v[196:199], v[120:123]\n"
    "v_mfma_f32_16x16x32_bf16 v[120:123], v[240:243], v[200:203], v[120:123]\n"
    "v_mfma_f32_16x16x32_bf16 v[124:127], v[212:215], v[204:207], v[124:127]\n"
    "v_mfma_f32_16x16x32_bf16 v[124:127], v[216:219], v[208:211], v[124:127]\n"
    "v_mfma_f32_16x16x32_bf16 v[128:131], v[220:223], v[204:207], v[128:131]\n"
    "v_mfma_f32_16x16x32_bf16 v[128:131], v[224:227], v[208:211], v[128:131]\n"
    "v_mfma_f32_16x16x32_bf16 v[132:135], v[228:231], v[204:207], v[132:135]\n"
    "v_mfma_f32_16x16x32_bf16 v[132:135], v[232:235], v[208:211], v[132:135]\n"
    "v_mfma_f32_16x16x32_bf16 v[136:139], v[236:239], v[204:207], v[136:139]\n"
    "v_mfma_f32_16x16x32_bf16 v[136:139], v[240:243], v[208:211], v[136:139]\n"
    "s_setprio 0\n"
    "s_barrier\n"
    "; phase 24: step 18 slot 0 stage 0\n"
    "s_bitcmp1_b32 s55, 8\n"
    "s_cbranch_scc1 .La25_%=\n"
    "s_sleep 2\n"
    "s_branch .La26_%=\n"
    ".La25_%=:\n"
    "s_sleep 4\n"
    ".La26_%=:\n"
    "ds_read_b128 v[180:183], v244 offset:0\n"
    "ds_read_b128 v[184:187], v245 offset:0\n"
    "ds_read_b128 v[188:191], v244 offset:2048\n"
    "ds_read_b128 v[192:195], v245 offset:2048\n"
    "s_add_u32 m0, s51, 0xc000\n"
    "s_xor_b32 s82, s65, 0x1300000\n"
    "buffer_load_dwordx4 v251, s[40:43], s82 offen  lds\n"
    "ds_read_b128 v[196:199], v244 offset:8192\n"
    "ds_read_b128 v[200:203], v245 offset:8192\n"
    "ds_read_b128 v[204:207], v244 offset:10240\n"
    "ds_read_b128 v[208:211], v245 offset:10240\n"
    "s_add_u32 m0, s51, 0xe000\n"
    "s_xor_b32 s82, s65, 0x1380000\n"
    "buffer_load_dwordx4 v251, s[40:43], s82 offen  lds\n"
    "ds_read_b64_tr_b16 v[212:213], v246 offset:0\n"
    "ds_read_b64_tr_b16 v[214:215], v246 offset:1024\n"
    "ds_read_b64_tr_b16 v[216:217], v246 offset:8192\n"
    "ds_read_b64_tr_b16 v[218:219], v246 offset:9216\n"
    "ds_read_b64_tr_b16 v[220:221], v247 offset:0\n"
    "ds_read_b64_tr_b16 v[222:223], v247 offset:1024\n"
    "ds_read_b64_tr_b16 v[224:225], v247 offset:8192\n"
    "ds_read_b64_tr_b16 v[226:227], v247 offset:9216\n"
    "ds_read_b64_tr_b16 v[228:229], v248 offset:0\n"
    "ds_read_b64_tr_b16 v[230:231], v248 offset:1024\n"
    "ds_read_b64_tr_b16 v[232:233], v248 offset:8192\n"
    "ds_read_b64_tr_b16 v[234:235], v248 offset:9216\n"
    "ds_read_b64_tr_b16 v[236:237], v249 offset:0\n"
    "ds_read_b64_tr_b16 v[238:239], v249 offset:1024\n"
    "ds_read_b64_tr_b16 v[240:241], v249 offset:8192\n"
    "ds_read_b64_tr_b16 v[242:243], v249 offset:9216\n"
    "s_waitcnt vmcnt(8) lgkmcnt(0)\n"
    "s_barrier\n"
    "s_setprio 1\n"
    "v_mfma_f32_16x16x32_bf16 v[12:15], v[212:215], v[180:183], v[12:15]\n"
    "v_mfma_f32_16x16x32_bf16 v[12:15], v[216:219], v[184:187], v[12:15]\n"
    "v_mfma_f32_16x16x32_bf16 v[16:19], v[220:223], v[180:183], v[16:19]\n"
    "v_mfma_f32_16x16x32_bf16 v[16:19], v[224:227], v[184:187], v[16:19]\n"
    "v_mfma_f32_16x16x32_bf16 v[20:23], v[228:231], v[180:183], v[20:23]\n"
    "v_mfma_f32_16x16x32_bf16 v[20:23], v[232:235], v[184:187], v[20:23]\n"
    "v_mfma_f32_16x16x32_bf16 v[24:27], v[236:239], v[180:183], v[24:27]\n"
    "v_mfma_f32_16x16x32_bf16 v[24:27], v[240:243], v[184:187], v[24:27]\n"
    "v_mfma_f32_16x16x32_bf16 v[28:31], v[212:215], v[188:191], v[28:31]\n"
    "v_mfma_f32_16x16x32_bf16 v[28:31], v[216:219], v[192:195], v[28:31]\n"
    "v_mfma_f32_16x16x32_bf16 v[32:35], v[220:223], v[188:191], v[32:35]\n"
    "v_mfma_f32_16x16x32_bf16 v[32:35], v[224:227], v[192:195], v[32:35]\n"
    "v_mfma_f32_16x16x32_bf16 v[36:39], v[228:231], v[188:191], v[36:39]\n"
    "v_mfma_f32_16x16x32_bf16 v[36:39], v[232:235], v[192:195], v[36:39]\n"
    "v_mfma_f32_16x16x32_bf16 v[40:43], v[236:239], v[188:191], v[40:43]\n"
    "v_mfma_f32_16x16x32_bf16 v[40:43], v[240:243], v[192:195], v[40:43]\n"
    "v_mfma_f32_16x16x32_bf16 v[44:47], v[212:215], v[196:199], v[44:47]\n"
    "v_mfma_f32_16x16x32_bf16 v[44:47], v[216:219], v[200:203], v[44:47]\n"
    "v_mfma_f32_16x16x32_bf16 v[48:51], v[220:223], v[196:199], v[48:51]\n"
    "v_mfma_f32_16x16x32_bf16 v[48:51], v[224:227], v[200:203], v[48:51]\n"
    "v_mfma_f32_16x16x32_bf16 v[52:55], v[228:231], v[196:199], v[52:55]\n"
    "v_mfma_f32_16x16x32_bf16 v[52:55], v[232:235], v[200:203], v[52:55]\n"
    "v_mfma_f32_16x16x32_bf16 v[56:59], v[236:239], v[196:199], v[56:59]\n"
    "v_mfma_f32_16x16x32_bf16 v[56:59], v[240:243], v[200:203], v[56:59]\n"
    "v_mfma_f32_16x16x32_bf16 v[60:63], v[212:215], v[204:207], v[60:63]\n"
    "v_mfma_f32_16x16x32_bf16 v[60:63], v[216:219], v[208:211], v[60:63]\n"
    "v_mfma_f32_16x16x32_bf16 v[64:67], v[220:223], v[204:207], v[64:67]\n"
    "v_mfma_f32_16x16x32_bf16 v[64:67], v[224:227], v[208:211], v[64:67]\n"
    "v_mfma_f32_16x16x32_bf16 v[68:71], v[228:231], v[204:207], v[68:71]\n"
    "v_mfma_f32_16x16x32_bf16 v[68:71], v[232:235], v[208:211], v[68:71]\n"
    "v_mfma_f32_16x16x32_bf16 v[72:75], v[236:239], v[204:207], v[72:75]\n"
    "v_mfma_f32_16x16x32_bf16 v[72:75], v[240:243], v[208:211], v[72:75]\n"
    "s_setprio 0\n"
    "s_barrier\n"
    "; phase 25: step 18 slot 1 stage 1\n"
    "s_bitcmp1_b32 s55, 8\n"
    "s_cbranch_scc1 .La27_%=\n"
    "s_sleep 2\n"
    "s_branch .La28_%=\n"
    ".La27_%=:\n"
    "s_sleep 4\n"
    ".La28_%=:\n"
    "ds_read_b64_tr_b16 v[212:213], v246 offset:16384\n"
    "ds_read_b64_tr_b16 v[214:215], v246 offset:17408\n"
    "s_add_u32 m0, s51, 0x0\n"
    "s_xor_b32 s82, s64, 0x1400000\n"
    "buffer_load_dwordx4 v251, s[40:43], s82 offen  lds\n"
    "ds_read_b64_tr_b16 v[216:217], v246 offset:24576\n"
    "ds_read_b64_tr_b16 v[218:219], v246 offset:25600\n"
    "s_add_u32 m0, s51, 0x2000\n"
    "s_xor_b32 s82, s64, 0x1480000\n"
    "buffer_load_dwordx4 v251, s[40:43], s82 offen  lds\n"
    "ds_read_b64_tr_b16 v[220:221], v247 offset:16384\n"
    "ds_read_b64_tr_b16 v[222:223], v247 offset:17408\n"
    "s_add_u32 m0, s50, 0x0\n"
    "s_xor_b32 s82, s48, 0x200a80\n"
    "buffer_load_dwordx4 v250, s[36:39], s82 offen  lds\n"
    "ds_read_b64_tr_b16 v[224:225], v247 offset:24576\n"
    "ds_read_b64_tr_b16 v[226:227], v247 offset:25600\n"
    "s_add_u32 m0, s50, 0x1000\n"
    "s_xor_b32 s82, s48, 0x280a80\n"
    "buffer_load_dwordx4 v250, s[36:39], s82 offen  lds\n"
    "ds_read_b64_tr_b16 v[228:229], v248 offset:16384\n"
    "ds_read_b64_tr_b16 v[230:231], v248 offset:17408\n"
    "s_add_u32 m0, s50, 0x2000\n"
    "s_xor_b32 s82, s48, 0xa80\n"
    "buffer_load_dwordx4 v250, s[36:39], s82 offen  lds\n"
    "ds_read_b64_tr_b16 v[232:233], v248 offset:24576\n"
    "ds_read_b64_tr_b16 v[234:235], v248 offset:25600\n"
    "s_add_u32 m0, s50, 0x3000\n"
    "s_xor_b32 s82, s48, 0x80a80\n"
    "buffer_load_dwordx4 v250, s[36:39], s82 offen  lds\n"
    "ds_read_b64_tr_b16 v[236:237], v249 offset:16384\n"
    "ds_read_b64_tr_b16 v[238:239], v249 offset:17408\n"
    "ds_read_b64_tr_b16 v[240:241], v249 offset:24576\n"
    "ds_read_b64_tr_b16 v[242:243], v249 offset:25600\n"
    "s_waitcnt vmcnt(12) lgkmcnt(0)\n"
    "s_barrier\n"
    "s_setprio 1\n"
    "v_mfma_f32_16x16x32_bf16 v[76:79], v[212:215], v[180:183], v[76:79]\n"
    "v_mfma_f32_16x16x32_bf16 v[76:79], v[216:219], v[184:187], v[76:79]\n"
    "v_mfma_f32_16x16x32_bf16 v[80:83], v[220:223], v[180:183], v[80:83]\n"
    "v_mfma_f32_16x16x32_bf16 v[80:83], v[224:227], v[184:187], v[80:83]\n"
    "v_mfma_f32_16x16x32_bf16 v[84:87], v[228:231], v[180:183], v[84:87]\n"
    "v_mfma_f32_16x16x32_bf16 v[84:87], v[232:235], v[184:187], v[84:87]\n"
    "v_mfma_f32_16x16x32_bf16 v[88:91], v[236:239], v[180:183], v[88:91]\n"
    "v_mfma_f32_16x16x32_bf16 v[88:91], v[240:243], v[184:187], v[88:91]\n"
    "v_mfma_f32_16x16x32_bf16 v[92:95], v[212:215], v[188:191], v[92:95]\n"
    "v_mfma_f32_16x16x32_bf16 v[92:95], v[216:219], v[192:195], v[92:95]\n"
    "v_mfma_f32_16x16x32_bf16 v[96:99], v[220:223], v[188:191], v[96:99]\n"
    "v_mfma_f32_16x16x32_bf16 v[96:99], v[224:227], v[192:195], v[96:99]\n"
    "v_mfma_f32_16x16x32_bf16 v[100:103], v[228:231], v[188:191], v[100:103]\n"
    "v_mfma_f32_16x16x32_bf16 v[100:103], v[232:235], v[192:195], v[100:103]\n"
    "v_mfma_f32_16x16x32_bf16 v[104:107], v[236:239], v[188:191], v[104:107]\n"
    "v_mfma_f32_16x16x32_bf16 v[104:107], v[240:243], v[192:195], v[104:107]\n"
    "v_mfma_f32_16x16x32_bf16 v[108:111], v[212:215], v[196:199], v[108:111]\n"
    "v_mfma_f32_16x16x32_bf16 v[108:111], v[216:219], v[200:203], v[108:111]\n"
    "v_mfma_f32_16x16x32_bf16 v[112:115], v[220:223], v[196:199], v[112:115]\n"
    "v_mfma_f32_16x16x32_bf16 v[112:115], v[224:227], v[200:203], v[112:115]\n"
    "v_mfma_f32_16x16x32_bf16 v[116:119], v[228:231], v[196:199], v[116:119]\n"
    "v_mfma_f32_16x16x32_bf16 v[116:119], v[232:235], v[200:203], v[116:119]\n"
    "v_mfma_f32_16x16x32_bf16 v[120:123], v[236:239], v[196:199], v[120:123]\n"
    "v_mfma_f32_16x16x32_bf16 v[120:123], v[240:243], v[200:203], v[120:123]\n"
    "v_mfma_f32_16x16x32_bf16 v[124:127], v[212:215], v[204:207], v[124:127]\n"
    "v_mfma_f32_16x16x32_bf16 v[124:127], v[216:219], v[208:211], v[124:127]\n"
    "v_mfma_f32_16x16x32_bf16 v[128:131], v[220:223], v[204:207], v[128:131]\n"
    "v_mfma_f32_16x16x32_bf16 v[128:131], v[224:227], v[208:211], v[128:131]\n"
    "v_mfma_f32_16x16x32_bf16 v[132:135], v[228:231], v[204:207], v[132:135]\n"
    "v_mfma_f32_16x16x32_bf16 v[132:135], v[232:235], v[208:211], v[132:135]\n"
    "v_mfma_f32_16x16x32_bf16 v[136:139], v[236:239], v[204:207], v[136:139]\n"
    "v_mfma_f32_16x16x32_bf16 v[136:139], v[240:243], v[208:211], v[136:139]\n"
    "s_setprio 0\n"
    "s_barrier\n"
    "; phase 26: step 19 slot 0 stage 0\n"
    "s_bitcmp1_b32 s55, 8\n"
    "s_cbranch_scc1 .La29_%=\n"
    "s_sleep 2\n"
    "s_branch .La30_%=\n"
    ".La29_%=:\n"
    "s_sleep 4\n"
    ".La30_%=:\n"
    "ds_read_b128 v[180:183], v244 offset:16384\n"
    "ds_read_b128 v[184:187], v245 offset:16384\n"
    "ds_read_b128 v[188:191], v244 offset:18432\n"
    "ds_read_b128 v[192:195], v245 offset:18432\n"
    "s_add_u32 m0, s51, 0x4000\n"
    "s_xor_b32 s82, s65, 0x1400000\n"
    "buffer_load_dwordx4 v251, s[40:43], s82 offen  lds\n"
    "ds_read_b128 v[196:199], v244 offset:24576\n"
    "ds_read_b128 v[200:203], v245 offset:24576\n"
    "ds_read_b128 v[204:207], v244 offset:26624\n"
    "ds_read_b128 v[208:211], v245 offset:26624\n"
    "s_add_u32 m0, s51, 0x6000\n"
    "s_xor_b32 s82, s65, 0x1480000\n"
    "buffer_load_dwordx4 v251, s[40:43], s82 offen  lds\n"
    "ds_read_b64_tr_b16 v[212:213], v246 offset:32768\n"
    "ds_read_b64_tr_b16 v[214:215], v246 offset:33792\n"
    "ds_read_b64_tr_b16 v[216:217], v246 offset:40960\n"
    "ds_read_b64_tr_b16 v[218:219], v246 offset:41984\n"
    "ds_read_b64_tr_b16 v[220:221], v247 offset:32768\n"
    "ds_read_b64_tr_b16 v[222:223], v247 offset:33792\n"
    "ds_read_b64_tr_b16 v[224:225], v247 offset:40960\n"
    "ds_read_b64_tr_b16 v[226:227], v247 offset:41984\n"
    "ds_read_b64_tr_b16 v[228:229], v248 offset:32768\n"
    "ds_read_b64_tr_b16 v[230:231], v248 offset:33792\n"
    "ds_read_b64_tr_b16 v[232:233], v248 offset:40960\n"
    "ds_read_b64_tr_b16 v[234:235], v248 offset:41984\n"
    "ds_read_b64_tr_b16 v[236:237], v249 offset:32768\n"
    "ds_read_b64_tr_b16 v[238:239], v249 offset:33792\n"
    "ds_read_b64_tr_b16 v[240:241], v249 offset:40960\n"
    "ds_read_b64_tr_b16 v[242:243], v249 offset:41984\n"
    "s_waitcnt vmcnt(8) lgkmcnt(0)\n"
    "s_barrier\n"
    "s_setprio 1\n"
    "v_mfma_f32_16x16x32_bf16 v[12:15], v[212:215], v[180:183], v[12:15]\n"
    "v_mfma_f32_16x16x32_bf16 v[12:15], v[216:219], v[184:187], v[12:15]\n"
    "v_mfma_f32_16x16x32_bf16 v[16:19], v[220:223], v[180:183], v[16:19]\n"
    "v_mfma_f32_16x16x32_bf16 v[16:19], v[224:227], v[184:187], v[16:19]\n"
    "v_mfma_f32_16x16x32_bf16 v[20:23], v[228:231], v[180:183], v[20:23]\n"
    "v_mfma_f32_16x16x32_bf16 v[20:23], v[232:235], v[184:187], v[20:23]\n"
    "v_mfma_f32_16x16x32_bf16 v[24:27], v[236:239], v[180:183], v[24:27]\n"
    "v_mfma_f32_16x16x32_bf16 v[24:27], v[240:243], v[184:187], v[24:27]\n"
    "v_mfma_f32_16x16x32_bf16 v[28:31], v[212:215], v[188:191], v[28:31]\n"
    "v_mfma_f32_16x16x32_bf16 v[28:31], v[216:219], v[192:195], v[28:31]\n"
    "v_mfma_f32_16x16x32_bf16 v[32:35], v[220:223], v[188:191], v[32:35]\n"
    "v_mfma_f32_16x16x32_bf16 v[32:35], v[224:227], v[192:195], v[32:35]\n"
    "v_mfma_f32_16x16x32_bf16 v[36:39], v[228:231], v[188:191], v[36:39]\n"
    "v_mfma_f32_16x16x32_bf16 v[36:39], v[232:235], v[192:195], v[36:39]\n"
    "v_mfma_f32_16x16x32_bf16 v[40:43], v[236:239], v[188:191], v[40:43]\n"
    "v_mfma_f32_16x16x32_bf16 v[40:43], v[240:243], v[192:195], v[40:43]\n"
    "v_mfma_f32_16x16x32_bf16 v[44:47], v[212:215], v[196:199], v[44:47]\n"
    "v_mfma_f32_16x16x32_bf16 v[44:47], v[216:219], v[200:203], v[44:47]\n"
    "v_mfma_f32_16x16x32_bf16 v[48:51], v[220:223], v[196:199], v[48:51]\n"
    "v_mfma_f32_16x16x32_bf16 v[48:51], v[224:227], v[200:203], v[48:51]\n"
    "v_mfma_f32_16x16x32_bf16 v[52:55], v[228:231], v[196:199], v[52:55]\n"
    "v_mfma_f32_16x16x32_bf16 v[52:55], v[232:235], v[200:203], v[52:55]\n"
    "v_mfma_f32_16x16x32_bf16 v[56:59], v[236:239], v[196:199], v[56:59]\n"
    "v_mfma_f32_16x16x32_bf16 v[56:59], v[240:243], v[200:203], v[56:59]\n"
    "v_mfma_f32_16x16x32_bf16 v[60:63], v[212:215], v[204:207], v[60:63]\n"
    "v_mfma_f32_16x16x32_bf16 v[60:63], v[216:219], v[208:211], v[60:63]\n"
    "v_mfma_f32_16x16x32_bf16 v[64:67], v[220:223], v[204:207], v[64:67]\n"
    "v_mfma_f32_16x16x32_bf16 v[64:67], v[224:227], v[208:211], v[64:67]\n"
    "v_mfma_f32_16x16x32_bf16 v[68:71], v[228:231], v[204:207], v[68:71]\n"
    "v_mfma_f32_16x16x32_bf16 v[68:71], v[232:235], v[208:211], v[68:71]\n"
    "v_mfma_f32_16x16x32_bf16 v[72:75], v[236:239], v[204:207], v[72:75]\n"
    "v_mfma_f32_16x16x32_bf16 v[72:75], v[240:243], v[208:211], v[72:75]\n"
    "s_setprio 0\n"
    "s_barrier\n"
    "; phase 27: step 19 slot 1 stage 1\n"
    "s_bitcmp1_b32 s55, 8\n"
    "s_cbranch_scc1 .La31_%=\n"
    "s_sleep 2\n"
    "s_branch .La32_%=\n"
    ".La31_%=:\n"
    "s_sleep 4\n"
    ".La32_%=:\n"
    "ds_read_b64_tr_b16 v[212:213], v246 offset:49152\n"
    "ds_read_b64_tr_b16 v[214:215], v246 offset:50176\n"
    "s_add_u32 m0, s51, 0x8000\n"
    "s_xor_b32 s82, s64, 0x1500000\n"
    "buffer_load_dwordx4 v251, s[40:43], s82 offen  lds\n"
    "ds_read_b64_tr_b16 v[216:217], v246 offset:57344\n"
    "ds_read_b64_tr_b16 v[218:219], v246 offset:58368\n"
    "s_add_u32 m0, s51, 0xa000\n"
    "s_xor_b32 s82, s64, 0x1580000\n"
    "buffer_load_dwordx4 v251, s[40:43], s82 offen  lds\n"
    "ds_read_b64_tr_b16 v[220:221], v247 offset:49152\n"
    "ds_read_b64_tr_b16 v[222:223], v247 offset:50176\n"
    "s_add_u32 m0, s50, 0x4000\n"
    "s_xor_b32 s82, s48, 0x200b00\n"
    "buffer_load_dwordx4 v250, s[36:39], s82 offen  lds\n"
    "ds_read_b64_tr_b16 v[224:225], v247 offset:57344\n"
    "ds_read_b64_tr_b16 v[226:227], v247 offset:58368\n"
    "s_add_u32 m0, s50, 0x5000\n"
    "s_xor_b32 s82, s48, 0x280b00\n"
    "buffer_load_dwordx4 v250, s[36:39], s82 offen  lds\n"
    "ds_read_b64_tr_b16 v[228:229], v248 offset:49152\n"
    "ds_read_b64_tr_b16 v[230:231], v248 offset:50176\n"
    "s_add_u32 m0, s50, 0x6000\n"
    "s_xor_b32 s82, s48, 0xb00\n"
    "buffer_load_dwordx4 v250, s[36:39], s82 offen  lds\n"
    "ds_read_b64_tr_b16 v[232:233], v248 offset:57344\n"
    "ds_read_b64_tr_b16 v[234:235], v248 offset:58368\n"
    "s_add_u32 m0, s50, 0x7000\n"
    "s_xor_b32 s82, s48, 0x80b00\n"
    "buffer_load_dwordx4 v250, s[36:39], s82 offen  lds\n"
    "ds_read_b64_tr_b16 v[236:237], v249 offset:49152\n"
    "ds_read_b64_tr_b16 v[238:239], v249 offset:50176\n"
    "ds_read_b64_tr_b16 v[240:241], v249 offset:57344\n"
    "ds_read_b64_tr_b16 v[242:243], v249 offset:58368\n"
    "s_waitcnt vmcnt(12) lgkmcnt(0)\n"
    "s_barrier\n"
    "s_setprio 1\n"
    "v_mfma_f32_16x16x32_bf16 v[76:79], v[212:215], v[180:183], v[76:79]\n"
    "v_mfma_f32_16x16x32_bf16 v[76:79], v[216:219], v[184:187], v[76:79]\n"
    "v_mfma_f32_16x16x32_bf16 v[80:83], v[220:223], v[180:183], v[80:83]\n"
    "v_mfma_f32_16x16x32_bf16 v[80:83], v[224:227], v[184:187], v[80:83]\n"
    "v_mfma_f32_16x16x32_bf16 v[84:87], v[228:231], v[180:183], v[84:87]\n"
    "v_mfma_f32_16x16x32_bf16 v[84:87], v[232:235], v[184:187], v[84:87]\n"
    "v_mfma_f32_16x16x32_bf16 v[88:91], v[236:239], v[180:183], v[88:91]\n"
    "v_mfma_f32_16x16x32_bf16 v[88:91], v[240:243], v[184:187], v[88:91]\n"
    "v_mfma_f32_16x16x32_bf16 v[92:95], v[212:215], v[188:191], v[92:95]\n"
    "v_mfma_f32_16x16x32_bf16 v[92:95], v[216:219], v[192:195], v[92:95]\n"
    "v_mfma_f32_16x16x32_bf16 v[96:99], v[220:223], v[188:191], v[96:99]\n"
    "v_mfma_f32_16x16x32_bf16 v[96:99], v[224:227], v[192:195], v[96:99]\n"
    "v_mfma_f32_16x16x32_bf16 v[100:103], v[228:231], v[188:191], v[100:103]\n"
    "v_mfma_f32_16x16x32_bf16 v[100:103], v[232:235], v[192:195], v[100:103]\n"
    "v_mfma_f32_16x16x32_bf16 v[104:107], v[236:239], v[188:191], v[104:107]\n"
    "v_mfma_f32_16x16x32_bf16 v[104:107], v[240:243], v[192:195], v[104:107]\n"
    "v_mfma_f32_16x16x32_bf16 v[108:111], v[212:215], v[196:199], v[108:111]\n"
    "v_mfma_f32_16x16x32_bf16 v[108:111], v[216:219], v[200:203], v[108:111]\n"
    "v_mfma_f32_16x16x32_bf16 v[112:115], v[220:223], v[196:199], v[112:115]\n"
    "v_mfma_f32_16x16x32_bf16 v[112:115], v[224:227], v[200:203], v[112:115]\n"
    "v_mfma_f32_16x16x32_bf16 v[116:119], v[228:231], v[196:199], v[116:119]\n"
    "v_mfma_f32_16x16x32_bf16 v[116:119], v[232:235], v[200:203], v[116:119]\n"
    "v_mfma_f32_16x16x32_bf16 v[120:123], v[236:239], v[196:199], v[120:123]\n"
    "v_mfma_f32_16x16x32_bf16 v[120:123], v[240:243], v[200:203], v[120:123]\n"
    "v_mfma_f32_16x16x32_bf16 v[124:127], v[212:215], v[204:207], v[124:127]\n"
    "v_mfma_f32_16x16x32_bf16 v[124:127], v[216:219], v[208:211], v[124:127]\n"
    "v_mfma_f32_16x16x32_bf16 v[128:131], v[220:223], v[204:207], v[128:131]\n"
    "v_mfma_f32_16x16x32_bf16 v[128:131], v[224:227], v[208:211], v[128:131]\n"
    "v_mfma_f32_16x16x32_bf16 v[132:135], v[228:231], v[204:207], v[132:135]\n"
    "v_mfma_f32_16x16x32_bf16 v[132:135], v[232:235], v[208:211], v[132:135]\n"
    "v_mfma_f32_16x16x32_bf16 v[136:139], v[236:239], v[204:207], v[136:139]\n"
    "v_mfma_f32_16x16x32_bf16 v[136:139], v[240:243], v[208:211], v[136:139]\n"
    "s_setprio 0\n"
    "s_barrier\n"
    "; phase 28: step 20 slot 0 stage 0\n"
    "s_bitcmp1_b32 s55, 8\n"
    "s_cbranch_scc1 .La33_%=\n"
    "s_sleep 2\n"
    "s_branch .La34_%=\n"
    ".La33_%=:\n"
    "s_sleep 4\n"
    ".La34_%=:\n"
    "ds_read_b128 v[180:183], v244 offset:32768\n"
    "ds_read_b128 v[184:187], v245 offset:32768\n"
    "ds_read_b128 v[188:191], v244 offset:34816\n"
    "ds_read_b128 v[192:195], v245 offset:34816\n"
    "s_add_u32 m0, s51, 0xc000\n"
    "s_xor_b32 s82, s65, 0x1500000\n"
    "buffer_load_dwordx4 v251, s[40:43], s82 offen  lds\n"
    "ds_read_b128 v[196:199], v244 offset:40960\n"
    "ds_read_b128 v[200:203], v245 offset:40960\n"
    "ds_read_b128 v[204:207], v244 offset:43008\n"
    "ds_read_b128 v[208:211], v245 offset:43008\n"
    "s_add_u32 m0, s51, 0xe000\n"
    "s_xor_b32 s82, s65, 0x1580000\n"
    "buffer_load_dwordx4 v251, s[40:43], s82 offen  lds\n"
    "ds_read_b64_tr_b16 v[212:213], v246 offset:0\n"
    "ds_read_b64_tr_b16 v[214:215], v246 offset:1024\n"
    "ds_read_b64_tr_b16 v[216:217], v246 offset:8192\n"
    "ds_read_b64_tr_b16 v[218:219], v246 offset:9216\n"
    "ds_read_b64_tr_b16 v[220:221], v247 offset:0\n"
    "ds_read_b64_tr_b16 v[222:223], v247 offset:1024\n"
    "ds_read_b64_tr_b16 v[224:225], v247 offset:8192\n"
    "ds_read_b64_tr_b16 v[226:227], v247 offset:9216\n"
    "ds_read_b64_tr_b16 v[228:229], v248 offset:0\n"
    "ds_read_b64_tr_b16 v[230:231], v248 offset:1024\n"
    "ds_read_b64_tr_b16 v[232:233], v248 offset:8192\n"
    "ds_read_b64_tr_b16 v[234:235], v248 offset:9216\n"
    "ds_read_b64_tr_b16 v[236:237], v249 offset:0\n"
    "ds_read_b64_tr_b16 v[238:239], v249 offset:1024\n"
    "ds_read_b64_tr_b16 v[240:241], v249 offset:8192\n"
    "ds_read_b64_tr_b16 v[242:243], v249 offset:9216\n"
    "s_waitcnt vmcnt(8) lgkmcnt(0)\n"
    "s_barrier\n"
    "s_setprio 1\n"
    "v_mfma_f32_16x16x32_bf16 v[12:15], v[212:215], v[180:183], v[12:15]\n"
    "v_mfma_f32_16x16x32_bf16 v[12:15], v[216:219], v[184:187], v[12:15]\n"
    "v_mfma_f32_16x16x32_bf16 v[16:19], v[220:223], v[180:183], v[16:19]\n"
    "v_mfma_f32_16x16x32_bf16 v[16:19], v[224:227], v[184:187], v[16:19]\n"
    "v_mfma_f32_16x16x32_bf16 v[20:23], v[228:231], v[180:183], v[20:23]\n"
    "v_mfma_f32_16x16x32_bf16 v[20:23], v[232:235], v[184:187], v[20:23]\n"
    "v_mfma_f32_16x16x32_bf16 v[24:27], v[236:239], v[180:183], v[24:27]\n"
    "v_mfma_f32_16x16x32_bf16 v[24:27], v[240:243], v[184:187], v[24:27]\n"
    "v_mfma_f32_16x16x32_bf16 v[28:31], v[212:215], v[188:191], v[28:31]\n"
    "v_mfma_f32_16x16x32_bf16 v[28:31], v[216:219], v[192:195], v[28:31]\n"
    "v_mfma_f32_16x16x32_bf16 v[32:35], v[220:223], v[188:191], v[32:35]\n"
    "v_mfma_f32_16x16x32_bf16 v[32:35], v[224:227], v[192:195], v[32:35]\n"
    "v_mfma_f32_16x16x32_bf16 v[36:39], v[228:231], v[188:191], v[36:39]\n"
    "v_mfma_f32_16x16x32_bf16 v[36:39], v[232:235], v[192:195], v[36:39]\n"
    "v_mfma_f32_16x16x32_bf16 v[40:43], v[236:239], v[188:191], v[40:43]\n"
    "v_mfma_f32_16x16x32_bf16 v[40:43], v[240:243], v[192:195], v[40:43]\n"
    "v_mfma_f32_16x16x32_bf16 v[44:47], v[212:215], v[196:199], v[44:47]\n"
    "v_mfma_f32_16x16x32_bf16 v[44:47], v[216:219], v[200:203], v[44:47]\n"
    "v_mfma_f32_16x16x32_bf16 v[48:51], v[220:223], v[196:199], v[48:51]\n"
    "v_mfma_f32_16x16x32_bf16 v[48:51], v[224:227], v[200:203], v[48:51]\n"
    "v_mfma_f32_16x16x32_bf16 v[52:55], v[228:231], v[196:199], v[52:55]\n"
    "v_mfma_f32_16x16x32_bf16 v[52:55], v[232:235], v[200:203], v[52:55]\n"
    "v_mfma_f32_16x16x32_bf16 v[56:59], v[236:239], v[196:199], v[56:59]\n"
    "v_mfma_f32_16x16x32_bf16 v[56:59], v[240:243], v[200:203], v[56:59]\n"
    "v_mfma_f32_16x16x32_bf16 v[60:63], v[212:215], v[204:207], v[60:63]\n"
    "v_mfma_f32_16x16x32_bf16 v[60:63], v[216:219], v[208:211], v[60:63]\n"
    "v_mfma_f32_16x16x32_bf16 v[64:67], v[220:223], v[204:207], v[64:67]\n"
    "v_mfma_f32_16x16x32_bf16 v[64:67], v[224:227], v[208:211], v[64:67]\n"
    "v_mfma_f32_16x16x32_bf16 v[68:71], v[228:231], v[204:207], v[68:71]\n"
    "v_mfma_f32_16x16x32_bf16 v[68:71], v[232:235], v[208:211], v[68:71]\n"
    "v_mfma_f32_16x16x32_bf16 v[72:75], v[236:239], v[204:207], v[72:75]\n"
    "v_mfma_f32_16x16x32_bf16 v[72:75], v[240:243], v[208:211], v[72:75]\n"
    "s_setprio 0\n"
    "s_barrier\n"
    "; phase 29: step 20 slot 1 stage 1\n"
    "s_bitcmp1_b32 s55, 8\n"
    "s_cbranch_scc1 .La35_%=\n"
    "s_sleep 2\n"
    "s_branch .La36_%=\n"
    ".La35_%=:\n"
    "s_sleep 4\n"
    ".La36_%=:\n"
    "ds_read_b64_tr_b16 v[212:213], v246 offset:16384\n"
    "ds_read_b64_tr_b16 v[214:215], v246 offset:17408\n"
    "s_add_u32 m0, s51, 0x0\n"
    "s_xor_b32 s82, s64, 0x1600000\n"
    "buffer_load_dwordx4 v251, s[40:43], s82 offen  lds\n"
    "ds_read_b64_tr_b16 v[216:217], v246 offset:24576\n"
    "ds_read_b64_tr_b16 v[218:219], v246 offset:25600\n"
    "s_add_u32 m0, s51, 0x2000\n"
    "s_xor_b32 s82, s64, 0x1680000\n"
    "buffer_load_dwordx4 v251, s[40:43], s82 offen  lds\n"
    "ds_read_b64_tr_b16 v[220:221], v247 offset:16384\n"
    "ds_read_b64_tr_b16 v[222:223], v247 offset:17408\n"
    "s_add_u32 m0, s50, 0x8000\n"
    "s_xor_b32 s82, s48, 0x200b80\n"
    "buffer_load_dwordx4 v250, s[36:39], s82 offen  lds\n"
    "ds_read_b64_tr_b16 v[224:225], v247 offset:24576\n"
    "ds_read_b64_tr_b16 v[226:227], v247 offset:25600\n"
    "s_add_u32 m0, s50, 0x9000\n"
    "s_xor_b32 s82, s48, 0x280b80\n"
    "buffer_load_dwordx4 v250, s[36:39], s82 offen  lds\n"
    "ds_read_b64_tr_b16 v[228:229], v248 offset:16384\n"
    "ds_read_b64_tr_b16 v[230:231], v248 offset:17408\n"
    "s_add_u32 m0, s50, 0xa000\n"
    "s_xor_b32 s82, s48, 0xb80\n"
    "buffer_load_dwordx4 v250, s[36:39], s82 offen  lds\n"
    "ds_read_b64_tr_b16 v[232:233], v248 offset:24576\n"
    "ds_read_b64_tr_b16 v[234:235], v248 offset:25600\n"
    "s_add_u32 m0, s50, 0xb000\n"
    "s_xor_b32 s82, s48, 0x80b80\n"
    "buffer_load_dwordx4 v250, s[36:39], s82 offen  lds\n"
    "ds_read_b64_tr_b16 v[236:237], v249 offset:16384\n"
    "ds_read_b64_tr_b16 v[238:239], v249 offset:17408\n"
    "ds_read_b64_tr_b16 v[240:241], v249 offset:24576\n"
    "ds_read_b64_tr_b16 v[242:243], v249 offset:25600\n"
    "s_waitcnt vmcnt(12) lgkmcnt(0)\n"
    "s_barrier\n"
    "s_setprio 1\n"
    "v_mfma_f32_16x16x32_bf16 v[76:79], v[212:215], v[180:183], v[76:79]\n"
    "v_mfma_f32_16x16x32_bf16 v[76:79], v[216:219], v[184:187], v[76:79]\n"
    "v_mfma_f32_16x16x32_bf16 v[80:83], v[220:223], v[180:183], v[80:83]\n"
    "v_mfma_f32_16x16x32_bf16 v[80:83], v[224:227], v[184:187], v[80:83]\n"
    "v_mfma_f32_16x16x32_bf16 v[84:87], v[228:231], v[180:183], v[84:87]\n"
    "v_mfma_f32_16x16x32_bf16 v[84:87], v[232:235], v[184:187], v[84:87]\n"
    "v_mfma_f32_16x16x32_bf16 v[88:91], v[236:239], v[180:183], v[88:91]\n"
    "v_mfma_f32_16x16x32_bf16 v[88:91], v[240:243], v[184:187], v[88:91]\n"
    "v_mfma_f32_16x16x32_bf16 v[92:95], v[212:215], v[188:191], v[92:95]\n"
    "v_mfma_f32_16x16x32_bf16 v[92:95], v[216:219], v[192:195], v[92:95]\n"
    "v_mfma_f32_16x16x32_bf16 v[96:99], v[220:223], v[188:191], v[96:99]\n"
    "v_mfma_f32_16x16x32_bf16 v[96:99], v[224:227], v[192:195], v[96:99]\n"
    "v_mfma_f32_16x16x32_bf16 v[100:103], v[228:231], v[188:191], v[100:103]\n"
    "v_mfma_f32_16x16x32_bf16 v[100:103], v[232:235], v[192:195], v[100:103]\n"
    "v_mfma_f32_16x16x32_bf16 v[104:107], v[236:239], v[188:191], v[104:107]\n"
    "v_mfma_f32_16x16x32_bf16 v[104:107], v[240:243], v[192:195], v[104:107]\n"
    "v_mfma_f32_16x16x32_bf16 v[108:111], v[212:215], v[196:199], v[108:111]\n"
    "v_mfma_f32_16x16x32_bf16 v[108:111], v[216:219], v[200:203], v[108:111]\n"
    "v_mfma_f32_16x16x32_bf16 v[112:115], v[220:223], v[196:199], v[112:115]\n"
    "v_mfma_f32_16x16x32_bf16 v[112:115], v[224:227], v[200:203], v[112:115]\n"
    "v_mfma_f32_16x16x32_bf16 v[116:119], v[228:231], v[196:199], v[116:119]\n"
    "v_mfma_f32_16x16x32_bf16 v[116:119], v[232:235], v[200:203], v[116:119]\n"
    "v_mfma_f32_16x16x32_bf16 v[120:123], v[236:239], v[196:199], v[120:123]\n"
    "v_mfma_f32_16x16x32_bf16 v[120:123], v[240:243], v[200:203], v[120:123]\n"
    "v_mfma_f32_16x16x32_bf16 v[124:127], v[212:215], v[204:207], v[124:127]\n"
    "v_mfma_f32_16x16x32_bf16 v[124:127], v[216:219], v[208:211], v[124:127]\n"
    "v_mfma_f32_16x16x32_bf16 v[128:131], v[220:223], v[204:207], v[128:131]\n"
    "v_mfma_f32_16x16x32_bf16 v[128:131], v[224:227], v[208:211], v[128:131]\n"
    "v_mfma_f32_16x16x32_bf16 v[132:135], v[228:231], v[204:207], v[132:135]\n"
    "v_mfma_f32_16x16x32_bf16 v[132:135], v[232:235], v[208:211], v[132:135]\n"
    "v_mfma_f32_16x16x32_bf16 v[136:139], v[236:239], v[204:207], v[136:139]\n"
    "v_mfma_f32_16x16x32_bf16 v[136:139], v[240:243], v[208:211], v[136:139]\n"
    "s_setprio 0\n"
    "s_barrier\n"
    "; phase 30: step 21 slot 0 stage 0\n"
    "s_bitcmp1_b32 s55, 8\n"
    "s_cbranch_scc1 .La37_%=\n"
    "s_sleep 2\n"
    "s_branch .La38_%=\n"
    ".La37_%=:\n"
    "s_sleep 4\n"
    ".La38_%=:\n"
    "ds_read_b128 v[180:183], v244 offset:0\n"
    "ds_read_b128 v[184:187], v245 offset:0\n"
    "ds_read_b128 v[188:191], v244 offset:2048\n"
    "ds_read_b128 v[192:195], v245 offset:2048\n"
    "s_add_u32 m0, s51, 0x4000\n"
    "s_xor_b32 s82, s65, 0x1600000\n"
    "buffer_load_dwordx4 v251, s[40:43], s82 offen  lds\n"
    "ds_read_b128 v[196:199], v244 offset:8192\n"
    "ds_read_b128 v[200:203], v245 offset:8192\n"
    "ds_read_b128 v[204:207], v244 offset:10240\n"
    "ds_read_b128 v[208:211], v245 offset:10240\n"
    "s_add_u32 m0, s51, 0x6000\n"
    "s_xor_b32 s82, s65, 0x1680000\n"
    "buffer_load_dwordx4 v251, s[40:43], s82 offen  lds\n"
    "ds_read_b64_tr_b16 v[212:213], v246 offset:32768\n"
    "ds_read_b64_tr_b16 v[214:215], v246 offset:33792\n"
    "ds_read_b64_tr_b16 v[216:217], v246 offset:40960\n"
    "ds_read_b64_tr_b16 v[218:219], v246 offset:41984\n"
    "ds_read_b64_tr_b16 v[220:221], v247 offset:32768\n"
    "ds_read_b64_tr_b16 v[222:223], v247 offset:33792\n"
    "ds_read_b64_tr_b16 v[224:225], v247 offset:40960\n"
    "ds_read_b64_tr_b16 v[226:227], v247 offset:41984\n"
    "ds_read_b64_tr_b16 v[228:229], v248 offset:32768\n"
    "ds_read_b64_tr_b16 v[230:231], v248 offset:33792\n"
    "ds_read_b64_tr_b16 v[232:233], v248 offset:40960\n"
    "ds_read_b64_tr_b16 v[234:235], v248 offset:41984\n"
    "ds_read_b64_tr_b16 v[236:237], v249 offset:32768\n"
    "ds_read_b64_tr_b16 v[238:239], v249 offset:33792\n"
    "ds_read_b64_tr_b16 v[240:241], v249 offset:40960\n"
    "ds_read_b64_tr_b16 v[242:243], v249 offset:41984\n"
    "s_waitcnt vmcnt(8) lgkmcnt(0)\n"
    "s_barrier\n"
    "s_setprio 1\n"
    "v_mfma_f32_16x16x32_bf16 v[12:15], v[212:215], v[180:183], v[12:15]\n"
    "v_mfma_f32_16x16x32_bf16 v[12:15], v[216:219], v[184:187], v[12:15]\n"
    "v_mfma_f32_16x16x32_bf16 v[16:19], v[220:223], v[180:183], v[16:19]\n"
    "v_mfma_f32_16x16x32_bf16 v[16:19], v[224:227], v[184:187], v[16:19]\n"
    "v_mfma_f32_16x16x32_bf16 v[20:23], v[228:231], v[180:183], v[20:23]\n"
    "v_mfma_f32_16x16x32_bf16 v[20:23], v[232:235], v[184:187], v[20:23]\n"
    "v_mfma_f32_16x16x32_bf16 v[24:27], v[236:239], v[180:183], v[24:27]\n"
    "v_mfma_f32_16x16x32_bf16 v[24:27], v[240:243], v[184:187], v[24:27]\n"
    "v_mfma_f32_16x16x32_bf16 v[28:31], v[212:215], v[188:191], v[28:31]\n"
    "v_mfma_f32_16x16x32_bf16 v[28:31], v[216:219], v[192:195], v[28:31]\n"
    "v_mfma_f32_16x16x32_bf16 v[32:35], v[220:223], v[188:191], v[32:35]\n"
    "v_mfma_f32_16x16x32_bf16 v[32:35], v[224:227], v[192:195], v[32:35]\n"
    "v_mfma_f32_16x16x32_bf16 v[36:39], v[228:231], v[188:191], v[36:39]\n"
    "v_mfma_f32_16x16x32_bf16 v[36:39], v[232:235], v[192:195], v[36:39]\n"
    "v_mfma_f32_16x16x32_bf16 v[40:43], v[236:239], v[188:191], v[40:43]\n"
    "v_mfma_f32_16x16x32_bf16 v[40:43], v[240:243], v[192:195], v[40:43]\n"
    "v_mfma_f32_16x16x32_bf16 v[44:47], v[212:215], v[196:199], v[44:47]\n"
    "v_mfma_f32_16x16x32_bf16 v[44:47], v[216:219], v[200:203], v[44:47]\n"
    "v_mfma_f32_16x16x32_bf16 v[48:51], v[220:223], v[196:199], v[48:51]\n"
    "v_mfma_f32_16x16x32_bf16 v[48:51], v[224:227], v[200:203], v[48:51]\n"
    "v_mfma_f32_16x16x32_bf16 v[52:55], v[228:231], v[196:199], v[52:55]\n"
    "v_mfma_f32_16x16x32_bf16 v[52:55], v[232:235], v[200:203], v[52:55]\n"
    "v_mfma_f32_16x16x32_bf16 v[56:59], v[236:239], v[196:199], v[56:59]\n"
    "v_mfma_f32_16x16x32_bf16 v[56:59], v[240:243], v[200:203], v[56:59]\n"
    "v_mfma_f32_16x16x32_bf16 v[60:63], v[212:215], v[204:207], v[60:63]\n"
    "v_mfma_f32_16x16x32_bf16 v[60:63], v[216:219], v[208:211], v[60:63]\n"
    "v_mfma_f32_16x16x32_bf16 v[64:67], v[220:223], v[204:207], v[64:67]\n"
    "v_mfma_f32_16x16x32_bf16 v[64:67], v[224:227], v[208:211], v[64:67]\n"
    "v_mfma_f32_16x16x32_bf16 v[68:71], v[228:231], v[204:207], v[68:71]\n"
    "v_mfma_f32_16x16x32_bf16 v[68:71], v[232:235], v[208:211], v[68:71]\n"
    "v_mfma_f32_16x16x32_bf16 v[72:75], v[236:239], v[204:207], v[72:75]\n"
    "v_mfma_f32_16x16x32_bf16 v[72:75], v[240:243], v[208:211], v[72:75]\n"
    "s_setprio 0\n"
    "s_barrier\n"
    "; phase 31: step 21 slot 1 stage 1\n"
    "s_bitcmp1_b32 s55, 8\n"
    "s_cbranch_scc1 .La39_%=\n"
    "s_sleep 2\n"
    "s_branch .La40_%=\n"
    ".La39_%=:\n"
    "s_sleep 4\n"
    ".La40_%=:\n"
    "ds_read_b64_tr_b16 v[212:213], v246 offset:49152\n"
    "ds_read_b64_tr_b16 v[214:215], v246 offset:50176\n"
    "s_add_u32 m0, s51, 0x8000\n"
    "s_xor_b32 s82, s64, 0x1700000\n"
    "buffer_load_dwordx4 v251, s[40:43], s82 offen  lds\n"
    "ds_read_b64_tr_b16 v[216:217], v246 offset:57344\n"
    "ds_read_b64_tr_b16 v[218:219], v246 offset:58368\n"
    "s_add_u32 m0, s51, 0xa000\n"
    "s_xor_b32 s82, s64, 0x1780000\n"
    "buffer_load_dwordx4 v251, s[40:43], s82 offen  lds\n"
    "ds_read_b64_tr_b16 v[220:221], v247 offset:49152\n"
    "ds_read_b64_tr_b16 v[222:223], v247 offset:50176\n"
    "s_add_u32 m0, s50, 0x0\n"
    "s_xor_b32 s82, s48, 0x200c00\n"
    "buffer_load_dwordx4 v250, s[36:39], s82 offen  lds\n"
    "ds_read_b64_tr_b16 v[224:225], v247 offset:57344\n"
    "ds_read_b64_tr_b16 v[226:227], v247 offset:58368\n"
    "s_add_u32 m0, s50, 0x1000\n"
    "s_xor_b32 s82, s48, 0x280c00\n"
    "buffer_load_dwordx4 v250, s[36:39], s82 offen  lds\n"
    "ds_read_b64_tr_b16 v[228:229], v248 offset:49152\n"
    "ds_read_b64_tr_b16 v[230:231], v248 offset:50176\n"
    "s_add_u32 m0, s50, 0x2000\n"
    "s_xor_b32 s82, s48, 0xc00\n"
    "buffer_load_dwordx4 v250, s[36:39], s82 offen  lds\n"
    "ds_read_b64_tr_b16 v[232:233], v248 offset:57344\n"
    "ds_read_b64_tr_b16 v[234:235], v248 offset:58368\n"
    "s_add_u32 m0, s50, 0x3000\n"
    "s_xor_b32 s82, s48, 0x80c00\n"
    "buffer_load_dwordx4 v250, s[36:39], s82 offen  lds\n"
    "ds_read_b64_tr_b16 v[236:237], v249 offset:49152\n"
    "ds_read_b64_tr_b16 v[238:239], v249 offset:50176\n"
    "ds_read_b64_tr_b16 v[240:241], v249 offset:57344\n"
    "ds_read_b64_tr_b16 v[242:243], v249 offset:58368\n"
    "s_waitcnt vmcnt(12) lgkmcnt(0)\n"
    "s_barrier\n"
    "s_setprio 1\n"
    "v_mfma_f32_16x16x32_bf16 v[76:79], v[212:215], v[180:183], v[76:79]\n"
    "v_mfma_f32_16x16x32_bf16 v[76:79], v[216:219], v[184:187], v[76:79]\n"
    "v_mfma_f32_16x16x32_bf16 v[80:83], v[220:223], v[180:183], v[80:83]\n"
    "v_mfma_f32_16x16x32_bf16 v[80:83], v[224:227], v[184:187], v[80:83]\n"
    "v_mfma_f32_16x16x32_bf16 v[84:87], v[228:231], v[180:183], v[84:87]\n"
    "v_mfma_f32_16x16x32_bf16 v[84:87], v[232:235], v[184:187], v[84:87]\n"
    "v_mfma_f32_16x16x32_bf16 v[88:91], v[236:239], v[180:183], v[88:91]\n"
    "v_mfma_f32_16x16x32_bf16 v[88:91], v[240:243], v[184:187], v[88:91]\n"
    "v_mfma_f32_16x16x32_bf16 v[92:95], v[212:215], v[188:191], v[92:95]\n"
    "v_mfma_f32_16x16x32_bf16 v[92:95], v[216:219], v[192:195], v[92:95]\n"
    "v_mfma_f32_16x16x32_bf16 v[96:99], v[220:223], v[188:191], v[96:99]\n"
    "v_mfma_f32_16x16x32_bf16 v[96:99], v[224:227], v[192:195], v[96:99]\n"
    "v_mfma_f32_16x16x32_bf16 v[100:103], v[228:231], v[188:191], v[100:103]\n"
    "v_mfma_f32_16x16x32_bf16 v[100:103], v[232:235], v[192:195], v[100:103]\n"
    "v_mfma_f32_16x16x32_bf16 v[104:107], v[236:239], v[188:191], v[104:107]\n"
    "v_mfma_f32_16x16x32_bf16 v[104:107], v[240:243], v[192:195], v[104:107]\n"
    "v_mfma_f32_16x16x32_bf16 v[108:111], v[212:215], v[196:199], v[108:111]\n"
    "v_mfma_f32_16x16x32_bf16 v[108:111], v[216:219], v[200:203], v[108:111]\n"
    "v_mfma_f32_16x16x32_bf16 v[112:115], v[220:223], v[196:199], v[112:115]\n"
    "v_mfma_f32_16x16x32_bf16 v[112:115], v[224:227], v[200:203], v[112:115]\n"
    "v_mfma_f32_16x16x32_bf16 v[116:119], v[228:231], v[196:199], v[116:119]\n"
    "v_mfma_f32_16x16x32_bf16 v[116:119], v[232:235], v[200:203], v[116:119]\n"
    "v_mfma_f32_16x16x32_bf16 v[120:123], v[236:239], v[196:199], v[120:123]\n"
    "v_mfma_f32_16x16x32_bf16 v[120:123], v[240:243], v[200:203], v[120:123]\n"
    "v_mfma_f32_16x16x32_bf16 v[124:127], v[212:215], v[204:207], v[124:127]\n"
    "v_mfma_f32_16x16x32_bf16 v[124:127], v[216:219], v[208:211], v[124:127]\n"
    "v_mfma_f32_16x16x32_bf16 v[128:131], v[220:223], v[204:207], v[128:131]\n"
    "v_mfma_f32_16x16x32_bf16 v[128:131], v[224:227], v[208:211], v[128:131]\n"
    "v_mfma_f32_16x16x32_bf16 v[132:135], v[228:231], v[204:207], v[132:135]\n"
    "v_mfma_f32_16x16x32_bf16 v[132:135], v[232:235], v[208:211], v[132:135]\n"
    "v_mfma_f32_16x16x32_bf16 v[136:139], v[236:239], v[204:207], v[136:139]\n"
    "v_mfma_f32_16x16x32_bf16 v[136:139], v[240:243], v[208:211], v[136:139]\n"
    "s_setprio 0\n"
    "s_barrier\n"
    "; phase 32: step 22 slot 0 stage 0\n"
    "s_bitcmp1_b32 s55, 8\n"
    "s_cbranch_scc1 .La41_%=\n"
    "s_sleep 2\n"
    "s_branch .La42_%=\n"
    ".La41_%=:\n"
    "s_sleep 4\n"
    ".La42_%=:\n"
    "ds_read_b128 v[180:183], v244 offset:16384\n"
    "ds_read_b128 v[184:187], v245 offset:16384\n"
    "ds_read_b128 v[188:191], v244 offset:18432\n"
    "ds_read_b128 v[192:195], v245 offset:18432\n"
    "s_add_u32 m0, s51, 0xc000\n"
    "s_xor_b32 s82, s65, 0x1700000\n"
    "buffer_load_dwordx4 v251, s[40:43], s82 offen  lds\n"
    "ds_read_b128 v[196:199], v244 offset:24576\n"
    "ds_read_b128 v[200:203], v245 offset:24576\n"
    "ds_read_b128 v[204:207], v244 offset:26624\n"
    "ds_read_b128 v[208:211], v245 offset:26624\n"
    "s_add_u32 m0, s51, 0xe000\n"
    "s_xor_b32 s82, s65, 0x1780000\n"
    "buffer_load_dwordx4 v251, s[40:43], s82 offen  lds\n"
    "ds_read_b64_tr_b16 v[212:213], v246 offset:0\n"
    "ds_read_b64_tr_b16 v[214:215], v246 offset:1024\n"
    "ds_read_b64_tr_b16 v[216:217], v246 offset:8192\n"
    "ds_read_b64_tr_b16 v[218:219], v246 offset:9216\n"
    "ds_read_b64_tr_b16 v[220:221], v247 offset:0\n"
    "ds_read_b64_tr_b16 v[222:223], v247 offset:1024\n"
    "ds_read_b64_tr_b16 v[224:225], v247 offset:8192\n"
    "ds_read_b64_tr_b16 v[226:227], v247 offset:9216\n"
    "ds_read_b64_tr_b16 v[228:229], v248 offset:0\n"
    "ds_read_b64_tr_b16 v[230:231], v248 offset:1024\n"
    "ds_read_b64_tr_b16 v[232:233], v248 offset:8192\n"
    "ds_read_b64_tr_b16 v[234:235], v248 offset:9216\n"
    "ds_read_b64_tr_b16 v[236:237], v249 offset:0\n"
    "ds_read_b64_tr_b16 v[238:239], v249 offset:1024\n"
    "ds_read_b64_tr_b16 v[240:241], v249 offset:8192\n"
    "ds_read_b64_tr_b16 v[242:243], v249 offset:9216\n"
    "s_waitcnt vmcnt(8) lgkmcnt(0)\n"
    "s_barrier\n"
    "s_setprio 1\n"
    "v_mfma_f32_16x16x32_bf16 v[12:15], v[212:215], v[180:183], v[12:15]\n"
    "v_mfma_f32_16x16x32_bf16 v[12:15], v[216:219], v[184:187], v[12:15]\n"
    "v_mfma_f32_16x16x32_bf16 v[16:19], v[220:223], v[180:183], v[16:19]\n"
    "v_mfma_f32_16x16x32_bf16 v[16:19], v[224:227], v[184:187], v[16:19]\n"
    "v_mfma_f32_16x16x32_bf16 v[20:23], v[228:231], v[180:183], v[20:23]\n"
    "v_mfma_f32_16x16x32_bf16 v[20:23], v[232:235], v[184:187], v[20:23]\n"
    "v_mfma_f32_16x16x32_bf16 v[24:27], v[236:239], v[180:183], v[24:27]\n"
    "v_mfma_f32_16x16x32_bf16 v[24:27], v[240:243], v[184:187], v[24:27]\n"
    "v_mfma_f32_16x16x32_bf16 v[28:31], v[212:215], v[188:191], v[28:31]\n"
    "v_mfma_f32_16x16x32_bf16 v[28:31], v[216:219], v[192:195], v[28:31]\n"
    "v_mfma_f32_16x16x32_bf16 v[32:35], v[220:223], v[188:191], v[32:35]\n"
    "v_mfma_f32_16x16x32_bf16 v[32:35], v[224:227], v[192:195], v[32:35]\n"
    "v_mfma_f32_16x16x32_bf16 v[36:39], v[228:231], v[188:191], v[36:39]\n"
    "v_mfma_f32_16x16x32_bf16 v[36:39], v[232:235], v[192:195], v[36:39]\n"
    "v_mfma_f32_16x16x32_bf16 v[40:43], v[236:239], v[188:191], v[40:43]\n"
    "v_mfma_f32_16x16x32_bf16 v[40:43], v[240:243], v[192:195], v[40:43]\n"
    "v_mfma_f32_16x16x32_bf16 v[44:47], v[212:215], v[196:199], v[44:47]\n"
    "v_mfma_f32_16x16x32_bf16 v[44:47], v[216:219], v[200:203], v[44:47]\n"
    "v_mfma_f32_16x16x32_bf16 v[48:51], v[220:223], v[196:199], v[48:51]\n"
    "v_mfma_f32_16x16x32_bf16 v[48:51], v[224:227], v[200:203], v[48:51]\n"
    "v_mfma_f32_16x16x32_bf16 v[52:55], v[228:231], v[196:199], v[52:55]\n"
    "v_mfma_f32_16x16x32_bf16 v[52:55], v[232:235], v[200:203], v[52:55]\n"
    "v_mfma_f32_16x16x32_bf16 v[56:59], v[236:239], v[196:199], v[56:59]\n"
    "v_mfma_f32_16x16x32_bf16 v[56:59], v[240:243], v[200:203], v[56:59]\n"
    "v_mfma_f32_16x16x32_bf16 v[60:63], v[212:215], v[204:207], v[60:63]\n"
    "v_mfma_f32_16x16x32_bf16 v[60:63], v[216:219], v[208:211], v[60:63]\n"
    "v_mfma_f32_16x16x32_bf16 v[64:67], v[220:223], v[204:207], v[64:67]\n"
    "v_mfma_f32_16x16x32_bf16 v[64:67], v[224:227], v[208:211], v[64:67]\n"
    "v_mfma_f32_16x16x32_bf16 v[68:71], v[228:231], v[204:207], v[68:71]\n"
    "v_mfma_f32_16x16x32_bf16 v[68:71], v[232:235], v[208:211], v[68:71]\n"
    "v_mfma_f32_16x16x32_bf16 v[72:75], v[236:239], v[204:207], v[72:75]\n"
    "v_mfma_f32_16x16x32_bf16 v[72:75], v[240:243], v[208:211], v[72:75]\n"
    "s_setprio 0\n"
    "s_barrier\n"
    "; phase 33: step 22 slot 1 stage 1\n"
    "s_bitcmp1_b32 s55, 8\n"
    "s_cbranch_scc1 .La43_%=\n"
    "s_sleep 2\n"
    "s_branch .La44_%=\n"
    ".La43_%=:\n"
    "s_sleep 4\n"
    ".La44_%=:\n"
    "ds_read_b64_tr_b16 v[212:213], v246 offset:16384\n"
    "ds_read_b64_tr_b16 v[214:215], v246 offset:17408\n"
    "s_add_u32 m0, s51, 0x0\n"
    "s_xor_b32 s82, s64, 0x1800000\n"
    "buffer_load_dwordx4 v251, s[40:43], s82 offen  lds\n"
    "ds_read_b64_tr_b16 v[216:217], v246 offset:24576\n"
    "ds_read_b64_tr_b16 v[218:219], v246 offset:25600\n"
    "s_add_u32 m0, s51, 0x2000\n"
    "s_xor_b32 s82, s64, 0x1880000\n"
    "buffer_load_dwordx4 v251, s[40:43], s82 offen  lds\n"
    "ds_read_b64_tr_b16 v[220:221], v247 offset:16384\n"
    "ds_read_b64_tr_b16 v[222:223], v247 offset:17408\n"
    "s_add_u32 m0, s50, 0x4000\n"
    "s_xor_b32 s82, s48, 0x200c80\n"
    "buffer_load_dwordx4 v250, s[36:39], s82 offen  lds\n"
    "ds_read_b64_tr_b16 v[224:225], v247 offset:24576\n"
    "ds_read_b64_tr_b16 v[226:227], v247 offset:25600\n"
    "s_add_u32 m0, s50, 0x5000\n"
    "s_xor_b32 s82, s48, 0x280c80\n"
    "buffer_load_dwordx4 v250, s[36:39], s82 offen  lds\n"
    "ds_read_b64_tr_b16 v[228:229], v248 offset:16384\n"
    "ds_read_b64_tr_b16 v[230:231], v248 offset:17408\n"
    "s_add_u32 m0, s50, 0x6000\n"
    "s_xor_b32 s82, s48, 0xc80\n"
    "buffer_load_dwordx4 v250, s[36:39], s82 offen  lds\n"
    "ds_read_b64_tr_b16 v[232:233], v248 offset:24576\n"
    "ds_read_b64_tr_b16 v[234:235], v248 offset:25600\n"
    "s_add_u32 m0, s50, 0x7000\n"
    "s_xor_b32 s82, s48, 0x80c80\n"
    "buffer_load_dwordx4 v250, s[36:39], s82 offen  lds\n"
    "ds_read_b64_tr_b16 v[236:237], v249 offset:16384\n"
    "ds_read_b64_tr_b16 v[238:239], v249 offset:17408\n"
    "ds_read_b64_tr_b16 v[240:241], v249 offset:24576\n"
    "ds_read_b64_tr_b16 v[242:243], v249 offset:25600\n"
    "s_waitcnt vmcnt(12) lgkmcnt(0)\n"
    "s_barrier\n"
    "s_setprio 1\n"
    "v_mfma_f32_16x16x32_bf16 v[76:79], v[212:215], v[180:183], v[76:79]\n"
    "v_mfma_f32_16x16x32_bf16 v[76:79], v[216:219], v[184:187], v[76:79]\n"
    "v_mfma_f32_16x16x32_bf16 v[80:83], v[220:223], v[180:183], v[80:83]\n"
    "v_mfma_f32_16x16x32_bf16 v[80:83], v[224:227], v[184:187], v[80:83]\n"
    "v_mfma_f32_16x16x32_bf16 v[84:87], v[228:231], v[180:183], v[84:87]\n"
    "v_mfma_f32_16x16x32_bf16 v[84:87], v[232:235], v[184:187], v[84:87]\n"
    "v_mfma_f32_16x16x32_bf16 v[88:91], v[236:239], v[180:183], v[88:91]\n"
    "v_mfma_f32_16x16x32_bf16 v[88:91], v[240:243], v[184:187], v[88:91]\n"
    "v_mfma_f32_16x16x32_bf16 v[92:95], v[212:215], v[188:191], v[92:95]\n"
    "v_mfma_f32_16x16x32_bf16 v[92:95], v[216:219], v[192:195], v[92:95]\n"
    "v_mfma_f32_16x16x32_bf16 v[96:99], v[220:223], v[188:191], v[96:99]\n"
    "v_mfma_f32_16x16x32_bf16 v[96:99], v[224:227], v[192:195], v[96:99]\n"
    "v_mfma_f32_16x16x32_bf16 v[100:103], v[228:231], v[188:191], v[100:103]\n"
    "v_mfma_f32_16x16x32_bf16 v[100:103], v[232:235], v[192:195], v[100:103]\n"
    "v_mfma_f32_16x16x32_bf16 v[104:107], v[236:239], v[188:191], v[104:107]\n"
    "v_mfma_f32_16x16x32_bf16 v[104:107], v[240:243], v[192:195], v[104:107]\n"
    "v_mfma_f32_16x16x32_bf16 v[108:111], v[212:215], v[196:199], v[108:111]\n"
    "v_mfma_f32_16x16x32_bf16 v[108:111], v[216:219], v[200:203], v[108:111]\n"
    "v_mfma_f32_16x16x32_bf16 v[112:115], v[220:223], v[196:199], v[112:115]\n"
    "v_mfma_f32_16x16x32_bf16 v[112:115], v[224:227], v[200:203], v[112:115]\n"
    "v_mfma_f32_16x16x32_bf16 v[116:119], v[228:231], v[196:199], v[116:119]\n"
    "v_mfma_f32_16x16x32_bf16 v[116:119], v[232:235], v[200:203], v[116:119]\n"
    "v_mfma_f32_16x16x32_bf16 v[120:123], v[236:239], v[196:199], v[120:123]\n"
    "v_mfma_f32_16x16x32_bf16 v[120:123], v[240:243], v[200:203], v[120:123]\n"
    "v_mfma_f32_16x16x32_bf16 v[124:127], v[212:215], v[204:207], v[124:127]\n"
    "v_mfma_f32_16x16x32_bf16 v[124:127], v[216:219], v[208:211], v[124:127]\n"
    "v_mfma_f32_16x16x32_bf16 v[128:131], v[220:223], v[204:207], v[128:131]\n"
    "v_mfma_f32_16x16x32_bf16 v[128:131], v[224:227], v[208:211], v[128:131]\n"
    "v_mfma_f32_16x16x32_bf16 v[132:135], v[228:231], v[204:207], v[132:135]\n"
    "v_mfma_f32_16x16x32_bf16 v[132:135], v[232:235], v[208:211], v[132:135]\n"
    "v_mfma_f32_16x16x32_bf16 v[136:139], v[236:239], v[204:207], v[136:139]\n"
    "v_mfma_f32_16x16x32_bf16 v[136:139], v[240:243], v[208:211], v[136:139]\n"
    "s_setprio 0\n"
    "s_barrier\n"
    "; phase 34: step 23 slot 0 stage 0\n"
    "s_bitcmp1_b32 s55, 8\n"
    "s_cbranch_scc1 .La45_%=\n"
    "s_sleep 2\n"
    "s_branch .La46_%=\n"
    ".La45_%=:\n"
    "s_sleep 4\n"
    ".La46_%=:\n"
    "ds_read_b128 v[180:183], v244 offset:32768\n"
    "ds_read_b128 v[184:187], v245 offset:32768\n"
    "ds_read_b128 v[188:191], v244 offset:34816\n"
    "ds_read_b128 v[192:195], v245 offset:34816\n"
    "s_add_u32 m0, s51, 0x4000\n"
    "s_xor_b32 s82, s65, 0x1800000\n"
    "buffer_load_dwordx4 v251, s[40:43], s82 offen  lds\n"
    "ds_read_b128 v[196:199], v244 offset:40960\n"
    "ds_read_b128 v[200:203], v245 offset:40960\n"
    "ds_read_b128 v[204:207], v244 offset:43008\n"
    "ds_read_b128 v[208:211], v245 offset:43008\n"
    "s_add_u32 m0, s51, 0x6000\n"
    "s_xor_b32 s82, s65, 0x1880000\n"
    "buffer_load_dwordx4 v251, s[40:43], s82 offen  lds\n"
    "ds_read_b64_tr_b16 v[212:213], v246 offset:32768\n"
    "ds_read_b64_tr_b16 v[214:215], v246 offset:33792\n"
    "ds_read_b64_tr_b16 v[216:217], v246 offset:40960\n"
    "ds_read_b64_tr_b16 v[218:219], v246 offset:41984\n"
    "ds_read_b64_tr_b16 v[220:221], v247 offset:32768\n"
    "ds_read_b64_tr_b16 v[222:223], v247 offset:33792\n"
    "ds_read_b64_tr_b16 v[224:225], v247 offset:40960\n"
    "ds_read_b64_tr_b16 v[226:227], v247 offset:41984\n"
    "ds_read_b64_tr_b16 v[228:229], v248 offset:32768\n"
    "ds_read_b64_tr_b16 v[230:231], v248 offset:33792\n"
    "ds_read_b64_tr_b16 v[232:233], v248 offset:40960\n"
    "ds_read_b64_tr_b16 v[234:235], v248 offset:41984\n"
    "ds_read_b64_tr_b16 v[236:237], v249 offset:32768\n"
    "ds_read_b64_tr_b16 v[238:239], v249 offset:33792\n"
    "ds_read_b64_tr_b16 v[240:241], v249 offset:40960\n"
    "ds_read_b64_tr_b16 v[242:243], v249 offset:41984\n"
    "s_waitcnt vmcnt(8) lgkmcnt(0)\n"
    "s_barrier\n"
    "s_setprio 1\n"
    "v_mfma_f32_16x16x32_bf16 v[12:15], v[212:215], v[180:183], v[12:15]\n"
    "v_mfma_f32_16x16x32_bf16 v[12:15], v[216:219], v[184:187], v[12:15]\n"
    "v_mfma_f32_16x16x32_bf16 v[16:19], v[220:223], v[180:183], v[16:19]\n"
    "v_mfma_f32_16x16x32_bf16 v[16:19], v[224:227], v[184:187], v[16:19]\n"
    "v_mfma_f32_16x16x32_bf16 v[20:23], v[228:231], v[180:183], v[20:23]\n"
    "v_mfma_f32_16x16x32_bf16 v[20:23], v[232:235], v[184:187], v[20:23]\n"
    "v_mfma_f32_16x16x32_bf16 v[24:27], v[236:239], v[180:183], v[24:27]\n"
    "v_mfma_f32_16x16x32_bf16 v[24:27], v[240:243], v[184:187], v[24:27]\n"
    "v_mfma_f32_16x16x32_bf16 v[28:31], v[212:215], v[188:191], v[28:31]\n"
    "v_mfma_f32_16x16x32_bf16 v[28:31], v[216:219], v[192:195], v[28:31]\n"
    "v_mfma_f32_16x16x32_bf16 v[32:35], v[220:223], v[188:191], v[32:35]\n"
    "v_mfma_f32_16x16x32_bf16 v[32:35], v[224:227], v[192:195], v[32:35]\n"
    "v_mfma_f32_16x16x32_bf16 v[36:39], v[228:231], v[188:191], v[36:39]\n"
    "v_mfma_f32_16x16x32_bf16 v[36:39], v[232:235], v[192:195], v[36:39]\n"
    "v_mfma_f32_16x16x32_bf16 v[40:43], v[236:239], v[188:191], v[40:43]\n"
    "v_mfma_f32_16x16x32_bf16 v[40:43], v[240:243], v[192:195], v[40:43]\n"
    "v_mfma_f32_16x16x32_bf16 v[44:47], v[212:215], v[196:199], v[44:47]\n"
    "v_mfma_f32_16x16x32_bf16 v[44:47], v[216:219], v[200:203], v[44:47]\n"
    "v_mfma_f32_16x16x32_bf16 v[48:51], v[220:223], v[196:199], v[48:51]\n"
    "v_mfma_f32_16x16x32_bf16 v[48:51], v[224:227], v[200:203], v[48:51]\n"
    "v_mfma_f32_16x16x32_bf16 v[52:55], v[228:231], v[196:199], v[52:55]\n"
    "v_mfma_f32_16x16x32_bf16 v[52:55], v[232:235], v[200:203], v[52:55]\n"
    "v_mfma_f32_16x16x32_bf16 v[56:59], v[236:239], v[196:199], v[56:59]\n"
    "v_mfma_f32_16x16x32_bf16 v[56:59], v[240:243], v[200:203], v[56:59]\n"
    "v_mfma_f32_16x16x32_bf16 v[60:63], v[212:215], v[204:207], v[60:63]\n"
    "v_mfma_f32_16x16x32_bf16 v[60:63], v[216:219], v[208:211], v[60:63]\n"
    "v_mfma_f32_16x16x32_bf16 v[64:67], v[220:223], v[204:207], v[64:67]\n"
    "v_mfma_f32_16x16x32_bf16 v[64:67], v[224:227], v[208:211], v[64:67]\n"
    "v_mfma_f32_16x16x32_bf16 v[68:71], v[228:231], v[204:207], v[68:71]\n"
    "v_mfma_f32_16x16x32_bf16 v[68:71], v[232:235], v[208:211], v[68:71]\n"
    "v_mfma_f32_16x16x32_bf16 v[72:75], v[236:239], v[204:207], v[72:75]\n"
    "v_mfma_f32_16x16x32_bf16 v[72:75], v[240:243], v[208:211], v[72:75]\n"
    "s_setprio 0\n"
    "s_barrier\n"
    "; phase 35: step 23 slot 1 stage 1\n"
    "s_bitcmp1_b32 s55, 8\n"
    "s_cbranch_scc1 .La47_%=\n"
    "s_sleep 2\n"
    "s_branch .La48_%=\n"
    ".La47_%=:\n"
    "s_sleep 4\n"
    ".La48_%=:\n"
    "ds_read_b64_tr_b16 v[212:213], v246 offset:49152\n"
    "ds_read_b64_tr_b16 v[214:215], v246 offset:50176\n"
    "s_add_u32 m0, s51, 0x8000\n"
    "s_xor_b32 s82, s64, 0x1900000\n"
    "buffer_load_dwordx4 v251, s[40:43], s82 offen  lds\n"
    "ds_read_b64_tr_b16 v[216:217], v246 offset:57344\n"
    "ds_read_b64_tr_b16 v[218:219], v246 offset:58368\n"
    "s_add_u32 m0, s51, 0xa000\n"
    "s_xor_b32 s82, s64, 0x1980000\n"
    "buffer_load_dwordx4 v251, s[40:43], s82 offen  lds\n"
    "ds_read_b64_tr_b16 v[220:221], v247 offset:49152\n"
    "ds_read_b64_tr_b16 v[222:223], v247 offset:50176\n"
    "s_add_u32 m0, s50, 0x8000\n"
    "s_xor_b32 s82, s48, 0x200d00\n"
    "buffer_load_dwordx4 v250, s[36:39], s82 offen  lds\n"
    "ds_read_b64_tr_b16 v[224:225], v247 offset:57344\n"
    "ds_read_b64_tr_b16 v[226:227], v247 offset:58368\n"
    "s_add_u32 m0, s50, 0x9000\n"
    "s_xor_b32 s82, s48, 0x280d00\n"
    "buffer_load_dwordx4 v250, s[36:39], s82 offen  lds\n"
    "ds_read_b64_tr_b16 v[228:229], v248 offset:49152\n"
    "ds_read_b64_tr_b16 v[230:231], v248 offset:50176\n"
    "s_add_u32 m0, s50, 0xa000\n"
    "s_xor_b32 s82, s48, 0xd00\n"
    "buffer_load_dwordx4 v250, s[36:39], s82 offen  lds\n"
    "ds_read_b64_tr_b16 v[232:233], v248 offset:57344\n"
    "ds_read_b64_tr_b16 v[234:235], v248 offset:58368\n"
    "s_add_u32 m0, s50, 0xb000\n"
    "s_xor_b32 s82, s48, 0x80d00\n"
    "buffer_load_dwordx4 v250, s[36:39], s82 offen  lds\n"
    "ds_read_b64_tr_b16 v[236:237], v249 offset:49152\n"
    "ds_read_b64_tr_b16 v[238:239], v249 offset:50176\n"
    "ds_read_b64_tr_b16 v[240:241], v249 offset:57344\n"
    "ds_read_b64_tr_b16 v[242:243], v249 offset:58368\n"
    "s_waitcnt vmcnt(12) lgkmcnt(0)\n"
    "s_barrier\n"
    "s_setprio 1\n"
    "v_mfma_f32_16x16x32_bf16 v[76:79], v[212:215], v[180:183], v[76:79]\n"
    "v_mfma_f32_16x16x32_bf16 v[76:79], v[216:219], v[184:187], v[76:79]\n"
    "v_mfma_f32_16x16x32_bf16 v[80:83], v[220:223], v[180:183], v[80:83]\n"
    "v_mfma_f32_16x16x32_bf16 v[80:83], v[224:227], v[184:187], v[80:83]\n"
    "v_mfma_f32_16x16x32_bf16 v[84:87], v[228:231], v[180:183], v[84:87]\n"
    "v_mfma_f32_16x16x32_bf16 v[84:87], v[232:235], v[184:187], v[84:87]\n"
    "v_mfma_f32_16x16x32_bf16 v[88:91], v[236:239], v[180:183], v[88:91]\n"
    "v_mfma_f32_16x16x32_bf16 v[88:91], v[240:243], v[184:187], v[88:91]\n"
    "v_mfma_f32_16x16x32_bf16 v[92:95], v[212:215], v[188:191], v[92:95]\n"
    "v_mfma_f32_16x16x32_bf16 v[92:95], v[216:219], v[192:195], v[92:95]\n"
    "v_mfma_f32_16x16x32_bf16 v[96:99], v[220:223], v[188:191], v[96:99]\n"
    "v_mfma_f32_16x16x32_bf16 v[96:99], v[224:227], v[192:195], v[96:99]\n"
    "v_mfma_f32_16x16x32_bf16 v[100:103], v[228:231], v[188:191], v[100:103]\n"
    "v_mfma_f32_16x16x32_bf16 v[100:103], v[232:235], v[192:195], v[100:103]\n"
    "v_mfma_f32_16x16x32_bf16 v[104:107], v[236:239], v[188:191], v[104:107]\n"
    "v_mfma_f32_16x16x32_bf16 v[104:107], v[240:243], v[192:195], v[104:107]\n"
    "v_mfma_f32_16x16x32_bf16 v[108:111], v[212:215], v[196:199], v[108:111]\n"
    "v_mfma_f32_16x16x32_bf16 v[108:111], v[216:219], v[200:203], v[108:111]\n"
    "v_mfma_f32_16x16x32_bf16 v[112:115], v[220:223], v[196:199], v[112:115]\n"
    "v_mfma_f32_16x16x32_bf16 v[112:115], v[224:227], v[200:203], v[112:115]\n"
    "v_mfma_f32_16x16x32_bf16 v[116:119], v[228:231], v[196:199], v[116:119]\n"
    "v_mfma_f32_16x16x32_bf16 v[116:119], v[232:235], v[200:203], v[116:119]\n"
    "v_mfma_f32_16x16x32_bf16 v[120:123], v[236:239], v[196:199], v[120:123]\n"
    "v_mfma_f32_16x16x32_bf16 v[120:123], v[240:243], v[200:203], v[120:123]\n"
    "v_mfma_f32_16x16x32_bf16 v[124:127], v[212:215], v[204:207], v[124:127]\n"
    "v_mfma_f32_16x16x32_bf16 v[124:127], v[216:219], v[208:211], v[124:127]\n"
    "v_mfma_f32_16x16x32_bf16 v[128:131], v[220:223], v[204:207], v[128:131]\n"
    "v_mfma_f32_16x16x32_bf16 v[128:131], v[224:227], v[208:211], v[128:131]\n"
    "v_mfma_f32_16x16x32_bf16 v[132:135], v[228:231], v[204:207], v[132:135]\n"
    "v_mfma_f32_16x16x32_bf16 v[132:135], v[232:235], v[208:211], v[132:135]\n"
    "v_mfma_f32_16x16x32_bf16 v[136:139], v[236:239], v[204:207], v[136:139]\n"
    "v_mfma_f32_16x16x32_bf16 v[136:139], v[240:243], v[208:211], v[136:139]\n"
    "s_setprio 0\n"
    "s_barrier\n"
    "; phase 36: step 24 slot 0 stage 0\n"
    "s_bitcmp1_b32 s55, 8\n"
    "s_cbranch_scc1 .La49_%=\n"
    "s_sleep 2\n"
    "s_branch .La50_%=\n"
    ".La49_%=:\n"
    "s_sleep 4\n"
    ".La50_%=:\n"
    "ds_read_b128 v[180:183], v244 offset:0\n"
    "ds_read_b128 v[184:187], v245 offset:0\n"
    "ds_read_b128 v[188:191], v244 offset:2048\n"
    "ds_read_b128 v[192:195], v245 offset:2048\n"
    "s_add_u32 m0, s51, 0xc000\n"
    "s_xor_b32 s82, s65, 0x1900000\n"
    "buffer_load_dwordx4 v251, s[40:43], s82 offen  lds\n"
    "ds_read_b128 v[196:199], v244 offset:8192\n"
    "ds_read_b128 v[200:203], v245 offset:8192\n"
    "ds_read_b128 v[204:207], v244 offset:10240\n"
    "ds_read_b128 v[208:211], v245 offset:10240\n"
    "s_add_u32 m0, s51, 0xe000\n"
    "s_xor_b32 s82, s65, 0x1980000\n"
    "buffer_load_dwordx4 v251, s[40:43], s82 offen  lds\n"
    "ds_read_b64_tr_b16 v[212:213], v246 offset:0\n"
    "ds_read_b64_tr_b16 v[214:215], v246 offset:1024\n"
    "ds_read_b64_tr_b16 v[216:217], v246 offset:8192\n"
    "ds_read_b64_tr_b16 v[218:219], v246 offset:9216\n"
    "s_cmp_lg_u32 s52, 0\n"
    "s_cbranch_scc1 .La51_%=\n"
    "s_xor_b32 s82, s66, 0x2c00000\n"
    "s_add_u32 s82, s82, s53\n"
    "buffer_load_dword v9, v8, s[40:43], s82 offen \n"
    ".La51_%=:\n"
    "ds_read_b64_tr_b16 v[220:221], v247 offset:0\n"
    "ds_read_b64_tr_b16 v[222:223], v247 offset:1024\n"
    "ds_read_b64_tr_b16 v[224:225], v247 offset:8192\n"
    "ds_read_b64_tr_b16 v[226:227], v247 offset:9216\n"
    "ds_read_b64_tr_b16 v[228:229], v248 offset:0\n"
    "ds_read_b64_tr_b16 v[230:231], v248 offset:1024\n"
    "ds_read_b64_tr_b16 v[232:233], v248 offset:8192\n"
    "ds_read_b64_tr_b16 v[234:235], v248 offset:9216\n"
    "ds_read_b64_tr_b16 v[236:237], v249 offset:0\n"
    "ds_read_b64_tr_b16 v[238:239], v249 offset:1024\n"
    "ds_read_b64_tr_b16 v[240:241], v249 offset:8192\n"
    "ds_read_b64_tr_b16 v[242:243], v249 offset:9216\n"
    "s_waitcnt vmcnt(8) lgkmcnt(0)\n"
    "s_barrier\n"
    "s_setprio 1\n"
    "v_mfma_f32_16x16x32_bf16 v[12:15], v[212:215], v[180:183], v[12:15]\n"
    "v_mfma_f32_16x16x32_bf16 v[12:15], v[216:219], v[184:187], v[12:15]\n"
    "v_mfma_f32_16x16x32_bf16 v[16:19], v[220:223], v[180:183], v[16:19]\n"
    "v_mfma_f32_16x16x32_bf16 v[16:19], v[224:227], v[184:187], v[16:19]\n"
    "v_mfma_f32_16x16x32_bf16 v[20:23], v[228:231], v[180:183], v[20:23]\n"
    "v_mfma_f32_16x16x32_bf16 v[20:23], v[232:235], v[184:187], v[20:23]\n"
    "v_mfma_f32_16x16x32_bf16 v[24:27], v[236:239], v[180:183], v[24:27]\n"
    "v_mfma_f32_16x16x32_bf16 v[24:27], v[240:243], v[184:187], v[24:27]\n"
    "v_mfma_f32_16x16x32_bf16 v[28:31], v[212:215], v[188:191], v[28:31]\n"
    "v_mfma_f32_16x16x32_bf16 v[28:31], v[216:219], v[192:195], v[28:31]\n"
    "v_mfma_f32_16x16x32_bf16 v[32:35], v[220:223], v[188:191], v[32:35]\n"
    "v_mfma_f32_16x16x32_bf16 v[32:35], v[224:227], v[192:195], v[32:35]\n"
    "v_mfma_f32_16x16x32_bf16 v[36:39], v[228:231], v[188:191], v[36:39]\n"
    "v_mfma_f32_16x16x32_bf16 v[36:39], v[232:235], v[192:195], v[36:39]\n"
    "v_mfma_f32_16x16x32_bf16 v[40:43], v[236:239], v[188:191], v[40:43]\n"
    "v_mfma_f32_16x16x32_bf16 v[40:43], v[240:243], v[192:195], v[40:43]\n"
    "v_mfma_f32_16x16x32_bf16 v[44:47], v[212:215], v[196:199], v[44:47]\n"
    "v_mfma_f32_16x16x32_bf16 v[44:47], v[216:219], v[200:203], v[44:47]\n"
    "v_mfma_f32_16x16x32_bf16 v[48:51], v[220:223], v[196:199], v[48:51]\n"
    "v_mfma_f32_16x16x32_bf16 v[48:51], v[224:227], v[200:203], v[48:51]\n"
    "v_mfma_f32_16x16x32_bf16 v[52:55], v[228:231], v[196:199], v[52:55]\n"
    "v_mfma_f32_16x16x32_bf16 v[52:55], v[232:235], v[200:203], v[52:55]\n"
    "v_mfma_f32_16x16x32_bf16 v[56:59], v[236:239], v[196:199], v[56:59]\n"
    "v_mfma_f32_16x16x32_bf16 v[56:59], v[240:243], v[200:203], v[56:59]\n"
    "v_mfma_f32_16x16x32_bf16 v[60:63], v[212:215], v[204:207], v[60:63]\n"
    "v_mfma_f32_16x16x32_bf16 v[60:63], v[216:219], v[208:211], v[60:63]\n"
    "v_mfma_f32_16x16x32_bf16 v[64:67], v[220:223], v[204:207], v[64:67]\n"
    "v_mfma_f32_16x16x32_bf16 v[64:67], v[224:227], v[208:211], v[64:67]\n"
    "v_mfma_f32_16x16x32_bf16 v[68:71], v[228:231], v[204:207], v[68:71]\n"
    "v_mfma_f32_16x16x32_bf16 v[68:71], v[232:235], v[208:211], v[68:71]\n"
    "v_mfma_f32_16x16x32_bf16 v[72:75], v[236:239], v[204:207], v[72:75]\n"
    "v_mfma_f32_16x16x32_bf16 v[72:75], v[240:243], v[208:211], v[72:75]\n"
    "s_setprio 0\n"
    "s_barrier\n"
    "; phase 37: step 24 slot 1 stage 1\n"
    "s_bitcmp1_b32 s55, 8\n"
    "s_cbranch_scc1 .La52_%=\n"
    "s_sleep 2\n"
    "s_branch .La53_%=\n"
    ".La52_%=:\n"
    "s_sleep 4\n"
    ".La53_%=:\n"
    "ds_read_b64_tr_b16 v[212:213], v246 offset:16384\n"
    "ds_read_b64_tr_b16 v[214:215], v246 offset:17408\n"
    "s_add_u32 m0, s51, 0x0\n"
    "s_xor_b32 s82, s64, 0x1a00000\n"
    "buffer_load_dwordx4 v251, s[40:43], s82 offen  lds\n"
    "ds_read_b64_tr_b16 v[216:217], v246 offset:24576\n"
    "ds_read_b64_tr_b16 v[218:219], v246 offset:25600\n"
    "s_add_u32 m0, s51, 0x2000\n"
    "s_xor_b32 s82, s64, 0x1a80000\n"
    "buffer_load_dwordx4 v251, s[40:43], s82 offen  lds\n"
    "ds_read_b64_tr_b16 v[220:221], v247 offset:16384\n"
    "ds_read_b64_tr_b16 v[222:223], v247 offset:17408\n"
    "s_add_u32 m0, s50, 0x0\n"
    "s_xor_b32 s82, s48, 0x200d80\n"
    "buffer_load_dwordx4 v250, s[36:39], s82 offen  lds\n"
    "ds_read_b64_tr_b16 v[224:225], v247 offset:24576\n"
    "ds_read_b64_tr_b16 v[226:227], v247 offset:25600\n"
    "s_add_u32 m0, s50, 0x1000\n"
    "s_xor_b32 s82, s48, 0x280d80\n"
    "buffer_load_dwordx4 v250, s[36:39], s82 offen  lds\n"
    "ds_read_b64_tr_b16 v[228:229], v248 offset:16384\n"
    "ds_read_b64_tr_b16 v[230:231], v248 offset:17408\n"
    "s_add_u32 m0, s50, 0x2000\n"
    "s_xor_b32 s82, s48, 0xd80\n"
    "buffer_load_dwordx4 v250, s[36:39], s82 offen  lds\n"
    "ds_read_b64_tr_b16 v[232:233], v248 offset:24576\n"
    "ds_read_b64_tr_b16 v[234:235], v248 offset:25600\n"
    "s_add_u32 m0, s50, 0x3000\n"
    "s_xor_b32 s82, s48, 0x80d80\n"
    "buffer_load_dwordx4 v250, s[36:39], s82 offen  lds\n"
    "ds_read_b64_tr_b16 v[236:237], v249 offset:16384\n"
    "ds_read_b64_tr_b16 v[238:239], v249 offset:17408\n"
    "s_cmp_lg_u32 s52, 1\n"
    "s_cbranch_scc1 .La54_%=\n"
    "s_xor_b32 s82, s66, 0x2d00000\n"
    "s_add_u32 s82, s82, s53\n"
    "buffer_load_dword v9, v8, s[40:43], s82 offen \n"
    ".La54_%=:\n"
    "ds_read_b64_tr_b16 v[240:241], v249 offset:24576\n"
    "ds_read_b64_tr_b16 v[242:243], v249 offset:25600\n"
    "s_waitcnt vmcnt(12) lgkmcnt(0)\n"
    "s_barrier\n"
    "s_setprio 1\n"
    "v_mfma_f32_16x16x32_bf16 v[76:79], v[212:215], v[180:183], v[76:79]\n"
    "v_mfma_f32_16x16x32_bf16 v[76:79], v[216:219], v[184:187], v[76:79]\n"
    "v_mfma_f32_16x16x32_bf16 v[80:83], v[220:223], v[180:183], v[80:83]\n"
    "v_mfma_f32_16x16x32_bf16 v[80:83], v[224:227], v[184:187], v[80:83]\n"
    "v_mfma_f32_16x16x32_bf16 v[84:87], v[228:231], v[180:183], v[84:87]\n"
    "v_mfma_f32_16x16x32_bf16 v[84:87], v[232:235], v[184:187], v[84:87]\n"
    "v_mfma_f32_16x16x32_bf16 v[88:91], v[236:239], v[180:183], v[88:91]\n"
    "v_mfma_f32_16x16x32_bf16 v[88:91], v[240:243], v[184:187], v[88:91]\n"
    "v_mfma_f32_16x16x32_bf16 v[92:95], v[212:215], v[188:191], v[92:95]\n"
    "v_mfma_f32_16x16x32_bf16 v[92:95], v[216:219], v[192:195], v[92:95]\n"
    "v_mfma_f32_16x16x32_bf16 v[96:99], v[220:223], v[188:191], v[96:99]\n"
    "v_mfma_f32_16x16x32_bf16 v[96:99], v[224:227], v[192:195], v[96:99]\n"
    "v_mfma_f32_16x16x32_bf16 v[100:103], v[228:231], v[188:191], v[100:103]\n"
    "v_mfma_f32_16x16x32_bf16 v[100:103], v[232:235], v[192:195], v[100:103]\n"
    "v_mfma_f32_16x16x32_bf16 v[104:107], v[236:239], v[188:191], v[104:107]\n"
    "v_mfma_f32_16x16x32_bf16 v[104:107], v[240:243], v[192:195], v[104:107]\n"
    "v_mfma_f32_16x16x32_bf16 v[108:111], v[212:215], v[196:199], v[108:111]\n"
    "v_mfma_f32_16x16x32_bf16 v[108:111], v[216:219], v[200:203], v[108:111]\n"
    "v_mfma_f32_16x16x32_bf16 v[112:115], v[220:223], v[196:199], v[112:115]\n"
    "v_mfma_f32_16x16x32_bf16 v[112:115], v[224:227], v[200:203], v[112:115]\n"
    "v_mfma_f32_16x16x32_bf16 v[116:119], v[228:231], v[196:199], v[116:119]\n"
    "v_mfma_f32_16x16x32_bf16 v[116:119], v[232:235], v[200:203], v[116:119]\n"
    "v_mfma_f32_16x16x32_bf16 v[120:123], v[236:239], v[196:199], v[120:123]\n"
    "v_mfma_f32_16x16x32_bf16 v[120:123], v[240:243], v[200:203], v[120:123]\n"
    "v_mfma_f32_16x16x32_bf16 v[124:127], v[212:215], v[204:207], v[124:127]\n"
    "v_mfma_f32_16x16x32_bf16 v[124:127], v[216:219], v[208:211], v[124:127]\n"
    "v_mfma_f32_16x16x32_bf16 v[128:131], v[220:223], v[204:207], v[128:131]\n"
    "v_mfma_f32_16x16x32_bf16 v[128:131], v[224:227], v[208:211], v[128:131]\n"
    "v_mfma_f32_16x16x32_bf16 v[132:135], v[228:231], v[204:207], v[132:135]\n"
    "v_mfma_f32_16x16x32_bf16 v[132:135], v[232:235], v[208:211], v[132:135]\n"
    "v_mfma_f32_16x16x32_bf16 v[136:139], v[236:239], v[204:207], v[136:139]\n"
    "v_mfma_f32_16x16x32_bf16 v[136:139], v[240:243], v[208:211], v[136:139]\n"
    "s_setprio 0\n"
    "s_barrier\n"
    "; phase 38: step 25 slot 0 stage 0\n"
    "s_bitcmp1_b32 s55, 8\n"
    "s_cbranch_scc1 .La55_%=\n"
    "s_sleep 2\n"
    "s_branch .La56_%=\n"
    ".La55_%=:\n"
    "s_sleep 4\n"
    ".La56_%=:\n"
    "ds_read_b128 v[180:183], v244 offset:16384\n"
    "ds_read_b128 v[184:187], v245 offset:16384\n"
    "ds_read_b128 v[188:191], v244 offset:18432\n"
    "ds_read_b128 v[192:195], v245 offset:18432\n"
    "s_add_u32 m0, s51, 0x4000\n"
    "s_xor_b32 s82, s65, 0x1a00000\n"
    "buffer_load_dwordx4 v251, s[40:43], s82 offen  lds\n"
    "ds_read_b128 v[196:199], v244 offset:24576\n"
    "ds_read_b128 v[200:203], v245 offset:24576\n"
    "ds_read_b128 v[204:207], v244 offset:26624\n"
    "ds_read_b128 v[208:211], v245 offset:26624\n"
    "s_add_u32 m0, s51, 0x6000\n"
    "s_xor_b32 s82, s65, 0x1a80000\n"
    "buffer_load_dwordx4 v251, s[40:43], s82 offen  lds\n"
    "ds_read_b64_tr_b16 v[212:213], v246 offset:32768\n"
    "ds_read_b64_tr_b16 v[214:215], v246 offset:33792\n"
    "ds_read_b64_tr_b16 v[216:217], v246 offset:40960\n"
    "ds_read_b64_tr_b16 v[218:219], v246 offset:41984\n"
    "s_cmp_lg_u32 s52, 2\n"
    "s_cbranch_scc1 .La57_%=\n"
    "s_xor_b32 s82, s66, 0x2e00000\n"
    "s_add_u32 s82, s82, s53\n"
    "buffer_load_dword v9, v8, s[40:43], s82 offen \n"
    ".La57_%=:\n"
    "ds_read_b64_tr_b16 v[220:221], v247 offset:32768\n"
    "ds_read_b64_tr_b16 v[222:223], v247 offset:33792\n"
    "ds_read_b64_tr_b16 v[224:225], v247 offset:40960\n"
    "ds_read_b64_tr_b16 v[226:227], v247 offset:41984\n"
    "ds_read_b64_tr_b16 v[228:229], v248 offset:32768\n"
    "ds_read_b64_tr_b16 v[230:231], v248 offset:33792\n"
    "ds_read_b64_tr_b16 v[232:233], v248 offset:40960\n"
    "ds_read_b64_tr_b16 v[234:235], v248 offset:41984\n"
    "ds_read_b64_tr_b16 v[236:237], v249 offset:32768\n"
    "ds_read_b64_tr_b16 v[238:239], v249 offset:33792\n"
    "ds_read_b64_tr_b16 v[240:241], v249 offset:40960\n"
    "ds_read_b64_tr_b16 v[242:243], v249 offset:41984\n"
    "s_waitcnt vmcnt(8) lgkmcnt(0)\n"
    "s_barrier\n"
    "s_setprio 1\n"
    "v_mfma_f32_16x16x32_bf16 v[12:15], v[212:215], v[180:183], v[12:15]\n"
    "v_mfma_f32_16x16x32_bf16 v[12:15], v[216:219], v[184:187], v[12:15]\n"
    "v_mfma_f32_16x16x32_bf16 v[16:19], v[220:223], v[180:183], v[16:19]\n"
    "v_mfma_f32_16x16x32_bf16 v[16:19], v[224:227], v[184:187], v[16:19]\n"
    "v_mfma_f32_16x16x32_bf16 v[20:23], v[228:231], v[180:183], v[20:23]\n"
    "v_mfma_f32_16x16x32_bf16 v[20:23], v[232:235], v[184:187], v[20:23]\n"
    "v_mfma_f32_16x16x32_bf16 v[24:27], v[236:239], v[180:183], v[24:27]\n"
    "v_mfma_f32_16x16x32_bf16 v[24:27], v[240:243], v[184:187], v[24:27]\n"
    "v_mfma_f32_16x16x32_bf16 v[28:31], v[212:215], v[188:191], v[28:31]\n"
    "v_mfma_f32_16x16x32_bf16 v[28:31], v[216:219], v[192:195], v[28:31]\n"
    "v_mfma_f32_16x16x32_bf16 v[32:35], v[220:223], v[188:191], v[32:35]\n"
    "v_mfma_f32_16x16x32_bf16 v[32:35], v[224:227], v[192:195], v[32:35]\n"
    "v_mfma_f32_16x16x32_bf16 v[36:39], v[228:231], v[188:191], v[36:39]\n"
    "v_mfma_f32_16x16x32_bf16 v[36:39], v[232:235], v[192:195], v[36:39]\n"
    "v_mfma_f32_16x16x32_bf16 v[40:43], v[236:239], v[188:191], v[40:43]\n"
    "v_mfma_f32_16x16x32_bf16 v[40:43], v[240:243], v[192:195], v[40:43]\n"
    "v_mfma_f32_16x16x32_bf16 v[44:47], v[212:215], v[196:199], v[44:47]\n"
    "v_mfma_f32_16x16x32_bf16 v[44:47], v[216:219], v[200:203], v[44:47]\n"
    "v_mfma_f32_16x16x32_bf16 v[48:51], v[220:223], v[196:199], v[48:51]\n"
    "v_mfma_f32_16x16x32_bf16 v[48:51], v[224:227], v[200:203], v[48:51]\n"
    "v_mfma_f32_16x16x32_bf16 v[52:55], v[228:231], v[196:199], v[52:55]\n"
    "v_mfma_f32_16x16x32_bf16 v[52:55], v[232:235], v[200:203], v[52:55]\n"
    "v_mfma_f32_16x16x32_bf16 v[56:59], v[236:239], v[196:199], v[56:59]\n"
    "v_mfma_f32_16x16x32_bf16 v[56:59], v[240:243], v[200:203], v[56:59]\n"
    "v_mfma_f32_16x16x32_bf16 v[60:63], v[212:215], v[204:207], v[60:63]\n"
    "v_mfma_f32_16x16x32_bf16 v[60:63], v[216:219], v[208:211], v[60:63]\n"
    "v_mfma_f32_16x16x32_bf16 v[64:67], v[220:223], v[204:207], v[64:67]\n"
    "v_mfma_f32_16x16x32_bf16 v[64:67], v[224:227], v[208:211], v[64:67]\n"
    "v_mfma_f32_16x16x32_bf16 v[68:71], v[228:231], v[204:207], v[68:71]\n"
    "v_mfma_f32_16x16x32_bf16 v[68:71], v[232:235], v[208:211], v[68:71]\n"
    "v_mfma_f32_16x16x32_bf16 v[72:75], v[236:239], v[204:207], v[72:75]\n"
    "v_mfma_f32_16x16x32_bf16 v[72:75], v[240:243], v[208:211], v[72:75]\n"
    "s_setprio 0\n"
    "s_barrier\n"
    "; phase 39: step 25 slot 1 stage 1\n"
    "s_bitcmp1_b32 s55, 8\n"
    "s_cbranch_scc1 .La58_%=\n"
    "s_sleep 2\n"
    "s_branch .La59_%=\n"
    ".La58_%=:\n"
    "s_sleep 4\n"
    ".La59_%=:\n"
    "ds_read_b64_tr_b16 v[212:213], v246 offset:49152\n"
    "ds_read_b64_tr_b16 v[214:215], v246 offset:50176\n"
    "s_add_u32 m0, s51, 0x8000\n"
    "s_xor_b32 s82, s64, 0x1b00000\n"
    "buffer_load_dwordx4 v251, s[40:43], s82 offen  lds\n"
    "ds_read_b64_tr_b16 v[216:217], v246 offset:57344\n"
    "ds_read_b64_tr_b16 v[218:219], v246 offset:58368\n"
    "s_add_u32 m0, s51, 0xa000\n"
    "s_xor_b32 s82, s64, 0x1b80000\n"
    "buffer_load_dwordx4 v251, s[40:43], s82 offen  lds\n"
    "ds_read_b64_tr_b16 v[220:221], v247 offset:49152\n"
    "ds_read_b64_tr_b16 v[222:223], v247 offset:50176\n"
    "s_add_u32 m0, s50, 0x4000\n"
    "s_xor_b32 s82, s48, 0x200e00\n"
    "buffer_load_dwordx4 v250, s[36:39], s82 offen  lds\n"
    "ds_read_b64_tr_b16 v[224:225], v247 offset:57344\n"
    "ds_read_b64_tr_b16 v[226:227], v247 offset:58368\n"
    "s_add_u32 m0, s50, 0x5000\n"
    "s_xor_b32 s82, s48, 0x280e00\n"
    "buffer_load_dwordx4 v250, s[36:39], s82 offen  lds\n"
    "ds_read_b64_tr_b16 v[228:229], v248 offset:49152\n"
    "ds_read_b64_tr_b16 v[230:231], v248 offset:50176\n"
    "s_add_u32 m0, s50, 0x6000\n"
    "s_xor_b32 s82, s48, 0xe00\n"
    "buffer_load_dwordx4 v250, s[36:39], s82 offen  lds\n"
    "ds_read_b64_tr_b16 v[232:233], v248 offset:57344\n"
    "ds_read_b64_tr_b16 v[234:235], v248 offset:58368\n"
    "s_add_u32 m0, s50, 0x7000\n"
    "s_xor_b32 s82, s48, 0x80e00\n"
    "buffer_load_dwordx4 v250, s[36:39], s82 offen  lds\n"
    "ds_read_b64_tr_b16 v[236:237], v249 offset:49152\n"
    "ds_read_b64_tr_b16 v[238:239], v249 offset:50176\n"
    "s_cmp_lg_u32 s52, 3\n"
    "s_cbranch_scc1 .La60_%=\n"
    "s_xor_b32 s82, s66, 0x2f00000\n"
    "s_add_u32 s82, s82, s53\n"
    "buffer_load_dword v9, v8, s[40:43], s82 offen \n"
    ".La60_%=:\n"
    "ds_read_b64_tr_b16 v[240:241], v249 offset:57344\n"
    "ds_read_b64_tr_b16 v[242:243], v249 offset:58368\n"
    "s_waitcnt vmcnt(12) lgkmcnt(0)\n"
    "s_barrier\n"
    "s_setprio 1\n"
    "v_mfma_f32_16x16x32_bf16 v[76:79], v[212:215], v[180:183], v[76:79]\n"
    "v_mfma_f32_16x16x32_bf16 v[76:79], v[216:219], v[184:187], v[76:79]\n"
    "v_mfma_f32_16x16x32_bf16 v[80:83], v[220:223], v[180:183], v[80:83]\n"
    "v_mfma_f32_16x16x32_bf16 v[80:83], v[224:227], v[184:187], v[80:83]\n"
    "v_mfma_f32_16x16x32_bf16 v[84:87], v[228:231], v[180:183], v[84:87]\n"
    "v_mfma_f32_16x16x32_bf16 v[84:87], v[232:235], v[184:187], v[84:87]\n"
    "v_mfma_f32_16x16x32_bf16 v[88:91], v[236:239], v[180:183], v[88:91]\n"
    "v_mfma_f32_16x16x32_bf16 v[88:91], v[240:243], v[184:187], v[88:91]\n"
    "v_mfma_f32_16x16x32_bf16 v[92:95], v[212:215], v[188:191], v[92:95]\n"
    "v_mfma_f32_16x16x32_bf16 v[92:95], v[216:219], v[192:195], v[92:95]\n"
    "v_mfma_f32_16x16x32_bf16 v[96:99], v[220:223], v[188:191], v[96:99]\n"
    "v_mfma_f32_16x16x32_bf16 v[96:99], v[224:227], v[192:195], v[96:99]\n"
    "v_mfma_f32_16x16x32_bf16 v[100:103], v[228:231], v[188:191], v[100:103]\n"
    "v_mfma_f32_16x16x32_bf16 v[100:103], v[232:235], v[192:195], v[100:103]\n"
    "v_mfma_f32_16x16x32_bf16 v[104:107], v[236:239], v[188:191], v[104:107]\n"
    "v_mfma_f32_16x16x32_bf16 v[104:107], v[240:243], v[192:195], v[104:107]\n"
    "v_mfma_f32_16x16x32_bf16 v[108:111], v[212:215], v[196:199], v[108:111]\n"
    "v_mfma_f32_16x16x32_bf16 v[108:111], v[216:219], v[200:203], v[108:111]\n"
    "v_mfma_f32_16x16x32_bf16 v[112:115], v[220:223], v[196:199], v[112:115]\n"
    "v_mfma_f32_16x16x32_bf16 v[112:115], v[224:227], v[200:203], v[112:115]\n"
    "v_mfma_f32_16x16x32_bf16 v[116:119], v[228:231], v[196:199], v[116:119]\n"
    "v_mfma_f32_16x16x32_bf16 v[116:119], v[232:235], v[200:203], v[116:119]\n"
    "v_mfma_f32_16x16x32_bf16 v[120:123], v[236:239], v[196:199], v[120:123]\n"
    "v_mfma_f32_16x16x32_bf16 v[120:123], v[240:243], v[200:203], v[120:123]\n"
    "v_mfma_f32_16x16x32_bf16 v[124:127], v[212:215], v[204:207], v[124:127]\n"
    "v_mfma_f32_16x16x32_bf16 v[124:127], v[216:219], v[208:211], v[124:127]\n"
    "v_mfma_f32_16x16x32_bf16 v[128:131], v[220:223], v[204:207], v[128:131]\n"
    "v_mfma_f32_16x16x32_bf16 v[128:131], v[224:227], v[208:211], v[128:131]\n"
    "v_mfma_f32_16x16x32_bf16 v[132:135], v[228:231], v[204:207], v[132:135]\n"
    "v_mfma_f32_16x16x32_bf16 v[132:135], v[232:235], v[208:211], v[132:135]\n"
    "v_mfma_f32_16x16x32_bf16 v[136:139], v[236:239], v[204:207], v[136:139]\n"
    "v_mfma_f32_16x16x32_bf16 v[136:139], v[240:243], v[208:211], v[136:139]\n"
    "s_setprio 0\n"
    "s_barrier\n"
    "; phase 40: step 26 slot 0 stage 0\n"
    "s_bitcmp1_b32 s55, 8\n"
    "s_cbranch_scc1 .La61_%=\n"
    "s_sleep 2\n"
    "s_branch .La62_%=\n"
    ".La61_%=:\n"
    "s_sleep 4\n"
    ".La62_%=:\n"
    "ds_read_b128 v[180:183], v244 offset:32768\n"
    "ds_read_b128 v[184:187], v245 offset:32768\n"
    "ds_read_b128 v[188:191], v244 offset:34816\n"
    "ds_read_b128 v[192:195], v245 offset:34816\n"
    "s_add_u32 m0, s51, 0xc000\n"
    "s_xor_b32 s82, s65, 0x1b00000\n"
    "buffer_load_dwordx4 v251, s[40:43], s82 offen  lds\n"
    "ds_read_b128 v[196:199], v244 offset:40960\n"
    "ds_read_b128 v[200:203], v245 offset:40960\n"
    "ds_read_b128 v[204:207], v244 offset:43008\n"
    "ds_read_b128 v[208:211], v245 offset:43008\n"
    "s_add_u32 m0, s51, 0xe000\n"
    "s_xor_b32 s82, s65, 0x1b80000\n"
    "buffer_load_dwordx4 v251, s[40:43], s82 offen  lds\n"
    "ds_read_b64_tr_b16 v[212:213], v246 offset:0\n"
    "ds_read_b64_tr_b16 v[214:215], v246 offset:1024\n"
    "ds_read_b64_tr_b16 v[216:217], v246 offset:8192\n"
    "ds_read_b64_tr_b16 v[218:219], v246 offset:9216\n"
    "s_cmp_lg_u32 s52, 4\n"
    "s_cbranch_scc1 .La63_%=\n"
    "s_xor_b32 s82, s66, 0x3000000\n"
    "s_add_u32 s82, s82, s53\n"
    "buffer_load_dword v9, v8, s[40:43], s82 offen \n"
    ".La63_%=:\n"
    "ds_read_b64_tr_b16 v[220:221], v247 offset:0\n"
    "ds_read_b64_tr_b16 v[222:223], v247 offset:1024\n"
    "ds_read_b64_tr_b16 v[224:225], v247 offset:8192\n"
    "ds_read_b64_tr_b16 v[226:227], v247 offset:9216\n"
    "ds_read_b64_tr_b16 v[228:229], v248 offset:0\n"
    "ds_read_b64_tr_b16 v[230:231], v248 offset:1024\n"
    "ds_read_b64_tr_b16 v[232:233], v248 offset:8192\n"
    "ds_read_b64_tr_b16 v[234:235], v248 offset:9216\n"
    "ds_read_b64_tr_b16 v[236:237], v249 offset:0\n"
    "ds_read_b64_tr_b16 v[238:239], v249 offset:1024\n"
    "ds_read_b64_tr_b16 v[240:241], v249 offset:8192\n"
    "ds_read_b64_tr_b16 v[242:243], v249 offset:9216\n"
    "s_waitcnt vmcnt(8) lgkmcnt(0)\n"
    "s_barrier\n"
    "s_setprio 1\n"
    "v_mfma_f32_16x16x32_bf16 v[12:15], v[212:215], v[180:183], v[12:15]\n"
    "v_mfma_f32_16x16x32_bf16 v[12:15], v[216:219], v[184:187], v[12:15]\n"
    "v_mfma_f32_16x16x32_bf16 v[16:19], v[220:223], v[180:183], v[16:19]\n"
    "v_mfma_f32_16x16x32_bf16 v[16:19], v[224:227], v[184:187], v[16:19]\n"
    "v_mfma_f32_16x16x32_bf16 v[20:23], v[228:231], v[180:183], v[20:23]\n"
    "v_mfma_f32_16x16x32_bf16 v[20:23], v[232:235], v[184:187], v[20:23]\n"
    "v_mfma_f32_16x16x32_bf16 v[24:27], v[236:239], v[180:183], v[24:27]\n"
    "v_mfma_f32_16x16x32_bf16 v[24:27], v[240:243], v[184:187], v[24:27]\n"
    "v_mfma_f32_16x16x32_bf16 v[28:31], v[212:215], v[188:191], v[28:31]\n"
    "v_mfma_f32_16x16x32_bf16 v[28:31], v[216:219], v[192:195], v[28:31]\n"
    "v_mfma_f32_16x16x32_bf16 v[32:35], v[220:223], v[188:191], v[32:35]\n"
    "v_mfma_f32_16x16x32_bf16 v[32:35], v[224:227], v[192:195], v[32:35]\n"
    "v_mfma_f32_16x16x32_bf16 v[36:39], v[228:231], v[188:191], v[36:39]\n"
    "v_mfma_f32_16x16x32_bf16 v[36:39], v[232:235], v[192:195], v[36:39]\n"
    "v_mfma_f32_16x16x32_bf16 v[40:43], v[236:239], v[188:191], v[40:43]\n"
    "v_mfma_f32_16x16x32_bf16 v[40:43], v[240:243], v[192:195], v[40:43]\n"
    "v_mfma_f32_16x16x32_bf16 v[44:47], v[212:215], v[196:199], v[44:47]\n"
    "v_mfma_f32_16x16x32_bf16 v[44:47], v[216:219], v[200:203], v[44:47]\n"
    "v_mfma_f32_16x16x32_bf16 v[48:51], v[220:223], v[196:199], v[48:51]\n"
    "v_mfma_f32_16x16x32_bf16 v[48:51], v[224:227], v[200:203], v[48:51]\n"
    "v_mfma_f32_16x16x32_bf16 v[52:55], v[228:231], v[196:199], v[52:55]\n"
    "v_mfma_f32_16x16x32_bf16 v[52:55], v[232:235], v[200:203], v[52:55]\n"
    "v_mfma_f32_16x16x32_bf16 v[56:59], v[236:239], v[196:199], v[56:59]\n"
    "v_mfma_f32_16x16x32_bf16 v[56:59], v[240:243], v[200:203], v[56:59]\n"
    "v_mfma_f32_16x16x32_bf16 v[60:63], v[212:215], v[204:207], v[60:63]\n"
    "v_mfma_f32_16x16x32_bf16 v[60:63], v[216:219], v[208:211], v[60:63]\n"
    "v_mfma_f32_16x16x32_bf16 v[64:67], v[220:223], v[204:207], v[64:67]\n"
    "v_mfma_f32_16x16x32_bf16 v[64:67], v[224:227], v[208:211], v[64:67]\n"
    "v_mfma_f32_16x16x32_bf16 v[68:71], v[228:231], v[204:207], v[68:71]\n"
    "v_mfma_f32_16x16x32_bf16 v[68:71], v[232:235], v[208:211], v[68:71]\n"
    "v_mfma_f32_16x16x32_bf16 v[72:75], v[236:239], v[204:207], v[72:75]\n"
    "v_mfma_f32_16x16x32_bf16 v[72:75], v[240:243], v[208:211], v[72:75]\n"
    "s_setprio 0\n"
    "s_barrier\n"
    "; phase 41: step 26 slot 1 stage 1\n"
    "s_bitcmp1_b32 s55, 8\n"
    "s_cbranch_scc1 .La64_%=\n"
    "s_sleep 2\n"
    "s_branch .La65_%=\n"
    ".La64_%=:\n"
    "s_sleep 4\n"
    ".La65_%=:\n"
    "ds_read_b64_tr_b16 v[212:213], v246 offset:16384\n"
    "ds_read_b64_tr_b16 v[214:215], v246 offset:17408\n"
    "s_add_u32 m0, s51, 0x0\n"
    "s_xor_b32 s82, s64, 0x1c00000\n"
    "buffer_load_dwordx4 v251, s[40:43], s82 offen  lds\n"
    "ds_read_b64_tr_b16 v[216:217], v246 offset:24576\n"
    "ds_read_b64_tr_b16 v[218:219], v246 offset:25600\n"
    "s_add_u32 m0, s51, 0x2000\n"
    "s_xor_b32 s82, s64, 0x1c80000\n"
    "buffer_load_dwordx4 v251, s[40:43], s82 offen  lds\n"
    "ds_read_b64_tr_b16 v[220:221], v247 offset:16384\n"
    "ds_read_b64_tr_b16 v[222:223], v247 offset:17408\n"
    "s_add_u32 m0, s50, 0x8000\n"
    "s_xor_b32 s82, s48, 0x200e80\n"
    "buffer_load_dwordx4 v250, s[36:39], s82 offen  lds\n"
    "ds_read_b64_tr_b16 v[224:225], v247 offset:24576\n"
    "ds_read_b64_tr_b16 v[226:227], v247 offset:25600\n"
    "s_add_u32 m0, s50, 0x9000\n"
    "s_xor_b32 s82, s48, 0x280e80\n"
    "buffer_load_dwordx4 v250, s[36:39], s82 offen  lds\n"
    "ds_read_b64_tr_b16 v[228:229], v248 offset:16384\n"
    "ds_read_b64_tr_b16 v[230:231], v248 offset:17408\n"
    "s_add_u32 m0, s50, 0xa000\n"
    "s_xor_b32 s82, s48, 0xe80\n"
    "buffer_load_dwordx4 v250, s[36:39], s82 offen  lds\n"
    "ds_read_b64_tr_b16 v[232:233], v248 offset:24576\n"
    "ds_read_b64_tr_b16 v[234:235], v248 offset:25600\n"
    "s_add_u32 m0, s50, 0xb000\n"
    "s_xor_b32 s82, s48, 0x80e80\n"
    "buffer_load_dwordx4 v250, s[36:39], s82 offen  lds\n"
    "ds_read_b64_tr_b16 v[236:237], v249 offset:16384\n"
    "ds_read_b64_tr_b16 v[238:239], v249 offset:17408\n"
    "s_cmp_lg_u32 s52, 5\n"
    "s_cbranch_scc1 .La66_%=\n"
    "s_xor_b32 s82, s66, 0x3100000\n"
    "s_add_u32 s82, s82, s53\n"
    "buffer_load_dword v9, v8, s[40:43], s82 offen \n"
    ".La66_%=:\n"
    "ds_read_b64_tr_b16 v[240:241], v249 offset:24576\n"
    "ds_read_b64_tr_b16 v[242:243], v249 offset:25600\n"
    "s_waitcnt vmcnt(12) lgkmcnt(0)\n"
    "s_barrier\n"
    "s_setprio 1\n"
    "v_mfma_f32_16x16x32_bf16 v[76:79], v[212:215], v[180:183], v[76:79]\n"
    "v_mfma_f32_16x16x32_bf16 v[76:79], v[216:219], v[184:187], v[76:79]\n"
    "v_mfma_f32_16x16x32_bf16 v[80:83], v[220:223], v[180:183], v[80:83]\n"
    "v_mfma_f32_16x16x32_bf16 v[80:83], v[224:227], v[184:187], v[80:83]\n"
    "v_mfma_f32_16x16x32_bf16 v[84:87], v[228:231], v[180:183], v[84:87]\n"
    "v_mfma_f32_16x16x32_bf16 v[84:87], v[232:235], v[184:187], v[84:87]\n"
    "v_mfma_f32_16x16x32_bf16 v[88:91], v[236:239], v[180:183], v[88:91]\n"
    "v_mfma_f32_16x16x32_bf16 v[88:91], v[240:243], v[184:187], v[88:91]\n"
    "v_mfma_f32_16x16x32_bf16 v[92:95], v[212:215], v[188:191], v[92:95]\n"
    "v_mfma_f32_16x16x32_bf16 v[92:95], v[216:219], v[192:195], v[92:95]\n"
    "v_mfma_f32_16x16x32_bf16 v[96:99], v[220:223], v[188:191], v[96:99]\n"
    "v_mfma_f32_16x16x32_bf16 v[96:99], v[224:227], v[192:195], v[96:99]\n"
    "v_mfma_f32_16x16x32_bf16 v[100:103], v[228:231], v[188:191], v[100:103]\n"
    "v_mfma_f32_16x16x32_bf16 v[100:103], v[232:235], v[192:195], v[100:103]\n"
    "v_mfma_f32_16x16x32_bf16 v[104:107], v[236:239], v[188:191], v[104:107]\n"
    "v_mfma_f32_16x16x32_bf16 v[104:107], v[240:243], v[192:195], v[104:107]\n"
    "v_mfma_f32_16x16x32_bf16 v[108:111], v[212:215], v[196:199], v[108:111]\n"
    "v_mfma_f32_16x16x32_bf16 v[108:111], v[216:219], v[200:203], v[108:111]\n"
    "v_mfma_f32_16x16x32_bf16 v[112:115], v[220:223], v[196:199], v[112:115]\n"
    "v_mfma_f32_16x16x32_bf16 v[112:115], v[224:227], v[200:203], v[112:115]\n"
    "v_mfma_f32_16x16x32_bf16 v[116:119], v[228:231], v[196:199], v[116:119]\n"
    "v_mfma_f32_16x16x32_bf16 v[116:119], v[232:235], v[200:203], v[116:119]\n"
    "v_mfma_f32_16x16x32_bf16 v[120:123], v[236:239], v[196:199], v[120:123]\n"
    "v_mfma_f32_16x16x32_bf16 v[120:123], v[240:243], v[200:203], v[120:123]\n"
    "v_mfma_f32_16x16x32_bf16 v[124:127], v[212:215], v[204:207], v[124:127]\n"
    "v_mfma_f32_16x16x32_bf16 v[124:127], v[216:219], v[208:211], v[124:127]\n"
    "v_mfma_f32_16x16x32_bf16 v[128:131], v[220:223], v[204:207], v[128:131]\n"
    "v_mfma_f32_16x16x32_bf16 v[128:131], v[224:227], v[208:211], v[128:131]\n"
    "v_mfma_f32_16x16x32_bf16 v[132:135], v[228:231], v[204:207], v[132:135]\n"
    "v_mfma_f32_16x16x32_bf16 v[132:135], v[232:235], v[208:211], v[132:135]\n"
    "v_mfma_f32_16x16x32_bf16 v[136:139], v[236:239], v[204:207], v[136:139]\n"
    "v_mfma_f32_16x16x32_bf16 v[136:139], v[240:243], v[208:211], v[136:139]\n"
    "s_setprio 0\n"
    "s_barrier\n"
    "; phase 42: step 27 slot 0 stage 0\n"
    "s_bitcmp1_b32 s55, 8\n"
    "s_cbranch_scc1 .La67_%=\n"
    "s_sleep 2\n"
    "s_branch .La68_%=\n"
    ".La67_%=:\n"
    "s_sleep 4\n"
    ".La68_%=:\n"
    "ds_read_b128 v[180:183], v244 offset:0\n"
    "ds_read_b128 v[184:187], v245 offset:0\n"
    "ds_read_b128 v[188:191], v244 offset:2048\n"
    "ds_read_b128 v[192:195], v245 offset:2048\n"
    "s_add_u32 m0, s51, 0x4000\n"
    "s_xor_b32 s82, s65, 0x1c00000\n"
    "buffer_load_dwordx4 v251, s[40:43], s82 offen  lds\n"
    "ds_read_b128 v[196:199], v244 offset:8192\n"
    "ds_read_b128 v[200:203], v245 offset:8192\n"
    "ds_read_b128 v[204:207], v244 offset:10240\n"
    "ds_read_b128 v[208:211], v245 offset:10240\n"
    "s_add_u32 m0, s51, 0x6000\n"
    "s_xor_b32 s82, s65, 0x1c80000\n"
    "buffer_load_dwordx4 v251, s[40:43], s82 offen  lds\n"
    "ds_read_b64_tr_b16 v[212:213], v246 offset:32768\n"
    "ds_read_b64_tr_b16 v[214:215], v246 offset:33792\n"
    "ds_read_b64_tr_b16 v[216:217], v246 offset:40960\n"
    "ds_read_b64_tr_b16 v[218:219], v246 offset:41984\n"
    "ds_read_b64_tr_b16 v[220:221], v247 offset:32768\n"
    "ds_read_b64_tr_b16 v[222:223], v247 offset:33792\n"
    "ds_read_b64_tr_b16 v[224:225], v247 offset:40960\n"
    "ds_read_b64_tr_b16 v[226:227], v247 offset:41984\n"
    "ds_read_b64_tr_b16 v[228:229], v248 offset:32768\n"
    "ds_read_b64_tr_b16 v[230:231], v248 offset:33792\n"
    "ds_read_b64_tr_b16 v[232:233], v248 offset:40960\n"
    "ds_read_b64_tr_b16 v[234:235], v248 offset:41984\n"
    "ds_read_b64_tr_b16 v[236:237], v249 offset:32768\n"
    "ds_read_b64_tr_b16 v[238:239], v249 offset:33792\n"
    "ds_read_b64_tr_b16 v[240:241], v249 offset:40960\n"
    "ds_read_b64_tr_b16 v[242:243], v249 offset:41984\n"
    "s_waitcnt vmcnt(8) lgkmcnt(0)\n"
    "s_barrier\n"
    "s_setprio 1\n"
    "v_mfma_f32_16x16x32_bf16 v[12:15], v[212:215], v[180:183], v[12:15]\n"
    "v_mfma_f32_16x16x32_bf16 v[12:15], v[216:219], v[184:187], v[12:15]\n"
    "v_mfma_f32_16x16x32_bf16 v[16:19], v[220:223], v[180:183], v[16:19]\n"
    "v_mfma_f32_16x16x32_bf16 v[16:19], v[224:227], v[184:187], v[16:19]\n"
    "v_mfma_f32_16x16x32_bf16 v[20:23], v[228:231], v[180:183], v[20:23]\n"
    "v_mfma_f32_16x16x32_bf16 v[20:23], v[232:235], v[184:187], v[20:23]\n"
    "v_mfma_f32_16x16x32_bf16 v[24:27], v[236:239], v[180:183], v[24:27]\n"
    "v_mfma_f32_16x16x32_bf16 v[24:27], v[240:243], v[184:187], v[24:27]\n"
    "v_mfma_f32_16x16x32_bf16 v[28:31], v[212:215], v[188:191], v[28:31]\n"
    "v_mfma_f32_16x16x32_bf16 v[28:31], v[216:219], v[192:195], v[28:31]\n"
    "v_mfma_f32_16x16x32_bf16 v[32:35], v[220:223], v[188:191], v[32:35]\n"
    "v_mfma_f32_16x16x32_bf16 v[32:35], v[224:227], v[192:195], v[32:35]\n"
    "v_mfma_f32_16x16x32_bf16 v[36:39], v[228:231], v[188:191], v[36:39]\n"
    "v_mfma_f32_16x16x32_bf16 v[36:39], v[232:235], v[192:195], v[36:39]\n"
    "v_mfma_f32_16x16x32_bf16 v[40:43], v[236:239], v[188:191], v[40:43]\n"
    "v_mfma_f32_16x16x32_bf16 v[40:43], v[240:243], v[192:195], v[40:43]\n"
    "v_mfma_f32_16x16x32_bf16 v[44:47], v[212:215], v[196:199], v[44:47]\n"
    "v_mfma_f32_16x16x32_bf16 v[44:47], v[216:219], v[200:203], v[44:47]\n"
    "v_mfma_f32_16x16x32_bf16 v[48:51], v[220:223], v[196:199], v[48:51]\n"
    "v_mfma_f32_16x16x32_bf16 v[48:51], v[224:227], v[200:203], v[48:51]\n"
    "v_mfma_f32_16x16x32_bf16 v[52:55], v[228:231], v[196:199], v[52:55]\n"
    "v_mfma_f32_16x16x32_bf16 v[52:55], v[232:235], v[200:203], v[52:55]\n"
    "v_mfma_f32_16x16x32_bf16 v[56:59], v[236:239], v[196:199], v[56:59]\n"
    "v_mfma_f32_16x16x32_bf16 v[56:59], v[240:243], v[200:203], v[56:59]\n"
    "v_mfma_f32_16x16x32_bf16 v[60:63], v[212:215], v[204:207], v[60:63]\n"
    "v_mfma_f32_16x16x32_bf16 v[60:63], v[216:219], v[208:211], v[60:63]\n"
    "v_mfma_f32_16x16x32_bf16 v[64:67], v[220:223], v[204:207], v[64:67]\n"
    "v_mfma_f32_16x16x32_bf16 v[64:67], v[224:227], v[208:211], v[64:67]\n"
    "v_mfma_f32_16x16x32_bf16 v[68:71], v[228:231], v[204:207], v[68:71]\n"
    "v_mfma_f32_16x16x32_bf16 v[68:71], v[232:235], v[208:211], v[68:71]\n"
    "v_mfma_f32_16x16x32_bf16 v[72:75], v[236:239], v[204:207], v[72:75]\n"
    "v_mfma_f32_16x16x32_bf16 v[72:75], v[240:243], v[208:211], v[72:75]\n"
    "s_setprio 0\n"
    "s_barrier\n"
    "; phase 43: step 27 slot 1 stage 1\n"
    "s_bitcmp1_b32 s55, 8\n"
    "s_cbranch_scc1 .La69_%=\n"
    "s_sleep 2\n"
    "s_branch .La70_%=\n"
    ".La69_%=:\n"
    "s_sleep 4\n"
    ".La70_%=:\n"
    "ds_read_b64_tr_b16 v[212:213], v246 offset:49152\n"
    "ds_read_b64_tr_b16 v[214:215], v246 offset:50176\n"
    "s_add_u32 m0, s51, 0x8000\n"
    "s_xor_b32 s82, s64, 0x1d00000\n"
    "buffer_load_dwordx4 v251, s[40:43], s82 offen  lds\n"
    "ds_read_b64_tr_b16 v[216:217], v246 offset:57344\n"
    "ds_read_b64_tr_b16 v[218:219], v246 offset:58368\n"
    "s_add_u32 m0, s51, 0xa000\n"
    "s_xor_b32 s82, s64, 0x1d80000\n"
    "buffer_load_dwordx4 v251, s[40:43], s82 offen  lds\n"
    "ds_read_b64_tr_b16 v[220:221], v247 offset:49152\n"
    "ds_read_b64_tr_b16 v[222:223], v247 offset:50176\n"
    "s_add_u32 m0, s50, 0x0\n"
    "s_xor_b32 s82, s48, 0x200f00\n"
    "buffer_load_dwordx4 v250, s[36:39], s82 offen  lds\n"
    "ds_read_b64_tr_b16 v[224:225], v247 offset:57344\n"
    "ds_read_b64_tr_b16 v[226:227], v247 offset:58368\n"
    "s_add_u32 m0, s50, 0x1000\n"
    "s_xor_b32 s82, s48, 0x280f00\n"
    "buffer_load_dwordx4 v250, s[36:39], s82 offen  lds\n"
    "ds_read_b64_tr_b16 v[228:229], v248 offset:49152\n"
    "ds_read_b64_tr_b16 v[230:231], v248 offset:50176\n"
    "s_add_u32 m0, s50, 0x2000\n"
    "s_xor_b32 s82, s48, 0xf00\n"
    "buffer_load_dwordx4 v250, s[36:39], s82 offen  lds\n"
    "ds_read_b64_tr_b16 v[232:233], v248 offset:57344\n"
    "ds_read_b64_tr_b16 v[234:235], v248 offset:58368\n"
    "s_add_u32 m0, s50, 0x3000\n"
    "s_xor_b32 s82, s48, 0x80f00\n"
    "buffer_load_dwordx4 v250, s[36:39], s82 offen  lds\n"
    "ds_read_b64_tr_b16 v[236:237], v249 offset:49152\n"
    "ds_read_b64_tr_b16 v[238:239], v249 offset:50176\n"
    "s_cmp_lg_u32 s52, 6\n"
    "s_cbranch_scc1 .La71_%=\n"
    "s_xor_b32 s82, s66, 0x3200000\n"
    "s_add_u32 s82, s82, s53\n"
    "buffer_load_dword v9, v8, s[40:43], s82 offen \n"
    ".La71_%=:\n"
    "ds_read_b64_tr_b16 v[240:241], v249 offset:57344\n"
    "ds_read_b64_tr_b16 v[242:243], v249 offset:58368\n"
    "s_waitcnt vmcnt(12) lgkmcnt(0)\n"
    "s_barrier\n"
    "s_setprio 1\n"
    "v_mfma_f32_16x16x32_bf16 v[76:79], v[212:215], v[180:183], v[76:79]\n"
    "v_mfma_f32_16x16x32_bf16 v[76:79], v[216:219], v[184:187], v[76:79]\n"
    "v_mfma_f32_16x16x32_bf16 v[80:83], v[220:223], v[180:183], v[80:83]\n"
    "v_mfma_f32_16x16x32_bf16 v[80:83], v[224:227], v[184:187], v[80:83]\n"
    "v_mfma_f32_16x16x32_bf16 v[84:87], v[228:231], v[180:183], v[84:87]\n"
    "v_mfma_f32_16x16x32_bf16 v[84:87], v[232:235], v[184:187], v[84:87]\n"
    "v_mfma_f32_16x16x32_bf16 v[88:91], v[236:239], v[180:183], v[88:91]\n"
    "v_mfma_f32_16x16x32_bf16 v[88:91], v[240:243], v[184:187], v[88:91]\n"
    "v_mfma_f32_16x16x32_bf16 v[92:95], v[212:215], v[188:191], v[92:95]\n"
    "v_mfma_f32_16x16x32_bf16 v[92:95], v[216:219], v[192:195], v[92:95]\n"
    "v_mfma_f32_16x16x32_bf16 v[96:99], v[220:223], v[188:191], v[96:99]\n"
    "v_mfma_f32_16x16x32_bf16 v[96:99], v[224:227], v[192:195], v[96:99]\n"
    "v_mfma_f32_16x16x32_bf16 v[100:103], v[228:231], v[188:191], v[100:103]\n"
    "v_mfma_f32_16x16x32_bf16 v[100:103], v[232:235], v[192:195], v[100:103]\n"
    "v_mfma_f32_16x16x32_bf16 v[104:107], v[236:239], v[188:191], v[104:107]\n"
    "v_mfma_f32_16x16x32_bf16 v[104:107], v[240:243], v[192:195], v[104:107]\n"
    "v_mfma_f32_16x16x32_bf16 v[108:111], v[212:215], v[196:199], v[108:111]\n"
    "v_mfma_f32_16x16x32_bf16 v[108:111], v[216:219], v[200:203], v[108:111]\n"
    "v_mfma_f32_16x16x32_bf16 v[112:115], v[220:223], v[196:199], v[112:115]\n"
    "v_mfma_f32_16x16x32_bf16 v[112:115], v[224:227], v[200:203], v[112:115]\n"
    "v_mfma_f32_16x16x32_bf16 v[116:119], v[228:231], v[196:199], v[116:119]\n"
    "v_mfma_f32_16x16x32_bf16 v[116:119], v[232:235], v[200:203], v[116:119]\n"
    "v_mfma_f32_16x16x32_bf16 v[120:123], v[236:239], v[196:199], v[120:123]\n"
    "v_mfma_f32_16x16x32_bf16 v[120:123], v[240:243], v[200:203], v[120:123]\n"
    "v_mfma_f32_16x16x32_bf16 v[124:127], v[212:215], v[204:207], v[124:127]\n"
    "v_mfma_f32_16x16x32_bf16 v[124:127], v[216:219], v[208:211], v[124:127]\n"
    "v_mfma_f32_16x16x32_bf16 v[128:131], v[220:223], v[204:207], v[128:131]\n"
    "v_mfma_f32_16x16x32_bf16 v[128:131], v[224:227], v[208:211], v[128:131]\n"
    "v_mfma_f32_16x16x32_bf16 v[132:135], v[228:231], v[204:207], v[132:135]\n"
    "v_mfma_f32_16x16x32_bf16 v[132:135], v[232:235], v[208:211], v[132:135]\n"
    "v_mfma_f32_16x16x32_bf16 v[136:139], v[236:239], v[204:207], v[136:139]\n"
    "v_mfma_f32_16x16x32_bf16 v[136:139], v[240:243], v[208:211], v[136:139]\n"
    "s_setprio 0\n"
    "s_barrier\n"
    "; phase 44: step 28 slot 0 stage 0\n"
    "s_bitcmp1_b32 s55, 8\n"
    "s_cbranch_scc1 .La72_%=\n"
    "s_sleep 2\n"
    "s_branch .La73_%=\n"
    ".La72_%=:\n"
    "s_sleep 4\n"
    ".La73_%=:\n"
    "ds_read_b128 v[180:183], v244 offset:16384\n"
    "ds_read_b128 v[184:187], v245 offset:16384\n"
    "ds_read_b128 v[188:191], v244 offset:18432\n"
    "ds_read_b128 v[192:195], v245 offset:18432\n"
    "s_add_u32 m0, s51, 0xc000\n"
    "s_xor_b32 s82, s65, 0x1d00000\n"
    "buffer_load_dwordx4 v251, s[40:43], s82 offen  lds\n"
    "ds_read_b128 v[196:199], v244 offset:24576\n"
    "ds_read_b128 v[200:203], v245 offset:24576\n"
    "ds_read_b128 v[204:207], v244 offset:26624\n"
    "ds_read_b128 v[208:211], v245 offset:26624\n"
    "s_add_u32 m0, s51, 0xe000\n"
    "s_xor_b32 s82, s65, 0x1d80000\n"
    "buffer_load_dwordx4 v251, s[40:43], s82 offen  lds\n"
    "ds_read_b64_tr_b16 v[212:213], v246 offset:0\n"
    "ds_read_b64_tr_b16 v[214:215], v246 offset:1024\n"
    "ds_read_b64_tr_b16 v[216:217], v246 offset:8192\n"
    "ds_read_b64_tr_b16 v[218:219], v246 offset:9216\n"
    "s_cmp_lg_u32 s52, 7\n"
    "s_cbranch_scc1 .La74_%=\n"
    "s_xor_b32 s82, s66, 0x3300000\n"
    "s_add_u32 s82, s82, s53\n"
    "buffer_load_dword v9, v8, s[40:43], s82 offen \n"
    ".La74_%=:\n"
    "ds_read_b64_tr_b16 v[220:221], v247 offset:0\n"
    "ds_read_b64_tr_b16 v[222:223], v247 offset:1024\n"
    "ds_read_b64_tr_b16 v[224:225], v247 offset:8192\n"
    "ds_read_b64_tr_b16 v[226:227], v247 offset:9216\n"
    "ds_read_b64_tr_b16 v[228:229], v248 offset:0\n"
    "ds_read_b64_tr_b16 v[230:231], v248 offset:1024\n"
    "ds_read_b64_tr_b16 v[232:233], v248 offset:8192\n"
    "ds_read_b64_tr_b16 v[234:235], v248 offset:9216\n"
    "ds_read_b64_tr_b16 v[236:237], v249 offset:0\n"
    "ds_read_b64_tr_b16 v[238:239], v249 offset:1024\n"
    "ds_read_b64_tr_b16 v[240:241], v249 offset:8192\n"
    "ds_read_b64_tr_b16 v[242:243], v249 offset:9216\n"
    "s_waitcnt vmcnt(8) lgkmcnt(0)\n"
    "s_barrier\n"
    "s_setprio 1\n"
    "v_mfma_f32_16x16x32_bf16 v[12:15], v[212:215], v[180:183], v[12:15]\n"
    "v_mfma_f32_16x16x32_bf16 v[12:15], v[216:219], v[184:187], v[12:15]\n"
    "v_mfma_f32_16x16x32_bf16 v[16:19], v[220:223], v[180:183], v[16:19]\n"
    "v_mfma_f32_16x16x32_bf16 v[16:19], v[224:227], v[184:187], v[16:19]\n"
    "v_mfma_f32_16x16x32_bf16 v[20:23], v[228:231], v[180:183], v[20:23]\n"
    "v_mfma_f32_16x16x32_bf16 v[20:23], v[232:235], v[184:187], v[20:23]\n"
    "v_mfma_f32_16x16x32_bf16 v[24:27], v[236:239], v[180:183], v[24:27]\n"
    "v_mfma_f32_16x16x32_bf16 v[24:27], v[240:243], v[184:187], v[24:27]\n"
    "v_mfma_f32_16x16x32_bf16 v[28:31], v[212:215], v[188:191], v[28:31]\n"
    "v_mfma_f32_16x16x32_bf16 v[28:31], v[216:219], v[192:195], v[28:31]\n"
    "v_mfma_f32_16x16x32_bf16 v[32:35], v[220:223], v[188:191], v[32:35]\n"
    "v_mfma_f32_16x16x32_bf16 v[32:35], v[224:227], v[192:195], v[32:35]\n"
    "v_mfma_f32_16x16x32_bf16 v[36:39], v[228:231], v[188:191], v[36:39]\n"
    "v_mfma_f32_16x16x32_bf16 v[36:39], v[232:235], v[192:195], v[36:39]\n"
    "v_mfma_f32_16x16x32_bf16 v[40:43], v[236:239], v[188:191], v[40:43]\n"
    "v_mfma_f32_16x16x32_bf16 v[40:43], v[240:243], v[192:195], v[40:43]\n"
    "v_mfma_f32_16x16x32_bf16 v[44:47], v[212:215], v[196:199], v[44:47]\n"
    "v_mfma_f32_16x16x32_bf16 v[44:47], v[216:219], v[200:203], v[44:47]\n"
    "v_mfma_f32_16x16x32_bf16 v[48:51], v[220:223], v[196:199], v[48:51]\n"
    "v_mfma_f32_16x16x32_bf16 v[48:51], v[224:227], v[200:203], v[48:51]\n"
    "v_mfma_f32_16x16x32_bf16 v[52:55], v[228:231], v[196:199], v[52:55]\n"
    "v_mfma_f32_16x16x32_bf16 v[52:55], v[232:235], v[200:203], v[52:55]\n"
    "v_mfma_f32_16x16x32_bf16 v[56:59], v[236:239], v[196:199], v[56:59]\n"
    "v_mfma_f32_16x16x32_bf16 v[56:59], v[240:243], v[200:203], v[56:59]\n"
    "v_mfma_f32_16x16x32_bf16 v[60:63], v[212:215], v[204:207], v[60:63]\n"
    "v_mfma_f32_16x16x32_bf16 v[60:63], v[216:219], v[208:211], v[60:63]\n"
    "v_mfma_f32_16x16x32_bf16 v[64:67], v[220:223], v[204:207], v[64:67]\n"
    "v_mfma_f32_16x16x32_bf16 v[64:67], v[224:227], v[208:211], v[64:67]\n"
    "v_mfma_f32_16x16x32_bf16 v[68:71], v[228:231], v[204:207], v[68:71]\n"
    "v_mfma_f32_16x16x32_bf16 v[68:71], v[232:235], v[208:211], v[68:71]\n"
    "v_mfma_f32_16x16x32_bf16 v[72:75], v[236:239], v[204:207], v[72:75]\n"
    "v_mfma_f32_16x16x32_bf16 v[72:75], v[240:243], v[208:211], v[72:75]\n"
    "s_setprio 0\n"
    "s_barrier\n"
    "; phase 45: step 28 slot 1 stage 1\n"
    "s_bitcmp1_b32 s55, 8\n"
    "s_cbranch_scc1 .La75_%=\n"
    "s_sleep 2\n"
    "s_branch .La76_%=\n"
    ".La75_%=:\n"
    "s_sleep 4\n"
    ".La76_%=:\n"
    "ds_read_b64_tr_b16 v[212:213], v246 offset:16384\n"
    "ds_read_b64_tr_b16 v[214:215], v246 offset:17408\n"
    "s_add_u32 m0, s51, 0x0\n"
    "s_xor_b32 s82, s64, 0x1e00000\n"
    "buffer_load_dwordx4 v251, s[40:43], s82 offen  lds\n"
    "ds_read_b64_tr_b16 v[216:217], v246 offset:24576\n"
    "ds_read_b64_tr_b16 v[218:219], v246 offset:25600\n"
    "s_add_u32 m0, s51, 0x2000\n"
    "s_xor_b32 s82, s64, 0x1e80000\n"
    "buffer_load_dwordx4 v251, s[40:43], s82 offen  lds\n"
    "ds_read_b64_tr_b16 v[220:221], v247 offset:16384\n"
    "ds_read_b64_tr_b16 v[222:223], v247 offset:17408\n"
    "s_add_u32 m0, s50, 0x4000\n"
    "s_xor_b32 s82, s48, 0x200f80\n"
    "buffer_load_dwordx4 v250, s[36:39], s82 offen  lds\n"
    "ds_read_b64_tr_b16 v[224:225], v247 offset:24576\n"
    "ds_read_b64_tr_b16 v[226:227], v247 offset:25600\n"
    "s_add_u32 m0, s50, 0x5000\n"
    "s_xor_b32 s82, s48, 0x280f80\n"
    "buffer_load_dwordx4 v250, s[36:39], s82 offen  lds\n"
    "ds_read_b64_tr_b16 v[228:229], v248 offset:16384\n"
    "ds_read_b64_tr_b16 v[230:231], v248 offset:17408\n"
    "s_add_u32 m0, s50, 0x6000\n"
    "s_xor_b32 s82, s48, 0xf80\n"
    "buffer_load_dwordx4 v250, s[36:39], s82 offen  lds\n"
    "ds_read_b64_tr_b16 v[232:233], v248 offset:24576\n"
    "ds_read_b64_tr_b16 v[234:235], v248 offset:25600\n"
    "s_add_u32 m0, s50, 0x7000\n"
    "s_xor_b32 s82, s48, 0x80f80\n"
    "buffer_load_dwordx4 v250, s[36:39], s82 offen  lds\n"
    "ds_read_b64_tr_b16 v[236:237], v249 offset:16384\n"
    "ds_read_b64_tr_b16 v[238:239], v249 offset:17408\n"
    "s_cmp_lg_u32 s52, 0\n"
    "s_cbranch_scc1 .La77_%=\n"
    "s_xor_b32 s82, s66, 0x3400000\n"
    "s_add_u32 s82, s82, s53\n"
    "buffer_load_dword v9, v8, s[40:43], s82 offen \n"
    ".La77_%=:\n"
    "ds_read_b64_tr_b16 v[240:241], v249 offset:24576\n"
    "ds_read_b64_tr_b16 v[242:243], v249 offset:25600\n"
    "s_waitcnt vmcnt(12) lgkmcnt(0)\n"
    "s_barrier\n"
    "s_setprio 1\n"
    "v_mfma_f32_16x16x32_bf16 v[76:79], v[212:215], v[180:183], v[76:79]\n"
    "v_mfma_f32_16x16x32_bf16 v[76:79], v[216:219], v[184:187], v[76:79]\n"
    "v_mfma_f32_16x16x32_bf16 v[80:83], v[220:223], v[180:183], v[80:83]\n"
    "v_mfma_f32_16x16x32_bf16 v[80:83], v[224:227], v[184:187], v[80:83]\n"
    "v_mfma_f32_16x16x32_bf16 v[84:87], v[228:231], v[180:183], v[84:87]\n"
    "v_mfma_f32_16x16x32_bf16 v[84:87], v[232:235], v[184:187], v[84:87]\n"
    "v_mfma_f32_16x16x32_bf16 v[88:91], v[236:239], v[180:183], v[88:91]\n"
    "v_mfma_f32_16x16x32_bf16 v[88:91], v[240:243], v[184:187], v[88:91]\n"
    "v_mfma_f32_16x16x32_bf16 v[92:95], v[212:215], v[188:191], v[92:95]\n"
    "v_mfma_f32_16x16x32_bf16 v[92:95], v[216:219], v[192:195], v[92:95]\n"
    "v_mfma_f32_16x16x32_bf16 v[96:99], v[220:223], v[188:191], v[96:99]\n"
    "v_mfma_f32_16x16x32_bf16 v[96:99], v[224:227], v[192:195], v[96:99]\n"
    "v_mfma_f32_16x16x32_bf16 v[100:103], v[228:231], v[188:191], v[100:103]\n"
    "v_mfma_f32_16x16x32_bf16 v[100:103], v[232:235], v[192:195], v[100:103]\n"
    "v_mfma_f32_16x16x32_bf16 v[104:107], v[236:239], v[188:191], v[104:107]\n"
    "v_mfma_f32_16x16x32_bf16 v[104:107], v[240:243], v[192:195], v[104:107]\n"
    "v_mfma_f32_16x16x32_bf16 v[108:111], v[212:215], v[196:199], v[108:111]\n"
    "v_mfma_f32_16x16x32_bf16 v[108:111], v[216:219], v[200:203], v[108:111]\n"
    "v_mfma_f32_16x16x32_bf16 v[112:115], v[220:223], v[196:199], v[112:115]\n"
    "v_mfma_f32_16x16x32_bf16 v[112:115], v[224:227], v[200:203], v[112:115]\n"
    "v_mfma_f32_16x16x32_bf16 v[116:119], v[228:231], v[196:199], v[116:119]\n"
    "v_mfma_f32_16x16x32_bf16 v[116:119], v[232:235], v[200:203], v[116:119]\n"
    "v_mfma_f32_16x16x32_bf16 v[120:123], v[236:239], v[196:199], v[120:123]\n"
    "v_mfma_f32_16x16x32_bf16 v[120:123], v[240:243], v[200:203], v[120:123]\n"
    "v_mfma_f32_16x16x32_bf16 v[124:127], v[212:215], v[204:207], v[124:127]\n"
    "v_mfma_f32_16x16x32_bf16 v[124:127], v[216:219], v[208:211], v[124:127]\n"
    "v_mfma_f32_16x16x32_bf16 v[128:131], v[220:223], v[204:207], v[128:131]\n"
    "v_mfma_f32_16x16x32_bf16 v[128:131], v[224:227], v[208:211], v[128:131]\n"
    "v_mfma_f32_16x16x32_bf16 v[132:135], v[228:231], v[204:207], v[132:135]\n"
    "v_mfma_f32_16x16x32_bf16 v[132:135], v[232:235], v[208:211], v[132:135]\n"
    "v_mfma_f32_16x16x32_bf16 v[136:139], v[236:239], v[204:207], v[136:139]\n"
    "v_mfma_f32_16x16x32_bf16 v[136:139], v[240:243], v[208:211], v[136:139]\n"
    "s_setprio 0\n"
    "s_barrier\n"
    "; phase 46: step 29 slot 0 stage 0\n"
    "s_bitcmp1_b32 s55, 8\n"
    "s_cbranch_scc1 .La78_%=\n"
    "s_sleep 2\n"
    "s_branch .La79_%=\n"
    ".La78_%=:\n"
    "s_sleep 4\n"
    ".La79_%=:\n"
    "ds_read_b128 v[180:183], v244 offset:32768\n"
    "ds_read_b128 v[184:187], v245 offset:32768\n"
    "ds_read_b128 v[188:191], v244 offset:34816\n"
    "ds_read_b128 v[192:195], v245 offset:34816\n"
    "s_add_u32 m0, s51, 0x4000\n"
    "s_xor_b32 s82, s65, 0x1e00000\n"
    "buffer_load_dwordx4 v251, s[40:43], s82 offen  lds\n"
    "ds_read_b128 v[196:199], v244 offset:40960\n"
    "ds_read_b128 v[200:203], v245 offset:40960\n"
    "ds_read_b128 v[204:207], v244 offset:43008\n"
    "ds_read_b128 v[208:211], v245 offset:43008\n"
    "s_add_u32 m0, s51, 0x6000\n"
    "s_xor_b32 s82, s65, 0x1e80000\n"
    "buffer_load_dwordx4 v251, s[40:43], s82 offen  lds\n"
    "ds_read_b64_tr_b16 v[212:213], v246 offset:32768\n"
    "ds_read_b64_tr_b16 v[214:215], v246 offset:33792\n"
    "ds_read_b64_tr_b16 v[216:217], v246 offset:40960\n"
    "ds_read_b64_tr_b16 v[218:219], v246 offset:41984\n"
    "s_cmp_lg_u32 s52, 1\n"
    "s_cbranch_scc1 .La80_%=\n"
    "s_xor_b32 s82, s66, 0x3500000\n"
    "s_add_u32 s82, s82, s53\n"
    "buffer_load_dword v9, v8, s[40:43], s82 offen \n"
    ".La80_%=:\n"
    "ds_read_b64_tr_b16 v[220:221], v247 offset:32768\n"
    "ds_read_b64_tr_b16 v[222:223], v247 offset:33792\n"
    "ds_read_b64_tr_b16 v[224:225], v247 offset:40960\n"
    "ds_read_b64_tr_b16 v[226:227], v247 offset:41984\n"
    "ds_read_b64_tr_b16 v[228:229], v248 offset:32768\n"
    "ds_read_b64_tr_b16 v[230:231], v248 offset:33792\n"
    "ds_read_b64_tr_b16 v[232:233], v248 offset:40960\n"
    "ds_read_b64_tr_b16 v[234:235], v248 offset:41984\n"
    "ds_read_b64_tr_b16 v[236:237], v249 offset:32768\n"
    "ds_read_b64_tr_b16 v[238:239], v249 offset:33792\n"
    "ds_read_b64_tr_b16 v[240:241], v249 offset:40960\n"
    "ds_read_b64_tr_b16 v[242:243], v249 offset:41984\n"
    "s_waitcnt vmcnt(8) lgkmcnt(0)\n"
    "s_barrier\n"
    "s_setprio 1\n"
    "v_mfma_f32_16x16x32_bf16 v[12:15], v[212:215], v[180:183], v[12:15]\n"
    "v_mfma_f32_16x16x32_bf16 v[12:15], v[216:219], v[184:187], v[12:15]\n"
    "v_mfma_f32_16x16x32_bf16 v[16:19], v[220:223], v[180:183], v[16:19]\n"
    "v_mfma_f32_16x16x32_bf16 v[16:19], v[224:227], v[184:187], v[16:19]\n"
    "v_mfma_f32_16x16x32_bf16 v[20:23], v[228:231], v[180:183], v[20:23]\n"
    "v_mfma_f32_16x16x32_bf16 v[20:23], v[232:235], v[184:187], v[20:23]\n"
    "v_mfma_f32_16x16x32_bf16 v[24:27], v[236:239], v[180:183], v[24:27]\n"
    "v_mfma_f32_16x16x32_bf16 v[24:27], v[240:243], v[184:187], v[24:27]\n"
    "v_mfma_f32_16x16x32_bf16 v[28:31], v[212:215], v[188:191], v[28:31]\n"
    "v_mfma_f32_16x16x32_bf16 v[28:31], v[216:219], v[192:195], v[28:31]\n"
    "v_mfma_f32_16x16x32_bf16 v[32:35], v[220:223], v[188:191], v[32:35]\n"
    "v_mfma_f32_16x16x32_bf16 v[32:35], v[224:227], v[192:195], v[32:35]\n"
    "v_mfma_f32_16x16x32_bf16 v[36:39], v[228:231], v[188:191], v[36:39]\n"
    "v_mfma_f32_16x16x32_bf16 v[36:39], v[232:235], v[192:195], v[36:39]\n"
    "v_mfma_f32_16x16x32_bf16 v[40:43], v[236:239], v[188:191], v[40:43]\n"
    "v_mfma_f32_16x16x32_bf16 v[40:43], v[240:243], v[192:195], v[40:43]\n"
    "v_mfma_f32_16x16x32_bf16 v[44:47], v[212:215], v[196:199], v[44:47]\n"
    "v_mfma_f32_16x16x32_bf16 v[44:47], v[216:219], v[200:203], v[44:47]\n"
    "v_mfma_f32_16x16x32_bf16 v[48:51], v[220:223], v[196:199], v[48:51]\n"
    "v_mfma_f32_16x16x32_bf16 v[48:51], v[224:227], v[200:203], v[48:51]\n"
    "v_mfma_f32_16x16x32_bf16 v[52:55], v[228:231], v[196:199], v[52:55]\n"
    "v_mfma_f32_16x16x32_bf16 v[52:55], v[232:235], v[200:203], v[52:55]\n"
    "v_mfma_f32_16x16x32_bf16 v[56:59], v[236:239], v[196:199], v[56:59]\n"
    "v_mfma_f32_16x16x32_bf16 v[56:59], v[240:243], v[200:203], v[56:59]\n"
    "v_mfma_f32_16x16x32_bf16 v[60:63], v[212:215], v[204:207], v[60:63]\n"
    "v_mfma_f32_16x16x32_bf16 v[60:63], v[216:219], v[208:211], v[60:63]\n"
    "v_mfma_f32_16x16x32_bf16 v[64:67], v[220:223], v[204:207], v[64:67]\n"
    "v_mfma_f32_16x16x32_bf16 v[64:67], v[224:227], v[208:211], v[64:67]\n"
    "v_mfma_f32_16x16x32_bf16 v[68:71], v[228:231], v[204:207], v[68:71]\n"
    "v_mfma_f32_16x16x32_bf16 v[68:71], v[232:235], v[208:211], v[68:71]\n"
    "v_mfma_f32_16x16x32_bf16 v[72:75], v[236:239], v[204:207], v[72:75]\n"
    "v_mfma_f32_16x16x32_bf16 v[72:75], v[240:243], v[208:211], v[72:75]\n"
    "s_setprio 0\n"
    "s_barrier\n"
    "; phase 47: step 29 slot 1 stage 1\n"
    "ds_read_b64_tr_b16 v[212:213], v246 offset:49152\n"
    "ds_read_b64_tr_b16 v[214:215], v246 offset:50176\n"
    "s_add_u32 m0, s51, 0x8000\n"
    "s_xor_b32 s82, s64, 0x1f00000\n"
    "buffer_load_dwordx4 v251, s[40:43], s82 offen  lds\n"
    "ds_read_b64_tr_b16 v[216:217], v246 offset:57344\n"
    "ds_read_b64_tr_b16 v[218:219], v246 offset:58368\n"
    "s_add_u32 m0, s51, 0xa000\n"
    "s_xor_b32 s82, s64, 0x1f80000\n"
    "buffer_load_dwordx4 v251, s[40:43], s82 offen  lds\n"
    "ds_read_b64_tr_b16 v[220:221], v247 offset:49152\n"
    "ds_read_b64_tr_b16 v[222:223], v247 offset:50176\n"
    "s_add_u32 m0, s50, 0x8000\n"
    "s_xor_b32 s82, s48, 0x201000\n"
    "buffer_load_dwordx4 v250, s[36:39], s82 offen  lds\n"
    "ds_read_b64_tr_b16 v[224:225], v247 offset:57344\n"
    "ds_read_b64_tr_b16 v[226:227], v247 offset:58368\n"
    "s_add_u32 m0, s50, 0x9000\n"
    "s_xor_b32 s82, s48, 0x281000\n"
    "buffer_load_dwordx4 v250, s[36:39], s82 offen  lds\n"
    "ds_read_b64_tr_b16 v[228:229], v248 offset:49152\n"
    "ds_read_b64_tr_b16 v[230:231], v248 offset:50176\n"
    "s_add_u32 m0, s50, 0xa000\n"
    "s_xor_b32 s82, s48, 0x1000\n"
    "buffer_load_dwordx4 v250, s[36:39], s82 offen  lds\n"
    "ds_read_b64_tr_b16 v[232:233], v248 offset:57344\n"
    "ds_read_b64_tr_b16 v[234:235], v248 offset:58368\n"
    "s_add_u32 m0, s50, 0xb000\n"
    "s_xor_b32 s82, s48, 0x81000\n"
    "buffer_load_dwordx4 v250, s[36:39], s82 offen  lds\n"
    "ds_read_b64_tr_b16 v[236:237], v249 offset:49152\n"
    "ds_read_b64_tr_b16 v[238:239], v249 offset:50176\n"
    "s_cmp_lg_u32 s52, 2\n"
    "s_cbranch_scc1 .La81_%=\n"
    "s_xor_b32 s82, s66, 0x3600000\n"
    "s_add_u32 s82, s82, s53\n"
    "buffer_load_dword v9, v8, s[40:43], s82 offen \n"
    ".La81_%=:\n"
    "ds_read_b64_tr_b16 v[240:241], v249 offset:57344\n"
    "ds_read_b64_tr_b16 v[242:243], v249 offset:58368\n"
    "s_waitcnt vmcnt(12) lgkmcnt(0)\n"
    "s_barrier\n"
    "s_setprio 1\n"
    "v_mfma_f32_16x16x32_bf16 v[76:79], v[212:215], v[180:183], v[76:79]\n"
    "v_mfma_f32_16x16x32_bf16 v[76:79], v[216:219], v[184:187], v[76:79]\n"
    "v_mfma_f32_16x16x32_bf16 v[80:83], v[220:223], v[180:183], v[80:83]\n"
    "v_mfma_f32_16x16x32_bf16 v[80:83], v[224:227], v[184:187], v[80:83]\n"
    "v_mfma_f32_16x16x32_bf16 v[84:87], v[228:231], v[180:183], v[84:87]\n"
    "v_mfma_f32_16x16x32_bf16 v[84:87], v[232:235], v[184:187], v[84:87]\n"
    "v_mfma_f32_16x16x32_bf16 v[88:91], v[236:239], v[180:183], v[88:91]\n"
    "v_mfma_f32_16x16x32_bf16 v[88:91], v[240:243], v[184:187], v[88:91]\n"
    "v_mfma_f32_16x16x32_bf16 v[92:95], v[212:215], v[188:191], v[92:95]\n"
    "v_mfma_f32_16x16x32_bf16 v[92:95], v[216:219], v[192:195], v[92:95]\n"
    "v_mfma_f32_16x16x32_bf16 v[96:99], v[220:223], v[188:191], v[96:99]\n"
    "v_mfma_f32_16x16x32_bf16 v[96:99], v[224:227], v[192:195], v[96:99]\n"
    "v_mfma_f32_16x16x32_bf16 v[100:103], v[228:231], v[188:191], v[100:103]\n"
    "v_mfma_f32_16x16x32_bf16 v[100:103], v[232:235], v[192:195], v[100:103]\n"
    "v_mfma_f32_16x16x32_bf16 v[104:107], v[236:239], v[188:191], v[104:107]\n"
    "v_mfma_f32_16x16x32_bf16 v[104:107], v[240:243], v[192:195], v[104:107]\n"
    "v_mfma_f32_16x16x32_bf16 v[108:111], v[212:215], v[196:199], v[108:111]\n"
    "v_mfma_f32_16x16x32_bf16 v[108:111], v[216:219], v[200:203], v[108:111]\n"
    "v_mfma_f32_16x16x32_bf16 v[112:115], v[220:223], v[196:199], v[112:115]\n"
    "v_mfma_f32_16x16x32_bf16 v[112:115], v[224:227], v[200:203], v[112:115]\n"
    "v_mfma_f32_16x16x32_bf16 v[116:119], v[228:231], v[196:199], v[116:119]\n"
    "v_mfma_f32_16x16x32_bf16 v[116:119], v[232:235], v[200:203], v[116:119]\n"
    "v_mfma_f32_16x16x32_bf16 v[120:123], v[236:239], v[196:199], v[120:123]\n"
    "v_mfma_f32_16x16x32_bf16 v[120:123], v[240:243], v[200:203], v[120:123]\n"
    "v_mfma_f32_16x16x32_bf16 v[124:127], v[212:215], v[204:207], v[124:127]\n"
    "v_mfma_f32_16x16x32_bf16 v[124:127], v[216:219], v[208:211], v[124:127]\n"
    "v_mfma_f32_16x16x32_bf16 v[128:131], v[220:223], v[204:207], v[128:131]\n"
    "v_mfma_f32_16x16x32_bf16 v[128:131], v[224:227], v[208:211], v[128:131]\n"
    "v_mfma_f32_16x16x32_bf16 v[132:135], v[228:231], v[204:207], v[132:135]\n"
    "v_mfma_f32_16x16x32_bf16 v[132:135], v[232:235], v[208:211], v[132:135]\n"
    "v_mfma_f32_16x16x32_bf16 v[136:139], v[236:239], v[204:207], v[136:139]\n"
    "v_mfma_f32_16x16x32_bf16 v[136:139], v[240:243], v[208:211], v[136:139]\n"
    "s_setprio 0\n"
    "s_barrier\n"
    "; phase 48: step 30 slot 0 stage 0\n"
    "ds_read_b128 v[180:183], v244 offset:0\n"
    "ds_read_b128 v[184:187], v245 offset:0\n"
    "ds_read_b128 v[188:191], v244 offset:2048\n"
    "ds_read_b128 v[192:195], v245 offset:2048\n"
    "s_add_u32 m0, s51, 0xc000\n"
    "s_xor_b32 s82, s65, 0x1f00000\n"
    "buffer_load_dwordx4 v251, s[40:43], s82 offen  lds\n"
    "ds_read_b128 v[196:199], v244 offset:8192\n"
    "ds_read_b128 v[200:203], v245 offset:8192\n"
    "ds_read_b128 v[204:207], v244 offset:10240\n"
    "ds_read_b128 v[208:211], v245 offset:10240\n"
    "s_add_u32 m0, s51, 0xe000\n"
    "s_xor_b32 s82, s65, 0x1f80000\n"
    "buffer_load_dwordx4 v251, s[40:43], s82 offen  lds\n"
    "ds_read_b64_tr_b16 v[212:213], v246 offset:0\n"
    "ds_read_b64_tr_b16 v[214:215], v246 offset:1024\n"
    "ds_read_b64_tr_b16 v[216:217], v246 offset:8192\n"
    "ds_read_b64_tr_b16 v[218:219], v246 offset:9216\n"
    "ds_read_b64_tr_b16 v[220:221], v247 offset:0\n"
    "ds_read_b64_tr_b16 v[222:223], v247 offset:1024\n"
    "ds_read_b64_tr_b16 v[224:225], v247 offset:8192\n"
    "ds_read_b64_tr_b16 v[226:227], v247 offset:9216\n"
    "ds_read_b64_tr_b16 v[228:229], v248 offset:0\n"
    "ds_read_b64_tr_b16 v[230:231], v248 offset:1024\n"
    "ds_read_b64_tr_b16 v[232:233], v248 offset:8192\n"
    "ds_read_b64_tr_b16 v[234:235], v248 offset:9216\n"
    "ds_read_b64_tr_b16 v[236:237], v249 offset:0\n"
    "ds_read_b64_tr_b16 v[238:239], v249 offset:1024\n"
    "ds_read_b64_tr_b16 v[240:241], v249 offset:8192\n"
    "ds_read_b64_tr_b16 v[242:243], v249 offset:9216\n"
    "s_waitcnt vmcnt(8) lgkmcnt(0)\n"
    "s_barrier\n"
    "s_setprio 1\n"
    "v_mfma_f32_16x16x32_bf16 v[12:15], v[212:215], v[180:183], v[12:15]\n"
    "v_mfma_f32_16x16x32_bf16 v[12:15], v[216:219], v[184:187], v[12:15]\n"
    "v_mfma_f32_16x16x32_bf16 v[16:19], v[220:223], v[180:183], v[16:19]\n"
    "v_mfma_f32_16x16x32_bf16 v[16:19], v[224:227], v[184:187], v[16:19]\n"
    "v_mfma_f32_16x16x32_bf16 v[20:23], v[228:231], v[180:183], v[20:23]\n"
    "v_mfma_f32_16x16x32_bf16 v[20:23], v[232:235], v[184:187], v[20:23]\n"
    "v_mfma_f32_16x16x32_bf16 v[24:27], v[236:239], v[180:183], v[24:27]\n"
    "v_mfma_f32_16x16x32_bf16 v[24:27], v[240:243], v[184:187], v[24:27]\n"
    "v_mfma_f32_16x16x32_bf16 v[28:31], v[212:215], v[188:191], v[28:31]\n"
    "v_mfma_f32_16x16x32_bf16 v[28:31], v[216:219], v[192:195], v[28:31]\n"
    "v_mfma_f32_16x16x32_bf16 v[32:35], v[220:223], v[188:191], v[32:35]\n"
    "v_mfma_f32_16x16x32_bf16 v[32:35], v[224:227], v[192:195], v[32:35]\n"
    "v_mfma_f32_16x16x32_bf16 v[36:39], v[228:231], v[188:191], v[36:39]\n"
    "v_mfma_f32_16x16x32_bf16 v[36:39], v[232:235], v[192:195], v[36:39]\n"
    "v_mfma_f32_16x16x32_bf16 v[40:43], v[236:239], v[188:191], v[40:43]\n"
    "v_mfma_f32_16x16x32_bf16 v[40:43], v[240:243], v[192:195], v[40:43]\n"
    "v_mfma_f32_16x16x32_bf16 v[44:47], v[212:215], v[196:199], v[44:47]\n"
    "v_mfma_f32_16x16x32_bf16 v[44:47], v[216:219], v[200:203], v[44:47]\n"
    "v_mfma_f32_16x16x32_bf16 v[48:51], v[220:223], v[196:199], v[48:51]\n"
    "v_mfma_f32_16x16x32_bf16 v[48:51], v[224:227], v[200:203], v[48:51]\n"
    "v_mfma_f32_16x16x32_bf16 v[52:55], v[228:231], v[196:199], v[52:55]\n"
    "v_mfma_f32_16x16x32_bf16 v[52:55], v[232:235], v[200:203], v[52:55]\n"
    "v_mfma_f32_16x16x32_bf16 v[56:59], v[236:239], v[196:199], v[56:59]\n"
    "v_mfma_f32_16x16x32_bf16 v[56:59], v[240:243], v[200:203], v[56:59]\n"
    "v_mfma_f32_16x16x32_bf16 v[60:63], v[212:215], v[204:207], v[60:63]\n"
    "v_mfma_f32_16x16x32_bf16 v[60:63], v[216:219], v[208:211], v[60:63]\n"
    "v_mfma_f32_16x16x32_bf16 v[64:67], v[220:223], v[204:207], v[64:67]\n"
    "v_mfma_f32_16x16x32_bf16 v[64:67], v[224:227], v[208:211], v[64:67]\n"
    "v_mfma_f32_16x16x32_bf16 v[68:71], v[228:231], v[204:207], v[68:71]\n"
    "v_mfma_f32_16x16x32_bf16 v[68:71], v[232:235], v[208:211], v[68:71]\n"
    "v_mfma_f32_16x16x32_bf16 v[72:75], v[236:239], v[204:207], v[72:75]\n"
    "v_mfma_f32_16x16x32_bf16 v[72:75], v[240:243], v[208:211], v[72:75]\n"
    "s_setprio 0\n"
    "s_barrier\n"
    "; phase 49: step 30 slot 1 stage 1\n"
    "ds_read_b64_tr_b16 v[212:213], v246 offset:16384\n"
    "ds_read_b64_tr_b16 v[214:215], v246 offset:17408\n"
    "s_add_u32 m0, s51, 0x0\n"
    "s_xor_b32 s82, s66, 0x2000000\n"
    "buffer_load_dwordx4 v251, s[40:43], s82 offen  lds\n"
    "ds_read_b64_tr_b16 v[216:217], v246 offset:24576\n"
    "ds_read_b64_tr_b16 v[218:219], v246 offset:25600\n"
    "s_add_u32 m0, s51, 0x2000\n"
    "s_xor_b32 s82, s66, 0x2080000\n"
    "buffer_load_dwordx4 v251, s[40:43], s82 offen  lds\n"
    "ds_read_b64_tr_b16 v[220:221], v247 offset:16384\n"
    "ds_read_b64_tr_b16 v[222:223], v247 offset:17408\n"
    "s_add_u32 m0, s50, 0x0\n"
    "s_xor_b32 s82, s48, 0x201080\n"
    "buffer_load_dwordx4 v250, s[36:39], s82 offen  lds\n"
    "ds_read_b64_tr_b16 v[224:225], v247 offset:24576\n"
    "ds_read_b64_tr_b16 v[226:227], v247 offset:25600\n"
    "s_add_u32 m0, s50, 0x1000\n"
    "s_xor_b32 s82, s48, 0x281080\n"
    "buffer_load_dwordx4 v250, s[36:39], s82 offen  lds\n"
    "ds_read_b64_tr_b16 v[228:229], v248 offset:16384\n"
    "ds_read_b64_tr_b16 v[230:231], v248 offset:17408\n"
    "s_add_u32 m0, s50, 0x2000\n"
    "s_xor_b32 s82, s48, 0x1080\n"
    "buffer_load_dwordx4 v250, s[36:39], s82 offen  lds\n"
    "ds_read_b64_tr_b16 v[232:233], v248 offset:24576\n"
    "ds_read_b64_tr_b16 v[234:235], v248 offset:25600\n"
    "s_add_u32 m0, s50, 0x3000\n"
    "s_xor_b32 s82, s48, 0x81080\n"
    "buffer_load_dwordx4 v250, s[36:39], s82 offen  lds\n"
    "ds_read_b64_tr_b16 v[236:237], v249 offset:16384\n"
    "ds_read_b64_tr_b16 v[238:239], v249 offset:17408\n"
    "s_cmp_lg_u32 s52, 3\n"
    "s_cbranch_scc1 .La82_%=\n"
    "s_xor_b32 s82, s66, 0x3700000\n"
    "s_add_u32 s82, s82, s53\n"
    "buffer_load_dword v9, v8, s[40:43], s82 offen \n"
    ".La82_%=:\n"
    "ds_read_b64_tr_b16 v[240:241], v249 offset:24576\n"
    "ds_read_b64_tr_b16 v[242:243], v249 offset:25600\n"
    "s_waitcnt vmcnt(12) lgkmcnt(0)\n"
    "s_barrier\n"
    "s_setprio 1\n"
    "v_mfma_f32_16x16x32_bf16 v[76:79], v[212:215], v[180:183], v[76:79]\n"
    "v_mfma_f32_16x16x32_bf16 v[76:79], v[216:219], v[184:187], v[76:79]\n"
    "v_mfma_f32_16x16x32_bf16 v[80:83], v[220:223], v[180:183], v[80:83]\n"
    "v_mfma_f32_16x16x32_bf16 v[80:83], v[224:227], v[184:187], v[80:83]\n"
    "v_mfma_f32_16x16x32_bf16 v[84:87], v[228:231], v[180:183], v[84:87]\n"
    "v_mfma_f32_16x16x32_bf16 v[84:87], v[232:235], v[184:187], v[84:87]\n"
    "v_mfma_f32_16x16x32_bf16 v[88:91], v[236:239], v[180:183], v[88:91]\n"
    "v_mfma_f32_16x16x32_bf16 v[88:91], v[240:243], v[184:187], v[88:91]\n"
    "v_mfma_f32_16x16x32_bf16 v[92:95], v[212:215], v[188:191], v[92:95]\n"
    "v_mfma_f32_16x16x32_bf16 v[92:95], v[216:219], v[192:195], v[92:95]\n"
    "v_mfma_f32_16x16x32_bf16 v[96:99], v[220:223], v[188:191], v[96:99]\n"
    "v_mfma_f32_16x16x32_bf16 v[96:99], v[224:227], v[192:195], v[96:99]\n"
    "v_mfma_f32_16x16x32_bf16 v[100:103], v[228:231], v[188:191], v[100:103]\n"
    "v_mfma_f32_16x16x32_bf16 v[100:103], v[232:235], v[192:195], v[100:103]\n"
    "v_mfma_f32_16x16x32_bf16 v[104:107], v[236:239], v[188:191], v[104:107]\n"
    "v_mfma_f32_16x16x32_bf16 v[104:107], v[240:243], v[192:195], v[104:107]\n"
    "v_mfma_f32_16x16x32_bf16 v[108:111], v[212:215], v[196:199], v[108:111]\n"
    "v_mfma_f32_16x16x32_bf16 v[108:111], v[216:219], v[200:203], v[108:111]\n"
    "v_mfma_f32_16x16x32_bf16 v[112:115], v[220:223], v[196:199], v[112:115]\n"
    "v_mfma_f32_16x16x32_bf16 v[112:115], v[224:227], v[200:203], v[112:115]\n"
    "v_mfma_f32_16x16x32_bf16 v[116:119], v[228:231], v[196:199], v[116:119]\n"
    "v_mfma_f32_16x16x32_bf16 v[116:119], v[232:235], v[200:203], v[116:119]\n"
    "v_mfma_f32_16x16x32_bf16 v[120:123], v[236:239], v[196:199], v[120:123]\n"
    "v_mfma_f32_16x16x32_bf16 v[120:123], v[240:243], v[200:203], v[120:123]\n"
    "v_mfma_f32_16x16x32_bf16 v[124:127], v[212:215], v[204:207], v[124:127]\n"
    "v_mfma_f32_16x16x32_bf16 v[124:127], v[216:219], v[208:211], v[124:127]\n"
    "v_mfma_f32_16x16x32_bf16 v[128:131], v[220:223], v[204:207], v[128:131]\n"
    "v_mfma_f32_16x16x32_bf16 v[128:131], v[224:227], v[208:211], v[128:131]\n"
    "v_mfma_f32_16x16x32_bf16 v[132:135], v[228:231], v[204:207], v[132:135]\n"
    "v_mfma_f32_16x16x32_bf16 v[132:135], v[232:235], v[208:211], v[132:135]\n"
    "v_mfma_f32_16x16x32_bf16 v[136:139], v[236:239], v[204:207], v[136:139]\n"
    "v_mfma_f32_16x16x32_bf16 v[136:139], v[240:243], v[208:211], v[136:139]\n"
    "s_setprio 0\n"
    "s_barrier\n"
    "; phase 50: step 31 slot 0 stage 0\n"
    "ds_read_b128 v[180:183], v244 offset:16384\n"
    "ds_read_b128 v[184:187], v245 offset:16384\n"
    "ds_read_b128 v[188:191], v244 offset:18432\n"
    "ds_read_b128 v[192:195], v245 offset:18432\n"
    "s_add_u32 m0, s51, 0x4000\n"
    "s_xor_b32 s82, s65, 0x2000000\n"
    "buffer_load_dwordx4 v251, s[40:43], s82 offen  lds\n"
    "ds_read_b128 v[196:199], v244 offset:24576\n"
    "ds_read_b128 v[200:203], v245 offset:24576\n"
    "ds_read_b128 v[204:207], v244 offset:26624\n"
    "ds_read_b128 v[208:211], v245 offset:26624\n"
    "s_add_u32 m0, s51, 0x6000\n"
    "s_xor_b32 s82, s65, 0x2080000\n"
    "buffer_load_dwordx4 v251, s[40:43], s82 offen  lds\n"
    "ds_read_b64_tr_b16 v[212:213], v246 offset:32768\n"
    "ds_read_b64_tr_b16 v[214:215], v246 offset:33792\n"
    "ds_read_b64_tr_b16 v[216:217], v246 offset:40960\n"
    "ds_read_b64_tr_b16 v[218:219], v246 offset:41984\n"
    "s_cmp_lg_u32 s52, 4\n"
    "s_cbranch_scc1 .La83_%=\n"
    "s_xor_b32 s82, s66, 0x3f00000\n"
    "s_add_u32 s82, s82, s53\n"
    "buffer_load_dword v9, v8, s[40:43], s82 offen \n"
    ".La83_%=:\n"
    "ds_read_b64_tr_b16 v[220:221], v247 offset:32768\n"
    "ds_read_b64_tr_b16 v[222:223], v247 offset:33792\n"
    "ds_read_b64_tr_b16 v[224:225], v247 offset:40960\n"
    "ds_read_b64_tr_b16 v[226:227], v247 offset:41984\n"
    "ds_read_b64_tr_b16 v[228:229], v248 offset:32768\n"
    "ds_read_b64_tr_b16 v[230:231], v248 offset:33792\n"
    "ds_read_b64_tr_b16 v[232:233], v248 offset:40960\n"
    "ds_read_b64_tr_b16 v[234:235], v248 offset:41984\n"
    "ds_read_b64_tr_b16 v[236:237], v249 offset:32768\n"
    "ds_read_b64_tr_b16 v[238:239], v249 offset:33792\n"
    "ds_read_b64_tr_b16 v[240:241], v249 offset:40960\n"
    "ds_read_b64_tr_b16 v[242:243], v249 offset:41984\n"
    "s_waitcnt vmcnt(8) lgkmcnt(0)\n"
    "s_barrier\n"
    "s_setprio 1\n"
    "v_mfma_f32_16x16x32_bf16 v[12:15], v[212:215], v[180:183], v[12:15]\n"
    "v_mfma_f32_16x16x32_bf16 v[12:15], v[216:219], v[184:187], v[12:15]\n"
    "v_mfma_f32_16x16x32_bf16 v[16:19], v[220:223], v[180:183], v[16:19]\n"
    "v_mfma_f32_16x16x32_bf16 v[16:19], v[224:227], v[184:187], v[16:19]\n"
    "v_mfma_f32_16x16x32_bf16 v[20:23], v[228:231], v[180:183], v[20:23]\n"
    "v_mfma_f32_16x16x32_bf16 v[20:23], v[232:235], v[184:187], v[20:23]\n"
    "v_mfma_f32_16x16x32_bf16 v[24:27], v[236:239], v[180:183], v[24:27]\n"
    "v_mfma_f32_16x16x32_bf16 v[24:27], v[240:243], v[184:187], v[24:27]\n"
    "v_mfma_f32_16x16x32_bf16 v[28:31], v[212:215], v[188:191], v[28:31]\n"
    "v_mfma_f32_16x16x32_bf16 v[28:31], v[216:219], v[192:195], v[28:31]\n"
    "v_mfma_f32_16x16x32_bf16 v[32:35], v[220:223], v[188:191], v[32:35]\n"
    "v_mfma_f32_16x16x32_bf16 v[32:35], v[224:227], v[192:195], v[32:35]\n"
    "v_mfma_f32_16x16x32_bf16 v[36:39], v[228:231], v[188:191], v[36:39]\n"
    "v_mfma_f32_16x16x32_bf16 v[36:39], v[232:235], v[192:195], v[36:39]\n"
    "v_mfma_f32_16x16x32_bf16 v[40:43], v[236:239], v[188:191], v[40:43]\n"
    "v_mfma_f32_16x16x32_bf16 v[40:43], v[240:243], v[192:195], v[40:43]\n"
    "v_mfma_f32_16x16x32_bf16 v[44:47], v[212:215], v[196:199], v[44:47]\n"
    "v_mfma_f32_16x16x32_bf16 v[44:47], v[216:219], v[200:203], v[44:47]\n"
    "v_mfma_f32_16x16x32_bf16 v[48:51], v[220:223], v[196:199], v[48:51]\n"
    "v_mfma_f32_16x16x32_bf16 v[48:51], v[224:227], v[200:203], v[48:51]\n"
    "v_mfma_f32_16x16x32_bf16 v[52:55], v[228:231], v[196:199], v[52:55]\n"
    "v_mfma_f32_16x16x32_bf16 v[52:55], v[232:235], v[200:203], v[52:55]\n"
    "v_mfma_f32_16x16x32_bf16 v[56:59], v[236:239], v[196:199], v[56:59]\n"
    "v_mfma_f32_16x16x32_bf16 v[56:59], v[240:243], v[200:203], v[56:59]\n"
    "v_cvt_pk_bf16_f32 v12, v12, v13\n"
    "v_cvt_pk_bf16_f32 v13, v14, v15\n"
    "v_mfma_f32_16x16x32_bf16 v[60:63], v[212:215], v[204:207], v[60:63]\n"
    "v_cvt_pk_bf16_f32 v14, v16, v17\n"
    "v_cvt_pk_bf16_f32 v15, v18, v19\n"
    "v_mfma_f32_16x16x32_bf16 v[60:63], v[216:219], v[208:211], v[60:63]\n"
    "v_cvt_pk_bf16_f32 v20, v20, v21\n"
    "v_cvt_pk_bf16_f32 v21, v22, v23\n"
    "v_mfma_f32_16x16x32_bf16 v[64:67], v[220:223], v[204:207], v[64:67]\n"
    "v_cvt_pk_bf16_f32 v22, v24, v25\n"
    "v_cvt_pk_bf16_f32 v23, v26, v27\n"
    "v_mfma_f32_16x16x32_bf16 v[64:67], v[224:227], v[208:211], v[64:67]\n"
    "v_cvt_pk_bf16_f32 v28, v28, v29\n"
    "v_cvt_pk_bf16_f32 v29, v30, v31\n"
    "v_mfma_f32_16x16x32_bf16 v[68:71], v[228:231], v[204:207], v[68:71]\n"
    "v_cvt_pk_bf16_f32 v30, v32, v33\n"
    "v_cvt_pk_bf16_f32 v31, v34, v35\n"
    "v_mfma_f32_16x16x32_bf16 v[68:71], v[232:235], v[208:211], v[68:71]\n"
    "v_cvt_pk_bf16_f32 v36, v36, v37\n"
    "v_cvt_pk_bf16_f32 v37, v38, v39\n"
    "v_mfma_f32_16x16x32_bf16 v[72:75], v[236:239], v[204:207], v[72:75]\n"
    "v_cvt_pk_bf16_f32 v38, v40, v41\n"
    "v_cvt_pk_bf16_f32 v39, v42, v43\n"
    "v_mfma_f32_16x16x32_bf16 v[72:75], v[240:243], v[208:211], v[72:75]\n"
    "s_setprio 0\n"
    "s_barrier\n"
    "; phase 51: step 31 slot 1 stage 1\n"
    "ds_read_b64_tr_b16 v[212:213], v246 offset:49152\n"
    "ds_read_b64_tr_b16 v[214:215], v246 offset:50176\n"
    "s_xor_b32 s82, s54, 0x6e000\n"
    "s_add_u32 s82, s82, 0x100000\n"
    "buffer_store_dwordx4 v[12:15], v252, s[44:47], s82 offen sc0 sc1\n"
    "ds_read_b64_tr_b16 v[216:217], v246 offset:57344\n"
    "ds_read_b64_tr_b16 v[218:219], v246 offset:58368\n"
    "s_add_u32 m0, s51, 0x8000\n"
    "s_xor_b32 s82, s66, 0x2100000\n"
    "buffer_load_dwordx4 v251, s[40:43], s82 offen  lds\n"
    "ds_read_b64_tr_b16 v[220:221], v247 offset:49152\n"
    "ds_read_b64_tr_b16 v[222:223], v247 offset:50176\n"
    "s_add_u32 m0, s51, 0xa000\n"
    "s_xor_b32 s82, s66, 0x2180000\n"
    "buffer_load_dwordx4 v251, s[40:43], s82 offen  lds\n"
    "ds_read_b64_tr_b16 v[224:225], v247 offset:57344\n"
    "ds_read_b64_tr_b16 v[226:227], v247 offset:58368\n"
    "s_add_u32 m0, s50, 0x4000\n"
    "s_xor_b32 s82, s48, 0x201100\n"
    "buffer_load_dwordx4 v250, s[36:39], s82 offen  lds\n"
    "ds_read_b64_tr_b16 v[228:229], v248 offset:49152\n"
    "ds_read_b64_tr_b16 v[230:231], v248 offset:50176\n"
    "s_add_u32 m0, s50, 0x5000\n"
    "s_xor_b32 s82, s48, 0x281100\n"
    "buffer_load_dwordx4 v250, s[36:39], s82 offen  lds\n"
    "ds_read_b64_tr_b16 v[232:233], v248 offset:57344\n"
    "ds_read_b64_tr_b16 v[234:235], v248 offset:58368\n"
    "s_add_u32 m0, s50, 0x6000\n"
    "s_xor_b32 s82, s48, 0x1100\n"
    "buffer_load_dwordx4 v250, s[36:39], s82 offen  lds\n"
    "ds_read_b64_tr_b16 v[236:237], v249 offset:49152\n"
    "ds_read_b64_tr_b16 v[238:239], v249 offset:50176\n"
    "s_add_u32 m0, s50, 0x7000\n"
    "s_xor_b32 s82, s48, 0x81100\n"
    "buffer_load_dwordx4 v250, s[36:39], s82 offen  lds\n"
    "ds_read_b64_tr_b16 v[240:241], v249 offset:57344\n"
    "ds_read_b64_tr_b16 v[242:243], v249 offset:58368\n"
    "s_cmp_lg_u32 s52, 5\n"
    "s_cbranch_scc1 .La84_%=\n"
    "s_xor_b32 s82, s67, 0x3f00000\n"
    "s_add_u32 s82, s82, s53\n"
    "buffer_load_dword v9, v8, s[40:43], s82 offen \n"
    ".La84_%=:\n"
    "s_waitcnt vmcnt(13) lgkmcnt(0)\n"
    "s_barrier\n"
    "s_setprio 1\n"
    "v_mfma_f32_16x16x32_bf16 v[76:79], v[212:215], v[180:183], v[76:79]\n"
    "v_cvt_pk_bf16_f32 v44, v44, v45\n"
    "v_cvt_pk_bf16_f32 v45, v46, v47\n"
    "v_mfma_f32_16x16x32_bf16 v[76:79], v[216:219], v[184:187], v[76:79]\n"
    "v_cvt_pk_bf16_f32 v46, v48, v49\n"
    "v_cvt_pk_bf16_f32 v47, v50, v51\n"
    "v_mfma_f32_16x16x32_bf16 v[80:83], v[220:223], v[180:183], v[80:83]\n"
    "v_cvt_pk_bf16_f32 v52, v52, v53\n"
    "v_cvt_pk_bf16_f32 v53, v54, v55\n"
    "v_mfma_f32_16x16x32_bf16 v[80:83], v[224:227], v[184:187], v[80:83]\n"
    "v_cvt_pk_bf16_f32 v54, v56, v57\n"
    "v_cvt_pk_bf16_f32 v55, v58, v59\n"
    "v_mfma_f32_16x16x32_bf16 v[84:87], v[228:231], v[180:183], v[84:87]\n"
    "v_mfma_f32_16x16x32_bf16 v[84:87], v[232:235], v[184:187], v[84:87]\n"
    "v_mfma_f32_16x16x32_bf16 v[88:91], v[236:239], v[180:183], v[88:91]\n"
    "v_mfma_f32_16x16x32_bf16 v[88:91], v[240:243], v[184:187], v[88:91]\n"
    "v_mfma_f32_16x16x32_bf16 v[92:95], v[212:215], v[188:191], v[92:95]\n"
    "v_mfma_f32_16x16x32_bf16 v[92:95], v[216:219], v[192:195], v[92:95]\n"
    "v_mfma_f32_16x16x32_bf16 v[96:99], v[220:223], v[188:191], v[96:99]\n"
    "v_mfma_f32_16x16x32_bf16 v[96:99], v[224:227], v[192:195], v[96:99]\n"
    "v_mfma_f32_16x16x32_bf16 v[100:103], v[228:231], v[188:191], v[100:103]\n"
    "v_mfma_f32_16x16x32_bf16 v[100:103], v[232:235], v[192:195], v[100:103]\n"
    "v_mfma_f32_16x16x32_bf16 v[104:107], v[236:239], v[188:191], v[104:107]\n"
    "v_mfma_f32_16x16x32_bf16 v[104:107], v[240:243], v[192:195], v[104:107]\n"
    "v_mfma_f32_16x16x32_bf16 v[108:111], v[212:215], v[196:199], v[108:111]\n"
    "v_mfma_f32_16x16x32_bf16 v[108:111], v[216:219], v[200:203], v[108:111]\n"
    "v_mfma_f32_16x16x32_bf16 v[112:115], v[220:223], v[196:199], v[112:115]\n"
    "v_mfma_f32_16x16x32_bf16 v[112:115], v[224:227], v[200:203], v[112:115]\n"
    "v_mfma_f32_16x16x32_bf16 v[116:119], v[228:231], v[196:199], v[116:119]\n"
    "v_mfma_f32_16x16x32_bf16 v[116:119], v[232:235], v[200:203], v[116:119]\n"
    "v_mfma_f32_16x16x32_bf16 v[120:123], v[236:239], v[196:199], v[120:123]\n"
    "v_mfma_f32_16x16x32_bf16 v[120:123], v[240:243], v[200:203], v[120:123]\n"
    "v_mfma_f32_16x16x32_bf16 v[124:127], v[212:215], v[204:207], v[124:127]\n"
    "v_mfma_f32_16x16x32_bf16 v[124:127], v[216:219], v[208:211], v[124:127]\n"
    "v_mfma_f32_16x16x32_bf16 v[128:131], v[220:223], v[204:207], v[128:131]\n"
    "v_mfma_f32_16x16x32_bf16 v[128:131], v[224:227], v[208:211], v[128:131]\n"
    "v_mfma_f32_16x16x32_bf16 v[132:135], v[228:231], v[204:207], v[132:135]\n"
    "v_mfma_f32_16x16x32_bf16 v[132:135], v[232:235], v[208:211], v[132:135]\n"
    "v_mfma_f32_16x16x32_bf16 v[136:139], v[236:239], v[204:207], v[136:139]\n"
    "v_mfma_f32_16x16x32_bf16 v[136:139], v[240:243], v[208:211], v[136:139]\n"
    "s_setprio 0\n"
    "s_barrier\n"
    "; phase 52: step 32 slot 0 stage 2\n"
    "ds_read_b128 v[180:183], v244 offset:32768\n"
    "ds_read_b128 v[184:187], v245 offset:32768\n"
    "ds_read_b128 v[188:191], v244 offset:34816\n"
    "ds_read_b128 v[192:195], v245 offset:34816\n"
    "s_add_u32 m0, s51, 0xc000\n"
    "s_xor_b32 s82, s65, 0x2100000\n"
    "buffer_load_dwordx4 v251, s[40:43], s82 offen  lds\n"
    "ds_read_b128 v[196:199], v244 offset:40960\n"
    "ds_read_b128 v[200:203], v245 offset:40960\n"
    "ds_read_b128 v[204:207], v244 offset:43008\n"
    "ds_read_b128 v[208:211], v245 offset:43008\n"
    "s_add_u32 m0, s51, 0xe000\n"
    "s_xor_b32 s82, s65, 0x2180000\n"
    "buffer_load_dwordx4 v251, s[40:43], s82 offen  lds\n"
    "ds_read_b64_tr_b16 v[212:213], v246 offset:0\n"
    "ds_read_b64_tr_b16 v[214:215], v246 offset:1024\n"
    "ds_read_b64_tr_b16 v[216:217], v246 offset:8192\n"
    "ds_read_b64_tr_b16 v[218:219], v246 offset:9216\n"
    "s_cmp_lg_u32 s52, 6\n"
    "s_cbranch_scc1 .La85_%=\n"
    "s_xor_b32 s82, s67, 0x4000000\n"
    "s_add_u32 s82, s82, s53\n"
    "buffer_load_dword v9, v8, s[40:43], s82 offen \n"
    ".La85_%=:\n"
    "ds_read_b64_tr_b16 v[220:221], v247 offset:0\n"
    "ds_read_b64_tr_b16 v[222:223], v247 offset:1024\n"
    "ds_read_b64_tr_b16 v[224:225], v247 offset:8192\n"
    "ds_read_b64_tr_b16 v[226:227], v247 offset:9216\n"
    "ds_read_b64_tr_b16 v[228:229], v248 offset:0\n"
    "ds_read_b64_tr_b16 v[230:231], v248 offset:1024\n"
    "ds_read_b64_tr_b16 v[232:233], v248 offset:8192\n"
    "ds_read_b64_tr_b16 v[234:235], v248 offset:9216\n"
    "ds_read_b64_tr_b16 v[236:237], v249 offset:0\n"
    "ds_read_b64_tr_b16 v[238:239], v249 offset:1024\n"
    "ds_read_b64_tr_b16 v[240:241], v249 offset:8192\n"
    "ds_read_b64_tr_b16 v[242:243], v249 offset:9216\n"
    "s_waitcnt vmcnt(9) lgkmcnt(0)\n"
    "s_barrier\n"
    "s_setprio 1\n"
    "v_mfma_f32_16x16x32_bf16 v[140:143], v[212:215], v[180:183], 0\n"
    "v_mfma_f32_16x16x32_bf16 v[140:143], v[216:219], v[184:187], v[140:143]\n"
    "v_mfma_f32_16x16x32_bf16 v[144:147], v[220:223], v[180:183], 0\n"
    "v_mfma_f32_16x16x32_bf16 v[144:147], v[224:227], v[184:187], v[144:147]\n"
    "v_mfma_f32_16x16x32_bf16 v[148:151], v[228:231], v[180:183], 0\n"
    "v_mfma_f32_16x16x32_bf16 v[148:151], v[232:235], v[184:187], v[148:151]\n"
    "v_mfma_f32_16x16x32_bf16 v[152:155], v[236:239], v[180:183], 0\n"
    "v_mfma_f32_16x16x32_bf16 v[152:155], v[240:243], v[184:187], v[152:155]\n"
    "v_mfma_f32_16x16x32_bf16 v[156:159], v[212:215], v[188:191], 0\n"
    "v_mfma_f32_16x16x32_bf16 v[156:159], v[216:219], v[192:195], v[156:159]\n"
    "v_mfma_f32_16x16x32_bf16 v[160:163], v[220:223], v[188:191], 0\n"
    "v_mfma_f32_16x16x32_bf16 v[160:163], v[224:227], v[192:195], v[160:163]\n"
    "v_mfma_f32_16x16x32_bf16 v[164:167], v[228:231], v[188:191], 0\n"
    "v_mfma_f32_16x16x32_bf16 v[164:167], v[232:235], v[192:195], v[164:167]\n"
    "v_mfma_f32_16x16x32_bf16 v[168:171], v[236:239], v[188:191], 0\n"
    "v_mfma_f32_16x16x32_bf16 v[168:171], v[240:243], v[192:195], v[168:171]\n"
    "v_mfma_f32_16x16x32_bf16 v[172:175], v[212:215], v[196:199], 0\n"
    "v_mfma_f32_16x16x32_bf16 v[172:175], v[216:219], v[200:203], v[172:175]\n"
    "v_mfma_f32_16x16x32_bf16 v[176:179], v[220:223], v[196:199], 0\n"
    "v_mfma_f32_16x16x32_bf16 v[176:179], v[224:227], v[200:203], v[176:179]\n"
    "v_mfma_f32_16x16x32_bf16 v[16:19], v[228:231], v[196:199], 0\n"
    "v_mfma_f32_16x16x32_bf16 v[16:19], v[232:235], v[200:203], v[16:19]\n"
    "v_mfma_f32_16x16x32_bf16 v[24:27], v[236:239], v[196:199], 0\n"
    "v_mfma_f32_16x16x32_bf16 v[24:27], v[240:243], v[200:203], v[24:27]\n"
    "v_mfma_f32_16x16x32_bf16 v[32:35], v[212:215], v[204:207], 0\n"
    "v_mfma_f32_16x16x32_bf16 v[32:35], v[216:219], v[208:211], v[32:35]\n"
    "v_mfma_f32_16x16x32_bf16 v[40:43], v[220:223], v[204:207], 0\n"
    "v_mfma_f32_16x16x32_bf16 v[40:43], v[224:227], v[208:211], v[40:43]\n"
    "v_mfma_f32_16x16x32_bf16 v[12:15], v[228:231], v[204:207], 0\n"
    "v_mfma_f32_16x16x32_bf16 v[12:15], v[232:235], v[208:211], v[12:15]\n"
    "v_mfma_f32_16x16x32_bf16 v[48:51], v[236:239], v[204:207], 0\n"
    "v_mfma_f32_16x16x32_bf16 v[48:51], v[240:243], v[208:211], v[48:51]\n"
    "s_setprio 0\n"
    "s_barrier\n"
    "; phase 53: step 32 slot 1 stage 1\n"
    "ds_read_b64_tr_b16 v[212:213], v246 offset:16384\n"
    "ds_read_b64_tr_b16 v[214:215], v246 offset:17408\n"
    "s_xor_b32 s82, s54, 0x6c000\n"
    "s_add_u32 s82, s82, 0x100000\n"
    "buffer_store_dwordx4 v[20:23], v252, s[44:47], s82 offen sc0 sc1\n"
    "ds_read_b64_tr_b16 v[216:217], v246 offset:24576\n"
    "ds_read_b64_tr_b16 v[218:219], v246 offset:25600\n"
    "s_add_u32 m0, s51, 0x0\n"
    "s_xor_b32 s82, s66, 0x2200000\n"
    "buffer_load_dwordx4 v251, s[40:43], s82 offen  lds\n"
    "ds_read_b64_tr_b16 v[220:221], v247 offset:16384\n"
    "ds_read_b64_tr_b16 v[222:223], v247 offset:17408\n"
    "s_add_u32 m0, s51, 0x2000\n"
    "s_xor_b32 s82, s66, 0x2280000\n"
    "buffer_load_dwordx4 v251, s[40:43], s82 offen  lds\n"
    "ds_read_b64_tr_b16 v[224:225], v247 offset:24576\n"
    "ds_read_b64_tr_b16 v[226:227], v247 offset:25600\n"
    "s_add_u32 m0, s50, 0x8000\n"
    "s_xor_b32 s82, s48, 0x201180\n"
    "buffer_load_dwordx4 v250, s[36:39], s82 offen  lds\n"
    "ds_read_b64_tr_b16 v[228:229], v248 offset:16384\n"
    "ds_read_b64_tr_b16 v[230:231], v248 offset:17408\n"
    "s_add_u32 m0, s50, 0x9000\n"
    "s_xor_b32 s82, s48, 0x281180\n"
    "buffer_load_dwordx4 v250, s[36:39], s82 offen  lds\n"
    "ds_read_b64_tr_b16 v[232:233], v248 offset:24576\n"
    "ds_read_b64_tr_b16 v[234:235], v248 offset:25600\n"
    "s_add_u32 m0, s50, 0xa000\n"
    "s_xor_b32 s82, s48, 0x1180\n"
    "buffer_load_dwordx4 v250, s[36:39], s82 offen  lds\n"
    "ds_read_b64_tr_b16 v[236:237], v249 offset:16384\n"
    "ds_read_b64_tr_b16 v[238:239], v249 offset:17408\n"
    "s_add_u32 m0, s50, 0xb000\n"
    "s_xor_b32 s82, s48, 0x81180\n"
    "buffer_load_dwordx4 v250, s[36:39], s82 offen  lds\n"
    "ds_read_b64_tr_b16 v[240:241], v249 offset:24576\n"
    "ds_read_b64_tr_b16 v[242:243], v249 offset:25600\n"
    "s_cmp_lg_u32 s52, 7\n"
    "s_cbranch_scc1 .La86_%=\n"
    "s_xor_b32 s82, s67, 0x4100000\n"
    "s_add_u32 s82, s82, s53\n"
    "buffer_load_dword v9, v8, s[40:43], s82 offen \n"
    ".La86_%=:\n"
    "s_waitcnt vmcnt(13) lgkmcnt(0)\n"
    "s_barrier\n"
    "s_setprio 1\n"
    "v_mfma_f32_16x16x32_bf16 v[76:79], v[212:215], v[180:183], v[76:79]\n"
    "v_mfma_f32_16x16x32_bf16 v[76:79], v[216:219], v[184:187], v[76:79]\n"
    "v_mfma_f32_16x16x32_bf16 v[80:83], v[220:223], v[180:183], v[80:83]\n"
    "v_mfma_f32_16x16x32_bf16 v[80:83], v[224:227], v[184:187], v[80:83]\n"
    "v_mfma_f32_16x16x32_bf16 v[84:87], v[228:231], v[180:183], v[84:87]\n"
    "v_mfma_f32_16x16x32_bf16 v[84:87], v[232:235], v[184:187], v[84:87]\n"
    "v_mfma_f32_16x16x32_bf16 v[88:91], v[236:239], v[180:183], v[88:91]\n"
    "v_mfma_f32_16x16x32_bf16 v[88:91], v[240:243], v[184:187], v[88:91]\n"
    "v_mfma_f32_16x16x32_bf16 v[92:95], v[212:215], v[188:191], v[92:95]\n"
    "v_mfma_f32_16x16x32_bf16 v[92:95], v[216:219], v[192:195], v[92:95]\n"
    "v_mfma_f32_16x16x32_bf16 v[96:99], v[220:223], v[188:191], v[96:99]\n"
    "v_mfma_f32_16x16x32_bf16 v[96:99], v[224:227], v[192:195], v[96:99]\n"
    "v_mfma_f32_16x16x32_bf16 v[100:103], v[228:231], v[188:191], v[100:103]\n"
    "v_mfma_f32_16x16x32_bf16 v[100:103], v[232:235], v[192:195], v[100:103]\n"
    "v_mfma_f32_16x16x32_bf16 v[104:107], v[236:239], v[188:191], v[104:107]\n"
    "v_mfma_f32_16x16x32_bf16 v[104:107], v[240:243], v[192:195], v[104:107]\n"
    "v_mfma_f32_16x16x32_bf16 v[108:111], v[212:215], v[196:199], v[108:111]\n"
    "v_mfma_f32_16x16x32_bf16 v[108:111], v[216:219], v[200:203], v[108:111]\n"
    "v_mfma_f32_16x16x32_bf16 v[112:115], v[220:223], v[196:199], v[112:115]\n"
    "v_mfma_f32_16x16x32_bf16 v[112:115], v[224:227], v[200:203], v[112:115]\n"
    "v_mfma_f32_16x16x32_bf16 v[116:119], v[228:231], v[196:199], v[116:119]\n"
    "v_mfma_f32_16x16x32_bf16 v[116:119], v[232:235], v[200:203], v[116:119]\n"
    "v_mfma_f32_16x16x32_bf16 v[120:123], v[236:239], v[196:199], v[120:123]\n"
    "v_mfma_f32_16x16x32_bf16 v[120:123], v[240:243], v[200:203], v[120:123]\n"
    "v_mfma_f32_16x16x32_bf16 v[124:127], v[212:215], v[204:207], v[124:127]\n"
    "v_mfma_f32_16x16x32_bf16 v[124:127], v[216:219], v[208:211], v[124:127]\n"
    "v_mfma_f32_16x16x32_bf16 v[128:131], v[220:223], v[204:207], v[128:131]\n"
    "v_mfma_f32_16x16x32_bf16 v[128:131], v[224:227], v[208:211], v[128:131]\n"
    "v_mfma_f32_16x16x32_bf16 v[132:135], v[228:231], v[204:207], v[132:135]\n"
    "v_mfma_f32_16x16x32_bf16 v[132:135], v[232:235], v[208:211], v[132:135]\n"
    "v_mfma_f32_16x16x32_bf16 v[136:139], v[236:239], v[204:207], v[136:139]\n"
    "v_mfma_f32_16x16x32_bf16 v[136:139], v[240:243], v[208:211], v[136:139]\n"
    "s_setprio 0\n"
    "s_barrier\n"
    "; phase 54: step 33 slot 0 stage 2\n"
    "ds_read_b128 v[180:183], v244 offset:0\n"
    "ds_read_b128 v[184:187], v245 offset:0\n"
    "ds_read_b128 v[188:191], v244 offset:2048\n"
    "ds_read_b128 v[192:195], v245 offset:2048\n"
    "s_add_u32 m0, s51, 0x4000\n"
    "s_xor_b32 s82, s65, 0x2200000\n"
    "buffer_load_dwordx4 v251, s[40:43], s82 offen  lds\n"
    "ds_read_b128 v[196:199], v244 offset:8192\n"
    "ds_read_b128 v[200:203], v245 offset:8192\n"
    "ds_read_b128 v[204:207], v244 offset:10240\n"
    "ds_read_b128 v[208:211], v245 offset:10240\n"
    "s_add_u32 m0, s51, 0x6000\n"
    "s_xor_b32 s82, s65, 0x2280000\n"
    "buffer_load_dwordx4 v251, s[40:43], s82 offen  lds\n"
    "ds_read_b64_tr_b16 v[212:213], v246 offset:32768\n"
    "ds_read_b64_tr_b16 v[214:215], v246 offset:33792\n"
    "ds_read_b64_tr_b16 v[216:217], v246 offset:40960\n"
    "ds_read_b64_tr_b16 v[218:219], v246 offset:41984\n"
    "ds_read_b64_tr_b16 v[220:221], v247 offset:32768\n"
    "ds_read_b64_tr_b16 v[222:223], v247 offset:33792\n"
    "ds_read_b64_tr_b16 v[224:225], v247 offset:40960\n"
    "ds_read_b64_tr_b16 v[226:227], v247 offset:41984\n"
    "ds_read_b64_tr_b16 v[228:229], v248 offset:32768\n"
    "ds_read_b64_tr_b16 v[230:231], v248 offset:33792\n"
    "ds_read_b64_tr_b16 v[232:233], v248 offset:40960\n"
    "ds_read_b64_tr_b16 v[234:235], v248 offset:41984\n"
    "ds_read_b64_tr_b16 v[236:237], v249 offset:32768\n"
    "ds_read_b64_tr_b16 v[238:239], v249 offset:33792\n"
    "ds_read_b64_tr_b16 v[240:241], v249 offset:40960\n"
    "ds_read_b64_tr_b16 v[242:243], v249 offset:41984\n"
    "s_waitcnt vmcnt(9) lgkmcnt(0)\n"
    "s_barrier\n"
    "s_setprio 1\n"
    "v_mfma_f32_16x16x32_bf16 v[140:143], v[212:215], v[180:183], v[140:143]\n"
    "v_mfma_f32_16x16x32_bf16 v[140:143], v[216:219], v[184:187], v[140:143]\n"
    "v_mfma_f32_16x16x32_bf16 v[144:147], v[220:223], v[180:183], v[144:147]\n"
    "v_mfma_f32_16x16x32_bf16 v[144:147], v[224:227], v[184:187], v[144:147]\n"
    "v_mfma_f32_16x16x32_bf16 v[148:151], v[228:231], v[180:183], v[148:151]\n"
    "v_mfma_f32_16x16x32_bf16 v[148:151], v[232:235], v[184:187], v[148:151]\n"
    "v_mfma_f32_16x16x32_bf16 v[152:155], v[236:239], v[180:183], v[152:155]\n"
    "v_mfma_f32_16x16x32_bf16 v[152:155], v[240:243], v[184:187], v[152:155]\n"
    "v_mfma_f32_16x16x32_bf16 v[156:159], v[212:215], v[188:191], v[156:159]\n"
    "v_mfma_f32_16x16x32_bf16 v[156:159], v[216:219], v[192:195], v[156:159]\n"
    "v_mfma_f32_16x16x32_bf16 v[160:163], v[220:223], v[188:191], v[160:163]\n"
    "v_mfma_f32_16x16x32_bf16 v[160:163], v[224:227], v[192:195], v[160:163]\n"
    "v_mfma_f32_16x16x32_bf16 v[164:167], v[228:231], v[188:191], v[164:167]\n"
    "v_mfma_f32_16x16x32_bf16 v[164:167], v[232:235], v[192:195], v[164:167]\n"
    "v_mfma_f32_16x16x32_bf16 v[168:171], v[236:239], v[188:191], v[168:171]\n"
    "v_mfma_f32_16x16x32_bf16 v[168:171], v[240:243], v[192:195], v[168:171]\n"
    "v_mfma_f32_16x16x32_bf16 v[172:175], v[212:215], v[196:199], v[172:175]\n"
    "v_mfma_f32_16x16x32_bf16 v[172:175], v[216:219], v[200:203], v[172:175]\n"
    "v_mfma_f32_16x16x32_bf16 v[176:179], v[220:223], v[196:199], v[176:179]\n"
    "v_mfma_f32_16x16x32_bf16 v[176:179], v[224:227], v[200:203], v[176:179]\n"
    "v_mfma_f32_16x16x32_bf16 v[16:19], v[228:231], v[196:199], v[16:19]\n"
    "v_mfma_f32_16x16x32_bf16 v[16:19], v[232:235], v[200:203], v[16:19]\n"
    "v_mfma_f32_16x16x32_bf16 v[24:27], v[236:239], v[196:199], v[24:27]\n"
    "v_mfma_f32_16x16x32_bf16 v[24:27], v[240:243], v[200:203], v[24:27]\n"
    "v_mfma_f32_16x16x32_bf16 v[32:35], v[212:215], v[204:207], v[32:35]\n"
    "v_mfma_f32_16x16x32_bf16 v[32:35], v[216:219], v[208:211], v[32:35]\n"
    "v_mfma_f32_16x16x32_bf16 v[40:43], v[220:223], v[204:207], v[40:43]\n"
    "v_mfma_f32_16x16x32_bf16 v[40:43], v[224:227], v[208:211], v[40:43]\n"
    "v_mfma_f32_16x16x32_bf16 v[12:15], v[228:231], v[204:207], v[12:15]\n"
    "v_mfma_f32_16x16x32_bf16 v[12:15], v[232:235], v[208:211], v[12:15]\n"
    "v_mfma_f32_16x16x32_bf16 v[48:51], v[236:239], v[204:207], v[48:51]\n"
    "v_mfma_f32_16x16x32_bf16 v[48:51], v[240:243], v[208:211], v[48:51]\n"
    "s_setprio 0\n"
    "s_barrier\n"
    "; phase 55: step 33 slot 1 stage 1\n"
    "ds_read_b64_tr_b16 v[212:213], v246 offset:49152\n"
    "ds_read_b64_tr_b16 v[214:215], v246 offset:50176\n"
    "s_xor_b32 s82, s54, 0x4a000\n"
    "s_add_u32 s82, s82, 0x100000\n"
    "buffer_store_dwordx4 v[28:31], v252, s[44:47], s82 offen sc0 sc1\n"
    "ds_read_b64_tr_b16 v[216:217], v246 offset:57344\n"
    "ds_read_b64_tr_b16 v[218:219], v246 offset:58368\n"
    "s_add_u32 m0, s51, 0x8000\n"
    "s_xor_b32 s82, s66, 0x2300000\n"
    "buffer_load_dwordx4 v251, s[40:43], s82 offen  lds\n"
    "ds_read_b64_tr_b16 v[220:221], v247 offset:49152\n"
    "ds_read_b64_tr_b16 v[222:223], v247 offset:50176\n"
    "s_add_u32 m0, s51, 0xa000\n"
    "s_xor_b32 s82, s66, 0x2380000\n"
    "buffer_load_dwordx4 v251, s[40:43], s82 offen  lds\n"
    "ds_read_b64_tr_b16 v[224:225], v247 offset:57344\n"
    "ds_read_b64_tr_b16 v[226:227], v247 offset:58368\n"
    "s_add_u32 m0, s50, 0x0\n"
    "s_xor_b32 s82, s48, 0x201200\n"
    "buffer_load_dwordx4 v250, s[36:39], s82 offen  lds\n"
    "ds_read_b64_tr_b16 v[228:229], v248 offset:49152\n"
    "ds_read_b64_tr_b16 v[230:231], v248 offset:50176\n"
    "s_add_u32 m0, s50, 0x1000\n"
    "s_xor_b32 s82, s48, 0x281200\n"
    "buffer_load_dwordx4 v250, s[36:39], s82 offen  lds\n"
    "ds_read_b64_tr_b16 v[232:233], v248 offset:57344\n"
    "ds_read_b64_tr_b16 v[234:235], v248 offset:58368\n"
    "s_add_u32 m0, s50, 0x2000\n"
    "s_xor_b32 s82, s48, 0x1200\n"
    "buffer_load_dwordx4 v250, s[36:39], s82 offen  lds\n"
    "ds_read_b64_tr_b16 v[236:237], v249 offset:49152\n"
    "ds_read_b64_tr_b16 v[238:239], v249 offset:50176\n"
    "s_add_u32 m0, s50, 0x3000\n"
    "s_xor_b32 s82, s48, 0x81200\n"
    "buffer_load_dwordx4 v250, s[36:39], s82 offen  lds\n"
    "ds_read_b64_tr_b16 v[240:241], v249 offset:57344\n"
    "ds_read_b64_tr_b16 v[242:243], v249 offset:58368\n"
    "s_cmp_lg_u32 s52, 0\n"
    "s_cbranch_scc1 .La87_%=\n"
    "s_xor_b32 s82, s67, 0x4200000\n"
    "s_add_u32 s82, s82, s53\n"
    "buffer_load_dword v9, v8, s[40:43], s82 offen \n"
    ".La87_%=:\n"
    "s_waitcnt vmcnt(13) lgkmcnt(0)\n"
    "s_barrier\n"
    "s_setprio 1\n"
    "v_mfma_f32_16x16x32_bf16 v[76:79], v[212:215], v[180:183], v[76:79]\n"
    "v_mfma_f32_16x16x32_bf16 v[76:79], v[216:219], v[184:187], v[76:79]\n"
    "v_mfma_f32_16x16x32_bf16 v[80:83], v[220:223], v[180:183], v[80:83]\n"
    "v_mfma_f32_16x16x32_bf16 v[80:83], v[224:227], v[184:187], v[80:83]\n"
    "v_mfma_f32_16x16x32_bf16 v[84:87], v[228:231], v[180:183], v[84:87]\n"
    "v_mfma_f32_16x16x32_bf16 v[84:87], v[232:235], v[184:187], v[84:87]\n"
    "v_mfma_f32_16x16x32_bf16 v[88:91], v[236:239], v[180:183], v[88:91]\n"
    "v_mfma_f32_16x16x32_bf16 v[88:91], v[240:243], v[184:187], v[88:91]\n"
    "v_mfma_f32_16x16x32_bf16 v[92:95], v[212:215], v[188:191], v[92:95]\n"
    "v_mfma_f32_16x16x32_bf16 v[92:95], v[216:219], v[192:195], v[92:95]\n"
    "v_mfma_f32_16x16x32_bf16 v[96:99], v[220:223], v[188:191], v[96:99]\n"
    "v_mfma_f32_16x16x32_bf16 v[96:99], v[224:227], v[192:195], v[96:99]\n"
    "v_mfma_f32_16x16x32_bf16 v[100:103], v[228:231], v[188:191], v[100:103]\n"
    "v_mfma_f32_16x16x32_bf16 v[100:103], v[232:235], v[192:195], v[100:103]\n"
    "v_mfma_f32_16x16x32_bf16 v[104:107], v[236:239], v[188:191], v[104:107]\n"
    "v_mfma_f32_16x16x32_bf16 v[104:107], v[240:243], v[192:195], v[104:107]\n"
    "v_mfma_f32_16x16x32_bf16 v[108:111], v[212:215], v[196:199], v[108:111]\n"
    "v_mfma_f32_16x16x32_bf16 v[108:111], v[216:219], v[200:203], v[108:111]\n"
    "v_mfma_f32_16x16x32_bf16 v[112:115], v[220:223], v[196:199], v[112:115]\n"
    "v_mfma_f32_16x16x32_bf16 v[112:115], v[224:227], v[200:203], v[112:115]\n"
    "v_mfma_f32_16x16x32_bf16 v[116:119], v[228:231], v[196:199], v[116:119]\n"
    "v_mfma_f32_16x16x32_bf16 v[116:119], v[232:235], v[200:203], v[116:119]\n"
    "v_mfma_f32_16x16x32_bf16 v[120:123], v[236:239], v[196:199], v[120:123]\n"
    "v_mfma_f32_16x16x32_bf16 v[120:123], v[240:243], v[200:203], v[120:123]\n"
    "v_mfma_f32_16x16x32_bf16 v[124:127], v[212:215], v[204:207], v[124:127]\n"
    "v_mfma_f32_16x16x32_bf16 v[124:127], v[216:219], v[208:211], v[124:127]\n"
    "v_mfma_f32_16x16x32_bf16 v[128:131], v[220:223], v[204:207], v[128:131]\n"
    "v_mfma_f32_16x16x32_bf16 v[128:131], v[224:227], v[208:211], v[128:131]\n"
    "v_mfma_f32_16x16x32_bf16 v[132:135], v[228:231], v[204:207], v[132:135]\n"
    "v_mfma_f32_16x16x32_bf16 v[132:135], v[232:235], v[208:211], v[132:135]\n"
    "v_mfma_f32_16x16x32_bf16 v[136:139], v[236:239], v[204:207], v[136:139]\n"
    "v_mfma_f32_16x16x32_bf16 v[136:139], v[240:243], v[208:211], v[136:139]\n"
    "s_setprio 0\n"
    "s_barrier\n"
    "; phase 56: step 34 slot 0 stage 2\n"
    "ds_read_b128 v[180:183], v244 offset:16384\n"
    "ds_read_b128 v[184:187], v245 offset:16384\n"
    "ds_read_b128 v[188:191], v244 offset:18432\n"
    "ds_read_b128 v[192:195], v245 offset:18432\n"
    "s_add_u32 m0, s51, 0xc000\n"
    "s_xor_b32 s82, s65, 0x2300000\n"
    "buffer_load_dwordx4 v251, s[40:43], s82 offen  lds\n"
    "ds_read_b128 v[196:199], v244 offset:24576\n"
    "ds_read_b128 v[200:203], v245 offset:24576\n"
    "ds_read_b128 v[204:207], v244 offset:26624\n"
    "ds_read_b128 v[208:211], v245 offset:26624\n"
    "s_add_u32 m0, s51, 0xe000\n"
    "s_xor_b32 s82, s65, 0x2380000\n"
    "buffer_load_dwordx4 v251, s[40:43], s82 offen  lds\n"
    "ds_read_b64_tr_b16 v[212:213], v246 offset:0\n"
    "ds_read_b64_tr_b16 v[214:215], v246 offset:1024\n"
    "ds_read_b64_tr_b16 v[216:217], v246 offset:8192\n"
    "ds_read_b64_tr_b16 v[218:219], v246 offset:9216\n"
    "s_cmp_lg_u32 s52, 1\n"
    "s_cbranch_scc1 .La88_%=\n"
    "s_xor_b32 s82, s67, 0x4300000\n"
    "s_add_u32 s82, s82, s53\n"
    "buffer_load_dword v9, v8, s[40:43], s82 offen \n"
    ".La88_%=:\n"
    "ds_read_b64_tr_b16 v[220:221], v247 offset:0\n"
    "ds_read_b64_tr_b16 v[222:223], v247 offset:1024\n"
    "ds_read_b64_tr_b16 v[224:225], v247 offset:8192\n"
    "ds_read_b64_tr_b16 v[226:227], v247 offset:9216\n"
    "ds_read_b64_tr_b16 v[228:229], v248 offset:0\n"
    "ds_read_b64_tr_b16 v[230:231], v248 offset:1024\n"
    "ds_read_b64_tr_b16 v[232:233], v248 offset:8192\n"
    "ds_read_b64_tr_b16 v[234:235], v248 offset:9216\n"
    "ds_read_b64_tr_b16 v[236:237], v249 offset:0\n"
    "ds_read_b64_tr_b16 v[238:239], v249 offset:1024\n"
    "ds_read_b64_tr_b16 v[240:241], v249 offset:8192\n"
    "ds_read_b64_tr_b16 v[242:243], v249 offset:9216\n"
    "s_waitcnt vmcnt(9) lgkmcnt(0)\n"
    "s_barrier\n"
    "s_setprio 1\n"
    "v_mfma_f32_16x16x32_bf16 v[140:143], v[212:215], v[180:183], v[140:143]\n"
    "v_mfma_f32_16x16x32_bf16 v[140:143], v[216:219], v[184:187], v[140:143]\n"
    "v_mfma_f32_16x16x32_bf16 v[144:147], v[220:223], v[180:183], v[144:147]\n"
    "v_mfma_f32_16x16x32_bf16 v[144:147], v[224:227], v[184:187], v[144:147]\n"
    "v_mfma_f32_16x16x32_bf16 v[148:151], v[228:231], v[180:183], v[148:151]\n"
    "v_mfma_f32_16x16x32_bf16 v[148:151], v[232:235], v[184:187], v[148:151]\n"
    "v_mfma_f32_16x16x32_bf16 v[152:155], v[236:239], v[180:183], v[152:155]\n"
    "v_mfma_f32_16x16x32_bf16 v[152:155], v[240:243], v[184:187], v[152:155]\n"
    "v_mfma_f32_16x16x32_bf16 v[156:159], v[212:215], v[188:191], v[156:159]\n"
    "v_mfma_f32_16x16x32_bf16 v[156:159], v[216:219], v[192:195], v[156:159]\n"
    "v_mfma_f32_16x16x32_bf16 v[160:163], v[220:223], v[188:191], v[160:163]\n"
    "v_mfma_f32_16x16x32_bf16 v[160:163], v[224:227], v[192:195], v[160:163]\n"
    "v_mfma_f32_16x16x32_bf16 v[164:167], v[228:231], v[188:191], v[164:167]\n"
    "v_mfma_f32_16x16x32_bf16 v[164:167], v[232:235], v[192:195], v[164:167]\n"
    "v_mfma_f32_16x16x32_bf16 v[168:171], v[236:239], v[188:191], v[168:171]\n"
    "v_mfma_f32_16x16x32_bf16 v[168:171], v[240:243], v[192:195], v[168:171]\n"
    "v_mfma_f32_16x16x32_bf16 v[172:175], v[212:215], v[196:199], v[172:175]\n"
    "v_mfma_f32_16x16x32_bf16 v[172:175], v[216:219], v[200:203], v[172:175]\n"
    "v_mfma_f32_16x16x32_bf16 v[176:179], v[220:223], v[196:199], v[176:179]\n"
    "v_mfma_f32_16x16x32_bf16 v[176:179], v[224:227], v[200:203], v[176:179]\n"
    "v_mfma_f32_16x16x32_bf16 v[16:19], v[228:231], v[196:199], v[16:19]\n"
    "v_mfma_f32_16x16x32_bf16 v[16:19], v[232:235], v[200:203], v[16:19]\n"
    "v_mfma_f32_16x16x32_bf16 v[24:27], v[236:239], v[196:199], v[24:27]\n"
    "v_mfma_f32_16x16x32_bf16 v[24:27], v[240:243], v[200:203], v[24:27]\n"
    "v_mfma_f32_16x16x32_bf16 v[32:35], v[212:215], v[204:207], v[32:35]\n"
    "v_mfma_f32_16x16x32_bf16 v[32:35], v[216:219], v[208:211], v[32:35]\n"
    "v_mfma_f32_16x16x32_bf16 v[40:43], v[220:223], v[204:207], v[40:43]\n"
    "v_mfma_f32_16x16x32_bf16 v[40:43], v[224:227], v[208:211], v[40:43]\n"
    "v_mfma_f32_16x16x32_bf16 v[12:15], v[228:231], v[204:207], v[12:15]\n"
    "v_mfma_f32_16x16x32_bf16 v[12:15], v[232:235], v[208:211], v[12:15]\n"
    "v_mfma_f32_16x16x32_bf16 v[48:51], v[236:239], v[204:207], v[48:51]\n"
    "v_mfma_f32_16x16x32_bf16 v[48:51], v[240:243], v[208:211], v[48:51]\n"
    "s_setprio 0\n"
    "s_barrier\n"
    "; phase 57: step 34 slot 1 stage 1\n"
    "ds_read_b64_tr_b16 v[212:213], v246 offset:16384\n"
    "ds_read_b64_tr_b16 v[214:215], v246 offset:17408\n"
    "s_xor_b32 s82, s54, 0x48000\n"
    "s_add_u32 s82, s82, 0x100000\n"
    "buffer_store_dwordx4 v[36:39], v252, s[44:47], s82 offen sc0 sc1\n"
    "ds_read_b64_tr_b16 v[216:217], v246 offset:24576\n"
    "ds_read_b64_tr_b16 v[218:219], v246 offset:25600\n"
    "s_add_u32 m0, s51, 0x0\n"
    "s_xor_b32 s82, s66, 0x2400000\n"
    "buffer_load_dwordx4 v251, s[40:43], s82 offen  lds\n"
    "ds_read_b64_tr_b16 v[220:221], v247 offset:16384\n"
    "ds_read_b64_tr_b16 v[222:223], v247 offset:17408\n"
    "s_add_u32 m0, s51, 0x2000\n"
    "s_xor_b32 s82, s66, 0x2480000\n"
    "buffer_load_dwordx4 v251, s[40:43], s82 offen  lds\n"
    "ds_read_b64_tr_b16 v[224:225], v247 offset:24576\n"
    "ds_read_b64_tr_b16 v[226:227], v247 offset:25600\n"
    "s_add_u32 m0, s50, 0x4000\n"
    "s_xor_b32 s82, s48, 0x201280\n"
    "buffer_load_dwordx4 v250, s[36:39], s82 offen  lds\n"
    "ds_read_b64_tr_b16 v[228:229], v248 offset:16384\n"
    "ds_read_b64_tr_b16 v[230:231], v248 offset:17408\n"
    "s_add_u32 m0, s50, 0x5000\n"
    "s_xor_b32 s82, s48, 0x281280\n"
    "buffer_load_dwordx4 v250, s[36:39], s82 offen  lds\n"
    "ds_read_b64_tr_b16 v[232:233], v248 offset:24576\n"
    "ds_read_b64_tr_b16 v[234:235], v248 offset:25600\n"
    "s_add_u32 m0, s50, 0x6000\n"
    "s_xor_b32 s82, s48, 0x1280\n"
    "buffer_load_dwordx4 v250, s[36:39], s82 offen  lds\n"
    "ds_read_b64_tr_b16 v[236:237], v249 offset:16384\n"
    "ds_read_b64_tr_b16 v[238:239], v249 offset:17408\n"
    "s_add_u32 m0, s50, 0x7000\n"
    "s_xor_b32 s82, s48, 0x81280\n"
    "buffer_load_dwordx4 v250, s[36:39], s82 offen  lds\n"
    "ds_read_b64_tr_b16 v[240:241], v249 offset:24576\n"
    "ds_read_b64_tr_b16 v[242:243], v249 offset:25600\n"
    "s_cmp_lg_u32 s52, 2\n"
    "s_cbranch_scc1 .La89_%=\n"
    "s_xor_b32 s82, s67, 0x4400000\n"
    "s_add_u32 s82, s82, s53\n"
    "buffer_load_dword v9, v8, s[40:43], s82 offen \n"
    ".La89_%=:\n"
    "s_waitcnt vmcnt(13) lgkmcnt(0)\n"
    "s_barrier\n"
    "s_setprio 1\n"
    "v_mfma_f32_16x16x32_bf16 v[76:79], v[212:215], v[180:183], v[76:79]\n"
    "v_mfma_f32_16x16x32_bf16 v[76:79], v[216:219], v[184:187], v[76:79]\n"
    "v_mfma_f32_16x16x32_bf16 v[80:83], v[220:223], v[180:183], v[80:83]\n"
    "v_mfma_f32_16x16x32_bf16 v[80:83], v[224:227], v[184:187], v[80:83]\n"
    "v_mfma_f32_16x16x32_bf16 v[84:87], v[228:231], v[180:183], v[84:87]\n"
    "v_mfma_f32_16x16x32_bf16 v[84:87], v[232:235], v[184:187], v[84:87]\n"
    "v_mfma_f32_16x16x32_bf16 v[88:91], v[236:239], v[180:183], v[88:91]\n"
    "v_mfma_f32_16x16x32_bf16 v[88:91], v[240:243], v[184:187], v[88:91]\n"
    "v_mfma_f32_16x16x32_bf16 v[92:95], v[212:215], v[188:191], v[92:95]\n"
    "v_mfma_f32_16x16x32_bf16 v[92:95], v[216:219], v[192:195], v[92:95]\n"
    "v_mfma_f32_16x16x32_bf16 v[96:99], v[220:223], v[188:191], v[96:99]\n"
    "v_mfma_f32_16x16x32_bf16 v[96:99], v[224:227], v[192:195], v[96:99]\n"
    "v_mfma_f32_16x16x32_bf16 v[100:103], v[228:231], v[188:191], v[100:103]\n"
    "v_mfma_f32_16x16x32_bf16 v[100:103], v[232:235], v[192:195], v[100:103]\n"
    "v_mfma_f32_16x16x32_bf16 v[104:107], v[236:239], v[188:191], v[104:107]\n"
    "v_mfma_f32_16x16x32_bf16 v[104:107], v[240:243], v[192:195], v[104:107]\n"
    "v_mfma_f32_16x16x32_bf16 v[108:111], v[212:215], v[196:199], v[108:111]\n"
    "v_mfma_f32_16x16x32_bf16 v[108:111], v[216:219], v[200:203], v[108:111]\n"
    "v_mfma_f32_16x16x32_bf16 v[112:115], v[220:223], v[196:199], v[112:115]\n"
    "v_mfma_f32_16x16x32_bf16 v[112:115], v[224:227], v[200:203], v[112:115]\n"
    "v_mfma_f32_16x16x32_bf16 v[116:119], v[228:231], v[196:199], v[116:119]\n"
    "v_mfma_f32_16x16x32_bf16 v[116:119], v[232:235], v[200:203], v[116:119]\n"
    "v_mfma_f32_16x16x32_bf16 v[120:123], v[236:239], v[196:199], v[120:123]\n"
    "v_mfma_f32_16x16x32_bf16 v[120:123], v[240:243], v[200:203], v[120:123]\n"
    "v_mfma_f32_16x16x32_bf16 v[124:127], v[212:215], v[204:207], v[124:127]\n"
    "v_mfma_f32_16x16x32_bf16 v[124:127], v[216:219], v[208:211], v[124:127]\n"
    "v_mfma_f32_16x16x32_bf16 v[128:131], v[220:223], v[204:207], v[128:131]\n"
    "v_mfma_f32_16x16x32_bf16 v[128:131], v[224:227], v[208:211], v[128:131]\n"
    "v_mfma_f32_16x16x32_bf16 v[132:135], v[228:231], v[204:207], v[132:135]\n"
    "v_mfma_f32_16x16x32_bf16 v[132:135], v[232:235], v[208:211], v[132:135]\n"
    "v_mfma_f32_16x16x32_bf16 v[136:139], v[236:239], v[204:207], v[136:139]\n"
    "v_mfma_f32_16x16x32_bf16 v[136:139], v[240:243], v[208:211], v[136:139]\n"
    "s_setprio 0\n"
    "s_barrier\n"
    "; phase 58: step 35 slot 0 stage 2\n"
    "ds_read_b128 v[180:183], v244 offset:32768\n"
    "ds_read_b128 v[184:187], v245 offset:32768\n"
    "ds_read_b128 v[188:191], v244 offset:34816\n"
    "ds_read_b128 v[192:195], v245 offset:34816\n"
    "s_add_u32 m0, s51, 0x4000\n"
    "s_xor_b32 s82, s65, 0x2400000\n"
    "buffer_load_dwordx4 v251, s[40:43], s82 offen  lds\n"
    "ds_read_b128 v[196:199], v244 offset:40960\n"
    "ds_read_b128 v[200:203], v245 offset:40960\n"
    "ds_read_b128 v[204:207], v244 offset:43008\n"
    "ds_read_b128 v[208:211], v245 offset:43008\n"
    "s_add_u32 m0, s51, 0x6000\n"
    "s_xor_b32 s82, s65, 0x2480000\n"
    "buffer_load_dwordx4 v251, s[40:43], s82 offen  lds\n"
    "ds_read_b64_tr_b16 v[212:213], v246 offset:32768\n"
    "ds_read_b64_tr_b16 v[214:215], v246 offset:33792\n"
    "ds_read_b64_tr_b16 v[216:217], v246 offset:40960\n"
    "ds_read_b64_tr_b16 v[218:219], v246 offset:41984\n"
    "s_cmp_lg_u32 s52, 3\n"
    "s_cbranch_scc1 .La90_%=\n"
    "s_xor_b32 s82, s67, 0x4500000\n"
    "s_add_u32 s82, s82, s53\n"
    "buffer_load_dword v9, v8, s[40:43], s82 offen \n"
    ".La90_%=:\n"
    "ds_read_b64_tr_b16 v[220:221], v247 offset:32768\n"
    "ds_read_b64_tr_b16 v[222:223], v247 offset:33792\n"
    "ds_read_b64_tr_b16 v[224:225], v247 offset:40960\n"
    "ds_read_b64_tr_b16 v[226:227], v247 offset:41984\n"
    "ds_read_b64_tr_b16 v[228:229], v248 offset:32768\n"
    "ds_read_b64_tr_b16 v[230:231], v248 offset:33792\n"
    "ds_read_b64_tr_b16 v[232:233], v248 offset:40960\n"
    "ds_read_b64_tr_b16 v[234:235], v248 offset:41984\n"
    "ds_read_b64_tr_b16 v[236:237], v249 offset:32768\n"
    "ds_read_b64_tr_b16 v[238:239], v249 offset:33792\n"
    "ds_read_b64_tr_b16 v[240:241], v249 offset:40960\n"
    "ds_read_b64_tr_b16 v[242:243], v249 offset:41984\n"
    "s_waitcnt vmcnt(9) lgkmcnt(0)\n"
    "s_barrier\n"
    "s_setprio 1\n"
    "v_mfma_f32_16x16x32_bf16 v[140:143], v[212:215], v[180:183], v[140:143]\n"
    "v_mfma_f32_16x16x32_bf16 v[140:143], v[216:219], v[184:187], v[140:143]\n"
    "v_mfma_f32_16x16x32_bf16 v[144:147], v[220:223], v[180:183], v[144:147]\n"
    "v_mfma_f32_16x16x32_bf16 v[144:147], v[224:227], v[184:187], v[144:147]\n"
    "v_mfma_f32_16x16x32_bf16 v[148:151], v[228:231], v[180:183], v[148:151]\n"
    "v_mfma_f32_16x16x32_bf16 v[148:151], v[232:235], v[184:187], v[148:151]\n"
    "v_mfma_f32_16x16x32_bf16 v[152:155], v[236:239], v[180:183], v[152:155]\n"
    "v_mfma_f32_16x16x32_bf16 v[152:155], v[240:243], v[184:187], v[152:155]\n"
    "v_mfma_f32_16x16x32_bf16 v[156:159], v[212:215], v[188:191], v[156:159]\n"
    "v_mfma_f32_16x16x32_bf16 v[156:159], v[216:219], v[192:195], v[156:159]\n"
    "v_mfma_f32_16x16x32_bf16 v[160:163], v[220:223], v[188:191], v[160:163]\n"
    "v_mfma_f32_16x16x32_bf16 v[160:163], v[224:227], v[192:195], v[160:163]\n"
    "v_mfma_f32_16x16x32_bf16 v[164:167], v[228:231], v[188:191], v[164:167]\n"
    "v_mfma_f32_16x16x32_bf16 v[164:167], v[232:235], v[192:195], v[164:167]\n"
    "v_mfma_f32_16x16x32_bf16 v[168:171], v[236:239], v[188:191], v[168:171]\n"
    "v_mfma_f32_16x16x32_bf16 v[168:171], v[240:243], v[192:195], v[168:171]\n"
    "v_mfma_f32_16x16x32_bf16 v[172:175], v[212:215], v[196:199], v[172:175]\n"
    "v_mfma_f32_16x16x32_bf16 v[172:175], v[216:219], v[200:203], v[172:175]\n"
    "v_mfma_f32_16x16x32_bf16 v[176:179], v[220:223], v[196:199], v[176:179]\n"
    "v_mfma_f32_16x16x32_bf16 v[176:179], v[224:227], v[200:203], v[176:179]\n"
    "v_mfma_f32_16x16x32_bf16 v[16:19], v[228:231], v[196:199], v[16:19]\n"
    "v_mfma_f32_16x16x32_bf16 v[16:19], v[232:235], v[200:203], v[16:19]\n"
    "v_mfma_f32_16x16x32_bf16 v[24:27], v[236:239], v[196:199], v[24:27]\n"
    "v_mfma_f32_16x16x32_bf16 v[24:27], v[240:243], v[200:203], v[24:27]\n"
    "v_mfma_f32_16x16x32_bf16 v[32:35], v[212:215], v[204:207], v[32:35]\n"
    "v_mfma_f32_16x16x32_bf16 v[32:35], v[216:219], v[208:211], v[32:35]\n"
    "v_mfma_f32_16x16x32_bf16 v[40:43], v[220:223], v[204:207], v[40:43]\n"
    "v_mfma_f32_16x16x32_bf16 v[40:43], v[224:227], v[208:211], v[40:43]\n"
    "v_mfma_f32_16x16x32_bf16 v[12:15], v[228:231], v[204:207], v[12:15]\n"
    "v_mfma_f32_16x16x32_bf16 v[12:15], v[232:235], v[208:211], v[12:15]\n"
    "v_mfma_f32_16x16x32_bf16 v[48:51], v[236:239], v[204:207], v[48:51]\n"
    "v_mfma_f32_16x16x32_bf16 v[48:51], v[240:243], v[208:211], v[48:51]\n"
    "s_setprio 0\n"
    "s_barrier\n"
    "; phase 59: step 35 slot 1 stage 1\n"
    "ds_read_b64_tr_b16 v[212:213], v246 offset:49152\n"
    "ds_read_b64_tr_b16 v[214:215], v246 offset:50176\n"
    "s_xor_b32 s82, s54, 0x26000\n"
    "s_add_u32 s82, s82, 0x100000\n"
    "buffer_store_dwordx4 v[44:47], v252, s[44:47], s82 offen sc0 sc1\n"
    "ds_read_b64_tr_b16 v[216:217], v246 offset:57344\n"
    "ds_read_b64_tr_b16 v[218:219], v246 offset:58368\n"
    "s_add_u32 m0, s51, 0x8000\n"
    "s_xor_b32 s82, s66, 0x2500000\n"
    "buffer_load_dwordx4 v251, s[40:43], s82 offen  lds\n"
    "ds_read_b64_tr_b16 v[220:221], v247 offset:49152\n"
    "ds_read_b64_tr_b16 v[222:223], v247 offset:50176\n"
    "s_add_u32 m0, s51, 0xa000\n"
    "s_xor_b32 s82, s66, 0x2580000\n"
    "buffer_load_dwordx4 v251, s[40:43], s82 offen  lds\n"
    "ds_read_b64_tr_b16 v[224:225], v247 offset:57344\n"
    "ds_read_b64_tr_b16 v[226:227], v247 offset:58368\n"
    "s_add_u32 m0, s50, 0x8000\n"
    "s_xor_b32 s82, s48, 0x201300\n"
    "buffer_load_dwordx4 v250, s[36:39], s82 offen  lds\n"
    "ds_read_b64_tr_b16 v[228:229], v248 offset:49152\n"
    "ds_read_b64_tr_b16 v[230:231], v248 offset:50176\n"
    "s_add_u32 m0, s50, 0x9000\n"
    "s_xor_b32 s82, s48, 0x281300\n"
    "buffer_load_dwordx4 v250, s[36:39], s82 offen  lds\n"
    "ds_read_b64_tr_b16 v[232:233], v248 offset:57344\n"
    "ds_read_b64_tr_b16 v[234:235], v248 offset:58368\n"
    "s_add_u32 m0, s50, 0xa000\n"
    "s_xor_b32 s82, s48, 0x1300\n"
    "buffer_load_dwordx4 v250, s[36:39], s82 offen  lds\n"
    "ds_read_b64_tr_b16 v[236:237], v249 offset:49152\n"
    "ds_read_b64_tr_b16 v[238:239], v249 offset:50176\n"
    "s_add_u32 m0, s50, 0xb000\n"
    "s_xor_b32 s82, s48, 0x81300\n"
    "buffer_load_dwordx4 v250, s[36:39], s82 offen  lds\n"
    "ds_read_b64_tr_b16 v[240:241], v249 offset:57344\n"
    "ds_read_b64_tr_b16 v[242:243], v249 offset:58368\n"
    "s_cmp_lg_u32 s52, 4\n"
    "s_cbranch_scc1 .La91_%=\n"
    "s_xor_b32 s82, s67, 0x4600000\n"
    "s_add_u32 s82, s82, s53\n"
    "buffer_load_dword v9, v8, s[40:43], s82 offen \n"
    ".La91_%=:\n"
    "s_waitcnt vmcnt(13) lgkmcnt(0)\n"
    "s_barrier\n"
    "s_setprio 1\n"
    "v_mfma_f32_16x16x32_bf16 v[76:79], v[212:215], v[180:183], v[76:79]\n"
    "v_mfma_f32_16x16x32_bf16 v[76:79], v[216:219], v[184:187], v[76:79]\n"
    "v_mfma_f32_16x16x32_bf16 v[80:83], v[220:223], v[180:183], v[80:83]\n"
    "v_mfma_f32_16x16x32_bf16 v[80:83], v[224:227], v[184:187], v[80:83]\n"
    "v_mfma_f32_16x16x32_bf16 v[84:87], v[228:231], v[180:183], v[84:87]\n"
    "v_mfma_f32_16x16x32_bf16 v[84:87], v[232:235], v[184:187], v[84:87]\n"
    "v_mfma_f32_16x16x32_bf16 v[88:91], v[236:239], v[180:183], v[88:91]\n"
    "v_mfma_f32_16x16x32_bf16 v[88:91], v[240:243], v[184:187], v[88:91]\n"
    "v_mfma_f32_16x16x32_bf16 v[92:95], v[212:215], v[188:191], v[92:95]\n"
    "v_mfma_f32_16x16x32_bf16 v[92:95], v[216:219], v[192:195], v[92:95]\n"
    "v_mfma_f32_16x16x32_bf16 v[96:99], v[220:223], v[188:191], v[96:99]\n"
    "v_mfma_f32_16x16x32_bf16 v[96:99], v[224:227], v[192:195], v[96:99]\n"
    "v_mfma_f32_16x16x32_bf16 v[100:103], v[228:231], v[188:191], v[100:103]\n"
    "v_mfma_f32_16x16x32_bf16 v[100:103], v[232:235], v[192:195], v[100:103]\n"
    "v_mfma_f32_16x16x32_bf16 v[104:107], v[236:239], v[188:191], v[104:107]\n"
    "v_mfma_f32_16x16x32_bf16 v[104:107], v[240:243], v[192:195], v[104:107]\n"
    "v_mfma_f32_16x16x32_bf16 v[108:111], v[212:215], v[196:199], v[108:111]\n"
    "v_mfma_f32_16x16x32_bf16 v[108:111], v[216:219], v[200:203], v[108:111]\n"
    "v_mfma_f32_16x16x32_bf16 v[112:115], v[220:223], v[196:199], v[112:115]\n"
    "v_mfma_f32_16x16x32_bf16 v[112:115], v[224:227], v[200:203], v[112:115]\n"
    "v_mfma_f32_16x16x32_bf16 v[116:119], v[228:231], v[196:199], v[116:119]\n"
    "v_mfma_f32_16x16x32_bf16 v[116:119], v[232:235], v[200:203], v[116:119]\n"
    "v_mfma_f32_16x16x32_bf16 v[120:123], v[236:239], v[196:199], v[120:123]\n"
    "v_mfma_f32_16x16x32_bf16 v[120:123], v[240:243], v[200:203], v[120:123]\n"
    "v_mfma_f32_16x16x32_bf16 v[124:127], v[212:215], v[204:207], v[124:127]\n"
    "v_mfma_f32_16x16x32_bf16 v[124:127], v[216:219], v[208:211], v[124:127]\n"
    "v_mfma_f32_16x16x32_bf16 v[128:131], v[220:223], v[204:207], v[128:131]\n"
    "v_mfma_f32_16x16x32_bf16 v[128:131], v[224:227], v[208:211], v[128:131]\n"
    "v_mfma_f32_16x16x32_bf16 v[132:135], v[228:231], v[204:207], v[132:135]\n"
    "v_mfma_f32_16x16x32_bf16 v[132:135], v[232:235], v[208:211], v[132:135]\n"
    "v_mfma_f32_16x16x32_bf16 v[136:139], v[236:239], v[204:207], v[136:139]\n"
    "v_mfma_f32_16x16x32_bf16 v[136:139], v[240:243], v[208:211], v[136:139]\n"
    "s_setprio 0\n"
    "s_barrier\n"
    "; phase 60: step 36 slot 0 stage 2\n"
    "ds_read_b128 v[180:183], v244 offset:0\n"
    "ds_read_b128 v[184:187], v245 offset:0\n"
    "ds_read_b128 v[188:191], v244 offset:2048\n"
    "ds_read_b128 v[192:195], v245 offset:2048\n"
    "s_add_u32 m0, s51, 0xc000\n"
    "s_xor_b32 s82, s65, 0x2500000\n"
    "buffer_load_dwordx4 v251, s[40:43], s82 offen  lds\n"
    "ds_read_b128 v[196:199], v244 offset:8192\n"
    "ds_read_b128 v[200:203], v245 offset:8192\n"
    "ds_read_b128 v[204:207], v244 offset:10240\n"
    "ds_read_b128 v[208:211], v245 offset:10240\n"
    "s_add_u32 m0, s51, 0xe000\n"
    "s_xor_b32 s82, s65, 0x2580000\n"
    "buffer_load_dwordx4 v251, s[40:43], s82 offen  lds\n"
    "ds_read_b64_tr_b16 v[212:213], v246 offset:0\n"
    "ds_read_b64_tr_b16 v[214:215], v246 offset:1024\n"
    "ds_read_b64_tr_b16 v[216:217], v246 offset:8192\n"
    "ds_read_b64_tr_b16 v[218:219], v246 offset:9216\n"
    "s_cmp_lg_u32 s52, 5\n"
    "s_cbranch_scc1 .La92_%=\n"
    "s_xor_b32 s82, s67, 0x4700000\n"
    "s_add_u32 s82, s82, s53\n"
    "buffer_load_dword v9, v8, s[40:43], s82 offen \n"
    ".La92_%=:\n"
    "ds_read_b64_tr_b16 v[220:221], v247 offset:0\n"
    "ds_read_b64_tr_b16 v[222:223], v247 offset:1024\n"
    "ds_read_b64_tr_b16 v[224:225], v247 offset:8192\n"
    "ds_read_b64_tr_b16 v[226:227], v247 offset:9216\n"
    "ds_read_b64_tr_b16 v[228:229], v248 offset:0\n"
    "ds_read_b64_tr_b16 v[230:231], v248 offset:1024\n"
    "ds_read_b64_tr_b16 v[232:233], v248 offset:8192\n"
    "ds_read_b64_tr_b16 v[234:235], v248 offset:9216\n"
    "ds_read_b64_tr_b16 v[236:237], v249 offset:0\n"
    "ds_read_b64_tr_b16 v[238:239], v249 offset:1024\n"
    "ds_read_b64_tr_b16 v[240:241], v249 offset:8192\n"
    "ds_read_b64_tr_b16 v[242:243], v249 offset:9216\n"
    "s_waitcnt vmcnt(9) lgkmcnt(0)\n"
    "s_barrier\n"
    "s_setprio 1\n"
    "v_mfma_f32_16x16x32_bf16 v[140:143], v[212:215], v[180:183], v[140:143]\n"
    "v_mfma_f32_16x16x32_bf16 v[140:143], v[216:219], v[184:187], v[140:143]\n"
    "v_mfma_f32_16x16x32_bf16 v[144:147], v[220:223], v[180:183], v[144:147]\n"
    "v_mfma_f32_16x16x32_bf16 v[144:147], v[224:227], v[184:187], v[144:147]\n"
    "v_mfma_f32_16x16x32_bf16 v[148:151], v[228:231], v[180:183], v[148:151]\n"
    "v_mfma_f32_16x16x32_bf16 v[148:151], v[232:235], v[184:187], v[148:151]\n"
    "v_mfma_f32_16x16x32_bf16 v[152:155], v[236:239], v[180:183], v[152:155]\n"
    "v_mfma_f32_16x16x32_bf16 v[152:155], v[240:243], v[184:187], v[152:155]\n"
    "v_mfma_f32_16x16x32_bf16 v[156:159], v[212:215], v[188:191], v[156:159]\n"
    "v_mfma_f32_16x16x32_bf16 v[156:159], v[216:219], v[192:195], v[156:159]\n"
    "v_mfma_f32_16x16x32_bf16 v[160:163], v[220:223], v[188:191], v[160:163]\n"
    "v_mfma_f32_16x16x32_bf16 v[160:163], v[224:227], v[192:195], v[160:163]\n"
    "v_mfma_f32_16x16x32_bf16 v[164:167], v[228:231], v[188:191], v[164:167]\n"
    "v_mfma_f32_16x16x32_bf16 v[164:167], v[232:235], v[192:195], v[164:167]\n"
    "v_mfma_f32_16x16x32_bf16 v[168:171], v[236:239], v[188:191], v[168:171]\n"
    "v_mfma_f32_16x16x32_bf16 v[168:171], v[240:243], v[192:195], v[168:171]\n"
    "v_mfma_f32_16x16x32_bf16 v[172:175], v[212:215], v[196:199], v[172:175]\n"
    "v_mfma_f32_16x16x32_bf16 v[172:175], v[216:219], v[200:203], v[172:175]\n"
    "v_mfma_f32_16x16x32_bf16 v[176:179], v[220:223], v[196:199], v[176:179]\n"
    "v_mfma_f32_16x16x32_bf16 v[176:179], v[224:227], v[200:203], v[176:179]\n"
    "v_mfma_f32_16x16x32_bf16 v[16:19], v[228:231], v[196:199], v[16:19]\n"
    "v_mfma_f32_16x16x32_bf16 v[16:19], v[232:235], v[200:203], v[16:19]\n"
    "v_mfma_f32_16x16x32_bf16 v[24:27], v[236:239], v[196:199], v[24:27]\n"
    "v_mfma_f32_16x16x32_bf16 v[24:27], v[240:243], v[200:203], v[24:27]\n"
    "v_mfma_f32_16x16x32_bf16 v[32:35], v[212:215], v[204:207], v[32:35]\n"
    "v_mfma_f32_16x16x32_bf16 v[32:35], v[216:219], v[208:211], v[32:35]\n"
    "v_mfma_f32_16x16x32_bf16 v[40:43], v[220:223], v[204:207], v[40:43]\n"
    "v_mfma_f32_16x16x32_bf16 v[40:43], v[224:227], v[208:211], v[40:43]\n"
    "v_mfma_f32_16x16x32_bf16 v[12:15], v[228:231], v[204:207], v[12:15]\n"
    "v_mfma_f32_16x16x32_bf16 v[12:15], v[232:235], v[208:211], v[12:15]\n"
    "v_mfma_f32_16x16x32_bf16 v[48:51], v[236:239], v[204:207], v[48:51]\n"
    "v_mfma_f32_16x16x32_bf16 v[48:51], v[240:243], v[208:211], v[48:51]\n"
    "s_setprio 0\n"
    "s_barrier\n"
    "; phase 61: step 36 slot 1 stage 1\n"
    "ds_read_b64_tr_b16 v[212:213], v246 offset:16384\n"
    "ds_read_b64_tr_b16 v[214:215], v246 offset:17408\n"
    "s_xor_b32 s82, s54, 0x24000\n"
    "s_add_u32 s82, s82, 0x100000\n"
    "buffer_store_dwordx4 v[52:55], v252, s[44:47], s82 offen sc0 sc1\n"
    "ds_read_b64_tr_b16 v[216:217], v246 offset:24576\n"
    "ds_read_b64_tr_b16 v[218:219], v246 offset:25600\n"
    "s_add_u32 m0, s51, 0x0\n"
    "s_xor_b32 s82, s66, 0x2600000\n"
    "buffer_load_dwordx4 v251, s[40:43], s82 offen  lds\n"
    "ds_read_b64_tr_b16 v[220:221], v247 offset:16384\n"
    "ds_read_b64_tr_b16 v[222:223], v247 offset:17408\n"
    "s_add_u32 m0, s51, 0x2000\n"
    "s_xor_b32 s82, s66, 0x2680000\n"
    "buffer_load_dwordx4 v251, s[40:43], s82 offen  lds\n"
    "ds_read_b64_tr_b16 v[224:225], v247 offset:24576\n"
    "ds_read_b64_tr_b16 v[226:227], v247 offset:25600\n"
    "s_add_u32 m0, s50, 0x0\n"
    "s_xor_b32 s82, s48, 0x201380\n"
    "buffer_load_dwordx4 v250, s[36:39], s82 offen  lds\n"
    "ds_read_b64_tr_b16 v[228:229], v248 offset:16384\n"
    "ds_read_b64_tr_b16 v[230:231], v248 offset:17408\n"
    "s_add_u32 m0, s50, 0x1000\n"
    "s_xor_b32 s82, s48, 0x281380\n"
    "buffer_load_dwordx4 v250, s[36:39], s82 offen  lds\n"
    "ds_read_b64_tr_b16 v[232:233], v248 offset:24576\n"
    "ds_read_b64_tr_b16 v[234:235], v248 offset:25600\n"
    "s_add_u32 m0, s50, 0x2000\n"
    "s_xor_b32 s82, s48, 0x1380\n"
    "buffer_load_dwordx4 v250, s[36:39], s82 offen  lds\n"
    "ds_read_b64_tr_b16 v[236:237], v249 offset:16384\n"
    "ds_read_b64_tr_b16 v[238:239], v249 offset:17408\n"
    "s_add_u32 m0, s50, 0x3000\n"
    "s_xor_b32 s82, s48, 0x81380\n"
    "buffer_load_dwordx4 v250, s[36:39], s82 offen  lds\n"
    "ds_read_b64_tr_b16 v[240:241], v249 offset:24576\n"
    "ds_read_b64_tr_b16 v[242:243], v249 offset:25600\n"
    "s_waitcnt vmcnt(13) lgkmcnt(0)\n"
    "s_barrier\n"
    "s_setprio 1\n"
    "v_mfma_f32_16x16x32_bf16 v[76:79], v[212:215], v[180:183], v[76:79]\n"
    "v_mfma_f32_16x16x32_bf16 v[76:79], v[216:219], v[184:187], v[76:79]\n"
    "v_mfma_f32_16x16x32_bf16 v[80:83], v[220:223], v[180:183], v[80:83]\n"
    "v_mfma_f32_16x16x32_bf16 v[80:83], v[224:227], v[184:187], v[80:83]\n"
    "v_mfma_f32_16x16x32_bf16 v[84:87], v[228:231], v[180:183], v[84:87]\n"
    "v_mfma_f32_16x16x32_bf16 v[84:87], v[232:235], v[184:187], v[84:87]\n"
    "v_mfma_f32_16x16x32_bf16 v[88:91], v[236:239], v[180:183], v[88:91]\n"
    "v_mfma_f32_16x16x32_bf16 v[88:91], v[240:243], v[184:187], v[88:91]\n"
    "v_mfma_f32_16x16x32_bf16 v[92:95], v[212:215], v[188:191], v[92:95]\n"
    "v_mfma_f32_16x16x32_bf16 v[92:95], v[216:219], v[192:195], v[92:95]\n"
    "v_mfma_f32_16x16x32_bf16 v[96:99], v[220:223], v[188:191], v[96:99]\n"
    "v_mfma_f32_16x16x32_bf16 v[96:99], v[224:227], v[192:195], v[96:99]\n"
    "v_mfma_f32_16x16x32_bf16 v[100:103], v[228:231], v[188:191], v[100:103]\n"
    "v_mfma_f32_16x16x32_bf16 v[100:103], v[232:235], v[192:195], v[100:103]\n"
    "v_mfma_f32_16x16x32_bf16 v[104:107], v[236:239], v[188:191], v[104:107]\n"
    "v_mfma_f32_16x16x32_bf16 v[104:107], v[240:243], v[192:195], v[104:107]\n"
    "v_mfma_f32_16x16x32_bf16 v[108:111], v[212:215], v[196:199], v[108:111]\n"
    "v_mfma_f32_16x16x32_bf16 v[108:111], v[216:219], v[200:203], v[108:111]\n"
    "v_mfma_f32_16x16x32_bf16 v[112:115], v[220:223], v[196:199], v[112:115]\n"
    "v_mfma_f32_16x16x32_bf16 v[112:115], v[224:227], v[200:203], v[112:115]\n"
    "v_mfma_f32_16x16x32_bf16 v[116:119], v[228:231], v[196:199], v[116:119]\n"
    "v_mfma_f32_16x16x32_bf16 v[116:119], v[232:235], v[200:203], v[116:119]\n"
    "v_mfma_f32_16x16x32_bf16 v[120:123], v[236:239], v[196:199], v[120:123]\n"
    "v_mfma_f32_16x16x32_bf16 v[120:123], v[240:243], v[200:203], v[120:123]\n"
    "v_mfma_f32_16x16x32_bf16 v[124:127], v[212:215], v[204:207], v[124:127]\n"
    "v_mfma_f32_16x16x32_bf16 v[124:127], v[216:219], v[208:211], v[124:127]\n"
    "v_mfma_f32_16x16x32_bf16 v[128:131], v[220:223], v[204:207], v[128:131]\n"
    "v_mfma_f32_16x16x32_bf16 v[128:131], v[224:227], v[208:211], v[128:131]\n"
    "v_mfma_f32_16x16x32_bf16 v[132:135], v[228:231], v[204:207], v[132:135]\n"
    "v_mfma_f32_16x16x32_bf16 v[132:135], v[232:235], v[208:211], v[132:135]\n"
    "v_mfma_f32_16x16x32_bf16 v[136:139], v[236:239], v[204:207], v[136:139]\n"
    "v_mfma_f32_16x16x32_bf16 v[136:139], v[240:243], v[208:211], v[136:139]\n"
    "s_setprio 0\n"
    "s_barrier\n"
    "; phase 62: step 37 slot 0 stage 2\n"
    "ds_read_b128 v[180:183], v244 offset:16384\n"
    "ds_read_b128 v[184:187], v245 offset:16384\n"
    "ds_read_b128 v[188:191], v244 offset:18432\n"
    "ds_read_b128 v[192:195], v245 offset:18432\n"
    "s_add_u32 m0, s51, 0x4000\n"
    "s_xor_b32 s82, s65, 0x2600000\n"
    "buffer_load_dwordx4 v251, s[40:43], s82 offen  lds\n"
    "ds_read_b128 v[196:199], v244 offset:24576\n"
    "ds_read_b128 v[200:203], v245 offset:24576\n"
    "ds_read_b128 v[204:207], v244 offset:26624\n"
    "ds_read_b128 v[208:211], v245 offset:26624\n"
    "s_add_u32 m0, s51, 0x6000\n"
    "s_xor_b32 s82, s65, 0x2680000\n"
    "buffer_load_dwordx4 v251, s[40:43], s82 offen  lds\n"
    "ds_read_b64_tr_b16 v[212:213], v246 offset:32768\n"
    "ds_read_b64_tr_b16 v[214:215], v246 offset:33792\n"
    "ds_read_b64_tr_b16 v[216:217], v246 offset:40960\n"
    "ds_read_b64_tr_b16 v[218:219], v246 offset:41984\n"
    "s_cmp_lg_u32 s52, 6\n"
    "s_cbranch_scc1 .La93_%=\n"
    "s_xor_b32 s82, s67, 0x4e00000\n"
    "s_add_u32 s82, s82, s53\n"
    "buffer_load_dword v9, v8, s[40:43], s82 offen \n"
    ".La93_%=:\n"
    "ds_read_b64_tr_b16 v[220:221], v247 offset:32768\n"
    "ds_read_b64_tr_b16 v[222:223], v247 offset:33792\n"
    "ds_read_b64_tr_b16 v[224:225], v247 offset:40960\n"
    "ds_read_b64_tr_b16 v[226:227], v247 offset:41984\n"
    "ds_read_b64_tr_b16 v[228:229], v248 offset:32768\n"
    "ds_read_b64_tr_b16 v[230:231], v248 offset:33792\n"
    "ds_read_b64_tr_b16 v[232:233], v248 offset:40960\n"
    "ds_read_b64_tr_b16 v[234:235], v248 offset:41984\n"
    "ds_read_b64_tr_b16 v[236:237], v249 offset:32768\n"
    "ds_read_b64_tr_b16 v[238:239], v249 offset:33792\n"
    "ds_read_b64_tr_b16 v[240:241], v249 offset:40960\n"
    "ds_read_b64_tr_b16 v[242:243], v249 offset:41984\n"
    "s_waitcnt vmcnt(9) lgkmcnt(0)\n"
    "s_barrier\n"
    "s_setprio 1\n"
    "v_mfma_f32_16x16x32_bf16 v[140:143], v[212:215], v[180:183], v[140:143]\n"
    "v_mfma_f32_16x16x32_bf16 v[140:143], v[216:219], v[184:187], v[140:143]\n"
    "v_mfma_f32_16x16x32_bf16 v[144:147], v[220:223], v[180:183], v[144:147]\n"
    "v_mfma_f32_16x16x32_bf16 v[144:147], v[224:227], v[184:187], v[144:147]\n"
    "v_mfma_f32_16x16x32_bf16 v[148:151], v[228:231], v[180:183], v[148:151]\n"
    "v_mfma_f32_16x16x32_bf16 v[148:151], v[232:235], v[184:187], v[148:151]\n"
    "v_mfma_f32_16x16x32_bf16 v[152:155], v[236:239], v[180:183], v[152:155]\n"
    "v_mfma_f32_16x16x32_bf16 v[152:155], v[240:243], v[184:187], v[152:155]\n"
    "v_mfma_f32_16x16x32_bf16 v[156:159], v[212:215], v[188:191], v[156:159]\n"
    "v_mfma_f32_16x16x32_bf16 v[156:159], v[216:219], v[192:195], v[156:159]\n"
    "v_mfma_f32_16x16x32_bf16 v[160:163], v[220:223], v[188:191], v[160:163]\n"
    "v_mfma_f32_16x16x32_bf16 v[160:163], v[224:227], v[192:195], v[160:163]\n"
    "v_mfma_f32_16x16x32_bf16 v[164:167], v[228:231], v[188:191], v[164:167]\n"
    "v_mfma_f32_16x16x32_bf16 v[164:167], v[232:235], v[192:195], v[164:167]\n"
    "v_mfma_f32_16x16x32_bf16 v[168:171], v[236:239], v[188:191], v[168:171]\n"
    "v_mfma_f32_16x16x32_bf16 v[168:171], v[240:243], v[192:195], v[168:171]\n"
    "v_mfma_f32_16x16x32_bf16 v[172:175], v[212:215], v[196:199], v[172:175]\n"
    "v_mfma_f32_16x16x32_bf16 v[172:175], v[216:219], v[200:203], v[172:175]\n"
    "v_mfma_f32_16x16x32_bf16 v[176:179], v[220:223], v[196:199], v[176:179]\n"
    "v_mfma_f32_16x16x32_bf16 v[176:179], v[224:227], v[200:203], v[176:179]\n"
    "v_mfma_f32_16x16x32_bf16 v[16:19], v[228:231], v[196:199], v[16:19]\n"
    "v_mfma_f32_16x16x32_bf16 v[16:19], v[232:235], v[200:203], v[16:19]\n"
    "v_mfma_f32_16x16x32_bf16 v[24:27], v[236:239], v[196:199], v[24:27]\n"
    "v_mfma_f32_16x16x32_bf16 v[24:27], v[240:243], v[200:203], v[24:27]\n"
    "v_mfma_f32_16x16x32_bf16 v[32:35], v[212:215], v[204:207], v[32:35]\n"
    "v_mfma_f32_16x16x32_bf16 v[32:35], v[216:219], v[208:211], v[32:35]\n"
    "v_mfma_f32_16x16x32_bf16 v[40:43], v[220:223], v[204:207], v[40:43]\n"
    "v_mfma_f32_16x16x32_bf16 v[40:43], v[224:227], v[208:211], v[40:43]\n"
    "v_mfma_f32_16x16x32_bf16 v[12:15], v[228:231], v[204:207], v[12:15]\n"
    "v_mfma_f32_16x16x32_bf16 v[12:15], v[232:235], v[208:211], v[12:15]\n"
    "v_mfma_f32_16x16x32_bf16 v[48:51], v[236:239], v[204:207], v[48:51]\n"
    "v_mfma_f32_16x16x32_bf16 v[48:51], v[240:243], v[208:211], v[48:51]\n"
    "s_setprio 0\n"
    "s_waitcnt vmcnt(8)\n"
    "s_barrier\n"
    "; phase 63: step 37 slot 1 stage 1\n"
    "ds_read_b64_tr_b16 v[212:213], v246 offset:49152\n"
    "ds_read_b64_tr_b16 v[214:215], v246 offset:50176\n"
    "s_mov_b32 s82, 0x2100000\n"
    "buffer_store_dwordx2 v[10:11], v253, s[44:47], s82 offen sc0 sc1\n"
    "ds_read_b64_tr_b16 v[216:217], v246 offset:57344\n"
    "ds_read_b64_tr_b16 v[218:219], v246 offset:58368\n"
    "s_add_u32 m0, s51, 0x8000\n"
    "s_xor_b32 s82, s66, 0x2700000\n"
    "buffer_load_dwordx4 v251, s[40:43], s82 offen  lds\n"
    "ds_read_b64_tr_b16 v[220:221], v247 offset:49152\n"
    "ds_read_b64_tr_b16 v[222:223], v247 offset:50176\n"
    "s_add_u32 m0, s51, 0xa000\n"
    "s_xor_b32 s82, s66, 0x2780000\n"
    "buffer_load_dwordx4 v251, s[40:43], s82 offen  lds\n"
    "ds_read_b64_tr_b16 v[224:225], v247 offset:57344\n"
    "ds_read_b64_tr_b16 v[226:227], v247 offset:58368\n"
    "s_add_u32 m0, s50, 0x4000\n"
    "s_xor_b32 s82, s48, 0x201400\n"
    "buffer_load_dwordx4 v250, s[36:39], s82 offen  lds\n"
    "ds_read_b64_tr_b16 v[228:229], v248 offset:49152\n"
    "ds_read_b64_tr_b16 v[230:231], v248 offset:50176\n"
    "s_add_u32 m0, s50, 0x5000\n"
    "s_xor_b32 s82, s48, 0x281400\n"
    "buffer_load_dwordx4 v250, s[36:39], s82 offen  lds\n"
    "ds_read_b64_tr_b16 v[232:233], v248 offset:57344\n"
    "ds_read_b64_tr_b16 v[234:235], v248 offset:58368\n"
    "s_add_u32 m0, s50, 0x6000\n"
    "s_xor_b32 s82, s48, 0x1400\n"
    "buffer_load_dwordx4 v250, s[36:39], s82 offen  lds\n"
    "ds_read_b64_tr_b16 v[236:237], v249 offset:49152\n"
    "ds_read_b64_tr_b16 v[238:239], v249 offset:50176\n"
    "s_add_u32 m0, s50, 0x7000\n"
    "s_xor_b32 s82, s48, 0x81400\n"
    "buffer_load_dwordx4 v250, s[36:39], s82 offen  lds\n"
    "ds_read_b64_tr_b16 v[240:241], v249 offset:57344\n"
    "ds_read_b64_tr_b16 v[242:243], v249 offset:58368\n"
    "s_cmp_lg_u32 s52, 7\n"
    "s_cbranch_scc1 .La94_%=\n"
    "s_xor_b32 s82, s67, 0x4f00000\n"
    "s_add_u32 s82, s82, s53\n"
    "buffer_load_dword v9, v8, s[40:43], s82 offen \n"
    ".La94_%=:\n"
    "s_waitcnt vmcnt(13) lgkmcnt(0)\n"
    "s_barrier\n"
    "s_setprio 1\n"
    "v_mfma_f32_16x16x32_bf16 v[76:79], v[212:215], v[180:183], v[76:79]\n"
    "v_mfma_f32_16x16x32_bf16 v[76:79], v[216:219], v[184:187], v[76:79]\n"
    "v_mfma_f32_16x16x32_bf16 v[80:83], v[220:223], v[180:183], v[80:83]\n"
    "v_mfma_f32_16x16x32_bf16 v[80:83], v[224:227], v[184:187], v[80:83]\n"
    "v_mfma_f32_16x16x32_bf16 v[84:87], v[228:231], v[180:183], v[84:87]\n"
    "v_mfma_f32_16x16x32_bf16 v[84:87], v[232:235], v[184:187], v[84:87]\n"
    "v_mfma_f32_16x16x32_bf16 v[88:91], v[236:239], v[180:183], v[88:91]\n"
    "v_mfma_f32_16x16x32_bf16 v[88:91], v[240:243], v[184:187], v[88:91]\n"
    "v_mfma_f32_16x16x32_bf16 v[92:95], v[212:215], v[188:191], v[92:95]\n"
    "v_mfma_f32_16x16x32_bf16 v[92:95], v[216:219], v[192:195], v[92:95]\n"
    "v_mfma_f32_16x16x32_bf16 v[96:99], v[220:223], v[188:191], v[96:99]\n"
    "v_mfma_f32_16x16x32_bf16 v[96:99], v[224:227], v[192:195], v[96:99]\n"
    "v_mfma_f32_16x16x32_bf16 v[100:103], v[228:231], v[188:191], v[100:103]\n"
    "v_mfma_f32_16x16x32_bf16 v[100:103], v[232:235], v[192:195], v[100:103]\n"
    "v_mfma_f32_16x16x32_bf16 v[104:107], v[236:239], v[188:191], v[104:107]\n"
    "v_mfma_f32_16x16x32_bf16 v[104:107], v[240:243], v[192:195], v[104:107]\n"
    "v_mfma_f32_16x16x32_bf16 v[108:111], v[212:215], v[196:199], v[108:111]\n"
    "v_mfma_f32_16x16x32_bf16 v[108:111], v[216:219], v[200:203], v[108:111]\n"
    "v_mfma_f32_16x16x32_bf16 v[112:115], v[220:223], v[196:199], v[112:115]\n"
    "v_mfma_f32_16x16x32_bf16 v[112:115], v[224:227], v[200:203], v[112:115]\n"
    "v_mfma_f32_16x16x32_bf16 v[116:119], v[228:231], v[196:199], v[116:119]\n"
    "v_mfma_f32_16x16x32_bf16 v[116:119], v[232:235], v[200:203], v[116:119]\n"
    "v_mfma_f32_16x16x32_bf16 v[120:123], v[236:239], v[196:199], v[120:123]\n"
    "v_mfma_f32_16x16x32_bf16 v[120:123], v[240:243], v[200:203], v[120:123]\n"
    "v_mfma_f32_16x16x32_bf16 v[124:127], v[212:215], v[204:207], v[124:127]\n"
    "v_mfma_f32_16x16x32_bf16 v[124:127], v[216:219], v[208:211], v[124:127]\n"
    "v_mfma_f32_16x16x32_bf16 v[128:131], v[220:223], v[204:207], v[128:131]\n"
    "v_mfma_f32_16x16x32_bf16 v[128:131], v[224:227], v[208:211], v[128:131]\n"
    "v_mfma_f32_16x16x32_bf16 v[132:135], v[228:231], v[204:207], v[132:135]\n"
    "v_mfma_f32_16x16x32_bf16 v[132:135], v[232:235], v[208:211], v[132:135]\n"
    "v_mfma_f32_16x16x32_bf16 v[136:139], v[236:239], v[204:207], v[136:139]\n"
    "v_mfma_f32_16x16x32_bf16 v[136:139], v[240:243], v[208:211], v[136:139]\n"
    "s_setprio 0\n"
    "s_barrier\n"
    "; phase 64: step 38 slot 0 stage 2\n"
    "ds_read_b128 v[180:183], v244 offset:32768\n"
    "ds_read_b128 v[184:187], v245 offset:32768\n"
    "ds_read_b128 v[188:191], v244 offset:34816\n"
    "ds_read_b128 v[192:195], v245 offset:34816\n"
    "s_add_u32 s82, s56, 0x0\n"
    "buffer_load_dword v6, v254, s[44:47], s82 offen sc0 sc1\n"
    "s_lshr_b32 s83, s64, 8\n"
    "s_and_b32 s82, s83, 0x38\n"
    "s_load_dwordx2 s[72:73], s[60:61], s82\n"
    "s_load_dwordx2 s[16:17], s[60:61], s82 offset:0x40\n"
    "s_and_b32 s83, s64, 0x700\n"
    "s_add_u32 s80, s57, s83\n"
    "ds_read_b128 v[196:199], v244 offset:40960\n"
    "ds_read_b128 v[200:203], v245 offset:40960\n"
    "ds_read_b128 v[204:207], v244 offset:43008\n"
    "ds_read_b128 v[208:211], v245 offset:43008\n"
    "s_add_u32 m0, s51, 0xc000\n"
    "s_xor_b32 s82, s65, 0x2700000\n"
    "buffer_load_dwordx4 v251, s[40:43], s82 offen  lds\n"
    "ds_read_b64_tr_b16 v[212:213], v246 offset:0\n"
    "ds_read_b64_tr_b16 v[214:215], v246 offset:1024\n"
    "ds_read_b64_tr_b16 v[216:217], v246 offset:8192\n"
    "ds_read_b64_tr_b16 v[218:219], v246 offset:9216\n"
    "s_add_u32 m0, s51, 0xe000\n"
    "s_xor_b32 s82, s65, 0x2780000\n"
    "buffer_load_dwordx4 v251, s[40:43], s82 offen  lds\n"
    "ds_read_b64_tr_b16 v[220:221], v247 offset:0\n"
    "ds_read_b64_tr_b16 v[222:223], v247 offset:1024\n"
    "ds_read_b64_tr_b16 v[224:225], v247 offset:8192\n"
    "ds_read_b64_tr_b16 v[226:227], v247 offset:9216\n"
    "s_cmp_lg_u32 s52, 0\n"
    "s_cbranch_scc1 .La95_%=\n"
    "s_xor_b32 s82, s67, 0x5000000\n"
    "s_add_u32 s82, s82, s53\n"
    "buffer_load_dword v9, v8, s[40:43], s82 offen \n"
    ".La95_%=:\n"
    "ds_read_b64_tr_b16 v[228:229], v248 offset:0\n"
    "ds_read_b64_tr_b16 v[230:231], v248 offset:1024\n"
    "ds_read_b64_tr_b16 v[232:233], v248 offset:8192\n"
    "ds_read_b64_tr_b16 v[234:235], v248 offset:9216\n"
    "ds_read_b64_tr_b16 v[236:237], v249 offset:0\n"
    "ds_read_b64_tr_b16 v[238:239], v249 offset:1024\n"
    "ds_read_b64_tr_b16 v[240:241], v249 offset:8192\n"
    "ds_read_b64_tr_b16 v[242:243], v249 offset:9216\n"
    "s_waitcnt vmcnt(10) lgkmcnt(0)\n"
    "s_barrier\n"
    "s_setprio 1\n"
    "v_mfma_f32_16x16x32_bf16 v[140:143], v[212:215], v[180:183], v[140:143]\n"
    "v_mfma_f32_16x16x32_bf16 v[140:143], v[216:219], v[184:187], v[140:143]\n"
    "v_mfma_f32_16x16x32_bf16 v[144:147], v[220:223], v[180:183], v[144:147]\n"
    "v_mfma_f32_16x16x32_bf16 v[144:147], v[224:227], v[184:187], v[144:147]\n"
    "v_mfma_f32_16x16x32_bf16 v[148:151], v[228:231], v[180:183], v[148:151]\n"
    "v_mfma_f32_16x16x32_bf16 v[148:151], v[232:235], v[184:187], v[148:151]\n"
    "v_mfma_f32_16x16x32_bf16 v[152:155], v[236:239], v[180:183], v[152:155]\n"
    "v_mfma_f32_16x16x32_bf16 v[152:155], v[240:243], v[184:187], v[152:155]\n"
    "v_mfma_f32_16x16x32_bf16 v[156:159], v[212:215], v[188:191], v[156:159]\n"
    "v_mfma_f32_16x16x32_bf16 v[156:159], v[216:219], v[192:195], v[156:159]\n"
    "v_mfma_f32_16x16x32_bf16 v[160:163], v[220:223], v[188:191], v[160:163]\n"
    "v_mfma_f32_16x16x32_bf16 v[160:163], v[224:227], v[192:195], v[160:163]\n"
    "v_mfma_f32_16x16x32_bf16 v[164:167], v[228:231], v[188:191], v[164:167]\n"
    "v_mfma_f32_16x16x32_bf16 v[164:167], v[232:235], v[192:195], v[164:167]\n"
    "v_mfma_f32_16x16x32_bf16 v[168:171], v[236:239], v[188:191], v[168:171]\n"
    "v_mfma_f32_16x16x32_bf16 v[168:171], v[240:243], v[192:195], v[168:171]\n"
    "v_mfma_f32_16x16x32_bf16 v[172:175], v[212:215], v[196:199], v[172:175]\n"
    "v_mfma_f32_16x16x32_bf16 v[172:175], v[216:219], v[200:203], v[172:175]\n"
    "v_mfma_f32_16x16x32_bf16 v[176:179], v[220:223], v[196:199], v[176:179]\n"
    "v_mfma_f32_16x16x32_bf16 v[176:179], v[224:227], v[200:203], v[176:179]\n"
    "v_mfma_f32_16x16x32_bf16 v[16:19], v[228:231], v[196:199], v[16:19]\n"
    "v_mfma_f32_16x16x32_bf16 v[16:19], v[232:235], v[200:203], v[16:19]\n"
    "v_mfma_f32_16x16x32_bf16 v[24:27], v[236:239], v[196:199], v[24:27]\n"
    "v_mfma_f32_16x16x32_bf16 v[24:27], v[240:243], v[200:203], v[24:27]\n"
    "v_mfma_f32_16x16x32_bf16 v[32:35], v[212:215], v[204:207], v[32:35]\n"
    "v_mfma_f32_16x16x32_bf16 v[32:35], v[216:219], v[208:211], v[32:35]\n"
    "v_mfma_f32_16x16x32_bf16 v[40:43], v[220:223], v[204:207], v[40:43]\n"
    "v_mfma_f32_16x16x32_bf16 v[40:43], v[224:227], v[208:211], v[40:43]\n"
    "v_mfma_f32_16x16x32_bf16 v[12:15], v[228:231], v[204:207], v[12:15]\n"
    "v_mfma_f32_16x16x32_bf16 v[12:15], v[232:235], v[208:211], v[12:15]\n"
    "v_mfma_f32_16x16x32_bf16 v[48:51], v[236:239], v[204:207], v[48:51]\n"
    "v_mfma_f32_16x16x32_bf16 v[48:51], v[240:243], v[208:211], v[48:51]\n"
    "s_setprio 0\n"
    "s_waitcnt vmcnt(2)\n"
    "s_barrier\n"
    "; phase 65: step 38 slot 1 stage 1\n"
    "v_cmp_ne_u32 vcc, s90, v6\n"
    "s_cbranch_vccz .La96_%=\n"
    "s_memrealtime s[86:87]\n"
    "s_add_u32 s82, s56, 0x0\n"
    ".La97_%=:\n"
    "s_sleep 2\n"
    "buffer_load_dword v6, v254, s[44:47], s82 offen sc0 sc1\n"
    "s_memrealtime s[88:89]\n"
    "s_waitcnt vmcnt(0) lgkmcnt(0)\n"
    "v_cmp_ne_u32 vcc, s90, v6\n"
    "s_cbranch_vccz .La96_%=\n"
    "s_sub_u32 s83, s88, s86\n"
    "s_cmp_lt_u32 s83, 0x2710\n"
    "s_cbranch_scc1 .La97_%=\n"
    "s_sleep 64\n"
    "buffer_load_dword v7, off, s[44:47], 0 sc0 sc1\n"
    "s_waitcnt vmcnt(0)\n"
    "v_cmp_eq_u32 vcc, s90, v7\n"
    "s_cbranch_vccnz .La98_%=\n"
    "s_cmp_lt_u32 s83, 0x2faf080\n"
    "s_cbranch_scc1 .La97_%=\n"
    ".La98_%=:\n"
    "v_mov_b32 v7, s90\n"
    "v_mov_b32 v60, 0x7fc00000\n"
    "v_mov_b32 v61, 0x7fc00000\n"
    "v_mov_b32 v62, 0x7fc00000\n"
    "v_mov_b32 v63, 0x7fc00000\n"
    "v_mov_b32 v64, 0x7fc00000\n"
    "v_mov_b32 v65, 0x7fc00000\n"
    "v_mov_b32 v66, 0x7fc00000\n"
    "v_mov_b32 v67, 0x7fc00000\n"
    "v_mov_b32 v68, 0x7fc00000\n"
    "v_mov_b32 v69, 0x7fc00000\n"
    "v_mov_b32 v70, 0x7fc00000\n"
    "v_mov_b32 v71, 0x7fc00000\n"
    "v_mov_b32 v72, 0x7fc00000\n"
    "v_mov_b32 v73, 0x7fc00000\n"
    "v_mov_b32 v74, 0x7fc00000\n"
    "v_mov_b32 v75, 0x7fc00000\n"
    "buffer_store_dword v7, off, s[44:47], 0 sc0 sc1\n"
    "s_waitcnt vmcnt(0)\n"
    ".La96_%=:\n"
    "ds_read_b64_tr_b16 v[212:213], v246 offset:16384\n"
    "s_add_u32 s82, s54, 0x104000\n"
    "buffer_load_dwordx4 v[56:59], v252, s[44:47], s82 offen sc0 sc1\n"
    "ds_read_b64_tr_b16 v[214:215], v246 offset:17408\n"
    "s_add_u32 s82, s54, 0x108000\n"
    "buffer_load_dwordx4 v[20:23], v252, s[44:47], s82 offen sc0 sc1\n"
    "ds_read_b64_tr_b16 v[216:217], v246 offset:24576\n"
    "s_add_u32 m0, s51, 0x0\n"
    "s_xor_b32 s82, s66, 0x2800000\n"
    "buffer_load_dwordx4 v251, s[40:43], s82 offen  lds\n"
    "ds_read_b64_tr_b16 v[218:219], v246 offset:25600\n"
    "s_add_u32 m0, s51, 0x2000\n"
    "s_xor_b32 s82, s66, 0x2880000\n"
    "buffer_load_dwordx4 v251, s[40:43], s82 offen  lds\n"
    "ds_read_b64_tr_b16 v[220:221], v247 offset:16384\n"
    "s_add_u32 m0, s50, 0x8000\n"
    "s_xor_b32 s82, s48, 0x201480\n"
    "buffer_load_dwordx4 v250, s[36:39], s82 offen  lds\n"
    "ds_read_b64_tr_b16 v[222:223], v247 offset:17408\n"
    "s_add_u32 m0, s50, 0x9000\n"
    "s_xor_b32 s82, s48, 0x281480\n"
    "buffer_load_dwordx4 v250, s[36:39], s82 offen  lds\n"
    "ds_read_b64_tr_b16 v[224:225], v247 offset:24576\n"
    "s_add_u32 m0, s50, 0xa000\n"
    "s_xor_b32 s82, s48, 0x1480\n"
    "buffer_load_dwordx4 v250, s[36:39], s82 offen  lds\n"
    "ds_read_b64_tr_b16 v[226:227], v247 offset:25600\n"
    "s_add_u32 m0, s50, 0xb000\n"
    "s_xor_b32 s82, s48, 0x81480\n"
    "buffer_load_dwordx4 v250, s[36:39], s82 offen  lds\n"
    "ds_read_b64_tr_b16 v[228:229], v248 offset:16384\n"
    "s_cmp_lg_u32 s52, 1\n"
    "s_cbranch_scc1 .La99_%=\n"
    "s_xor_b32 s82, s67, 0x5100000\n"
    "s_add_u32 s82, s82, s53\n"
    "buffer_load_dword v9, v8, s[40:43], s82 offen \n"
    ".La99_%=:\n"
    "ds_read_b64_tr_b16 v[230:231], v248 offset:17408\n"
    "ds_read_b64_tr_b16 v[232:233], v248 offset:24576\n"
    "ds_read_b64_tr_b16 v[234:235], v248 offset:25600\n"
    "ds_read_b64_tr_b16 v[236:237], v249 offset:16384\n"
    "ds_read_b64_tr_b16 v[238:239], v249 offset:17408\n"
    "ds_read_b64_tr_b16 v[240:241], v249 offset:24576\n"
    "ds_read_b64_tr_b16 v[242:243], v249 offset:25600\n"
    "s_waitcnt lgkmcnt(0)\n"
    "s_barrier\n"
    "s_setprio 1\n"
    "v_mfma_f32_16x16x32_bf16 v[76:79], v[212:215], v[180:183], v[76:79]\n"
    "v_mfma_f32_16x16x32_bf16 v[76:79], v[216:219], v[184:187], v[76:79]\n"
    "v_mfma_f32_16x16x32_bf16 v[80:83], v[220:223], v[180:183], v[80:83]\n"
    "v_mfma_f32_16x16x32_bf16 v[80:83], v[224:227], v[184:187], v[80:83]\n"
    "v_mfma_f32_16x16x32_bf16 v[84:87], v[228:231], v[180:183], v[84:87]\n"
    "v_mfma_f32_16x16x32_bf16 v[84:87], v[232:235], v[184:187], v[84:87]\n"
    "v_mfma_f32_16x16x32_bf16 v[88:91], v[236:239], v[180:183], v[88:91]\n"
    "v_mfma_f32_16x16x32_bf16 v[88:91], v[240:243], v[184:187], v[88:91]\n"
    "v_mfma_f32_16x16x32_bf16 v[92:95], v[212:215], v[188:191], v[92:95]\n"
    "v_mfma_f32_16x16x32_bf16 v[92:95], v[216:219], v[192:195], v[92:95]\n"
    "v_mfma_f32_16x16x32_bf16 v[96:99], v[220:223], v[188:191], v[96:99]\n"
    "v_mfma_f32_16x16x32_bf16 v[96:99], v[224:227], v[192:195], v[96:99]\n"
    "v_mfma_f32_16x16x32_bf16 v[100:103], v[228:231], v[188:191], v[100:103]\n"
    "v_mfma_f32_16x16x32_bf16 v[100:103], v[232:235], v[192:195], v[100:103]\n"
    "v_mfma_f32_16x16x32_bf16 v[104:107], v[236:239], v[188:191], v[104:107]\n"
    "v_mfma_f32_16x16x32_bf16 v[104:107], v[240:243], v[192:195], v[104:107]\n"
    "v_mfma_f32_16x16x32_bf16 v[108:111], v[212:215], v[196:199], v[108:111]\n"
    "v_mfma_f32_16x16x32_bf16 v[108:111], v[216:219], v[200:203], v[108:111]\n"
    "v_mfma_f32_16x16x32_bf16 v[112:115], v[220:223], v[196:199], v[112:115]\n"
    "v_mfma_f32_16x16x32_bf16 v[112:115], v[224:227], v[200:203], v[112:115]\n"
    "v_mfma_f32_16x16x32_bf16 v[116:119], v[228:231], v[196:199], v[116:119]\n"
    "v_mfma_f32_16x16x32_bf16 v[116:119], v[232:235], v[200:203], v[116:119]\n"
    "v_mfma_f32_16x16x32_bf16 v[120:123], v[236:239], v[196:199], v[120:123]\n"
    "v_mfma_f32_16x16x32_bf16 v[120:123], v[240:243], v[200:203], v[120:123]\n"
    "v_mfma_f32_16x16x32_bf16 v[124:127], v[212:215], v[204:207], v[124:127]\n"
    "v_mfma_f32_16x16x32_bf16 v[124:127], v[216:219], v[208:211], v[124:127]\n"
    "v_mfma_f32_16x16x32_bf16 v[128:131], v[220:223], v[204:207], v[128:131]\n"
    "v_mfma_f32_16x16x32_bf16 v[128:131], v[224:227], v[208:211], v[128:131]\n"
    "v_mfma_f32_16x16x32_bf16 v[132:135], v[228:231], v[204:207], v[132:135]\n"
    "v_mfma_f32_16x16x32_bf16 v[132:135], v[232:235], v[208:211], v[132:135]\n"
    "v_mfma_f32_16x16x32_bf16 v[136:139], v[236:239], v[204:207], v[136:139]\n"
    "v_mfma_f32_16x16x32_bf16 v[136:139], v[240:243], v[208:211], v[136:139]\n"
    "s_setprio 0\n"
    "s_barrier\n"
    "; phase 66: step 39 slot 0 stage 2\n"
    "ds_read_b128 v[180:183], v244 offset:0\n"
    "ds_read_b128 v[184:187], v245 offset:0\n"
    "ds_read_b128 v[188:191], v244 offset:2048\n"
    "ds_read_b128 v[192:195], v245 offset:2048\n"
    "s_add_u32 s82, s54, 0x10c000\n"
    "buffer_load_dwordx4 v[28:31], v252, s[44:47], s82 offen sc0 sc1\n"
    "ds_read_b128 v[196:199], v244 offset:8192\n"
    "ds_read_b128 v[200:203], v245 offset:8192\n"
    "ds_read_b128 v[204:207], v244 offset:10240\n"
    "ds_read_b128 v[208:211], v245 offset:10240\n"
    "s_add_u32 s82, s54, 0x106000\n"
    "buffer_load_dwordx4 v[36:39], v252, s[44:47], s82 offen sc0 sc1\n"
    "ds_read_b64_tr_b16 v[212:213], v246 offset:32768\n"
    "ds_read_b64_tr_b16 v[214:215], v246 offset:33792\n"
    "ds_read_b64_tr_b16 v[216:217], v246 offset:40960\n"
    "ds_read_b64_tr_b16 v[218:219], v246 offset:41984\n"
    "s_add_u32 m0, s51, 0x4000\n"
    "s_xor_b32 s82, s65, 0x2800000\n"
    "buffer_load_dwordx4 v251, s[40:43], s82 offen  lds\n"
    "ds_read_b64_tr_b16 v[220:221], v247 offset:32768\n"
    "ds_read_b64_tr_b16 v[222:223], v247 offset:33792\n"
    "ds_read_b64_tr_b16 v[224:225], v247 offset:40960\n"
    "ds_read_b64_tr_b16 v[226:227], v247 offset:41984\n"
    "s_add_u32 m0, s51, 0x6000\n"
    "s_xor_b32 s82, s65, 0x2880000\n"
    "buffer_load_dwordx4 v251, s[40:43], s82 offen  lds\n"
    "ds_read_b64_tr_b16 v[228:229], v248 offset:32768\n"
    "ds_read_b64_tr_b16 v[230:231], v248 offset:33792\n"
    "ds_read_b64_tr_b16 v[232:233], v248 offset:40960\n"
    "ds_read_b64_tr_b16 v[234:235], v248 offset:41984\n"
    "s_cmp_lg_u32 s52, 2\n"
    "s_cbranch_scc1 .La100_%=\n"
    "s_xor_b32 s82, s67, 0x5200000\n"
    "s_add_u32 s82, s82, s53\n"
    "buffer_load_dword v9, v8, s[40:43], s82 offen \n"
    ".La100_%=:\n"
    "ds_read_b64_tr_b16 v[236:237], v249 offset:32768\n"
    "ds_read_b64_tr_b16 v[238:239], v249 offset:33792\n"
    "ds_read_b64_tr_b16 v[240:241], v249 offset:40960\n"
    "ds_read_b64_tr_b16 v[242:243], v249 offset:41984\n"
    "s_waitcnt vmcnt(10) lgkmcnt(0)\n"
    "s_barrier\n"
    "s_setprio 1\n"
    "v_mfma_f32_16x16x32_bf16 v[140:143], v[212:215], v[180:183], v[140:143]\n"
    "v_mfma_f32_16x16x32_bf16 v[68:71], v[2:5], v[56:59], v[68:71]\n"
    "v_mfma_f32_16x16x32_bf16 v[140:143], v[216:219], v[184:187], v[140:143]\n"
    "v_mfma_f32_16x16x32_bf16 v[72:75], v[0:3], v[56:59], v[72:75]\n"
    "v_mfma_f32_16x16x32_bf16 v[144:147], v[220:223], v[180:183], v[144:147]\n"
    "v_mfma_f32_16x16x32_bf16 v[68:71], v[2:5], v[20:23], v[68:71]\n"
    "v_mfma_f32_16x16x32_bf16 v[144:147], v[224:227], v[184:187], v[144:147]\n"
    "v_mfma_f32_16x16x32_bf16 v[72:75], v[0:3], v[20:23], v[72:75]\n"
    "v_mfma_f32_16x16x32_bf16 v[148:151], v[228:231], v[180:183], v[148:151]\n"
    "v_mfma_f32_16x16x32_bf16 v[148:151], v[232:235], v[184:187], v[148:151]\n"
    "v_mfma_f32_16x16x32_bf16 v[152:155], v[236:239], v[180:183], v[152:155]\n"
    "v_mfma_f32_16x16x32_bf16 v[152:155], v[240:243], v[184:187], v[152:155]\n"
    "v_mfma_f32_16x16x32_bf16 v[156:159], v[212:215], v[188:191], v[156:159]\n"
    "v_mfma_f32_16x16x32_bf16 v[156:159], v[216:219], v[192:195], v[156:159]\n"
    "v_mfma_f32_16x16x32_bf16 v[160:163], v[220:223], v[188:191], v[160:163]\n"
    "v_mfma_f32_16x16x32_bf16 v[160:163], v[224:227], v[192:195], v[160:163]\n"
    "v_mfma_f32_16x16x32_bf16 v[164:167], v[228:231], v[188:191], v[164:167]\n"
    "v_mfma_f32_16x16x32_bf16 v[164:167], v[232:235], v[192:195], v[164:167]\n"
    "v_mfma_f32_16x16x32_bf16 v[168:171], v[236:239], v[188:191], v[168:171]\n"
    "v_mfma_f32_16x16x32_bf16 v[168:171], v[240:243], v[192:195], v[168:171]\n"
    "v_mfma_f32_16x16x32_bf16 v[172:175], v[212:215], v[196:199], v[172:175]\n"
    "v_mfma_f32_16x16x32_bf16 v[172:175], v[216:219], v[200:203], v[172:175]\n"
    "v_mfma_f32_16x16x32_bf16 v[176:179], v[220:223], v[196:199], v[176:179]\n"
    "v_mfma_f32_16x16x32_bf16 v[176:179], v[224:227], v[200:203], v[176:179]\n"
    "v_mfma_f32_16x16x32_bf16 v[16:19], v[228:231], v[196:199], v[16:19]\n"
    "v_mfma_f32_16x16x32_bf16 v[16:19], v[232:235], v[200:203], v[16:19]\n"
    "v_mfma_f32_16x16x32_bf16 v[24:27], v[236:239], v[196:199], v[24:27]\n"
    "v_mfma_f32_16x16x32_bf16 v[24:27], v[240:243], v[200:203], v[24:27]\n"
    "v_mfma_f32_16x16x32_bf16 v[32:35], v[212:215], v[204:207], v[32:35]\n"
    "v_mfma_f32_16x16x32_bf16 v[32:35], v[216:219], v[208:211], v[32:35]\n"
    "v_mfma_f32_16x16x32_bf16 v[40:43], v[220:223], v[204:207], v[40:43]\n"
    "v_mfma_f32_16x16x32_bf16 v[40:43], v[224:227], v[208:211], v[40:43]\n"
    "v_mfma_f32_16x16x32_bf16 v[12:15], v[228:231], v[204:207], v[12:15]\n"
    "v_mfma_f32_16x16x32_bf16 v[12:15], v[232:235], v[208:211], v[12:15]\n"
    "v_mfma_f32_16x16x32_bf16 v[48:51], v[236:239], v[204:207], v[48:51]\n"
    "v_mfma_f32_16x16x32_bf16 v[48:51], v[240:243], v[208:211], v[48:51]\n"
    "s_setprio 0\n"
    "s_barrier\n"
    "; phase 67: step 39 slot 1 stage 1\n"
    "ds_read_b64_tr_b16 v[212:213], v246 offset:49152\n"
    "ds_read_b64_tr_b16 v[214:215], v246 offset:50176\n"
    "s_add_u32 s82, s54, 0x10a000\n"
    "buffer_load_dwordx4 v[44:47], v252, s[44:47], s82 offen sc0 sc1\n"
    "ds_read_b64_tr_b16 v[216:217], v246 offset:57344\n"
    "ds_read_b64_tr_b16 v[218:219], v246 offset:58368\n"
    "s_add_u32 s82, s54, 0x10e000\n"
    "buffer_load_dwordx4 v[52:55], v252, s[44:47], s82 offen sc0 sc1\n"
    "ds_read_b64_tr_b16 v[220:221], v247 offset:49152\n"
    "ds_read_b64_tr_b16 v[222:223], v247 offset:50176\n"
    "s_add_u32 m0, s51, 0x8000\n"
    "s_xor_b32 s82, s66, 0x2900000\n"
    "buffer_load_dwordx4 v251, s[40:43], s82 offen  lds\n"
    "ds_read_b64_tr_b16 v[224:225], v247 offset:57344\n"
    "ds_read_b64_tr_b16 v[226:227], v247 offset:58368\n"
    "s_add_u32 m0, s51, 0xa000\n"
    "s_xor_b32 s82, s66, 0x2980000\n"
    "buffer_load_dwordx4 v251, s[40:43], s82 offen  lds\n"
    "ds_read_b64_tr_b16 v[228:229], v248 offset:49152\n"
    "ds_read_b64_tr_b16 v[230:231], v248 offset:50176\n"
    "s_add_u32 m0, s50, 0x0\n"
    "s_xor_b32 s82, s48, 0x201500\n"
    "buffer_load_dwordx4 v250, s[36:39], s82 offen  lds\n"
    "ds_read_b64_tr_b16 v[232:233], v248 offset:57344\n"
    "ds_read_b64_tr_b16 v[234:235], v248 offset:58368\n"
    "s_add_u32 m0, s50, 0x1000\n"
    "s_xor_b32 s82, s48, 0x281500\n"
    "buffer_load_dwordx4 v250, s[36:39], s82 offen  lds\n"
    "ds_read_b64_tr_b16 v[236:237], v249 offset:49152\n"
    "ds_read_b64_tr_b16 v[238:239], v249 offset:50176\n"
    "s_add_u32 m0, s50, 0x2000\n"
    "s_xor_b32 s82, s48, 0x1500\n"
    "buffer_load_dwordx4 v250, s[36:39], s82 offen  lds\n"
    "ds_read_b64_tr_b16 v[240:241], v249 offset:57344\n"
    "ds_read_b64_tr_b16 v[242:243], v249 offset:58368\n"
    "s_add_u32 m0, s50, 0x3000\n"
    "s_xor_b32 s82, s48, 0x81500\n"
    "buffer_load_dwordx4 v250, s[36:39], s82 offen  lds\n"
    "s_waitcnt vmcnt(10) lgkmcnt(0)\n"
    "s_barrier\n"
    "s_setprio 1\n"
    "v_mfma_f32_16x16x32_bf16 v[76:79], v[212:215], v[180:183], v[76:79]\n"
    "v_mfma_f32_16x16x32_bf16 v[68:71], v[2:5], v[28:31], v[68:71]\n"
    "v_mfma_f32_16x16x32_bf16 v[76:79], v[216:219], v[184:187], v[76:79]\n"
    "v_mfma_f32_16x16x32_bf16 v[72:75], v[0:3], v[28:31], v[72:75]\n"
    "v_mfma_f32_16x16x32_bf16 v[80:83], v[220:223], v[180:183], v[80:83]\n"
    "v_mfma_f32_16x16x32_bf16 v[60:63], v[2:5], v[36:39], v[60:63]\n"
    "v_mfma_f32_16x16x32_bf16 v[80:83], v[224:227], v[184:187], v[80:83]\n"
    "v_mfma_f32_16x16x32_bf16 v[64:67], v[0:3], v[36:39], v[64:67]\n"
    "v_mfma_f32_16x16x32_bf16 v[84:87], v[228:231], v[180:183], v[84:87]\n"
    "v_mfma_f32_16x16x32_bf16 v[84:87], v[232:235], v[184:187], v[84:87]\n"
    "v_mfma_f32_16x16x32_bf16 v[88:91], v[236:239], v[180:183], v[88:91]\n"
    "v_mfma_f32_16x16x32_bf16 v[88:91], v[240:243], v[184:187], v[88:91]\n"
    "v_med3_f32 v28, v68, -s62, s62\n"
    "v_med3_f32 v29, v69, -s62, s62\n"
    "v_mfma_f32_16x16x32_bf16 v[92:95], v[212:215], v[188:191], v[92:95]\n"
    "v_med3_f32 v30, v70, -s62, s62\n"
    "v_med3_f32 v31, v71, -s62, s62\n"
    "v_mfma_f32_16x16x32_bf16 v[92:95], v[216:219], v[192:195], v[92:95]\n"
    "v_mul_f32 v28, s63, v28\n"
    "v_mul_f32 v29, s63, v29\n"
    "v_mfma_f32_16x16x32_bf16 v[96:99], v[220:223], v[188:191], v[96:99]\n"
    "v_mul_f32 v30, s63, v30\n"
    "v_mul_f32 v31, s63, v31\n"
    "v_mfma_f32_16x16x32_bf16 v[96:99], v[224:227], v[192:195], v[96:99]\n"
    "v_exp_f32 v28, v28\n"
    "v_mfma_f32_16x16x32_bf16 v[100:103], v[228:231], v[188:191], v[100:103]\n"
    "v_exp_f32 v29, v29\n"
    "v_mfma_f32_16x16x32_bf16 v[100:103], v[232:235], v[192:195], v[100:103]\n"
    "v_exp_f32 v30, v30\n"
    "v_mfma_f32_16x16x32_bf16 v[104:107], v[236:239], v[188:191], v[104:107]\n"
    "v_exp_f32 v31, v31\n"
    "v_mfma_f32_16x16x32_bf16 v[104:107], v[240:243], v[192:195], v[104:107]\n"
    "v_add_f32 v28, 1.0, v28\n"
    "v_add_f32 v29, 1.0, v29\n"
    "v_mfma_f32_16x16x32_bf16 v[108:111], v[212:215], v[196:199], v[108:111]\n"
    "v_add_f32 v30, 1.0, v30\n"
    "v_add_f32 v31, 1.0, v31\n"
    "v_mfma_f32_16x16x32_bf16 v[108:111], v[216:219], v[200:203], v[108:111]\n"
    "v_rcp_f32 v28, v28\n"
    "v_mfma_f32_16x16x32_bf16 v[112:115], v[220:223], v[196:199], v[112:115]\n"
    "v_rcp_f32 v29, v29\n"
    "v_mfma_f32_16x16x32_bf16 v[112:115], v[224:227], v[200:203], v[112:115]\n"
    "v_rcp_f32 v30, v30\n"
    "v_mfma_f32_16x16x32_bf16 v[116:119], v[228:231], v[196:199], v[116:119]\n"
    "v_rcp_f32 v31, v31\n"
    "v_mfma_f32_16x16x32_bf16 v[116:119], v[232:235], v[200:203], v[116:119]\n"
    "v_mul_f32 v68, v68, v28\n"
    "v_mul_f32 v69, v69, v29\n"
    "v_mfma_f32_16x16x32_bf16 v[120:123], v[236:239], v[196:199], v[120:123]\n"
    "v_mul_f32 v70, v70, v30\n"
    "v_mul_f32 v71, v71, v31\n"
    "v_mfma_f32_16x16x32_bf16 v[120:123], v[240:243], v[200:203], v[120:123]\n"
    "v_med3_f32 v28, v72, -s62, s62\n"
    "v_med3_f32 v29, v73, -s62, s62\n"
    "v_mfma_f32_16x16x32_bf16 v[124:127], v[212:215], v[204:207], v[124:127]\n"
    "v_med3_f32 v30, v74, -s62, s62\n"
    "v_med3_f32 v31, v75, -s62, s62\n"
    "v_mfma_f32_16x16x32_bf16 v[124:127], v[216:219], v[208:211], v[124:127]\n"
    "v_mul_f32 v28, s63, v28\n"
    "v_mul_f32 v29, s63, v29\n"
    "v_mfma_f32_16x16x32_bf16 v[128:131], v[220:223], v[204:207], v[128:131]\n"
    "v_mul_f32 v30, s63, v30\n"
    "v_mul_f32 v31, s63, v31\n"
    "v_mfma_f32_16x16x32_bf16 v[128:131], v[224:227], v[208:211], v[128:131]\n"
    "v_exp_f32 v28, v28\n"
    "v_mfma_f32_16x16x32_bf16 v[132:135], v[228:231], v[204:207], v[132:135]\n"
    "v_exp_f32 v29, v29\n"
    "v_mfma_f32_16x16x32_bf16 v[132:135], v[232:235], v[208:211], v[132:135]\n"
    "v_exp_f32 v30, v30\n"
    "v_mfma_f32_16x16x32_bf16 v[136:139], v[236:239], v[204:207], v[136:139]\n"
    "v_exp_f32 v31, v31\n"
    "v_mfma_f32_16x16x32_bf16 v[136:139], v[240:243], v[208:211], v[136:139]\n"
    "s_setprio 0\n"
    "s_barrier\n"
    "; phase 68: step 40 slot 0 stage 2\n"
    "ds_read_b128 v[180:183], v244 offset:16384\n"
    "ds_read_b128 v[184:187], v245 offset:16384\n"
    "ds_read_b128 v[188:191], v244 offset:18432\n"
    "ds_read_b128 v[192:195], v245 offset:18432\n"
    "s_add_u32 m0, s51, 0xc000\n"
    "s_xor_b32 s82, s65, 0x2900000\n"
    "buffer_load_dwordx4 v251, s[40:43], s82 offen  lds\n"
    "ds_read_b128 v[196:199], v244 offset:24576\n"
    "ds_read_b128 v[200:203], v245 offset:24576\n"
    "ds_read_b128 v[204:207], v244 offset:26624\n"
    "ds_read_b128 v[208:211], v245 offset:26624\n"
    "s_add_u32 m0, s51, 0xe000\n"
    "s_xor_b32 s82, s65, 0x2980000\n"
    "buffer_load_dwordx4 v251, s[40:43], s82 offen  lds\n"
    "ds_read_b64_tr_b16 v[212:213], v246 offset:0\n"
    "ds_read_b64_tr_b16 v[214:215], v246 offset:1024\n"
    "ds_read_b64_tr_b16 v[216:217], v246 offset:8192\n"
    "ds_read_b64_tr_b16 v[218:219], v246 offset:9216\n"
    "s_cmp_lg_u32 s52, 3\n"
    "s_cbranch_scc1 .La101_%=\n"
    "s_xor_b32 s82, s67, 0x5300000\n"
    "s_add_u32 s82, s82, s53\n"
    "buffer_load_dword v9, v8, s[40:43], s82 offen \n"
    ".La101_%=:\n"
    "ds_read_b64_tr_b16 v[220:221], v247 offset:0\n"
    "ds_read_b64_tr_b16 v[222:223], v247 offset:1024\n"
    "ds_read_b64_tr_b16 v[224:225], v247 offset:8192\n"
    "ds_read_b64_tr_b16 v[226:227], v247 offset:9216\n"
    "ds_read_b64_tr_b16 v[228:229], v248 offset:0\n"
    "ds_read_b64_tr_b16 v[230:231], v248 offset:1024\n"
    "ds_read_b64_tr_b16 v[232:233], v248 offset:8192\n"
    "ds_read_b64_tr_b16 v[234:235], v248 offset:9216\n"
    "ds_read_b64_tr_b16 v[236:237], v249 offset:0\n"
    "ds_read_b64_tr_b16 v[238:239], v249 offset:1024\n"
    "ds_read_b64_tr_b16 v[240:241], v249 offset:8192\n"
    "ds_read_b64_tr_b16 v[242:243], v249 offset:9216\n"
    "s_waitcnt vmcnt(8) lgkmcnt(0)\n"
    "s_barrier\n"
    "s_setprio 1\n"
    "v_mfma_f32_16x16x32_bf16 v[140:143], v[212:215], v[180:183], v[140:143]\n"
    "v_add_f32 v28, 1.0, v28\n"
    "v_add_f32 v29, 1.0, v29\n"
    "v_mfma_f32_16x16x32_bf16 v[140:143], v[216:219], v[184:187], v[140:143]\n"
    "v_add_f32 v30, 1.0, v30\n"
    "v_add_f32 v31, 1.0, v31\n"
    "v_mfma_f32_16x16x32_bf16 v[144:147], v[220:223], v[180:183], v[144:147]\n"
    "v_rcp_f32 v28, v28\n"
    "v_mfma_f32_16x16x32_bf16 v[144:147], v[224:227], v[184:187], v[144:147]\n"
    "v_rcp_f32 v29, v29\n"
    "v_mfma_f32_16x16x32_bf16 v[148:151], v[228:231], v[180:183], v[148:151]\n"
    "v_rcp_f32 v30, v30\n"
    "v_mfma_f32_16x16x32_bf16 v[148:151], v[232:235], v[184:187], v[148:151]\n"
    "v_rcp_f32 v31, v31\n"
    "v_mfma_f32_16x16x32_bf16 v[152:155], v[236:239], v[180:183], v[152:155]\n"
    "v_mul_f32 v72, v72, v28\n"
    "v_mul_f32 v73, v73, v29\n"
    "v_mfma_f32_16x16x32_bf16 v[152:155], v[240:243], v[184:187], v[152:155]\n"
    "v_mul_f32 v74, v74, v30\n"
    "v_mul_f32 v75, v75, v31\n"
    "v_mfma_f32_16x16x32_bf16 v[156:159], v[212:215], v[188:191], v[156:159]\n"
    "v_cvt_pk_bf16_f32 v28, v68, v69\n"
    "v_cvt_pk_bf16_f32 v29, v70, v71\n"
    "v_mfma_f32_16x16x32_bf16 v[156:159], v[216:219], v[192:195], v[156:159]\n"
    "v_cvt_pk_bf16_f32 v30, v72, v73\n"
    "v_cvt_pk_bf16_f32 v31, v74, v75\n"
    "v_mfma_f32_16x16x32_bf16 v[160:163], v[220:223], v[188:191], v[160:163]\n"
    "v_permlane16_swap_b32 v28, v30\n"
    "v_permlane16_swap_b32 v29, v31\n"
    "v_mfma_f32_16x16x32_bf16 v[160:163], v[224:227], v[192:195], v[160:163]\n"
    "v_mfma_f32_16x16x32_bf16 v[60:63], v[2:5], v[44:47], v[60:63]\n"
    "v_mfma_f32_16x16x32_bf16 v[164:167], v[228:231], v[188:191], v[164:167]\n"
    "v_mfma_f32_16x16x32_bf16 v[64:67], v[0:3], v[44:47], v[64:67]\n"
    "v_mfma_f32_16x16x32_bf16 v[164:167], v[232:235], v[192:195], v[164:167]\n"
    "v_mfma_f32_16x16x32_bf16 v[60:63], v[2:5], v[52:55], v[60:63]\n"
    "v_mfma_f32_16x16x32_bf16 v[168:171], v[236:239], v[188:191], v[168:171]\n"
    "v_mfma_f32_16x16x32_bf16 v[64:67], v[0:3], v[52:55], v[64:67]\n"
    "v_mfma_f32_16x16x32_bf16 v[168:171], v[240:243], v[192:195], v[168:171]\n"
    "v_mfma_f32_16x16x32_bf16 v[172:175], v[212:215], v[196:199], v[172:175]\n"
    "v_mfma_f32_16x16x32_bf16 v[172:175], v[216:219], v[200:203], v[172:175]\n"
    "v_mfma_f32_16x16x32_bf16 v[176:179], v[220:223], v[196:199], v[176:179]\n"
    "v_mfma_f32_16x16x32_bf16 v[176:179], v[224:227], v[200:203], v[176:179]\n"
    "v_mfma_f32_16x16x32_bf16 v[16:19], v[228:231], v[196:199], v[16:19]\n"
    "v_mfma_f32_16x16x32_bf16 v[16:19], v[232:235], v[200:203], v[16:19]\n"
    "v_mfma_f32_16x16x32_bf16 v[24:27], v[236:239], v[196:199], v[24:27]\n"
    "v_med3_f32 v52, v60, -s62, s62\n"
    "v_med3_f32 v53, v61, -s62, s62\n"
    "v_mfma_f32_16x16x32_bf16 v[24:27], v[240:243], v[200:203], v[24:27]\n"
    "v_med3_f32 v54, v62, -s62, s62\n"
    "v_med3_f32 v55, v63, -s62, s62\n"
    "v_mfma_f32_16x16x32_bf16 v[32:35], v[212:215], v[204:207], v[32:35]\n"
    "v_mul_f32 v52, s63, v52\n"
    "v_mul_f32 v53, s63, v53\n"
    "v_mfma_f32_16x16x32_bf16 v[32:35], v[216:219], v[208:211], v[32:35]\n"
    "v_mul_f32 v54, s63, v54\n"
    "v_mul_f32 v55, s63, v55\n"
    "v_mfma_f32_16x16x32_bf16 v[40:43], v[220:223], v[204:207], v[40:43]\n"
    "v_exp_f32 v52, v52\n"
    "v_mfma_f32_16x16x32_bf16 v[40:43], v[224:227], v[208:211], v[40:43]\n"
    "v_exp_f32 v53, v53\n"
    "v_mfma_f32_16x16x32_bf16 v[12:15], v[228:231], v[204:207], v[12:15]\n"
    "v_exp_f32 v54, v54\n"
    "v_mfma_f32_16x16x32_bf16 v[12:15], v[232:235], v[208:211], v[12:15]\n"
    "v_exp_f32 v55, v55\n"
    "v_mfma_f32_16x16x32_bf16 v[48:51], v[236:239], v[204:207], v[48:51]\n"
    "v_add_f32 v52, 1.0, v52\n"
    "v_add_f32 v53, 1.0, v53\n"
    "v_mfma_f32_16x16x32_bf16 v[48:51], v[240:243], v[208:211], v[48:51]\n"
    "s_setprio 0\n"
    "s_barrier\n"
    "; phase 69: step 40 slot 1 stage 1\n"
    "ds_read_b64_tr_b16 v[212:213], v246 offset:16384\n"
    "ds_read_b64_tr_b16 v[214:215], v246 offset:17408\n"
    "s_add_u32 m0, s51, 0x0\n"
    "s_xor_b32 s82, s66, 0x2a00000\n"
    "buffer_load_dwordx4 v251, s[40:43], s82 offen  lds\n"
    "ds_read_b64_tr_b16 v[216:217], v246 offset:24576\n"
    "ds_read_b64_tr_b16 v[218:219], v246 offset:25600\n"
    "s_add_u32 m0, s51, 0x2000\n"
    "s_xor_b32 s82, s66, 0x2a80000\n"
    "buffer_load_dwordx4 v251, s[40:43], s82 offen  lds\n"
    "ds_read_b64_tr_b16 v[220:221], v247 offset:16384\n"
    "ds_read_b64_tr_b16 v[222:223], v247 offset:17408\n"
    "s_add_u32 m0, s50, 0x4000\n"
    "s_xor_b32 s82, s48, 0x201580\n"
    "buffer_load_dwordx4 v250, s[36:39], s82 offen  lds\n"
    "ds_read_b64_tr_b16 v[224:225], v247 offset:24576\n"
    "ds_read_b64_tr_b16 v[226:227], v247 offset:25600\n"
    "s_add_u32 m0, s50, 0x5000\n"
    "s_xor_b32 s82, s48, 0x281580\n"
    "buffer_load_dwordx4 v250, s[36:39], s82 offen  lds\n"
    "ds_read_b64_tr_b16 v[228:229], v248 offset:16384\n"
    "ds_read_b64_tr_b16 v[230:231], v248 offset:17408\n"
    "s_add_u32 m0, s50, 0x6000\n"
    "s_xor_b32 s82, s48, 0x1580\n"
    "buffer_load_dwordx4 v250, s[36:39], s82 offen  lds\n"
    "ds_read_b64_tr_b16 v[232:233], v248 offset:24576\n"
    "ds_read_b64_tr_b16 v[234:235], v248 offset:25600\n"
    "s_add_u32 m0, s50, 0x7000\n"
    "s_xor_b32 s82, s48, 0x81580\n"
    "buffer_load_dwordx4 v250, s[36:39], s82 offen  lds\n"
    "ds_read_b64_tr_b16 v[236:237], v249 offset:16384\n"
    "ds_read_b64_tr_b16 v[238:239], v249 offset:17408\n"
    "s_cmp_lg_u32 s52, 4\n"
    "s_cbranch_scc1 .La102_%=\n"
    "s_xor_b32 s82, s67, 0x5400000\n"
    "s_add_u32 s82, s82, s53\n"
    "buffer_load_dword v9, v8, s[40:43], s82 offen \n"
    ".La102_%=:\n"
    "ds_read_b64_tr_b16 v[240:241], v249 offset:24576\n"
    "ds_read_b64_tr_b16 v[242:243], v249 offset:25600\n"
    "s_waitcnt vmcnt(12) lgkmcnt(0)\n"
    "s_barrier\n"
    "s_setprio 1\n"
    "v_mfma_f32_16x16x32_bf16 v[76:79], v[212:215], v[180:183], v[76:79]\n"
    "v_add_f32 v54, 1.0, v54\n"
    "v_add_f32 v55, 1.0, v55\n"
    "v_mfma_f32_16x16x32_bf16 v[76:79], v[216:219], v[184:187], v[76:79]\n"
    "v_rcp_f32 v52, v52\n"
    "v_mfma_f32_16x16x32_bf16 v[80:83], v[220:223], v[180:183], v[80:83]\n"
    "v_rcp_f32 v53, v53\n"
    "v_mfma_f32_16x16x32_bf16 v[80:83], v[224:227], v[184:187], v[80:83]\n"
    "v_rcp_f32 v54, v54\n"
    "v_mfma_f32_16x16x32_bf16 v[84:87], v[228:231], v[180:183], v[84:87]\n"
    "v_rcp_f32 v55, v55\n"
    "v_mfma_f32_16x16x32_bf16 v[84:87], v[232:235], v[184:187], v[84:87]\n"
    "v_mul_f32 v60, v60, v52\n"
    "v_mul_f32 v61, v61, v53\n"
    "v_mfma_f32_16x16x32_bf16 v[88:91], v[236:239], v[180:183], v[88:91]\n"
    "v_mul_f32 v62, v62, v54\n"
    "v_mul_f32 v63, v63, v55\n"
    "v_mfma_f32_16x16x32_bf16 v[88:91], v[240:243], v[184:187], v[88:91]\n"
    "v_med3_f32 v52, v64, -s62, s62\n"
    "v_med3_f32 v53, v65, -s62, s62\n"
    "v_mfma_f32_16x16x32_bf16 v[92:95], v[212:215], v[188:191], v[92:95]\n"
    "v_med3_f32 v54, v66, -s62, s62\n"
    "v_med3_f32 v55, v67, -s62, s62\n"
    "v_mfma_f32_16x16x32_bf16 v[92:95], v[216:219], v[192:195], v[92:95]\n"
    "v_mul_f32 v52, s63, v52\n"
    "v_mul_f32 v53, s63, v53\n"
    "v_mfma_f32_16x16x32_bf16 v[96:99], v[220:223], v[188:191], v[96:99]\n"
    "v_mul_f32 v54, s63, v54\n"
    "v_mul_f32 v55, s63, v55\n"
    "v_mfma_f32_16x16x32_bf16 v[96:99], v[224:227], v[192:195], v[96:99]\n"
    "v_exp_f32 v52, v52\n"
    "v_mfma_f32_16x16x32_bf16 v[100:103], v[228:231], v[188:191], v[100:103]\n"
    "v_exp_f32 v53, v53\n"
    "v_mfma_f32_16x16x32_bf16 v[100:103], v[232:235], v[192:195], v[100:103]\n"
    "v_exp_f32 v54, v54\n"
    "v_mfma_f32_16x16x32_bf16 v[104:107], v[236:239], v[188:191], v[104:107]\n"
    "v_exp_f32 v55, v55\n"
    "v_mfma_f32_16x16x32_bf16 v[104:107], v[240:243], v[192:195], v[104:107]\n"
    "v_add_f32 v52, 1.0, v52\n"
    "v_add_f32 v53, 1.0, v53\n"
    "v_mfma_f32_16x16x32_bf16 v[108:111], v[212:215], v[196:199], v[108:111]\n"
    "v_add_f32 v54, 1.0, v54\n"
    "v_add_f32 v55, 1.0, v55\n"
    "v_mfma_f32_16x16x32_bf16 v[108:111], v[216:219], v[200:203], v[108:111]\n"
    "v_rcp_f32 v52, v52\n"
    "v_mfma_f32_16x16x32_bf16 v[112:115], v[220:223], v[196:199], v[112:115]\n"
    "v_rcp_f32 v53, v53\n"
    "v_mfma_f32_16x16x32_bf16 v[112:115], v[224:227], v[200:203], v[112:115]\n"
    "v_rcp_f32 v54, v54\n"
    "v_mfma_f32_16x16x32_bf16 v[116:119], v[228:231], v[196:199], v[116:119]\n"
    "v_rcp_f32 v55, v55\n"
    "v_mfma_f32_16x16x32_bf16 v[116:119], v[232:235], v[200:203], v[116:119]\n"
    "v_mul_f32 v64, v64, v52\n"
    "v_mul_f32 v65, v65, v53\n"
    "v_mfma_f32_16x16x32_bf16 v[120:123], v[236:239], v[196:199], v[120:123]\n"
    "v_mul_f32 v66, v66, v54\n"
    "v_mul_f32 v67, v67, v55\n"
    "v_mfma_f32_16x16x32_bf16 v[120:123], v[240:243], v[200:203], v[120:123]\n"
    "v_cvt_pk_bf16_f32 v52, v60, v61\n"
    "v_cvt_pk_bf16_f32 v53, v62, v63\n"
    "v_mfma_f32_16x16x32_bf16 v[124:127], v[212:215], v[204:207], v[124:127]\n"
    "v_cvt_pk_bf16_f32 v54, v64, v65\n"
    "v_cvt_pk_bf16_f32 v55, v66, v67\n"
    "v_mfma_f32_16x16x32_bf16 v[124:127], v[216:219], v[208:211], v[124:127]\n"
    "v_permlane16_swap_b32 v52, v54\n"
    "v_permlane16_swap_b32 v53, v55\n"
    "v_mfma_f32_16x16x32_bf16 v[128:131], v[220:223], v[204:207], v[128:131]\n"
    "v_mov_b32 v60, v28\n"
    "v_mov_b32 v61, v29\n"
    "v_mfma_f32_16x16x32_bf16 v[128:131], v[224:227], v[208:211], v[128:131]\n"
    "v_mov_b32 v62, v30\n"
    "v_mov_b32 v63, v31\n"
    "v_mfma_f32_16x16x32_bf16 v[132:135], v[228:231], v[204:207], v[132:135]\n"
    "v_mov_b32_dpp v28, v52 row_ror:8 row_mask:0xf bank_mask:0xc\n"
    "v_mov_b32_dpp v29, v53 row_ror:8 row_mask:0xf bank_mask:0xc\n"
    "v_mfma_f32_16x16x32_bf16 v[132:135], v[232:235], v[208:211], v[132:135]\n"
    "v_mov_b32_dpp v30, v54 row_ror:8 row_mask:0xf bank_mask:0xc\n"
    "v_mov_b32_dpp v31, v55 row_ror:8 row_mask:0xf bank_mask:0xc\n"
    "v_mfma_f32_16x16x32_bf16 v[136:139], v[236:239], v[204:207], v[136:139]\n"
    "v_mov_b32_dpp v52, v60 row_ror:8 row_mask:0xf bank_mask:0x3\n"
    "v_mov_b32_dpp v53, v61 row_ror:8 row_mask:0xf bank_mask:0x3\n"
    "v_mfma_f32_16x16x32_bf16 v[136:139], v[240:243], v[208:211], v[136:139]\n"
    "s_setprio 0\n"
    "s_barrier\n"
    "; phase 70: step 41 slot 0 stage 2\n"
    "ds_read_b128 v[180:183], v244 offset:32768\n"
    "ds_read_b128 v[184:187], v245 offset:32768\n"
    "ds_read_b128 v[188:191], v244 offset:34816\n"
    "ds_read_b128 v[192:195], v245 offset:34816\n"
    "s_add_u32 m0, s51, 0x4000\n"
    "s_xor_b32 s82, s65, 0x2a00000\n"
    "buffer_load_dwordx4 v251, s[40:43], s82 offen  lds\n"
    "ds_read_b128 v[196:199], v244 offset:40960\n"
    "ds_read_b128 v[200:203], v245 offset:40960\n"
    "ds_read_b128 v[204:207], v244 offset:43008\n"
    "ds_read_b128 v[208:211], v245 offset:43008\n"
    "s_add_u32 m0, s51, 0x6000\n"
    "s_xor_b32 s82, s65, 0x2a80000\n"
    "buffer_load_dwordx4 v251, s[40:43], s82 offen  lds\n"
    "ds_read_b64_tr_b16 v[212:213], v246 offset:32768\n"
    "ds_read_b64_tr_b16 v[214:215], v246 offset:33792\n"
    "ds_read_b64_tr_b16 v[216:217], v246 offset:40960\n"
    "ds_read_b64_tr_b16 v[218:219], v246 offset:41984\n"
    "s_cmp_lg_u32 s52, 5\n"
    "s_cbranch_scc1 .La103_%=\n"
    "s_xor_b32 s82, s67, 0x5500000\n"
    "s_add_u32 s82, s82, s53\n"
    "buffer_load_dword v9, v8, s[40:43], s82 offen \n"
    ".La103_%=:\n"
    "ds_read_b64_tr_b16 v[220:221], v247 offset:32768\n"
    "ds_read_b64_tr_b16 v[222:223], v247 offset:33792\n"
    "ds_read_b64_tr_b16 v[224:225], v247 offset:40960\n"
    "ds_read_b64_tr_b16 v[226:227], v247 offset:41984\n"
    "ds_read_b64_tr_b16 v[228:229], v248 offset:32768\n"
    "ds_read_b64_tr_b16 v[230:231], v248 offset:33792\n"
    "ds_read_b64_tr_b16 v[232:233], v248 offset:40960\n"
    "ds_read_b64_tr_b16 v[234:235], v248 offset:41984\n"
    "ds_read_b64_tr_b16 v[236:237], v249 offset:32768\n"
    "ds_read_b64_tr_b16 v[238:239], v249 offset:33792\n"
    "ds_read_b64_tr_b16 v[240:241], v249 offset:40960\n"
    "ds_read_b64_tr_b16 v[242:243], v249 offset:41984\n"
    "s_waitcnt vmcnt(8) lgkmcnt(0)\n"
    "s_barrier\n"
    "s_setprio 1\n"
    "v_mfma_f32_16x16x32_bf16 v[140:143], v[212:215], v[180:183], v[140:143]\n"
    "v_mov_b32_dpp v54, v62 row_ror:8 row_mask:0xf bank_mask:0x3\n"
    "v_mov_b32_dpp v55, v63 row_ror:8 row_mask:0xf bank_mask:0x3\n"
    "v_mfma_f32_16x16x32_bf16 v[140:143], v[216:219], v[184:187], v[140:143]\n"
    "v_mfma_f32_16x16x32_bf16 v[144:147], v[220:223], v[180:183], v[144:147]\n"
    "v_mfma_f32_16x16x32_bf16 v[144:147], v[224:227], v[184:187], v[144:147]\n"
    "v_mfma_f32_16x16x32_bf16 v[148:151], v[228:231], v[180:183], v[148:151]\n"
    "v_mfma_f32_16x16x32_bf16 v[148:151], v[232:235], v[184:187], v[148:151]\n"
    "v_mfma_f32_16x16x32_bf16 v[152:155], v[236:239], v[180:183], v[152:155]\n"
    "v_mfma_f32_16x16x32_bf16 v[152:155], v[240:243], v[184:187], v[152:155]\n"
    "v_mfma_f32_16x16x32_bf16 v[156:159], v[212:215], v[188:191], v[156:159]\n"
    "v_mfma_f32_16x16x32_bf16 v[156:159], v[216:219], v[192:195], v[156:159]\n"
    "v_mfma_f32_16x16x32_bf16 v[160:163], v[220:223], v[188:191], v[160:163]\n"
    "v_mfma_f32_16x16x32_bf16 v[160:163], v[224:227], v[192:195], v[160:163]\n"
    "v_mfma_f32_16x16x32_bf16 v[164:167], v[228:231], v[188:191], v[164:167]\n"
    "v_mfma_f32_16x16x32_bf16 v[164:167], v[232:235], v[192:195], v[164:167]\n"
    "v_mfma_f32_16x16x32_bf16 v[168:171], v[236:239], v[188:191], v[168:171]\n"
    "v_mfma_f32_16x16x32_bf16 v[168:171], v[240:243], v[192:195], v[168:171]\n"
    "v_mfma_f32_16x16x32_bf16 v[172:175], v[212:215], v[196:199], v[172:175]\n"
    "v_mfma_f32_16x16x32_bf16 v[172:175], v[216:219], v[200:203], v[172:175]\n"
    "v_mfma_f32_16x16x32_bf16 v[176:179], v[220:223], v[196:199], v[176:179]\n"
    "v_mfma_f32_16x16x32_bf16 v[176:179], v[224:227], v[200:203], v[176:179]\n"
    "v_mfma_f32_16x16x32_bf16 v[16:19], v[228:231], v[196:199], v[16:19]\n"
    "v_mfma_f32_16x16x32_bf16 v[16:19], v[232:235], v[200:203], v[16:19]\n"
    "v_mfma_f32_16x16x32_bf16 v[24:27], v[236:239], v[196:199], v[24:27]\n"
    "v_mfma_f32_16x16x32_bf16 v[24:27], v[240:243], v[200:203], v[24:27]\n"
    "v_mfma_f32_16x16x32_bf16 v[32:35], v[212:215], v[204:207], v[32:35]\n"
    "v_mfma_f32_16x16x32_bf16 v[32:35], v[216:219], v[208:211], v[32:35]\n"
    "v_mfma_f32_16x16x32_bf16 v[40:43], v[220:223], v[204:207], v[40:43]\n"
    "v_mfma_f32_16x16x32_bf16 v[40:43], v[224:227], v[208:211], v[40:43]\n"
    "v_mfma_f32_16x16x32_bf16 v[12:15], v[228:231], v[204:207], v[12:15]\n"
    "v_mfma_f32_16x16x32_bf16 v[12:15], v[232:235], v[208:211], v[12:15]\n"
    "v_mfma_f32_16x16x32_bf16 v[48:51], v[236:239], v[204:207], v[48:51]\n"
    "v_mfma_f32_16x16x32_bf16 v[48:51], v[240:243], v[208:211], v[48:51]\n"
    "s_setprio 0\n"
    "s_barrier\n"
    "; phase 71: step 41 slot 1 stage 1\n"
    "ds_read_b64_tr_b16 v[212:213], v246 offset:49152\n"
    "s_cmp_lg_u32 s52, 0\n"
    "s_cbranch_scc1 .La104_%=\n"
    "buffer_store_dwordx4 v[28:31], v255, s[72:75], s80 offen nt\n"
    ".La104_%=:\n"
    "ds_read_b64_tr_b16 v[214:215], v246 offset:50176\n"
    "s_cmp_lg_u32 s52, 1\n"
    "s_cbranch_scc1 .La105_%=\n"
    "buffer_store_dwordx4 v[28:31], v255, s[72:75], s80 offen nt\n"
    ".La105_%=:\n"
    "ds_read_b64_tr_b16 v[216:217], v246 offset:57344\n"
    "s_add_u32 m0, s51, 0x8000\n"
    "s_xor_b32 s82, s66, 0x2b00000\n"
    "buffer_load_dwordx4 v251, s[40:43], s82 offen  lds\n"
    "ds_read_b64_tr_b16 v[218:219], v246 offset:58368\n"
    "s_add_u32 m0, s51, 0xa000\n"
    "s_xor_b32 s82, s66, 0x2b80000\n"
    "buffer_load_dwordx4 v251, s[40:43], s82 offen  lds\n"
    "ds_read_b64_tr_b16 v[220:221], v247 offset:49152\n"
    "s_add_u32 m0, s50, 0x8000\n"
    "s_xor_b32 s82, s48, 0x201600\n"
    "buffer_load_dwordx4 v250, s[36:39], s82 offen  lds\n"
    "ds_read_b64_tr_b16 v[222:223], v247 offset:50176\n"
    "s_add_u32 m0, s50, 0x9000\n"
    "s_xor_b32 s82, s48, 0x281600\n"
    "buffer_load_dwordx4 v250, s[36:39], s82 offen  lds\n"
    "ds_read_b64_tr_b16 v[224:225], v247 offset:57344\n"
    "s_add_u32 m0, s50, 0xa000\n"
    "s_xor_b32 s82, s48, 0x1600\n"
    "buffer_load_dwordx4 v250, s[36:39], s82 offen  lds\n"
    "ds_read_b64_tr_b16 v[226:227], v247 offset:58368\n"
    "s_add_u32 m0, s50, 0xb000\n"
    "s_xor_b32 s82, s48, 0x81600\n"
    "buffer_load_dwordx4 v250, s[36:39], s82 offen  lds\n"
    "ds_read_b64_tr_b16 v[228:229], v248 offset:49152\n"
    "s_cmp_lg_u32 s52, 6\n"
    "s_cbranch_scc1 .La106_%=\n"
    "s_xor_b32 s82, s67, 0x5600000\n"
    "s_add_u32 s82, s82, s53\n"
    "buffer_load_dword v9, v8, s[40:43], s82 offen \n"
    ".La106_%=:\n"
    "ds_read_b64_tr_b16 v[230:231], v248 offset:50176\n"
    "ds_read_b64_tr_b16 v[232:233], v248 offset:57344\n"
    "ds_read_b64_tr_b16 v[234:235], v248 offset:58368\n"
    "ds_read_b64_tr_b16 v[236:237], v249 offset:49152\n"
    "ds_read_b64_tr_b16 v[238:239], v249 offset:50176\n"
    "ds_read_b64_tr_b16 v[240:241], v249 offset:57344\n"
    "ds_read_b64_tr_b16 v[242:243], v249 offset:58368\n"
    "s_waitcnt vmcnt(12) lgkmcnt(0)\n"
    "s_barrier\n"
    "s_setprio 1\n"
    "v_mfma_f32_16x16x32_bf16 v[76:79], v[212:215], v[180:183], v[76:79]\n"
    "v_mfma_f32_16x16x32_bf16 v[76:79], v[216:219], v[184:187], v[76:79]\n"
    "v_mfma_f32_16x16x32_bf16 v[80:83], v[220:223], v[180:183], v[80:83]\n"
    "v_mfma_f32_16x16x32_bf16 v[80:83], v[224:227], v[184:187], v[80:83]\n"
    "v_mfma_f32_16x16x32_bf16 v[84:87], v[228:231], v[180:183], v[84:87]\n"
    "v_mfma_f32_16x16x32_bf16 v[84:87], v[232:235], v[184:187], v[84:87]\n"
    "v_mfma_f32_16x16x32_bf16 v[88:91], v[236:239], v[180:183], v[88:91]\n"
    "v_mfma_f32_16x16x32_bf16 v[88:91], v[240:243], v[184:187], v[88:91]\n"
    "v_mfma_f32_16x16x32_bf16 v[92:95], v[212:215], v[188:191], v[92:95]\n"
    "v_mfma_f32_16x16x32_bf16 v[92:95], v[216:219], v[192:195], v[92:95]\n"
    "v_mfma_f32_16x16x32_bf16 v[96:99], v[220:223], v[188:191], v[96:99]\n"
    "v_mfma_f32_16x16x32_bf16 v[96:99], v[224:227], v[192:195], v[96:99]\n"
    "v_mfma_f32_16x16x32_bf16 v[100:103], v[228:231], v[188:191], v[100:103]\n"
    "v_mfma_f32_16x16x32_bf16 v[100:103], v[232:235], v[192:195], v[100:103]\n"
    "v_mfma_f32_16x16x32_bf16 v[104:107], v[236:239], v[188:191], v[104:107]\n"
    "v_mfma_f32_16x16x32_bf16 v[104:107], v[240:243], v[192:195], v[104:107]\n"
    "v_mfma_f32_16x16x32_bf16 v[108:111], v[212:215], v[196:199], v[108:111]\n"
    "v_mfma_f32_16x16x32_bf16 v[108:111], v[216:219], v[200:203], v[108:111]\n"
    "v_mfma_f32_16x16x32_bf16 v[112:115], v[220:223], v[196:199], v[112:115]\n"
    "v_mfma_f32_16x16x32_bf16 v[112:115], v[224:227], v[200:203], v[112:115]\n"
    "v_mfma_f32_16x16x32_bf16 v[116:119], v[228:231], v[196:199], v[116:119]\n"
    "v_mfma_f32_16x16x32_bf16 v[116:119], v[232:235], v[200:203], v[116:119]\n"
    "v_mfma_f32_16x16x32_bf16 v[120:123], v[236:239], v[196:199], v[120:123]\n"
    "v_mfma_f32_16x16x32_bf16 v[120:123], v[240:243], v[200:203], v[120:123]\n"
    "v_mfma_f32_16x16x32_bf16 v[124:127], v[212:215], v[204:207], v[124:127]\n"
    "v_mfma_f32_16x16x32_bf16 v[124:127], v[216:219], v[208:211], v[124:127]\n"
    "v_mfma_f32_16x16x32_bf16 v[128:131], v[220:223], v[204:207], v[128:131]\n"
    "v_mfma_f32_16x16x32_bf16 v[128:131], v[224:227], v[208:211], v[128:131]\n"
    "v_mfma_f32_16x16x32_bf16 v[132:135], v[228:231], v[204:207], v[132:135]\n"
    "v_mfma_f32_16x16x32_bf16 v[132:135], v[232:235], v[208:211], v[132:135]\n"
    "v_mfma_f32_16x16x32_bf16 v[136:139], v[236:239], v[204:207], v[136:139]\n"
    "v_mfma_f32_16x16x32_bf16 v[136:139], v[240:243], v[208:211], v[136:139]\n"
    "s_setprio 0\n"
    "s_barrier\n"
    "; phase 72: step 42 slot 0 stage 2\n"
    "ds_read_b128 v[180:183], v244 offset:0\n"
    "ds_read_b128 v[184:187], v245 offset:0\n"
    "ds_read_b128 v[188:191], v244 offset:2048\n"
    "ds_read_b128 v[192:195], v245 offset:2048\n"
    "s_cmp_lg_u32 s52, 2\n"
    "s_cbranch_scc1 .La107_%=\n"
    "buffer_store_dwordx4 v[28:31], v255, s[72:75], s80 offen nt\n"
    ".La107_%=:\n"
    "ds_read_b128 v[196:199], v244 offset:8192\n"
    "ds_read_b128 v[200:203], v245 offset:8192\n"
    "ds_read_b128 v[204:207], v244 offset:10240\n"
    "ds_read_b128 v[208:211], v245 offset:10240\n"
    "s_add_u32 m0, s51, 0xc000\n"
    "s_xor_b32 s82, s65, 0x2b00000\n"
    "buffer_load_dwordx4 v251, s[40:43], s82 offen  lds\n"
    "ds_read_b64_tr_b16 v[212:213], v246 offset:0\n"
    "ds_read_b64_tr_b16 v[214:215], v246 offset:1024\n"
    "ds_read_b64_tr_b16 v[216:217], v246 offset:8192\n"
    "ds_read_b64_tr_b16 v[218:219], v246 offset:9216\n"
    "s_add_u32 m0, s51, 0xe000\n"
    "s_xor_b32 s82, s65, 0x2b80000\n"
    "buffer_load_dwordx4 v251, s[40:43], s82 offen  lds\n"
    "ds_read_b64_tr_b16 v[220:221], v247 offset:0\n"
    "ds_read_b64_tr_b16 v[222:223], v247 offset:1024\n"
    "ds_read_b64_tr_b16 v[224:225], v247 offset:8192\n"
    "ds_read_b64_tr_b16 v[226:227], v247 offset:9216\n"
    "s_cmp_lg_u32 s52, 7\n"
    "s_cbranch_scc1 .La108_%=\n"
    "s_xor_b32 s82, s67, 0x5700000\n"
    "s_add_u32 s82, s82, s53\n"
    "buffer_load_dword v9, v8, s[40:43], s82 offen \n"
    ".La108_%=:\n"
    "ds_read_b64_tr_b16 v[228:229], v248 offset:0\n"
    "ds_read_b64_tr_b16 v[230:231], v248 offset:1024\n"
    "ds_read_b64_tr_b16 v[232:233], v248 offset:8192\n"
    "ds_read_b64_tr_b16 v[234:235], v248 offset:9216\n"
    "ds_read_b64_tr_b16 v[236:237], v249 offset:0\n"
    "ds_read_b64_tr_b16 v[238:239], v249 offset:1024\n"
    "ds_read_b64_tr_b16 v[240:241], v249 offset:8192\n"
    "ds_read_b64_tr_b16 v[242:243], v249 offset:9216\n"
    "s_waitcnt vmcnt(8) lgkmcnt(0)\n"
    "s_barrier\n"
    "s_setprio 1\n"
    "v_mfma_f32_16x16x32_bf16 v[140:143], v[212:215], v[180:183], v[140:143]\n"
    "v_mfma_f32_16x16x32_bf16 v[140:143], v[216:219], v[184:187], v[140:143]\n"
    "v_mfma_f32_16x16x32_bf16 v[144:147], v[220:223], v[180:183], v[144:147]\n"
    "v_mfma_f32_16x16x32_bf16 v[144:147], v[224:227], v[184:187], v[144:147]\n"
    "v_mfma_f32_16x16x32_bf16 v[148:151], v[228:231], v[180:183], v[148:151]\n"
    "v_mfma_f32_16x16x32_bf16 v[148:151], v[232:235], v[184:187], v[148:151]\n"
    "v_mfma_f32_16x16x32_bf16 v[152:155], v[236:239], v[180:183], v[152:155]\n"
    "v_mfma_f32_16x16x32_bf16 v[152:155], v[240:243], v[184:187], v[152:155]\n"
    "v_mfma_f32_16x16x32_bf16 v[156:159], v[212:215], v[188:191], v[156:159]\n"
    "v_mfma_f32_16x16x32_bf16 v[156:159], v[216:219], v[192:195], v[156:159]\n"
    "v_mfma_f32_16x16x32_bf16 v[160:163], v[220:223], v[188:191], v[160:163]\n"
    "v_mfma_f32_16x16x32_bf16 v[160:163], v[224:227], v[192:195], v[160:163]\n"
    "v_mfma_f32_16x16x32_bf16 v[164:167], v[228:231], v[188:191], v[164:167]\n"
    "v_mfma_f32_16x16x32_bf16 v[164:167], v[232:235], v[192:195], v[164:167]\n"
    "v_mfma_f32_16x16x32_bf16 v[168:171], v[236:239], v[188:191], v[168:171]\n"
    "v_mfma_f32_16x16x32_bf16 v[168:171], v[240:243], v[192:195], v[168:171]\n"
    "v_mfma_f32_16x16x32_bf16 v[172:175], v[212:215], v[196:199], v[172:175]\n"
    "v_mfma_f32_16x16x32_bf16 v[172:175], v[216:219], v[200:203], v[172:175]\n"
    "v_mfma_f32_16x16x32_bf16 v[176:179], v[220:223], v[196:199], v[176:179]\n"
    "v_mfma_f32_16x16x32_bf16 v[176:179], v[224:227], v[200:203], v[176:179]\n"
    "v_mfma_f32_16x16x32_bf16 v[16:19], v[228:231], v[196:199], v[16:19]\n"
    "v_mfma_f32_16x16x32_bf16 v[16:19], v[232:235], v[200:203], v[16:19]\n"
    "v_mfma_f32_16x16x32_bf16 v[24:27], v[236:239], v[196:199], v[24:27]\n"
    "v_mfma_f32_16x16x32_bf16 v[24:27], v[240:243], v[200:203], v[24:27]\n"
    "v_mfma_f32_16x16x32_bf16 v[32:35], v[212:215], v[204:207], v[32:35]\n"
    "v_mfma_f32_16x16x32_bf16 v[32:35], v[216:219], v[208:211], v[32:35]\n"
    "v_mfma_f32_16x16x32_bf16 v[40:43], v[220:223], v[204:207], v[40:43]\n"
    "v_mfma_f32_16x16x32_bf16 v[40:43], v[224:227], v[208:211], v[40:43]\n"
    "v_mfma_f32_16x16x32_bf16 v[12:15], v[228:231], v[204:207], v[12:15]\n"
    "v_mfma_f32_16x16x32_bf16 v[12:15], v[232:235], v[208:211], v[12:15]\n"
    "v_mfma_f32_16x16x32_bf16 v[48:51], v[236:239], v[204:207], v[48:51]\n"
    "v_mfma_f32_16x16x32_bf16 v[48:51], v[240:243], v[208:211], v[48:51]\n"
    "s_setprio 0\n"
    "s_barrier\n"
    "; phase 73: step 42 slot 1 stage 1\n"
    "ds_read_b64_tr_b16 v[212:213], v246 offset:16384\n"
    "ds_read_b64_tr_b16 v[214:215], v246 offset:17408\n"
    "s_cmp_lg_u32 s52, 3\n"
    "s_cbranch_scc1 .La109_%=\n"
    "buffer_store_dwordx4 v[28:31], v255, s[72:75], s80 offen nt\n"
    ".La109_%=:\n"
    "ds_read_b64_tr_b16 v[216:217], v246 offset:24576\n"
    "ds_read_b64_tr_b16 v[218:219], v246 offset:25600\n"
    "s_add_u32 m0, s51, 0x0\n"
    "s_xor_b32 s82, s66, 0x2c00000\n"
    "buffer_load_dwordx4 v251, s[40:43], s82 offen  lds\n"
    "ds_read_b64_tr_b16 v[220:221], v247 offset:16384\n"
    "ds_read_b64_tr_b16 v[222:223], v247 offset:17408\n"
    "s_add_u32 m0, s51, 0x2000\n"
    "s_xor_b32 s82, s66, 0x2c80000\n"
    "buffer_load_dwordx4 v251, s[40:43], s82 offen  lds\n"
    "ds_read_b64_tr_b16 v[224:225], v247 offset:24576\n"
    "ds_read_b64_tr_b16 v[226:227], v247 offset:25600\n"
    "s_add_u32 m0, s50, 0x0\n"
    "s_xor_b32 s82, s48, 0x201680\n"
    "buffer_load_dwordx4 v250, s[36:39], s82 offen  lds\n"
    "ds_read_b64_tr_b16 v[228:229], v248 offset:16384\n"
    "ds_read_b64_tr_b16 v[230:231], v248 offset:17408\n"
    "s_add_u32 m0, s50, 0x1000\n"
    "s_xor_b32 s82, s48, 0x281680\n"
    "buffer_load_dwordx4 v250, s[36:39], s82 offen  lds\n"
    "ds_read_b64_tr_b16 v[232:233], v248 offset:24576\n"
    "ds_read_b64_tr_b16 v[234:235], v248 offset:25600\n"
    "s_add_u32 m0, s50, 0x2000\n"
    "s_xor_b32 s82, s48, 0x1680\n"
    "buffer_load_dwordx4 v250, s[36:39], s82 offen  lds\n"
    "ds_read_b64_tr_b16 v[236:237], v249 offset:16384\n"
    "ds_read_b64_tr_b16 v[238:239], v249 offset:17408\n"
    "s_add_u32 m0, s50, 0x3000\n"
    "s_xor_b32 s82, s48, 0x81680\n"
    "buffer_load_dwordx4 v250, s[36:39], s82 offen  lds\n"
    "ds_read_b64_tr_b16 v[240:241], v249 offset:24576\n"
    "ds_read_b64_tr_b16 v[242:243], v249 offset:25600\n"
    "s_waitcnt vmcnt(12) lgkmcnt(0)\n"
    "s_barrier\n"
    "s_setprio 1\n"
    "v_mfma_f32_16x16x32_bf16 v[76:79], v[212:215], v[180:183], v[76:79]\n"
    "v_mfma_f32_16x16x32_bf16 v[76:79], v[216:219], v[184:187], v[76:79]\n"
    "v_mfma_f32_16x16x32_bf16 v[80:83], v[220:223], v[180:183], v[80:83]\n"
    "v_mfma_f32_16x16x32_bf16 v[80:83], v[224:227], v[184:187], v[80:83]\n"
    "v_mfma_f32_16x16x32_bf16 v[84:87], v[228:231], v[180:183], v[84:87]\n"
    "v_mfma_f32_16x16x32_bf16 v[84:87], v[232:235], v[184:187], v[84:87]\n"
    "v_mfma_f32_16x16x32_bf16 v[88:91], v[236:239], v[180:183], v[88:91]\n"
    "v_mfma_f32_16x16x32_bf16 v[88:91], v[240:243], v[184:187], v[88:91]\n"
    "v_mfma_f32_16x16x32_bf16 v[92:95], v[212:215], v[188:191], v[92:95]\n"
    "v_mfma_f32_16x16x32_bf16 v[92:95], v[216:219], v[192:195], v[92:95]\n"
    "v_mfma_f32_16x16x32_bf16 v[96:99], v[220:223], v[188:191], v[96:99]\n"
    "v_mfma_f32_16x16x32_bf16 v[96:99], v[224:227], v[192:195], v[96:99]\n"
    "v_mfma_f32_16x16x32_bf16 v[100:103], v[228:231], v[188:191], v[100:103]\n"
    "v_mfma_f32_16x16x32_bf16 v[100:103], v[232:235], v[192:195], v[100:103]\n"
    "v_mfma_f32_16x16x32_bf16 v[104:107], v[236:239], v[188:191], v[104:107]\n"
    "v_mfma_f32_16x16x32_bf16 v[104:107], v[240:243], v[192:195], v[104:107]\n"
    "v_mfma_f32_16x16x32_bf16 v[108:111], v[212:215], v[196:199], v[108:111]\n"
    "v_mfma_f32_16x16x32_bf16 v[108:111], v[216:219], v[200:203], v[108:111]\n"
    "v_mfma_f32_16x16x32_bf16 v[112:115], v[220:223], v[196:199], v[112:115]\n"
    "v_mfma_f32_16x16x32_bf16 v[112:115], v[224:227], v[200:203], v[112:115]\n"
    "v_mfma_f32_16x16x32_bf16 v[116:119], v[228:231], v[196:199], v[116:119]\n"
    "v_mfma_f32_16x16x32_bf16 v[116:119], v[232:235], v[200:203], v[116:119]\n"
    "v_mfma_f32_16x16x32_bf16 v[120:123], v[236:239], v[196:199], v[120:123]\n"
    "v_mfma_f32_16x16x32_bf16 v[120:123], v[240:243], v[200:203], v[120:123]\n"
    "v_mfma_f32_16x16x32_bf16 v[124:127], v[212:215], v[204:207], v[124:127]\n"
    "v_mfma_f32_16x16x32_bf16 v[124:127], v[216:219], v[208:211], v[124:127]\n"
    "v_mfma_f32_16x16x32_bf16 v[128:131], v[220:223], v[204:207], v[128:131]\n"
    "v_mfma_f32_16x16x32_bf16 v[128:131], v[224:227], v[208:211], v[128:131]\n"
    "v_mfma_f32_16x16x32_bf16 v[132:135], v[228:231], v[204:207], v[132:135]\n"
    "v_mfma_f32_16x16x32_bf16 v[132:135], v[232:235], v[208:211], v[132:135]\n"
    "v_mfma_f32_16x16x32_bf16 v[136:139], v[236:239], v[204:207], v[136:139]\n"
    "v_mfma_f32_16x16x32_bf16 v[136:139], v[240:243], v[208:211], v[136:139]\n"
    "s_setprio 0\n"
    "s_barrier\n"
    "; phase 74: step 43 slot 0 stage 2\n"
    "ds_read_b128 v[180:183], v244 offset:16384\n"
    "ds_read_b128 v[184:187], v245 offset:16384\n"
    "ds_read_b128 v[188:191], v244 offset:18432\n"
    "ds_read_b128 v[192:195], v245 offset:18432\n"
    "s_cmp_lg_u32 s52, 4\n"
    "s_cbranch_scc1 .La110_%=\n"
    "buffer_store_dwordx4 v[28:31], v255, s[72:75], s80 offen nt\n"
    ".La110_%=:\n"
    "ds_read_b128 v[196:199], v244 offset:24576\n"
    "ds_read_b128 v[200:203], v245 offset:24576\n"
    "ds_read_b128 v[204:207], v244 offset:26624\n"
    "ds_read_b128 v[208:211], v245 offset:26624\n"
    "s_add_u32 m0, s51, 0x4000\n"
    "s_xor_b32 s82, s66, 0x2d00000\n"
    "buffer_load_dwordx4 v251, s[40:43], s82 offen  lds\n"
    "ds_read_b64_tr_b16 v[212:213], v246 offset:32768\n"
    "ds_read_b64_tr_b16 v[214:215], v246 offset:33792\n"
    "ds_read_b64_tr_b16 v[216:217], v246 offset:40960\n"
    "ds_read_b64_tr_b16 v[218:219], v246 offset:41984\n"
    "s_add_u32 m0, s51, 0x6000\n"
    "s_xor_b32 s82, s66, 0x2d80000\n"
    "buffer_load_dwordx4 v251, s[40:43], s82 offen  lds\n"
    "ds_read_b64_tr_b16 v[220:221], v247 offset:32768\n"
    "ds_read_b64_tr_b16 v[222:223], v247 offset:33792\n"
    "ds_read_b64_tr_b16 v[224:225], v247 offset:40960\n"
    "ds_read_b64_tr_b16 v[226:227], v247 offset:41984\n"
    "ds_read_b64_tr_b16 v[228:229], v248 offset:32768\n"
    "ds_read_b64_tr_b16 v[230:231], v248 offset:33792\n"
    "ds_read_b64_tr_b16 v[232:233], v248 offset:40960\n"
    "ds_read_b64_tr_b16 v[234:235], v248 offset:41984\n"
    "ds_read_b64_tr_b16 v[236:237], v249 offset:32768\n"
    "ds_read_b64_tr_b16 v[238:239], v249 offset:33792\n"
    "ds_read_b64_tr_b16 v[240:241], v249 offset:40960\n"
    "ds_read_b64_tr_b16 v[242:243], v249 offset:41984\n"
    "s_waitcnt vmcnt(8) lgkmcnt(0)\n"
    "s_barrier\n"
    "s_setprio 1\n"
    "v_mfma_f32_16x16x32_bf16 v[140:143], v[212:215], v[180:183], v[140:143]\n"
    "v_mfma_f32_16x16x32_bf16 v[140:143], v[216:219], v[184:187], v[140:143]\n"
    "v_mfma_f32_16x16x32_bf16 v[144:147], v[220:223], v[180:183], v[144:147]\n"
    "v_mfma_f32_16x16x32_bf16 v[144:147], v[224:227], v[184:187], v[144:147]\n"
    "v_mfma_f32_16x16x32_bf16 v[148:151], v[228:231], v[180:183], v[148:151]\n"
    "v_mfma_f32_16x16x32_bf16 v[148:151], v[232:235], v[184:187], v[148:151]\n"
    "v_mfma_f32_16x16x32_bf16 v[152:155], v[236:239], v[180:183], v[152:155]\n"
    "v_mfma_f32_16x16x32_bf16 v[152:155], v[240:243], v[184:187], v[152:155]\n"
    "v_mfma_f32_16x16x32_bf16 v[156:159], v[212:215], v[188:191], v[156:159]\n"
    "v_mfma_f32_16x16x32_bf16 v[156:159], v[216:219], v[192:195], v[156:159]\n"
    "v_mfma_f32_16x16x32_bf16 v[160:163], v[220:223], v[188:191], v[160:163]\n"
    "v_mfma_f32_16x16x32_bf16 v[160:163], v[224:227], v[192:195], v[160:163]\n"
    "v_mfma_f32_16x16x32_bf16 v[164:167], v[228:231], v[188:191], v[164:167]\n"
    "v_mfma_f32_16x16x32_bf16 v[164:167], v[232:235], v[192:195], v[164:167]\n"
    "v_mfma_f32_16x16x32_bf16 v[168:171], v[236:239], v[188:191], v[168:171]\n"
    "v_mfma_f32_16x16x32_bf16 v[168:171], v[240:243], v[192:195], v[168:171]\n"
    "v_mfma_f32_16x16x32_bf16 v[172:175], v[212:215], v[196:199], v[172:175]\n"
    "v_mfma_f32_16x16x32_bf16 v[172:175], v[216:219], v[200:203], v[172:175]\n"
    "v_mfma_f32_16x16x32_bf16 v[176:179], v[220:223], v[196:199], v[176:179]\n"
    "v_mfma_f32_16x16x32_bf16 v[176:179], v[224:227], v[200:203], v[176:179]\n"
    "v_mfma_f32_16x16x32_bf16 v[16:19], v[228:231], v[196:199], v[16:19]\n"
    "v_mfma_f32_16x16x32_bf16 v[16:19], v[232:235], v[200:203], v[16:19]\n"
    "v_mfma_f32_16x16x32_bf16 v[24:27], v[236:239], v[196:199], v[24:27]\n"
    "v_mfma_f32_16x16x32_bf16 v[24:27], v[240:243], v[200:203], v[24:27]\n"
    "v_mfma_f32_16x16x32_bf16 v[32:35], v[212:215], v[204:207], v[32:35]\n"
    "v_mfma_f32_16x16x32_bf16 v[32:35], v[216:219], v[208:211], v[32:35]\n"
    "v_mfma_f32_16x16x32_bf16 v[40:43], v[220:223], v[204:207], v[40:43]\n"
    "v_mfma_f32_16x16x32_bf16 v[40:43], v[224:227], v[208:211], v[40:43]\n"
    "v_mfma_f32_16x16x32_bf16 v[12:15], v[228:231], v[204:207], v[12:15]\n"
    "v_mfma_f32_16x16x32_bf16 v[12:15], v[232:235], v[208:211], v[12:15]\n"
    "v_mfma_f32_16x16x32_bf16 v[48:51], v[236:239], v[204:207], v[48:51]\n"
    "v_mfma_f32_16x16x32_bf16 v[48:51], v[240:243], v[208:211], v[48:51]\n"
    "s_setprio 0\n"
    "s_barrier\n"
    "; phase 75: step 43 slot 1 stage 1\n"
    "ds_read_b64_tr_b16 v[212:213], v246 offset:49152\n"
    "ds_read_b64_tr_b16 v[214:215], v246 offset:50176\n"
    "s_cmp_lg_u32 s52, 5\n"
    "s_cbranch_scc1 .La111_%=\n"
    "buffer_store_dwordx4 v[28:31], v255, s[72:75], s80 offen nt\n"
    ".La111_%=:\n"
    "ds_read_b64_tr_b16 v[216:217], v246 offset:57344\n"
    "ds_read_b64_tr_b16 v[218:219], v246 offset:58368\n"
    "s_add_u32 m0, s51, 0x8000\n"
    "s_xor_b32 s82, s66, 0x2e00000\n"
    "buffer_load_dwordx4 v251, s[40:43], s82 offen  lds\n"
    "ds_read_b64_tr_b16 v[220:221], v247 offset:49152\n"
    "ds_read_b64_tr_b16 v[222:223], v247 offset:50176\n"
    "s_add_u32 m0, s51, 0xa000\n"
    "s_xor_b32 s82, s66, 0x2e80000\n"
    "buffer_load_dwordx4 v251, s[40:43], s82 offen  lds\n"
    "ds_read_b64_tr_b16 v[224:225], v247 offset:57344\n"
    "ds_read_b64_tr_b16 v[226:227], v247 offset:58368\n"
    "s_add_u32 m0, s50, 0x4000\n"
    "s_xor_b32 s82, s48, 0x201700\n"
    "buffer_load_dwordx4 v250, s[36:39], s82 offen  lds\n"
    "ds_read_b64_tr_b16 v[228:229], v248 offset:49152\n"
    "ds_read_b64_tr_b16 v[230:231], v248 offset:50176\n"
    "s_add_u32 m0, s50, 0x5000\n"
    "s_xor_b32 s82, s48, 0x281700\n"
    "buffer_load_dwordx4 v250, s[36:39], s82 offen  lds\n"
    "ds_read_b64_tr_b16 v[232:233], v248 offset:57344\n"
    "ds_read_b64_tr_b16 v[234:235], v248 offset:58368\n"
    "s_add_u32 m0, s50, 0x6000\n"
    "s_xor_b32 s82, s48, 0x1700\n"
    "buffer_load_dwordx4 v250, s[36:39], s82 offen  lds\n"
    "ds_read_b64_tr_b16 v[236:237], v249 offset:49152\n"
    "ds_read_b64_tr_b16 v[238:239], v249 offset:50176\n"
    "s_add_u32 m0, s50, 0x7000\n"
    "s_xor_b32 s82, s48, 0x81700\n"
    "buffer_load_dwordx4 v250, s[36:39], s82 offen  lds\n"
    "ds_read_b64_tr_b16 v[240:241], v249 offset:57344\n"
    "ds_read_b64_tr_b16 v[242:243], v249 offset:58368\n"
    "s_waitcnt vmcnt(12) lgkmcnt(0)\n"
    "s_barrier\n"
    "s_setprio 1\n"
    "v_mfma_f32_16x16x32_bf16 v[76:79], v[212:215], v[180:183], v[76:79]\n"
    "v_mfma_f32_16x16x32_bf16 v[76:79], v[216:219], v[184:187], v[76:79]\n"
    "v_mfma_f32_16x16x32_bf16 v[80:83], v[220:223], v[180:183], v[80:83]\n"
    "v_mfma_f32_16x16x32_bf16 v[80:83], v[224:227], v[184:187], v[80:83]\n"
    "v_mfma_f32_16x16x32_bf16 v[84:87], v[228:231], v[180:183], v[84:87]\n"
    "v_mfma_f32_16x16x32_bf16 v[84:87], v[232:235], v[184:187], v[84:87]\n"
    "v_mfma_f32_16x16x32_bf16 v[88:91], v[236:239], v[180:183], v[88:91]\n"
    "v_mfma_f32_16x16x32_bf16 v[88:91], v[240:243], v[184:187], v[88:91]\n"
    "v_mfma_f32_16x16x32_bf16 v[92:95], v[212:215], v[188:191], v[92:95]\n"
    "v_mfma_f32_16x16x32_bf16 v[92:95], v[216:219], v[192:195], v[92:95]\n"
    "v_mfma_f32_16x16x32_bf16 v[96:99], v[220:223], v[188:191], v[96:99]\n"
    "v_mfma_f32_16x16x32_bf16 v[96:99], v[224:227], v[192:195], v[96:99]\n"
    "v_mfma_f32_16x16x32_bf16 v[100:103], v[228:231], v[188:191], v[100:103]\n"
    "v_mfma_f32_16x16x32_bf16 v[100:103], v[232:235], v[192:195], v[100:103]\n"
    "v_mfma_f32_16x16x32_bf16 v[104:107], v[236:239], v[188:191], v[104:107]\n"
    "v_mfma_f32_16x16x32_bf16 v[104:107], v[240:243], v[192:195], v[104:107]\n"
    "v_mfma_f32_16x16x32_bf16 v[108:111], v[212:215], v[196:199], v[108:111]\n"
    "v_mfma_f32_16x16x32_bf16 v[108:111], v[216:219], v[200:203], v[108:111]\n"
    "v_mfma_f32_16x16x32_bf16 v[112:115], v[220:223], v[196:199], v[112:115]\n"
    "v_mfma_f32_16x16x32_bf16 v[112:115], v[224:227], v[200:203], v[112:115]\n"
    "v_mfma_f32_16x16x32_bf16 v[116:119], v[228:231], v[196:199], v[116:119]\n"
    "v_mfma_f32_16x16x32_bf16 v[116:119], v[232:235], v[200:203], v[116:119]\n"
    "v_mfma_f32_16x16x32_bf16 v[120:123], v[236:239], v[196:199], v[120:123]\n"
    "v_mfma_f32_16x16x32_bf16 v[120:123], v[240:243], v[200:203], v[120:123]\n"
    "v_cvt_pk_bf16_f32 v76, v76, v77\n"
    "v_cvt_pk_bf16_f32 v77, v78, v79\n"
    "v_mfma_f32_16x16x32_bf16 v[124:127], v[212:215], v[204:207], v[124:127]\n"
    "v_cvt_pk_bf16_f32 v78, v80, v81\n"
    "v_cvt_pk_bf16_f32 v79, v82, v83\n"
    "v_mfma_f32_16x16x32_bf16 v[124:127], v[216:219], v[208:211], v[124:127]\n"
    "v_cvt_pk_bf16_f32 v84, v84, v85\n"
    "v_cvt_pk_bf16_f32 v85, v86, v87\n"
    "v_mfma_f32_16x16x32_bf16 v[128:131], v[220:223], v[204:207], v[128:131]\n"
    "v_cvt_pk_bf16_f32 v86, v88, v89\n"
    "v_cvt_pk_bf16_f32 v87, v90, v91\n"
    "v_mfma_f32_16x16x32_bf16 v[128:131], v[224:227], v[208:211], v[128:131]\n"
    "v_cvt_pk_bf16_f32 v92, v92, v93\n"
    "v_cvt_pk_bf16_f32 v93, v94, v95\n"
    "v_mfma_f32_16x16x32_bf16 v[132:135], v[228:231], v[204:207], v[132:135]\n"
    "v_cvt_pk_bf16_f32 v94, v96, v97\n"
    "v_cvt_pk_bf16_f32 v95, v98, v99\n"
    "v_mfma_f32_16x16x32_bf16 v[132:135], v[232:235], v[208:211], v[132:135]\n"
    "v_cvt_pk_bf16_f32 v100, v100, v101\n"
    "v_cvt_pk_bf16_f32 v101, v102, v103\n"
    "v_mfma_f32_16x16x32_bf16 v[136:139], v[236:239], v[204:207], v[136:139]\n"
    "v_cvt_pk_bf16_f32 v102, v104, v105\n"
    "v_cvt_pk_bf16_f32 v103, v106, v107\n"
    "v_mfma_f32_16x16x32_bf16 v[136:139], v[240:243], v[208:211], v[136:139]\n"
    "s_setprio 0\n"
    "s_barrier\n"
    "; phase 76: step 44 slot 0 stage 2\n"
    "ds_read_b128 v[180:183], v244 offset:32768\n"
    "ds_read_b128 v[184:187], v245 offset:32768\n"
    "ds_read_b128 v[188:191], v244 offset:34816\n"
    "ds_read_b128 v[192:195], v245 offset:34816\n"
    "s_xor_b32 s82, s54, 0x6e000\n"
    "s_add_u32 s82, s82, 0x1100000\n"
    "buffer_store_dwordx4 v[76:79], v252, s[44:47], s82 offen sc0 sc1\n"
    "ds_read_b128 v[196:199], v244 offset:40960\n"
    "ds_read_b128 v[200:203], v245 offset:40960\n"
    "ds_read_b128 v[204:207], v244 offset:43008\n"
    "ds_read_b128 v[208:211], v245 offset:43008\n"
    "s_cmp_lg_u32 s52, 6\n"
    "s_cbranch_scc1 .La112_%=\n"
    "buffer_store_dwordx4 v[28:31], v255, s[72:75], s80 offen nt\n"
    ".La112_%=:\n"
    "ds_read_b64_tr_b16 v[212:213], v246 offset:0\n"
    "ds_read_b64_tr_b16 v[214:215], v246 offset:1024\n"
    "ds_read_b64_tr_b16 v[216:217], v246 offset:8192\n"
    "ds_read_b64_tr_b16 v[218:219], v246 offset:9216\n"
    "s_add_u32 m0, s51, 0xc000\n"
    "s_xor_b32 s82, s66, 0x2f00000\n"
    "buffer_load_dwordx4 v251, s[40:43], s82 offen  lds\n"
    "ds_read_b64_tr_b16 v[220:221], v247 offset:0\n"
    "ds_read_b64_tr_b16 v[222:223], v247 offset:1024\n"
    "ds_read_b64_tr_b16 v[224:225], v247 offset:8192\n"
    "ds_read_b64_tr_b16 v[226:227], v247 offset:9216\n"
    "s_add_u32 m0, s51, 0xe000\n"
    "s_xor_b32 s82, s66, 0x2f80000\n"
    "buffer_load_dwordx4 v251, s[40:43], s82 offen  lds\n"
    "ds_read_b64_tr_b16 v[228:229], v248 offset:0\n"
    "ds_read_b64_tr_b16 v[230:231], v248 offset:1024\n"
    "ds_read_b64_tr_b16 v[232:233], v248 offset:8192\n"
    "ds_read_b64_tr_b16 v[234:235], v248 offset:9216\n"
    "ds_read_b64_tr_b16 v[236:237], v249 offset:0\n"
    "ds_read_b64_tr_b16 v[238:239], v249 offset:1024\n"
    "ds_read_b64_tr_b16 v[240:241], v249 offset:8192\n"
    "ds_read_b64_tr_b16 v[242:243], v249 offset:9216\n"
    "s_waitcnt vmcnt(9) lgkmcnt(0)\n"
    "s_barrier\n"
    "s_setprio 1\n"
    "v_mfma_f32_16x16x32_bf16 v[140:143], v[212:215], v[180:183], v[140:143]\n"
    "v_cvt_pk_bf16_f32 v108, v108, v109\n"
    "v_cvt_pk_bf16_f32 v109, v110, v111\n"
    "v_mfma_f32_16x16x32_bf16 v[140:143], v[216:219], v[184:187], v[140:143]\n"
    "v_cvt_pk_bf16_f32 v110, v112, v113\n"
    "v_cvt_pk_bf16_f32 v111, v114, v115\n"
    "v_mfma_f32_16x16x32_bf16 v[144:147], v[220:223], v[180:183], v[144:147]\n"
    "v_cvt_pk_bf16_f32 v116, v116, v117\n"
    "v_cvt_pk_bf16_f32 v117, v118, v119\n"
    "v_mfma_f32_16x16x32_bf16 v[144:147], v[224:227], v[184:187], v[144:147]\n"
    "v_cvt_pk_bf16_f32 v118, v120, v121\n"
    "v_cvt_pk_bf16_f32 v119, v122, v123\n"
    "v_mfma_f32_16x16x32_bf16 v[148:151], v[228:231], v[180:183], v[148:151]\n"
    "v_mfma_f32_16x16x32_bf16 v[148:151], v[232:235], v[184:187], v[148:151]\n"
    "v_mfma_f32_16x16x32_bf16 v[152:155], v[236:239], v[180:183], v[152:155]\n"
    "v_mfma_f32_16x16x32_bf16 v[152:155], v[240:243], v[184:187], v[152:155]\n"
    "v_mfma_f32_16x16x32_bf16 v[156:159], v[212:215], v[188:191], v[156:159]\n"
    "v_mfma_f32_16x16x32_bf16 v[156:159], v[216:219], v[192:195], v[156:159]\n"
    "v_mfma_f32_16x16x32_bf16 v[160:163], v[220:223], v[188:191], v[160:163]\n"
    "v_mfma_f32_16x16x32_bf16 v[160:163], v[224:227], v[192:195], v[160:163]\n"
    "v_mfma_f32_16x16x32_bf16 v[164:167], v[228:231], v[188:191], v[164:167]\n"
    "v_mfma_f32_16x16x32_bf16 v[164:167], v[232:235], v[192:195], v[164:167]\n"
    "v_mfma_f32_16x16x32_bf16 v[168:171], v[236:239], v[188:191], v[168:171]\n"
    "v_mfma_f32_16x16x32_bf16 v[168:171], v[240:243], v[192:195], v[168:171]\n"
    "v_mfma_f32_16x16x32_bf16 v[172:175], v[212:215], v[196:199], v[172:175]\n"
    "v_mfma_f32_16x16x32_bf16 v[172:175], v[216:219], v[200:203], v[172:175]\n"
    "v_mfma_f32_16x16x32_bf16 v[176:179], v[220:223], v[196:199], v[176:179]\n"
    "v_mfma_f32_16x16x32_bf16 v[176:179], v[224:227], v[200:203], v[176:179]\n"
    "v_mfma_f32_16x16x32_bf16 v[16:19], v[228:231], v[196:199], v[16:19]\n"
    "v_mfma_f32_16x16x32_bf16 v[16:19], v[232:235], v[200:203], v[16:19]\n"
    "v_mfma_f32_16x16x32_bf16 v[24:27], v[236:239], v[196:199], v[24:27]\n"
    "v_mfma_f32_16x16x32_bf16 v[24:27], v[240:243], v[200:203], v[24:27]\n"
    "v_mfma_f32_16x16x32_bf16 v[32:35], v[212:215], v[204:207], v[32:35]\n"
    "v_mfma_f32_16x16x32_bf16 v[32:35], v[216:219], v[208:211], v[32:35]\n"
    "v_mfma_f32_16x16x32_bf16 v[40:43], v[220:223], v[204:207], v[40:43]\n"
    "v_mfma_f32_16x16x32_bf16 v[40:43], v[224:227], v[208:211], v[40:43]\n"
    "v_mfma_f32_16x16x32_bf16 v[12:15], v[228:231], v[204:207], v[12:15]\n"
    "v_mfma_f32_16x16x32_bf16 v[12:15], v[232:235], v[208:211], v[12:15]\n"
    "v_mfma_f32_16x16x32_bf16 v[48:51], v[236:239], v[204:207], v[48:51]\n"
    "v_mfma_f32_16x16x32_bf16 v[48:51], v[240:243], v[208:211], v[48:51]\n"
    "s_setprio 0\n"
    "s_barrier\n"
    "; phase 77: step 45 slot 0 stage 2\n"
    "ds_read_b128 v[180:183], v244 offset:0\n"
    "ds_read_b128 v[184:187], v245 offset:0\n"
    "ds_read_b128 v[188:191], v244 offset:2048\n"
    "s_xor_b32 s82, s54, 0x6c000\n"
    "s_add_u32 s82, s82, 0x1100000\n"
    "buffer_store_dwordx4 v[84:87], v252, s[44:47], s82 offen sc0 sc1\n"
    "ds_read_b128 v[192:195], v245 offset:2048\n"
    "ds_read_b128 v[196:199], v244 offset:8192\n"
    "ds_read_b128 v[200:203], v245 offset:8192\n"
    "s_cmp_lg_u32 s52, 7\n"
    "s_cbranch_scc1 .La113_%=\n"
    "buffer_store_dwordx4 v[28:31], v255, s[72:75], s80 offen nt\n"
    ".La113_%=:\n"
    "ds_read_b128 v[204:207], v244 offset:10240\n"
    "ds_read_b128 v[208:211], v245 offset:10240\n"
    "ds_read_b64_tr_b16 v[212:213], v246 offset:16384\n"
    "s_add_u32 m0, s50, 0x8000\n"
    "s_xor_b32 s82, s48, 0x201780\n"
    "buffer_load_dwordx4 v250, s[36:39], s82 offen  lds\n"
    "ds_read_b64_tr_b16 v[214:215], v246 offset:17408\n"
    "ds_read_b64_tr_b16 v[216:217], v246 offset:24576\n"
    "ds_read_b64_tr_b16 v[218:219], v246 offset:25600\n"
    "s_add_u32 m0, s50, 0x9000\n"
    "s_xor_b32 s82, s48, 0x281780\n"
    "buffer_load_dwordx4 v250, s[36:39], s82 offen  lds\n"
    "ds_read_b64_tr_b16 v[220:221], v247 offset:16384\n"
    "ds_read_b64_tr_b16 v[222:223], v247 offset:17408\n"
    "ds_read_b64_tr_b16 v[224:225], v247 offset:24576\n"
    "s_add_u32 m0, s50, 0xa000\n"
    "s_xor_b32 s82, s48, 0x1780\n"
    "buffer_load_dwordx4 v250, s[36:39], s82 offen  lds\n"
    "ds_read_b64_tr_b16 v[226:227], v247 offset:25600\n"
    "ds_read_b64_tr_b16 v[228:229], v248 offset:16384\n"
    "ds_read_b64_tr_b16 v[230:231], v248 offset:17408\n"
    "s_add_u32 m0, s50, 0xb000\n"
    "s_xor_b32 s82, s48, 0x81780\n"
    "buffer_load_dwordx4 v250, s[36:39], s82 offen  lds\n"
    "ds_read_b64_tr_b16 v[232:233], v248 offset:24576\n"
    "ds_read_b64_tr_b16 v[234:235], v248 offset:25600\n"
    "ds_read_b64_tr_b16 v[236:237], v249 offset:16384\n"
    "s_add_u32 m0, s51, 0x0\n"
    "s_xor_b32 s82, s66, 0x3000000\n"
    "buffer_load_dwordx4 v251, s[40:43], s82 offen  lds\n"
    "ds_read_b64_tr_b16 v[238:239], v249 offset:17408\n"
    "ds_read_b64_tr_b16 v[240:241], v249 offset:24576\n"
    "ds_read_b64_tr_b16 v[242:243], v249 offset:25600\n"
    "s_add_u32 m0, s51, 0x2000\n"
    "s_xor_b32 s82, s66, 0x3080000\n"
    "buffer_load_dwordx4 v251, s[40:43], s82 offen  lds\n"
    "s_waitcnt vmcnt(14) lgkmcnt(0)\n"
    "s_barrier\n"
    "s_setprio 1\n"
    "v_mfma_f32_16x16x32_bf16 v[140:143], v[212:215], v[180:183], v[140:143]\n"
    "v_mfma_f32_16x16x32_bf16 v[140:143], v[216:219], v[184:187], v[140:143]\n"
    "v_mfma_f32_16x16x32_bf16 v[144:147], v[220:223], v[180:183], v[144:147]\n"
    "v_mfma_f32_16x16x32_bf16 v[144:147], v[224:227], v[184:187], v[144:147]\n"
    "v_mfma_f32_16x16x32_bf16 v[148:151], v[228:231], v[180:183], v[148:151]\n"
    "v_mfma_f32_16x16x32_bf16 v[148:151], v[232:235], v[184:187], v[148:151]\n"
    "v_mfma_f32_16x16x32_bf16 v[152:155], v[236:239], v[180:183], v[152:155]\n"
    "v_mfma_f32_16x16x32_bf16 v[152:155], v[240:243], v[184:187], v[152:155]\n"
    "v_mfma_f32_16x16x32_bf16 v[156:159], v[212:215], v[188:191], v[156:159]\n"
    "v_mfma_f32_16x16x32_bf16 v[156:159], v[216:219], v[192:195], v[156:159]\n"
    "v_mfma_f32_16x16x32_bf16 v[160:163], v[220:223], v[188:191], v[160:163]\n"
    "v_mfma_f32_16x16x32_bf16 v[160:163], v[224:227], v[192:195], v[160:163]\n"
    "v_mfma_f32_16x16x32_bf16 v[164:167], v[228:231], v[188:191], v[164:167]\n"
    "v_mfma_f32_16x16x32_bf16 v[164:167], v[232:235], v[192:195], v[164:167]\n"
    "v_mfma_f32_16x16x32_bf16 v[168:171], v[236:239], v[188:191], v[168:171]\n"
    "v_mfma_f32_16x16x32_bf16 v[168:171], v[240:243], v[192:195], v[168:171]\n"
    "v_mfma_f32_16x16x32_bf16 v[172:175], v[212:215], v[196:199], v[172:175]\n"
    "v_mfma_f32_16x16x32_bf16 v[172:175], v[216:219], v[200:203], v[172:175]\n"
    "v_mfma_f32_16x16x32_bf16 v[176:179], v[220:223], v[196:199], v[176:179]\n"
    "v_mfma_f32_16x16x32_bf16 v[176:179], v[224:227], v[200:203], v[176:179]\n"
    "v_mfma_f32_16x16x32_bf16 v[16:19], v[228:231], v[196:199], v[16:19]\n"
    "v_mfma_f32_16x16x32_bf16 v[16:19], v[232:235], v[200:203], v[16:19]\n"
    "v_mfma_f32_16x16x32_bf16 v[24:27], v[236:239], v[196:199], v[24:27]\n"
    "v_mfma_f32_16x16x32_bf16 v[24:27], v[240:243], v[200:203], v[24:27]\n"
    "v_mfma_f32_16x16x32_bf16 v[32:35], v[212:215], v[204:207], v[32:35]\n"
    "v_mfma_f32_16x16x32_bf16 v[32:35], v[216:219], v[208:211], v[32:35]\n"
    "v_mfma_f32_16x16x32_bf16 v[40:43], v[220:223], v[204:207], v[40:43]\n"
    "v_mfma_f32_16x16x32_bf16 v[40:43], v[224:227], v[208:211], v[40:43]\n"
    "v_mfma_f32_16x16x32_bf16 v[12:15], v[228:231], v[204:207], v[12:15]\n"
    "v_mfma_f32_16x16x32_bf16 v[12:15], v[232:235], v[208:211], v[12:15]\n"
    "v_mfma_f32_16x16x32_bf16 v[48:51], v[236:239], v[204:207], v[48:51]\n"
    "v_mfma_f32_16x16x32_bf16 v[48:51], v[240:243], v[208:211], v[48:51]\n"
    "s_setprio 0\n"
    "s_waitcnt vmcnt(10)\n"
    "s_barrier\n"
    "; phase 78: step 46 slot 0 stage 2\n"
    "ds_read_b128 v[180:183], v244 offset:16384\n"
    "ds_read_b128 v[184:187], v245 offset:16384\n"
    "ds_read_b128 v[188:191], v244 offset:18432\n"
    "s_xor_b32 s82, s54, 0x4a000\n"
    "s_add_u32 s82, s82, 0x1100000\n"
    "buffer_store_dwordx4 v[92:95], v252, s[44:47], s82 offen sc0 sc1\n"
    "ds_read_b128 v[192:195], v245 offset:18432\n"
    "ds_read_b128 v[196:199], v244 offset:24576\n"
    "ds_read_b128 v[200:203], v245 offset:24576\n"
    "s_cmp_lg_u32 s52, 0\n"
    "s_cbranch_scc1 .La114_%=\n"
    "s_xor_b32 s82, s80, 0x4000\n"
    "buffer_store_dwordx4 v[52:55], v255, s[72:75], s82 offen nt\n"
    ".La114_%=:\n"
    "ds_read_b128 v[204:207], v244 offset:26624\n"
    "ds_read_b128 v[208:211], v245 offset:26624\n"
    "ds_read_b64_tr_b16 v[212:213], v246 offset:32768\n"
    "s_add_u32 m0, s50, 0x0\n"
    "s_xor_b32 s82, s48, 0x201800\n"
    "buffer_load_dwordx4 v250, s[36:39], s82 offen  lds\n"
    "ds_read_b64_tr_b16 v[214:215], v246 offset:33792\n"
    "ds_read_b64_tr_b16 v[216:217], v246 offset:40960\n"
    "ds_read_b64_tr_b16 v[218:219], v246 offset:41984\n"
    "s_add_u32 m0, s50, 0x1000\n"
    "s_xor_b32 s82, s48, 0x281800\n"
    "buffer_load_dwordx4 v250, s[36:39], s82 offen  lds\n"
    "ds_read_b64_tr_b16 v[220:221], v247 offset:32768\n"
    "ds_read_b64_tr_b16 v[222:223], v247 offset:33792\n"
    "ds_read_b64_tr_b16 v[224:225], v247 offset:40960\n"
    "s_add_u32 m0, s50, 0x2000\n"
    "s_xor_b32 s82, s48, 0x1800\n"
    "buffer_load_dwordx4 v250, s[36:39], s82 offen  lds\n"
    "ds_read_b64_tr_b16 v[226:227], v247 offset:41984\n"
    "ds_read_b64_tr_b16 v[228:229], v248 offset:32768\n"
    "ds_read_b64_tr_b16 v[230:231], v248 offset:33792\n"
    "s_add_u32 m0, s50, 0x3000\n"
    "s_xor_b32 s82, s48, 0x81800\n"
    "buffer_load_dwordx4 v250, s[36:39], s82 offen  lds\n"
    "ds_read_b64_tr_b16 v[232:233], v248 offset:40960\n"
    "ds_read_b64_tr_b16 v[234:235], v248 offset:41984\n"
    "ds_read_b64_tr_b16 v[236:237], v249 offset:32768\n"
    "s_add_u32 m0, s51, 0x4000\n"
    "s_xor_b32 s82, s66, 0x3100000\n"
    "buffer_load_dwordx4 v251, s[40:43], s82 offen  lds\n"
    "ds_read_b64_tr_b16 v[238:239], v249 offset:33792\n"
    "ds_read_b64_tr_b16 v[240:241], v249 offset:40960\n"
    "ds_read_b64_tr_b16 v[242:243], v249 offset:41984\n"
    "s_add_u32 m0, s51, 0x6000\n"
    "s_xor_b32 s82, s66, 0x3180000\n"
    "buffer_load_dwordx4 v251, s[40:43], s82 offen  lds\n"
    "s_waitcnt vmcnt(14) lgkmcnt(0)\n"
    "s_barrier\n"
    "s_setprio 1\n"
    "v_mfma_f32_16x16x32_bf16 v[140:143], v[212:215], v[180:183], v[140:143]\n"
    "v_mfma_f32_16x16x32_bf16 v[140:143], v[216:219], v[184:187], v[140:143]\n"
    "v_mfma_f32_16x16x32_bf16 v[144:147], v[220:223], v[180:183], v[144:147]\n"
    "v_mfma_f32_16x16x32_bf16 v[144:147], v[224:227], v[184:187], v[144:147]\n"
    "v_mfma_f32_16x16x32_bf16 v[148:151], v[228:231], v[180:183], v[148:151]\n"
    "v_mfma_f32_16x16x32_bf16 v[148:151], v[232:235], v[184:187], v[148:151]\n"
    "v_mfma_f32_16x16x32_bf16 v[152:155], v[236:239], v[180:183], v[152:155]\n"
    "v_mfma_f32_16x16x32_bf16 v[152:155], v[240:243], v[184:187], v[152:155]\n"
    "v_mfma_f32_16x16x32_bf16 v[156:159], v[212:215], v[188:191], v[156:159]\n"
    "v_mfma_f32_16x16x32_bf16 v[156:159], v[216:219], v[192:195], v[156:159]\n"
    "v_mfma_f32_16x16x32_bf16 v[160:163], v[220:223], v[188:191], v[160:163]\n"
    "v_mfma_f32_16x16x32_bf16 v[160:163], v[224:227], v[192:195], v[160:163]\n"
    "v_mfma_f32_16x16x32_bf16 v[164:167], v[228:231], v[188:191], v[164:167]\n"
    "v_mfma_f32_16x16x32_bf16 v[164:167], v[232:235], v[192:195], v[164:167]\n"
    "v_mfma_f32_16x16x32_bf16 v[168:171], v[236:239], v[188:191], v[168:171]\n"
    "v_mfma_f32_16x16x32_bf16 v[168:171], v[240:243], v[192:195], v[168:171]\n"
    "v_mfma_f32_16x16x32_bf16 v[172:175], v[212:215], v[196:199], v[172:175]\n"
    "v_mfma_f32_16x16x32_bf16 v[172:175], v[216:219], v[200:203], v[172:175]\n"
    "v_mfma_f32_16x16x32_bf16 v[176:179], v[220:223], v[196:199], v[176:179]\n"
    "v_mfma_f32_16x16x32_bf16 v[176:179], v[224:227], v[200:203], v[176:179]\n"
    "v_mfma_f32_16x16x32_bf16 v[16:19], v[228:231], v[196:199], v[16:19]\n"
    "v_mfma_f32_16x16x32_bf16 v[16:19], v[232:235], v[200:203], v[16:19]\n"
    "v_mfma_f32_16x16x32_bf16 v[24:27], v[236:239], v[196:199], v[24:27]\n"
    "v_mfma_f32_16x16x32_bf16 v[24:27], v[240:243], v[200:203], v[24:27]\n"
    "v_mfma_f32_16x16x32_bf16 v[32:35], v[212:215], v[204:207], v[32:35]\n"
    "v_mfma_f32_16x16x32_bf16 v[32:35], v[216:219], v[208:211], v[32:35]\n"
    "v_mfma_f32_16x16x32_bf16 v[40:43], v[220:223], v[204:207], v[40:43]\n"
    "v_mfma_f32_16x16x32_bf16 v[40:43], v[224:227], v[208:211], v[40:43]\n"
    "v_mfma_f32_16x16x32_bf16 v[12:15], v[228:231], v[204:207], v[12:15]\n"
    "v_mfma_f32_16x16x32_bf16 v[12:15], v[232:235], v[208:211], v[12:15]\n"
    "v_mfma_f32_16x16x32_bf16 v[48:51], v[236:239], v[204:207], v[48:51]\n"
    "v_mfma_f32_16x16x32_bf16 v[48:51], v[240:243], v[208:211], v[48:51]\n"
    "s_setprio 0\n"
    "s_waitcnt vmcnt(9)\n"
    "s_barrier\n"
    "; phase 79: step 47 slot 0 stage 2\n"
    "ds_read_b128 v[180:183], v244 offset:32768\n"
    "ds_read_b128 v[184:187], v245 offset:32768\n"
    "ds_read_b128 v[188:191], v244 offset:34816\n"
    "s_xor_b32 s82, s54, 0x48000\n"
    "s_add_u32 s82, s82, 0x1100000\n"
    "buffer_store_dwordx4 v[100:103], v252, s[44:47], s82 offen sc0 sc1\n"
    "ds_read_b128 v[192:195], v245 offset:34816\n"
    "ds_read_b128 v[196:199], v244 offset:40960\n"
    "ds_read_b128 v[200:203], v245 offset:40960\n"
    "s_cmp_lg_u32 s52, 1\n"
    "s_cbranch_scc1 .La115_%=\n"
    "s_xor_b32 s82, s80, 0x4000\n"
    "buffer_store_dwordx4 v[52:55], v255, s[72:75], s82 offen nt\n"
    ".La115_%=:\n"
    "ds_read_b128 v[204:207], v244 offset:43008\n"
    "ds_read_b128 v[208:211], v245 offset:43008\n"
    "ds_read_b64_tr_b16 v[212:213], v246 offset:49152\n"
    "s_add_u32 m0, s50, 0x4000\n"
    "s_xor_b32 s82, s48, 0x201880\n"
    "buffer_load_dwordx4 v250, s[36:39], s82 offen  lds\n"
    "ds_read_b64_tr_b16 v[214:215], v246 offset:50176\n"
    "ds_read_b64_tr_b16 v[216:217], v246 offset:57344\n"
    "ds_read_b64_tr_b16 v[218:219], v246 offset:58368\n"
    "s_add_u32 m0, s50, 0x5000\n"
    "s_xor_b32 s82, s48, 0x281880\n"
    "buffer_load_dwordx4 v250, s[36:39], s82 offen  lds\n"
    "ds_read_b64_tr_b16 v[220:221], v247 offset:49152\n"
    "ds_read_b64_tr_b16 v[222:223], v247 offset:50176\n"
    "ds_read_b64_tr_b16 v[224:225], v247 offset:57344\n"
    "s_add_u32 m0, s50, 0x6000\n"
    "s_xor_b32 s82, s48, 0x1880\n"
    "buffer_load_dwordx4 v250, s[36:39], s82 offen  lds\n"
    "ds_read_b64_tr_b16 v[226:227], v247 offset:58368\n"
    "ds_read_b64_tr_b16 v[228:229], v248 offset:49152\n"
    "ds_read_b64_tr_b16 v[230:231], v248 offset:50176\n"
    "s_add_u32 m0, s50, 0x7000\n"
    "s_xor_b32 s82, s48, 0x81880\n"
    "buffer_load_dwordx4 v250, s[36:39], s82 offen  lds\n"
    "ds_read_b64_tr_b16 v[232:233], v248 offset:57344\n"
    "ds_read_b64_tr_b16 v[234:235], v248 offset:58368\n"
    "ds_read_b64_tr_b16 v[236:237], v249 offset:49152\n"
    "s_add_u32 m0, s51, 0x8000\n"
    "s_xor_b32 s82, s66, 0x3200000\n"
    "buffer_load_dwordx4 v251, s[40:43], s82 offen  lds\n"
    "ds_read_b64_tr_b16 v[238:239], v249 offset:50176\n"
    "ds_read_b64_tr_b16 v[240:241], v249 offset:57344\n"
    "ds_read_b64_tr_b16 v[242:243], v249 offset:58368\n"
    "s_add_u32 m0, s51, 0xa000\n"
    "s_xor_b32 s82, s66, 0x3280000\n"
    "buffer_load_dwordx4 v251, s[40:43], s82 offen  lds\n"
    "s_waitcnt vmcnt(14) lgkmcnt(0)\n"
    "s_barrier\n"
    "s_setprio 1\n"
    "v_mfma_f32_16x16x32_bf16 v[140:143], v[212:215], v[180:183], v[140:143]\n"
    "v_mfma_f32_16x16x32_bf16 v[140:143], v[216:219], v[184:187], v[140:143]\n"
    "v_mfma_f32_16x16x32_bf16 v[144:147], v[220:223], v[180:183], v[144:147]\n"
    "v_mfma_f32_16x16x32_bf16 v[144:147], v[224:227], v[184:187], v[144:147]\n"
    "v_mfma_f32_16x16x32_bf16 v[148:151], v[228:231], v[180:183], v[148:151]\n"
    "v_mfma_f32_16x16x32_bf16 v[148:151], v[232:235], v[184:187], v[148:151]\n"
    "v_mfma_f32_16x16x32_bf16 v[152:155], v[236:239], v[180:183], v[152:155]\n"
    "v_mfma_f32_16x16x32_bf16 v[152:155], v[240:243], v[184:187], v[152:155]\n"
    "v_mfma_f32_16x16x32_bf16 v[156:159], v[212:215], v[188:191], v[156:159]\n"
    "v_mfma_f32_16x16x32_bf16 v[156:159], v[216:219], v[192:195], v[156:159]\n"
    "v_mfma_f32_16x16x32_bf16 v[160:163], v[220:223], v[188:191], v[160:163]\n"
    "v_mfma_f32_16x16x32_bf16 v[160:163], v[224:227], v[192:195], v[160:163]\n"
    "v_mfma_f32_16x16x32_bf16 v[164:167], v[228:231], v[188:191], v[164:167]\n"
    "v_mfma_f32_16x16x32_bf16 v[164:167], v[232:235], v[192:195], v[164:167]\n"
    "v_mfma_f32_16x16x32_bf16 v[168:171], v[236:239], v[188:191], v[168:171]\n"
    "v_mfma_f32_16x16x32_bf16 v[168:171], v[240:243], v[192:195], v[168:171]\n"
    "v_mfma_f32_16x16x32_bf16 v[172:175], v[212:215], v[196:199], v[172:175]\n"
    "v_mfma_f32_16x16x32_bf16 v[172:175], v[216:219], v[200:203], v[172:175]\n"
    "v_mfma_f32_16x16x32_bf16 v[176:179], v[220:223], v[196:199], v[176:179]\n"
    "v_mfma_f32_16x16x32_bf16 v[176:179], v[224:227], v[200:203], v[176:179]\n"
    "v_mfma_f32_16x16x32_bf16 v[16:19], v[228:231], v[196:199], v[16:19]\n"
    "v_mfma_f32_16x16x32_bf16 v[16:19], v[232:235], v[200:203], v[16:19]\n"
    "v_mfma_f32_16x16x32_bf16 v[24:27], v[236:239], v[196:199], v[24:27]\n"
    "v_mfma_f32_16x16x32_bf16 v[24:27], v[240:243], v[200:203], v[24:27]\n"
    "v_mfma_f32_16x16x32_bf16 v[32:35], v[212:215], v[204:207], v[32:35]\n"
    "v_mfma_f32_16x16x32_bf16 v[32:35], v[216:219], v[208:211], v[32:35]\n"
    "v_mfma_f32_16x16x32_bf16 v[40:43], v[220:223], v[204:207], v[40:43]\n"
    "v_mfma_f32_16x16x32_bf16 v[40:43], v[224:227], v[208:211], v[40:43]\n"
    "v_mfma_f32_16x16x32_bf16 v[12:15], v[228:231], v[204:207], v[12:15]\n"
    "v_mfma_f32_16x16x32_bf16 v[12:15], v[232:235], v[208:211], v[12:15]\n"
    "v_mfma_f32_16x16x32_bf16 v[48:51], v[236:239], v[204:207], v[48:51]\n"
    "v_mfma_f32_16x16x32_bf16 v[48:51], v[240:243], v[208:211], v[48:51]\n"
    "s_setprio 0\n"
    "s_waitcnt vmcnt(9)\n"
    "s_barrier\n"
    "; phase 80: step 48 slot 0 stage 2\n"
    "ds_read_b128 v[180:183], v244 offset:0\n"
    "ds_read_b128 v[184:187], v245 offset:0\n"
    "ds_read_b128 v[188:191], v244 offset:2048\n"
    "s_xor_b32 s82, s54, 0x26000\n"
    "s_add_u32 s82, s82, 0x1100000\n"
    "buffer_store_dwordx4 v[108:111], v252, s[44:47], s82 offen sc0 sc1\n"
    "ds_read_b128 v[192:195], v245 offset:2048\n"
    "ds_read_b128 v[196:199], v244 offset:8192\n"
    "ds_read_b128 v[200:203], v245 offset:8192\n"
    "s_cmp_lg_u32 s52, 2\n"
    "s_cbranch_scc1 .La116_%=\n"
    "s_xor_b32 s82, s80, 0x4000\n"
    "buffer_store_dwordx4 v[52:55], v255, s[72:75], s82 offen nt\n"
    ".La116_%=:\n"
    "ds_read_b128 v[204:207], v244 offset:10240\n"
    "ds_read_b128 v[208:211], v245 offset:10240\n"
    "ds_read_b64_tr_b16 v[212:213], v246 offset:0\n"
    "s_add_u32 m0, s50, 0x8000\n"
    "s_xor_b32 s82, s48, 0x201900\n"
    "buffer_load_dwordx4 v250, s[36:39], s82 offen  lds\n"
    "ds_read_b64_tr_b16 v[214:215], v246 offset:1024\n"
    "ds_read_b64_tr_b16 v[216:217], v246 offset:8192\n"
    "ds_read_b64_tr_b16 v[218:219], v246 offset:9216\n"
    "s_add_u32 m0, s50, 0x9000\n"
    "s_xor_b32 s82, s48, 0x281900\n"
    "buffer_load_dwordx4 v250, s[36:39], s82 offen  lds\n"
    "ds_read_b64_tr_b16 v[220:221], v247 offset:0\n"
    "ds_read_b64_tr_b16 v[222:223], v247 offset:1024\n"
    "ds_read_b64_tr_b16 v[224:225], v247 offset:8192\n"
    "s_add_u32 m0, s50, 0xa000\n"
    "s_xor_b32 s82, s48, 0x1900\n"
    "buffer_load_dwordx4 v250, s[36:39], s82 offen  lds\n"
    "ds_read_b64_tr_b16 v[226:227], v247 offset:9216\n"
    "ds_read_b64_tr_b16 v[228:229], v248 offset:0\n"
    "ds_read_b64_tr_b16 v[230:231], v248 offset:1024\n"
    "s_add_u32 m0, s50, 0xb000\n"
    "s_xor_b32 s82, s48, 0x81900\n"
    "buffer_load_dwordx4 v250, s[36:39], s82 offen  lds\n"
    "ds_read_b64_tr_b16 v[232:233], v248 offset:8192\n"
    "ds_read_b64_tr_b16 v[234:235], v248 offset:9216\n"
    "ds_read_b64_tr_b16 v[236:237], v249 offset:0\n"
    "s_add_u32 m0, s51, 0xc000\n"
    "s_xor_b32 s82, s66, 0x3300000\n"
    "buffer_load_dwordx4 v251, s[40:43], s82 offen  lds\n"
    "ds_read_b64_tr_b16 v[238:239], v249 offset:1024\n"
    "ds_read_b64_tr_b16 v[240:241], v249 offset:8192\n"
    "ds_read_b64_tr_b16 v[242:243], v249 offset:9216\n"
    "s_add_u32 m0, s51, 0xe000\n"
    "s_xor_b32 s82, s66, 0x3380000\n"
    "buffer_load_dwordx4 v251, s[40:43], s82 offen  lds\n"
    "s_waitcnt vmcnt(14) lgkmcnt(0)\n"
    "s_barrier\n"
    "s_setprio 1\n"
    "v_mfma_f32_16x16x32_bf16 v[140:143], v[212:215], v[180:183], v[140:143]\n"
    "v_mfma_f32_16x16x32_bf16 v[140:143], v[216:219], v[184:187], v[140:143]\n"
    "v_mfma_f32_16x16x32_bf16 v[144:147], v[220:223], v[180:183], v[144:147]\n"
    "v_mfma_f32_16x16x32_bf16 v[144:147], v[224:227], v[184:187], v[144:147]\n"
    "v_mfma_f32_16x16x32_bf16 v[148:151], v[228:231], v[180:183], v[148:151]\n"
    "v_mfma_f32_16x16x32_bf16 v[148:151], v[232:235], v[184:187], v[148:151]\n"
    "v_mfma_f32_16x16x32_bf16 v[152:155], v[236:239], v[180:183], v[152:155]\n"
    "v_mfma_f32_16x16x32_bf16 v[152:155], v[240:243], v[184:187], v[152:155]\n"
    "v_mfma_f32_16x16x32_bf16 v[156:159], v[212:215], v[188:191], v[156:159]\n"
    "v_mfma_f32_16x16x32_bf16 v[156:159], v[216:219], v[192:195], v[156:159]\n"
    "v_mfma_f32_16x16x32_bf16 v[160:163], v[220:223], v[188:191], v[160:163]\n"
    "v_mfma_f32_16x16x32_bf16 v[160:163], v[224:227], v[192:195], v[160:163]\n"
    "v_mfma_f32_16x16x32_bf16 v[164:167], v[228:231], v[188:191], v[164:167]\n"
    "v_mfma_f32_16x16x32_bf16 v[164:167], v[232:235], v[192:195], v[164:167]\n"
    "v_mfma_f32_16x16x32_bf16 v[168:171], v[236:239], v[188:191], v[168:171]\n"
    "v_mfma_f32_16x16x32_bf16 v[168:171], v[240:243], v[192:195], v[168:171]\n"
    "v_mfma_f32_16x16x32_bf16 v[172:175], v[212:215], v[196:199], v[172:175]\n"
    "v_mfma_f32_16x16x32_bf16 v[172:175], v[216:219], v[200:203], v[172:175]\n"
    "v_mfma_f32_16x16x32_bf16 v[176:179], v[220:223], v[196:199], v[176:179]\n"
    "v_mfma_f32_16x16x32_bf16 v[176:179], v[224:227], v[200:203], v[176:179]\n"
    "v_mfma_f32_16x16x32_bf16 v[16:19], v[228:231], v[196:199], v[16:19]\n"
    "v_mfma_f32_16x16x32_bf16 v[16:19], v[232:235], v[200:203], v[16:19]\n"
    "v_mfma_f32_16x16x32_bf16 v[24:27], v[236:239], v[196:199], v[24:27]\n"
    "v_mfma_f32_16x16x32_bf16 v[24:27], v[240:243], v[200:203], v[24:27]\n"
    "v_mfma_f32_16x16x32_bf16 v[32:35], v[212:215], v[204:207], v[32:35]\n"
    "v_mfma_f32_16x16x32_bf16 v[32:35], v[216:219], v[208:211], v[32:35]\n"
    "v_mfma_f32_16x16x32_bf16 v[40:43], v[220:223], v[204:207], v[40:43]\n"
    "v_mfma_f32_16x16x32_bf16 v[40:43], v[224:227], v[208:211], v[40:43]\n"
    "v_mfma_f32_16x16x32_bf16 v[12:15], v[228:231], v[204:207], v[12:15]\n"
    "v_mfma_f32_16x16x32_bf16 v[12:15], v[232:235], v[208:211], v[12:15]\n"
    "v_mfma_f32_16x16x32_bf16 v[48:51], v[236:239], v[204:207], v[48:51]\n"
    "v_mfma_f32_16x16x32_bf16 v[48:51], v[240:243], v[208:211], v[48:51]\n"
    "s_setprio 0\n"
    "s_waitcnt vmcnt(9)\n"
    "s_barrier\n"
    "; phase 81: step 49 slot 0 stage 2\n"
    "ds_read_b128 v[180:183], v244 offset:16384\n"
    "ds_read_b128 v[184:187], v245 offset:16384\n"
    "ds_read_b128 v[188:191], v244 offset:18432\n"
    "s_xor_b32 s82, s54, 0x24000\n"
    "s_add_u32 s82, s82, 0x1100000\n"
    "buffer_store_dwordx4 v[116:119], v252, s[44:47], s82 offen sc0 sc1\n"
    "ds_read_b128 v[192:195], v245 offset:18432\n"
    "ds_read_b128 v[196:199], v244 offset:24576\n"
    "ds_read_b128 v[200:203], v245 offset:24576\n"
    "s_cmp_lg_u32 s52, 3\n"
    "s_cbranch_scc1 .La117_%=\n"
    "s_xor_b32 s82, s80, 0x4000\n"
    "buffer_store_dwordx4 v[52:55], v255, s[72:75], s82 offen nt\n"
    ".La117_%=:\n"
    "ds_read_b128 v[204:207], v244 offset:26624\n"
    "ds_read_b128 v[208:211], v245 offset:26624\n"
    "ds_read_b64_tr_b16 v[212:213], v246 offset:16384\n"
    "s_add_u32 m0, s50, 0x0\n"
    "s_xor_b32 s82, s48, 0x201980\n"
    "buffer_load_dwordx4 v250, s[36:39], s82 offen  lds\n"
    "ds_read_b64_tr_b16 v[214:215], v246 offset:17408\n"
    "ds_read_b64_tr_b16 v[216:217], v246 offset:24576\n"
    "ds_read_b64_tr_b16 v[218:219], v246 offset:25600\n"
    "s_add_u32 m0, s50, 0x1000\n"
    "s_xor_b32 s82, s48, 0x281980\n"
    "buffer_load_dwordx4 v250, s[36:39], s82 offen  lds\n"
    "ds_read_b64_tr_b16 v[220:221], v247 offset:16384\n"
    "ds_read_b64_tr_b16 v[222:223], v247 offset:17408\n"
    "ds_read_b64_tr_b16 v[224:225], v247 offset:24576\n"
    "s_add_u32 m0, s50, 0x2000\n"
    "s_xor_b32 s82, s48, 0x1980\n"
    "buffer_load_dwordx4 v250, s[36:39], s82 offen  lds\n"
    "ds_read_b64_tr_b16 v[226:227], v247 offset:25600\n"
    "ds_read_b64_tr_b16 v[228:229], v248 offset:16384\n"
    "ds_read_b64_tr_b16 v[230:231], v248 offset:17408\n"
    "s_add_u32 m0, s50, 0x3000\n"
    "s_xor_b32 s82, s48, 0x81980\n"
    "buffer_load_dwordx4 v250, s[36:39], s82 offen  lds\n"
    "ds_read_b64_tr_b16 v[232:233], v248 offset:24576\n"
    "ds_read_b64_tr_b16 v[234:235], v248 offset:25600\n"
    "ds_read_b64_tr_b16 v[236:237], v249 offset:16384\n"
    "s_add_u32 m0, s51, 0x0\n"
    "s_xor_b32 s82, s66, 0x3400000\n"
    "buffer_load_dwordx4 v251, s[40:43], s82 offen  lds\n"
    "ds_read_b64_tr_b16 v[238:239], v249 offset:17408\n"
    "ds_read_b64_tr_b16 v[240:241], v249 offset:24576\n"
    "ds_read_b64_tr_b16 v[242:243], v249 offset:25600\n"
    "s_add_u32 m0, s51, 0x2000\n"
    "s_xor_b32 s82, s66, 0x3480000\n"
    "buffer_load_dwordx4 v251, s[40:43], s82 offen  lds\n"
    "s_waitcnt vmcnt(14) lgkmcnt(0)\n"
    "s_barrier\n"
    "s_setprio 1\n"
    "v_mfma_f32_16x16x32_bf16 v[140:143], v[212:215], v[180:183], v[140:143]\n"
    "v_mfma_f32_16x16x32_bf16 v[140:143], v[216:219], v[184:187], v[140:143]\n"
    "v_mfma_f32_16x16x32_bf16 v[144:147], v[220:223], v[180:183], v[144:147]\n"
    "v_mfma_f32_16x16x32_bf16 v[144:147], v[224:227], v[184:187], v[144:147]\n"
    "v_mfma_f32_16x16x32_bf16 v[148:151], v[228:231], v[180:183], v[148:151]\n"
    "v_mfma_f32_16x16x32_bf16 v[148:151], v[232:235], v[184:187], v[148:151]\n"
    "v_mfma_f32_16x16x32_bf16 v[152:155], v[236:239], v[180:183], v[152:155]\n"
    "v_mfma_f32_16x16x32_bf16 v[152:155], v[240:243], v[184:187], v[152:155]\n"
    "v_mfma_f32_16x16x32_bf16 v[156:159], v[212:215], v[188:191], v[156:159]\n"
    "v_mfma_f32_16x16x32_bf16 v[156:159], v[216:219], v[192:195], v[156:159]\n"
    "v_mfma_f32_16x16x32_bf16 v[160:163], v[220:223], v[188:191], v[160:163]\n"
    "v_mfma_f32_16x16x32_bf16 v[160:163], v[224:227], v[192:195], v[160:163]\n"
    "v_mfma_f32_16x16x32_bf16 v[164:167], v[228:231], v[188:191], v[164:167]\n"
    "v_mfma_f32_16x16x32_bf16 v[164:167], v[232:235], v[192:195], v[164:167]\n"
    "v_mfma_f32_16x16x32_bf16 v[168:171], v[236:239], v[188:191], v[168:171]\n"
    "v_mfma_f32_16x16x32_bf16 v[168:171], v[240:243], v[192:195], v[168:171]\n"
    "v_mfma_f32_16x16x32_bf16 v[172:175], v[212:215], v[196:199], v[172:175]\n"
    "v_mfma_f32_16x16x32_bf16 v[172:175], v[216:219], v[200:203], v[172:175]\n"
    "v_mfma_f32_16x16x32_bf16 v[176:179], v[220:223], v[196:199], v[176:179]\n"
    "v_mfma_f32_16x16x32_bf16 v[176:179], v[224:227], v[200:203], v[176:179]\n"
    "v_mfma_f32_16x16x32_bf16 v[16:19], v[228:231], v[196:199], v[16:19]\n"
    "v_mfma_f32_16x16x32_bf16 v[16:19], v[232:235], v[200:203], v[16:19]\n"
    "v_mfma_f32_16x16x32_bf16 v[24:27], v[236:239], v[196:199], v[24:27]\n"
    "v_mfma_f32_16x16x32_bf16 v[24:27], v[240:243], v[200:203], v[24:27]\n"
    "v_mfma_f32_16x16x32_bf16 v[32:35], v[212:215], v[204:207], v[32:35]\n"
    "v_mfma_f32_16x16x32_bf16 v[32:35], v[216:219], v[208:211], v[32:35]\n"
    "v_mfma_f32_16x16x32_bf16 v[40:43], v[220:223], v[204:207], v[40:43]\n"
    "v_mfma_f32_16x16x32_bf16 v[40:43], v[224:227], v[208:211], v[40:43]\n"
    "v_mfma_f32_16x16x32_bf16 v[12:15], v[228:231], v[204:207], v[12:15]\n"
    "v_mfma_f32_16x16x32_bf16 v[12:15], v[232:235], v[208:211], v[12:15]\n"
    "v_mfma_f32_16x16x32_bf16 v[48:51], v[236:239], v[204:207], v[48:51]\n"
    "v_mfma_f32_16x16x32_bf16 v[48:51], v[240:243], v[208:211], v[48:51]\n"
    "s_setprio 0\n"
    "s_waitcnt vmcnt(9)\n"
    "s_barrier\n"
    "; phase 82: step 50 slot 0 stage 2\n"
    "ds_read_b128 v[180:183], v244 offset:32768\n"
    "ds_read_b128 v[184:187], v245 offset:32768\n"
    "ds_read_b128 v[188:191], v244 offset:34816\n"
    "s_cmp_lg_u32 s52, 4\n"
    "s_cbranch_scc1 .La118_%=\n"
    "s_xor_b32 s82, s80, 0x4000\n"
    "buffer_store_dwordx4 v[52:55], v255, s[72:75], s82 offen nt\n"
    ".La118_%=:\n"
    "ds_read_b128 v[192:195], v245 offset:34816\n"
    "ds_read_b128 v[196:199], v244 offset:40960\n"
    "ds_read_b128 v[200:203], v245 offset:40960\n"
    "s_add_u32 m0, s50, 0x4000\n"
    "s_xor_b32 s82, s48, 0x201a00\n"
    "buffer_load_dwordx4 v250, s[36:39], s82 offen  lds\n"
    "ds_read_b128 v[204:207], v244 offset:43008\n"
    "ds_read_b128 v[208:211], v245 offset:43008\n"
    "ds_read_b64_tr_b16 v[212:213], v246 offset:32768\n"
    "s_add_u32 m0, s50, 0x5000\n"
    "s_xor_b32 s82, s48, 0x281a00\n"
    "buffer_load_dwordx4 v250, s[36:39], s82 offen  lds\n"
    "ds_read_b64_tr_b16 v[214:215], v246 offset:33792\n"
    "ds_read_b64_tr_b16 v[216:217], v246 offset:40960\n"
    "ds_read_b64_tr_b16 v[218:219], v246 offset:41984\n"
    "s_add_u32 m0, s50, 0x6000\n"
    "s_xor_b32 s82, s48, 0x1a00\n"
    "buffer_load_dwordx4 v250, s[36:39], s82 offen  lds\n"
    "ds_read_b64_tr_b16 v[220:221], v247 offset:32768\n"
    "ds_read_b64_tr_b16 v[222:223], v247 offset:33792\n"
    "ds_read_b64_tr_b16 v[224:225], v247 offset:40960\n"
    "s_add_u32 m0, s50, 0x7000\n"
    "s_xor_b32 s82, s48, 0x81a00\n"
    "buffer_load_dwordx4 v250, s[36:39], s82 offen  lds\n"
    "ds_read_b64_tr_b16 v[226:227], v247 offset:41984\n"
    "ds_read_b64_tr_b16 v[228:229], v248 offset:32768\n"
    "ds_read_b64_tr_b16 v[230:231], v248 offset:33792\n"
    "s_add_u32 m0, s51, 0x4000\n"
    "s_xor_b32 s82, s66, 0x3500000\n"
    "buffer_load_dwordx4 v251, s[40:43], s82 offen  lds\n"
    "ds_read_b64_tr_b16 v[232:233], v248 offset:40960\n"
    "ds_read_b64_tr_b16 v[234:235], v248 offset:41984\n"
    "ds_read_b64_tr_b16 v[236:237], v249 offset:32768\n"
    "s_add_u32 m0, s51, 0x6000\n"
    "s_xor_b32 s82, s66, 0x3580000\n"
    "buffer_load_dwordx4 v251, s[40:43], s82 offen  lds\n"
    "ds_read_b64_tr_b16 v[238:239], v249 offset:33792\n"
    "ds_read_b64_tr_b16 v[240:241], v249 offset:40960\n"
    "ds_read_b64_tr_b16 v[242:243], v249 offset:41984\n"
    "s_waitcnt vmcnt(13) lgkmcnt(0)\n"
    "s_barrier\n"
    "s_setprio 1\n"
    "v_mfma_f32_16x16x32_bf16 v[140:143], v[212:215], v[180:183], v[140:143]\n"
    "v_mfma_f32_16x16x32_bf16 v[140:143], v[216:219], v[184:187], v[140:143]\n"
    "v_mfma_f32_16x16x32_bf16 v[144:147], v[220:223], v[180:183], v[144:147]\n"
    "v_mfma_f32_16x16x32_bf16 v[144:147], v[224:227], v[184:187], v[144:147]\n"
    "v_mfma_f32_16x16x32_bf16 v[148:151], v[228:231], v[180:183], v[148:151]\n"
    "v_mfma_f32_16x16x32_bf16 v[148:151], v[232:235], v[184:187], v[148:151]\n"
    "v_mfma_f32_16x16x32_bf16 v[152:155], v[236:239], v[180:183], v[152:155]\n"
    "v_mfma_f32_16x16x32_bf16 v[152:155], v[240:243], v[184:187], v[152:155]\n"
    "v_mfma_f32_16x16x32_bf16 v[156:159], v[212:215], v[188:191], v[156:159]\n"
    "v_mfma_f32_16x16x32_bf16 v[156:159], v[216:219], v[192:195], v[156:159]\n"
    "v_mfma_f32_16x16x32_bf16 v[160:163], v[220:223], v[188:191], v[160:163]\n"
    "v_mfma_f32_16x16x32_bf16 v[160:163], v[224:227], v[192:195], v[160:163]\n"
    "v_mfma_f32_16x16x32_bf16 v[164:167], v[228:231], v[188:191], v[164:167]\n"
    "v_mfma_f32_16x16x32_bf16 v[164:167], v[232:235], v[192:195], v[164:167]\n"
    "v_mfma_f32_16x16x32_bf16 v[168:171], v[236:239], v[188:191], v[168:171]\n"
    "v_mfma_f32_16x16x32_bf16 v[168:171], v[240:243], v[192:195], v[168:171]\n"
    "v_mfma_f32_16x16x32_bf16 v[172:175], v[212:215], v[196:199], v[172:175]\n"
    "v_mfma_f32_16x16x32_bf16 v[172:175], v[216:219], v[200:203], v[172:175]\n"
    "v_mfma_f32_16x16x32_bf16 v[176:179], v[220:223], v[196:199], v[176:179]\n"
    "v_mfma_f32_16x16x32_bf16 v[176:179], v[224:227], v[200:203], v[176:179]\n"
    "v_mfma_f32_16x16x32_bf16 v[16:19], v[228:231], v[196:199], v[16:19]\n"
    "v_mfma_f32_16x16x32_bf16 v[16:19], v[232:235], v[200:203], v[16:19]\n"
    "v_mfma_f32_16x16x32_bf16 v[24:27], v[236:239], v[196:199], v[24:27]\n"
    "v_mfma_f32_16x16x32_bf16 v[24:27], v[240:243], v[200:203], v[24:27]\n"
    "v_mfma_f32_16x16x32_bf16 v[32:35], v[212:215], v[204:207], v[32:35]\n"
    "v_mfma_f32_16x16x32_bf16 v[32:35], v[216:219], v[208:211], v[32:35]\n"
    "v_mfma_f32_16x16x32_bf16 v[40:43], v[220:223], v[204:207], v[40:43]\n"
    "v_mfma_f32_16x16x32_bf16 v[40:43], v[224:227], v[208:211], v[40:43]\n"
    "v_mfma_f32_16x16x32_bf16 v[12:15], v[228:231], v[204:207], v[12:15]\n"
    "v_mfma_f32_16x16x32_bf16 v[12:15], v[232:235], v[208:211], v[12:15]\n"
    "v_mfma_f32_16x16x32_bf16 v[48:51], v[236:239], v[204:207], v[48:51]\n"
    "v_mfma_f32_16x16x32_bf16 v[48:51], v[240:243], v[208:211], v[48:51]\n"
    "s_setprio 0\n"
    "s_waitcnt vmcnt(8)\n"
    "s_barrier\n"
    "; phase 83: step 51 slot 0 stage 2\n"
    "ds_read_b128 v[180:183], v244 offset:0\n"
    "ds_read_b128 v[184:187], v245 offset:0\n"
    "ds_read_b128 v[188:191], v244 offset:2048\n"
    "s_mov_b32 s82, 0x2200000\n"
    "buffer_store_dwordx2 v[10:11], v253, s[44:47], s82 offen sc0 sc1\n"
    "ds_read_b128 v[192:195], v245 offset:2048\n"
    "ds_read_b128 v[196:199], v244 offset:8192\n"
    "ds_read_b128 v[200:203], v245 offset:8192\n"
    "s_cmp_lg_u32 s52, 5\n"
    "s_cbranch_scc1 .La119_%=\n"
    "s_xor_b32 s82, s80, 0x4000\n"
    "buffer_store_dwordx4 v[52:55], v255, s[72:75], s82 offen nt\n"
    ".La119_%=:\n"
    "ds_read_b128 v[204:207], v244 offset:10240\n"
    "ds_read_b128 v[208:211], v245 offset:10240\n"
    "ds_read_b64_tr_b16 v[212:213], v246 offset:49152\n"
    "s_add_u32 m0, s50, 0x8000\n"
    "s_xor_b32 s82, s48, 0x201a80\n"
    "buffer_load_dwordx4 v250, s[36:39], s82 offen  lds\n"
    "ds_read_b64_tr_b16 v[214:215], v246 offset:50176\n"
    "ds_read_b64_tr_b16 v[216:217], v246 offset:57344\n"
    "ds_read_b64_tr_b16 v[218:219], v246 offset:58368\n"
    "s_add_u32 m0, s50, 0x9000\n"
    "s_xor_b32 s82, s48, 0x281a80\n"
    "buffer_load_dwordx4 v250, s[36:39], s82 offen  lds\n"
    "ds_read_b64_tr_b16 v[220:221], v247 offset:49152\n"
    "ds_read_b64_tr_b16 v[222:223], v247 offset:50176\n"
    "ds_read_b64_tr_b16 v[224:225], v247 offset:57344\n"
    "s_add_u32 m0, s50, 0xa000\n"
    "s_xor_b32 s82, s48, 0x1a80\n"
    "buffer_load_dwordx4 v250, s[36:39], s82 offen  lds\n"
    "ds_read_b64_tr_b16 v[226:227], v247 offset:58368\n"
    "ds_read_b64_tr_b16 v[228:229], v248 offset:49152\n"
    "ds_read_b64_tr_b16 v[230:231], v248 offset:50176\n"
    "s_add_u32 m0, s50, 0xb000\n"
    "s_xor_b32 s82, s48, 0x81a80\n"
    "buffer_load_dwordx4 v250, s[36:39], s82 offen  lds\n"
    "ds_read_b64_tr_b16 v[232:233], v248 offset:57344\n"
    "ds_read_b64_tr_b16 v[234:235], v248 offset:58368\n"
    "ds_read_b64_tr_b16 v[236:237], v249 offset:49152\n"
    "s_add_u32 m0, s51, 0x8000\n"
    "s_xor_b32 s82, s66, 0x3600000\n"
    "buffer_load_dwordx4 v251, s[40:43], s82 offen  lds\n"
    "ds_read_b64_tr_b16 v[238:239], v249 offset:50176\n"
    "ds_read_b64_tr_b16 v[240:241], v249 offset:57344\n"
    "ds_read_b64_tr_b16 v[242:243], v249 offset:58368\n"
    "s_add_u32 m0, s51, 0xa000\n"
    "s_xor_b32 s82, s66, 0x3680000\n"
    "buffer_load_dwordx4 v251, s[40:43], s82 offen  lds\n"
    "s_waitcnt vmcnt(13) lgkmcnt(0)\n"
    "s_barrier\n"
    "s_setprio 1\n"
    "v_mfma_f32_16x16x32_bf16 v[140:143], v[212:215], v[180:183], v[140:143]\n"
    "v_mfma_f32_16x16x32_bf16 v[140:143], v[216:219], v[184:187], v[140:143]\n"
    "v_mfma_f32_16x16x32_bf16 v[144:147], v[220:223], v[180:183], v[144:147]\n"
    "v_mfma_f32_16x16x32_bf16 v[144:147], v[224:227], v[184:187], v[144:147]\n"
    "v_mfma_f32_16x16x32_bf16 v[148:151], v[228:231], v[180:183], v[148:151]\n"
    "v_mfma_f32_16x16x32_bf16 v[148:151], v[232:235], v[184:187], v[148:151]\n"
    "v_mfma_f32_16x16x32_bf16 v[152:155], v[236:239], v[180:183], v[152:155]\n"
    "v_mfma_f32_16x16x32_bf16 v[152:155], v[240:243], v[184:187], v[152:155]\n"
    "v_mfma_f32_16x16x32_bf16 v[156:159], v[212:215], v[188:191], v[156:159]\n"
    "v_mfma_f32_16x16x32_bf16 v[156:159], v[216:219], v[192:195], v[156:159]\n"
    "v_mfma_f32_16x16x32_bf16 v[160:163], v[220:223], v[188:191], v[160:163]\n"
    "v_mfma_f32_16x16x32_bf16 v[160:163], v[224:227], v[192:195], v[160:163]\n"
    "v_mfma_f32_16x16x32_bf16 v[164:167], v[228:231], v[188:191], v[164:167]\n"
    "v_mfma_f32_16x16x32_bf16 v[164:167], v[232:235], v[192:195], v[164:167]\n"
    "v_mfma_f32_16x16x32_bf16 v[168:171], v[236:239], v[188:191], v[168:171]\n"
    "v_mfma_f32_16x16x32_bf16 v[168:171], v[240:243], v[192:195], v[168:171]\n"
    "v_mfma_f32_16x16x32_bf16 v[172:175], v[212:215], v[196:199], v[172:175]\n"
    "v_mfma_f32_16x16x32_bf16 v[172:175], v[216:219], v[200:203], v[172:175]\n"
    "v_mfma_f32_16x16x32_bf16 v[176:179], v[220:223], v[196:199], v[176:179]\n"
    "v_mfma_f32_16x16x32_bf16 v[176:179], v[224:227], v[200:203], v[176:179]\n"
    "v_mfma_f32_16x16x32_bf16 v[16:19], v[228:231], v[196:199], v[16:19]\n"
    "v_mfma_f32_16x16x32_bf16 v[16:19], v[232:235], v[200:203], v[16:19]\n"
    "v_mfma_f32_16x16x32_bf16 v[24:27], v[236:239], v[196:199], v[24:27]\n"
    "v_mfma_f32_16x16x32_bf16 v[24:27], v[240:243], v[200:203], v[24:27]\n"
    "v_mfma_f32_16x16x32_bf16 v[32:35], v[212:215], v[204:207], v[32:35]\n"
    "v_mfma_f32_16x16x32_bf16 v[32:35], v[216:219], v[208:211], v[32:35]\n"
    "v_mfma_f32_16x16x32_bf16 v[40:43], v[220:223], v[204:207], v[40:43]\n"
    "v_mfma_f32_16x16x32_bf16 v[40:43], v[224:227], v[208:211], v[40:43]\n"
    "v_mfma_f32_16x16x32_bf16 v[12:15], v[228:231], v[204:207], v[12:15]\n"
    "v_mfma_f32_16x16x32_bf16 v[12:15], v[232:235], v[208:211], v[12:15]\n"
    "v_mfma_f32_16x16x32_bf16 v[48:51], v[236:239], v[204:207], v[48:51]\n"
    "v_mfma_f32_16x16x32_bf16 v[48:51], v[240:243], v[208:211], v[48:51]\n"
    "s_setprio 0\n"
    "s_waitcnt vmcnt(9)\n"
    "s_barrier\n"
    "; phase 84: step 52 slot 0 stage 2\n"
    "ds_read_b128 v[180:183], v244 offset:16384\n"
    "ds_read_b128 v[184:187], v245 offset:16384\n"
    "ds_read_b128 v[188:191], v244 offset:18432\n"
    "s_add_u32 s82, s56, 0x100000\n"
    "buffer_load_dword v6, v254, s[44:47], s82 offen sc0 sc1\n"
    "s_lshr_b32 s83, s65, 8\n"
    "s_and_b32 s82, s83, 0x38\n"
    "s_load_dwordx2 s[76:77], s[60:61], s82\n"
    "s_load_dwordx2 s[18:19], s[60:61], s82 offset:0x40\n"
    "s_and_b32 s83, s65, 0x700\n"
    "s_add_u32 s81, s57, s83\n"
    "ds_read_b128 v[192:195], v245 offset:18432\n"
    "ds_read_b128 v[196:199], v244 offset:24576\n"
    "ds_read_b128 v[200:203], v245 offset:24576\n"
    "s_cmp_lg_u32 s52, 6\n"
    "s_cbranch_scc1 .La120_%=\n"
    "s_xor_b32 s82, s80, 0x4000\n"
    "buffer_store_dwordx4 v[52:55], v255, s[72:75], s82 offen nt\n"
    ".La120_%=:\n"
    "ds_read_b128 v[204:207], v244 offset:26624\n"
    "ds_read_b128 v[208:211], v245 offset:26624\n"
    "ds_read_b64_tr_b16 v[212:213], v246 offset:0\n"
    "s_add_u32 m0, s50, 0x0\n"
    "s_xor_b32 s82, s48, 0x201b00\n"
    "buffer_load_dwordx4 v250, s[36:39], s82 offen  lds\n"
    "ds_read_b64_tr_b16 v[214:215], v246 offset:1024\n"
    "ds_read_b64_tr_b16 v[216:217], v246 offset:8192\n"
    "ds_read_b64_tr_b16 v[218:219], v246 offset:9216\n"
    "s_add_u32 m0, s50, 0x1000\n"
    "s_xor_b32 s82, s48, 0x281b00\n"
    "buffer_load_dwordx4 v250, s[36:39], s82 offen  lds\n"
    "ds_read_b64_tr_b16 v[220:221], v247 offset:0\n"
    "ds_read_b64_tr_b16 v[222:223], v247 offset:1024\n"
    "ds_read_b64_tr_b16 v[224:225], v247 offset:8192\n"
    "s_add_u32 m0, s50, 0x2000\n"
    "s_xor_b32 s82, s48, 0x1b00\n"
    "buffer_load_dwordx4 v250, s[36:39], s82 offen  lds\n"
    "ds_read_b64_tr_b16 v[226:227], v247 offset:9216\n"
    "ds_read_b64_tr_b16 v[228:229], v248 offset:0\n"
    "ds_read_b64_tr_b16 v[230:231], v248 offset:1024\n"
    "s_add_u32 m0, s50, 0x3000\n"
    "s_xor_b32 s82, s48, 0x81b00\n"
    "buffer_load_dwordx4 v250, s[36:39], s82 offen  lds\n"
    "ds_read_b64_tr_b16 v[232:233], v248 offset:8192\n"
    "ds_read_b64_tr_b16 v[234:235], v248 offset:9216\n"
    "ds_read_b64_tr_b16 v[236:237], v249 offset:0\n"
    "s_add_u32 m0, s51, 0xc000\n"
    "s_xor_b32 s82, s66, 0x3700000\n"
    "buffer_load_dwordx4 v251, s[40:43], s82 offen  lds\n"
    "ds_read_b64_tr_b16 v[238:239], v249 offset:1024\n"
    "ds_read_b64_tr_b16 v[240:241], v249 offset:8192\n"
    "ds_read_b64_tr_b16 v[242:243], v249 offset:9216\n"
    "s_add_u32 m0, s51, 0xe000\n"
    "s_xor_b32 s82, s66, 0x3780000\n"
    "buffer_load_dwordx4 v251, s[40:43], s82 offen  lds\n"
    "s_waitcnt vmcnt(14) lgkmcnt(0)\n"
    "s_barrier\n"
    "s_setprio 1\n"
    "v_mfma_f32_16x16x32_bf16 v[140:143], v[212:215], v[180:183], v[140:143]\n"
    "v_mfma_f32_16x16x32_bf16 v[140:143], v[216:219], v[184:187], v[140:143]\n"
    "v_mfma_f32_16x16x32_bf16 v[144:147], v[220:223], v[180:183], v[144:147]\n"
    "v_mfma_f32_16x16x32_bf16 v[144:147], v[224:227], v[184:187], v[144:147]\n"
    "v_mfma_f32_16x16x32_bf16 v[148:151], v[228:231], v[180:183], v[148:151]\n"
    "v_mfma_f32_16x16x32_bf16 v[148:151], v[232:235], v[184:187], v[148:151]\n"
    "v_mfma_f32_16x16x32_bf16 v[152:155], v[236:239], v[180:183], v[152:155]\n"
    "v_mfma_f32_16x16x32_bf16 v[152:155], v[240:243], v[184:187], v[152:155]\n"
    "v_mfma_f32_16x16x32_bf16 v[156:159], v[212:215], v[188:191], v[156:159]\n"
    "v_mfma_f32_16x16x32_bf16 v[156:159], v[216:219], v[192:195], v[156:159]\n"
    "v_mfma_f32_16x16x32_bf16 v[160:163], v[220:223], v[188:191], v[160:163]\n"
    "v_mfma_f32_16x16x32_bf16 v[160:163], v[224:227], v[192:195], v[160:163]\n"
    "v_mfma_f32_16x16x32_bf16 v[164:167], v[228:231], v[188:191], v[164:167]\n"
    "v_mfma_f32_16x16x32_bf16 v[164:167], v[232:235], v[192:195], v[164:167]\n"
    "v_mfma_f32_16x16x32_bf16 v[168:171], v[236:239], v[188:191], v[168:171]\n"
    "v_mfma_f32_16x16x32_bf16 v[168:171], v[240:243], v[192:195], v[168:171]\n"
    "v_mfma_f32_16x16x32_bf16 v[172:175], v[212:215], v[196:199], v[172:175]\n"
    "v_mfma_f32_16x16x32_bf16 v[172:175], v[216:219], v[200:203], v[172:175]\n"
    "v_mfma_f32_16x16x32_bf16 v[176:179], v[220:223], v[196:199], v[176:179]\n"
    "v_mfma_f32_16x16x32_bf16 v[176:179], v[224:227], v[200:203], v[176:179]\n"
    "v_mfma_f32_16x16x32_bf16 v[16:19], v[228:231], v[196:199], v[16:19]\n"
    "v_mfma_f32_16x16x32_bf16 v[16:19], v[232:235], v[200:203], v[16:19]\n"
    "v_mfma_f32_16x16x32_bf16 v[24:27], v[236:239], v[196:199], v[24:27]\n"
    "v_mfma_f32_16x16x32_bf16 v[24:27], v[240:243], v[200:203], v[24:27]\n"
    "v_mfma_f32_16x16x32_bf16 v[32:35], v[212:215], v[204:207], v[32:35]\n"
    "v_mfma_f32_16x16x32_bf16 v[32:35], v[216:219], v[208:211], v[32:35]\n"
    "v_mfma_f32_16x16x32_bf16 v[40:43], v[220:223], v[204:207], v[40:43]\n"
    "v_mfma_f32_16x16x32_bf16 v[40:43], v[224:227], v[208:211], v[40:43]\n"
    "v_mfma_f32_16x16x32_bf16 v[12:15], v[228:231], v[204:207], v[12:15]\n"
    "v_mfma_f32_16x16x32_bf16 v[12:15], v[232:235], v[208:211], v[12:15]\n"
    "v_mfma_f32_16x16x32_bf16 v[48:51], v[236:239], v[204:207], v[48:51]\n"
    "v_mfma_f32_16x16x32_bf16 v[48:51], v[240:243], v[208:211], v[48:51]\n"
    "s_setprio 0\n"
    "s_waitcnt vmcnt(6)\n"
    "s_barrier\n"
    "; phase 85: step 53 slot 0 stage 2\n"
    "v_cmp_ne_u32 vcc, s90, v6\n"
    "s_cbranch_vccz .La121_%=\n"
    "s_memrealtime s[86:87]\n"
    "s_add_u32 s82, s56, 0x100000\n"
    ".La122_%=:\n"
    "s_sleep 2\n"
    "buffer_load_dword v6, v254, s[44:47], s82 offen sc0 sc1\n"
    "s_memrealtime s[88:89]\n"
    "s_waitcnt vmcnt(0) lgkmcnt(0)\n"
    "v_cmp_ne_u32 vcc, s90, v6\n"
    "s_cbranch_vccz .La121_%=\n"
    "s_sub_u32 s83, s88, s86\n"
    "s_cmp_lt_u32 s83, 0x2710\n"
    "s_cbranch_scc1 .La122_%=\n"
    "s_sleep 64\n"
    "buffer_load_dword v7, off, s[44:47], 0 sc0 sc1\n"
    "s_waitcnt vmcnt(0)\n"
    "v_cmp_eq_u32 vcc, s90, v7\n"
    "s_cbranch_vccnz .La123_%=\n"
    "s_cmp_lt_u32 s83, 0x2faf080\n"
    "s_cbranch_scc1 .La122_%=\n"
    ".La123_%=:\n"
    "v_mov_b32 v7, s90\n"
    "v_mov_b32 v124, 0x7fc00000\n"
    "v_mov_b32 v125, 0x7fc00000\n"
    "v_mov_b32 v126, 0x7fc00000\n"
    "v_mov_b32 v127, 0x7fc00000\n"
    "v_mov_b32 v128, 0x7fc00000\n"
    "v_mov_b32 v129, 0x7fc00000\n"
    "v_mov_b32 v130, 0x7fc00000\n"
    "v_mov_b32 v131, 0x7fc00000\n"
    "v_mov_b32 v132, 0x7fc00000\n"
    "v_mov_b32 v133, 0x7fc00000\n"
    "v_mov_b32 v134, 0x7fc00000\n"
    "v_mov_b32 v135, 0x7fc00000\n"
    "v_mov_b32 v136, 0x7fc00000\n"
    "v_mov_b32 v137, 0x7fc00000\n"
    "v_mov_b32 v138, 0x7fc00000\n"
    "v_mov_b32 v139, 0x7fc00000\n"
    "buffer_store_dword v7, off, s[44:47], 0 sc0 sc1\n"
    "s_waitcnt vmcnt(0)\n"
    ".La121_%=:\n"
    "ds_read_b128 v[180:183], v244 offset:32768\n"
    "ds_read_b128 v[184:187], v245 offset:32768\n"
    "s_add_u32 s82, s54, 0x1104000\n"
    "buffer_load_dwordx4 v[56:59], v252, s[44:47], s82 offen sc0 sc1\n"
    "ds_read_b128 v[188:191], v244 offset:34816\n"
    "ds_read_b128 v[192:195], v245 offset:34816\n"
    "s_add_u32 s82, s54, 0x1108000\n"
    "buffer_load_dwordx4 v[20:23], v252, s[44:47], s82 offen sc0 sc1\n"
    "ds_read_b128 v[196:199], v244 offset:40960\n"
    "ds_read_b128 v[200:203], v245 offset:40960\n"
    "s_cmp_lg_u32 s52, 7\n"
    "s_cbranch_scc1 .La124_%=\n"
    "s_xor_b32 s82, s80, 0x4000\n"
    "buffer_store_dwordx4 v[52:55], v255, s[72:75], s82 offen nt\n"
    ".La124_%=:\n"
    "ds_read_b128 v[204:207], v244 offset:43008\n"
    "ds_read_b128 v[208:211], v245 offset:43008\n"
    "s_add_u32 m0, s50, 0x4000\n"
    "s_xor_b32 s82, s48, 0x201b80\n"
    "buffer_load_dwordx4 v250, s[36:39], s82 offen  lds\n"
    "ds_read_b64_tr_b16 v[212:213], v246 offset:16384\n"
    "ds_read_b64_tr_b16 v[214:215], v246 offset:17408\n"
    "s_add_u32 m0, s50, 0x5000\n"
    "s_xor_b32 s82, s48, 0x281b80\n"
    "buffer_load_dwordx4 v250, s[36:39], s82 offen  lds\n"
    "ds_read_b64_tr_b16 v[216:217], v246 offset:24576\n"
    "ds_read_b64_tr_b16 v[218:219], v246 offset:25600\n"
    "s_add_u32 m0, s50, 0x6000\n"
    "s_xor_b32 s82, s48, 0x1b80\n"
    "buffer_load_dwordx4 v250, s[36:39], s82 offen  lds\n"
    "ds_read_b64_tr_b16 v[220:221], v247 offset:16384\n"
    "ds_read_b64_tr_b16 v[222:223], v247 offset:17408\n"
    "s_add_u32 m0, s50, 0x7000\n"
    "s_xor_b32 s82, s48, 0x81b80\n"
    "buffer_load_dwordx4 v250, s[36:39], s82 offen  lds\n"
    "ds_read_b64_tr_b16 v[224:225], v247 offset:24576\n"
    "ds_read_b64_tr_b16 v[226:227], v247 offset:25600\n"
    "s_add_u32 m0, s51, 0x0\n"
    "s_xor_b32 s82, s66, 0x3800000\n"
    "buffer_load_dwordx4 v251, s[40:43], s82 offen  lds\n"
    "ds_read_b64_tr_b16 v[228:229], v248 offset:16384\n"
    "ds_read_b64_tr_b16 v[230:231], v248 offset:17408\n"
    "s_add_u32 m0, s51, 0x2000\n"
    "s_xor_b32 s82, s66, 0x3880000\n"
    "buffer_load_dwordx4 v251, s[40:43], s82 offen  lds\n"
    "ds_read_b64_tr_b16 v[232:233], v248 offset:24576\n"
    "ds_read_b64_tr_b16 v[234:235], v248 offset:25600\n"
    "ds_read_b64_tr_b16 v[236:237], v249 offset:16384\n"
    "ds_read_b64_tr_b16 v[238:239], v249 offset:17408\n"
    "ds_read_b64_tr_b16 v[240:241], v249 offset:24576\n"
    "ds_read_b64_tr_b16 v[242:243], v249 offset:25600\n"
    "s_waitcnt lgkmcnt(0)\n"
    "s_barrier\n"
    "s_setprio 1\n"
    "v_mfma_f32_16x16x32_bf16 v[140:143], v[212:215], v[180:183], v[140:143]\n"
    "v_mfma_f32_16x16x32_bf16 v[140:143], v[216:219], v[184:187], v[140:143]\n"
    "v_mfma_f32_16x16x32_bf16 v[144:147], v[220:223], v[180:183], v[144:147]\n"
    "v_mfma_f32_16x16x32_bf16 v[144:147], v[224:227], v[184:187], v[144:147]\n"
    "v_mfma_f32_16x16x32_bf16 v[148:151], v[228:231], v[180:183], v[148:151]\n"
    "v_mfma_f32_16x16x32_bf16 v[148:151], v[232:235], v[184:187], v[148:151]\n"
    "v_mfma_f32_16x16x32_bf16 v[152:155], v[236:239], v[180:183], v[152:155]\n"
    "v_mfma_f32_16x16x32_bf16 v[152:155], v[240:243], v[184:187], v[152:155]\n"
    "v_mfma_f32_16x16x32_bf16 v[156:159], v[212:215], v[188:191], v[156:159]\n"
    "v_mfma_f32_16x16x32_bf16 v[156:159], v[216:219], v[192:195], v[156:159]\n"
    "v_mfma_f32_16x16x32_bf16 v[160:163], v[220:223], v[188:191], v[160:163]\n"
    "v_mfma_f32_16x16x32_bf16 v[160:163], v[224:227], v[192:195], v[160:163]\n"
    "v_mfma_f32_16x16x32_bf16 v[164:167], v[228:231], v[188:191], v[164:167]\n"
    "v_mfma_f32_16x16x32_bf16 v[164:167], v[232:235], v[192:195], v[164:167]\n"
    "v_mfma_f32_16x16x32_bf16 v[168:171], v[236:239], v[188:191], v[168:171]\n"
    "v_mfma_f32_16x16x32_bf16 v[168:171], v[240:243], v[192:195], v[168:171]\n"
    "v_mfma_f32_16x16x32_bf16 v[172:175], v[212:215], v[196:199], v[172:175]\n"
    "v_mfma_f32_16x16x32_bf16 v[172:175], v[216:219], v[200:203], v[172:175]\n"
    "v_mfma_f32_16x16x32_bf16 v[176:179], v[220:223], v[196:199], v[176:179]\n"
    "v_mfma_f32_16x16x32_bf16 v[176:179], v[224:227], v[200:203], v[176:179]\n"
    "v_mfma_f32_16x16x32_bf16 v[16:19], v[228:231], v[196:199], v[16:19]\n"
    "v_mfma_f32_16x16x32_bf16 v[16:19], v[232:235], v[200:203], v[16:19]\n"
    "v_mfma_f32_16x16x32_bf16 v[24:27], v[236:239], v[196:199], v[24:27]\n"
    "v_mfma_f32_16x16x32_bf16 v[24:27], v[240:243], v[200:203], v[24:27]\n"
    "v_mfma_f32_16x16x32_bf16 v[32:35], v[212:215], v[204:207], v[32:35]\n"
    "v_mfma_f32_16x16x32_bf16 v[32:35], v[216:219], v[208:211], v[32:35]\n"
    "v_mfma_f32_16x16x32_bf16 v[40:43], v[220:223], v[204:207], v[40:43]\n"
    "v_mfma_f32_16x16x32_bf16 v[40:43], v[224:227], v[208:211], v[40:43]\n"
    "v_mfma_f32_16x16x32_bf16 v[12:15], v[228:231], v[204:207], v[12:15]\n"
    "v_mfma_f32_16x16x32_bf16 v[12:15], v[232:235], v[208:211], v[12:15]\n"
    "v_mfma_f32_16x16x32_bf16 v[48:51], v[236:239], v[204:207], v[48:51]\n"
    "v_mfma_f32_16x16x32_bf16 v[48:51], v[240:243], v[208:211], v[48:51]\n"
    "s_setprio 0\n"
    "s_waitcnt vmcnt(10)\n"
    "s_barrier\n"
    "; phase 86: step 54 slot 0 stage 2\n"
    "ds_read_b128 v[180:183], v244 offset:0\n"
    "ds_read_b128 v[184:187], v245 offset:0\n"
    "ds_read_b128 v[188:191], v244 offset:2048\n"
    "s_add_u32 s82, s54, 0x110c000\n"
    "buffer_load_dwordx4 v[36:39], v252, s[44:47], s82 offen sc0 sc1\n"
    "ds_read_b128 v[192:195], v245 offset:2048\n"
    "ds_read_b128 v[196:199], v244 offset:8192\n"
    "ds_read_b128 v[200:203], v245 offset:8192\n"
    "s_add_u32 s82, s54, 0x1106000\n"
    "buffer_load_dwordx4 v[68:71], v252, s[44:47], s82 offen sc0 sc1\n"
    "ds_read_b128 v[204:207], v244 offset:10240\n"
    "ds_read_b128 v[208:211], v245 offset:10240\n"
    "ds_read_b64_tr_b16 v[212:213], v246 offset:32768\n"
    "s_add_u32 m0, s50, 0x8000\n"
    "s_xor_b32 s82, s48, 0x201c00\n"
    "buffer_load_dwordx4 v250, s[36:39], s82 offen  lds\n"
    "ds_read_b64_tr_b16 v[214:215], v246 offset:33792\n"
    "ds_read_b64_tr_b16 v[216:217], v246 offset:40960\n"
    "ds_read_b64_tr_b16 v[218:219], v246 offset:41984\n"
    "s_add_u32 m0, s50, 0x9000\n"
    "s_xor_b32 s82, s48, 0x281c00\n"
    "buffer_load_dwordx4 v250, s[36:39], s82 offen  lds\n"
    "ds_read_b64_tr_b16 v[220:221], v247 offset:32768\n"
    "ds_read_b64_tr_b16 v[222:223], v247 offset:33792\n"
    "ds_read_b64_tr_b16 v[224:225], v247 offset:40960\n"
    "s_add_u32 m0, s50, 0xa000\n"
    "s_xor_b32 s82, s48, 0x1c00\n"
    "buffer_load_dwordx4 v250, s[36:39], s82 offen  lds\n"
    "ds_read_b64_tr_b16 v[226:227], v247 offset:41984\n"
    "ds_read_b64_tr_b16 v[228:229], v248 offset:32768\n"
    "ds_read_b64_tr_b16 v[230:231], v248 offset:33792\n"
    "s_add_u32 m0, s50, 0xb000\n"
    "s_xor_b32 s82, s48, 0x81c00\n"
    "buffer_load_dwordx4 v250, s[36:39], s82 offen  lds\n"
    "ds_read_b64_tr_b16 v[232:233], v248 offset:40960\n"
    "ds_read_b64_tr_b16 v[234:235], v248 offset:41984\n"
    "ds_read_b64_tr_b16 v[236:237], v249 offset:32768\n"
    "s_add_u32 m0, s51, 0x4000\n"
    "s_xor_b32 s82, s67, 0x3800000\n"
    "buffer_load_dwordx4 v251, s[40:43], s82 offen  lds\n"
    "ds_read_b64_tr_b16 v[238:239], v249 offset:33792\n"
    "ds_read_b64_tr_b16 v[240:241], v249 offset:40960\n"
    "ds_read_b64_tr_b16 v[242:243], v249 offset:41984\n"
    "s_add_u32 m0, s51, 0x6000\n"
    "s_xor_b32 s82, s67, 0x3880000\n"
    "buffer_load_dwordx4 v251, s[40:43], s82 offen  lds\n"
    "s_waitcnt vmcnt(14) lgkmcnt(0)\n"
    "s_barrier\n"
    "s_setprio 1\n"
    "v_mfma_f32_16x16x32_bf16 v[140:143], v[212:215], v[180:183], v[140:143]\n"
    "v_mfma_f32_16x16x32_bf16 v[132:135], v[2:5], v[56:59], v[132:135]\n"
    "v_mfma_f32_16x16x32_bf16 v[140:143], v[216:219], v[184:187], v[140:143]\n"
    "v_mfma_f32_16x16x32_bf16 v[136:139], v[0:3], v[56:59], v[136:139]\n"
    "v_mfma_f32_16x16x32_bf16 v[144:147], v[220:223], v[180:183], v[144:147]\n"
    "v_mfma_f32_16x16x32_bf16 v[132:135], v[2:5], v[20:23], v[132:135]\n"
    "v_mfma_f32_16x16x32_bf16 v[144:147], v[224:227], v[184:187], v[144:147]\n"
    "v_mfma_f32_16x16x32_bf16 v[136:139], v[0:3], v[20:23], v[136:139]\n"
    "v_mfma_f32_16x16x32_bf16 v[148:151], v[228:231], v[180:183], v[148:151]\n"
    "v_mfma_f32_16x16x32_bf16 v[148:151], v[232:235], v[184:187], v[148:151]\n"
    "v_mfma_f32_16x16x32_bf16 v[152:155], v[236:239], v[180:183], v[152:155]\n"
    "v_mfma_f32_16x16x32_bf16 v[152:155], v[240:243], v[184:187], v[152:155]\n"
    "v_mfma_f32_16x16x32_bf16 v[156:159], v[212:215], v[188:191], v[156:159]\n"
    "v_mfma_f32_16x16x32_bf16 v[156:159], v[216:219], v[192:195], v[156:159]\n"
    "v_mfma_f32_16x16x32_bf16 v[160:163], v[220:223], v[188:191], v[160:163]\n"
    "v_mfma_f32_16x16x32_bf16 v[160:163], v[224:227], v[192:195], v[160:163]\n"
    "v_mfma_f32_16x16x32_bf16 v[164:167], v[228:231], v[188:191], v[164:167]\n"
    "v_mfma_f32_16x16x32_bf16 v[164:167], v[232:235], v[192:195], v[164:167]\n"
    "v_mfma_f32_16x16x32_bf16 v[168:171], v[236:239], v[188:191], v[168:171]\n"
    "v_mfma_f32_16x16x32_bf16 v[168:171], v[240:243], v[192:195], v[168:171]\n"
    "v_mfma_f32_16x16x32_bf16 v[172:175], v[212:215], v[196:199], v[172:175]\n"
    "v_mfma_f32_16x16x32_bf16 v[172:175], v[216:219], v[200:203], v[172:175]\n"
    "v_mfma_f32_16x16x32_bf16 v[176:179], v[220:223], v[196:199], v[176:179]\n"
    "v_mfma_f32_16x16x32_bf16 v[176:179], v[224:227], v[200:203], v[176:179]\n"
    "v_mfma_f32_16x16x32_bf16 v[16:19], v[228:231], v[196:199], v[16:19]\n"
    "v_mfma_f32_16x16x32_bf16 v[16:19], v[232:235], v[200:203], v[16:19]\n"
    "v_mfma_f32_16x16x32_bf16 v[24:27], v[236:239], v[196:199], v[24:27]\n"
    "v_mfma_f32_16x16x32_bf16 v[24:27], v[240:243], v[200:203], v[24:27]\n"
    "v_mfma_f32_16x16x32_bf16 v[32:35], v[212:215], v[204:207], v[32:35]\n"
    "v_mfma_f32_16x16x32_bf16 v[32:35], v[216:219], v[208:211], v[32:35]\n"
    "v_mfma_f32_16x16x32_bf16 v[40:43], v[220:223], v[204:207], v[40:43]\n"
    "v_mfma_f32_16x16x32_bf16 v[40:43], v[224:227], v[208:211], v[40:43]\n"
    "v_mfma_f32_16x16x32_bf16 v[12:15], v[228:231], v[204:207], v[12:15]\n"
    "v_mfma_f32_16x16x32_bf16 v[12:15], v[232:235], v[208:211], v[12:15]\n"
    "v_mfma_f32_16x16x32_bf16 v[48:51], v[236:239], v[204:207], v[48:51]\n"
    "v_mfma_f32_16x16x32_bf16 v[48:51], v[240:243], v[208:211], v[48:51]\n"
    "s_setprio 0\n"
    "s_waitcnt vmcnt(10)\n"
    "s_barrier\n"
    "; phase 87: step 55 slot 0 stage 2\n"
    "ds_read_b128 v[180:183], v244 offset:16384\n"
    "ds_read_b128 v[184:187], v245 offset:16384\n"
    "ds_read_b128 v[188:191], v244 offset:18432\n"
    "s_add_u32 s82, s54, 0x110a000\n"
    "buffer_load_dwordx4 v[72:75], v252, s[44:47], s82 offen sc0 sc1\n"
    "ds_read_b128 v[192:195], v245 offset:18432\n"
    "ds_read_b128 v[196:199], v244 offset:24576\n"
    "ds_read_b128 v[200:203], v245 offset:24576\n"
    "s_add_u32 s82, s54, 0x110e000\n"
    "buffer_load_dwordx4 v[44:47], v252, s[44:47], s82 offen sc0 sc1\n"
    "ds_read_b128 v[204:207], v244 offset:26624\n"
    "ds_read_b128 v[208:211], v245 offset:26624\n"
    "ds_read_b64_tr_b16 v[212:213], v246 offset:49152\n"
    "s_add_u32 m0, s51, 0x8000\n"
    "s_xor_b32 s82, s66, 0x3900000\n"
    "buffer_load_dwordx4 v251, s[40:43], s82 offen  lds\n"
    "ds_read_b64_tr_b16 v[214:215], v246 offset:50176\n"
    "ds_read_b64_tr_b16 v[216:217], v246 offset:57344\n"
    "ds_read_b64_tr_b16 v[218:219], v246 offset:58368\n"
    "s_add_u32 m0, s51, 0xa000\n"
    "s_xor_b32 s82, s66, 0x3980000\n"
    "buffer_load_dwordx4 v251, s[40:43], s82 offen  lds\n"
    "ds_read_b64_tr_b16 v[220:221], v247 offset:49152\n"
    "ds_read_b64_tr_b16 v[222:223], v247 offset:50176\n"
    "ds_read_b64_tr_b16 v[224:225], v247 offset:57344\n"
    "s_add_u32 m0, s50, 0x0\n"
    "s_xor_b32 s82, s48, 0x201c80\n"
    "buffer_load_dwordx4 v250, s[36:39], s82 offen  lds\n"
    "ds_read_b64_tr_b16 v[226:227], v247 offset:58368\n"
    "ds_read_b64_tr_b16 v[228:229], v248 offset:49152\n"
    "ds_read_b64_tr_b16 v[230:231], v248 offset:50176\n"
    "s_add_u32 m0, s50, 0x1000\n"
    "s_xor_b32 s82, s48, 0x281c80\n"
    "buffer_load_dwordx4 v250, s[36:39], s82 offen  lds\n"
    "ds_read_b64_tr_b16 v[232:233], v248 offset:57344\n"
    "ds_read_b64_tr_b16 v[234:235], v248 offset:58368\n"
    "ds_read_b64_tr_b16 v[236:237], v249 offset:49152\n"
    "s_add_u32 m0, s50, 0x2000\n"
    "s_xor_b32 s82, s48, 0x1c80\n"
    "buffer_load_dwordx4 v250, s[36:39], s82 offen  lds\n"
    "ds_read_b64_tr_b16 v[238:239], v249 offset:50176\n"
    "ds_read_b64_tr_b16 v[240:241], v249 offset:57344\n"
    "ds_read_b64_tr_b16 v[242:243], v249 offset:58368\n"
    "s_add_u32 m0, s50, 0x3000\n"
    "s_xor_b32 s82, s48, 0x81c80\n"
    "buffer_load_dwordx4 v250, s[36:39], s82 offen  lds\n"
    "s_waitcnt vmcnt(14) lgkmcnt(0)\n"
    "s_barrier\n"
    "s_setprio 1\n"
    "v_mfma_f32_16x16x32_bf16 v[140:143], v[212:215], v[180:183], v[140:143]\n"
    "v_mfma_f32_16x16x32_bf16 v[132:135], v[2:5], v[36:39], v[132:135]\n"
    "v_mfma_f32_16x16x32_bf16 v[140:143], v[216:219], v[184:187], v[140:143]\n"
    "v_mfma_f32_16x16x32_bf16 v[136:139], v[0:3], v[36:39], v[136:139]\n"
    "v_mfma_f32_16x16x32_bf16 v[144:147], v[220:223], v[180:183], v[144:147]\n"
    "v_mfma_f32_16x16x32_bf16 v[124:127], v[2:5], v[68:71], v[124:127]\n"
    "v_mfma_f32_16x16x32_bf16 v[144:147], v[224:227], v[184:187], v[144:147]\n"
    "v_mfma_f32_16x16x32_bf16 v[128:131], v[0:3], v[68:71], v[128:131]\n"
    "v_mfma_f32_16x16x32_bf16 v[148:151], v[228:231], v[180:183], v[148:151]\n"
    "v_mfma_f32_16x16x32_bf16 v[148:151], v[232:235], v[184:187], v[148:151]\n"
    "v_mfma_f32_16x16x32_bf16 v[152:155], v[236:239], v[180:183], v[152:155]\n"
    "v_mfma_f32_16x16x32_bf16 v[152:155], v[240:243], v[184:187], v[152:155]\n"
    "v_med3_f32 v36, v132, -s62, s62\n"
    "v_med3_f32 v37, v133, -s62, s62\n"
    "v_mfma_f32_16x16x32_bf16 v[156:159], v[212:215], v[188:191], v[156:159]\n"
    "v_med3_f32 v38, v134, -s62, s62\n"
    "v_med3_f32 v39, v135, -s62, s62\n"
    "v_mfma_f32_16x16x32_bf16 v[156:159], v[216:219], v[192:195], v[156:159]\n"
    "v_mul_f32 v36, s63, v36\n"
    "v_mul_f32 v37, s63, v37\n"
    "v_mfma_f32_16x16x32_bf16 v[160:163], v[220:223], v[188:191], v[160:163]\n"
    "v_mul_f32 v38, s63, v38\n"
    "v_mul_f32 v39, s63, v39\n"
    "v_mfma_f32_16x16x32_bf16 v[160:163], v[224:227], v[192:195], v[160:163]\n"
    "v_exp_f32 v36, v36\n"
    "v_mfma_f32_16x16x32_bf16 v[164:167], v[228:231], v[188:191], v[164:167]\n"
    "v_exp_f32 v37, v37\n"
    "v_mfma_f32_16x16x32_bf16 v[164:167], v[232:235], v[192:195], v[164:167]\n"
    "v_exp_f32 v38, v38\n"
    "v_mfma_f32_16x16x32_bf16 v[168:171], v[236:239], v[188:191], v[168:171]\n"
    "v_exp_f32 v39, v39\n"
    "v_mfma_f32_16x16x32_bf16 v[168:171], v[240:243], v[192:195], v[168:171]\n"
    "v_add_f32 v36, 1.0, v36\n"
    "v_add_f32 v37, 1.0, v37\n"
    "v_mfma_f32_16x16x32_bf16 v[172:175], v[212:215], v[196:199], v[172:175]\n"
    "v_add_f32 v38, 1.0, v38\n"
    "v_add_f32 v39, 1.0, v39\n"
    "v_mfma_f32_16x16x32_bf16 v[172:175], v[216:219], v[200:203], v[172:175]\n"
    "v_rcp_f32 v36, v36\n"
    "v_mfma_f32_16x16x32_bf16 v[176:179], v[220:223], v[196:199], v[176:179]\n"
    "v_rcp_f32 v37, v37\n"
    "v_mfma_f32_16x16x32_bf16 v[176:179], v[224:227], v[200:203], v[176:179]\n"
    "v_rcp_f32 v38, v38\n"
    "v_mfma_f32_16x16x32_bf16 v[16:19], v[228:231], v[196:199], v[16:19]\n"
    "v_rcp_f32 v39, v39\n"
    "v_mfma_f32_16x16x32_bf16 v[16:19], v[232:235], v[200:203], v[16:19]\n"
    "v_mul_f32 v132, v132, v36\n"
    "v_mul_f32 v133, v133, v37\n"
    "v_mfma_f32_16x16x32_bf16 v[24:27], v[236:239], v[196:199], v[24:27]\n"
    "v_mul_f32 v134, v134, v38\n"
    "v_mul_f32 v135, v135, v39\n"
    "v_mfma_f32_16x16x32_bf16 v[24:27], v[240:243], v[200:203], v[24:27]\n"
    "v_med3_f32 v36, v136, -s62, s62\n"
    "v_med3_f32 v37, v137, -s62, s62\n"
    "v_mfma_f32_16x16x32_bf16 v[32:35], v[212:215], v[204:207], v[32:35]\n"
    "v_med3_f32 v38, v138, -s62, s62\n"
    "v_med3_f32 v39, v139, -s62, s62\n"
    "v_mfma_f32_16x16x32_bf16 v[32:35], v[216:219], v[208:211], v[32:35]\n"
    "v_mul_f32 v36, s63, v36\n"
    "v_mul_f32 v37, s63, v37\n"
    "v_mfma_f32_16x16x32_bf16 v[40:43], v[220:223], v[204:207], v[40:43]\n"
    "v_mul_f32 v38, s63, v38\n"
    "v_mul_f32 v39, s63, v39\n"
    "v_mfma_f32_16x16x32_bf16 v[40:43], v[224:227], v[208:211], v[40:43]\n"
    "v_exp_f32 v36, v36\n"
    "v_mfma_f32_16x16x32_bf16 v[12:15], v[228:231], v[204:207], v[12:15]\n"
    "v_exp_f32 v37, v37\n"
    "v_mfma_f32_16x16x32_bf16 v[12:15], v[232:235], v[208:211], v[12:15]\n"
    "v_exp_f32 v38, v38\n"
    "v_mfma_f32_16x16x32_bf16 v[48:51], v[236:239], v[204:207], v[48:51]\n"
    "v_exp_f32 v39, v39\n"
    "v_mfma_f32_16x16x32_bf16 v[48:51], v[240:243], v[208:211], v[48:51]\n"
    "s_setprio 0\n"
    "s_waitcnt vmcnt(10)\n"
    "s_barrier\n"
    "; phase 88: step 56 slot 0 stage 2\n"
    "ds_read_b128 v[180:183], v244 offset:32768\n"
    "ds_read_b128 v[184:187], v245 offset:32768\n"
    "ds_read_b128 v[188:191], v244 offset:34816\n"
    "ds_read_b128 v[192:195], v245 offset:34816\n"
    "s_add_u32 m0, s51, 0xc000\n"
    "s_xor_b32 s82, s67, 0x3900000\n"
    "buffer_load_dwordx4 v251, s[40:43], s82 offen  lds\n"
    "ds_read_b128 v[196:199], v244 offset:40960\n"
    "ds_read_b128 v[200:203], v245 offset:40960\n"
    "ds_read_b128 v[204:207], v244 offset:43008\n"
    "ds_read_b128 v[208:211], v245 offset:43008\n"
    "s_add_u32 m0, s51, 0xe000\n"
    "s_xor_b32 s82, s67, 0x3980000\n"
    "buffer_load_dwordx4 v251, s[40:43], s82 offen  lds\n"
    "ds_read_b64_tr_b16 v[212:213], v246 offset:0\n"
    "ds_read_b64_tr_b16 v[214:215], v246 offset:1024\n"
    "ds_read_b64_tr_b16 v[216:217], v246 offset:8192\n"
    "ds_read_b64_tr_b16 v[218:219], v246 offset:9216\n"
    "s_add_u32 m0, s50, 0x4000\n"
    "s_xor_b32 s82, s48, 0x201d00\n"
    "buffer_load_dwordx4 v250, s[36:39], s82 offen  lds\n"
    "ds_read_b64_tr_b16 v[220:221], v247 offset:0\n"
    "ds_read_b64_tr_b16 v[222:223], v247 offset:1024\n"
    "ds_read_b64_tr_b16 v[224:225], v247 offset:8192\n"
    "ds_read_b64_tr_b16 v[226:227], v247 offset:9216\n"
    "s_add_u32 m0, s50, 0x5000\n"
    "s_xor_b32 s82, s48, 0x281d00\n"
    "buffer_load_dwordx4 v250, s[36:39], s82 offen  lds\n"
    "ds_read_b64_tr_b16 v[228:229], v248 offset:0\n"
    "ds_read_b64_tr_b16 v[230:231], v248 offset:1024\n"
    "ds_read_b64_tr_b16 v[232:233], v248 offset:8192\n"
    "ds_read_b64_tr_b16 v[234:235], v248 offset:9216\n"
    "s_add_u32 m0, s50, 0x6000\n"
    "s_xor_b32 s82, s48, 0x1d00\n"
    "buffer_load_dwordx4 v250, s[36:39], s82 offen  lds\n"
    "ds_read_b64_tr_b16 v[236:237], v249 offset:0\n"
    "ds_read_b64_tr_b16 v[238:239], v249 offset:1024\n"
    "ds_read_b64_tr_b16 v[240:241], v249 offset:8192\n"
    "ds_read_b64_tr_b16 v[242:243], v249 offset:9216\n"
    "s_add_u32 m0, s50, 0x7000\n"
    "s_xor_b32 s82, s48, 0x81d00\n"
    "buffer_load_dwordx4 v250, s[36:39], s82 offen  lds\n"
    "s_waitcnt vmcnt(12) lgkmcnt(0)\n"
    "s_barrier\n"
    "s_setprio 1\n"
    "v_mfma_f32_16x16x32_bf16 v[140:143], v[212:215], v[180:183], v[140:143]\n"
    "v_add_f32 v36, 1.0, v36\n"
    "v_add_f32 v37, 1.0, v37\n"
    "v_mfma_f32_16x16x32_bf16 v[140:143], v[216:219], v[184:187], v[140:143]\n"
    "v_add_f32 v38, 1.0, v38\n"
    "v_add_f32 v39, 1.0, v39\n"
    "v_mfma_f32_16x16x32_bf16 v[144:147], v[220:223], v[180:183], v[144:147]\n"
    "v_rcp_f32 v36, v36\n"
    "v_mfma_f32_16x16x32_bf16 v[144:147], v[224:227], v[184:187], v[144:147]\n"
    "v_rcp_f32 v37, v37\n"
    "v_mfma_f32_16x16x32_bf16 v[148:151], v[228:231], v[180:183], v[148:151]\n"
    "v_rcp_f32 v38, v38\n"
    "v_mfma_f32_16x16x32_bf16 v[148:151], v[232:235], v[184:187], v[148:151]\n"
    "v_rcp_f32 v39, v39\n"
    "v_mfma_f32_16x16x32_bf16 v[152:155], v[236:239], v[180:183], v[152:155]\n"
    "v_mul_f32 v136, v136, v36\n"
    "v_mul_f32 v137, v137, v37\n"
    "v_mfma_f32_16x16x32_bf16 v[152:155], v[240:243], v[184:187], v[152:155]\n"
    "v_mul_f32 v138, v138, v38\n"
    "v_mul_f32 v139, v139, v39\n"
    "v_mfma_f32_16x16x32_bf16 v[156:159], v[212:215], v[188:191], v[156:159]\n"
    "v_cvt_pk_bf16_f32 v36, v132, v133\n"
    "v_cvt_pk_bf16_f32 v37, v134, v135\n"
    "v_mfma_f32_16x16x32_bf16 v[156:159], v[216:219], v[192:195], v[156:159]\n"
    "v_cvt_pk_bf16_f32 v38, v136, v137\n"
    "v_cvt_pk_bf16_f32 v39, v138, v139\n"
    "v_mfma_f32_16x16x32_bf16 v[160:163], v[220:223], v[188:191], v[160:163]\n"
    "v_permlane16_swap_b32 v36, v38\n"
    "v_permlane16_swap_b32 v37, v39\n"
    "v_mfma_f32_16x16x32_bf16 v[160:163], v[224:227], v[192:195], v[160:163]\n"
    "v_mfma_f32_16x16x32_bf16 v[124:127], v[2:5], v[72:75], v[124:127]\n"
    "v_mfma_f32_16x16x32_bf16 v[164:167], v[228:231], v[188:191], v[164:167]\n"
    "v_mfma_f32_16x16x32_bf16 v[128:131], v[0:3], v[72:75], v[128:131]\n"
    "v_mfma_f32_16x16x32_bf16 v[164:167], v[232:235], v[192:195], v[164:167]\n"
    "v_mfma_f32_16x16x32_bf16 v[124:127], v[2:5], v[44:47], v[124:127]\n"
    "v_mfma_f32_16x16x32_bf16 v[168:171], v[236:239], v[188:191], v[168:171]\n"
    "v_mfma_f32_16x16x32_bf16 v[128:131], v[0:3], v[44:47], v[128:131]\n"
    "v_mfma_f32_16x16x32_bf16 v[168:171], v[240:243], v[192:195], v[168:171]\n"
    "v_mfma_f32_16x16x32_bf16 v[172:175], v[212:215], v[196:199], v[172:175]\n"
    "v_mfma_f32_16x16x32_bf16 v[172:175], v[216:219], v[200:203], v[172:175]\n"
    "v_mfma_f32_16x16x32_bf16 v[176:179], v[220:223], v[196:199], v[176:179]\n"
    "v_mfma_f32_16x16x32_bf16 v[176:179], v[224:227], v[200:203], v[176:179]\n"
    "v_mfma_f32_16x16x32_bf16 v[16:19], v[228:231], v[196:199], v[16:19]\n"
    "v_mfma_f32_16x16x32_bf16 v[16:19], v[232:235], v[200:203], v[16:19]\n"
    "v_mfma_f32_16x16x32_bf16 v[24:27], v[236:239], v[196:199], v[24:27]\n"
    "v_med3_f32 v44, v124, -s62, s62\n"
    "v_med3_f32 v45, v125, -s62, s62\n"
    "v_mfma_f32_16x16x32_bf16 v[24:27], v[240:243], v[200:203], v[24:27]\n"
    "v_med3_f32 v46, v126, -s62, s62\n"
    "v_med3_f32 v47, v127, -s62, s62\n"
    "v_mfma_f32_16x16x32_bf16 v[32:35], v[212:215], v[204:207], v[32:35]\n"
    "v_mul_f32 v44, s63, v44\n"
    "v_mul_f32 v45, s63, v45\n"
    "v_mfma_f32_16x16x32_bf16 v[32:35], v[216:219], v[208:211], v[32:35]\n"
    "v_mul_f32 v46, s63, v46\n"
    "v_mul_f32 v47, s63, v47\n"
    "v_mfma_f32_16x16x32_bf16 v[40:43], v[220:223], v[204:207], v[40:43]\n"
    "v_exp_f32 v44, v44\n"
    "v_mfma_f32_16x16x32_bf16 v[40:43], v[224:227], v[208:211], v[40:43]\n"
    "v_exp_f32 v45, v45\n"
    "v_mfma_f32_16x16x32_bf16 v[12:15], v[228:231], v[204:207], v[12:15]\n"
    "v_exp_f32 v46, v46\n"
    "v_mfma_f32_16x16x32_bf16 v[12:15], v[232:235], v[208:211], v[12:15]\n"
    "v_exp_f32 v47, v47\n"
    "v_mfma_f32_16x16x32_bf16 v[48:51], v[236:239], v[204:207], v[48:51]\n"
    "v_add_f32 v44, 1.0, v44\n"
    "v_add_f32 v45, 1.0, v45\n"
    "v_mfma_f32_16x16x32_bf16 v[48:51], v[240:243], v[208:211], v[48:51]\n"
    "s_setprio 0\n"
    "s_barrier\n"
    "; phase 89: step 56 slot 1 stage 3\n"
    "ds_read_b64_tr_b16 v[212:213], v246 offset:16384\n"
    "ds_read_b64_tr_b16 v[214:215], v246 offset:17408\n"
    "s_add_u32 m0, s51, 0x0\n"
    "s_xor_b32 s82, s66, 0x3a00000\n"
    "buffer_load_dwordx4 v251, s[40:43], s82 offen  lds\n"
    "ds_read_b64_tr_b16 v[216:217], v246 offset:24576\n"
    "ds_read_b64_tr_b16 v[218:219], v246 offset:25600\n"
    "s_add_u32 m0, s51, 0x2000\n"
    "s_xor_b32 s82, s66, 0x3a80000\n"
    "buffer_load_dwordx4 v251, s[40:43], s82 offen  lds\n"
    "ds_read_b64_tr_b16 v[220:221], v247 offset:16384\n"
    "ds_read_b64_tr_b16 v[222:223], v247 offset:17408\n"
    "s_add_u32 m0, s50, 0x8000\n"
    "s_xor_b32 s82, s48, 0x201d80\n"
    "buffer_load_dwordx4 v250, s[36:39], s82 offen  lds\n"
    "ds_read_b64_tr_b16 v[224:225], v247 offset:24576\n"
    "ds_read_b64_tr_b16 v[226:227], v247 offset:25600\n"
    "s_add_u32 m0, s50, 0x9000\n"
    "s_xor_b32 s82, s48, 0x281d80\n"
    "buffer_load_dwordx4 v250, s[36:39], s82 offen  lds\n"
    "ds_read_b64_tr_b16 v[228:229], v248 offset:16384\n"
    "ds_read_b64_tr_b16 v[230:231], v248 offset:17408\n"
    "s_add_u32 m0, s50, 0xa000\n"
    "s_xor_b32 s82, s48, 0x1d80\n"
    "buffer_load_dwordx4 v250, s[36:39], s82 offen  lds\n"
    "ds_read_b64_tr_b16 v[232:233], v248 offset:24576\n"
    "ds_read_b64_tr_b16 v[234:235], v248 offset:25600\n"
    "s_add_u32 m0, s50, 0xb000\n"
    "s_xor_b32 s82, s48, 0x81d80\n"
    "buffer_load_dwordx4 v250, s[36:39], s82 offen  lds\n"
    "ds_read_b64_tr_b16 v[236:237], v249 offset:16384\n"
    "ds_read_b64_tr_b16 v[238:239], v249 offset:17408\n"
    "ds_read_b64_tr_b16 v[240:241], v249 offset:24576\n"
    "ds_read_b64_tr_b16 v[242:243], v249 offset:25600\n"
    "s_waitcnt vmcnt(16) lgkmcnt(0)\n"
    "s_barrier\n"
    "s_setprio 1\n"
    "v_mfma_f32_16x16x32_bf16 v[64:67], v[212:215], v[180:183], 0\n"
    "v_add_f32 v46, 1.0, v46\n"
    "v_add_f32 v47, 1.0, v47\n"
    "v_mfma_f32_16x16x32_bf16 v[64:67], v[216:219], v[184:187], v[64:67]\n"
    "v_rcp_f32 v44, v44\n"
    "v_mfma_f32_16x16x32_bf16 v[60:63], v[220:223], v[180:183], 0\n"
    "v_rcp_f32 v45, v45\n"
    "v_mfma_f32_16x16x32_bf16 v[60:63], v[224:227], v[184:187], v[60:63]\n"
    "v_rcp_f32 v46, v46\n"
    "v_mfma_f32_16x16x32_bf16 v[80:83], v[228:231], v[180:183], 0\n"
    "v_rcp_f32 v47, v47\n"
    "v_mfma_f32_16x16x32_bf16 v[80:83], v[232:235], v[184:187], v[80:83]\n"
    "v_mul_f32 v124, v124, v44\n"
    "v_mul_f32 v125, v125, v45\n"
    "v_mfma_f32_16x16x32_bf16 v[88:91], v[236:239], v[180:183], 0\n"
    "v_mul_f32 v126, v126, v46\n"
    "v_mul_f32 v127, v127, v47\n"
    "v_mfma_f32_16x16x32_bf16 v[88:91], v[240:243], v[184:187], v[88:91]\n"
    "v_med3_f32 v44, v128, -s62, s62\n"
    "v_med3_f32 v45, v129, -s62, s62\n"
    "v_mfma_f32_16x16x32_bf16 v[96:99], v[212:215], v[188:191], 0\n"
    "v_med3_f32 v46, v130, -s62, s62\n"
    "v_med3_f32 v47, v131, -s62, s62\n"
    "v_mfma_f32_16x16x32_bf16 v[96:99], v[216:219], v[192:195], v[96:99]\n"
    "v_mul_f32 v44, s63, v44\n"
    "v_mul_f32 v45, s63, v45\n"
    "v_mfma_f32_16x16x32_bf16 v[104:107], v[220:223], v[188:191], 0\n"
    "v_mul_f32 v46, s63, v46\n"
    "v_mul_f32 v47, s63, v47\n"
    "v_mfma_f32_16x16x32_bf16 v[104:107], v[224:227], v[192:195], v[104:107]\n"
    "v_exp_f32 v44, v44\n"
    "v_mfma_f32_16x16x32_bf16 v[76:79], v[228:231], v[188:191], 0\n"
    "v_exp_f32 v45, v45\n"
    "v_mfma_f32_16x16x32_bf16 v[76:79], v[232:235], v[192:195], v[76:79]\n"
    "v_exp_f32 v46, v46\n"
    "v_mfma_f32_16x16x32_bf16 v[112:115], v[236:239], v[188:191], 0\n"
    "v_exp_f32 v47, v47\n"
    "v_mfma_f32_16x16x32_bf16 v[112:115], v[240:243], v[192:195], v[112:115]\n"
    "v_add_f32 v44, 1.0, v44\n"
    "v_add_f32 v45, 1.0, v45\n"
    "v_mfma_f32_16x16x32_bf16 v[120:123], v[212:215], v[196:199], 0\n"
    "v_add_f32 v46, 1.0, v46\n"
    "v_add_f32 v47, 1.0, v47\n"
    "v_mfma_f32_16x16x32_bf16 v[120:123], v[216:219], v[200:203], v[120:123]\n"
    "v_rcp_f32 v44, v44\n"
    "v_mfma_f32_16x16x32_bf16 v[84:87], v[220:223], v[196:199], 0\n"
    "v_rcp_f32 v45, v45\n"
    "v_mfma_f32_16x16x32_bf16 v[84:87], v[224:227], v[200:203], v[84:87]\n"
    "v_rcp_f32 v46, v46\n"
    "v_mfma_f32_16x16x32_bf16 v[28:31], v[228:231], v[196:199], 0\n"
    "v_rcp_f32 v47, v47\n"
    "v_mfma_f32_16x16x32_bf16 v[28:31], v[232:235], v[200:203], v[28:31]\n"
    "v_mul_f32 v128, v128, v44\n"
    "v_mul_f32 v129, v129, v45\n"
    "v_mfma_f32_16x16x32_bf16 v[92:95], v[236:239], v[196:199], 0\n"
    "v_mul_f32 v130, v130, v46\n"
    "v_mul_f32 v131, v131, v47\n"
    "v_mfma_f32_16x16x32_bf16 v[92:95], v[240:243], v[200:203], v[92:95]\n"
    "v_cvt_pk_bf16_f32 v44, v124, v125\n"
    "v_cvt_pk_bf16_f32 v45, v126, v127\n"
    "v_mfma_f32_16x16x32_bf16 v[100:103], v[212:215], v[204:207], 0\n"
    "v_cvt_pk_bf16_f32 v46, v128, v129\n"
    "v_cvt_pk_bf16_f32 v47, v130, v131\n"
    "v_mfma_f32_16x16x32_bf16 v[100:103], v[216:219], v[208:211], v[100:103]\n"
    "v_permlane16_swap_b32 v44, v46\n"
    "v_permlane16_swap_b32 v45, v47\n"
    "v_mfma_f32_16x16x32_bf16 v[108:111], v[220:223], v[204:207], 0\n"
    "v_mov_b32 v124, v36\n"
    "v_mov_b32 v125, v37\n"
    "v_mfma_f32_16x16x32_bf16 v[108:111], v[224:227], v[208:211], v[108:111]\n"
    "v_mov_b32 v126, v38\n"
    "v_mov_b32 v127, v39\n"
    "v_mfma_f32_16x16x32_bf16 v[116:119], v[228:231], v[204:207], 0\n"
    "v_mov_b32_dpp v36, v44 row_ror:8 row_mask:0xf bank_mask:0xc\n"
    "v_mov_b32_dpp v37, v45 row_ror:8 row_mask:0xf bank_mask:0xc\n"
    "v_mfma_f32_16x16x32_bf16 v[116:119], v[232:235], v[208:211], v[116:119]\n"
    "v_mov_b32_dpp v38, v46 row_ror:8 row_mask:0xf bank_mask:0xc\n"
    "v_mov_b32_dpp v39, v47 row_ror:8 row_mask:0xf bank_mask:0xc\n"
    "v_mfma_f32_16x16x32_bf16 v[52:55], v[236:239], v[204:207], 0\n"
    "v_mov_b32_dpp v44, v124 row_ror:8 row_mask:0xf bank_mask:0x3\n"
    "v_mov_b32_dpp v45, v125 row_ror:8 row_mask:0xf bank_mask:0x3\n"
    "v_mfma_f32_16x16x32_bf16 v[52:55], v[240:243], v[208:211], v[52:55]\n"
    "s_setprio 0\n"
    "s_waitcnt vmcnt(12)\n"
    "s_barrier\n"
    "; phase 90: step 57 slot 0 stage 2\n"
    "ds_read_b128 v[180:183], v244 offset:0\n"
    "ds_read_b128 v[184:187], v245 offset:0\n"
    "ds_read_b128 v[188:191], v244 offset:2048\n"
    "ds_read_b128 v[192:195], v245 offset:2048\n"
    "s_add_u32 m0, s51, 0x4000\n"
    "s_xor_b32 s82, s67, 0x3a00000\n"
    "buffer_load_dwordx4 v251, s[40:43], s82 offen  lds\n"
    "ds_read_b128 v[196:199], v244 offset:8192\n"
    "ds_read_b128 v[200:203], v245 offset:8192\n"
    "ds_read_b128 v[204:207], v244 offset:10240\n"
    "ds_read_b128 v[208:211], v245 offset:10240\n"
    "s_add_u32 m0, s51, 0x6000\n"
    "s_xor_b32 s82, s67, 0x3a80000\n"
    "buffer_load_dwordx4 v251, s[40:43], s82 offen  lds\n"
    "ds_read_b64_tr_b16 v[212:213], v246 offset:32768\n"
    "ds_read_b64_tr_b16 v[214:215], v246 offset:33792\n"
    "ds_read_b64_tr_b16 v[216:217], v246 offset:40960\n"
    "ds_read_b64_tr_b16 v[218:219], v246 offset:41984\n"
    "ds_read_b64_tr_b16 v[220:221], v247 offset:32768\n"
    "ds_read_b64_tr_b16 v[222:223], v247 offset:33792\n"
    "ds_read_b64_tr_b16 v[224:225], v247 offset:40960\n"
    "ds_read_b64_tr_b16 v[226:227], v247 offset:41984\n"
    "ds_read_b64_tr_b16 v[228:229], v248 offset:32768\n"
    "ds_read_b64_tr_b16 v[230:231], v248 offset:33792\n"
    "ds_read_b64_tr_b16 v[232:233], v248 offset:40960\n"
    "ds_read_b64_tr_b16 v[234:235], v248 offset:41984\n"
    "ds_read_b64_tr_b16 v[236:237], v249 offset:32768\n"
    "ds_read_b64_tr_b16 v[238:239], v249 offset:33792\n"
    "ds_read_b64_tr_b16 v[240:241], v249 offset:40960\n"
    "ds_read_b64_tr_b16 v[242:243], v249 offset:41984\n"
    "s_waitcnt vmcnt(12) lgkmcnt(0)\n"
    "s_barrier\n"
    "s_setprio 1\n"
    "v_mfma_f32_16x16x32_bf16 v[140:143], v[212:215], v[180:183], v[140:143]\n"
    "v_mov_b32_dpp v46, v126 row_ror:8 row_mask:0xf bank_mask:0x3\n"
    "v_mov_b32_dpp v47, v127 row_ror:8 row_mask:0xf bank_mask:0x3\n"
    "v_mfma_f32_16x16x32_bf16 v[140:143], v[216:219], v[184:187], v[140:143]\n"
    "v_mfma_f32_16x16x32_bf16 v[144:147], v[220:223], v[180:183], v[144:147]\n"
    "v_mfma_f32_16x16x32_bf16 v[144:147], v[224:227], v[184:187], v[144:147]\n"
    "v_mfma_f32_16x16x32_bf16 v[148:151], v[228:231], v[180:183], v[148:151]\n"
    "v_mfma_f32_16x16x32_bf16 v[148:151], v[232:235], v[184:187], v[148:151]\n"
    "v_mfma_f32_16x16x32_bf16 v[152:155], v[236:239], v[180:183], v[152:155]\n"
    "v_mfma_f32_16x16x32_bf16 v[152:155], v[240:243], v[184:187], v[152:155]\n"
    "v_mfma_f32_16x16x32_bf16 v[156:159], v[212:215], v[188:191], v[156:159]\n"
    "v_mfma_f32_16x16x32_bf16 v[156:159], v[216:219], v[192:195], v[156:159]\n"
    "v_mfma_f32_16x16x32_bf16 v[160:163], v[220:223], v[188:191], v[160:163]\n"
    "v_mfma_f32_16x16x32_bf16 v[160:163], v[224:227], v[192:195], v[160:163]\n"
    "v_mfma_f32_16x16x32_bf16 v[164:167], v[228:231], v[188:191], v[164:167]\n"
    "v_mfma_f32_16x16x32_bf16 v[164:167], v[232:235], v[192:195], v[164:167]\n"
    "v_mfma_f32_16x16x32_bf16 v[168:171], v[236:239], v[188:191], v[168:171]\n"
    "v_mfma_f32_16x16x32_bf16 v[168:171], v[240:243], v[192:195], v[168:171]\n"
    "v_mfma_f32_16x16x32_bf16 v[172:175], v[212:215], v[196:199], v[172:175]\n"
    "v_mfma_f32_16x16x32_bf16 v[172:175], v[216:219], v[200:203], v[172:175]\n"
    "v_mfma_f32_16x16x32_bf16 v[176:179], v[220:223], v[196:199], v[176:179]\n"
    "v_mfma_f32_16x16x32_bf16 v[176:179], v[224:227], v[200:203], v[176:179]\n"
    "v_mfma_f32_16x16x32_bf16 v[16:19], v[228:231], v[196:199], v[16:19]\n"
    "v_mfma_f32_16x16x32_bf16 v[16:19], v[232:235], v[200:203], v[16:19]\n"
    "v_mfma_f32_16x16x32_bf16 v[24:27], v[236:239], v[196:199], v[24:27]\n"
    "v_mfma_f32_16x16x32_bf16 v[24:27], v[240:243], v[200:203], v[24:27]\n"
    "v_mfma_f32_16x16x32_bf16 v[32:35], v[212:215], v[204:207], v[32:35]\n"
    "v_mfma_f32_16x16x32_bf16 v[32:35], v[216:219], v[208:211], v[32:35]\n"
    "v_mfma_f32_16x16x32_bf16 v[40:43], v[220:223], v[204:207], v[40:43]\n"
    "v_mfma_f32_16x16x32_bf16 v[40:43], v[224:227], v[208:211], v[40:43]\n"
    "v_mfma_f32_16x16x32_bf16 v[12:15], v[228:231], v[204:207], v[12:15]\n"
    "v_mfma_f32_16x16x32_bf16 v[12:15], v[232:235], v[208:211], v[12:15]\n"
    "v_mfma_f32_16x16x32_bf16 v[48:51], v[236:239], v[204:207], v[48:51]\n"
    "v_mfma_f32_16x16x32_bf16 v[48:51], v[240:243], v[208:211], v[48:51]\n"
    "s_setprio 0\n"
    "s_barrier\n"
    "; phase 91: step 57 slot 1 stage 3\n"
    "ds_read_b64_tr_b16 v[212:213], v246 offset:49152\n"
    "ds_read_b64_tr_b16 v[214:215], v246 offset:50176\n"
    "s_cmp_lg_u32 s52, 0\n"
    "s_cbranch_scc1 .La125_%=\n"
    "buffer_store_dwordx4 v[36:39], v255, s[76:79], s81 offen nt\n"
    ".La125_%=:\n"
    "ds_read_b64_tr_b16 v[216:217], v246 offset:57344\n"
    "ds_read_b64_tr_b16 v[218:219], v246 offset:58368\n"
    "s_cmp_lg_u32 s52, 1\n"
    "s_cbranch_scc1 .La126_%=\n"
    "buffer_store_dwordx4 v[36:39], v255, s[76:79], s81 offen nt\n"
    ".La126_%=:\n"
    "ds_read_b64_tr_b16 v[220:221], v247 offset:49152\n"
    "ds_read_b64_tr_b16 v[222:223], v247 offset:50176\n"
    "s_add_u32 m0, s51, 0x8000\n"
    "s_xor_b32 s82, s66, 0x3b00000\n"
    "buffer_load_dwordx4 v251, s[40:43], s82 offen  lds\n"
    "ds_read_b64_tr_b16 v[224:225], v247 offset:57344\n"
    "ds_read_b64_tr_b16 v[226:227], v247 offset:58368\n"
    "s_add_u32 m0, s51, 0xa000\n"
    "s_xor_b32 s82, s66, 0x3b80000\n"
    "buffer_load_dwordx4 v251, s[40:43], s82 offen  lds\n"
    "ds_read_b64_tr_b16 v[228:229], v248 offset:49152\n"
    "ds_read_b64_tr_b16 v[230:231], v248 offset:50176\n"
    "s_add_u32 m0, s50, 0x0\n"
    "s_xor_b32 s82, s48, 0x201e00\n"
    "buffer_load_dwordx4 v250, s[36:39], s82 offen  lds\n"
    "ds_read_b64_tr_b16 v[232:233], v248 offset:57344\n"
    "ds_read_b64_tr_b16 v[234:235], v248 offset:58368\n"
    "s_add_u32 m0, s50, 0x1000\n"
    "s_xor_b32 s82, s48, 0x281e00\n"
    "buffer_load_dwordx4 v250, s[36:39], s82 offen  lds\n"
    "ds_read_b64_tr_b16 v[236:237], v249 offset:49152\n"
    "ds_read_b64_tr_b16 v[238:239], v249 offset:50176\n"
    "s_add_u32 m0, s50, 0x2000\n"
    "s_xor_b32 s82, s48, 0x1e00\n"
    "buffer_load_dwordx4 v250, s[36:39], s82 offen  lds\n"
    "ds_read_b64_tr_b16 v[240:241], v249 offset:57344\n"
    "ds_read_b64_tr_b16 v[242:243], v249 offset:58368\n"
    "s_add_u32 m0, s50, 0x3000\n"
    "s_xor_b32 s82, s48, 0x81e00\n"
    "buffer_load_dwordx4 v250, s[36:39], s82 offen  lds\n"
    "s_waitcnt vmcnt(12) lgkmcnt(0)\n"
    "s_barrier\n"
    "s_setprio 1\n"
    "v_mfma_f32_16x16x32_bf16 v[64:67], v[212:215], v[180:183], v[64:67]\n"
    "v_mfma_f32_16x16x32_bf16 v[64:67], v[216:219], v[184:187], v[64:67]\n"
    "v_mfma_f32_16x16x32_bf16 v[60:63], v[220:223], v[180:183], v[60:63]\n"
    "v_mfma_f32_16x16x32_bf16 v[60:63], v[224:227], v[184:187], v[60:63]\n"
    "v_mfma_f32_16x16x32_bf16 v[80:83], v[228:231], v[180:183], v[80:83]\n"
    "v_mfma_f32_16x16x32_bf16 v[80:83], v[232:235], v[184:187], v[80:83]\n"
    "v_mfma_f32_16x16x32_bf16 v[88:91], v[236:239], v[180:183], v[88:91]\n"
    "v_mfma_f32_16x16x32_bf16 v[88:91], v[240:243], v[184:187], v[88:91]\n"
    "v_mfma_f32_16x16x32_bf16 v[96:99], v[212:215], v[188:191], v[96:99]\n"
    "v_mfma_f32_16x16x32_bf16 v[96:99], v[216:219], v[192:195], v[96:99]\n"
    "v_mfma_f32_16x16x32_bf16 v[104:107], v[220:223], v[188:191], v[104:107]\n"
    "v_mfma_f32_16x16x32_bf16 v[104:107], v[224:227], v[192:195], v[104:107]\n"
    "v_mfma_f32_16x16x32_bf16 v[76:79], v[228:231], v[188:191], v[76:79]\n"
    "v_mfma_f32_16x16x32_bf16 v[76:79], v[232:235], v[192:195], v[76:79]\n"
    "v_mfma_f32_16x16x32_bf16 v[112:115], v[236:239], v[188:191], v[112:115]\n"
    "v_mfma_f32_16x16x32_bf16 v[112:115], v[240:243], v[192:195], v[112:115]\n"
    "v_mfma_f32_16x16x32_bf16 v[120:123], v[212:215], v[196:199], v[120:123]\n"
    "v_mfma_f32_16x16x32_bf16 v[120:123], v[216:219], v[200:203], v[120:123]\n"
    "v_mfma_f32_16x16x32_bf16 v[84:87], v[220:223], v[196:199], v[84:87]\n"
    "v_mfma_f32_16x16x32_bf16 v[84:87], v[224:227], v[200:203], v[84:87]\n"
    "v_mfma_f32_16x16x32_bf16 v[28:31], v[228:231], v[196:199], v[28:31]\n"
    "v_mfma_f32_16x16x32_bf16 v[28:31], v[232:235], v[200:203], v[28:31]\n"
    "v_mfma_f32_16x16x32_bf16 v[92:95], v[236:239], v[196:199], v[92:95]\n"
    "v_mfma_f32_16x16x32_bf16 v[92:95], v[240:243], v[200:203], v[92:95]\n"
    "v_mfma_f32_16x16x32_bf16 v[100:103], v[212:215], v[204:207], v[100:103]\n"
    "v_mfma_f32_16x16x32_bf16 v[100:103], v[216:219], v[208:211], v[100:103]\n"
    "v_mfma_f32_16x16x32_bf16 v[108:111], v[220:223], v[204:207], v[108:111]\n"
    "v_mfma_f32_16x16x32_bf16 v[108:111], v[224:227], v[208:211], v[108:111]\n"
    "v_mfma_f32_16x16x32_bf16 v[116:119], v[228:231], v[204:207], v[116:119]\n"
    "v_mfma_f32_16x16x32_bf16 v[116:119], v[232:235], v[208:211], v[116:119]\n"
    "v_mfma_f32_16x16x32_bf16 v[52:55], v[236:239], v[204:207], v[52:55]\n"
    "v_mfma_f32_16x16x32_bf16 v[52:55], v[240:243], v[208:211], v[52:55]\n"
    "s_setprio 0\n"
    "s_barrier\n"
    "; phase 92: step 58 slot 0 stage 2\n"
    "ds_read_b128 v[180:183], v244 offset:16384\n"
    "ds_read_b128 v[184:187], v245 offset:16384\n"
    "ds_read_b128 v[188:191], v244 offset:18432\n"
    "ds_read_b128 v[192:195], v245 offset:18432\n"
    "s_cmp_lg_u32 s52, 2\n"
    "s_cbranch_scc1 .La127_%=\n"
    "buffer_store_dwordx4 v[36:39], v255, s[76:79], s81 offen nt\n"
    ".La127_%=:\n"
    "ds_read_b128 v[196:199], v244 offset:24576\n"
    "ds_read_b128 v[200:203], v245 offset:24576\n"
    "ds_read_b128 v[204:207], v244 offset:26624\n"
    "ds_read_b128 v[208:211], v245 offset:26624\n"
    "s_add_u32 m0, s51, 0xc000\n"
    "s_xor_b32 s82, s67, 0x3b00000\n"
    "buffer_load_dwordx4 v251, s[40:43], s82 offen  lds\n"
    "ds_read_b64_tr_b16 v[212:213], v246 offset:0\n"
    "ds_read_b64_tr_b16 v[214:215], v246 offset:1024\n"
    "ds_read_b64_tr_b16 v[216:217], v246 offset:8192\n"
    "ds_read_b64_tr_b16 v[218:219], v246 offset:9216\n"
    "s_add_u32 m0, s51, 0xe000\n"
    "s_xor_b32 s82, s67, 0x3b80000\n"
    "buffer_load_dwordx4 v251, s[40:43], s82 offen  lds\n"
    "ds_read_b64_tr_b16 v[220:221], v247 offset:0\n"
    "ds_read_b64_tr_b16 v[222:223], v247 offset:1024\n"
    "ds_read_b64_tr_b16 v[224:225], v247 offset:8192\n"
    "ds_read_b64_tr_b16 v[226:227], v247 offset:9216\n"
    "ds_read_b64_tr_b16 v[228:229], v248 offset:0\n"
    "ds_read_b64_tr_b16 v[230:231], v248 offset:1024\n"
    "ds_read_b64_tr_b16 v[232:233], v248 offset:8192\n"
    "ds_read_b64_tr_b16 v[234:235], v248 offset:9216\n"
    "ds_read_b64_tr_b16 v[236:237], v249 offset:0\n"
    "ds_read_b64_tr_b16 v[238:239], v249 offset:1024\n"
    "ds_read_b64_tr_b16 v[240:241], v249 offset:8192\n"
    "ds_read_b64_tr_b16 v[242:243], v249 offset:9216\n"
    "s_waitcnt vmcnt(8) lgkmcnt(0)\n"
    "s_barrier\n"
    "s_setprio 1\n"
    "v_mfma_f32_16x16x32_bf16 v[140:143], v[212:215], v[180:183], v[140:143]\n"
    "v_mfma_f32_16x16x32_bf16 v[140:143], v[216:219], v[184:187], v[140:143]\n"
    "v_mfma_f32_16x16x32_bf16 v[144:147], v[220:223], v[180:183], v[144:147]\n"
    "v_mfma_f32_16x16x32_bf16 v[144:147], v[224:227], v[184:187], v[144:147]\n"
    "v_mfma_f32_16x16x32_bf16 v[148:151], v[228:231], v[180:183], v[148:151]\n"
    "v_mfma_f32_16x16x32_bf16 v[148:151], v[232:235], v[184:187], v[148:151]\n"
    "v_mfma_f32_16x16x32_bf16 v[152:155], v[236:239], v[180:183], v[152:155]\n"
    "v_mfma_f32_16x16x32_bf16 v[152:155], v[240:243], v[184:187], v[152:155]\n"
    "v_mfma_f32_16x16x32_bf16 v[156:159], v[212:215], v[188:191], v[156:159]\n"
    "v_mfma_f32_16x16x32_bf16 v[156:159], v[216:219], v[192:195], v[156:159]\n"
    "v_mfma_f32_16x16x32_bf16 v[160:163], v[220:223], v[188:191], v[160:163]\n"
    "v_mfma_f32_16x16x32_bf16 v[160:163], v[224:227], v[192:195], v[160:163]\n"
    "v_mfma_f32_16x16x32_bf16 v[164:167], v[228:231], v[188:191], v[164:167]\n"
    "v_mfma_f32_16x16x32_bf16 v[164:167], v[232:235], v[192:195], v[164:167]\n"
    "v_mfma_f32_16x16x32_bf16 v[168:171], v[236:239], v[188:191], v[168:171]\n"
    "v_mfma_f32_16x16x32_bf16 v[168:171], v[240:243], v[192:195], v[168:171]\n"
    "v_mfma_f32_16x16x32_bf16 v[172:175], v[212:215], v[196:199], v[172:175]\n"
    "v_mfma_f32_16x16x32_bf16 v[172:175], v[216:219], v[200:203], v[172:175]\n"
    "v_mfma_f32_16x16x32_bf16 v[176:179], v[220:223], v[196:199], v[176:179]\n"
    "v_mfma_f32_16x16x32_bf16 v[176:179], v[224:227], v[200:203], v[176:179]\n"
    "v_mfma_f32_16x16x32_bf16 v[16:19], v[228:231], v[196:199], v[16:19]\n"
    "v_mfma_f32_16x16x32_bf16 v[16:19], v[232:235], v[200:203], v[16:19]\n"
    "v_mfma_f32_16x16x32_bf16 v[24:27], v[236:239], v[196:199], v[24:27]\n"
    "v_mfma_f32_16x16x32_bf16 v[24:27], v[240:243], v[200:203], v[24:27]\n"
    "v_mfma_f32_16x16x32_bf16 v[32:35], v[212:215], v[204:207], v[32:35]\n"
    "v_mfma_f32_16x16x32_bf16 v[32:35], v[216:219], v[208:211], v[32:35]\n"
    "v_mfma_f32_16x16x32_bf16 v[40:43], v[220:223], v[204:207], v[40:43]\n"
    "v_mfma_f32_16x16x32_bf16 v[40:43], v[224:227], v[208:211], v[40:43]\n"
    "v_mfma_f32_16x16x32_bf16 v[12:15], v[228:231], v[204:207], v[12:15]\n"
    "v_mfma_f32_16x16x32_bf16 v[12:15], v[232:235], v[208:211], v[12:15]\n"
    "v_mfma_f32_16x16x32_bf16 v[48:51], v[236:239], v[204:207], v[48:51]\n"
    "v_mfma_f32_16x16x32_bf16 v[48:51], v[240:243], v[208:211], v[48:51]\n"
    "s_setprio 0\n"
    "s_barrier\n"
    "; phase 93: step 58 slot 1 stage 3\n"
    "ds_read_b64_tr_b16 v[212:213], v246 offset:16384\n"
    "ds_read_b64_tr_b16 v[214:215], v246 offset:17408\n"
    "s_cmp_lg_u32 s52, 3\n"
    "s_cbranch_scc1 .La128_%=\n"
    "buffer_store_dwordx4 v[36:39], v255, s[76:79], s81 offen nt\n"
    ".La128_%=:\n"
    "ds_read_b64_tr_b16 v[216:217], v246 offset:24576\n"
    "ds_read_b64_tr_b16 v[218:219], v246 offset:25600\n"
    "s_add_u32 m0, s51, 0x0\n"
    "s_xor_b32 s82, s66, 0x3c00000\n"
    "buffer_load_dwordx4 v251, s[40:43], s82 offen  lds\n"
    "ds_read_b64_tr_b16 v[220:221], v247 offset:16384\n"
    "ds_read_b64_tr_b16 v[222:223], v247 offset:17408\n"
    "s_add_u32 m0, s51, 0x2000\n"
    "s_xor_b32 s82, s66, 0x3c80000\n"
    "buffer_load_dwordx4 v251, s[40:43], s82 offen  lds\n"
    "ds_read_b64_tr_b16 v[224:225], v247 offset:24576\n"
    "ds_read_b64_tr_b16 v[226:227], v247 offset:25600\n"
    "s_add_u32 m0, s50, 0x4000\n"
    "s_xor_b32 s82, s48, 0x201e80\n"
    "buffer_load_dwordx4 v250, s[36:39], s82 offen  lds\n"
    "ds_read_b64_tr_b16 v[228:229], v248 offset:16384\n"
    "ds_read_b64_tr_b16 v[230:231], v248 offset:17408\n"
    "s_add_u32 m0, s50, 0x5000\n"
    "s_xor_b32 s82, s48, 0x281e80\n"
    "buffer_load_dwordx4 v250, s[36:39], s82 offen  lds\n"
    "ds_read_b64_tr_b16 v[232:233], v248 offset:24576\n"
    "ds_read_b64_tr_b16 v[234:235], v248 offset:25600\n"
    "s_add_u32 m0, s50, 0x6000\n"
    "s_xor_b32 s82, s48, 0x1e80\n"
    "buffer_load_dwordx4 v250, s[36:39], s82 offen  lds\n"
    "ds_read_b64_tr_b16 v[236:237], v249 offset:16384\n"
    "ds_read_b64_tr_b16 v[238:239], v249 offset:17408\n"
    "s_add_u32 m0, s50, 0x7000\n"
    "s_xor_b32 s82, s48, 0x81e80\n"
    "buffer_load_dwordx4 v250, s[36:39], s82 offen  lds\n"
    "ds_read_b64_tr_b16 v[240:241], v249 offset:24576\n"
    "ds_read_b64_tr_b16 v[242:243], v249 offset:25600\n"
    "s_waitcnt vmcnt(12) lgkmcnt(0)\n"
    "s_barrier\n"
    "s_setprio 1\n"
    "v_mfma_f32_16x16x32_bf16 v[64:67], v[212:215], v[180:183], v[64:67]\n"
    "v_mfma_f32_16x16x32_bf16 v[64:67], v[216:219], v[184:187], v[64:67]\n"
    "v_mfma_f32_16x16x32_bf16 v[60:63], v[220:223], v[180:183], v[60:63]\n"
    "v_mfma_f32_16x16x32_bf16 v[60:63], v[224:227], v[184:187], v[60:63]\n"
    "v_mfma_f32_16x16x32_bf16 v[80:83], v[228:231], v[180:183], v[80:83]\n"
    "v_mfma_f32_16x16x32_bf16 v[80:83], v[232:235], v[184:187], v[80:83]\n"
    "v_mfma_f32_16x16x32_bf16 v[88:91], v[236:239], v[180:183], v[88:91]\n"
    "v_mfma_f32_16x16x32_bf16 v[88:91], v[240:243], v[184:187], v[88:91]\n"
    "v_mfma_f32_16x16x32_bf16 v[96:99], v[212:215], v[188:191], v[96:99]\n"
    "v_mfma_f32_16x16x32_bf16 v[96:99], v[216:219], v[192:195], v[96:99]\n"
    "v_mfma_f32_16x16x32_bf16 v[104:107], v[220:223], v[188:191], v[104:107]\n"
    "v_mfma_f32_16x16x32_bf16 v[104:107], v[224:227], v[192:195], v[104:107]\n"
    "v_mfma_f32_16x16x32_bf16 v[76:79], v[228:231], v[188:191], v[76:79]\n"
    "v_mfma_f32_16x16x32_bf16 v[76:79], v[232:235], v[192:195], v[76:79]\n"
    "v_mfma_f32_16x16x32_bf16 v[112:115], v[236:239], v[188:191], v[112:115]\n"
    "v_mfma_f32_16x16x32_bf16 v[112:115], v[240:243], v[192:195], v[112:115]\n"
    "v_mfma_f32_16x16x32_bf16 v[120:123], v[212:215], v[196:199], v[120:123]\n"
    "v_mfma_f32_16x16x32_bf16 v[120:123], v[216:219], v[200:203], v[120:123]\n"
    "v_mfma_f32_16x16x32_bf16 v[84:87], v[220:223], v[196:199], v[84:87]\n"
    "v_mfma_f32_16x16x32_bf16 v[84:87], v[224:227], v[200:203], v[84:87]\n"
    "v_mfma_f32_16x16x32_bf16 v[28:31], v[228:231], v[196:199], v[28:31]\n"
    "v_mfma_f32_16x16x32_bf16 v[28:31], v[232:235], v[200:203], v[28:31]\n"
    "v_mfma_f32_16x16x32_bf16 v[92:95], v[236:239], v[196:199], v[92:95]\n"
    "v_mfma_f32_16x16x32_bf16 v[92:95], v[240:243], v[200:203], v[92:95]\n"
    "v_mfma_f32_16x16x32_bf16 v[100:103], v[212:215], v[204:207], v[100:103]\n"
    "v_mfma_f32_16x16x32_bf16 v[100:103], v[216:219], v[208:211], v[100:103]\n"
    "v_mfma_f32_16x16x32_bf16 v[108:111], v[220:223], v[204:207], v[108:111]\n"
    "v_mfma_f32_16x16x32_bf16 v[108:111], v[224:227], v[208:211], v[108:111]\n"
    "v_mfma_f32_16x16x32_bf16 v[116:119], v[228:231], v[204:207], v[116:119]\n"
    "v_mfma_f32_16x16x32_bf16 v[116:119], v[232:235], v[208:211], v[116:119]\n"
    "v_mfma_f32_16x16x32_bf16 v[52:55], v[236:239], v[204:207], v[52:55]\n"
    "v_mfma_f32_16x16x32_bf16 v[52:55], v[240:243], v[208:211], v[52:55]\n"
    "s_setprio 0\n"
    "s_barrier\n"
    "; phase 94: step 59 slot 0 stage 2\n"
    "ds_read_b128 v[180:183], v244 offset:32768\n"
    "ds_read_b128 v[184:187], v245 offset:32768\n"
    "ds_read_b128 v[188:191], v244 offset:34816\n"
    "ds_read_b128 v[192:195], v245 offset:34816\n"
    "s_cmp_lg_u32 s52, 4\n"
    "s_cbranch_scc1 .La129_%=\n"
    "buffer_store_dwordx4 v[36:39], v255, s[76:79], s81 offen nt\n"
    ".La129_%=:\n"
    "ds_read_b128 v[196:199], v244 offset:40960\n"
    "ds_read_b128 v[200:203], v245 offset:40960\n"
    "ds_read_b128 v[204:207], v244 offset:43008\n"
    "ds_read_b128 v[208:211], v245 offset:43008\n"
    "s_add_u32 m0, s51, 0x4000\n"
    "s_xor_b32 s82, s67, 0x3c00000\n"
    "buffer_load_dwordx4 v251, s[40:43], s82 offen  lds\n"
    "ds_read_b64_tr_b16 v[212:213], v246 offset:32768\n"
    "ds_read_b64_tr_b16 v[214:215], v246 offset:33792\n"
    "ds_read_b64_tr_b16 v[216:217], v246 offset:40960\n"
    "ds_read_b64_tr_b16 v[218:219], v246 offset:41984\n"
    "s_add_u32 m0, s51, 0x6000\n"
    "s_xor_b32 s82, s67, 0x3c80000\n"
    "buffer_load_dwordx4 v251, s[40:43], s82 offen  lds\n"
    "ds_read_b64_tr_b16 v[220:221], v247 offset:32768\n"
    "ds_read_b64_tr_b16 v[222:223], v247 offset:33792\n"
    "ds_read_b64_tr_b16 v[224:225], v247 offset:40960\n"
    "ds_read_b64_tr_b16 v[226:227], v247 offset:41984\n"
    "ds_read_b64_tr_b16 v[228:229], v248 offset:32768\n"
    "ds_read_b64_tr_b16 v[230:231], v248 offset:33792\n"
    "ds_read_b64_tr_b16 v[232:233], v248 offset:40960\n"
    "ds_read_b64_tr_b16 v[234:235], v248 offset:41984\n"
    "ds_read_b64_tr_b16 v[236:237], v249 offset:32768\n"
    "ds_read_b64_tr_b16 v[238:239], v249 offset:33792\n"
    "ds_read_b64_tr_b16 v[240:241], v249 offset:40960\n"
    "ds_read_b64_tr_b16 v[242:243], v249 offset:41984\n"
    "s_waitcnt vmcnt(8) lgkmcnt(0)\n"
    "s_barrier\n"
    "s_setprio 1\n"
    "v_mfma_f32_16x16x32_bf16 v[140:143], v[212:215], v[180:183], v[140:143]\n"
    "v_mfma_f32_16x16x32_bf16 v[140:143], v[216:219], v[184:187], v[140:143]\n"
    "v_mfma_f32_16x16x32_bf16 v[144:147], v[220:223], v[180:183], v[144:147]\n"
    "v_mfma_f32_16x16x32_bf16 v[144:147], v[224:227], v[184:187], v[144:147]\n"
    "v_mfma_f32_16x16x32_bf16 v[148:151], v[228:231], v[180:183], v[148:151]\n"
    "v_mfma_f32_16x16x32_bf16 v[148:151], v[232:235], v[184:187], v[148:151]\n"
    "v_mfma_f32_16x16x32_bf16 v[152:155], v[236:239], v[180:183], v[152:155]\n"
    "v_mfma_f32_16x16x32_bf16 v[152:155], v[240:243], v[184:187], v[152:155]\n"
    "v_mfma_f32_16x16x32_bf16 v[156:159], v[212:215], v[188:191], v[156:159]\n"
    "v_mfma_f32_16x16x32_bf16 v[156:159], v[216:219], v[192:195], v[156:159]\n"
    "v_mfma_f32_16x16x32_bf16 v[160:163], v[220:223], v[188:191], v[160:163]\n"
    "v_mfma_f32_16x16x32_bf16 v[160:163], v[224:227], v[192:195], v[160:163]\n"
    "v_mfma_f32_16x16x32_bf16 v[164:167], v[228:231], v[188:191], v[164:167]\n"
    "v_mfma_f32_16x16x32_bf16 v[164:167], v[232:235], v[192:195], v[164:167]\n"
    "v_mfma_f32_16x16x32_bf16 v[168:171], v[236:239], v[188:191], v[168:171]\n"
    "v_mfma_f32_16x16x32_bf16 v[168:171], v[240:243], v[192:195], v[168:171]\n"
    "v_mfma_f32_16x16x32_bf16 v[172:175], v[212:215], v[196:199], v[172:175]\n"
    "v_mfma_f32_16x16x32_bf16 v[172:175], v[216:219], v[200:203], v[172:175]\n"
    "v_mfma_f32_16x16x32_bf16 v[176:179], v[220:223], v[196:199], v[176:179]\n"
    "v_mfma_f32_16x16x32_bf16 v[176:179], v[224:227], v[200:203], v[176:179]\n"
    "v_mfma_f32_16x16x32_bf16 v[16:19], v[228:231], v[196:199], v[16:19]\n"
    "v_mfma_f32_16x16x32_bf16 v[16:19], v[232:235], v[200:203], v[16:19]\n"
    "v_mfma_f32_16x16x32_bf16 v[24:27], v[236:239], v[196:199], v[24:27]\n"
    "v_mfma_f32_16x16x32_bf16 v[24:27], v[240:243], v[200:203], v[24:27]\n"
    "v_mfma_f32_16x16x32_bf16 v[32:35], v[212:215], v[204:207], v[32:35]\n"
    "v_mfma_f32_16x16x32_bf16 v[32:35], v[216:219], v[208:211], v[32:35]\n"
    "v_mfma_f32_16x16x32_bf16 v[40:43], v[220:223], v[204:207], v[40:43]\n"
    "v_mfma_f32_16x16x32_bf16 v[40:43], v[224:227], v[208:211], v[40:43]\n"
    "v_mfma_f32_16x16x32_bf16 v[12:15], v[228:231], v[204:207], v[12:15]\n"
    "v_mfma_f32_16x16x32_bf16 v[12:15], v[232:235], v[208:211], v[12:15]\n"
    "v_mfma_f32_16x16x32_bf16 v[48:51], v[236:239], v[204:207], v[48:51]\n"
    "v_mfma_f32_16x16x32_bf16 v[48:51], v[240:243], v[208:211], v[48:51]\n"
    "s_setprio 0\n"
    "s_barrier\n"
    "; phase 95: step 59 slot 1 stage 3\n"
    "ds_read_b64_tr_b16 v[212:213], v246 offset:49152\n"
    "ds_read_b64_tr_b16 v[214:215], v246 offset:50176\n"
    "s_cmp_lg_u32 s52, 5\n"
    "s_cbranch_scc1 .La130_%=\n"
    "buffer_store_dwordx4 v[36:39], v255, s[76:79], s81 offen nt\n"
    ".La130_%=:\n"
    "ds_read_b64_tr_b16 v[216:217], v246 offset:57344\n"
    "ds_read_b64_tr_b16 v[218:219], v246 offset:58368\n"
    "s_add_u32 m0, s51, 0x8000\n"
    "s_xor_b32 s82, s66, 0x3d00000\n"
    "buffer_load_dwordx4 v251, s[40:43], s82 offen  lds\n"
    "ds_read_b64_tr_b16 v[220:221], v247 offset:49152\n"
    "ds_read_b64_tr_b16 v[222:223], v247 offset:50176\n"
    "s_add_u32 m0, s51, 0xa000\n"
    "s_xor_b32 s82, s66, 0x3d80000\n"
    "buffer_load_dwordx4 v251, s[40:43], s82 offen  lds\n"
    "ds_read_b64_tr_b16 v[224:225], v247 offset:57344\n"
    "ds_read_b64_tr_b16 v[226:227], v247 offset:58368\n"
    "s_add_u32 m0, s50, 0x8000\n"
    "s_xor_b32 s82, s48, 0x201f00\n"
    "buffer_load_dwordx4 v250, s[36:39], s82 offen  lds\n"
    "ds_read_b64_tr_b16 v[228:229], v248 offset:49152\n"
    "ds_read_b64_tr_b16 v[230:231], v248 offset:50176\n"
    "s_add_u32 m0, s50, 0x9000\n"
    "s_xor_b32 s82, s48, 0x281f00\n"
    "buffer_load_dwordx4 v250, s[36:39], s82 offen  lds\n"
    "ds_read_b64_tr_b16 v[232:233], v248 offset:57344\n"
    "ds_read_b64_tr_b16 v[234:235], v248 offset:58368\n"
    "s_add_u32 m0, s50, 0xa000\n"
    "s_xor_b32 s82, s48, 0x1f00\n"
    "buffer_load_dwordx4 v250, s[36:39], s82 offen  lds\n"
    "ds_read_b64_tr_b16 v[236:237], v249 offset:49152\n"
    "ds_read_b64_tr_b16 v[238:239], v249 offset:50176\n"
    "s_add_u32 m0, s50, 0xb000\n"
    "s_xor_b32 s82, s48, 0x81f00\n"
    "buffer_load_dwordx4 v250, s[36:39], s82 offen  lds\n"
    "ds_read_b64_tr_b16 v[240:241], v249 offset:57344\n"
    "ds_read_b64_tr_b16 v[242:243], v249 offset:58368\n"
    "s_waitcnt vmcnt(12) lgkmcnt(0)\n"
    "s_barrier\n"
    "s_setprio 1\n"
    "v_mfma_f32_16x16x32_bf16 v[64:67], v[212:215], v[180:183], v[64:67]\n"
    "v_mfma_f32_16x16x32_bf16 v[64:67], v[216:219], v[184:187], v[64:67]\n"
    "v_mfma_f32_16x16x32_bf16 v[60:63], v[220:223], v[180:183], v[60:63]\n"
    "v_mfma_f32_16x16x32_bf16 v[60:63], v[224:227], v[184:187], v[60:63]\n"
    "v_mfma_f32_16x16x32_bf16 v[80:83], v[228:231], v[180:183], v[80:83]\n"
    "v_mfma_f32_16x16x32_bf16 v[80:83], v[232:235], v[184:187], v[80:83]\n"
    "v_mfma_f32_16x16x32_bf16 v[88:91], v[236:239], v[180:183], v[88:91]\n"
    "v_mfma_f32_16x16x32_bf16 v[88:91], v[240:243], v[184:187], v[88:91]\n"
    "v_mfma_f32_16x16x32_bf16 v[96:99], v[212:215], v[188:191], v[96:99]\n"
    "v_mfma_f32_16x16x32_bf16 v[96:99], v[216:219], v[192:195], v[96:99]\n"
    "v_mfma_f32_16x16x32_bf16 v[104:107], v[220:223], v[188:191], v[104:107]\n"
    "v_mfma_f32_16x16x32_bf16 v[104:107], v[224:227], v[192:195], v[104:107]\n"
    "v_mfma_f32_16x16x32_bf16 v[76:79], v[228:231], v[188:191], v[76:79]\n"
    "v_mfma_f32_16x16x32_bf16 v[76:79], v[232:235], v[192:195], v[76:79]\n"
    "v_mfma_f32_16x16x32_bf16 v[112:115], v[236:239], v[188:191], v[112:115]\n"
    "v_mfma_f32_16x16x32_bf16 v[112:115], v[240:243], v[192:195], v[112:115]\n"
    "v_mfma_f32_16x16x32_bf16 v[120:123], v[212:215], v[196:199], v[120:123]\n"
    "v_mfma_f32_16x16x32_bf16 v[120:123], v[216:219], v[200:203], v[120:123]\n"
    "v_mfma_f32_16x16x32_bf16 v[84:87], v[220:223], v[196:199], v[84:87]\n"
    "v_mfma_f32_16x16x32_bf16 v[84:87], v[224:227], v[200:203], v[84:87]\n"
    "v_mfma_f32_16x16x32_bf16 v[28:31], v[228:231], v[196:199], v[28:31]\n"
    "v_mfma_f32_16x16x32_bf16 v[28:31], v[232:235], v[200:203], v[28:31]\n"
    "v_mfma_f32_16x16x32_bf16 v[92:95], v[236:239], v[196:199], v[92:95]\n"
    "v_mfma_f32_16x16x32_bf16 v[92:95], v[240:243], v[200:203], v[92:95]\n"
    "v_mfma_f32_16x16x32_bf16 v[100:103], v[212:215], v[204:207], v[100:103]\n"
    "v_mfma_f32_16x16x32_bf16 v[100:103], v[216:219], v[208:211], v[100:103]\n"
    "v_mfma_f32_16x16x32_bf16 v[108:111], v[220:223], v[204:207], v[108:111]\n"
    "v_mfma_f32_16x16x32_bf16 v[108:111], v[224:227], v[208:211], v[108:111]\n"
    "v_mfma_f32_16x16x32_bf16 v[116:119], v[228:231], v[204:207], v[116:119]\n"
    "v_mfma_f32_16x16x32_bf16 v[116:119], v[232:235], v[208:211], v[116:119]\n"
    "v_mfma_f32_16x16x32_bf16 v[52:55], v[236:239], v[204:207], v[52:55]\n"
    "v_mfma_f32_16x16x32_bf16 v[52:55], v[240:243], v[208:211], v[52:55]\n"
    "s_setprio 0\n"
    "s_barrier\n"
    "; phase 96: step 60 slot 0 stage 2\n"
    "ds_read_b128 v[180:183], v244 offset:0\n"
    "ds_read_b128 v[184:187], v245 offset:0\n"
    "ds_read_b128 v[188:191], v244 offset:2048\n"
    "ds_read_b128 v[192:195], v245 offset:2048\n"
    "s_cmp_lg_u32 s52, 6\n"
    "s_cbranch_scc1 .La131_%=\n"
    "buffer_store_dwordx4 v[36:39], v255, s[76:79], s81 offen nt\n"
    ".La131_%=:\n"
    "ds_read_b128 v[196:199], v244 offset:8192\n"
    "ds_read_b128 v[200:203], v245 offset:8192\n"
    "ds_read_b128 v[204:207], v244 offset:10240\n"
    "ds_read_b128 v[208:211], v245 offset:10240\n"
    "s_add_u32 m0, s51, 0xc000\n"
    "s_xor_b32 s82, s67, 0x3d00000\n"
    "buffer_load_dwordx4 v251, s[40:43], s82 offen  lds\n"
    "ds_read_b64_tr_b16 v[212:213], v246 offset:0\n"
    "ds_read_b64_tr_b16 v[214:215], v246 offset:1024\n"
    "ds_read_b64_tr_b16 v[216:217], v246 offset:8192\n"
    "ds_read_b64_tr_b16 v[218:219], v246 offset:9216\n"
    "s_add_u32 m0, s51, 0xe000\n"
    "s_xor_b32 s82, s67, 0x3d80000\n"
    "buffer_load_dwordx4 v251, s[40:43], s82 offen  lds\n"
    "ds_read_b64_tr_b16 v[220:221], v247 offset:0\n"
    "ds_read_b64_tr_b16 v[222:223], v247 offset:1024\n"
    "ds_read_b64_tr_b16 v[224:225], v247 offset:8192\n"
    "ds_read_b64_tr_b16 v[226:227], v247 offset:9216\n"
    "ds_read_b64_tr_b16 v[228:229], v248 offset:0\n"
    "ds_read_b64_tr_b16 v[230:231], v248 offset:1024\n"
    "ds_read_b64_tr_b16 v[232:233], v248 offset:8192\n"
    "ds_read_b64_tr_b16 v[234:235], v248 offset:9216\n"
    "ds_read_b64_tr_b16 v[236:237], v249 offset:0\n"
    "ds_read_b64_tr_b16 v[238:239], v249 offset:1024\n"
    "ds_read_b64_tr_b16 v[240:241], v249 offset:8192\n"
    "ds_read_b64_tr_b16 v[242:243], v249 offset:9216\n"
    "s_waitcnt vmcnt(8) lgkmcnt(0)\n"
    "s_barrier\n"
    "s_setprio 1\n"
    "v_mfma_f32_16x16x32_bf16 v[140:143], v[212:215], v[180:183], v[140:143]\n"
    "v_mfma_f32_16x16x32_bf16 v[140:143], v[216:219], v[184:187], v[140:143]\n"
    "v_mfma_f32_16x16x32_bf16 v[144:147], v[220:223], v[180:183], v[144:147]\n"
    "v_mfma_f32_16x16x32_bf16 v[144:147], v[224:227], v[184:187], v[144:147]\n"
    "v_mfma_f32_16x16x32_bf16 v[148:151], v[228:231], v[180:183], v[148:151]\n"
    "v_mfma_f32_16x16x32_bf16 v[148:151], v[232:235], v[184:187], v[148:151]\n"
    "v_mfma_f32_16x16x32_bf16 v[152:155], v[236:239], v[180:183], v[152:155]\n"
    "v_mfma_f32_16x16x32_bf16 v[152:155], v[240:243], v[184:187], v[152:155]\n"
    "v_mfma_f32_16x16x32_bf16 v[156:159], v[212:215], v[188:191], v[156:159]\n"
    "v_mfma_f32_16x16x32_bf16 v[156:159], v[216:219], v[192:195], v[156:159]\n"
    "v_mfma_f32_16x16x32_bf16 v[160:163], v[220:223], v[188:191], v[160:163]\n"
    "v_mfma_f32_16x16x32_bf16 v[160:163], v[224:227], v[192:195], v[160:163]\n"
    "v_mfma_f32_16x16x32_bf16 v[164:167], v[228:231], v[188:191], v[164:167]\n"
    "v_mfma_f32_16x16x32_bf16 v[164:167], v[232:235], v[192:195], v[164:167]\n"
    "v_mfma_f32_16x16x32_bf16 v[168:171], v[236:239], v[188:191], v[168:171]\n"
    "v_mfma_f32_16x16x32_bf16 v[168:171], v[240:243], v[192:195], v[168:171]\n"
    "v_mfma_f32_16x16x32_bf16 v[172:175], v[212:215], v[196:199], v[172:175]\n"
    "v_mfma_f32_16x16x32_bf16 v[172:175], v[216:219], v[200:203], v[172:175]\n"
    "v_mfma_f32_16x16x32_bf16 v[176:179], v[220:223], v[196:199], v[176:179]\n"
    "v_mfma_f32_16x16x32_bf16 v[176:179], v[224:227], v[200:203], v[176:179]\n"
    "v_mfma_f32_16x16x32_bf16 v[16:19], v[228:231], v[196:199], v[16:19]\n"
    "v_mfma_f32_16x16x32_bf16 v[16:19], v[232:235], v[200:203], v[16:19]\n"
    "v_mfma_f32_16x16x32_bf16 v[24:27], v[236:239], v[196:199], v[24:27]\n"
    "v_mfma_f32_16x16x32_bf16 v[24:27], v[240:243], v[200:203], v[24:27]\n"
    "v_mfma_f32_16x16x32_bf16 v[32:35], v[212:215], v[204:207], v[32:35]\n"
    "v_mfma_f32_16x16x32_bf16 v[32:35], v[216:219], v[208:211], v[32:35]\n"
    "v_mfma_f32_16x16x32_bf16 v[40:43], v[220:223], v[204:207], v[40:43]\n"
    "v_mfma_f32_16x16x32_bf16 v[40:43], v[224:227], v[208:211], v[40:43]\n"
    "v_mfma_f32_16x16x32_bf16 v[12:15], v[228:231], v[204:207], v[12:15]\n"
    "v_mfma_f32_16x16x32_bf16 v[12:15], v[232:235], v[208:211], v[12:15]\n"
    "v_mfma_f32_16x16x32_bf16 v[48:51], v[236:239], v[204:207], v[48:51]\n"
    "v_mfma_f32_16x16x32_bf16 v[48:51], v[240:243], v[208:211], v[48:51]\n"
    "s_setprio 0\n"
    "s_barrier\n"
    "; phase 97: step 60 slot 1 stage 3\n"
    "ds_read_b64_tr_b16 v[212:213], v246 offset:16384\n"
    "ds_read_b64_tr_b16 v[214:215], v246 offset:17408\n"
    "s_cmp_lg_u32 s52, 7\n"
    "s_cbranch_scc1 .La132_%=\n"
    "buffer_store_dwordx4 v[36:39], v255, s[76:79], s81 offen nt\n"
    ".La132_%=:\n"
    "ds_read_b64_tr_b16 v[216:217], v246 offset:24576\n"
    "ds_read_b64_tr_b16 v[218:219], v246 offset:25600\n"
    "s_add_u32 m0, s51, 0x0\n"
    "s_xor_b32 s82, s66, 0x3e00000\n"
    "buffer_load_dwordx4 v251, s[40:43], s82 offen  lds\n"
    "ds_read_b64_tr_b16 v[220:221], v247 offset:16384\n"
    "ds_read_b64_tr_b16 v[222:223], v247 offset:17408\n"
    "s_add_u32 m0, s51, 0x2000\n"
    "s_xor_b32 s82, s66, 0x3e80000\n"
    "buffer_load_dwordx4 v251, s[40:43], s82 offen  lds\n"
    "ds_read_b64_tr_b16 v[224:225], v247 offset:24576\n"
    "ds_read_b64_tr_b16 v[226:227], v247 offset:25600\n"
    "s_add_u32 m0, s50, 0x0\n"
    "s_xor_b32 s82, s48, 0x201f80\n"
    "buffer_load_dwordx4 v250, s[36:39], s82 offen  lds\n"
    "ds_read_b64_tr_b16 v[228:229], v248 offset:16384\n"
    "ds_read_b64_tr_b16 v[230:231], v248 offset:17408\n"
    "s_add_u32 m0, s50, 0x1000\n"
    "s_xor_b32 s82, s48, 0x281f80\n"
    "buffer_load_dwordx4 v250, s[36:39], s82 offen  lds\n"
    "ds_read_b64_tr_b16 v[232:233], v248 offset:24576\n"
    "ds_read_b64_tr_b16 v[234:235], v248 offset:25600\n"
    "s_add_u32 m0, s50, 0x2000\n"
    "s_xor_b32 s82, s48, 0x1f80\n"
    "buffer_load_dwordx4 v250, s[36:39], s82 offen  lds\n"
    "ds_read_b64_tr_b16 v[236:237], v249 offset:16384\n"
    "ds_read_b64_tr_b16 v[238:239], v249 offset:17408\n"
    "s_add_u32 m0, s50, 0x3000\n"
    "s_xor_b32 s82, s48, 0x81f80\n"
    "buffer_load_dwordx4 v250, s[36:39], s82 offen  lds\n"
    "ds_read_b64_tr_b16 v[240:241], v249 offset:24576\n"
    "ds_read_b64_tr_b16 v[242:243], v249 offset:25600\n"
    "s_waitcnt vmcnt(12) lgkmcnt(0)\n"
    "s_barrier\n"
    "s_setprio 1\n"
    "v_mfma_f32_16x16x32_bf16 v[64:67], v[212:215], v[180:183], v[64:67]\n"
    "v_mfma_f32_16x16x32_bf16 v[64:67], v[216:219], v[184:187], v[64:67]\n"
    "v_mfma_f32_16x16x32_bf16 v[60:63], v[220:223], v[180:183], v[60:63]\n"
    "v_mfma_f32_16x16x32_bf16 v[60:63], v[224:227], v[184:187], v[60:63]\n"
    "v_mfma_f32_16x16x32_bf16 v[80:83], v[228:231], v[180:183], v[80:83]\n"
    "v_mfma_f32_16x16x32_bf16 v[80:83], v[232:235], v[184:187], v[80:83]\n"
    "v_mfma_f32_16x16x32_bf16 v[88:91], v[236:239], v[180:183], v[88:91]\n"
    "v_mfma_f32_16x16x32_bf16 v[88:91], v[240:243], v[184:187], v[88:91]\n"
    "v_mfma_f32_16x16x32_bf16 v[96:99], v[212:215], v[188:191], v[96:99]\n"
    "v_mfma_f32_16x16x32_bf16 v[96:99], v[216:219], v[192:195], v[96:99]\n"
    "v_mfma_f32_16x16x32_bf16 v[104:107], v[220:223], v[188:191], v[104:107]\n"
    "v_mfma_f32_16x16x32_bf16 v[104:107], v[224:227], v[192:195], v[104:107]\n"
    "v_mfma_f32_16x16x32_bf16 v[76:79], v[228:231], v[188:191], v[76:79]\n"
    "v_mfma_f32_16x16x32_bf16 v[76:79], v[232:235], v[192:195], v[76:79]\n"
    "v_mfma_f32_16x16x32_bf16 v[112:115], v[236:239], v[188:191], v[112:115]\n"
    "v_mfma_f32_16x16x32_bf16 v[112:115], v[240:243], v[192:195], v[112:115]\n"
    "v_mfma_f32_16x16x32_bf16 v[120:123], v[212:215], v[196:199], v[120:123]\n"
    "v_mfma_f32_16x16x32_bf16 v[120:123], v[216:219], v[200:203], v[120:123]\n"
    "v_mfma_f32_16x16x32_bf16 v[84:87], v[220:223], v[196:199], v[84:87]\n"
    "v_mfma_f32_16x16x32_bf16 v[84:87], v[224:227], v[200:203], v[84:87]\n"
    "v_mfma_f32_16x16x32_bf16 v[28:31], v[228:231], v[196:199], v[28:31]\n"
    "v_mfma_f32_16x16x32_bf16 v[28:31], v[232:235], v[200:203], v[28:31]\n"
    "v_mfma_f32_16x16x32_bf16 v[92:95], v[236:239], v[196:199], v[92:95]\n"
    "v_mfma_f32_16x16x32_bf16 v[92:95], v[240:243], v[200:203], v[92:95]\n"
    "v_mfma_f32_16x16x32_bf16 v[100:103], v[212:215], v[204:207], v[100:103]\n"
    "v_mfma_f32_16x16x32_bf16 v[100:103], v[216:219], v[208:211], v[100:103]\n"
    "v_mfma_f32_16x16x32_bf16 v[108:111], v[220:223], v[204:207], v[108:111]\n"
    "v_mfma_f32_16x16x32_bf16 v[108:111], v[224:227], v[208:211], v[108:111]\n"
    "v_mfma_f32_16x16x32_bf16 v[116:119], v[228:231], v[204:207], v[116:119]\n"
    "v_mfma_f32_16x16x32_bf16 v[116:119], v[232:235], v[208:211], v[116:119]\n"
    "v_mfma_f32_16x16x32_bf16 v[52:55], v[236:239], v[204:207], v[52:55]\n"
    "v_mfma_f32_16x16x32_bf16 v[52:55], v[240:243], v[208:211], v[52:55]\n"
    "s_setprio 0\n"
    "s_barrier\n"
    "; phase 98: step 61 slot 0 stage 2\n"
    "ds_read_b128 v[180:183], v244 offset:16384\n"
    "ds_read_b128 v[184:187], v245 offset:16384\n"
    "ds_read_b128 v[188:191], v244 offset:18432\n"
    "ds_read_b128 v[192:195], v245 offset:18432\n"
    "s_cmp_lg_u32 s52, 0\n"
    "s_cbranch_scc1 .La133_%=\n"
    "s_xor_b32 s82, s81, 0x4000\n"
    "buffer_store_dwordx4 v[44:47], v255, s[76:79], s82 offen nt\n"
    ".La133_%=:\n"
    "ds_read_b128 v[196:199], v244 offset:24576\n"
    "ds_read_b128 v[200:203], v245 offset:24576\n"
    "ds_read_b128 v[204:207], v244 offset:26624\n"
    "ds_read_b128 v[208:211], v245 offset:26624\n"
    "s_add_u32 m0, s51, 0x4000\n"
    "s_xor_b32 s82, s67, 0x3e00000\n"
    "buffer_load_dwordx4 v251, s[40:43], s82 offen  lds\n"
    "ds_read_b64_tr_b16 v[212:213], v246 offset:32768\n"
    "ds_read_b64_tr_b16 v[214:215], v246 offset:33792\n"
    "ds_read_b64_tr_b16 v[216:217], v246 offset:40960\n"
    "ds_read_b64_tr_b16 v[218:219], v246 offset:41984\n"
    "s_add_u32 m0, s51, 0x6000\n"
    "s_xor_b32 s82, s67, 0x3e80000\n"
    "buffer_load_dwordx4 v251, s[40:43], s82 offen  lds\n"
    "ds_read_b64_tr_b16 v[220:221], v247 offset:32768\n"
    "ds_read_b64_tr_b16 v[222:223], v247 offset:33792\n"
    "ds_read_b64_tr_b16 v[224:225], v247 offset:40960\n"
    "ds_read_b64_tr_b16 v[226:227], v247 offset:41984\n"
    "ds_read_b64_tr_b16 v[228:229], v248 offset:32768\n"
    "ds_read_b64_tr_b16 v[230:231], v248 offset:33792\n"
    "ds_read_b64_tr_b16 v[232:233], v248 offset:40960\n"
    "ds_read_b64_tr_b16 v[234:235], v248 offset:41984\n"
    "ds_read_b64_tr_b16 v[236:237], v249 offset:32768\n"
    "ds_read_b64_tr_b16 v[238:239], v249 offset:33792\n"
    "ds_read_b64_tr_b16 v[240:241], v249 offset:40960\n"
    "ds_read_b64_tr_b16 v[242:243], v249 offset:41984\n"
    "s_waitcnt vmcnt(8) lgkmcnt(0)\n"
    "s_barrier\n"
    "s_setprio 1\n"
    "v_mfma_f32_16x16x32_bf16 v[140:143], v[212:215], v[180:183], v[140:143]\n"
    "v_mfma_f32_16x16x32_bf16 v[140:143], v[216:219], v[184:187], v[140:143]\n"
    "v_mfma_f32_16x16x32_bf16 v[144:147], v[220:223], v[180:183], v[144:147]\n"
    "v_mfma_f32_16x16x32_bf16 v[144:147], v[224:227], v[184:187], v[144:147]\n"
    "v_mfma_f32_16x16x32_bf16 v[148:151], v[228:231], v[180:183], v[148:151]\n"
    "v_mfma_f32_16x16x32_bf16 v[148:151], v[232:235], v[184:187], v[148:151]\n"
    "v_mfma_f32_16x16x32_bf16 v[152:155], v[236:239], v[180:183], v[152:155]\n"
    "v_mfma_f32_16x16x32_bf16 v[152:155], v[240:243], v[184:187], v[152:155]\n"
    "v_mfma_f32_16x16x32_bf16 v[156:159], v[212:215], v[188:191], v[156:159]\n"
    "v_mfma_f32_16x16x32_bf16 v[156:159], v[216:219], v[192:195], v[156:159]\n"
    "v_mfma_f32_16x16x32_bf16 v[160:163], v[220:223], v[188:191], v[160:163]\n"
    "v_mfma_f32_16x16x32_bf16 v[160:163], v[224:227], v[192:195], v[160:163]\n"
    "v_mfma_f32_16x16x32_bf16 v[164:167], v[228:231], v[188:191], v[164:167]\n"
    "v_mfma_f32_16x16x32_bf16 v[164:167], v[232:235], v[192:195], v[164:167]\n"
    "v_mfma_f32_16x16x32_bf16 v[168:171], v[236:239], v[188:191], v[168:171]\n"
    "v_mfma_f32_16x16x32_bf16 v[168:171], v[240:243], v[192:195], v[168:171]\n"
    "v_mfma_f32_16x16x32_bf16 v[172:175], v[212:215], v[196:199], v[172:175]\n"
    "v_mfma_f32_16x16x32_bf16 v[172:175], v[216:219], v[200:203], v[172:175]\n"
    "v_mfma_f32_16x16x32_bf16 v[176:179], v[220:223], v[196:199], v[176:179]\n"
    "v_mfma_f32_16x16x32_bf16 v[176:179], v[224:227], v[200:203], v[176:179]\n"
    "v_mfma_f32_16x16x32_bf16 v[16:19], v[228:231], v[196:199], v[16:19]\n"
    "v_mfma_f32_16x16x32_bf16 v[16:19], v[232:235], v[200:203], v[16:19]\n"
    "v_mfma_f32_16x16x32_bf16 v[24:27], v[236:239], v[196:199], v[24:27]\n"
    "v_mfma_f32_16x16x32_bf16 v[24:27], v[240:243], v[200:203], v[24:27]\n"
    "v_mfma_f32_16x16x32_bf16 v[32:35], v[212:215], v[204:207], v[32:35]\n"
    "v_mfma_f32_16x16x32_bf16 v[32:35], v[216:219], v[208:211], v[32:35]\n"
    "v_mfma_f32_16x16x32_bf16 v[40:43], v[220:223], v[204:207], v[40:43]\n"
    "v_mfma_f32_16x16x32_bf16 v[40:43], v[224:227], v[208:211], v[40:43]\n"
    "v_mfma_f32_16x16x32_bf16 v[12:15], v[228:231], v[204:207], v[12:15]\n"
    "v_mfma_f32_16x16x32_bf16 v[12:15], v[232:235], v[208:211], v[12:15]\n"
    "v_mfma_f32_16x16x32_bf16 v[48:51], v[236:239], v[204:207], v[48:51]\n"
    "v_mfma_f32_16x16x32_bf16 v[48:51], v[240:243], v[208:211], v[48:51]\n"
    "s_setprio 0\n"
    "s_barrier\n"
    "; phase 99: step 61 slot 1 stage 3\n"
    "ds_read_b64_tr_b16 v[212:213], v246 offset:49152\n"
    "ds_read_b64_tr_b16 v[214:215], v246 offset:50176\n"
    "s_cmp_lg_u32 s52, 1\n"
    "s_cbranch_scc1 .La134_%=\n"
    "s_xor_b32 s82, s81, 0x4000\n"
    "buffer_store_dwordx4 v[44:47], v255, s[76:79], s82 offen nt\n"
    ".La134_%=:\n"
    "ds_read_b64_tr_b16 v[216:217], v246 offset:57344\n"
    "ds_read_b64_tr_b16 v[218:219], v246 offset:58368\n"
    "s_add_u32 m0, s51, 0x8000\n"
    "s_xor_b32 s82, s66, 0x3f00000\n"
    "buffer_load_dwordx4 v251, s[40:43], s82 offen  lds\n"
    "ds_read_b64_tr_b16 v[220:221], v247 offset:49152\n"
    "ds_read_b64_tr_b16 v[222:223], v247 offset:50176\n"
    "s_add_u32 m0, s51, 0xa000\n"
    "s_xor_b32 s82, s66, 0x3f80000\n"
    "buffer_load_dwordx4 v251, s[40:43], s82 offen  lds\n"
    "ds_read_b64_tr_b16 v[224:225], v247 offset:57344\n"
    "ds_read_b64_tr_b16 v[226:227], v247 offset:58368\n"
    "s_add_u32 m0, s50, 0x4000\n"
    "s_xor_b32 s82, s48, 0x202000\n"
    "buffer_load_dwordx4 v250, s[36:39], s82 offen  lds\n"
    "ds_read_b64_tr_b16 v[228:229], v248 offset:49152\n"
    "ds_read_b64_tr_b16 v[230:231], v248 offset:50176\n"
    "s_add_u32 m0, s50, 0x5000\n"
    "s_xor_b32 s82, s48, 0x282000\n"
    "buffer_load_dwordx4 v250, s[36:39], s82 offen  lds\n"
    "ds_read_b64_tr_b16 v[232:233], v248 offset:57344\n"
    "ds_read_b64_tr_b16 v[234:235], v248 offset:58368\n"
    "s_add_u32 m0, s50, 0x6000\n"
    "s_xor_b32 s82, s48, 0x2000\n"
    "buffer_load_dwordx4 v250, s[36:39], s82 offen  lds\n"
    "ds_read_b64_tr_b16 v[236:237], v249 offset:49152\n"
    "ds_read_b64_tr_b16 v[238:239], v249 offset:50176\n"
    "s_add_u32 m0, s50, 0x7000\n"
    "s_xor_b32 s82, s48, 0x82000\n"
    "buffer_load_dwordx4 v250, s[36:39], s82 offen  lds\n"
    "ds_read_b64_tr_b16 v[240:241], v249 offset:57344\n"
    "ds_read_b64_tr_b16 v[242:243], v249 offset:58368\n"
    "s_waitcnt vmcnt(12) lgkmcnt(0)\n"
    "s_barrier\n"
    "s_setprio 1\n"
    "v_mfma_f32_16x16x32_bf16 v[64:67], v[212:215], v[180:183], v[64:67]\n"
    "v_mfma_f32_16x16x32_bf16 v[64:67], v[216:219], v[184:187], v[64:67]\n"
    "v_mfma_f32_16x16x32_bf16 v[60:63], v[220:223], v[180:183], v[60:63]\n"
    "v_mfma_f32_16x16x32_bf16 v[60:63], v[224:227], v[184:187], v[60:63]\n"
    "v_mfma_f32_16x16x32_bf16 v[80:83], v[228:231], v[180:183], v[80:83]\n"
    "v_mfma_f32_16x16x32_bf16 v[80:83], v[232:235], v[184:187], v[80:83]\n"
    "v_mfma_f32_16x16x32_bf16 v[88:91], v[236:239], v[180:183], v[88:91]\n"
    "v_mfma_f32_16x16x32_bf16 v[88:91], v[240:243], v[184:187], v[88:91]\n"
    "v_mfma_f32_16x16x32_bf16 v[96:99], v[212:215], v[188:191], v[96:99]\n"
    "v_mfma_f32_16x16x32_bf16 v[96:99], v[216:219], v[192:195], v[96:99]\n"
    "v_mfma_f32_16x16x32_bf16 v[104:107], v[220:223], v[188:191], v[104:107]\n"
    "v_mfma_f32_16x16x32_bf16 v[104:107], v[224:227], v[192:195], v[104:107]\n"
    "v_mfma_f32_16x16x32_bf16 v[76:79], v[228:231], v[188:191], v[76:79]\n"
    "v_mfma_f32_16x16x32_bf16 v[76:79], v[232:235], v[192:195], v[76:79]\n"
    "v_mfma_f32_16x16x32_bf16 v[112:115], v[236:239], v[188:191], v[112:115]\n"
    "v_mfma_f32_16x16x32_bf16 v[112:115], v[240:243], v[192:195], v[112:115]\n"
    "v_mfma_f32_16x16x32_bf16 v[120:123], v[212:215], v[196:199], v[120:123]\n"
    "v_mfma_f32_16x16x32_bf16 v[120:123], v[216:219], v[200:203], v[120:123]\n"
    "v_mfma_f32_16x16x32_bf16 v[84:87], v[220:223], v[196:199], v[84:87]\n"
    "v_mfma_f32_16x16x32_bf16 v[84:87], v[224:227], v[200:203], v[84:87]\n"
    "v_mfma_f32_16x16x32_bf16 v[28:31], v[228:231], v[196:199], v[28:31]\n"
    "v_mfma_f32_16x16x32_bf16 v[28:31], v[232:235], v[200:203], v[28:31]\n"
    "v_mfma_f32_16x16x32_bf16 v[92:95], v[236:239], v[196:199], v[92:95]\n"
    "v_mfma_f32_16x16x32_bf16 v[92:95], v[240:243], v[200:203], v[92:95]\n"
    "v_mfma_f32_16x16x32_bf16 v[100:103], v[212:215], v[204:207], v[100:103]\n"
    "v_mfma_f32_16x16x32_bf16 v[100:103], v[216:219], v[208:211], v[100:103]\n"
    "v_mfma_f32_16x16x32_bf16 v[108:111], v[220:223], v[204:207], v[108:111]\n"
    "v_mfma_f32_16x16x32_bf16 v[108:111], v[224:227], v[208:211], v[108:111]\n"
    "v_mfma_f32_16x16x32_bf16 v[116:119], v[228:231], v[204:207], v[116:119]\n"
    "v_mfma_f32_16x16x32_bf16 v[116:119], v[232:235], v[208:211], v[116:119]\n"
    "v_mfma_f32_16x16x32_bf16 v[52:55], v[236:239], v[204:207], v[52:55]\n"
    "v_mfma_f32_16x16x32_bf16 v[52:55], v[240:243], v[208:211], v[52:55]\n"
    "s_setprio 0\n"
    "s_barrier\n"
    "; phase 100: step 62 slot 0 stage 2\n"
    "ds_read_b128 v[180:183], v244 offset:32768\n"
    "ds_read_b128 v[184:187], v245 offset:32768\n"
    "ds_read_b128 v[188:191], v244 offset:34816\n"
    "ds_read_b128 v[192:195], v245 offset:34816\n"
    "s_cmp_lg_u32 s52, 2\n"
    "s_cbranch_scc1 .La135_%=\n"
    "s_xor_b32 s82, s81, 0x4000\n"
    "buffer_store_dwordx4 v[44:47], v255, s[76:79], s82 offen nt\n"
    ".La135_%=:\n"
    "ds_read_b128 v[196:199], v244 offset:40960\n"
    "ds_read_b128 v[200:203], v245 offset:40960\n"
    "ds_read_b128 v[204:207], v244 offset:43008\n"
    "ds_read_b128 v[208:211], v245 offset:43008\n"
    "s_add_u32 m0, s51, 0xc000\n"
    "s_xor_b32 s82, s67, 0x3f00000\n"
    "buffer_load_dwordx4 v251, s[40:43], s82 offen  lds\n"
    "ds_read_b64_tr_b16 v[212:213], v246 offset:0\n"
    "ds_read_b64_tr_b16 v[214:215], v246 offset:1024\n"
    "ds_read_b64_tr_b16 v[216:217], v246 offset:8192\n"
    "ds_read_b64_tr_b16 v[218:219], v246 offset:9216\n"
    "s_add_u32 m0, s51, 0xe000\n"
    "s_xor_b32 s82, s67, 0x3f80000\n"
    "buffer_load_dwordx4 v251, s[40:43], s82 offen  lds\n"
    "ds_read_b64_tr_b16 v[220:221], v247 offset:0\n"
    "ds_read_b64_tr_b16 v[222:223], v247 offset:1024\n"
    "ds_read_b64_tr_b16 v[224:225], v247 offset:8192\n"
    "ds_read_b64_tr_b16 v[226:227], v247 offset:9216\n"
    "ds_read_b64_tr_b16 v[228:229], v248 offset:0\n"
    "ds_read_b64_tr_b16 v[230:231], v248 offset:1024\n"
    "ds_read_b64_tr_b16 v[232:233], v248 offset:8192\n"
    "ds_read_b64_tr_b16 v[234:235], v248 offset:9216\n"
    "ds_read_b64_tr_b16 v[236:237], v249 offset:0\n"
    "ds_read_b64_tr_b16 v[238:239], v249 offset:1024\n"
    "ds_read_b64_tr_b16 v[240:241], v249 offset:8192\n"
    "ds_read_b64_tr_b16 v[242:243], v249 offset:9216\n"
    "s_waitcnt vmcnt(8) lgkmcnt(0)\n"
    "s_barrier\n"
    "s_setprio 1\n"
    "v_mfma_f32_16x16x32_bf16 v[140:143], v[212:215], v[180:183], v[140:143]\n"
    "v_mfma_f32_16x16x32_bf16 v[140:143], v[216:219], v[184:187], v[140:143]\n"
    "v_mfma_f32_16x16x32_bf16 v[144:147], v[220:223], v[180:183], v[144:147]\n"
    "v_mfma_f32_16x16x32_bf16 v[144:147], v[224:227], v[184:187], v[144:147]\n"
    "v_mfma_f32_16x16x32_bf16 v[148:151], v[228:231], v[180:183], v[148:151]\n"
    "v_mfma_f32_16x16x32_bf16 v[148:151], v[232:235], v[184:187], v[148:151]\n"
    "v_mfma_f32_16x16x32_bf16 v[152:155], v[236:239], v[180:183], v[152:155]\n"
    "v_mfma_f32_16x16x32_bf16 v[152:155], v[240:243], v[184:187], v[152:155]\n"
    "v_mfma_f32_16x16x32_bf16 v[156:159], v[212:215], v[188:191], v[156:159]\n"
    "v_mfma_f32_16x16x32_bf16 v[156:159], v[216:219], v[192:195], v[156:159]\n"
    "v_mfma_f32_16x16x32_bf16 v[160:163], v[220:223], v[188:191], v[160:163]\n"
    "v_mfma_f32_16x16x32_bf16 v[160:163], v[224:227], v[192:195], v[160:163]\n"
    "v_mfma_f32_16x16x32_bf16 v[164:167], v[228:231], v[188:191], v[164:167]\n"
    "v_mfma_f32_16x16x32_bf16 v[164:167], v[232:235], v[192:195], v[164:167]\n"
    "v_mfma_f32_16x16x32_bf16 v[168:171], v[236:239], v[188:191], v[168:171]\n"
    "v_mfma_f32_16x16x32_bf16 v[168:171], v[240:243], v[192:195], v[168:171]\n"
    "v_mfma_f32_16x16x32_bf16 v[172:175], v[212:215], v[196:199], v[172:175]\n"
    "v_mfma_f32_16x16x32_bf16 v[172:175], v[216:219], v[200:203], v[172:175]\n"
    "v_mfma_f32_16x16x32_bf16 v[176:179], v[220:223], v[196:199], v[176:179]\n"
    "v_mfma_f32_16x16x32_bf16 v[176:179], v[224:227], v[200:203], v[176:179]\n"
    "v_mfma_f32_16x16x32_bf16 v[16:19], v[228:231], v[196:199], v[16:19]\n"
    "v_mfma_f32_16x16x32_bf16 v[16:19], v[232:235], v[200:203], v[16:19]\n"
    "v_mfma_f32_16x16x32_bf16 v[24:27], v[236:239], v[196:199], v[24:27]\n"
    "v_mfma_f32_16x16x32_bf16 v[24:27], v[240:243], v[200:203], v[24:27]\n"
    "v_mfma_f32_16x16x32_bf16 v[32:35], v[212:215], v[204:207], v[32:35]\n"
    "v_mfma_f32_16x16x32_bf16 v[32:35], v[216:219], v[208:211], v[32:35]\n"
    "v_mfma_f32_16x16x32_bf16 v[40:43], v[220:223], v[204:207], v[40:43]\n"
    "v_mfma_f32_16x16x32_bf16 v[40:43], v[224:227], v[208:211], v[40:43]\n"
    "v_mfma_f32_16x16x32_bf16 v[12:15], v[228:231], v[204:207], v[12:15]\n"
    "v_mfma_f32_16x16x32_bf16 v[12:15], v[232:235], v[208:211], v[12:15]\n"
    "v_mfma_f32_16x16x32_bf16 v[48:51], v[236:239], v[204:207], v[48:51]\n"
    "v_mfma_f32_16x16x32_bf16 v[48:51], v[240:243], v[208:211], v[48:51]\n"
    "s_setprio 0\n"
    "s_barrier\n"
    "; phase 101: step 62 slot 1 stage 3\n"
    "ds_read_b64_tr_b16 v[212:213], v246 offset:16384\n"
    "ds_read_b64_tr_b16 v[214:215], v246 offset:17408\n"
    "s_cmp_lg_u32 s52, 3\n"
    "s_cbranch_scc1 .La136_%=\n"
    "s_xor_b32 s82, s81, 0x4000\n"
    "buffer_store_dwordx4 v[44:47], v255, s[76:79], s82 offen nt\n"
    ".La136_%=:\n"
    "ds_read_b64_tr_b16 v[216:217], v246 offset:24576\n"
    "ds_read_b64_tr_b16 v[218:219], v246 offset:25600\n"
    "s_add_u32 m0, s51, 0x0\n"
    "s_xor_b32 s82, s67, 0x4000000\n"
    "buffer_load_dwordx4 v251, s[40:43], s82 offen  lds\n"
    "ds_read_b64_tr_b16 v[220:221], v247 offset:16384\n"
    "ds_read_b64_tr_b16 v[222:223], v247 offset:17408\n"
    "s_add_u32 m0, s51, 0x2000\n"
    "s_xor_b32 s82, s67, 0x4080000\n"
    "buffer_load_dwordx4 v251, s[40:43], s82 offen  lds\n"
    "ds_read_b64_tr_b16 v[224:225], v247 offset:24576\n"
    "ds_read_b64_tr_b16 v[226:227], v247 offset:25600\n"
    "s_add_u32 m0, s50, 0x8000\n"
    "s_xor_b32 s82, s48, 0x202080\n"
    "buffer_load_dwordx4 v250, s[36:39], s82 offen  lds\n"
    "ds_read_b64_tr_b16 v[228:229], v248 offset:16384\n"
    "ds_read_b64_tr_b16 v[230:231], v248 offset:17408\n"
    "s_add_u32 m0, s50, 0x9000\n"
    "s_xor_b32 s82, s48, 0x282080\n"
    "buffer_load_dwordx4 v250, s[36:39], s82 offen  lds\n"
    "ds_read_b64_tr_b16 v[232:233], v248 offset:24576\n"
    "ds_read_b64_tr_b16 v[234:235], v248 offset:25600\n"
    "s_add_u32 m0, s50, 0xa000\n"
    "s_xor_b32 s82, s48, 0x2080\n"
    "buffer_load_dwordx4 v250, s[36:39], s82 offen  lds\n"
    "ds_read_b64_tr_b16 v[236:237], v249 offset:16384\n"
    "ds_read_b64_tr_b16 v[238:239], v249 offset:17408\n"
    "s_add_u32 m0, s50, 0xb000\n"
    "s_xor_b32 s82, s48, 0x82080\n"
    "buffer_load_dwordx4 v250, s[36:39], s82 offen  lds\n"
    "ds_read_b64_tr_b16 v[240:241], v249 offset:24576\n"
    "ds_read_b64_tr_b16 v[242:243], v249 offset:25600\n"
    "s_waitcnt vmcnt(12) lgkmcnt(0)\n"
    "s_barrier\n"
    "s_setprio 1\n"
    "v_mfma_f32_16x16x32_bf16 v[64:67], v[212:215], v[180:183], v[64:67]\n"
    "v_mfma_f32_16x16x32_bf16 v[64:67], v[216:219], v[184:187], v[64:67]\n"
    "v_mfma_f32_16x16x32_bf16 v[60:63], v[220:223], v[180:183], v[60:63]\n"
    "v_mfma_f32_16x16x32_bf16 v[60:63], v[224:227], v[184:187], v[60:63]\n"
    "v_mfma_f32_16x16x32_bf16 v[80:83], v[228:231], v[180:183], v[80:83]\n"
    "v_mfma_f32_16x16x32_bf16 v[80:83], v[232:235], v[184:187], v[80:83]\n"
    "v_mfma_f32_16x16x32_bf16 v[88:91], v[236:239], v[180:183], v[88:91]\n"
    "v_mfma_f32_16x16x32_bf16 v[88:91], v[240:243], v[184:187], v[88:91]\n"
    "v_mfma_f32_16x16x32_bf16 v[96:99], v[212:215], v[188:191], v[96:99]\n"
    "v_mfma_f32_16x16x32_bf16 v[96:99], v[216:219], v[192:195], v[96:99]\n"
    "v_mfma_f32_16x16x32_bf16 v[104:107], v[220:223], v[188:191], v[104:107]\n"
    "v_mfma_f32_16x16x32_bf16 v[104:107], v[224:227], v[192:195], v[104:107]\n"
    "v_mfma_f32_16x16x32_bf16 v[76:79], v[228:231], v[188:191], v[76:79]\n"
    "v_mfma_f32_16x16x32_bf16 v[76:79], v[232:235], v[192:195], v[76:79]\n"
    "v_mfma_f32_16x16x32_bf16 v[112:115], v[236:239], v[188:191], v[112:115]\n"
    "v_mfma_f32_16x16x32_bf16 v[112:115], v[240:243], v[192:195], v[112:115]\n"
    "v_mfma_f32_16x16x32_bf16 v[120:123], v[212:215], v[196:199], v[120:123]\n"
    "v_mfma_f32_16x16x32_bf16 v[120:123], v[216:219], v[200:203], v[120:123]\n"
    "v_mfma_f32_16x16x32_bf16 v[84:87], v[220:223], v[196:199], v[84:87]\n"
    "v_mfma_f32_16x16x32_bf16 v[84:87], v[224:227], v[200:203], v[84:87]\n"
    "v_mfma_f32_16x16x32_bf16 v[28:31], v[228:231], v[196:199], v[28:31]\n"
    "v_mfma_f32_16x16x32_bf16 v[28:31], v[232:235], v[200:203], v[28:31]\n"
    "v_mfma_f32_16x16x32_bf16 v[92:95], v[236:239], v[196:199], v[92:95]\n"
    "v_mfma_f32_16x16x32_bf16 v[92:95], v[240:243], v[200:203], v[92:95]\n"
    "v_mfma_f32_16x16x32_bf16 v[100:103], v[212:215], v[204:207], v[100:103]\n"
    "v_mfma_f32_16x16x32_bf16 v[100:103], v[216:219], v[208:211], v[100:103]\n"
    "v_mfma_f32_16x16x32_bf16 v[108:111], v[220:223], v[204:207], v[108:111]\n"
    "v_mfma_f32_16x16x32_bf16 v[108:111], v[224:227], v[208:211], v[108:111]\n"
    "v_mfma_f32_16x16x32_bf16 v[116:119], v[228:231], v[204:207], v[116:119]\n"
    "v_mfma_f32_16x16x32_bf16 v[116:119], v[232:235], v[208:211], v[116:119]\n"
    "v_mfma_f32_16x16x32_bf16 v[52:55], v[236:239], v[204:207], v[52:55]\n"
    "v_mfma_f32_16x16x32_bf16 v[52:55], v[240:243], v[208:211], v[52:55]\n"
    "s_setprio 0\n"
    "s_barrier\n"
    "; phase 102: step 63 slot 0 stage 2\n"
    "ds_read_b128 v[180:183], v244 offset:0\n"
    "ds_read_b128 v[184:187], v245 offset:0\n"
    "ds_read_b128 v[188:191], v244 offset:2048\n"
    "ds_read_b128 v[192:195], v245 offset:2048\n"
    "s_cmp_lg_u32 s52, 4\n"
    "s_cbranch_scc1 .La137_%=\n"
    "s_xor_b32 s82, s81, 0x4000\n"
    "buffer_store_dwordx4 v[44:47], v255, s[76:79], s82 offen nt\n"
    ".La137_%=:\n"
    "ds_read_b128 v[196:199], v244 offset:8192\n"
    "ds_read_b128 v[200:203], v245 offset:8192\n"
    "ds_read_b128 v[204:207], v244 offset:10240\n"
    "ds_read_b128 v[208:211], v245 offset:10240\n"
    "s_add_u32 m0, s51, 0x4000\n"
    "s_xor_b32 s82, s67, 0x4100000\n"
    "buffer_load_dwordx4 v251, s[40:43], s82 offen  lds\n"
    "ds_read_b64_tr_b16 v[212:213], v246 offset:32768\n"
    "ds_read_b64_tr_b16 v[214:215], v246 offset:33792\n"
    "ds_read_b64_tr_b16 v[216:217], v246 offset:40960\n"
    "ds_read_b64_tr_b16 v[218:219], v246 offset:41984\n"
    "s_add_u32 m0, s51, 0x6000\n"
    "s_xor_b32 s82, s67, 0x4180000\n"
    "buffer_load_dwordx4 v251, s[40:43], s82 offen  lds\n"
    "ds_read_b64_tr_b16 v[220:221], v247 offset:32768\n"
    "ds_read_b64_tr_b16 v[222:223], v247 offset:33792\n"
    "ds_read_b64_tr_b16 v[224:225], v247 offset:40960\n"
    "ds_read_b64_tr_b16 v[226:227], v247 offset:41984\n"
    "ds_read_b64_tr_b16 v[228:229], v248 offset:32768\n"
    "ds_read_b64_tr_b16 v[230:231], v248 offset:33792\n"
    "ds_read_b64_tr_b16 v[232:233], v248 offset:40960\n"
    "ds_read_b64_tr_b16 v[234:235], v248 offset:41984\n"
    "ds_read_b64_tr_b16 v[236:237], v249 offset:32768\n"
    "ds_read_b64_tr_b16 v[238:239], v249 offset:33792\n"
    "ds_read_b64_tr_b16 v[240:241], v249 offset:40960\n"
    "ds_read_b64_tr_b16 v[242:243], v249 offset:41984\n"
    "s_waitcnt vmcnt(8) lgkmcnt(0)\n"
    "s_barrier\n"
    "s_setprio 1\n"
    "v_mfma_f32_16x16x32_bf16 v[140:143], v[212:215], v[180:183], v[140:143]\n"
    "v_mfma_f32_16x16x32_bf16 v[140:143], v[216:219], v[184:187], v[140:143]\n"
    "v_mfma_f32_16x16x32_bf16 v[144:147], v[220:223], v[180:183], v[144:147]\n"
    "v_mfma_f32_16x16x32_bf16 v[144:147], v[224:227], v[184:187], v[144:147]\n"
    "v_mfma_f32_16x16x32_bf16 v[148:151], v[228:231], v[180:183], v[148:151]\n"
    "v_mfma_f32_16x16x32_bf16 v[148:151], v[232:235], v[184:187], v[148:151]\n"
    "v_mfma_f32_16x16x32_bf16 v[152:155], v[236:239], v[180:183], v[152:155]\n"
    "v_mfma_f32_16x16x32_bf16 v[152:155], v[240:243], v[184:187], v[152:155]\n"
    "v_mfma_f32_16x16x32_bf16 v[156:159], v[212:215], v[188:191], v[156:159]\n"
    "v_mfma_f32_16x16x32_bf16 v[156:159], v[216:219], v[192:195], v[156:159]\n"
    "v_mfma_f32_16x16x32_bf16 v[160:163], v[220:223], v[188:191], v[160:163]\n"
    "v_mfma_f32_16x16x32_bf16 v[160:163], v[224:227], v[192:195], v[160:163]\n"
    "v_mfma_f32_16x16x32_bf16 v[164:167], v[228:231], v[188:191], v[164:167]\n"
    "v_mfma_f32_16x16x32_bf16 v[164:167], v[232:235], v[192:195], v[164:167]\n"
    "v_mfma_f32_16x16x32_bf16 v[168:171], v[236:239], v[188:191], v[168:171]\n"
    "v_mfma_f32_16x16x32_bf16 v[168:171], v[240:243], v[192:195], v[168:171]\n"
    "v_mfma_f32_16x16x32_bf16 v[172:175], v[212:215], v[196:199], v[172:175]\n"
    "v_mfma_f32_16x16x32_bf16 v[172:175], v[216:219], v[200:203], v[172:175]\n"
    "v_mfma_f32_16x16x32_bf16 v[176:179], v[220:223], v[196:199], v[176:179]\n"
    "v_mfma_f32_16x16x32_bf16 v[176:179], v[224:227], v[200:203], v[176:179]\n"
    "v_mfma_f32_16x16x32_bf16 v[16:19], v[228:231], v[196:199], v[16:19]\n"
    "v_mfma_f32_16x16x32_bf16 v[16:19], v[232:235], v[200:203], v[16:19]\n"
    "v_mfma_f32_16x16x32_bf16 v[24:27], v[236:239], v[196:199], v[24:27]\n"
    "v_mfma_f32_16x16x32_bf16 v[24:27], v[240:243], v[200:203], v[24:27]\n"
    "v_cvt_pk_bf16_f32 v140, v140, v141\n"
    "v_cvt_pk_bf16_f32 v141, v142, v143\n"
    "v_mfma_f32_16x16x32_bf16 v[32:35], v[212:215], v[204:207], v[32:35]\n"
    "v_cvt_pk_bf16_f32 v142, v144, v145\n"
    "v_cvt_pk_bf16_f32 v143, v146, v147\n"
    "v_mfma_f32_16x16x32_bf16 v[32:35], v[216:219], v[208:211], v[32:35]\n"
    "v_cvt_pk_bf16_f32 v148, v148, v149\n"
    "v_cvt_pk_bf16_f32 v149, v150, v151\n"
    "v_mfma_f32_16x16x32_bf16 v[40:43], v[220:223], v[204:207], v[40:43]\n"
    "v_cvt_pk_bf16_f32 v150, v152, v153\n"
    "v_cvt_pk_bf16_f32 v151, v154, v155\n"
    "v_mfma_f32_16x16x32_bf16 v[40:43], v[224:227], v[208:211], v[40:43]\n"
    "v_cvt_pk_bf16_f32 v156, v156, v157\n"
    "v_cvt_pk_bf16_f32 v157, v158, v159\n"
    "v_mfma_f32_16x16x32_bf16 v[12:15], v[228:231], v[204:207], v[12:15]\n"
    "v_cvt_pk_bf16_f32 v158, v160, v161\n"
    "v_cvt_pk_bf16_f32 v159, v162, v163\n"
    "v_mfma_f32_16x16x32_bf16 v[12:15], v[232:235], v[208:211], v[12:15]\n"
    "v_cvt_pk_bf16_f32 v164, v164, v165\n"
    "v_cvt_pk_bf16_f32 v165, v166, v167\n"
    "v_mfma_f32_16x16x32_bf16 v[48:51], v[236:239], v[204:207], v[48:51]\n"
    "v_cvt_pk_bf16_f32 v166, v168, v169\n"
    "v_cvt_pk_bf16_f32 v167, v170, v171\n"
    "v_mfma_f32_16x16x32_bf16 v[48:51], v[240:243], v[208:211], v[48:51]\n"
    "s_setprio 0\n"
    "s_barrier\n"
    "; phase 103: step 63 slot 1 stage 3\n"
    "ds_read_b64_tr_b16 v[212:213], v246 offset:49152\n"
    "s_xor_b32 s82, s54, 0x6e000\n"
    "s_add_u32 s82, s82, 0x100000\n"
    "buffer_store_dwordx4 v[140:143], v252, s[44:47], s82 offen sc0 sc1\n"
    "ds_read_b64_tr_b16 v[214:215], v246 offset:50176\n"
    "s_xor_b32 s82, s54, 0x6c000\n"
    "s_add_u32 s82, s82, 0x100000\n"
    "buffer_store_dwordx4 v[148:151], v252, s[44:47], s82 offen sc0 sc1\n"
    "ds_read_b64_tr_b16 v[216:217], v246 offset:57344\n"
    "s_xor_b32 s82, s54, 0x4a000\n"
    "s_add_u32 s82, s82, 0x100000\n"
    "buffer_store_dwordx4 v[156:159], v252, s[44:47], s82 offen sc0 sc1\n"
    "ds_read_b64_tr_b16 v[218:219], v246 offset:58368\n"
    "s_cmp_lg_u32 s52, 5\n"
    "s_cbranch_scc1 .La138_%=\n"
    "s_xor_b32 s82, s81, 0x4000\n"
    "buffer_store_dwordx4 v[44:47], v255, s[76:79], s82 offen nt\n"
    ".La138_%=:\n"
    "ds_read_b64_tr_b16 v[220:221], v247 offset:49152\n"
    "s_add_u32 m0, s51, 0x8000\n"
    "s_xor_b32 s82, s67, 0x4200000\n"
    "buffer_load_dwordx4 v251, s[40:43], s82 offen  lds\n"
    "ds_read_b64_tr_b16 v[222:223], v247 offset:50176\n"
    "s_add_u32 m0, s51, 0xa000\n"
    "s_xor_b32 s82, s67, 0x4280000\n"
    "buffer_load_dwordx4 v251, s[40:43], s82 offen  lds\n"
    "ds_read_b64_tr_b16 v[224:225], v247 offset:57344\n"
    "s_add_u32 m0, s50, 0x0\n"
    "s_xor_b32 s82, s48, 0x202100\n"
    "buffer_load_dwordx4 v250, s[36:39], s82 offen  lds\n"
    "ds_read_b64_tr_b16 v[226:227], v247 offset:58368\n"
    "s_add_u32 m0, s50, 0x1000\n"
    "s_xor_b32 s82, s48, 0x282100\n"
    "buffer_load_dwordx4 v250, s[36:39], s82 offen  lds\n"
    "ds_read_b64_tr_b16 v[228:229], v248 offset:49152\n"
    "s_add_u32 m0, s50, 0x2000\n"
    "s_xor_b32 s82, s48, 0x2100\n"
    "buffer_load_dwordx4 v250, s[36:39], s82 offen  lds\n"
    "ds_read_b64_tr_b16 v[230:231], v248 offset:50176\n"
    "s_add_u32 m0, s50, 0x3000\n"
    "s_xor_b32 s82, s48, 0x82100\n"
    "buffer_load_dwordx4 v250, s[36:39], s82 offen  lds\n"
    "ds_read_b64_tr_b16 v[232:233], v248 offset:57344\n"
    "ds_read_b64_tr_b16 v[234:235], v248 offset:58368\n"
    "ds_read_b64_tr_b16 v[236:237], v249 offset:49152\n"
    "ds_read_b64_tr_b16 v[238:239], v249 offset:50176\n"
    "ds_read_b64_tr_b16 v[240:241], v249 offset:57344\n"
    "ds_read_b64_tr_b16 v[242:243], v249 offset:58368\n"
    "s_waitcnt vmcnt(15) lgkmcnt(0)\n"
    "s_barrier\n"
    "s_setprio 1\n"
    "v_mfma_f32_16x16x32_bf16 v[64:67], v[212:215], v[180:183], v[64:67]\n"
    "v_cvt_pk_bf16_f32 v172, v172, v173\n"
    "v_cvt_pk_bf16_f32 v173, v174, v175\n"
    "v_mfma_f32_16x16x32_bf16 v[64:67], v[216:219], v[184:187], v[64:67]\n"
    "v_cvt_pk_bf16_f32 v174, v176, v177\n"
    "v_cvt_pk_bf16_f32 v175, v178, v179\n"
    "v_mfma_f32_16x16x32_bf16 v[60:63], v[220:223], v[180:183], v[60:63]\n"
    "v_cvt_pk_bf16_f32 v16, v16, v17\n"
    "v_cvt_pk_bf16_f32 v17, v18, v19\n"
    "v_mfma_f32_16x16x32_bf16 v[60:63], v[224:227], v[184:187], v[60:63]\n"
    "v_cvt_pk_bf16_f32 v18, v24, v25\n"
    "v_cvt_pk_bf16_f32 v19, v26, v27\n"
    "v_mfma_f32_16x16x32_bf16 v[80:83], v[228:231], v[180:183], v[80:83]\n"
    "v_mfma_f32_16x16x32_bf16 v[80:83], v[232:235], v[184:187], v[80:83]\n"
    "v_mfma_f32_16x16x32_bf16 v[88:91], v[236:239], v[180:183], v[88:91]\n"
    "v_mfma_f32_16x16x32_bf16 v[88:91], v[240:243], v[184:187], v[88:91]\n"
    "v_mfma_f32_16x16x32_bf16 v[96:99], v[212:215], v[188:191], v[96:99]\n"
    "v_mfma_f32_16x16x32_bf16 v[96:99], v[216:219], v[192:195], v[96:99]\n"
    "v_mfma_f32_16x16x32_bf16 v[104:107], v[220:223], v[188:191], v[104:107]\n"
    "v_mfma_f32_16x16x32_bf16 v[104:107], v[224:227], v[192:195], v[104:107]\n"
    "v_mfma_f32_16x16x32_bf16 v[76:79], v[228:231], v[188:191], v[76:79]\n"
    "v_mfma_f32_16x16x32_bf16 v[76:79], v[232:235], v[192:195], v[76:79]\n"
    "v_mfma_f32_16x16x32_bf16 v[112:115], v[236:239], v[188:191], v[112:115]\n"
    "v_mfma_f32_16x16x32_bf16 v[112:115], v[240:243], v[192:195], v[112:115]\n"
    "v_mfma_f32_16x16x32_bf16 v[120:123], v[212:215], v[196:199], v[120:123]\n"
    "v_mfma_f32_16x16x32_bf16 v[120:123], v[216:219], v[200:203], v[120:123]\n"
    "v_mfma_f32_16x16x32_bf16 v[84:87], v[220:223], v[196:199], v[84:87]\n"
    "v_mfma_f32_16x16x32_bf16 v[84:87], v[224:227], v[200:203], v[84:87]\n"
    "v_mfma_f32_16x16x32_bf16 v[28:31], v[228:231], v[196:199], v[28:31]\n"
    "v_mfma_f32_16x16x32_bf16 v[28:31], v[232:235], v[200:203], v[28:31]\n"
    "v_mfma_f32_16x16x32_bf16 v[92:95], v[236:239], v[196:199], v[92:95]\n"
    "v_mfma_f32_16x16x32_bf16 v[92:95], v[240:243], v[200:203], v[92:95]\n"
    "v_mfma_f32_16x16x32_bf16 v[100:103], v[212:215], v[204:207], v[100:103]\n"
    "v_mfma_f32_16x16x32_bf16 v[100:103], v[216:219], v[208:211], v[100:103]\n"
    "v_mfma_f32_16x16x32_bf16 v[108:111], v[220:223], v[204:207], v[108:111]\n"
    "v_mfma_f32_16x16x32_bf16 v[108:111], v[224:227], v[208:211], v[108:111]\n"
    "v_mfma_f32_16x16x32_bf16 v[116:119], v[228:231], v[204:207], v[116:119]\n"
    "v_mfma_f32_16x16x32_bf16 v[116:119], v[232:235], v[208:211], v[116:119]\n"
    "v_mfma_f32_16x16x32_bf16 v[52:55], v[236:239], v[204:207], v[52:55]\n"
    "v_mfma_f32_16x16x32_bf16 v[52:55], v[240:243], v[208:211], v[52:55]\n"
    "s_setprio 0\n"
    "s_barrier\n"
    "; phase 104: step 64 slot 1 stage 3\n"
    "ds_read_b128 v[180:183], v244 offset:16384\n"
    "ds_read_b128 v[184:187], v245 offset:16384\n"
    "ds_read_b128 v[188:191], v244 offset:18432\n"
    "ds_read_b128 v[192:195], v245 offset:18432\n"
    "s_xor_b32 s82, s54, 0x48000\n"
    "s_add_u32 s82, s82, 0x100000\n"
    "buffer_store_dwordx4 v[164:167], v252, s[44:47], s82 offen sc0 sc1\n"
    "ds_read_b128 v[196:199], v244 offset:24576\n"
    "ds_read_b128 v[200:203], v245 offset:24576\n"
    "ds_read_b128 v[204:207], v244 offset:26624\n"
    "ds_read_b128 v[208:211], v245 offset:26624\n"
    "s_xor_b32 s82, s54, 0x26000\n"
    "s_add_u32 s82, s82, 0x100000\n"
    "buffer_store_dwordx4 v[172:175], v252, s[44:47], s82 offen sc0 sc1\n"
    "ds_read_b64_tr_b16 v[212:213], v246 offset:0\n"
    "ds_read_b64_tr_b16 v[214:215], v246 offset:1024\n"
    "ds_read_b64_tr_b16 v[216:217], v246 offset:8192\n"
    "ds_read_b64_tr_b16 v[218:219], v246 offset:9216\n"
    "s_xor_b32 s82, s54, 0x24000\n"
    "s_add_u32 s82, s82, 0x100000\n"
    "buffer_store_dwordx4 v[16:19], v252, s[44:47], s82 offen sc0 sc1\n"
    "ds_read_b64_tr_b16 v[220:221], v247 offset:0\n"
    "ds_read_b64_tr_b16 v[222:223], v247 offset:1024\n"
    "ds_read_b64_tr_b16 v[224:225], v247 offset:8192\n"
    "ds_read_b64_tr_b16 v[226:227], v247 offset:9216\n"
    "s_cmp_lg_u32 s52, 6\n"
    "s_cbranch_scc1 .La139_%=\n"
    "s_xor_b32 s82, s81, 0x4000\n"
    "buffer_store_dwordx4 v[44:47], v255, s[76:79], s82 offen nt\n"
    ".La139_%=:\n"
    "ds_read_b64_tr_b16 v[228:229], v248 offset:0\n"
    "ds_read_b64_tr_b16 v[230:231], v248 offset:1024\n"
    "ds_read_b64_tr_b16 v[232:233], v248 offset:8192\n"
    "ds_read_b64_tr_b16 v[234:235], v248 offset:9216\n"
    "s_add_u32 m0, s51, 0xc000\n"
    "s_xor_b32 s82, s67, 0x4300000\n"
    "buffer_load_dwordx4 v251, s[40:43], s82 offen  lds\n"
    "ds_read_b64_tr_b16 v[236:237], v249 offset:0\n"
    "ds_read_b64_tr_b16 v[238:239], v249 offset:1024\n"
    "ds_read_b64_tr_b16 v[240:241], v249 offset:8192\n"
    "ds_read_b64_tr_b16 v[242:243], v249 offset:9216\n"
    "s_add_u32 m0, s51, 0xe000\n"
    "s_xor_b32 s82, s67, 0x4380000\n"
    "buffer_load_dwordx4 v251, s[40:43], s82 offen  lds\n"
    "s_waitcnt vmcnt(14) lgkmcnt(0)\n"
    "s_barrier\n"
    "s_setprio 1\n"
    "v_mfma_f32_16x16x32_bf16 v[64:67], v[212:215], v[180:183], v[64:67]\n"
    "v_mfma_f32_16x16x32_bf16 v[64:67], v[216:219], v[184:187], v[64:67]\n"
    "v_mfma_f32_16x16x32_bf16 v[60:63], v[220:223], v[180:183], v[60:63]\n"
    "v_mfma_f32_16x16x32_bf16 v[60:63], v[224:227], v[184:187], v[60:63]\n"
    "v_mfma_f32_16x16x32_bf16 v[80:83], v[228:231], v[180:183], v[80:83]\n"
    "v_mfma_f32_16x16x32_bf16 v[80:83], v[232:235], v[184:187], v[80:83]\n"
    "v_mfma_f32_16x16x32_bf16 v[88:91], v[236:239], v[180:183], v[88:91]\n"
    "v_mfma_f32_16x16x32_bf16 v[88:91], v[240:243], v[184:187], v[88:91]\n"
    "v_mfma_f32_16x16x32_bf16 v[96:99], v[212:215], v[188:191], v[96:99]\n"
    "v_mfma_f32_16x16x32_bf16 v[96:99], v[216:219], v[192:195], v[96:99]\n"
    "v_mfma_f32_16x16x32_bf16 v[104:107], v[220:223], v[188:191], v[104:107]\n"
    "v_mfma_f32_16x16x32_bf16 v[104:107], v[224:227], v[192:195], v[104:107]\n"
    "v_mfma_f32_16x16x32_bf16 v[76:79], v[228:231], v[188:191], v[76:79]\n"
    "v_mfma_f32_16x16x32_bf16 v[76:79], v[232:235], v[192:195], v[76:79]\n"
    "v_mfma_f32_16x16x32_bf16 v[112:115], v[236:239], v[188:191], v[112:115]\n"
    "v_mfma_f32_16x16x32_bf16 v[112:115], v[240:243], v[192:195], v[112:115]\n"
    "v_mfma_f32_16x16x32_bf16 v[120:123], v[212:215], v[196:199], v[120:123]\n"
    "v_mfma_f32_16x16x32_bf16 v[120:123], v[216:219], v[200:203], v[120:123]\n"
    "v_mfma_f32_16x16x32_bf16 v[84:87], v[220:223], v[196:199], v[84:87]\n"
    "v_mfma_f32_16x16x32_bf16 v[84:87], v[224:227], v[200:203], v[84:87]\n"
    "v_mfma_f32_16x16x32_bf16 v[28:31], v[228:231], v[196:199], v[28:31]\n"
    "v_mfma_f32_16x16x32_bf16 v[28:31], v[232:235], v[200:203], v[28:31]\n"
    "v_mfma_f32_16x16x32_bf16 v[92:95], v[236:239], v[196:199], v[92:95]\n"
    "v_mfma_f32_16x16x32_bf16 v[92:95], v[240:243], v[200:203], v[92:95]\n"
    "v_mfma_f32_16x16x32_bf16 v[100:103], v[212:215], v[204:207], v[100:103]\n"
    "v_mfma_f32_16x16x32_bf16 v[100:103], v[216:219], v[208:211], v[100:103]\n"
    "v_mfma_f32_16x16x32_bf16 v[108:111], v[220:223], v[204:207], v[108:111]\n"
    "v_mfma_f32_16x16x32_bf16 v[108:111], v[224:227], v[208:211], v[108:111]\n"
    "v_mfma_f32_16x16x32_bf16 v[116:119], v[228:231], v[204:207], v[116:119]\n"
    "v_mfma_f32_16x16x32_bf16 v[116:119], v[232:235], v[208:211], v[116:119]\n"
    "v_mfma_f32_16x16x32_bf16 v[52:55], v[236:239], v[204:207], v[52:55]\n"
    "v_mfma_f32_16x16x32_bf16 v[52:55], v[240:243], v[208:211], v[52:55]\n"
    "s_setprio 0\n"
    "s_barrier\n"
    "; phase 105: step 65 slot 1 stage 3\n"
    "ds_read_b128 v[180:183], v244 offset:32768\n"
    "ds_read_b128 v[184:187], v245 offset:32768\n"
    "ds_read_b128 v[188:191], v244 offset:34816\n"
    "s_cmp_lg_u32 s52, 7\n"
    "s_cbranch_scc1 .La140_%=\n"
    "s_xor_b32 s82, s81, 0x4000\n"
    "buffer_store_dwordx4 v[44:47], v255, s[76:79], s82 offen nt\n"
    ".La140_%=:\n"
    "ds_read_b128 v[192:195], v245 offset:34816\n"
    "ds_read_b128 v[196:199], v244 offset:40960\n"
    "ds_read_b128 v[200:203], v245 offset:40960\n"
    "s_add_u32 m0, s50, 0x4000\n"
    "s_xor_b32 s82, s48, 0x202180\n"
    "buffer_load_dwordx4 v250, s[36:39], s82 offen  lds\n"
    "ds_read_b128 v[204:207], v244 offset:43008\n"
    "ds_read_b128 v[208:211], v245 offset:43008\n"
    "ds_read_b64_tr_b16 v[212:213], v246 offset:16384\n"
    "s_add_u32 m0, s50, 0x5000\n"
    "s_xor_b32 s82, s48, 0x282180\n"
    "buffer_load_dwordx4 v250, s[36:39], s82 offen  lds\n"
    "ds_read_b64_tr_b16 v[214:215], v246 offset:17408\n"
    "ds_read_b64_tr_b16 v[216:217], v246 offset:24576\n"
    "ds_read_b64_tr_b16 v[218:219], v246 offset:25600\n"
    "s_add_u32 m0, s50, 0x6000\n"
    "s_xor_b32 s82, s48, 0x2180\n"
    "buffer_load_dwordx4 v250, s[36:39], s82 offen  lds\n"
    "ds_read_b64_tr_b16 v[220:221], v247 offset:16384\n"
    "ds_read_b64_tr_b16 v[222:223], v247 offset:17408\n"
    "ds_read_b64_tr_b16 v[224:225], v247 offset:24576\n"
    "s_add_u32 m0, s50, 0x7000\n"
    "s_xor_b32 s82, s48, 0x82180\n"
    "buffer_load_dwordx4 v250, s[36:39], s82 offen  lds\n"
    "ds_read_b64_tr_b16 v[226:227], v247 offset:25600\n"
    "ds_read_b64_tr_b16 v[228:229], v248 offset:16384\n"
    "ds_read_b64_tr_b16 v[230:231], v248 offset:17408\n"
    "s_add_u32 m0, s51, 0x0\n"
    "s_xor_b32 s82, s67, 0x4400000\n"
    "buffer_load_dwordx4 v251, s[40:43], s82 offen  lds\n"
    "ds_read_b64_tr_b16 v[232:233], v248 offset:24576\n"
    "ds_read_b64_tr_b16 v[234:235], v248 offset:25600\n"
    "ds_read_b64_tr_b16 v[236:237], v249 offset:16384\n"
    "s_add_u32 m0, s51, 0x2000\n"
    "s_xor_b32 s82, s67, 0x4480000\n"
    "buffer_load_dwordx4 v251, s[40:43], s82 offen  lds\n"
    "ds_read_b64_tr_b16 v[238:239], v249 offset:17408\n"
    "ds_read_b64_tr_b16 v[240:241], v249 offset:24576\n"
    "ds_read_b64_tr_b16 v[242:243], v249 offset:25600\n"
    "s_waitcnt vmcnt(15) lgkmcnt(0)\n"
    "s_barrier\n"
    "s_setprio 1\n"
    "v_mfma_f32_16x16x32_bf16 v[64:67], v[212:215], v[180:183], v[64:67]\n"
    "v_mfma_f32_16x16x32_bf16 v[64:67], v[216:219], v[184:187], v[64:67]\n"
    "v_mfma_f32_16x16x32_bf16 v[60:63], v[220:223], v[180:183], v[60:63]\n"
    "v_mfma_f32_16x16x32_bf16 v[60:63], v[224:227], v[184:187], v[60:63]\n"
    "v_mfma_f32_16x16x32_bf16 v[80:83], v[228:231], v[180:183], v[80:83]\n"
    "v_mfma_f32_16x16x32_bf16 v[80:83], v[232:235], v[184:187], v[80:83]\n"
    "v_mfma_f32_16x16x32_bf16 v[88:91], v[236:239], v[180:183], v[88:91]\n"
    "v_mfma_f32_16x16x32_bf16 v[88:91], v[240:243], v[184:187], v[88:91]\n"
    "v_mfma_f32_16x16x32_bf16 v[96:99], v[212:215], v[188:191], v[96:99]\n"
    "v_mfma_f32_16x16x32_bf16 v[96:99], v[216:219], v[192:195], v[96:99]\n"
    "v_mfma_f32_16x16x32_bf16 v[104:107], v[220:223], v[188:191], v[104:107]\n"
    "v_mfma_f32_16x16x32_bf16 v[104:107], v[224:227], v[192:195], v[104:107]\n"
    "v_mfma_f32_16x16x32_bf16 v[76:79], v[228:231], v[188:191], v[76:79]\n"
    "v_mfma_f32_16x16x32_bf16 v[76:79], v[232:235], v[192:195], v[76:79]\n"
    "v_mfma_f32_16x16x32_bf16 v[112:115], v[236:239], v[188:191], v[112:115]\n"
    "v_mfma_f32_16x16x32_bf16 v[112:115], v[240:243], v[192:195], v[112:115]\n"
    "v_mfma_f32_16x16x32_bf16 v[120:123], v[212:215], v[196:199], v[120:123]\n"
    "v_mfma_f32_16x16x32_bf16 v[120:123], v[216:219], v[200:203], v[120:123]\n"
    "v_mfma_f32_16x16x32_bf16 v[84:87], v[220:223], v[196:199], v[84:87]\n"
    "v_mfma_f32_16x16x32_bf16 v[84:87], v[224:227], v[200:203], v[84:87]\n"
    "v_mfma_f32_16x16x32_bf16 v[28:31], v[228:231], v[196:199], v[28:31]\n"
    "v_mfma_f32_16x16x32_bf16 v[28:31], v[232:235], v[200:203], v[28:31]\n"
    "v_mfma_f32_16x16x32_bf16 v[92:95], v[236:239], v[196:199], v[92:95]\n"
    "v_mfma_f32_16x16x32_bf16 v[92:95], v[240:243], v[200:203], v[92:95]\n"
    "v_mfma_f32_16x16x32_bf16 v[100:103], v[212:215], v[204:207], v[100:103]\n"
    "v_mfma_f32_16x16x32_bf16 v[100:103], v[216:219], v[208:211], v[100:103]\n"
    "v_mfma_f32_16x16x32_bf16 v[108:111], v[220:223], v[204:207], v[108:111]\n"
    "v_mfma_f32_16x16x32_bf16 v[108:111], v[224:227], v[208:211], v[108:111]\n"
    "v_mfma_f32_16x16x32_bf16 v[116:119], v[228:231], v[204:207], v[116:119]\n"
    "v_mfma_f32_16x16x32_bf16 v[116:119], v[232:235], v[208:211], v[116:119]\n"
    "v_mfma_f32_16x16x32_bf16 v[52:55], v[236:239], v[204:207], v[52:55]\n"
    "v_mfma_f32_16x16x32_bf16 v[52:55], v[240:243], v[208:211], v[52:55]\n"
    "s_setprio 0\n"
    "s_waitcnt vmcnt(8)\n"
    "s_barrier\n"
    "; phase 106: step 66 slot 1 stage 3\n"
    "ds_read_b128 v[180:183], v244 offset:0\n"
    "ds_read_b128 v[184:187], v245 offset:0\n"
    "ds_read_b128 v[188:191], v244 offset:2048\n"
    "s_mov_b32 s82, 0x2300000\n"
    "buffer_store_dwordx2 v[10:11], v253, s[44:47], s82 offen sc0 sc1\n"
    "ds_read_b128 v[192:195], v245 offset:2048\n"
    "ds_read_b128 v[196:199], v244 offset:8192\n"
    "ds_read_b128 v[200:203], v245 offset:8192\n"
    "s_add_u32 m0, s50, 0x8000\n"
    "s_xor_b32 s82, s48, 0x202200\n"
    "buffer_load_dwordx4 v250, s[36:39], s82 offen  lds\n"
    "ds_read_b128 v[204:207], v244 offset:10240\n"
    "ds_read_b128 v[208:211], v245 offset:10240\n"
    "ds_read_b64_tr_b16 v[212:213], v246 offset:32768\n"
    "s_add_u32 m0, s50, 0x9000\n"
    "s_xor_b32 s82, s48, 0x282200\n"
    "buffer_load_dwordx4 v250, s[36:39], s82 offen  lds\n"
    "ds_read_b64_tr_b16 v[214:215], v246 offset:33792\n"
    "ds_read_b64_tr_b16 v[216:217], v246 offset:40960\n"
    "ds_read_b64_tr_b16 v[218:219], v246 offset:41984\n"
    "s_add_u32 m0, s50, 0xa000\n"
    "s_xor_b32 s82, s48, 0x2200\n"
    "buffer_load_dwordx4 v250, s[36:39], s82 offen  lds\n"
    "ds_read_b64_tr_b16 v[220:221], v247 offset:32768\n"
    "ds_read_b64_tr_b16 v[222:223], v247 offset:33792\n"
    "ds_read_b64_tr_b16 v[224:225], v247 offset:40960\n"
    "s_add_u32 m0, s50, 0xb000\n"
    "s_xor_b32 s82, s48, 0x82200\n"
    "buffer_load_dwordx4 v250, s[36:39], s82 offen  lds\n"
    "ds_read_b64_tr_b16 v[226:227], v247 offset:41984\n"
    "ds_read_b64_tr_b16 v[228:229], v248 offset:32768\n"
    "ds_read_b64_tr_b16 v[230:231], v248 offset:33792\n"
    "s_add_u32 m0, s51, 0x4000\n"
    "s_xor_b32 s82, s67, 0x4500000\n"
    "buffer_load_dwordx4 v251, s[40:43], s82 offen  lds\n"
    "ds_read_b64_tr_b16 v[232:233], v248 offset:40960\n"
    "ds_read_b64_tr_b16 v[234:235], v248 offset:41984\n"
    "ds_read_b64_tr_b16 v[236:237], v249 offset:32768\n"
    "s_add_u32 m0, s51, 0x6000\n"
    "s_xor_b32 s82, s67, 0x4580000\n"
    "buffer_load_dwordx4 v251, s[40:43], s82 offen  lds\n"
    "ds_read_b64_tr_b16 v[238:239], v249 offset:33792\n"
    "ds_read_b64_tr_b16 v[240:241], v249 offset:40960\n"
    "ds_read_b64_tr_b16 v[242:243], v249 offset:41984\n"
    "s_waitcnt vmcnt(13) lgkmcnt(0)\n"
    "s_barrier\n"
    "s_setprio 1\n"
    "v_mfma_f32_16x16x32_bf16 v[64:67], v[212:215], v[180:183], v[64:67]\n"
    "v_mfma_f32_16x16x32_bf16 v[64:67], v[216:219], v[184:187], v[64:67]\n"
    "v_mfma_f32_16x16x32_bf16 v[60:63], v[220:223], v[180:183], v[60:63]\n"
    "v_mfma_f32_16x16x32_bf16 v[60:63], v[224:227], v[184:187], v[60:63]\n"
    "v_mfma_f32_16x16x32_bf16 v[80:83], v[228:231], v[180:183], v[80:83]\n"
    "v_mfma_f32_16x16x32_bf16 v[80:83], v[232:235], v[184:187], v[80:83]\n"
    "v_mfma_f32_16x16x32_bf16 v[88:91], v[236:239], v[180:183], v[88:91]\n"
    "v_mfma_f32_16x16x32_bf16 v[88:91], v[240:243], v[184:187], v[88:91]\n"
    "v_mfma_f32_16x16x32_bf16 v[96:99], v[212:215], v[188:191], v[96:99]\n"
    "v_mfma_f32_16x16x32_bf16 v[96:99], v[216:219], v[192:195], v[96:99]\n"
    "v_mfma_f32_16x16x32_bf16 v[104:107], v[220:223], v[188:191], v[104:107]\n"
    "v_mfma_f32_16x16x32_bf16 v[104:107], v[224:227], v[192:195], v[104:107]\n"
    "v_mfma_f32_16x16x32_bf16 v[76:79], v[228:231], v[188:191], v[76:79]\n"
    "v_mfma_f32_16x16x32_bf16 v[76:79], v[232:235], v[192:195], v[76:79]\n"
    "v_mfma_f32_16x16x32_bf16 v[112:115], v[236:239], v[188:191], v[112:115]\n"
    "v_mfma_f32_16x16x32_bf16 v[112:115], v[240:243], v[192:195], v[112:115]\n"
    "v_mfma_f32_16x16x32_bf16 v[120:123], v[212:215], v[196:199], v[120:123]\n"
    "v_mfma_f32_16x16x32_bf16 v[120:123], v[216:219], v[200:203], v[120:123]\n"
    "v_mfma_f32_16x16x32_bf16 v[84:87], v[220:223], v[196:199], v[84:87]\n"
    "v_mfma_f32_16x16x32_bf16 v[84:87], v[224:227], v[200:203], v[84:87]\n"
    "v_mfma_f32_16x16x32_bf16 v[28:31], v[228:231], v[196:199], v[28:31]\n"
    "v_mfma_f32_16x16x32_bf16 v[28:31], v[232:235], v[200:203], v[28:31]\n"
    "v_mfma_f32_16x16x32_bf16 v[92:95], v[236:239], v[196:199], v[92:95]\n"
    "v_mfma_f32_16x16x32_bf16 v[92:95], v[240:243], v[200:203], v[92:95]\n"
    "v_mfma_f32_16x16x32_bf16 v[100:103], v[212:215], v[204:207], v[100:103]\n"
    "v_mfma_f32_16x16x32_bf16 v[100:103], v[216:219], v[208:211], v[100:103]\n"
    "v_mfma_f32_16x16x32_bf16 v[108:111], v[220:223], v[204:207], v[108:111]\n"
    "v_mfma_f32_16x16x32_bf16 v[108:111], v[224:227], v[208:211], v[108:111]\n"
    "v_mfma_f32_16x16x32_bf16 v[116:119], v[228:231], v[204:207], v[116:119]\n"
    "v_mfma_f32_16x16x32_bf16 v[116:119], v[232:235], v[208:211], v[116:119]\n"
    "v_mfma_f32_16x16x32_bf16 v[52:55], v[236:239], v[204:207], v[52:55]\n"
    "v_mfma_f32_16x16x32_bf16 v[52:55], v[240:243], v[208:211], v[52:55]\n"
    "s_setprio 0\n"
    "s_waitcnt vmcnt(9)\n"
    "s_barrier\n"
    "; phase 107: step 67 slot 1 stage 3\n"
    "ds_read_b128 v[180:183], v244 offset:16384\n"
    "ds_read_b128 v[184:187], v245 offset:16384\n"
    "ds_read_b128 v[188:191], v244 offset:18432\n"
    "s_add_u32 s82, s56, 0x200000\n"
    "buffer_load_dword v6, v254, s[44:47], s82 offen sc0 sc1\n"
    "s_lshr_b32 s83, s66, 8\n"
    "s_and_b32 s82, s83, 0x38\n"
    "s_load_dwordx2 s[72:73], s[60:61], s82\n"
    "s_load_dwordx2 s[20:21], s[60:61], s82 offset:0x40\n"
    "s_and_b32 s83, s66, 0x700\n"
    "s_add_u32 s80, s57, s83\n"
    "ds_read_b128 v[192:195], v245 offset:18432\n"
    "ds_read_b128 v[196:199], v244 offset:24576\n"
    "ds_read_b128 v[200:203], v245 offset:24576\n"
    "s_add_u32 m0, s50, 0x0\n"
    "s_xor_b32 s82, s48, 0x202280\n"
    "buffer_load_dwordx4 v250, s[36:39], s82 offen  lds\n"
    "ds_read_b128 v[204:207], v244 offset:26624\n"
    "ds_read_b128 v[208:211], v245 offset:26624\n"
    "ds_read_b64_tr_b16 v[212:213], v246 offset:49152\n"
    "s_add_u32 m0, s50, 0x1000\n"
    "s_xor_b32 s82, s48, 0x282280\n"
    "buffer_load_dwordx4 v250, s[36:39], s82 offen  lds\n"
    "ds_read_b64_tr_b16 v[214:215], v246 offset:50176\n"
    "ds_read_b64_tr_b16 v[216:217], v246 offset:57344\n"
    "ds_read_b64_tr_b16 v[218:219], v246 offset:58368\n"
    "s_add_u32 m0, s50, 0x2000\n"
    "s_xor_b32 s82, s48, 0x2280\n"
    "buffer_load_dwordx4 v250, s[36:39], s82 offen  lds\n"
    "ds_read_b64_tr_b16 v[220:221], v247 offset:49152\n"
    "ds_read_b64_tr_b16 v[222:223], v247 offset:50176\n"
    "ds_read_b64_tr_b16 v[224:225], v247 offset:57344\n"
    "s_add_u32 m0, s50, 0x3000\n"
    "s_xor_b32 s82, s48, 0x82280\n"
    "buffer_load_dwordx4 v250, s[36:39], s82 offen  lds\n"
    "ds_read_b64_tr_b16 v[226:227], v247 offset:58368\n"
    "ds_read_b64_tr_b16 v[228:229], v248 offset:49152\n"
    "ds_read_b64_tr_b16 v[230:231], v248 offset:50176\n"
    "s_add_u32 m0, s51, 0x8000\n"
    "s_xor_b32 s82, s67, 0x4600000\n"
    "buffer_load_dwordx4 v251, s[40:43], s82 offen  lds\n"
    "ds_read_b64_tr_b16 v[232:233], v248 offset:57344\n"
    "ds_read_b64_tr_b16 v[234:235], v248 offset:58368\n"
    "ds_read_b64_tr_b16 v[236:237], v249 offset:49152\n"
    "s_add_u32 m0, s51, 0xa000\n"
    "s_xor_b32 s82, s67, 0x4680000\n"
    "buffer_load_dwordx4 v251, s[40:43], s82 offen  lds\n"
    "ds_read_b64_tr_b16 v[238:239], v249 offset:50176\n"
    "ds_read_b64_tr_b16 v[240:241], v249 offset:57344\n"
    "ds_read_b64_tr_b16 v[242:243], v249 offset:58368\n"
    "s_waitcnt vmcnt(14) lgkmcnt(0)\n"
    "s_barrier\n"
    "s_setprio 1\n"
    "v_mfma_f32_16x16x32_bf16 v[64:67], v[212:215], v[180:183], v[64:67]\n"
    "v_mfma_f32_16x16x32_bf16 v[64:67], v[216:219], v[184:187], v[64:67]\n"
    "v_mfma_f32_16x16x32_bf16 v[60:63], v[220:223], v[180:183], v[60:63]\n"
    "v_mfma_f32_16x16x32_bf16 v[60:63], v[224:227], v[184:187], v[60:63]\n"
    "v_mfma_f32_16x16x32_bf16 v[80:83], v[228:231], v[180:183], v[80:83]\n"
    "v_mfma_f32_16x16x32_bf16 v[80:83], v[232:235], v[184:187], v[80:83]\n"
    "v_mfma_f32_16x16x32_bf16 v[88:91], v[236:239], v[180:183], v[88:91]\n"
    "v_mfma_f32_16x16x32_bf16 v[88:91], v[240:243], v[184:187], v[88:91]\n"
    "v_mfma_f32_16x16x32_bf16 v[96:99], v[212:215], v[188:191], v[96:99]\n"
    "v_mfma_f32_16x16x32_bf16 v[96:99], v[216:219], v[192:195], v[96:99]\n"
    "v_mfma_f32_16x16x32_bf16 v[104:107], v[220:223], v[188:191], v[104:107]\n"
    "v_mfma_f32_16x16x32_bf16 v[104:107], v[224:227], v[192:195], v[104:107]\n"
    "v_mfma_f32_16x16x32_bf16 v[76:79], v[228:231], v[188:191], v[76:79]\n"
    "v_mfma_f32_16x16x32_bf16 v[76:79], v[232:235], v[192:195], v[76:79]\n"
    "v_mfma_f32_16x16x32_bf16 v[112:115], v[236:239], v[188:191], v[112:115]\n"
    "v_mfma_f32_16x16x32_bf16 v[112:115], v[240:243], v[192:195], v[112:115]\n"
    "v_mfma_f32_16x16x32_bf16 v[120:123], v[212:215], v[196:199], v[120:123]\n"
    "v_mfma_f32_16x16x32_bf16 v[120:123], v[216:219], v[200:203], v[120:123]\n"
    "v_mfma_f32_16x16x32_bf16 v[84:87], v[220:223], v[196:199], v[84:87]\n"
    "v_mfma_f32_16x16x32_bf16 v[84:87], v[224:227], v[200:203], v[84:87]\n"
    "v_mfma_f32_16x16x32_bf16 v[28:31], v[228:231], v[196:199], v[28:31]\n"
    "v_mfma_f32_16x16x32_bf16 v[28:31], v[232:235], v[200:203], v[28:31]\n"
    "v_mfma_f32_16x16x32_bf16 v[92:95], v[236:239], v[196:199], v[92:95]\n"
    "v_mfma_f32_16x16x32_bf16 v[92:95], v[240:243], v[200:203], v[92:95]\n"
    "v_mfma_f32_16x16x32_bf16 v[100:103], v[212:215], v[204:207], v[100:103]\n"
    "v_mfma_f32_16x16x32_bf16 v[100:103], v[216:219], v[208:211], v[100:103]\n"
    "v_mfma_f32_16x16x32_bf16 v[108:111], v[220:223], v[204:207], v[108:111]\n"
    "v_mfma_f32_16x16x32_bf16 v[108:111], v[224:227], v[208:211], v[108:111]\n"
    "v_mfma_f32_16x16x32_bf16 v[116:119], v[228:231], v[204:207], v[116:119]\n"
    "v_mfma_f32_16x16x32_bf16 v[116:119], v[232:235], v[208:211], v[116:119]\n"
    "v_mfma_f32_16x16x32_bf16 v[52:55], v[236:239], v[204:207], v[52:55]\n"
    "v_mfma_f32_16x16x32_bf16 v[52:55], v[240:243], v[208:211], v[52:55]\n"
    "s_setprio 0\n"
    "s_waitcnt vmcnt(6)\n"
    "s_barrier\n"
    "; phase 108: step 68 slot 1 stage 3\n"
    "v_cmp_ne_u32 vcc, s90, v6\n"
    "s_cbranch_vccz .La141_%=\n"
    "s_memrealtime s[86:87]\n"
    "s_add_u32 s82, s56, 0x200000\n"
    ".La142_%=:\n"
    "s_sleep 2\n"
    "buffer_load_dword v6, v254, s[44:47], s82 offen sc0 sc1\n"
    "s_memrealtime s[88:89]\n"
    "s_waitcnt vmcnt(0) lgkmcnt(0)\n"
    "v_cmp_ne_u32 vcc, s90, v6\n"
    "s_cbranch_vccz .La141_%=\n"
    "s_sub_u32 s83, s88, s86\n"
    "s_cmp_lt_u32 s83, 0x2710\n"
    "s_cbranch_scc1 .La142_%=\n"
    "s_sleep 64\n"
    "buffer_load_dword v7, off, s[44:47], 0 sc0 sc1\n"
    "s_waitcnt vmcnt(0)\n"
    "v_cmp_eq_u32 vcc, s90, v7\n"
    "s_cbranch_vccnz .La143_%=\n"
    "s_cmp_lt_u32 s83, 0x2faf080\n"
    "s_cbranch_scc1 .La142_%=\n"
    ".La143_%=:\n"
    "v_mov_b32 v7, s90\n"
    "v_mov_b32 v32, 0x7fc00000\n"
    "v_mov_b32 v33, 0x7fc00000\n"
    "v_mov_b32 v34, 0x7fc00000\n"
    "v_mov_b32 v35, 0x7fc00000\n"
    "v_mov_b32 v40, 0x7fc00000\n"
    "v_mov_b32 v41, 0x7fc00000\n"
    "v_mov_b32 v42, 0x7fc00000\n"
    "v_mov_b32 v43, 0x7fc00000\n"
    "v_mov_b32 v12, 0x7fc00000\n"
    "v_mov_b32 v13, 0x7fc00000\n"
    "v_mov_b32 v14, 0x7fc00000\n"
    "v_mov_b32 v15, 0x7fc00000\n"
    "v_mov_b32 v48, 0x7fc00000\n"
    "v_mov_b32 v49, 0x7fc00000\n"
    "v_mov_b32 v50, 0x7fc00000\n"
    "v_mov_b32 v51, 0x7fc00000\n"
    "buffer_store_dword v7, off, s[44:47], 0 sc0 sc1\n"
    "s_waitcnt vmcnt(0)\n"
    ".La141_%=:\n"
    "ds_read_b128 v[180:183], v244 offset:32768\n"
    "ds_read_b128 v[184:187], v245 offset:32768\n"
    "s_add_u32 s82, s54, 0x104000\n"
    "buffer_load_dwordx4 v[56:59], v252, s[44:47], s82 offen sc0 sc1\n"
    "ds_read_b128 v[188:191], v244 offset:34816\n"
    "ds_read_b128 v[192:195], v245 offset:34816\n"
    "s_add_u32 s82, s54, 0x108000\n"
    "buffer_load_dwordx4 v[20:23], v252, s[44:47], s82 offen sc0 sc1\n"
    "ds_read_b128 v[196:199], v244 offset:40960\n"
    "ds_read_b128 v[200:203], v245 offset:40960\n"
    "s_add_u32 s82, s54, 0x10c000\n"
    "buffer_load_dwordx4 v[68:71], v252, s[44:47], s82 offen sc0 sc1\n"
    "ds_read_b128 v[204:207], v244 offset:43008\n"
    "ds_read_b128 v[208:211], v245 offset:43008\n"
    "s_add_u32 s82, s54, 0x106000\n"
    "buffer_load_dwordx4 v[132:135], v252, s[44:47], s82 offen sc0 sc1\n"
    "ds_read_b64_tr_b16 v[212:213], v246 offset:0\n"
    "ds_read_b64_tr_b16 v[214:215], v246 offset:1024\n"
    "s_add_u32 s82, s54, 0x10a000\n"
    "buffer_load_dwordx4 v[136:139], v252, s[44:47], s82 offen sc0 sc1\n"
    "ds_read_b64_tr_b16 v[216:217], v246 offset:8192\n"
    "ds_read_b64_tr_b16 v[218:219], v246 offset:9216\n"
    "s_add_u32 s82, s54, 0x10e000\n"
    "buffer_load_dwordx4 v[72:75], v252, s[44:47], s82 offen sc0 sc1\n"
    "ds_read_b64_tr_b16 v[220:221], v247 offset:0\n"
    "ds_read_b64_tr_b16 v[222:223], v247 offset:1024\n"
    "s_add_u32 m0, s50, 0x4000\n"
    "s_xor_b32 s82, s48, 0x202300\n"
    "buffer_load_dwordx4 v250, s[36:39], s82 offen  lds\n"
    "ds_read_b64_tr_b16 v[224:225], v247 offset:8192\n"
    "ds_read_b64_tr_b16 v[226:227], v247 offset:9216\n"
    "s_add_u32 m0, s50, 0x5000\n"
    "s_xor_b32 s82, s48, 0x282300\n"
    "buffer_load_dwordx4 v250, s[36:39], s82 offen  lds\n"
    "ds_read_b64_tr_b16 v[228:229], v248 offset:0\n"
    "ds_read_b64_tr_b16 v[230:231], v248 offset:1024\n"
    "s_add_u32 m0, s50, 0x6000\n"
    "s_xor_b32 s82, s48, 0x2300\n"
    "buffer_load_dwordx4 v250, s[36:39], s82 offen  lds\n"
    "ds_read_b64_tr_b16 v[232:233], v248 offset:8192\n"
    "ds_read_b64_tr_b16 v[234:235], v248 offset:9216\n"
    "s_add_u32 m0, s50, 0x7000\n"
    "s_xor_b32 s82, s48, 0x82300\n"
    "buffer_load_dwordx4 v250, s[36:39], s82 offen  lds\n"
    "ds_read_b64_tr_b16 v[236:237], v249 offset:0\n"
    "ds_read_b64_tr_b16 v[238:239], v249 offset:1024\n"
    "s_add_u32 m0, s51, 0xc000\n"
    "s_xor_b32 s82, s67, 0x4700000\n"
    "buffer_load_dwordx4 v251, s[40:43], s82 offen  lds\n"
    "ds_read_b64_tr_b16 v[240:241], v249 offset:8192\n"
    "ds_read_b64_tr_b16 v[242:243], v249 offset:9216\n"
    "s_add_u32 m0, s51, 0xe000\n"
    "s_xor_b32 s82, s67, 0x4780000\n"
    "buffer_load_dwordx4 v251, s[40:43], s82 offen  lds\n"
    "s_waitcnt lgkmcnt(0)\n"
    "s_barrier\n"
    "s_setprio 1\n"
    "v_mfma_f32_16x16x32_bf16 v[64:67], v[212:215], v[180:183], v[64:67]\n"
    "v_mfma_f32_16x16x32_bf16 v[64:67], v[216:219], v[184:187], v[64:67]\n"
    "v_mfma_f32_16x16x32_bf16 v[60:63], v[220:223], v[180:183], v[60:63]\n"
    "v_mfma_f32_16x16x32_bf16 v[60:63], v[224:227], v[184:187], v[60:63]\n"
    "v_mfma_f32_16x16x32_bf16 v[80:83], v[228:231], v[180:183], v[80:83]\n"
    "v_mfma_f32_16x16x32_bf16 v[80:83], v[232:235], v[184:187], v[80:83]\n"
    "v_mfma_f32_16x16x32_bf16 v[88:91], v[236:239], v[180:183], v[88:91]\n"
    "v_mfma_f32_16x16x32_bf16 v[88:91], v[240:243], v[184:187], v[88:91]\n"
    "v_mfma_f32_16x16x32_bf16 v[96:99], v[212:215], v[188:191], v[96:99]\n"
    "v_mfma_f32_16x16x32_bf16 v[96:99], v[216:219], v[192:195], v[96:99]\n"
    "v_mfma_f32_16x16x32_bf16 v[104:107], v[220:223], v[188:191], v[104:107]\n"
    "v_mfma_f32_16x16x32_bf16 v[104:107], v[224:227], v[192:195], v[104:107]\n"
    "v_mfma_f32_16x16x32_bf16 v[76:79], v[228:231], v[188:191], v[76:79]\n"
    "v_mfma_f32_16x16x32_bf16 v[76:79], v[232:235], v[192:195], v[76:79]\n"
    "v_mfma_f32_16x16x32_bf16 v[112:115], v[236:239], v[188:191], v[112:115]\n"
    "v_mfma_f32_16x16x32_bf16 v[112:115], v[240:243], v[192:195], v[112:115]\n"
    "v_mfma_f32_16x16x32_bf16 v[120:123], v[212:215], v[196:199], v[120:123]\n"
    "v_mfma_f32_16x16x32_bf16 v[120:123], v[216:219], v[200:203], v[120:123]\n"
    "v_mfma_f32_16x16x32_bf16 v[84:87], v[220:223], v[196:199], v[84:87]\n"
    "v_mfma_f32_16x16x32_bf16 v[84:87], v[224:227], v[200:203], v[84:87]\n"
    "v_mfma_f32_16x16x32_bf16 v[28:31], v[228:231], v[196:199], v[28:31]\n"
    "v_mfma_f32_16x16x32_bf16 v[28:31], v[232:235], v[200:203], v[28:31]\n"
    "v_mfma_f32_16x16x32_bf16 v[92:95], v[236:239], v[196:199], v[92:95]\n"
    "v_mfma_f32_16x16x32_bf16 v[92:95], v[240:243], v[200:203], v[92:95]\n"
    "v_mfma_f32_16x16x32_bf16 v[100:103], v[212:215], v[204:207], v[100:103]\n"
    "v_mfma_f32_16x16x32_bf16 v[100:103], v[216:219], v[208:211], v[100:103]\n"
    "v_mfma_f32_16x16x32_bf16 v[108:111], v[220:223], v[204:207], v[108:111]\n"
    "v_mfma_f32_16x16x32_bf16 v[108:111], v[224:227], v[208:211], v[108:111]\n"
    "v_mfma_f32_16x16x32_bf16 v[116:119], v[228:231], v[204:207], v[116:119]\n"
    "v_mfma_f32_16x16x32_bf16 v[116:119], v[232:235], v[208:211], v[116:119]\n"
    "v_mfma_f32_16x16x32_bf16 v[52:55], v[236:239], v[204:207], v[52:55]\n"
    "v_mfma_f32_16x16x32_bf16 v[52:55], v[240:243], v[208:211], v[52:55]\n"
    "s_setprio 0\n"
    "s_waitcnt vmcnt(14)\n"
    "s_barrier\n"
    "; phase 109: step 69 slot 1 stage 3\n"
    "ds_read_b128 v[180:183], v244 offset:0\n"
    "ds_read_b128 v[184:187], v245 offset:0\n"
    "ds_read_b128 v[188:191], v244 offset:2048\n"
    "ds_read_b128 v[192:195], v245 offset:2048\n"
    "s_add_u32 m0, s50, 0x8000\n"
    "s_xor_b32 s82, s48, 0x202380\n"
    "buffer_load_dwordx4 v250, s[36:39], s82 offen  lds\n"
    "ds_read_b128 v[196:199], v244 offset:8192\n"
    "ds_read_b128 v[200:203], v245 offset:8192\n"
    "ds_read_b128 v[204:207], v244 offset:10240\n"
    "ds_read_b128 v[208:211], v245 offset:10240\n"
    "s_add_u32 m0, s50, 0x9000\n"
    "s_xor_b32 s82, s48, 0x282380\n"
    "buffer_load_dwordx4 v250, s[36:39], s82 offen  lds\n"
    "ds_read_b64_tr_b16 v[212:213], v246 offset:16384\n"
    "ds_read_b64_tr_b16 v[214:215], v246 offset:17408\n"
    "ds_read_b64_tr_b16 v[216:217], v246 offset:24576\n"
    "ds_read_b64_tr_b16 v[218:219], v246 offset:25600\n"
    "s_add_u32 m0, s50, 0xa000\n"
    "s_xor_b32 s82, s48, 0x2380\n"
    "buffer_load_dwordx4 v250, s[36:39], s82 offen  lds\n"
    "ds_read_b64_tr_b16 v[220:221], v247 offset:16384\n"
    "ds_read_b64_tr_b16 v[222:223], v247 offset:17408\n"
    "ds_read_b64_tr_b16 v[224:225], v247 offset:24576\n"
    "ds_read_b64_tr_b16 v[226:227], v247 offset:25600\n"
    "s_add_u32 m0, s50, 0xb000\n"
    "s_xor_b32 s82, s48, 0x82380\n"
    "buffer_load_dwordx4 v250, s[36:39], s82 offen  lds\n"
    "ds_read_b64_tr_b16 v[228:229], v248 offset:16384\n"
    "ds_read_b64_tr_b16 v[230:231], v248 offset:17408\n"
    "ds_read_b64_tr_b16 v[232:233], v248 offset:24576\n"
    "ds_read_b64_tr_b16 v[234:235], v248 offset:25600\n"
    "s_add_u32 m0, s51, 0x0\n"
    "s_xor_b32 s82, s67, 0x4800000\n"
    "buffer_load_dwordx4 v251, s[40:43], s82 offen  lds\n"
    "ds_read_b64_tr_b16 v[236:237], v249 offset:16384\n"
    "ds_read_b64_tr_b16 v[238:239], v249 offset:17408\n"
    "ds_read_b64_tr_b16 v[240:241], v249 offset:24576\n"
    "ds_read_b64_tr_b16 v[242:243], v249 offset:25600\n"
    "s_add_u32 m0, s51, 0x2000\n"
    "s_xor_b32 s82, s67, 0x4880000\n"
    "buffer_load_dwordx4 v251, s[40:43], s82 offen  lds\n"
    "s_waitcnt vmcnt(12) lgkmcnt(0)\n"
    "s_barrier\n"
    "s_setprio 1\n"
    "v_mfma_f32_16x16x32_bf16 v[64:67], v[212:215], v[180:183], v[64:67]\n"
    "v_mfma_f32_16x16x32_bf16 v[12:15], v[2:5], v[56:59], v[12:15]\n"
    "v_mfma_f32_16x16x32_bf16 v[64:67], v[216:219], v[184:187], v[64:67]\n"
    "v_mfma_f32_16x16x32_bf16 v[48:51], v[0:3], v[56:59], v[48:51]\n"
    "v_mfma_f32_16x16x32_bf16 v[60:63], v[220:223], v[180:183], v[60:63]\n"
    "v_mfma_f32_16x16x32_bf16 v[12:15], v[2:5], v[20:23], v[12:15]\n"
    "v_mfma_f32_16x16x32_bf16 v[60:63], v[224:227], v[184:187], v[60:63]\n"
    "v_mfma_f32_16x16x32_bf16 v[48:51], v[0:3], v[20:23], v[48:51]\n"
    "v_mfma_f32_16x16x32_bf16 v[80:83], v[228:231], v[180:183], v[80:83]\n"
    "v_mfma_f32_16x16x32_bf16 v[12:15], v[2:5], v[68:71], v[12:15]\n"
    "v_mfma_f32_16x16x32_bf16 v[80:83], v[232:235], v[184:187], v[80:83]\n"
    "v_mfma_f32_16x16x32_bf16 v[48:51], v[0:3], v[68:71], v[48:51]\n"
    "v_mfma_f32_16x16x32_bf16 v[88:91], v[236:239], v[180:183], v[88:91]\n"
    "v_mfma_f32_16x16x32_bf16 v[32:35], v[2:5], v[132:135], v[32:35]\n"
    "v_mfma_f32_16x16x32_bf16 v[88:91], v[240:243], v[184:187], v[88:91]\n"
    "v_mfma_f32_16x16x32_bf16 v[40:43], v[0:3], v[132:135], v[40:43]\n"
    "v_mfma_f32_16x16x32_bf16 v[96:99], v[212:215], v[188:191], v[96:99]\n"
    "v_mfma_f32_16x16x32_bf16 v[32:35], v[2:5], v[136:139], v[32:35]\n"
    "v_mfma_f32_16x16x32_bf16 v[96:99], v[216:219], v[192:195], v[96:99]\n"
    "v_mfma_f32_16x16x32_bf16 v[40:43], v[0:3], v[136:139], v[40:43]\n"
    "v_mfma_f32_16x16x32_bf16 v[104:107], v[220:223], v[188:191], v[104:107]\n"
    "v_mfma_f32_16x16x32_bf16 v[32:35], v[2:5], v[72:75], v[32:35]\n"
    "v_mfma_f32_16x16x32_bf16 v[104:107], v[224:227], v[192:195], v[104:107]\n"
    "v_mfma_f32_16x16x32_bf16 v[40:43], v[0:3], v[72:75], v[40:43]\n"
    "v_mfma_f32_16x16x32_bf16 v[76:79], v[228:231], v[188:191], v[76:79]\n"
    "v_med3_f32 v68, v12, -s62, s62\n"
    "v_med3_f32 v69, v13, -s62, s62\n"
    "v_mfma_f32_16x16x32_bf16 v[76:79], v[232:235], v[192:195], v[76:79]\n"
    "v_med3_f32 v70, v14, -s62, s62\n"
    "v_med3_f32 v71, v15, -s62, s62\n"
    "v_mfma_f32_16x16x32_bf16 v[112:115], v[236:239], v[188:191], v[112:115]\n"
    "v_mul_f32 v68, s63, v68\n"
    "v_mul_f32 v69, s63, v69\n"
    "v_mfma_f32_16x16x32_bf16 v[112:115], v[240:243], v[192:195], v[112:115]\n"
    "v_mul_f32 v70, s63, v70\n"
    "v_mul_f32 v71, s63, v71\n"
    "v_mfma_f32_16x16x32_bf16 v[120:123], v[212:215], v[196:199], v[120:123]\n"
    "v_exp_f32 v68, v68\n"
    "v_mfma_f32_16x16x32_bf16 v[120:123], v[216:219], v[200:203], v[120:123]\n"
    "v_exp_f32 v69, v69\n"
    "v_mfma_f32_16x16x32_bf16 v[84:87], v[220:223], v[196:199], v[84:87]\n"
    "v_exp_f32 v70, v70\n"
    "v_mfma_f32_16x16x32_bf16 v[84:87], v[224:227], v[200:203], v[84:87]\n"
    "v_exp_f32 v71, v71\n"
    "v_mfma_f32_16x16x32_bf16 v[28:31], v[228:231], v[196:199], v[28:31]\n"
    "v_add_f32 v68, 1.0, v68\n"
    "v_add_f32 v69, 1.0, v69\n"
    "v_mfma_f32_16x16x32_bf16 v[28:31], v[232:235], v[200:203], v[28:31]\n"
    "v_add_f32 v70, 1.0, v70\n"
    "v_add_f32 v71, 1.0, v71\n"
    "v_mfma_f32_16x16x32_bf16 v[92:95], v[236:239], v[196:199], v[92:95]\n"
    "v_rcp_f32 v68, v68\n"
    "v_mfma_f32_16x16x32_bf16 v[92:95], v[240:243], v[200:203], v[92:95]\n"
    "v_rcp_f32 v69, v69\n"
    "v_mfma_f32_16x16x32_bf16 v[100:103], v[212:215], v[204:207], v[100:103]\n"
    "v_rcp_f32 v70, v70\n"
    "v_mfma_f32_16x16x32_bf16 v[100:103], v[216:219], v[208:211], v[100:103]\n"
    "v_rcp_f32 v71, v71\n"
    "v_mfma_f32_16x16x32_bf16 v[108:111], v[220:223], v[204:207], v[108:111]\n"
    "v_mul_f32 v12, v12, v68\n"
    "v_mul_f32 v13, v13, v69\n"
    "v_mfma_f32_16x16x32_bf16 v[108:111], v[224:227], v[208:211], v[108:111]\n"
    "v_mul_f32 v14, v14, v70\n"
    "v_mul_f32 v15, v15, v71\n"
    "v_mfma_f32_16x16x32_bf16 v[116:119], v[228:231], v[204:207], v[116:119]\n"
    "v_med3_f32 v68, v48, -s62, s62\n"
    "v_med3_f32 v69, v49, -s62, s62\n"
    "v_mfma_f32_16x16x32_bf16 v[116:119], v[232:235], v[208:211], v[116:119]\n"
    "v_med3_f32 v70, v50, -s62, s62\n"
    "v_med3_f32 v71, v51, -s62, s62\n"
    "v_mfma_f32_16x16x32_bf16 v[52:55], v[236:239], v[204:207], v[52:55]\n"
    "v_mul_f32 v68, s63, v68\n"
    "v_mul_f32 v69, s63, v69\n"
    "v_mfma_f32_16x16x32_bf16 v[52:55], v[240:243], v[208:211], v[52:55]\n"
    "s_setprio 0\n"
    "s_waitcnt vmcnt(8)\n"
    "s_barrier\n"
    "; phase 110: step 70 slot 1 stage 3\n"
    "ds_read_b128 v[180:183], v244 offset:16384\n"
    "ds_read_b128 v[184:187], v245 offset:16384\n"
    "ds_read_b128 v[188:191], v244 offset:18432\n"
    "ds_read_b128 v[192:195], v245 offset:18432\n"
    "s_add_u32 m0, s50, 0x0\n"
    "s_xor_b32 s82, s48, 0x202400\n"
    "buffer_load_dwordx4 v250, s[36:39], s82 offen  lds\n"
    "ds_read_b128 v[196:199], v244 offset:24576\n"
    "ds_read_b128 v[200:203], v245 offset:24576\n"
    "ds_read_b128 v[204:207], v244 offset:26624\n"
    "ds_read_b128 v[208:211], v245 offset:26624\n"
    "s_add_u32 m0, s50, 0x1000\n"
    "s_xor_b32 s82, s48, 0x282400\n"
    "buffer_load_dwordx4 v250, s[36:39], s82 offen  lds\n"
    "ds_read_b64_tr_b16 v[212:213], v246 offset:32768\n"
    "ds_read_b64_tr_b16 v[214:215], v246 offset:33792\n"
    "ds_read_b64_tr_b16 v[216:217], v246 offset:40960\n"
    "ds_read_b64_tr_b16 v[218:219], v246 offset:41984\n"
    "s_add_u32 m0, s50, 0x2000\n"
    "s_xor_b32 s82, s48, 0x2400\n"
    "buffer_load_dwordx4 v250, s[36:39], s82 offen  lds\n"
    "ds_read_b64_tr_b16 v[220:221], v247 offset:32768\n"
    "ds_read_b64_tr_b16 v[222:223], v247 offset:33792\n"
    "ds_read_b64_tr_b16 v[224:225], v247 offset:40960\n"
    "ds_read_b64_tr_b16 v[226:227], v247 offset:41984\n"
    "s_add_u32 m0, s50, 0x3000\n"
    "s_xor_b32 s82, s48, 0x82400\n"
    "buffer_load_dwordx4 v250, s[36:39], s82 offen  lds\n"
    "ds_read_b64_tr_b16 v[228:229], v248 offset:32768\n"
    "ds_read_b64_tr_b16 v[230:231], v248 offset:33792\n"
    "ds_read_b64_tr_b16 v[232:233], v248 offset:40960\n"
    "ds_read_b64_tr_b16 v[234:235], v248 offset:41984\n"
    "s_add_u32 m0, s51, 0x4000\n"
    "s_xor_b32 s82, s67, 0x4900000\n"
    "buffer_load_dwordx4 v251, s[40:43], s82 offen  lds\n"
    "ds_read_b64_tr_b16 v[236:237], v249 offset:32768\n"
    "ds_read_b64_tr_b16 v[238:239], v249 offset:33792\n"
    "ds_read_b64_tr_b16 v[240:241], v249 offset:40960\n"
    "ds_read_b64_tr_b16 v[242:243], v249 offset:41984\n"
    "s_add_u32 m0, s51, 0x6000\n"
    "s_xor_b32 s82, s67, 0x4980000\n"
    "buffer_load_dwordx4 v251, s[40:43], s82 offen  lds\n"
    "s_waitcnt vmcnt(12) lgkmcnt(0)\n"
    "s_barrier\n"
    "s_setprio 1\n"
    "v_mfma_f32_16x16x32_bf16 v[64:67], v[212:215], v[180:183], v[64:67]\n"
    "v_mul_f32 v70, s63, v70\n"
    "v_mul_f32 v71, s63, v71\n"
    "v_mfma_f32_16x16x32_bf16 v[64:67], v[216:219], v[184:187], v[64:67]\n"
    "v_exp_f32 v68, v68\n"
    "v_mfma_f32_16x16x32_bf16 v[60:63], v[220:223], v[180:183], v[60:63]\n"
    "v_exp_f32 v69, v69\n"
    "v_mfma_f32_16x16x32_bf16 v[60:63], v[224:227], v[184:187], v[60:63]\n"
    "v_exp_f32 v70, v70\n"
    "v_mfma_f32_16x16x32_bf16 v[80:83], v[228:231], v[180:183], v[80:83]\n"
    "v_exp_f32 v71, v71\n"
    "v_mfma_f32_16x16x32_bf16 v[80:83], v[232:235], v[184:187], v[80:83]\n"
    "v_add_f32 v68, 1.0, v68\n"
    "v_add_f32 v69, 1.0, v69\n"
    "v_mfma_f32_16x16x32_bf16 v[88:91], v[236:239], v[180:183], v[88:91]\n"
    "v_add_f32 v70, 1.0, v70\n"
    "v_add_f32 v71, 1.0, v71\n"
    "v_mfma_f32_16x16x32_bf16 v[88:91], v[240:243], v[184:187], v[88:91]\n"
    "v_rcp_f32 v68, v68\n"
    "v_mfma_f32_16x16x32_bf16 v[96:99], v[212:215], v[188:191], v[96:99]\n"
    "v_rcp_f32 v69, v69\n"
    "v_mfma_f32_16x16x32_bf16 v[96:99], v[216:219], v[192:195], v[96:99]\n"
    "v_rcp_f32 v70, v70\n"
    "v_mfma_f32_16x16x32_bf16 v[104:107], v[220:223], v[188:191], v[104:107]\n"
    "v_rcp_f32 v71, v71\n"
    "v_mfma_f32_16x16x32_bf16 v[104:107], v[224:227], v[192:195], v[104:107]\n"
    "v_mul_f32 v48, v48, v68\n"
    "v_mul_f32 v49, v49, v69\n"
    "v_mfma_f32_16x16x32_bf16 v[76:79], v[228:231], v[188:191], v[76:79]\n"
    "v_mul_f32 v50, v50, v70\n"
    "v_mul_f32 v51, v51, v71\n"
    "v_mfma_f32_16x16x32_bf16 v[76:79], v[232:235], v[192:195], v[76:79]\n"
    "v_cvt_pk_bf16_f32 v68, v12, v13\n"
    "v_cvt_pk_bf16_f32 v69, v14, v15\n"
    "v_mfma_f32_16x16x32_bf16 v[112:115], v[236:239], v[188:191], v[112:115]\n"
    "v_cvt_pk_bf16_f32 v70, v48, v49\n"
    "v_cvt_pk_bf16_f32 v71, v50, v51\n"
    "v_mfma_f32_16x16x32_bf16 v[112:115], v[240:243], v[192:195], v[112:115]\n"
    "v_permlane16_swap_b32 v68, v70\n"
    "v_permlane16_swap_b32 v69, v71\n"
    "v_mfma_f32_16x16x32_bf16 v[120:123], v[212:215], v[196:199], v[120:123]\n"
    "v_med3_f32 v72, v32, -s62, s62\n"
    "v_med3_f32 v73, v33, -s62, s62\n"
    "v_mfma_f32_16x16x32_bf16 v[120:123], v[216:219], v[200:203], v[120:123]\n"
    "v_med3_f32 v74, v34, -s62, s62\n"
    "v_med3_f32 v75, v35, -s62, s62\n"
    "v_mfma_f32_16x16x32_bf16 v[84:87], v[220:223], v[196:199], v[84:87]\n"
    "v_mul_f32 v72, s63, v72\n"
    "v_mul_f32 v73, s63, v73\n"
    "v_mfma_f32_16x16x32_bf16 v[84:87], v[224:227], v[200:203], v[84:87]\n"
    "v_mul_f32 v74, s63, v74\n"
    "v_mul_f32 v75, s63, v75\n"
    "v_mfma_f32_16x16x32_bf16 v[28:31], v[228:231], v[196:199], v[28:31]\n"
    "v_exp_f32 v72, v72\n"
    "v_mfma_f32_16x16x32_bf16 v[28:31], v[232:235], v[200:203], v[28:31]\n"
    "v_exp_f32 v73, v73\n"
    "v_mfma_f32_16x16x32_bf16 v[92:95], v[236:239], v[196:199], v[92:95]\n"
    "v_exp_f32 v74, v74\n"
    "v_mfma_f32_16x16x32_bf16 v[92:95], v[240:243], v[200:203], v[92:95]\n"
    "v_exp_f32 v75, v75\n"
    "v_mfma_f32_16x16x32_bf16 v[100:103], v[212:215], v[204:207], v[100:103]\n"
    "v_add_f32 v72, 1.0, v72\n"
    "v_add_f32 v73, 1.0, v73\n"
    "v_mfma_f32_16x16x32_bf16 v[100:103], v[216:219], v[208:211], v[100:103]\n"
    "v_add_f32 v74, 1.0, v74\n"
    "v_add_f32 v75, 1.0, v75\n"
    "v_mfma_f32_16x16x32_bf16 v[108:111], v[220:223], v[204:207], v[108:111]\n"
    "v_rcp_f32 v72, v72\n"
    "v_mfma_f32_16x16x32_bf16 v[108:111], v[224:227], v[208:211], v[108:111]\n"
    "v_rcp_f32 v73, v73\n"
    "v_mfma_f32_16x16x32_bf16 v[116:119], v[228:231], v[204:207], v[116:119]\n"
    "v_rcp_f32 v74, v74\n"
    "v_mfma_f32_16x16x32_bf16 v[116:119], v[232:235], v[208:211], v[116:119]\n"
    "v_rcp_f32 v75, v75\n"
    "v_mfma_f32_16x16x32_bf16 v[52:55], v[236:239], v[204:207], v[52:55]\n"
    "v_mul_f32 v32, v32, v72\n"
    "v_mul_f32 v33, v33, v73\n"
    "v_mfma_f32_16x16x32_bf16 v[52:55], v[240:243], v[208:211], v[52:55]\n"
    "s_setprio 0\n"
    "s_waitcnt vmcnt(8)\n"
    "s_barrier\n"
    "; phase 111: step 71 slot 1 stage 3\n"
    "ds_read_b128 v[180:183], v244 offset:32768\n"
    "ds_read_b128 v[184:187], v245 offset:32768\n"
    "ds_read_b128 v[188:191], v244 offset:34816\n"
    "ds_read_b128 v[192:195], v245 offset:34816\n"
    "s_add_u32 m0, s50, 0x4000\n"
    "s_xor_b32 s82, s48, 0x202480\n"
    "buffer_load_dwordx4 v250, s[36:39], s82 offen  lds\n"
    "ds_read_b128 v[196:199], v244 offset:40960\n"
    "ds_read_b128 v[200:203], v245 offset:40960\n"
    "ds_read_b128 v[204:207], v244 offset:43008\n"
    "ds_read_b128 v[208:211], v245 offset:43008\n"
    "s_add_u32 m0, s50, 0x5000\n"
    "s_xor_b32 s82, s48, 0x282480\n"
    "buffer_load_dwordx4 v250, s[36:39], s82 offen  lds\n"
    "ds_read_b64_tr_b16 v[212:213], v246 offset:49152\n"
    "ds_read_b64_tr_b16 v[214:215], v246 offset:50176\n"
    "ds_read_b64_tr_b16 v[216:217], v246 offset:57344\n"
    "ds_read_b64_tr_b16 v[218:219], v246 offset:58368\n"
    "s_add_u32 m0, s50, 0x6000\n"
    "s_xor_b32 s82, s48, 0x2480\n"
    "buffer_load_dwordx4 v250, s[36:39], s82 offen  lds\n"
    "ds_read_b64_tr_b16 v[220:221], v247 offset:49152\n"
    "ds_read_b64_tr_b16 v[222:223], v247 offset:50176\n"
    "ds_read_b64_tr_b16 v[224:225], v247 offset:57344\n"
    "ds_read_b64_tr_b16 v[226:227], v247 offset:58368\n"
    "s_add_u32 m0, s50, 0x7000\n"
    "s_xor_b32 s82, s48, 0x82480\n"
    "buffer_load_dwordx4 v250, s[36:39], s82 offen  lds\n"
    "ds_read_b64_tr_b16 v[228:229], v248 offset:49152\n"
    "ds_read_b64_tr_b16 v[230:231], v248 offset:50176\n"
    "ds_read_b64_tr_b16 v[232:233], v248 offset:57344\n"
    "ds_read_b64_tr_b16 v[234:235], v248 offset:58368\n"
    "s_add_u32 m0, s51, 0x8000\n"
    "s_xor_b32 s82, s67, 0x4a00000\n"
    "buffer_load_dwordx4 v251, s[40:43], s82 offen  lds\n"
    "ds_read_b64_tr_b16 v[236:237], v249 offset:49152\n"
    "ds_read_b64_tr_b16 v[238:239], v249 offset:50176\n"
    "ds_read_b64_tr_b16 v[240:241], v249 offset:57344\n"
    "ds_read_b64_tr_b16 v[242:243], v249 offset:58368\n"
    "s_add_u32 m0, s51, 0xa000\n"
    "s_xor_b32 s82, s67, 0x4a80000\n"
    "buffer_load_dwordx4 v251, s[40:43], s82 offen  lds\n"
    "s_waitcnt vmcnt(12) lgkmcnt(0)\n"
    "s_barrier\n"
    "s_setprio 1\n"
    "v_mfma_f32_16x16x32_bf16 v[64:67], v[212:215], v[180:183], v[64:67]\n"
    "v_mul_f32 v34, v34, v74\n"
    "v_mul_f32 v35, v35, v75\n"
    "v_mfma_f32_16x16x32_bf16 v[64:67], v[216:219], v[184:187], v[64:67]\n"
    "v_med3_f32 v72, v40, -s62, s62\n"
    "v_med3_f32 v73, v41, -s62, s62\n"
    "v_mfma_f32_16x16x32_bf16 v[60:63], v[220:223], v[180:183], v[60:63]\n"
    "v_med3_f32 v74, v42, -s62, s62\n"
    "v_med3_f32 v75, v43, -s62, s62\n"
    "v_mfma_f32_16x16x32_bf16 v[60:63], v[224:227], v[184:187], v[60:63]\n"
    "v_mul_f32 v72, s63, v72\n"
    "v_mul_f32 v73, s63, v73\n"
    "v_mfma_f32_16x16x32_bf16 v[80:83], v[228:231], v[180:183], v[80:83]\n"
    "v_mul_f32 v74, s63, v74\n"
    "v_mul_f32 v75, s63, v75\n"
    "v_mfma_f32_16x16x32_bf16 v[80:83], v[232:235], v[184:187], v[80:83]\n"
    "v_exp_f32 v72, v72\n"
    "v_mfma_f32_16x16x32_bf16 v[88:91], v[236:239], v[180:183], v[88:91]\n"
    "v_exp_f32 v73, v73\n"
    "v_mfma_f32_16x16x32_bf16 v[88:91], v[240:243], v[184:187], v[88:91]\n"
    "v_exp_f32 v74, v74\n"
    "v_mfma_f32_16x16x32_bf16 v[96:99], v[212:215], v[188:191], v[96:99]\n"
    "v_exp_f32 v75, v75\n"
    "v_mfma_f32_16x16x32_bf16 v[96:99], v[216:219], v[192:195], v[96:99]\n"
    "v_add_f32 v72, 1.0, v72\n"
    "v_add_f32 v73, 1.0, v73\n"
    "v_mfma_f32_16x16x32_bf16 v[104:107], v[220:223], v[188:191], v[104:107]\n"
    "v_add_f32 v74, 1.0, v74\n"
    "v_add_f32 v75, 1.0, v75\n"
    "v_mfma_f32_16x16x32_bf16 v[104:107], v[224:227], v[192:195], v[104:107]\n"
    "v_rcp_f32 v72, v72\n"
    "v_mfma_f32_16x16x32_bf16 v[76:79], v[228:231], v[188:191], v[76:79]\n"
    "v_rcp_f32 v73, v73\n"
    "v_mfma_f32_16x16x32_bf16 v[76:79], v[232:235], v[192:195], v[76:79]\n"
    "v_rcp_f32 v74, v74\n"
    "v_mfma_f32_16x16x32_bf16 v[112:115], v[236:239], v[188:191], v[112:115]\n"
    "v_rcp_f32 v75, v75\n"
    "v_mfma_f32_16x16x32_bf16 v[112:115], v[240:243], v[192:195], v[112:115]\n"
    "v_mul_f32 v40, v40, v72\n"
    "v_mul_f32 v41, v41, v73\n"
    "v_mfma_f32_16x16x32_bf16 v[120:123], v[212:215], v[196:199], v[120:123]\n"
    "v_mul_f32 v42, v42, v74\n"
    "v_mul_f32 v43, v43, v75\n"
    "v_mfma_f32_16x16x32_bf16 v[120:123], v[216:219], v[200:203], v[120:123]\n"
    "v_cvt_pk_bf16_f32 v72, v32, v33\n"
    "v_cvt_pk_bf16_f32 v73, v34, v35\n"
    "v_mfma_f32_16x16x32_bf16 v[84:87], v[220:223], v[196:199], v[84:87]\n"
    "v_cvt_pk_bf16_f32 v74, v40, v41\n"
    "v_cvt_pk_bf16_f32 v75, v42, v43\n"
    "v_mfma_f32_16x16x32_bf16 v[84:87], v[224:227], v[200:203], v[84:87]\n"
    "v_permlane16_swap_b32 v72, v74\n"
    "v_permlane16_swap_b32 v73, v75\n"
    "v_mfma_f32_16x16x32_bf16 v[28:31], v[228:231], v[196:199], v[28:31]\n"
    "v_mov_b32 v32, v68\n"
    "v_mov_b32 v33, v69\n"
    "v_mfma_f32_16x16x32_bf16 v[28:31], v[232:235], v[200:203], v[28:31]\n"
    "v_mov_b32 v34, v70\n"
    "v_mov_b32 v35, v71\n"
    "v_mfma_f32_16x16x32_bf16 v[92:95], v[236:239], v[196:199], v[92:95]\n"
    "v_mov_b32_dpp v68, v72 row_ror:8 row_mask:0xf bank_mask:0xc\n"
    "v_mov_b32_dpp v69, v73 row_ror:8 row_mask:0xf bank_mask:0xc\n"
    "v_mfma_f32_16x16x32_bf16 v[92:95], v[240:243], v[200:203], v[92:95]\n"
    "v_mov_b32_dpp v70, v74 row_ror:8 row_mask:0xf bank_mask:0xc\n"
    "v_mov_b32_dpp v71, v75 row_ror:8 row_mask:0xf bank_mask:0xc\n"
    "v_mfma_f32_16x16x32_bf16 v[100:103], v[212:215], v[204:207], v[100:103]\n"
    "v_mov_b32_dpp v72, v32 row_ror:8 row_mask:0xf bank_mask:0x3\n"
    "v_mov_b32_dpp v73, v33 row_ror:8 row_mask:0xf bank_mask:0x3\n"
    "v_mfma_f32_16x16x32_bf16 v[100:103], v[216:219], v[208:211], v[100:103]\n"
    "v_mov_b32_dpp v74, v34 row_ror:8 row_mask:0xf bank_mask:0x3\n"
    "v_mov_b32_dpp v75, v35 row_ror:8 row_mask:0xf bank_mask:0x3\n"
    "v_mfma_f32_16x16x32_bf16 v[108:111], v[220:223], v[204:207], v[108:111]\n"
    "v_mfma_f32_16x16x32_bf16 v[108:111], v[224:227], v[208:211], v[108:111]\n"
    "v_mfma_f32_16x16x32_bf16 v[116:119], v[228:231], v[204:207], v[116:119]\n"
    "v_mfma_f32_16x16x32_bf16 v[116:119], v[232:235], v[208:211], v[116:119]\n"
    "v_mfma_f32_16x16x32_bf16 v[52:55], v[236:239], v[204:207], v[52:55]\n"
    "v_mfma_f32_16x16x32_bf16 v[52:55], v[240:243], v[208:211], v[52:55]\n"
    "s_setprio 0\n"
    "s_waitcnt vmcnt(8)\n"
    "s_barrier\n"
    "; phase 112: step 72 slot 1 stage 3\n"
    "ds_read_b128 v[180:183], v244 offset:0\n"
    "ds_read_b128 v[184:187], v245 offset:0\n"
    "ds_read_b128 v[188:191], v244 offset:2048\n"
    "s_cmp_lg_u32 s52, 0\n"
    "s_cbranch_scc1 .La144_%=\n"
    "buffer_store_dwordx4 v[68:71], v255, s[72:75], s80 offen nt\n"
    ".La144_%=:\n"
    "ds_read_b128 v[192:195], v245 offset:2048\n"
    "ds_read_b128 v[196:199], v244 offset:8192\n"
    "ds_read_b128 v[200:203], v245 offset:8192\n"
    "s_cmp_lg_u32 s52, 1\n"
    "s_cbranch_scc1 .La145_%=\n"
    "buffer_store_dwordx4 v[68:71], v255, s[72:75], s80 offen nt\n"
    ".La145_%=:\n"
    "ds_read_b128 v[204:207], v244 offset:10240\n"
    "ds_read_b128 v[208:211], v245 offset:10240\n"
    "ds_read_b64_tr_b16 v[212:213], v246 offset:0\n"
    "s_add_u32 m0, s50, 0x8000\n"
    "s_xor_b32 s82, s48, 0x202500\n"
    "buffer_load_dwordx4 v250, s[36:39], s82 offen  lds\n"
    "ds_read_b64_tr_b16 v[214:215], v246 offset:1024\n"
    "ds_read_b64_tr_b16 v[216:217], v246 offset:8192\n"
    "ds_read_b64_tr_b16 v[218:219], v246 offset:9216\n"
    "s_add_u32 m0, s50, 0x9000\n"
    "s_xor_b32 s82, s48, 0x282500\n"
    "buffer_load_dwordx4 v250, s[36:39], s82 offen  lds\n"
    "ds_read_b64_tr_b16 v[220:221], v247 offset:0\n"
    "ds_read_b64_tr_b16 v[222:223], v247 offset:1024\n"
    "ds_read_b64_tr_b16 v[224:225], v247 offset:8192\n"
    "s_add_u32 m0, s50, 0xa000\n"
    "s_xor_b32 s82, s48, 0x2500\n"
    "buffer_load_dwordx4 v250, s[36:39], s82 offen  lds\n"
    "ds_read_b64_tr_b16 v[226:227], v247 offset:9216\n"
    "ds_read_b64_tr_b16 v[228:229], v248 offset:0\n"
    "ds_read_b64_tr_b16 v[230:231], v248 offset:1024\n"
    "s_add_u32 m0, s50, 0xb000\n"
    "s_xor_b32 s82, s48, 0x82500\n"
    "buffer_load_dwordx4 v250, s[36:39], s82 offen  lds\n"
    "ds_read_b64_tr_b16 v[232:233], v248 offset:8192\n"
    "ds_read_b64_tr_b16 v[234:235], v248 offset:9216\n"
    "ds_read_b64_tr_b16 v[236:237], v249 offset:0\n"
    "s_add_u32 m0, s51, 0xc000\n"
    "s_xor_b32 s82, s67, 0x4b00000\n"
    "buffer_load_dwordx4 v251, s[40:43], s82 offen  lds\n"
    "ds_read_b64_tr_b16 v[238:239], v249 offset:1024\n"
    "ds_read_b64_tr_b16 v[240:241], v249 offset:8192\n"
    "ds_read_b64_tr_b16 v[242:243], v249 offset:9216\n"
    "s_add_u32 m0, s51, 0xe000\n"
    "s_xor_b32 s82, s67, 0x4b80000\n"
    "buffer_load_dwordx4 v251, s[40:43], s82 offen  lds\n"
    "s_waitcnt vmcnt(12) lgkmcnt(0)\n"
    "s_barrier\n"
    "s_setprio 1\n"
    "v_mfma_f32_16x16x32_bf16 v[64:67], v[212:215], v[180:183], v[64:67]\n"
    "v_mfma_f32_16x16x32_bf16 v[64:67], v[216:219], v[184:187], v[64:67]\n"
    "v_mfma_f32_16x16x32_bf16 v[60:63], v[220:223], v[180:183], v[60:63]\n"
    "v_mfma_f32_16x16x32_bf16 v[60:63], v[224:227], v[184:187], v[60:63]\n"
    "v_mfma_f32_16x16x32_bf16 v[80:83], v[228:231], v[180:183], v[80:83]\n"
    "v_mfma_f32_16x16x32_bf16 v[80:83], v[232:235], v[184:187], v[80:83]\n"
    "v_mfma_f32_16x16x32_bf16 v[88:91], v[236:239], v[180:183], v[88:91]\n"
    "v_mfma_f32_16x16x32_bf16 v[88:91], v[240:243], v[184:187], v[88:91]\n"
    "v_mfma_f32_16x16x32_bf16 v[96:99], v[212:215], v[188:191], v[96:99]\n"
    "v_mfma_f32_16x16x32_bf16 v[96:99], v[216:219], v[192:195], v[96:99]\n"
    "v_mfma_f32_16x16x32_bf16 v[104:107], v[220:223], v[188:191], v[104:107]\n"
    "v_mfma_f32_16x16x32_bf16 v[104:107], v[224:227], v[192:195], v[104:107]\n"
    "v_mfma_f32_16x16x32_bf16 v[76:79], v[228:231], v[188:191], v[76:79]\n"
    "v_mfma_f32_16x16x32_bf16 v[76:79], v[232:235], v[192:195], v[76:79]\n"
    "v_mfma_f32_16x16x32_bf16 v[112:115], v[236:239], v[188:191], v[112:115]\n"
    "v_mfma_f32_16x16x32_bf16 v[112:115], v[240:243], v[192:195], v[112:115]\n"
    "v_mfma_f32_16x16x32_bf16 v[120:123], v[212:215], v[196:199], v[120:123]\n"
    "v_mfma_f32_16x16x32_bf16 v[120:123], v[216:219], v[200:203], v[120:123]\n"
    "v_mfma_f32_16x16x32_bf16 v[84:87], v[220:223], v[196:199], v[84:87]\n"
    "v_mfma_f32_16x16x32_bf16 v[84:87], v[224:227], v[200:203], v[84:87]\n"
    "v_mfma_f32_16x16x32_bf16 v[28:31], v[228:231], v[196:199], v[28:31]\n"
    "v_mfma_f32_16x16x32_bf16 v[28:31], v[232:235], v[200:203], v[28:31]\n"
    "v_mfma_f32_16x16x32_bf16 v[92:95], v[236:239], v[196:199], v[92:95]\n"
    "v_mfma_f32_16x16x32_bf16 v[92:95], v[240:243], v[200:203], v[92:95]\n"
    "v_mfma_f32_16x16x32_bf16 v[100:103], v[212:215], v[204:207], v[100:103]\n"
    "v_mfma_f32_16x16x32_bf16 v[100:103], v[216:219], v[208:211], v[100:103]\n"
    "v_mfma_f32_16x16x32_bf16 v[108:111], v[220:223], v[204:207], v[108:111]\n"
    "v_mfma_f32_16x16x32_bf16 v[108:111], v[224:227], v[208:211], v[108:111]\n"
    "v_mfma_f32_16x16x32_bf16 v[116:119], v[228:231], v[204:207], v[116:119]\n"
    "v_mfma_f32_16x16x32_bf16 v[116:119], v[232:235], v[208:211], v[116:119]\n"
    "v_mfma_f32_16x16x32_bf16 v[52:55], v[236:239], v[204:207], v[52:55]\n"
    "v_mfma_f32_16x16x32_bf16 v[52:55], v[240:243], v[208:211], v[52:55]\n"
    "s_setprio 0\n"
    "s_waitcnt vmcnt(8)\n"
    "s_barrier\n"
    "; phase 113: step 73 slot 1 stage 3\n"
    "ds_read_b128 v[180:183], v244 offset:16384\n"
    "ds_read_b128 v[184:187], v245 offset:16384\n"
    "ds_read_b128 v[188:191], v244 offset:18432\n"
    "s_cmp_lg_u32 s52, 2\n"
    "s_cbranch_scc1 .La146_%=\n"
    "buffer_store_dwordx4 v[68:71], v255, s[72:75], s80 offen nt\n"
    ".La146_%=:\n"
    "ds_read_b128 v[192:195], v245 offset:18432\n"
    "ds_read_b128 v[196:199], v244 offset:24576\n"
    "ds_read_b128 v[200:203], v245 offset:24576\n"
    "s_add_u32 m0, s50, 0x0\n"
    "s_xor_b32 s82, s48, 0x202580\n"
    "buffer_load_dwordx4 v250, s[36:39], s82 offen  lds\n"
    "ds_read_b128 v[204:207], v244 offset:26624\n"
    "ds_read_b128 v[208:211], v245 offset:26624\n"
    "ds_read_b64_tr_b16 v[212:213], v246 offset:16384\n"
    "s_add_u32 m0, s50, 0x1000\n"
    "s_xor_b32 s82, s48, 0x282580\n"
    "buffer_load_dwordx4 v250, s[36:39], s82 offen  lds\n"
    "ds_read_b64_tr_b16 v[214:215], v246 offset:17408\n"
    "ds_read_b64_tr_b16 v[216:217], v246 offset:24576\n"
    "ds_read_b64_tr_b16 v[218:219], v246 offset:25600\n"
    "s_add_u32 m0, s50, 0x2000\n"
    "s_xor_b32 s82, s48, 0x2580\n"
    "buffer_load_dwordx4 v250, s[36:39], s82 offen  lds\n"
    "ds_read_b64_tr_b16 v[220:221], v247 offset:16384\n"
    "ds_read_b64_tr_b16 v[222:223], v247 offset:17408\n"
    "ds_read_b64_tr_b16 v[224:225], v247 offset:24576\n"
    "s_add_u32 m0, s50, 0x3000\n"
    "s_xor_b32 s82, s48, 0x82580\n"
    "buffer_load_dwordx4 v250, s[36:39], s82 offen  lds\n"
    "ds_read_b64_tr_b16 v[226:227], v247 offset:25600\n"
    "ds_read_b64_tr_b16 v[228:229], v248 offset:16384\n"
    "ds_read_b64_tr_b16 v[230:231], v248 offset:17408\n"
    "s_add_u32 m0, s51, 0x0\n"
    "s_xor_b32 s82, s67, 0x4c00000\n"
    "buffer_load_dwordx4 v251, s[40:43], s82 offen  lds\n"
    "ds_read_b64_tr_b16 v[232:233], v248 offset:24576\n"
    "ds_read_b64_tr_b16 v[234:235], v248 offset:25600\n"
    "ds_read_b64_tr_b16 v[236:237], v249 offset:16384\n"
    "s_add_u32 m0, s51, 0x2000\n"
    "s_xor_b32 s82, s67, 0x4c80000\n"
    "buffer_load_dwordx4 v251, s[40:43], s82 offen  lds\n"
    "ds_read_b64_tr_b16 v[238:239], v249 offset:17408\n"
    "ds_read_b64_tr_b16 v[240:241], v249 offset:24576\n"
    "ds_read_b64_tr_b16 v[242:243], v249 offset:25600\n"
    "s_waitcnt vmcnt(12) lgkmcnt(0)\n"
    "s_barrier\n"
    "s_setprio 1\n"
    "v_mfma_f32_16x16x32_bf16 v[64:67], v[212:215], v[180:183], v[64:67]\n"
    "v_mfma_f32_16x16x32_bf16 v[64:67], v[216:219], v[184:187], v[64:67]\n"
    "v_mfma_f32_16x16x32_bf16 v[60:63], v[220:223], v[180:183], v[60:63]\n"
    "v_mfma_f32_16x16x32_bf16 v[60:63], v[224:227], v[184:187], v[60:63]\n"
    "v_mfma_f32_16x16x32_bf16 v[80:83], v[228:231], v[180:183], v[80:83]\n"
    "v_mfma_f32_16x16x32_bf16 v[80:83], v[232:235], v[184:187], v[80:83]\n"
    "v_mfma_f32_16x16x32_bf16 v[88:91], v[236:239], v[180:183], v[88:91]\n"
    "v_mfma_f32_16x16x32_bf16 v[88:91], v[240:243], v[184:187], v[88:91]\n"
    "v_mfma_f32_16x16x32_bf16 v[96:99], v[212:215], v[188:191], v[96:99]\n"
    "v_mfma_f32_16x16x32_bf16 v[96:99], v[216:219], v[192:195], v[96:99]\n"
    "v_mfma_f32_16x16x32_bf16 v[104:107], v[220:223], v[188:191], v[104:107]\n"
    "v_mfma_f32_16x16x32_bf16 v[104:107], v[224:227], v[192:195], v[104:107]\n"
    "v_mfma_f32_16x16x32_bf16 v[76:79], v[228:231], v[188:191], v[76:79]\n"
    "v_mfma_f32_16x16x32_bf16 v[76:79], v[232:235], v[192:195], v[76:79]\n"
    "v_mfma_f32_16x16x32_bf16 v[112:115], v[236:239], v[188:191], v[112:115]\n"
    "v_mfma_f32_16x16x32_bf16 v[112:115], v[240:243], v[192:195], v[112:115]\n"
    "v_mfma_f32_16x16x32_bf16 v[120:123], v[212:215], v[196:199], v[120:123]\n"
    "v_mfma_f32_16x16x32_bf16 v[120:123], v[216:219], v[200:203], v[120:123]\n"
    "v_mfma_f32_16x16x32_bf16 v[84:87], v[220:223], v[196:199], v[84:87]\n"
    "v_mfma_f32_16x16x32_bf16 v[84:87], v[224:227], v[200:203], v[84:87]\n"
    "v_mfma_f32_16x16x32_bf16 v[28:31], v[228:231], v[196:199], v[28:31]\n"
    "v_mfma_f32_16x16x32_bf16 v[28:31], v[232:235], v[200:203], v[28:31]\n"
    "v_mfma_f32_16x16x32_bf16 v[92:95], v[236:239], v[196:199], v[92:95]\n"
    "v_mfma_f32_16x16x32_bf16 v[92:95], v[240:243], v[200:203], v[92:95]\n"
    "v_mfma_f32_16x16x32_bf16 v[100:103], v[212:215], v[204:207], v[100:103]\n"
    "v_mfma_f32_16x16x32_bf16 v[100:103], v[216:219], v[208:211], v[100:103]\n"
    "v_mfma_f32_16x16x32_bf16 v[108:111], v[220:223], v[204:207], v[108:111]\n"
    "v_mfma_f32_16x16x32_bf16 v[108:111], v[224:227], v[208:211], v[108:111]\n"
    "v_mfma_f32_16x16x32_bf16 v[116:119], v[228:231], v[204:207], v[116:119]\n"
    "v_mfma_f32_16x16x32_bf16 v[116:119], v[232:235], v[208:211], v[116:119]\n"
    "v_mfma_f32_16x16x32_bf16 v[52:55], v[236:239], v[204:207], v[52:55]\n"
    "v_mfma_f32_16x16x32_bf16 v[52:55], v[240:243], v[208:211], v[52:55]\n"
    "s_setprio 0\n"
    "s_waitcnt vmcnt(8)\n"
    "s_barrier\n"
    "; phase 114: step 74 slot 1 stage 3\n"
    "ds_read_b128 v[180:183], v244 offset:32768\n"
    "ds_read_b128 v[184:187], v245 offset:32768\n"
    "ds_read_b128 v[188:191], v244 offset:34816\n"
    "s_cmp_lg_u32 s52, 3\n"
    "s_cbranch_scc1 .La147_%=\n"
    "buffer_store_dwordx4 v[68:71], v255, s[72:75], s80 offen nt\n"
    ".La147_%=:\n"
    "ds_read_b128 v[192:195], v245 offset:34816\n"
    "ds_read_b128 v[196:199], v244 offset:40960\n"
    "ds_read_b128 v[200:203], v245 offset:40960\n"
    "s_add_u32 m0, s50, 0x4000\n"
    "s_xor_b32 s82, s48, 0x202600\n"
    "buffer_load_dwordx4 v250, s[36:39], s82 offen  lds\n"
    "ds_read_b128 v[204:207], v244 offset:43008\n"
    "ds_read_b128 v[208:211], v245 offset:43008\n"
    "ds_read_b64_tr_b16 v[212:213], v246 offset:32768\n"
    "s_add_u32 m0, s50, 0x5000\n"
    "s_xor_b32 s82, s48, 0x282600\n"
    "buffer_load_dwordx4 v250, s[36:39], s82 offen  lds\n"
    "ds_read_b64_tr_b16 v[214:215], v246 offset:33792\n"
    "ds_read_b64_tr_b16 v[216:217], v246 offset:40960\n"
    "ds_read_b64_tr_b16 v[218:219], v246 offset:41984\n"
    "s_add_u32 m0, s50, 0x6000\n"
    "s_xor_b32 s82, s48, 0x2600\n"
    "buffer_load_dwordx4 v250, s[36:39], s82 offen  lds\n"
    "ds_read_b64_tr_b16 v[220:221], v247 offset:32768\n"
    "ds_read_b64_tr_b16 v[222:223], v247 offset:33792\n"
    "ds_read_b64_tr_b16 v[224:225], v247 offset:40960\n"
    "s_add_u32 m0, s50, 0x7000\n"
    "s_xor_b32 s82, s48, 0x82600\n"
    "buffer_load_dwordx4 v250, s[36:39], s82 offen  lds\n"
    "ds_read_b64_tr_b16 v[226:227], v247 offset:41984\n"
    "ds_read_b64_tr_b16 v[228:229], v248 offset:32768\n"
    "ds_read_b64_tr_b16 v[230:231], v248 offset:33792\n"
    "s_add_u32 m0, s51, 0x4000\n"
    "s_xor_b32 s82, s67, 0x4d00000\n"
    "buffer_load_dwordx4 v251, s[40:43], s82 offen  lds\n"
    "ds_read_b64_tr_b16 v[232:233], v248 offset:40960\n"
    "ds_read_b64_tr_b16 v[234:235], v248 offset:41984\n"
    "ds_read_b64_tr_b16 v[236:237], v249 offset:32768\n"
    "s_add_u32 m0, s51, 0x6000\n"
    "s_xor_b32 s82, s67, 0x4d80000\n"
    "buffer_load_dwordx4 v251, s[40:43], s82 offen  lds\n"
    "ds_read_b64_tr_b16 v[238:239], v249 offset:33792\n"
    "ds_read_b64_tr_b16 v[240:241], v249 offset:40960\n"
    "ds_read_b64_tr_b16 v[242:243], v249 offset:41984\n"
    "s_waitcnt vmcnt(12) lgkmcnt(0)\n"
    "s_barrier\n"
    "s_setprio 1\n"
    "v_mfma_f32_16x16x32_bf16 v[64:67], v[212:215], v[180:183], v[64:67]\n"
    "v_mfma_f32_16x16x32_bf16 v[64:67], v[216:219], v[184:187], v[64:67]\n"
    "v_mfma_f32_16x16x32_bf16 v[60:63], v[220:223], v[180:183], v[60:63]\n"
    "v_mfma_f32_16x16x32_bf16 v[60:63], v[224:227], v[184:187], v[60:63]\n"
    "v_mfma_f32_16x16x32_bf16 v[80:83], v[228:231], v[180:183], v[80:83]\n"
    "v_mfma_f32_16x16x32_bf16 v[80:83], v[232:235], v[184:187], v[80:83]\n"
    "v_mfma_f32_16x16x32_bf16 v[88:91], v[236:239], v[180:183], v[88:91]\n"
    "v_mfma_f32_16x16x32_bf16 v[88:91], v[240:243], v[184:187], v[88:91]\n"
    "v_mfma_f32_16x16x32_bf16 v[96:99], v[212:215], v[188:191], v[96:99]\n"
    "v_mfma_f32_16x16x32_bf16 v[96:99], v[216:219], v[192:195], v[96:99]\n"
    "v_mfma_f32_16x16x32_bf16 v[104:107], v[220:223], v[188:191], v[104:107]\n"
    "v_mfma_f32_16x16x32_bf16 v[104:107], v[224:227], v[192:195], v[104:107]\n"
    "v_mfma_f32_16x16x32_bf16 v[76:79], v[228:231], v[188:191], v[76:79]\n"
    "v_mfma_f32_16x16x32_bf16 v[76:79], v[232:235], v[192:195], v[76:79]\n"
    "v_mfma_f32_16x16x32_bf16 v[112:115], v[236:239], v[188:191], v[112:115]\n"
    "v_mfma_f32_16x16x32_bf16 v[112:115], v[240:243], v[192:195], v[112:115]\n"
    "v_mfma_f32_16x16x32_bf16 v[120:123], v[212:215], v[196:199], v[120:123]\n"
    "v_mfma_f32_16x16x32_bf16 v[120:123], v[216:219], v[200:203], v[120:123]\n"
    "v_mfma_f32_16x16x32_bf16 v[84:87], v[220:223], v[196:199], v[84:87]\n"
    "v_mfma_f32_16x16x32_bf16 v[84:87], v[224:227], v[200:203], v[84:87]\n"
    "v_mfma_f32_16x16x32_bf16 v[28:31], v[228:231], v[196:199], v[28:31]\n"
    "v_mfma_f32_16x16x32_bf16 v[28:31], v[232:235], v[200:203], v[28:31]\n"
    "v_mfma_f32_16x16x32_bf16 v[92:95], v[236:239], v[196:199], v[92:95]\n"
    "v_mfma_f32_16x16x32_bf16 v[92:95], v[240:243], v[200:203], v[92:95]\n"
    "v_mfma_f32_16x16x32_bf16 v[100:103], v[212:215], v[204:207], v[100:103]\n"
    "v_mfma_f32_16x16x32_bf16 v[100:103], v[216:219], v[208:211], v[100:103]\n"
    "v_mfma_f32_16x16x32_bf16 v[108:111], v[220:223], v[204:207], v[108:111]\n"
    "v_mfma_f32_16x16x32_bf16 v[108:111], v[224:227], v[208:211], v[108:111]\n"
    "v_mfma_f32_16x16x32_bf16 v[116:119], v[228:231], v[204:207], v[116:119]\n"
    "v_mfma_f32_16x16x32_bf16 v[116:119], v[232:235], v[208:211], v[116:119]\n"
    "v_mfma_f32_16x16x32_bf16 v[52:55], v[236:239], v[204:207], v[52:55]\n"
    "v_mfma_f32_16x16x32_bf16 v[52:55], v[240:243], v[208:211], v[52:55]\n"
    "s_setprio 0\n"
    "s_waitcnt vmcnt(8)\n"
    "s_barrier\n"
    "; phase 115: step 75 slot 1 stage 3\n"
    "ds_read_b128 v[180:183], v244 offset:0\n"
    "ds_read_b128 v[184:187], v245 offset:0\n"
    "ds_read_b128 v[188:191], v244 offset:2048\n"
    "s_cmp_lg_u32 s52, 4\n"
    "s_cbranch_scc1 .La148_%=\n"
    "buffer_store_dwordx4 v[68:71], v255, s[72:75], s80 offen nt\n"
    ".La148_%=:\n"
    "ds_read_b128 v[192:195], v245 offset:2048\n"
    "ds_read_b128 v[196:199], v244 offset:8192\n"
    "ds_read_b128 v[200:203], v245 offset:8192\n"
    "s_add_u32 m0, s50, 0x8000\n"
    "s_xor_b32 s82, s48, 0x202680\n"
    "buffer_load_dwordx4 v250, s[36:39], s82 offen  lds\n"
    "ds_read_b128 v[204:207], v244 offset:10240\n"
    "ds_read_b128 v[208:211], v245 offset:10240\n"
    "ds_read_b64_tr_b16 v[212:213], v246 offset:49152\n"
    "s_add_u32 m0, s50, 0x9000\n"
    "s_xor_b32 s82, s48, 0x282680\n"
    "buffer_load_dwordx4 v250, s[36:39], s82 offen  lds\n"
    "ds_read_b64_tr_b16 v[214:215], v246 offset:50176\n"
    "ds_read_b64_tr_b16 v[216:217], v246 offset:57344\n"
    "ds_read_b64_tr_b16 v[218:219], v246 offset:58368\n"
    "s_add_u32 m0, s50, 0xa000\n"
    "s_xor_b32 s82, s48, 0x2680\n"
    "buffer_load_dwordx4 v250, s[36:39], s82 offen  lds\n"
    "ds_read_b64_tr_b16 v[220:221], v247 offset:49152\n"
    "ds_read_b64_tr_b16 v[222:223], v247 offset:50176\n"
    "ds_read_b64_tr_b16 v[224:225], v247 offset:57344\n"
    "s_add_u32 m0, s50, 0xb000\n"
    "s_xor_b32 s82, s48, 0x82680\n"
    "buffer_load_dwordx4 v250, s[36:39], s82 offen  lds\n"
    "ds_read_b64_tr_b16 v[226:227], v247 offset:58368\n"
    "ds_read_b64_tr_b16 v[228:229], v248 offset:49152\n"
    "ds_read_b64_tr_b16 v[230:231], v248 offset:50176\n"
    "s_add_u32 m0, s51, 0x8000\n"
    "s_xor_b32 s82, s67, 0x4e00000\n"
    "buffer_load_dwordx4 v251, s[40:43], s82 offen  lds\n"
    "ds_read_b64_tr_b16 v[232:233], v248 offset:57344\n"
    "ds_read_b64_tr_b16 v[234:235], v248 offset:58368\n"
    "ds_read_b64_tr_b16 v[236:237], v249 offset:49152\n"
    "s_add_u32 m0, s51, 0xa000\n"
    "s_xor_b32 s82, s67, 0x4e80000\n"
    "buffer_load_dwordx4 v251, s[40:43], s82 offen  lds\n"
    "ds_read_b64_tr_b16 v[238:239], v249 offset:50176\n"
    "ds_read_b64_tr_b16 v[240:241], v249 offset:57344\n"
    "ds_read_b64_tr_b16 v[242:243], v249 offset:58368\n"
    "s_waitcnt vmcnt(12) lgkmcnt(0)\n"
    "s_barrier\n"
    "s_setprio 1\n"
    "v_mfma_f32_16x16x32_bf16 v[64:67], v[212:215], v[180:183], v[64:67]\n"
    "v_mfma_f32_16x16x32_bf16 v[64:67], v[216:219], v[184:187], v[64:67]\n"
    "v_mfma_f32_16x16x32_bf16 v[60:63], v[220:223], v[180:183], v[60:63]\n"
    "v_mfma_f32_16x16x32_bf16 v[60:63], v[224:227], v[184:187], v[60:63]\n"
    "v_mfma_f32_16x16x32_bf16 v[80:83], v[228:231], v[180:183], v[80:83]\n"
    "v_mfma_f32_16x16x32_bf16 v[80:83], v[232:235], v[184:187], v[80:83]\n"
    "v_mfma_f32_16x16x32_bf16 v[88:91], v[236:239], v[180:183], v[88:91]\n"
    "v_mfma_f32_16x16x32_bf16 v[88:91], v[240:243], v[184:187], v[88:91]\n"
    "v_mfma_f32_16x16x32_bf16 v[96:99], v[212:215], v[188:191], v[96:99]\n"
    "v_mfma_f32_16x16x32_bf16 v[96:99], v[216:219], v[192:195], v[96:99]\n"
    "v_mfma_f32_16x16x32_bf16 v[104:107], v[220:223], v[188:191], v[104:107]\n"
    "v_mfma_f32_16x16x32_bf16 v[104:107], v[224:227], v[192:195], v[104:107]\n"
    "v_mfma_f32_16x16x32_bf16 v[76:79], v[228:231], v[188:191], v[76:79]\n"
    "v_mfma_f32_16x16x32_bf16 v[76:79], v[232:235], v[192:195], v[76:79]\n"
    "v_mfma_f32_16x16x32_bf16 v[112:115], v[236:239], v[188:191], v[112:115]\n"
    "v_mfma_f32_16x16x32_bf16 v[112:115], v[240:243], v[192:195], v[112:115]\n"
    "v_mfma_f32_16x16x32_bf16 v[120:123], v[212:215], v[196:199], v[120:123]\n"
    "v_mfma_f32_16x16x32_bf16 v[120:123], v[216:219], v[200:203], v[120:123]\n"
    "v_mfma_f32_16x16x32_bf16 v[84:87], v[220:223], v[196:199], v[84:87]\n"
    "v_mfma_f32_16x16x32_bf16 v[84:87], v[224:227], v[200:203], v[84:87]\n"
    "v_mfma_f32_16x16x32_bf16 v[28:31], v[228:231], v[196:199], v[28:31]\n"
    "v_mfma_f32_16x16x32_bf16 v[28:31], v[232:235], v[200:203], v[28:31]\n"
    "v_mfma_f32_16x16x32_bf16 v[92:95], v[236:239], v[196:199], v[92:95]\n"
    "v_mfma_f32_16x16x32_bf16 v[92:95], v[240:243], v[200:203], v[92:95]\n"
    "v_mfma_f32_16x16x32_bf16 v[100:103], v[212:215], v[204:207], v[100:103]\n"
    "v_mfma_f32_16x16x32_bf16 v[100:103], v[216:219], v[208:211], v[100:103]\n"
    "v_mfma_f32_16x16x32_bf16 v[108:111], v[220:223], v[204:207], v[108:111]\n"
    "v_mfma_f32_16x16x32_bf16 v[108:111], v[224:227], v[208:211], v[108:111]\n"
    "v_mfma_f32_16x16x32_bf16 v[116:119], v[228:231], v[204:207], v[116:119]\n"
    "v_mfma_f32_16x16x32_bf16 v[116:119], v[232:235], v[208:211], v[116:119]\n"
    "v_mfma_f32_16x16x32_bf16 v[52:55], v[236:239], v[204:207], v[52:55]\n"
    "v_mfma_f32_16x16x32_bf16 v[52:55], v[240:243], v[208:211], v[52:55]\n"
    "s_setprio 0\n"
    "s_waitcnt vmcnt(8)\n"
    "s_barrier\n"
    "; phase 116: step 76 slot 1 stage 3\n"
    "ds_read_b128 v[180:183], v244 offset:16384\n"
    "ds_read_b128 v[184:187], v245 offset:16384\n"
    "ds_read_b128 v[188:191], v244 offset:18432\n"
    "s_cmp_lg_u32 s52, 5\n"
    "s_cbranch_scc1 .La149_%=\n"
    "buffer_store_dwordx4 v[68:71], v255, s[72:75], s80 offen nt\n"
    ".La149_%=:\n"
    "ds_read_b128 v[192:195], v245 offset:18432\n"
    "ds_read_b128 v[196:199], v244 offset:24576\n"
    "ds_read_b128 v[200:203], v245 offset:24576\n"
    "s_add_u32 m0, s50, 0x0\n"
    "s_xor_b32 s82, s48, 0x202700\n"
    "buffer_load_dwordx4 v250, s[36:39], s82 offen  lds\n"
    "ds_read_b128 v[204:207], v244 offset:26624\n"
    "ds_read_b128 v[208:211], v245 offset:26624\n"
    "ds_read_b64_tr_b16 v[212:213], v246 offset:0\n"
    "s_add_u32 m0, s50, 0x1000\n"
    "s_xor_b32 s82, s48, 0x282700\n"
    "buffer_load_dwordx4 v250, s[36:39], s82 offen  lds\n"
    "ds_read_b64_tr_b16 v[214:215], v246 offset:1024\n"
    "ds_read_b64_tr_b16 v[216:217], v246 offset:8192\n"
    "ds_read_b64_tr_b16 v[218:219], v246 offset:9216\n"
    "s_add_u32 m0, s50, 0x2000\n"
    "s_xor_b32 s82, s48, 0x2700\n"
    "buffer_load_dwordx4 v250, s[36:39], s82 offen  lds\n"
    "ds_read_b64_tr_b16 v[220:221], v247 offset:0\n"
    "ds_read_b64_tr_b16 v[222:223], v247 offset:1024\n"
    "ds_read_b64_tr_b16 v[224:225], v247 offset:8192\n"
    "s_add_u32 m0, s50, 0x3000\n"
    "s_xor_b32 s82, s48, 0x82700\n"
    "buffer_load_dwordx4 v250, s[36:39], s82 offen  lds\n"
    "ds_read_b64_tr_b16 v[226:227], v247 offset:9216\n"
    "ds_read_b64_tr_b16 v[228:229], v248 offset:0\n"
    "ds_read_b64_tr_b16 v[230:231], v248 offset:1024\n"
    "s_add_u32 m0, s51, 0xc000\n"
    "s_xor_b32 s82, s67, 0x4f00000\n"
    "buffer_load_dwordx4 v251, s[40:43], s82 offen  lds\n"
    "ds_read_b64_tr_b16 v[232:233], v248 offset:8192\n"
    "ds_read_b64_tr_b16 v[234:235], v248 offset:9216\n"
    "ds_read_b64_tr_b16 v[236:237], v249 offset:0\n"
    "s_add_u32 m0, s51, 0xe000\n"
    "s_xor_b32 s82, s67, 0x4f80000\n"
    "buffer_load_dwordx4 v251, s[40:43], s82 offen  lds\n"
    "ds_read_b64_tr_b16 v[238:239], v249 offset:1024\n"
    "ds_read_b64_tr_b16 v[240:241], v249 offset:8192\n"
    "ds_read_b64_tr_b16 v[242:243], v249 offset:9216\n"
    "s_waitcnt vmcnt(12) lgkmcnt(0)\n"
    "s_barrier\n"
    "s_setprio 1\n"
    "v_mfma_f32_16x16x32_bf16 v[64:67], v[212:215], v[180:183], v[64:67]\n"
    "v_mfma_f32_16x16x32_bf16 v[64:67], v[216:219], v[184:187], v[64:67]\n"
    "v_mfma_f32_16x16x32_bf16 v[60:63], v[220:223], v[180:183], v[60:63]\n"
    "v_mfma_f32_16x16x32_bf16 v[60:63], v[224:227], v[184:187], v[60:63]\n"
    "v_mfma_f32_16x16x32_bf16 v[80:83], v[228:231], v[180:183], v[80:83]\n"
    "v_mfma_f32_16x16x32_bf16 v[80:83], v[232:235], v[184:187], v[80:83]\n"
    "v_mfma_f32_16x16x32_bf16 v[88:91], v[236:239], v[180:183], v[88:91]\n"
    "v_mfma_f32_16x16x32_bf16 v[88:91], v[240:243], v[184:187], v[88:91]\n"
    "v_mfma_f32_16x16x32_bf16 v[96:99], v[212:215], v[188:191], v[96:99]\n"
    "v_mfma_f32_16x16x32_bf16 v[96:99], v[216:219], v[192:195], v[96:99]\n"
    "v_mfma_f32_16x16x32_bf16 v[104:107], v[220:223], v[188:191], v[104:107]\n"
    "v_mfma_f32_16x16x32_bf16 v[104:107], v[224:227], v[192:195], v[104:107]\n"
    "v_mfma_f32_16x16x32_bf16 v[76:79], v[228:231], v[188:191], v[76:79]\n"
    "v_mfma_f32_16x16x32_bf16 v[76:79], v[232:235], v[192:195], v[76:79]\n"
    "v_mfma_f32_16x16x32_bf16 v[112:115], v[236:239], v[188:191], v[112:115]\n"
    "v_mfma_f32_16x16x32_bf16 v[112:115], v[240:243], v[192:195], v[112:115]\n"
    "v_mfma_f32_16x16x32_bf16 v[120:123], v[212:215], v[196:199], v[120:123]\n"
    "v_mfma_f32_16x16x32_bf16 v[120:123], v[216:219], v[200:203], v[120:123]\n"
    "v_mfma_f32_16x16x32_bf16 v[84:87], v[220:223], v[196:199], v[84:87]\n"
    "v_mfma_f32_16x16x32_bf16 v[84:87], v[224:227], v[200:203], v[84:87]\n"
    "v_mfma_f32_16x16x32_bf16 v[28:31], v[228:231], v[196:199], v[28:31]\n"
    "v_mfma_f32_16x16x32_bf16 v[28:31], v[232:235], v[200:203], v[28:31]\n"
    "v_mfma_f32_16x16x32_bf16 v[92:95], v[236:239], v[196:199], v[92:95]\n"
    "v_mfma_f32_16x16x32_bf16 v[92:95], v[240:243], v[200:203], v[92:95]\n"
    "v_mfma_f32_16x16x32_bf16 v[100:103], v[212:215], v[204:207], v[100:103]\n"
    "v_mfma_f32_16x16x32_bf16 v[100:103], v[216:219], v[208:211], v[100:103]\n"
    "v_mfma_f32_16x16x32_bf16 v[108:111], v[220:223], v[204:207], v[108:111]\n"
    "v_mfma_f32_16x16x32_bf16 v[108:111], v[224:227], v[208:211], v[108:111]\n"
    "v_mfma_f32_16x16x32_bf16 v[116:119], v[228:231], v[204:207], v[116:119]\n"
    "v_mfma_f32_16x16x32_bf16 v[116:119], v[232:235], v[208:211], v[116:119]\n"
    "v_mfma_f32_16x16x32_bf16 v[52:55], v[236:239], v[204:207], v[52:55]\n"
    "v_mfma_f32_16x16x32_bf16 v[52:55], v[240:243], v[208:211], v[52:55]\n"
    "s_setprio 0\n"
    "s_waitcnt vmcnt(8)\n"
    "s_barrier\n"
    "; phase 117: step 77 slot 1 stage 3\n"
    "ds_read_b128 v[180:183], v244 offset:32768\n"
    "ds_read_b128 v[184:187], v245 offset:32768\n"
    "ds_read_b128 v[188:191], v244 offset:34816\n"
    "s_cmp_lg_u32 s52, 6\n"
    "s_cbranch_scc1 .La150_%=\n"
    "buffer_store_dwordx4 v[68:71], v255, s[72:75], s80 offen nt\n"
    ".La150_%=:\n"
    "ds_read_b128 v[192:195], v245 offset:34816\n"
    "ds_read_b128 v[196:199], v244 offset:40960\n"
    "ds_read_b128 v[200:203], v245 offset:40960\n"
    "s_cmp_lg_u32 s52, 7\n"
    "s_cbranch_scc1 .La151_%=\n"
    "buffer_store_dwordx4 v[68:71], v255, s[72:75], s80 offen nt\n"
    ".La151_%=:\n"
    "ds_read_b128 v[204:207], v244 offset:43008\n"
    "ds_read_b128 v[208:211], v245 offset:43008\n"
    "ds_read_b64_tr_b16 v[212:213], v246 offset:16384\n"
    "s_add_u32 m0, s50, 0x4000\n"
    "s_xor_b32 s82, s48, 0x202780\n"
    "buffer_load_dwordx4 v250, s[36:39], s82 offen  lds\n"
    "ds_read_b64_tr_b16 v[214:215], v246 offset:17408\n"
    "ds_read_b64_tr_b16 v[216:217], v246 offset:24576\n"
    "ds_read_b64_tr_b16 v[218:219], v246 offset:25600\n"
    "s_add_u32 m0, s50, 0x5000\n"
    "s_xor_b32 s82, s48, 0x282780\n"
    "buffer_load_dwordx4 v250, s[36:39], s82 offen  lds\n"
    "ds_read_b64_tr_b16 v[220:221], v247 offset:16384\n"
    "ds_read_b64_tr_b16 v[222:223], v247 offset:17408\n"
    "ds_read_b64_tr_b16 v[224:225], v247 offset:24576\n"
    "s_add_u32 m0, s50, 0x6000\n"
    "s_xor_b32 s82, s48, 0x2780\n"
    "buffer_load_dwordx4 v250, s[36:39], s82 offen  lds\n"
    "ds_read_b64_tr_b16 v[226:227], v247 offset:25600\n"
    "ds_read_b64_tr_b16 v[228:229], v248 offset:16384\n"
    "ds_read_b64_tr_b16 v[230:231], v248 offset:17408\n"
    "s_add_u32 m0, s50, 0x7000\n"
    "s_xor_b32 s82, s48, 0x82780\n"
    "buffer_load_dwordx4 v250, s[36:39], s82 offen  lds\n"
    "ds_read_b64_tr_b16 v[232:233], v248 offset:24576\n"
    "ds_read_b64_tr_b16 v[234:235], v248 offset:25600\n"
    "ds_read_b64_tr_b16 v[236:237], v249 offset:16384\n"
    "s_add_u32 m0, s51, 0x0\n"
    "s_xor_b32 s82, s67, 0x5000000\n"
    "buffer_load_dwordx4 v251, s[40:43], s82 offen  lds\n"
    "ds_read_b64_tr_b16 v[238:239], v249 offset:17408\n"
    "ds_read_b64_tr_b16 v[240:241], v249 offset:24576\n"
    "ds_read_b64_tr_b16 v[242:243], v249 offset:25600\n"
    "s_add_u32 m0, s51, 0x2000\n"
    "s_xor_b32 s82, s67, 0x5080000\n"
    "buffer_load_dwordx4 v251, s[40:43], s82 offen  lds\n"
    "s_waitcnt vmcnt(12) lgkmcnt(0)\n"
    "s_barrier\n"
    "s_setprio 1\n"
    "v_mfma_f32_16x16x32_bf16 v[64:67], v[212:215], v[180:183], v[64:67]\n"
    "v_mfma_f32_16x16x32_bf16 v[64:67], v[216:219], v[184:187], v[64:67]\n"
    "v_mfma_f32_16x16x32_bf16 v[60:63], v[220:223], v[180:183], v[60:63]\n"
    "v_mfma_f32_16x16x32_bf16 v[60:63], v[224:227], v[184:187], v[60:63]\n"
    "v_mfma_f32_16x16x32_bf16 v[80:83], v[228:231], v[180:183], v[80:83]\n"
    "v_mfma_f32_16x16x32_bf16 v[80:83], v[232:235], v[184:187], v[80:83]\n"
    "v_mfma_f32_16x16x32_bf16 v[88:91], v[236:239], v[180:183], v[88:91]\n"
    "v_mfma_f32_16x16x32_bf16 v[88:91], v[240:243], v[184:187], v[88:91]\n"
    "v_mfma_f32_16x16x32_bf16 v[96:99], v[212:215], v[188:191], v[96:99]\n"
    "v_mfma_f32_16x16x32_bf16 v[96:99], v[216:219], v[192:195], v[96:99]\n"
    "v_mfma_f32_16x16x32_bf16 v[104:107], v[220:223], v[188:191], v[104:107]\n"
    "v_mfma_f32_16x16x32_bf16 v[104:107], v[224:227], v[192:195], v[104:107]\n"
    "v_mfma_f32_16x16x32_bf16 v[76:79], v[228:231], v[188:191], v[76:79]\n"
    "v_mfma_f32_16x16x32_bf16 v[76:79], v[232:235], v[192:195], v[76:79]\n"
    "v_mfma_f32_16x16x32_bf16 v[112:115], v[236:239], v[188:191], v[112:115]\n"
    "v_mfma_f32_16x16x32_bf16 v[112:115], v[240:243], v[192:195], v[112:115]\n"
    "v_mfma_f32_16x16x32_bf16 v[120:123], v[212:215], v[196:199], v[120:123]\n"
    "v_mfma_f32_16x16x32_bf16 v[120:123], v[216:219], v[200:203], v[120:123]\n"
    "v_mfma_f32_16x16x32_bf16 v[84:87], v[220:223], v[196:199], v[84:87]\n"
    "v_mfma_f32_16x16x32_bf16 v[84:87], v[224:227], v[200:203], v[84:87]\n"
    "v_mfma_f32_16x16x32_bf16 v[28:31], v[228:231], v[196:199], v[28:31]\n"
    "v_mfma_f32_16x16x32_bf16 v[28:31], v[232:235], v[200:203], v[28:31]\n"
    "v_mfma_f32_16x16x32_bf16 v[92:95], v[236:239], v[196:199], v[92:95]\n"
    "v_mfma_f32_16x16x32_bf16 v[92:95], v[240:243], v[200:203], v[92:95]\n"
    "v_mfma_f32_16x16x32_bf16 v[100:103], v[212:215], v[204:207], v[100:103]\n"
    "v_mfma_f32_16x16x32_bf16 v[100:103], v[216:219], v[208:211], v[100:103]\n"
    "v_mfma_f32_16x16x32_bf16 v[108:111], v[220:223], v[204:207], v[108:111]\n"
    "v_mfma_f32_16x16x32_bf16 v[108:111], v[224:227], v[208:211], v[108:111]\n"
    "v_mfma_f32_16x16x32_bf16 v[116:119], v[228:231], v[204:207], v[116:119]\n"
    "v_mfma_f32_16x16x32_bf16 v[116:119], v[232:235], v[208:211], v[116:119]\n"
    "v_mfma_f32_16x16x32_bf16 v[52:55], v[236:239], v[204:207], v[52:55]\n"
    "v_mfma_f32_16x16x32_bf16 v[52:55], v[240:243], v[208:211], v[52:55]\n"
    "s_setprio 0\n"
    "s_waitcnt vmcnt(8)\n"
    "s_barrier\n"
    "; phase 118: step 78 slot 1 stage 3\n"
    "ds_read_b128 v[180:183], v244 offset:0\n"
    "ds_read_b128 v[184:187], v245 offset:0\n"
    "ds_read_b128 v[188:191], v244 offset:2048\n"
    "s_cmp_lg_u32 s52, 0\n"
    "s_cbranch_scc1 .La152_%=\n"
    "s_xor_b32 s82, s80, 0x4000\n"
    "buffer_store_dwordx4 v[72:75], v255, s[72:75], s82 offen nt\n"
    ".La152_%=:\n"
    "ds_read_b128 v[192:195], v245 offset:2048\n"
    "ds_read_b128 v[196:199], v244 offset:8192\n"
    "ds_read_b128 v[200:203], v245 offset:8192\n"
    "s_add_u32 m0, s50, 0x8000\n"
    "s_xor_b32 s82, s48, 0x202800\n"
    "buffer_load_dwordx4 v250, s[36:39], s82 offen  lds\n"
    "ds_read_b128 v[204:207], v244 offset:10240\n"
    "ds_read_b128 v[208:211], v245 offset:10240\n"
    "ds_read_b64_tr_b16 v[212:213], v246 offset:32768\n"
    "s_add_u32 m0, s50, 0x9000\n"
    "s_xor_b32 s82, s48, 0x282800\n"
    "buffer_load_dwordx4 v250, s[36:39], s82 offen  lds\n"
    "ds_read_b64_tr_b16 v[214:215], v246 offset:33792\n"
    "ds_read_b64_tr_b16 v[216:217], v246 offset:40960\n"
    "ds_read_b64_tr_b16 v[218:219], v246 offset:41984\n"
    "s_add_u32 m0, s50, 0xa000\n"
    "s_xor_b32 s82, s48, 0x2800\n"
    "buffer_load_dwordx4 v250, s[36:39], s82 offen  lds\n"
    "ds_read_b64_tr_b16 v[220:221], v247 offset:32768\n"
    "ds_read_b64_tr_b16 v[222:223], v247 offset:33792\n"
    "ds_read_b64_tr_b16 v[224:225], v247 offset:40960\n"
    "s_add_u32 m0, s50, 0xb000\n"
    "s_xor_b32 s82, s48, 0x82800\n"
    "buffer_load_dwordx4 v250, s[36:39], s82 offen  lds\n"
    "ds_read_b64_tr_b16 v[226:227], v247 offset:41984\n"
    "ds_read_b64_tr_b16 v[228:229], v248 offset:32768\n"
    "ds_read_b64_tr_b16 v[230:231], v248 offset:33792\n"
    "s_add_u32 m0, s51, 0x4000\n"
    "s_xor_b32 s82, s67, 0x5100000\n"
    "buffer_load_dwordx4 v251, s[40:43], s82 offen  lds\n"
    "ds_read_b64_tr_b16 v[232:233], v248 offset:40960\n"
    "ds_read_b64_tr_b16 v[234:235], v248 offset:41984\n"
    "ds_read_b64_tr_b16 v[236:237], v249 offset:32768\n"
    "s_add_u32 m0, s51, 0x6000\n"
    "s_xor_b32 s82, s67, 0x5180000\n"
    "buffer_load_dwordx4 v251, s[40:43], s82 offen  lds\n"
    "ds_read_b64_tr_b16 v[238:239], v249 offset:33792\n"
    "ds_read_b64_tr_b16 v[240:241], v249 offset:40960\n"
    "ds_read_b64_tr_b16 v[242:243], v249 offset:41984\n"
    "s_waitcnt vmcnt(12) lgkmcnt(0)\n"
    "s_barrier\n"
    "s_setprio 1\n"
    "v_mfma_f32_16x16x32_bf16 v[64:67], v[212:215], v[180:183], v[64:67]\n"
    "v_mfma_f32_16x16x32_bf16 v[64:67], v[216:219], v[184:187], v[64:67]\n"
    "v_mfma_f32_16x16x32_bf16 v[60:63], v[220:223], v[180:183], v[60:63]\n"
    "v_mfma_f32_16x16x32_bf16 v[60:63], v[224:227], v[184:187], v[60:63]\n"
    "v_mfma_f32_16x16x32_bf16 v[80:83], v[228:231], v[180:183], v[80:83]\n"
    "v_mfma_f32_16x16x32_bf16 v[80:83], v[232:235], v[184:187], v[80:83]\n"
    "v_mfma_f32_16x16x32_bf16 v[88:91], v[236:239], v[180:183], v[88:91]\n"
    "v_mfma_f32_16x16x32_bf16 v[88:91], v[240:243], v[184:187], v[88:91]\n"
    "v_mfma_f32_16x16x32_bf16 v[96:99], v[212:215], v[188:191], v[96:99]\n"
    "v_mfma_f32_16x16x32_bf16 v[96:99], v[216:219], v[192:195], v[96:99]\n"
    "v_mfma_f32_16x16x32_bf16 v[104:107], v[220:223], v[188:191], v[104:107]\n"
    "v_mfma_f32_16x16x32_bf16 v[104:107], v[224:227], v[192:195], v[104:107]\n"
    "v_mfma_f32_16x16x32_bf16 v[76:79], v[228:231], v[188:191], v[76:79]\n"
    "v_mfma_f32_16x16x32_bf16 v[76:79], v[232:235], v[192:195], v[76:79]\n"
    "v_mfma_f32_16x16x32_bf16 v[112:115], v[236:239], v[188:191], v[112:115]\n"
    "v_mfma_f32_16x16x32_bf16 v[112:115], v[240:243], v[192:195], v[112:115]\n"
    "v_mfma_f32_16x16x32_bf16 v[120:123], v[212:215], v[196:199], v[120:123]\n"
    "v_mfma_f32_16x16x32_bf16 v[120:123], v[216:219], v[200:203], v[120:123]\n"
    "v_mfma_f32_16x16x32_bf16 v[84:87], v[220:223], v[196:199], v[84:87]\n"
    "v_mfma_f32_16x16x32_bf16 v[84:87], v[224:227], v[200:203], v[84:87]\n"
    "v_mfma_f32_16x16x32_bf16 v[28:31], v[228:231], v[196:199], v[28:31]\n"
    "v_mfma_f32_16x16x32_bf16 v[28:31], v[232:235], v[200:203], v[28:31]\n"
    "v_mfma_f32_16x16x32_bf16 v[92:95], v[236:239], v[196:199], v[92:95]\n"
    "v_mfma_f32_16x16x32_bf16 v[92:95], v[240:243], v[200:203], v[92:95]\n"
    "v_mfma_f32_16x16x32_bf16 v[100:103], v[212:215], v[204:207], v[100:103]\n"
    "v_mfma_f32_16x16x32_bf16 v[100:103], v[216:219], v[208:211], v[100:103]\n"
    "v_mfma_f32_16x16x32_bf16 v[108:111], v[220:223], v[204:207], v[108:111]\n"
    "v_mfma_f32_16x16x32_bf16 v[108:111], v[224:227], v[208:211], v[108:111]\n"
    "v_mfma_f32_16x16x32_bf16 v[116:119], v[228:231], v[204:207], v[116:119]\n"
    "v_mfma_f32_16x16x32_bf16 v[116:119], v[232:235], v[208:211], v[116:119]\n"
    "v_mfma_f32_16x16x32_bf16 v[52:55], v[236:239], v[204:207], v[52:55]\n"
    "v_mfma_f32_16x16x32_bf16 v[52:55], v[240:243], v[208:211], v[52:55]\n"
    "s_setprio 0\n"
    "s_waitcnt vmcnt(8)\n"
    "s_barrier\n"
    "; phase 119: step 79 slot 1 stage 3\n"
    "ds_read_b128 v[180:183], v244 offset:16384\n"
    "ds_read_b128 v[184:187], v245 offset:16384\n"
    "ds_read_b128 v[188:191], v244 offset:18432\n"
    "s_cmp_lg_u32 s52, 1\n"
    "s_cbranch_scc1 .La153_%=\n"
    "s_xor_b32 s82, s80, 0x4000\n"
    "buffer_store_dwordx4 v[72:75], v255, s[72:75], s82 offen nt\n"
    ".La153_%=:\n"
    "ds_read_b128 v[192:195], v245 offset:18432\n"
    "ds_read_b128 v[196:199], v244 offset:24576\n"
    "ds_read_b128 v[200:203], v245 offset:24576\n"
    "s_add_u32 m0, s50, 0x0\n"
    "s_xor_b32 s82, s48, 0x202880\n"
    "buffer_load_dwordx4 v250, s[36:39], s82 offen  lds\n"
    "ds_read_b128 v[204:207], v244 offset:26624\n"
    "ds_read_b128 v[208:211], v245 offset:26624\n"
    "ds_read_b64_tr_b16 v[212:213], v246 offset:49152\n"
    "s_add_u32 m0, s50, 0x1000\n"
    "s_xor_b32 s82, s48, 0x282880\n"
    "buffer_load_dwordx4 v250, s[36:39], s82 offen  lds\n"
    "ds_read_b64_tr_b16 v[214:215], v246 offset:50176\n"
    "ds_read_b64_tr_b16 v[216:217], v246 offset:57344\n"
    "ds_read_b64_tr_b16 v[218:219], v246 offset:58368\n"
    "s_add_u32 m0, s50, 0x2000\n"
    "s_xor_b32 s82, s48, 0x2880\n"
    "buffer_load_dwordx4 v250, s[36:39], s82 offen  lds\n"
    "ds_read_b64_tr_b16 v[220:221], v247 offset:49152\n"
    "ds_read_b64_tr_b16 v[222:223], v247 offset:50176\n"
    "ds_read_b64_tr_b16 v[224:225], v247 offset:57344\n"
    "s_add_u32 m0, s50, 0x3000\n"
    "s_xor_b32 s82, s48, 0x82880\n"
    "buffer_load_dwordx4 v250, s[36:39], s82 offen  lds\n"
    "ds_read_b64_tr_b16 v[226:227], v247 offset:58368\n"
    "ds_read_b64_tr_b16 v[228:229], v248 offset:49152\n"
    "ds_read_b64_tr_b16 v[230:231], v248 offset:50176\n"
    "s_add_u32 m0, s51, 0x8000\n"
    "s_xor_b32 s82, s67, 0x5200000\n"
    "buffer_load_dwordx4 v251, s[40:43], s82 offen  lds\n"
    "ds_read_b64_tr_b16 v[232:233], v248 offset:57344\n"
    "ds_read_b64_tr_b16 v[234:235], v248 offset:58368\n"
    "ds_read_b64_tr_b16 v[236:237], v249 offset:49152\n"
    "s_add_u32 m0, s51, 0xa000\n"
    "s_xor_b32 s82, s67, 0x5280000\n"
    "buffer_load_dwordx4 v251, s[40:43], s82 offen  lds\n"
    "ds_read_b64_tr_b16 v[238:239], v249 offset:50176\n"
    "ds_read_b64_tr_b16 v[240:241], v249 offset:57344\n"
    "ds_read_b64_tr_b16 v[242:243], v249 offset:58368\n"
    "s_waitcnt vmcnt(12) lgkmcnt(0)\n"
    "s_barrier\n"
    "s_setprio 1\n"
    "v_mfma_f32_16x16x32_bf16 v[64:67], v[212:215], v[180:183], v[64:67]\n"
    "v_mfma_f32_16x16x32_bf16 v[64:67], v[216:219], v[184:187], v[64:67]\n"
    "v_mfma_f32_16x16x32_bf16 v[60:63], v[220:223], v[180:183], v[60:63]\n"
    "v_mfma_f32_16x16x32_bf16 v[60:63], v[224:227], v[184:187], v[60:63]\n"
    "v_mfma_f32_16x16x32_bf16 v[80:83], v[228:231], v[180:183], v[80:83]\n"
    "v_mfma_f32_16x16x32_bf16 v[80:83], v[232:235], v[184:187], v[80:83]\n"
    "v_mfma_f32_16x16x32_bf16 v[88:91], v[236:239], v[180:183], v[88:91]\n"
    "v_mfma_f32_16x16x32_bf16 v[88:91], v[240:243], v[184:187], v[88:91]\n"
    "v_mfma_f32_16x16x32_bf16 v[96:99], v[212:215], v[188:191], v[96:99]\n"
    "v_mfma_f32_16x16x32_bf16 v[96:99], v[216:219], v[192:195], v[96:99]\n"
    "v_mfma_f32_16x16x32_bf16 v[104:107], v[220:223], v[188:191], v[104:107]\n"
    "v_mfma_f32_16x16x32_bf16 v[104:107], v[224:227], v[192:195], v[104:107]\n"
    "v_mfma_f32_16x16x32_bf16 v[76:79], v[228:231], v[188:191], v[76:79]\n"
    "v_mfma_f32_16x16x32_bf16 v[76:79], v[232:235], v[192:195], v[76:79]\n"
    "v_mfma_f32_16x16x32_bf16 v[112:115], v[236:239], v[188:191], v[112:115]\n"
    "v_mfma_f32_16x16x32_bf16 v[112:115], v[240:243], v[192:195], v[112:115]\n"
    "v_mfma_f32_16x16x32_bf16 v[120:123], v[212:215], v[196:199], v[120:123]\n"
    "v_mfma_f32_16x16x32_bf16 v[120:123], v[216:219], v[200:203], v[120:123]\n"
    "v_mfma_f32_16x16x32_bf16 v[84:87], v[220:223], v[196:199], v[84:87]\n"
    "v_mfma_f32_16x16x32_bf16 v[84:87], v[224:227], v[200:203], v[84:87]\n"
    "v_mfma_f32_16x16x32_bf16 v[28:31], v[228:231], v[196:199], v[28:31]\n"
    "v_mfma_f32_16x16x32_bf16 v[28:31], v[232:235], v[200:203], v[28:31]\n"
    "v_mfma_f32_16x16x32_bf16 v[92:95], v[236:239], v[196:199], v[92:95]\n"
    "v_mfma_f32_16x16x32_bf16 v[92:95], v[240:243], v[200:203], v[92:95]\n"
    "v_mfma_f32_16x16x32_bf16 v[100:103], v[212:215], v[204:207], v[100:103]\n"
    "v_mfma_f32_16x16x32_bf16 v[100:103], v[216:219], v[208:211], v[100:103]\n"
    "v_mfma_f32_16x16x32_bf16 v[108:111], v[220:223], v[204:207], v[108:111]\n"
    "v_mfma_f32_16x16x32_bf16 v[108:111], v[224:227], v[208:211], v[108:111]\n"
    "v_mfma_f32_16x16x32_bf16 v[116:119], v[228:231], v[204:207], v[116:119]\n"
    "v_mfma_f32_16x16x32_bf16 v[116:119], v[232:235], v[208:211], v[116:119]\n"
    "v_mfma_f32_16x16x32_bf16 v[52:55], v[236:239], v[204:207], v[52:55]\n"
    "v_mfma_f32_16x16x32_bf16 v[52:55], v[240:243], v[208:211], v[52:55]\n"
    "s_setprio 0\n"
    "s_waitcnt vmcnt(8)\n"
    "s_barrier\n"
    "; phase 120: step 80 slot 1 stage 3\n"
    "ds_read_b128 v[180:183], v244 offset:32768\n"
    "ds_read_b128 v[184:187], v245 offset:32768\n"
    "ds_read_b128 v[188:191], v244 offset:34816\n"
    "s_cmp_lg_u32 s52, 2\n"
    "s_cbranch_scc1 .La154_%=\n"
    "s_xor_b32 s82, s80, 0x4000\n"
    "buffer_store_dwordx4 v[72:75], v255, s[72:75], s82 offen nt\n"
    ".La154_%=:\n"
    "ds_read_b128 v[192:195], v245 offset:34816\n"
    "ds_read_b128 v[196:199], v244 offset:40960\n"
    "ds_read_b128 v[200:203], v245 offset:40960\n"
    "s_add_u32 m0, s50, 0x4000\n"
    "s_xor_b32 s82, s48, 0x202900\n"
    "buffer_load_dwordx4 v250, s[36:39], s82 offen  lds\n"
    "ds_read_b128 v[204:207], v244 offset:43008\n"
    "ds_read_b128 v[208:211], v245 offset:43008\n"
    "ds_read_b64_tr_b16 v[212:213], v246 offset:0\n"
    "s_add_u32 m0, s50, 0x5000\n"
    "s_xor_b32 s82, s48, 0x282900\n"
    "buffer_load_dwordx4 v250, s[36:39], s82 offen  lds\n"
    "ds_read_b64_tr_b16 v[214:215], v246 offset:1024\n"
    "ds_read_b64_tr_b16 v[216:217], v246 offset:8192\n"
    "ds_read_b64_tr_b16 v[218:219], v246 offset:9216\n"
    "s_add_u32 m0, s50, 0x6000\n"
    "s_xor_b32 s82, s48, 0x2900\n"
    "buffer_load_dwordx4 v250, s[36:39], s82 offen  lds\n"
    "ds_read_b64_tr_b16 v[220:221], v247 offset:0\n"
    "ds_read_b64_tr_b16 v[222:223], v247 offset:1024\n"
    "ds_read_b64_tr_b16 v[224:225], v247 offset:8192\n"
    "s_add_u32 m0, s50, 0x7000\n"
    "s_xor_b32 s82, s48, 0x82900\n"
    "buffer_load_dwordx4 v250, s[36:39], s82 offen  lds\n"
    "ds_read_b64_tr_b16 v[226:227], v247 offset:9216\n"
    "ds_read_b64_tr_b16 v[228:229], v248 offset:0\n"
    "ds_read_b64_tr_b16 v[230:231], v248 offset:1024\n"
    "s_add_u32 m0, s51, 0xc000\n"
    "s_xor_b32 s82, s67, 0x5300000\n"
    "buffer_load_dwordx4 v251, s[40:43], s82 offen  lds\n"
    "ds_read_b64_tr_b16 v[232:233], v248 offset:8192\n"
    "ds_read_b64_tr_b16 v[234:235], v248 offset:9216\n"
    "ds_read_b64_tr_b16 v[236:237], v249 offset:0\n"
    "s_add_u32 m0, s51, 0xe000\n"
    "s_xor_b32 s82, s67, 0x5380000\n"
    "buffer_load_dwordx4 v251, s[40:43], s82 offen  lds\n"
    "ds_read_b64_tr_b16 v[238:239], v249 offset:1024\n"
    "ds_read_b64_tr_b16 v[240:241], v249 offset:8192\n"
    "ds_read_b64_tr_b16 v[242:243], v249 offset:9216\n"
    "s_waitcnt vmcnt(12) lgkmcnt(0)\n"
    "s_barrier\n"
    "s_setprio 1\n"
    "v_mfma_f32_16x16x32_bf16 v[64:67], v[212:215], v[180:183], v[64:67]\n"
    "v_mfma_f32_16x16x32_bf16 v[64:67], v[216:219], v[184:187], v[64:67]\n"
    "v_mfma_f32_16x16x32_bf16 v[60:63], v[220:223], v[180:183], v[60:63]\n"
    "v_mfma_f32_16x16x32_bf16 v[60:63], v[224:227], v[184:187], v[60:63]\n"
    "v_mfma_f32_16x16x32_bf16 v[80:83], v[228:231], v[180:183], v[80:83]\n"
    "v_mfma_f32_16x16x32_bf16 v[80:83], v[232:235], v[184:187], v[80:83]\n"
    "v_mfma_f32_16x16x32_bf16 v[88:91], v[236:239], v[180:183], v[88:91]\n"
    "v_mfma_f32_16x16x32_bf16 v[88:91], v[240:243], v[184:187], v[88:91]\n"
    "v_mfma_f32_16x16x32_bf16 v[96:99], v[212:215], v[188:191], v[96:99]\n"
    "v_mfma_f32_16x16x32_bf16 v[96:99], v[216:219], v[192:195], v[96:99]\n"
    "v_mfma_f32_16x16x32_bf16 v[104:107], v[220:223], v[188:191], v[104:107]\n"
    "v_mfma_f32_16x16x32_bf16 v[104:107], v[224:227], v[192:195], v[104:107]\n"
    "v_mfma_f32_16x16x32_bf16 v[76:79], v[228:231], v[188:191], v[76:79]\n"
    "v_mfma_f32_16x16x32_bf16 v[76:79], v[232:235], v[192:195], v[76:79]\n"
    "v_mfma_f32_16x16x32_bf16 v[112:115], v[236:239], v[188:191], v[112:115]\n"
    "v_mfma_f32_16x16x32_bf16 v[112:115], v[240:243], v[192:195], v[112:115]\n"
    "v_mfma_f32_16x16x32_bf16 v[120:123], v[212:215], v[196:199], v[120:123]\n"
    "v_mfma_f32_16x16x32_bf16 v[120:123], v[216:219], v[200:203], v[120:123]\n"
    "v_mfma_f32_16x16x32_bf16 v[84:87], v[220:223], v[196:199], v[84:87]\n"
    "v_mfma_f32_16x16x32_bf16 v[84:87], v[224:227], v[200:203], v[84:87]\n"
    "v_mfma_f32_16x16x32_bf16 v[28:31], v[228:231], v[196:199], v[28:31]\n"
    "v_mfma_f32_16x16x32_bf16 v[28:31], v[232:235], v[200:203], v[28:31]\n"
    "v_mfma_f32_16x16x32_bf16 v[92:95], v[236:239], v[196:199], v[92:95]\n"
    "v_mfma_f32_16x16x32_bf16 v[92:95], v[240:243], v[200:203], v[92:95]\n"
    "v_mfma_f32_16x16x32_bf16 v[100:103], v[212:215], v[204:207], v[100:103]\n"
    "v_mfma_f32_16x16x32_bf16 v[100:103], v[216:219], v[208:211], v[100:103]\n"
    "v_mfma_f32_16x16x32_bf16 v[108:111], v[220:223], v[204:207], v[108:111]\n"
    "v_mfma_f32_16x16x32_bf16 v[108:111], v[224:227], v[208:211], v[108:111]\n"
    "v_mfma_f32_16x16x32_bf16 v[116:119], v[228:231], v[204:207], v[116:119]\n"
    "v_mfma_f32_16x16x32_bf16 v[116:119], v[232:235], v[208:211], v[116:119]\n"
    "v_mfma_f32_16x16x32_bf16 v[52:55], v[236:239], v[204:207], v[52:55]\n"
    "v_mfma_f32_16x16x32_bf16 v[52:55], v[240:243], v[208:211], v[52:55]\n"
    "s_setprio 0\n"
    "s_waitcnt vmcnt(8)\n"
    "s_barrier\n"
    "; phase 121: step 81 slot 1 stage 3\n"
    "ds_read_b128 v[180:183], v244 offset:0\n"
    "ds_read_b128 v[184:187], v245 offset:0\n"
    "ds_read_b128 v[188:191], v244 offset:2048\n"
    "s_cmp_lg_u32 s52, 3\n"
    "s_cbranch_scc1 .La155_%=\n"
    "s_xor_b32 s82, s80, 0x4000\n"
    "buffer_store_dwordx4 v[72:75], v255, s[72:75], s82 offen nt\n"
    ".La155_%=:\n"
    "ds_read_b128 v[192:195], v245 offset:2048\n"
    "ds_read_b128 v[196:199], v244 offset:8192\n"
    "ds_read_b128 v[200:203], v245 offset:8192\n"
    "s_add_u32 m0, s50, 0x8000\n"
    "s_xor_b32 s82, s48, 0x202980\n"
    "buffer_load_dwordx4 v250, s[36:39], s82 offen  lds\n"
    "ds_read_b128 v[204:207], v244 offset:10240\n"
    "ds_read_b128 v[208:211], v245 offset:10240\n"
    "ds_read_b64_tr_b16 v[212:213], v246 offset:16384\n"
    "s_add_u32 m0, s50, 0x9000\n"
    "s_xor_b32 s82, s48, 0x282980\n"
    "buffer_load_dwordx4 v250, s[36:39], s82 offen  lds\n"
    "ds_read_b64_tr_b16 v[214:215], v246 offset:17408\n"
    "ds_read_b64_tr_b16 v[216:217], v246 offset:24576\n"
    "ds_read_b64_tr_b16 v[218:219], v246 offset:25600\n"
    "s_add_u32 m0, s50, 0xa000\n"
    "s_xor_b32 s82, s48, 0x2980\n"
    "buffer_load_dwordx4 v250, s[36:39], s82 offen  lds\n"
    "ds_read_b64_tr_b16 v[220:221], v247 offset:16384\n"
    "ds_read_b64_tr_b16 v[222:223], v247 offset:17408\n"
    "ds_read_b64_tr_b16 v[224:225], v247 offset:24576\n"
    "s_add_u32 m0, s50, 0xb000\n"
    "s_xor_b32 s82, s48, 0x82980\n"
    "buffer_load_dwordx4 v250, s[36:39], s82 offen  lds\n"
    "ds_read_b64_tr_b16 v[226:227], v247 offset:25600\n"
    "ds_read_b64_tr_b16 v[228:229], v248 offset:16384\n"
    "ds_read_b64_tr_b16 v[230:231], v248 offset:17408\n"
    "s_add_u32 m0, s51, 0x0\n"
    "s_xor_b32 s82, s67, 0x5400000\n"
    "buffer_load_dwordx4 v251, s[40:43], s82 offen  lds\n"
    "ds_read_b64_tr_b16 v[232:233], v248 offset:24576\n"
    "ds_read_b64_tr_b16 v[234:235], v248 offset:25600\n"
    "ds_read_b64_tr_b16 v[236:237], v249 offset:16384\n"
    "s_add_u32 m0, s51, 0x2000\n"
    "s_xor_b32 s82, s67, 0x5480000\n"
    "buffer_load_dwordx4 v251, s[40:43], s82 offen  lds\n"
    "ds_read_b64_tr_b16 v[238:239], v249 offset:17408\n"
    "ds_read_b64_tr_b16 v[240:241], v249 offset:24576\n"
    "ds_read_b64_tr_b16 v[242:243], v249 offset:25600\n"
    "s_waitcnt vmcnt(12) lgkmcnt(0)\n"
    "s_barrier\n"
    "s_setprio 1\n"
    "v_mfma_f32_16x16x32_bf16 v[64:67], v[212:215], v[180:183], v[64:67]\n"
    "v_mfma_f32_16x16x32_bf16 v[64:67], v[216:219], v[184:187], v[64:67]\n"
    "v_mfma_f32_16x16x32_bf16 v[60:63], v[220:223], v[180:183], v[60:63]\n"
    "v_mfma_f32_16x16x32_bf16 v[60:63], v[224:227], v[184:187], v[60:63]\n"
    "v_mfma_f32_16x16x32_bf16 v[80:83], v[228:231], v[180:183], v[80:83]\n"
    "v_mfma_f32_16x16x32_bf16 v[80:83], v[232:235], v[184:187], v[80:83]\n"
    "v_mfma_f32_16x16x32_bf16 v[88:91], v[236:239], v[180:183], v[88:91]\n"
    "v_mfma_f32_16x16x32_bf16 v[88:91], v[240:243], v[184:187], v[88:91]\n"
    "v_mfma_f32_16x16x32_bf16 v[96:99], v[212:215], v[188:191], v[96:99]\n"
    "v_mfma_f32_16x16x32_bf16 v[96:99], v[216:219], v[192:195], v[96:99]\n"
    "v_mfma_f32_16x16x32_bf16 v[104:107], v[220:223], v[188:191], v[104:107]\n"
    "v_mfma_f32_16x16x32_bf16 v[104:107], v[224:227], v[192:195], v[104:107]\n"
    "v_mfma_f32_16x16x32_bf16 v[76:79], v[228:231], v[188:191], v[76:79]\n"
    "v_mfma_f32_16x16x32_bf16 v[76:79], v[232:235], v[192:195], v[76:79]\n"
    "v_mfma_f32_16x16x32_bf16 v[112:115], v[236:239], v[188:191], v[112:115]\n"
    "v_mfma_f32_16x16x32_bf16 v[112:115], v[240:243], v[192:195], v[112:115]\n"
    "v_mfma_f32_16x16x32_bf16 v[120:123], v[212:215], v[196:199], v[120:123]\n"
    "v_mfma_f32_16x16x32_bf16 v[120:123], v[216:219], v[200:203], v[120:123]\n"
    "v_mfma_f32_16x16x32_bf16 v[84:87], v[220:223], v[196:199], v[84:87]\n"
    "v_mfma_f32_16x16x32_bf16 v[84:87], v[224:227], v[200:203], v[84:87]\n"
    "v_mfma_f32_16x16x32_bf16 v[28:31], v[228:231], v[196:199], v[28:31]\n"
    "v_mfma_f32_16x16x32_bf16 v[28:31], v[232:235], v[200:203], v[28:31]\n"
    "v_mfma_f32_16x16x32_bf16 v[92:95], v[236:239], v[196:199], v[92:95]\n"
    "v_mfma_f32_16x16x32_bf16 v[92:95], v[240:243], v[200:203], v[92:95]\n"
    "v_mfma_f32_16x16x32_bf16 v[100:103], v[212:215], v[204:207], v[100:103]\n"
    "v_mfma_f32_16x16x32_bf16 v[100:103], v[216:219], v[208:211], v[100:103]\n"
    "v_mfma_f32_16x16x32_bf16 v[108:111], v[220:223], v[204:207], v[108:111]\n"
    "v_mfma_f32_16x16x32_bf16 v[108:111], v[224:227], v[208:211], v[108:111]\n"
    "v_mfma_f32_16x16x32_bf16 v[116:119], v[228:231], v[204:207], v[116:119]\n"
    "v_mfma_f32_16x16x32_bf16 v[116:119], v[232:235], v[208:211], v[116:119]\n"
    "v_mfma_f32_16x16x32_bf16 v[52:55], v[236:239], v[204:207], v[52:55]\n"
    "v_mfma_f32_16x16x32_bf16 v[52:55], v[240:243], v[208:211], v[52:55]\n"
    "s_setprio 0\n"
    "s_waitcnt vmcnt(8)\n"
    "s_barrier\n"
    "; phase 122: step 82 slot 1 stage 3\n"
    "ds_read_b128 v[180:183], v244 offset:16384\n"
    "ds_read_b128 v[184:187], v245 offset:16384\n"
    "ds_read_b128 v[188:191], v244 offset:18432\n"
    "s_cmp_lg_u32 s52, 4\n"
    "s_cbranch_scc1 .La156_%=\n"
    "s_xor_b32 s82, s80, 0x4000\n"
    "buffer_store_dwordx4 v[72:75], v255, s[72:75], s82 offen nt\n"
    ".La156_%=:\n"
    "ds_read_b128 v[192:195], v245 offset:18432\n"
    "ds_read_b128 v[196:199], v244 offset:24576\n"
    "ds_read_b128 v[200:203], v245 offset:24576\n"
    "s_cmp_lg_u32 s52, 5\n"
    "s_cbranch_scc1 .La157_%=\n"
    "s_xor_b32 s82, s80, 0x4000\n"
    "buffer_store_dwordx4 v[72:75], v255, s[72:75], s82 offen nt\n"
    ".La157_%=:\n"
    "ds_read_b128 v[204:207], v244 offset:26624\n"
    "ds_read_b128 v[208:211], v245 offset:26624\n"
    "ds_read_b64_tr_b16 v[212:213], v246 offset:32768\n"
    "s_add_u32 m0, s50, 0x0\n"
    "s_xor_b32 s82, s48, 0x202a00\n"
    "buffer_load_dwordx4 v250, s[36:39], s82 offen  lds\n"
    "ds_read_b64_tr_b16 v[214:215], v246 offset:33792\n"
    "ds_read_b64_tr_b16 v[216:217], v246 offset:40960\n"
    "ds_read_b64_tr_b16 v[218:219], v246 offset:41984\n"
    "s_add_u32 m0, s50, 0x1000\n"
    "s_xor_b32 s82, s48, 0x282a00\n"
    "buffer_load_dwordx4 v250, s[36:39], s82 offen  lds\n"
    "ds_read_b64_tr_b16 v[220:221], v247 offset:32768\n"
    "ds_read_b64_tr_b16 v[222:223], v247 offset:33792\n"
    "ds_read_b64_tr_b16 v[224:225], v247 offset:40960\n"
    "s_add_u32 m0, s50, 0x2000\n"
    "s_xor_b32 s82, s48, 0x2a00\n"
    "buffer_load_dwordx4 v250, s[36:39], s82 offen  lds\n"
    "ds_read_b64_tr_b16 v[226:227], v247 offset:41984\n"
    "ds_read_b64_tr_b16 v[228:229], v248 offset:32768\n"
    "ds_read_b64_tr_b16 v[230:231], v248 offset:33792\n"
    "s_add_u32 m0, s50, 0x3000\n"
    "s_xor_b32 s82, s48, 0x82a00\n"
    "buffer_load_dwordx4 v250, s[36:39], s82 offen  lds\n"
    "ds_read_b64_tr_b16 v[232:233], v248 offset:40960\n"
    "ds_read_b64_tr_b16 v[234:235], v248 offset:41984\n"
    "ds_read_b64_tr_b16 v[236:237], v249 offset:32768\n"
    "s_add_u32 m0, s51, 0x4000\n"
    "s_xor_b32 s82, s67, 0x5500000\n"
    "buffer_load_dwordx4 v251, s[40:43], s82 offen  lds\n"
    "ds_read_b64_tr_b16 v[238:239], v249 offset:33792\n"
    "ds_read_b64_tr_b16 v[240:241], v249 offset:40960\n"
    "ds_read_b64_tr_b16 v[242:243], v249 offset:41984\n"
    "s_add_u32 m0, s51, 0x6000\n"
    "s_xor_b32 s82, s67, 0x5580000\n"
    "buffer_load_dwordx4 v251, s[40:43], s82 offen  lds\n"
    "s_waitcnt vmcnt(12) lgkmcnt(0)\n"
    "s_barrier\n"
    "s_setprio 1\n"
    "v_mfma_f32_16x16x32_bf16 v[64:67], v[212:215], v[180:183], v[64:67]\n"
    "v_mfma_f32_16x16x32_bf16 v[64:67], v[216:219], v[184:187], v[64:67]\n"
    "v_mfma_f32_16x16x32_bf16 v[60:63], v[220:223], v[180:183], v[60:63]\n"
    "v_mfma_f32_16x16x32_bf16 v[60:63], v[224:227], v[184:187], v[60:63]\n"
    "v_mfma_f32_16x16x32_bf16 v[80:83], v[228:231], v[180:183], v[80:83]\n"
    "v_mfma_f32_16x16x32_bf16 v[80:83], v[232:235], v[184:187], v[80:83]\n"
    "v_mfma_f32_16x16x32_bf16 v[88:91], v[236:239], v[180:183], v[88:91]\n"
    "v_mfma_f32_16x16x32_bf16 v[88:91], v[240:243], v[184:187], v[88:91]\n"
    "v_mfma_f32_16x16x32_bf16 v[96:99], v[212:215], v[188:191], v[96:99]\n"
    "v_mfma_f32_16x16x32_bf16 v[96:99], v[216:219], v[192:195], v[96:99]\n"
    "v_mfma_f32_16x16x32_bf16 v[104:107], v[220:223], v[188:191], v[104:107]\n"
    "v_mfma_f32_16x16x32_bf16 v[104:107], v[224:227], v[192:195], v[104:107]\n"
    "v_mfma_f32_16x16x32_bf16 v[76:79], v[228:231], v[188:191], v[76:79]\n"
    "v_mfma_f32_16x16x32_bf16 v[76:79], v[232:235], v[192:195], v[76:79]\n"
    "v_mfma_f32_16x16x32_bf16 v[112:115], v[236:239], v[188:191], v[112:115]\n"
    "v_mfma_f32_16x16x32_bf16 v[112:115], v[240:243], v[192:195], v[112:115]\n"
    "v_mfma_f32_16x16x32_bf16 v[120:123], v[212:215], v[196:199], v[120:123]\n"
    "v_mfma_f32_16x16x32_bf16 v[120:123], v[216:219], v[200:203], v[120:123]\n"
    "v_mfma_f32_16x16x32_bf16 v[84:87], v[220:223], v[196:199], v[84:87]\n"
    "v_mfma_f32_16x16x32_bf16 v[84:87], v[224:227], v[200:203], v[84:87]\n"
    "v_mfma_f32_16x16x32_bf16 v[28:31], v[228:231], v[196:199], v[28:31]\n"
    "v_mfma_f32_16x16x32_bf16 v[28:31], v[232:235], v[200:203], v[28:31]\n"
    "v_mfma_f32_16x16x32_bf16 v[92:95], v[236:239], v[196:199], v[92:95]\n"
    "v_mfma_f32_16x16x32_bf16 v[92:95], v[240:243], v[200:203], v[92:95]\n"
    "v_mfma_f32_16x16x32_bf16 v[100:103], v[212:215], v[204:207], v[100:103]\n"
    "v_mfma_f32_16x16x32_bf16 v[100:103], v[216:219], v[208:211], v[100:103]\n"
    "v_mfma_f32_16x16x32_bf16 v[108:111], v[220:223], v[204:207], v[108:111]\n"
    "v_mfma_f32_16x16x32_bf16 v[108:111], v[224:227], v[208:211], v[108:111]\n"
    "v_mfma_f32_16x16x32_bf16 v[116:119], v[228:231], v[204:207], v[116:119]\n"
    "v_mfma_f32_16x16x32_bf16 v[116:119], v[232:235], v[208:211], v[116:119]\n"
    "v_mfma_f32_16x16x32_bf16 v[52:55], v[236:239], v[204:207], v[52:55]\n"
    "v_mfma_f32_16x16x32_bf16 v[52:55], v[240:243], v[208:211], v[52:55]\n"
    "s_setprio 0\n"
    "s_waitcnt vmcnt(8)\n"
    "s_barrier\n"
    "; phase 123: step 83 slot 1 stage 3\n"
    "ds_read_b128 v[180:183], v244 offset:32768\n"
    "ds_read_b128 v[184:187], v245 offset:32768\n"
    "ds_read_b128 v[188:191], v244 offset:34816\n"
    "s_cmp_lg_u32 s52, 6\n"
    "s_cbranch_scc1 .La158_%=\n"
    "s_xor_b32 s82, s80, 0x4000\n"
    "buffer_store_dwordx4 v[72:75], v255, s[72:75], s82 offen nt\n"
    ".La158_%=:\n"
    "ds_read_b128 v[192:195], v245 offset:34816\n"
    "ds_read_b128 v[196:199], v244 offset:40960\n"
    "ds_read_b128 v[200:203], v245 offset:40960\n"
    "s_add_u32 m0, s50, 0x4000\n"
    "s_xor_b32 s82, s48, 0x202a80\n"
    "buffer_load_dwordx4 v250, s[36:39], s82 offen  lds\n"
    "ds_read_b128 v[204:207], v244 offset:43008\n"
    "ds_read_b128 v[208:211], v245 offset:43008\n"
    "ds_read_b64_tr_b16 v[212:213], v246 offset:49152\n"
    "s_add_u32 m0, s50, 0x5000\n"
    "s_xor_b32 s82, s48, 0x282a80\n"
    "buffer_load_dwordx4 v250, s[36:39], s82 offen  lds\n"
    "ds_read_b64_tr_b16 v[214:215], v246 offset:50176\n"
    "ds_read_b64_tr_b16 v[216:217], v246 offset:57344\n"
    "ds_read_b64_tr_b16 v[218:219], v246 offset:58368\n"
    "s_add_u32 m0, s50, 0x6000\n"
    "s_xor_b32 s82, s48, 0x2a80\n"
    "buffer_load_dwordx4 v250, s[36:39], s82 offen  lds\n"
    "ds_read_b64_tr_b16 v[220:221], v247 offset:49152\n"
    "ds_read_b64_tr_b16 v[222:223], v247 offset:50176\n"
    "ds_read_b64_tr_b16 v[224:225], v247 offset:57344\n"
    "s_add_u32 m0, s50, 0x7000\n"
    "s_xor_b32 s82, s48, 0x82a80\n"
    "buffer_load_dwordx4 v250, s[36:39], s82 offen  lds\n"
    "ds_read_b64_tr_b16 v[226:227], v247 offset:58368\n"
    "ds_read_b64_tr_b16 v[228:229], v248 offset:49152\n"
    "ds_read_b64_tr_b16 v[230:231], v248 offset:50176\n"
    "s_add_u32 m0, s51, 0x8000\n"
    "s_xor_b32 s82, s67, 0x5600000\n"
    "buffer_load_dwordx4 v251, s[40:43], s82 offen  lds\n"
    "ds_read_b64_tr_b16 v[232:233], v248 offset:57344\n"
    "ds_read_b64_tr_b16 v[234:235], v248 offset:58368\n"
    "ds_read_b64_tr_b16 v[236:237], v249 offset:49152\n"
    "s_add_u32 m0, s51, 0xa000\n"
    "s_xor_b32 s82, s67, 0x5680000\n"
    "buffer_load_dwordx4 v251, s[40:43], s82 offen  lds\n"
    "ds_read_b64_tr_b16 v[238:239], v249 offset:50176\n"
    "ds_read_b64_tr_b16 v[240:241], v249 offset:57344\n"
    "ds_read_b64_tr_b16 v[242:243], v249 offset:58368\n"
    "s_waitcnt vmcnt(12) lgkmcnt(0)\n"
    "s_barrier\n"
    "s_setprio 1\n"
    "v_mfma_f32_16x16x32_bf16 v[64:67], v[212:215], v[180:183], v[64:67]\n"
    "v_mfma_f32_16x16x32_bf16 v[64:67], v[216:219], v[184:187], v[64:67]\n"
    "v_mfma_f32_16x16x32_bf16 v[60:63], v[220:223], v[180:183], v[60:63]\n"
    "v_mfma_f32_16x16x32_bf16 v[60:63], v[224:227], v[184:187], v[60:63]\n"
    "v_mfma_f32_16x16x32_bf16 v[80:83], v[228:231], v[180:183], v[80:83]\n"
    "v_mfma_f32_16x16x32_bf16 v[80:83], v[232:235], v[184:187], v[80:83]\n"
    "v_mfma_f32_16x16x32_bf16 v[88:91], v[236:239], v[180:183], v[88:91]\n"
    "v_mfma_f32_16x16x32_bf16 v[88:91], v[240:243], v[184:187], v[88:91]\n"
    "v_mfma_f32_16x16x32_bf16 v[96:99], v[212:215], v[188:191], v[96:99]\n"
    "v_mfma_f32_16x16x32_bf16 v[96:99], v[216:219], v[192:195], v[96:99]\n"
    "v_mfma_f32_16x16x32_bf16 v[104:107], v[220:223], v[188:191], v[104:107]\n"
    "v_mfma_f32_16x16x32_bf16 v[104:107], v[224:227], v[192:195], v[104:107]\n"
    "v_mfma_f32_16x16x32_bf16 v[76:79], v[228:231], v[188:191], v[76:79]\n"
    "v_mfma_f32_16x16x32_bf16 v[76:79], v[232:235], v[192:195], v[76:79]\n"
    "v_mfma_f32_16x16x32_bf16 v[112:115], v[236:239], v[188:191], v[112:115]\n"
    "v_mfma_f32_16x16x32_bf16 v[112:115], v[240:243], v[192:195], v[112:115]\n"
    "v_mfma_f32_16x16x32_bf16 v[120:123], v[212:215], v[196:199], v[120:123]\n"
    "v_mfma_f32_16x16x32_bf16 v[120:123], v[216:219], v[200:203], v[120:123]\n"
    "v_mfma_f32_16x16x32_bf16 v[84:87], v[220:223], v[196:199], v[84:87]\n"
    "v_mfma_f32_16x16x32_bf16 v[84:87], v[224:227], v[200:203], v[84:87]\n"
    "v_mfma_f32_16x16x32_bf16 v[28:31], v[228:231], v[196:199], v[28:31]\n"
    "v_mfma_f32_16x16x32_bf16 v[28:31], v[232:235], v[200:203], v[28:31]\n"
    "v_mfma_f32_16x16x32_bf16 v[92:95], v[236:239], v[196:199], v[92:95]\n"
    "v_mfma_f32_16x16x32_bf16 v[92:95], v[240:243], v[200:203], v[92:95]\n"
    "v_mfma_f32_16x16x32_bf16 v[100:103], v[212:215], v[204:207], v[100:103]\n"
    "v_mfma_f32_16x16x32_bf16 v[100:103], v[216:219], v[208:211], v[100:103]\n"
    "v_mfma_f32_16x16x32_bf16 v[108:111], v[220:223], v[204:207], v[108:111]\n"
    "v_mfma_f32_16x16x32_bf16 v[108:111], v[224:227], v[208:211], v[108:111]\n"
    "v_mfma_f32_16x16x32_bf16 v[116:119], v[228:231], v[204:207], v[116:119]\n"
    "v_mfma_f32_16x16x32_bf16 v[116:119], v[232:235], v[208:211], v[116:119]\n"
    "v_mfma_f32_16x16x32_bf16 v[52:55], v[236:239], v[204:207], v[52:55]\n"
    "v_mfma_f32_16x16x32_bf16 v[52:55], v[240:243], v[208:211], v[52:55]\n"
    "s_setprio 0\n"
    "s_waitcnt vmcnt(8)\n"
    "s_barrier\n"
    "; phase 124: step 84 slot 1 stage 3\n"
    "ds_read_b128 v[180:183], v244 offset:0\n"
    "ds_read_b128 v[184:187], v245 offset:0\n"
    "ds_read_b128 v[188:191], v244 offset:2048\n"
    "s_add_u32 m0, s50, 0x8000\n"
    "s_xor_b32 s82, s48, 0x202b00\n"
    "buffer_load_dwordx4 v250, s[36:39], s82 offen  lds\n"
    "ds_read_b128 v[192:195], v245 offset:2048\n"
    "ds_read_b128 v[196:199], v244 offset:8192\n"
    "ds_read_b128 v[200:203], v245 offset:8192\n"
    "s_add_u32 m0, s50, 0x9000\n"
    "s_xor_b32 s82, s48, 0x282b00\n"
    "buffer_load_dwordx4 v250, s[36:39], s82 offen  lds\n"
    "ds_read_b128 v[204:207], v244 offset:10240\n"
    "ds_read_b128 v[208:211], v245 offset:10240\n"
    "ds_read_b64_tr_b16 v[212:213], v246 offset:0\n"
    "s_add_u32 m0, s50, 0xa000\n"
    "s_xor_b32 s82, s48, 0x2b00\n"
    "buffer_load_dwordx4 v250, s[36:39], s82 offen  lds\n"
    "ds_read_b64_tr_b16 v[214:215], v246 offset:1024\n"
    "ds_read_b64_tr_b16 v[216:217], v246 offset:8192\n"
    "ds_read_b64_tr_b16 v[218:219], v246 offset:9216\n"
    "s_add_u32 m0, s50, 0xb000\n"
    "s_xor_b32 s82, s48, 0x82b00\n"
    "buffer_load_dwordx4 v250, s[36:39], s82 offen  lds\n"
    "ds_read_b64_tr_b16 v[220:221], v247 offset:0\n"
    "ds_read_b64_tr_b16 v[222:223], v247 offset:1024\n"
    "ds_read_b64_tr_b16 v[224:225], v247 offset:8192\n"
    "s_add_u32 m0, s51, 0xc000\n"
    "s_xor_b32 s82, s67, 0x5700000\n"
    "buffer_load_dwordx4 v251, s[40:43], s82 offen  lds\n"
    "ds_read_b64_tr_b16 v[226:227], v247 offset:9216\n"
    "ds_read_b64_tr_b16 v[228:229], v248 offset:0\n"
    "ds_read_b64_tr_b16 v[230:231], v248 offset:1024\n"
    "s_add_u32 m0, s51, 0xe000\n"
    "s_xor_b32 s82, s67, 0x5780000\n"
    "buffer_load_dwordx4 v251, s[40:43], s82 offen  lds\n"
    "ds_read_b64_tr_b16 v[232:233], v248 offset:8192\n"
    "ds_read_b64_tr_b16 v[234:235], v248 offset:9216\n"
    "ds_read_b64_tr_b16 v[236:237], v249 offset:0\n"
    "s_cmp_lg_u32 s52, 7\n"
    "s_cbranch_scc1 .La159_%=\n"
    "s_xor_b32 s82, s80, 0x4000\n"
    "buffer_store_dwordx4 v[72:75], v255, s[72:75], s82 offen sc0 sc1\n"
    ".La159_%=:\n"
    "ds_read_b64_tr_b16 v[238:239], v249 offset:1024\n"
    "ds_read_b64_tr_b16 v[240:241], v249 offset:8192\n"
    "ds_read_b64_tr_b16 v[242:243], v249 offset:9216\n"
    "s_waitcnt vmcnt(12) lgkmcnt(0)\n"
    "s_barrier\n"
    "s_setprio 1\n"
    "v_mfma_f32_16x16x32_bf16 v[64:67], v[212:215], v[180:183], v[64:67]\n"
    "v_mfma_f32_16x16x32_bf16 v[64:67], v[216:219], v[184:187], v[64:67]\n"
    "v_mfma_f32_16x16x32_bf16 v[60:63], v[220:223], v[180:183], v[60:63]\n"
    "v_mfma_f32_16x16x32_bf16 v[60:63], v[224:227], v[184:187], v[60:63]\n"
    "v_mfma_f32_16x16x32_bf16 v[80:83], v[228:231], v[180:183], v[80:83]\n"
    "v_mfma_f32_16x16x32_bf16 v[80:83], v[232:235], v[184:187], v[80:83]\n"
    "v_mfma_f32_16x16x32_bf16 v[88:91], v[236:239], v[180:183], v[88:91]\n"
    "v_mfma_f32_16x16x32_bf16 v[88:91], v[240:243], v[184:187], v[88:91]\n"
    "v_mfma_f32_16x16x32_bf16 v[96:99], v[212:215], v[188:191], v[96:99]\n"
    "v_mfma_f32_16x16x32_bf16 v[96:99], v[216:219], v[192:195], v[96:99]\n"
    "v_mfma_f32_16x16x32_bf16 v[104:107], v[220:223], v[188:191], v[104:107]\n"
    "v_mfma_f32_16x16x32_bf16 v[104:107], v[224:227], v[192:195], v[104:107]\n"
    "v_mfma_f32_16x16x32_bf16 v[76:79], v[228:231], v[188:191], v[76:79]\n"
    "v_mfma_f32_16x16x32_bf16 v[76:79], v[232:235], v[192:195], v[76:79]\n"
    "v_mfma_f32_16x16x32_bf16 v[112:115], v[236:239], v[188:191], v[112:115]\n"
    "v_mfma_f32_16x16x32_bf16 v[112:115], v[240:243], v[192:195], v[112:115]\n"
    "v_mfma_f32_16x16x32_bf16 v[120:123], v[212:215], v[196:199], v[120:123]\n"
    "v_mfma_f32_16x16x32_bf16 v[120:123], v[216:219], v[200:203], v[120:123]\n"
    "v_mfma_f32_16x16x32_bf16 v[84:87], v[220:223], v[196:199], v[84:87]\n"
    "v_mfma_f32_16x16x32_bf16 v[84:87], v[224:227], v[200:203], v[84:87]\n"
    "v_mfma_f32_16x16x32_bf16 v[28:31], v[228:231], v[196:199], v[28:31]\n"
    "v_mfma_f32_16x16x32_bf16 v[28:31], v[232:235], v[200:203], v[28:31]\n"
    "v_mfma_f32_16x16x32_bf16 v[92:95], v[236:239], v[196:199], v[92:95]\n"
    "v_mfma_f32_16x16x32_bf16 v[92:95], v[240:243], v[200:203], v[92:95]\n"
    "v_mfma_f32_16x16x32_bf16 v[100:103], v[212:215], v[204:207], v[100:103]\n"
    "v_mfma_f32_16x16x32_bf16 v[100:103], v[216:219], v[208:211], v[100:103]\n"
    "v_mfma_f32_16x16x32_bf16 v[108:111], v[220:223], v[204:207], v[108:111]\n"
    "v_mfma_f32_16x16x32_bf16 v[108:111], v[224:227], v[208:211], v[108:111]\n"
    "v_mfma_f32_16x16x32_bf16 v[116:119], v[228:231], v[204:207], v[116:119]\n"
    "v_mfma_f32_16x16x32_bf16 v[116:119], v[232:235], v[208:211], v[116:119]\n"
    "v_mfma_f32_16x16x32_bf16 v[52:55], v[236:239], v[204:207], v[52:55]\n"
    "v_mfma_f32_16x16x32_bf16 v[52:55], v[240:243], v[208:211], v[52:55]\n"
    "s_setprio 0\n"
    "s_waitcnt vmcnt(8)\n"
    "s_barrier\n"
    "; phase 125: step 85 slot 1 stage 3\n"
    "ds_read_b128 v[180:183], v244 offset:16384\n"
    "ds_read_b128 v[184:187], v245 offset:16384\n"
    "ds_read_b128 v[188:191], v244 offset:18432\n"
    "ds_read_b128 v[192:195], v245 offset:18432\n"
    "s_add_u32 m0, s50, 0x0\n"
    "s_xor_b32 s82, s48, 0x202b80\n"
    "buffer_load_dwordx4 v250, s[36:39], s82 offen  lds\n"
    "ds_read_b128 v[196:199], v244 offset:24576\n"
    "ds_read_b128 v[200:203], v245 offset:24576\n"
    "ds_read_b128 v[204:207], v244 offset:26624\n"
    "ds_read_b128 v[208:211], v245 offset:26624\n"
    "s_add_u32 m0, s50, 0x1000\n"
    "s_xor_b32 s82, s48, 0x282b80\n"
    "buffer_load_dwordx4 v250, s[36:39], s82 offen  lds\n"
    "ds_read_b64_tr_b16 v[212:213], v246 offset:16384\n"
    "ds_read_b64_tr_b16 v[214:215], v246 offset:17408\n"
    "ds_read_b64_tr_b16 v[216:217], v246 offset:24576\n"
    "ds_read_b64_tr_b16 v[218:219], v246 offset:25600\n"
    "s_add_u32 m0, s50, 0x2000\n"
    "s_xor_b32 s82, s48, 0x2b80\n"
    "buffer_load_dwordx4 v250, s[36:39], s82 offen  lds\n"
    "ds_read_b64_tr_b16 v[220:221], v247 offset:16384\n"
    "ds_read_b64_tr_b16 v[222:223], v247 offset:17408\n"
    "ds_read_b64_tr_b16 v[224:225], v247 offset:24576\n"
    "ds_read_b64_tr_b16 v[226:227], v247 offset:25600\n"
    "s_add_u32 m0, s50, 0x3000\n"
    "s_xor_b32 s82, s48, 0x82b80\n"
    "buffer_load_dwordx4 v250, s[36:39], s82 offen  lds\n"
    "ds_read_b64_tr_b16 v[228:229], v248 offset:16384\n"
    "ds_read_b64_tr_b16 v[230:231], v248 offset:17408\n"
    "ds_read_b64_tr_b16 v[232:233], v248 offset:24576\n"
    "ds_read_b64_tr_b16 v[234:235], v248 offset:25600\n"
    "ds_read_b64_tr_b16 v[236:237], v249 offset:16384\n"
    "ds_read_b64_tr_b16 v[238:239], v249 offset:17408\n"
    "ds_read_b64_tr_b16 v[240:241], v249 offset:24576\n"
    "ds_read_b64_tr_b16 v[242:243], v249 offset:25600\n"
    "s_waitcnt vmcnt(10) lgkmcnt(0)\n"
    "s_barrier\n"
    "s_setprio 1\n"
    "v_mfma_f32_16x16x32_bf16 v[64:67], v[212:215], v[180:183], v[64:67]\n"
    "v_mfma_f32_16x16x32_bf16 v[64:67], v[216:219], v[184:187], v[64:67]\n"
    "v_mfma_f32_16x16x32_bf16 v[60:63], v[220:223], v[180:183], v[60:63]\n"
    "v_mfma_f32_16x16x32_bf16 v[60:63], v[224:227], v[184:187], v[60:63]\n"
    "v_mfma_f32_16x16x32_bf16 v[80:83], v[228:231], v[180:183], v[80:83]\n"
    "v_mfma_f32_16x16x32_bf16 v[80:83], v[232:235], v[184:187], v[80:83]\n"
    "v_mfma_f32_16x16x32_bf16 v[88:91], v[236:239], v[180:183], v[88:91]\n"
    "v_mfma_f32_16x16x32_bf16 v[88:91], v[240:243], v[184:187], v[88:91]\n"
    "v_mfma_f32_16x16x32_bf16 v[96:99], v[212:215], v[188:191], v[96:99]\n"
    "v_mfma_f32_16x16x32_bf16 v[96:99], v[216:219], v[192:195], v[96:99]\n"
    "v_mfma_f32_16x16x32_bf16 v[104:107], v[220:223], v[188:191], v[104:107]\n"
    "v_mfma_f32_16x16x32_bf16 v[104:107], v[224:227], v[192:195], v[104:107]\n"
    "v_mfma_f32_16x16x32_bf16 v[76:79], v[228:231], v[188:191], v[76:79]\n"
    "v_mfma_f32_16x16x32_bf16 v[76:79], v[232:235], v[192:195], v[76:79]\n"
    "v_mfma_f32_16x16x32_bf16 v[112:115], v[236:239], v[188:191], v[112:115]\n"
    "v_mfma_f32_16x16x32_bf16 v[112:115], v[240:243], v[192:195], v[112:115]\n"
    "v_mfma_f32_16x16x32_bf16 v[120:123], v[212:215], v[196:199], v[120:123]\n"
    "v_mfma_f32_16x16x32_bf16 v[120:123], v[216:219], v[200:203], v[120:123]\n"
    "v_mfma_f32_16x16x32_bf16 v[84:87], v[220:223], v[196:199], v[84:87]\n"
    "v_mfma_f32_16x16x32_bf16 v[84:87], v[224:227], v[200:203], v[84:87]\n"
    "v_mfma_f32_16x16x32_bf16 v[28:31], v[228:231], v[196:199], v[28:31]\n"
    "v_mfma_f32_16x16x32_bf16 v[28:31], v[232:235], v[200:203], v[28:31]\n"
    "v_mfma_f32_16x16x32_bf16 v[92:95], v[236:239], v[196:199], v[92:95]\n"
    "v_mfma_f32_16x16x32_bf16 v[92:95], v[240:243], v[200:203], v[92:95]\n"
    "v_mfma_f32_16x16x32_bf16 v[100:103], v[212:215], v[204:207], v[100:103]\n"
    "v_mfma_f32_16x16x32_bf16 v[100:103], v[216:219], v[208:211], v[100:103]\n"
    "v_mfma_f32_16x16x32_bf16 v[108:111], v[220:223], v[204:207], v[108:111]\n"
    "v_mfma_f32_16x16x32_bf16 v[108:111], v[224:227], v[208:211], v[108:111]\n"
    "v_mfma_f32_16x16x32_bf16 v[116:119], v[228:231], v[204:207], v[116:119]\n"
    "v_mfma_f32_16x16x32_bf16 v[116:119], v[232:235], v[208:211], v[116:119]\n"
    "v_mfma_f32_16x16x32_bf16 v[52:55], v[236:239], v[204:207], v[52:55]\n"
    "v_mfma_f32_16x16x32_bf16 v[52:55], v[240:243], v[208:211], v[52:55]\n"
    "s_setprio 0\n"
    "s_waitcnt vmcnt(6)\n"
    "s_barrier\n"
    "; phase 126: step 86 slot 1 stage 3\n"
    "ds_read_b128 v[180:183], v244 offset:32768\n"
    "ds_read_b128 v[184:187], v245 offset:32768\n"
    "ds_read_b128 v[188:191], v244 offset:34816\n"
    "ds_read_b128 v[192:195], v245 offset:34816\n"
    "s_cmp_lg_u32 s58, 0\n"
    "s_cbranch_scc1 .La160_%=\n"
    "v_mov_b32 v128, 1\n"
    "s_mov_b64 exec, 1\n"
    "buffer_atomic_add v128, off, s[44:47], s55 sc0 sc1\n"
    "s_mov_b64 exec, -1\n"
    ".La160_%=:\n"
    "ds_read_b128 v[196:199], v244 offset:40960\n"
    "ds_read_b128 v[200:203], v245 offset:40960\n"
    "ds_read_b128 v[204:207], v244 offset:43008\n"
    "ds_read_b128 v[208:211], v245 offset:43008\n"
    "ds_read_b64_tr_b16 v[212:213], v246 offset:32768\n"
    "ds_read_b64_tr_b16 v[214:215], v246 offset:33792\n"
    "ds_read_b64_tr_b16 v[216:217], v246 offset:40960\n"
    "ds_read_b64_tr_b16 v[218:219], v246 offset:41984\n"
    "ds_read_b64_tr_b16 v[220:221], v247 offset:32768\n"
    "ds_read_b64_tr_b16 v[222:223], v247 offset:33792\n"
    "ds_read_b64_tr_b16 v[224:225], v247 offset:40960\n"
    "ds_read_b64_tr_b16 v[226:227], v247 offset:41984\n"
    "ds_read_b64_tr_b16 v[228:229], v248 offset:32768\n"
    "ds_read_b64_tr_b16 v[230:231], v248 offset:33792\n"
    "ds_read_b64_tr_b16 v[232:233], v248 offset:40960\n"
    "ds_read_b64_tr_b16 v[234:235], v248 offset:41984\n"
    "ds_read_b64_tr_b16 v[236:237], v249 offset:32768\n"
    "ds_read_b64_tr_b16 v[238:239], v249 offset:33792\n"
    "ds_read_b64_tr_b16 v[240:241], v249 offset:40960\n"
    "ds_read_b64_tr_b16 v[242:243], v249 offset:41984\n"
    "s_waitcnt vmcnt(4) lgkmcnt(0)\n"
    "s_barrier\n"
    "s_setprio 1\n"
    "v_mfma_f32_16x16x32_bf16 v[64:67], v[212:215], v[180:183], v[64:67]\n"
    "v_mfma_f32_16x16x32_bf16 v[64:67], v[216:219], v[184:187], v[64:67]\n"
    "v_mfma_f32_16x16x32_bf16 v[60:63], v[220:223], v[180:183], v[60:63]\n"
    "v_mfma_f32_16x16x32_bf16 v[60:63], v[224:227], v[184:187], v[60:63]\n"
    "v_mfma_f32_16x16x32_bf16 v[80:83], v[228:231], v[180:183], v[80:83]\n"
    "v_mfma_f32_16x16x32_bf16 v[80:83], v[232:235], v[184:187], v[80:83]\n"
    "v_mfma_f32_16x16x32_bf16 v[88:91], v[236:239], v[180:183], v[88:91]\n"
    "v_mfma_f32_16x16x32_bf16 v[88:91], v[240:243], v[184:187], v[88:91]\n"
    "v_mfma_f32_16x16x32_bf16 v[96:99], v[212:215], v[188:191], v[96:99]\n"
    "v_mfma_f32_16x16x32_bf16 v[96:99], v[216:219], v[192:195], v[96:99]\n"
    "v_mfma_f32_16x16x32_bf16 v[104:107], v[220:223], v[188:191], v[104:107]\n"
    "v_mfma_f32_16x16x32_bf16 v[104:107], v[224:227], v[192:195], v[104:107]\n"
    "v_mfma_f32_16x16x32_bf16 v[76:79], v[228:231], v[188:191], v[76:79]\n"
    "v_mfma_f32_16x16x32_bf16 v[76:79], v[232:235], v[192:195], v[76:79]\n"
    "v_mfma_f32_16x16x32_bf16 v[112:115], v[236:239], v[188:191], v[112:115]\n"
    "v_mfma_f32_16x16x32_bf16 v[112:115], v[240:243], v[192:195], v[112:115]\n"
    "v_mfma_f32_16x16x32_bf16 v[120:123], v[212:215], v[196:199], v[120:123]\n"
    "v_mfma_f32_16x16x32_bf16 v[120:123], v[216:219], v[200:203], v[120:123]\n"
    "v_mfma_f32_16x16x32_bf16 v[84:87], v[220:223], v[196:199], v[84:87]\n"
    "v_mfma_f32_16x16x32_bf16 v[84:87], v[224:227], v[200:203], v[84:87]\n"
    "v_mfma_f32_16x16x32_bf16 v[28:31], v[228:231], v[196:199], v[28:31]\n"
    "v_mfma_f32_16x16x32_bf16 v[28:31], v[232:235], v[200:203], v[28:31]\n"
    "v_mfma_f32_16x16x32_bf16 v[92:95], v[236:239], v[196:199], v[92:95]\n"
    "v_mfma_f32_16x16x32_bf16 v[92:95], v[240:243], v[200:203], v[92:95]\n"
    "v_mfma_f32_16x16x32_bf16 v[100:103], v[212:215], v[204:207], v[100:103]\n"
    "v_mfma_f32_16x16x32_bf16 v[100:103], v[216:219], v[208:211], v[100:103]\n"
    "v_mfma_f32_16x16x32_bf16 v[108:111], v[220:223], v[204:207], v[108:111]\n"
    "v_mfma_f32_16x16x32_bf16 v[108:111], v[224:227], v[208:211], v[108:111]\n"
    "v_mfma_f32_16x16x32_bf16 v[116:119], v[228:231], v[204:207], v[116:119]\n"
    "v_mfma_f32_16x16x32_bf16 v[116:119], v[232:235], v[208:211], v[116:119]\n"
    "v_mfma_f32_16x16x32_bf16 v[52:55], v[236:239], v[204:207], v[52:55]\n"
    "v_mfma_f32_16x16x32_bf16 v[52:55], v[240:243], v[208:211], v[52:55]\n"
    "s_setprio 0\n"
    "s_waitcnt vmcnt(0)\n"
    "s_barrier\n"
    "; phase 127: step 87 slot 1 stage 3\n"
    "ds_read_b128 v[180:183], v244 offset:0\n"
    "ds_read_b128 v[184:187], v245 offset:0\n"
    "ds_read_b128 v[188:191], v244 offset:2048\n"
    "ds_read_b128 v[192:195], v245 offset:2048\n"
    "ds_read_b128 v[196:199], v244 offset:8192\n"
    "ds_read_b128 v[200:203], v245 offset:8192\n"
    "ds_read_b128 v[204:207], v244 offset:10240\n"
    "ds_read_b128 v[208:211], v245 offset:10240\n"
    "ds_read_b64_tr_b16 v[212:213], v246 offset:49152\n"
    "ds_read_b64_tr_b16 v[214:215], v246 offset:50176\n"
    "ds_read_b64_tr_b16 v[216:217], v246 offset:57344\n"
    "ds_read_b64_tr_b16 v[218:219], v246 offset:58368\n"
    "ds_read_b64_tr_b16 v[220:221], v247 offset:49152\n"
    "ds_read_b64_tr_b16 v[222:223], v247 offset:50176\n"
    "ds_read_b64_tr_b16 v[224:225], v247 offset:57344\n"
    "ds_read_b64_tr_b16 v[226:227], v247 offset:58368\n"
    "ds_read_b64_tr_b16 v[228:229], v248 offset:49152\n"
    "ds_read_b64_tr_b16 v[230:231], v248 offset:50176\n"
    "ds_read_b64_tr_b16 v[232:233], v248 offset:57344\n"
    "ds_read_b64_tr_b16 v[234:235], v248 offset:58368\n"
    "ds_read_b64_tr_b16 v[236:237], v249 offset:49152\n"
    "ds_read_b64_tr_b16 v[238:239], v249 offset:50176\n"
    "ds_read_b64_tr_b16 v[240:241], v249 offset:57344\n"
    "ds_read_b64_tr_b16 v[242:243], v249 offset:58368\n"
    "s_waitcnt lgkmcnt(0)\n"
    "s_barrier\n"
    "s_setprio 1\n"
    "v_mfma_f32_16x16x32_bf16 v[64:67], v[212:215], v[180:183], v[64:67]\n"
    "v_mfma_f32_16x16x32_bf16 v[64:67], v[216:219], v[184:187], v[64:67]\n"
    "v_mfma_f32_16x16x32_bf16 v[60:63], v[220:223], v[180:183], v[60:63]\n"
    "v_mfma_f32_16x16x32_bf16 v[60:63], v[224:227], v[184:187], v[60:63]\n"
    "v_mfma_f32_16x16x32_bf16 v[80:83], v[228:231], v[180:183], v[80:83]\n"
    "v_mfma_f32_16x16x32_bf16 v[80:83], v[232:235], v[184:187], v[80:83]\n"
    "v_mfma_f32_16x16x32_bf16 v[88:91], v[236:239], v[180:183], v[88:91]\n"
    "v_mfma_f32_16x16x32_bf16 v[88:91], v[240:243], v[184:187], v[88:91]\n"
    "v_mfma_f32_16x16x32_bf16 v[96:99], v[212:215], v[188:191], v[96:99]\n"
    "v_mfma_f32_16x16x32_bf16 v[96:99], v[216:219], v[192:195], v[96:99]\n"
    "v_mfma_f32_16x16x32_bf16 v[104:107], v[220:223], v[188:191], v[104:107]\n"
    "v_mfma_f32_16x16x32_bf16 v[104:107], v[224:227], v[192:195], v[104:107]\n"
    "v_mfma_f32_16x16x32_bf16 v[76:79], v[228:231], v[188:191], v[76:79]\n"
    "v_mfma_f32_16x16x32_bf16 v[76:79], v[232:235], v[192:195], v[76:79]\n"
    "v_mfma_f32_16x16x32_bf16 v[112:115], v[236:239], v[188:191], v[112:115]\n"
    "v_mfma_f32_16x16x32_bf16 v[112:115], v[240:243], v[192:195], v[112:115]\n"
    "v_mfma_f32_16x16x32_bf16 v[120:123], v[212:215], v[196:199], v[120:123]\n"
    "v_mfma_f32_16x16x32_bf16 v[120:123], v[216:219], v[200:203], v[120:123]\n"
    "v_mfma_f32_16x16x32_bf16 v[84:87], v[220:223], v[196:199], v[84:87]\n"
    "v_mfma_f32_16x16x32_bf16 v[84:87], v[224:227], v[200:203], v[84:87]\n"
    "v_mfma_f32_16x16x32_bf16 v[28:31], v[228:231], v[196:199], v[28:31]\n"
    "v_mfma_f32_16x16x32_bf16 v[28:31], v[232:235], v[200:203], v[28:31]\n"
    "v_mfma_f32_16x16x32_bf16 v[92:95], v[236:239], v[196:199], v[92:95]\n"
    "v_mfma_f32_16x16x32_bf16 v[92:95], v[240:243], v[200:203], v[92:95]\n"
    "v_cvt_pk_bf16_f32 v64, v64, v65\n"
    "v_cvt_pk_bf16_f32 v65, v66, v67\n"
    "v_mfma_f32_16x16x32_bf16 v[100:103], v[212:215], v[204:207], v[100:103]\n"
    "v_cvt_pk_bf16_f32 v66, v60, v61\n"
    "v_cvt_pk_bf16_f32 v67, v62, v63\n"
    "v_mfma_f32_16x16x32_bf16 v[100:103], v[216:219], v[208:211], v[100:103]\n"
    "v_cvt_pk_bf16_f32 v80, v80, v81\n"
    "v_cvt_pk_bf16_f32 v81, v82, v83\n"
    "v_mfma_f32_16x16x32_bf16 v[108:111], v[220:223], v[204:207], v[108:111]\n"
    "v_cvt_pk_bf16_f32 v82, v88, v89\n"
    "v_cvt_pk_bf16_f32 v83, v90, v91\n"
    "v_mfma_f32_16x16x32_bf16 v[108:111], v[224:227], v[208:211], v[108:111]\n"
    "v_cvt_pk_bf16_f32 v96, v96, v97\n"
    "v_cvt_pk_bf16_f32 v97, v98, v99\n"
    "v_mfma_f32_16x16x32_bf16 v[116:119], v[228:231], v[204:207], v[116:119]\n"
    "v_cvt_pk_bf16_f32 v98, v104, v105\n"
    "v_cvt_pk_bf16_f32 v99, v106, v107\n"
    "v_mfma_f32_16x16x32_bf16 v[116:119], v[232:235], v[208:211], v[116:119]\n"
    "v_cvt_pk_bf16_f32 v76, v76, v77\n"
    "v_cvt_pk_bf16_f32 v77, v78, v79\n"
    "v_mfma_f32_16x16x32_bf16 v[52:55], v[236:239], v[204:207], v[52:55]\n"
    "v_cvt_pk_bf16_f32 v78, v112, v113\n"
    "v_cvt_pk_bf16_f32 v79, v114, v115\n"
    "v_mfma_f32_16x16x32_bf16 v[52:55], v[240:243], v[208:211], v[52:55]\n"
    "s_setprio 0\n"
    "s_cmp_lg_u32 s59, 0\n"
    "s_cbranch_scc1 .La161_%=\n"
    "s_barrier\n"
    ".La161_%=:\n"
    "; tail\n"
    "v_cvt_pk_bf16_f32 v120, v120, v121\n"
    "v_cvt_pk_bf16_f32 v121, v122, v123\n"
    "v_cvt_pk_bf16_f32 v122, v84, v85\n"
    "v_cvt_pk_bf16_f32 v123, v86, v87\n"
    "v_cvt_pk_bf16_f32 v28, v28, v29\n"
    "v_cvt_pk_bf16_f32 v29, v30, v31\n"
    "v_cvt_pk_bf16_f32 v30, v92, v93\n"
    "v_cvt_pk_bf16_f32 v31, v94, v95\n"
    "s_xor_b32 s82, s54, 0x6e000\n"
    "s_add_u32 s82, s82, 0x1100000\n"
    "buffer_store_dwordx4 v[64:67], v252, s[44:47], s82 offen sc0 sc1\n"
    "s_xor_b32 s82, s54, 0x6c000\n"
    "s_add_u32 s82, s82, 0x1100000\n"
    "buffer_store_dwordx4 v[80:83], v252, s[44:47], s82 offen sc0 sc1\n"
    "s_xor_b32 s82, s54, 0x4a000\n"
    "s_add_u32 s82, s82, 0x1100000\n"
    "buffer_store_dwordx4 v[96:99], v252, s[44:47], s82 offen sc0 sc1\n"
    "s_xor_b32 s82, s54, 0x48000\n"
    "s_add_u32 s82, s82, 0x1100000\n"
    "buffer_store_dwordx4 v[76:79], v252, s[44:47], s82 offen sc0 sc1\n"
    "s_xor_b32 s82, s54, 0x26000\n"
    "s_add_u32 s82, s82, 0x1100000\n"
    "buffer_store_dwordx4 v[120:123], v252, s[44:47], s82 offen sc0 sc1\n"
    "s_xor_b32 s82, s54, 0x24000\n"
    "s_add_u32 s82, s82, 0x1100000\n"
    "buffer_store_dwordx4 v[28:31], v252, s[44:47], s82 offen sc0 sc1\n"
    "s_waitcnt vmcnt(0) lgkmcnt(0)\n"
    "s_mov_b32 s82, 0x2400000\n"
    "buffer_store_dwordx2 v[10:11], v253, s[44:47], s82 offen sc0 sc1\n"
    "v_readfirstlane_b32 s91, v128\n"
    "s_cmp_lg_u32 s58, 0\n"
    "s_cbranch_scc1 .La162_%=\n"
    "s_and_b32 s82, s91, 31\n"
    "s_cmp_lg_u32 s82, 31\n"
    "s_cbranch_scc1 .La162_%=\n"
    "buffer_wbl2 sc0 sc1\n"
    "s_branch .La163_%=\n"
    ".La162_%=:\n"
    "s_mov_b32 s82, 0x40000000\n"
    "buffer_store_dword v7, off, s[44:47], s82\n"
    ".La163_%=:\n"
    "s_add_u32 s82, s56, 0x300000\n"
    "buffer_load_dword v6, v254, s[44:47], s82 offen sc0 sc1\n"
    "s_lshr_b32 s83, s67, 8\n"
    "s_and_b32 s82, s83, 0x38\n"
    "s_load_dwordx2 s[72:73], s[60:61], s82\n"
    "s_load_dwordx2 s[22:23], s[60:61], s82 offset:0x40\n"
    "s_and_b32 s83, s67, 0x700\n"
    "s_add_u32 s80, s57, s83\n"
    "s_waitcnt vmcnt(0) lgkmcnt(0)\n"
    "v_cmp_ne_u32 vcc, s90, v6\n"
    "s_cbranch_vccz .La164_%=\n"
    "s_memrealtime s[86:87]\n"
    "s_add_u32 s82, s56, 0x300000\n"
    ".La165_%=:\n"
    "s_sleep 2\n"
    "buffer_load_dword v6, v254, s[44:47], s82 offen sc0 sc1\n"
    "s_memrealtime s[88:89]\n"
    "s_waitcnt vmcnt(0) lgkmcnt(0)\n"
    "v_cmp_ne_u32 vcc, s90, v6\n"
    "s_cbranch_vccz .La164_%=\n"
    "s_sub_u32 s83, s88, s86\n"
    "s_cmp_lt_u32 s83, 0x2710\n"
    "s_cbranch_scc1 .La165_%=\n"
    "s_sleep 64\n"
    "buffer_load_dword v7, off, s[44:47], 0 sc0 sc1\n"
    "s_waitcnt vmcnt(0)\n"
    "v_cmp_eq_u32 vcc, s90, v7\n"
    "s_cbranch_vccnz .La166_%=\n"
    "s_cmp_lt_u32 s83, 0x2faf080\n"
    "s_cbranch_scc1 .La165_%=\n"
    ".La166_%=:\n"
    "v_mov_b32 v7, s90\n"
    "v_mov_b32 v100, 0x7fc00000\n"
    "v_mov_b32 v101, 0x7fc00000\n"
    "v_mov_b32 v102, 0x7fc00000\n"
    "v_mov_b32 v103, 0x7fc00000\n"
    "v_mov_b32 v108, 0x7fc00000\n"
    "v_mov_b32 v109, 0x7fc00000\n"
    "v_mov_b32 v110, 0x7fc00000\n"
    "v_mov_b32 v111, 0x7fc00000\n"
    "v_mov_b32 v116, 0x7fc00000\n"
    "v_mov_b32 v117, 0x7fc00000\n"
    "v_mov_b32 v118, 0x7fc00000\n"
    "v_mov_b32 v119, 0x7fc00000\n"
    "v_mov_b32 v52, 0x7fc00000\n"
    "v_mov_b32 v53, 0x7fc00000\n"
    "v_mov_b32 v54, 0x7fc00000\n"
    "v_mov_b32 v55, 0x7fc00000\n"
    "buffer_store_dword v7, off, s[44:47], 0 sc0 sc1\n"
    "s_waitcnt vmcnt(0)\n"
    ".La164_%=:\n"
    "s_add_u32 s82, s54, 0x1104000\n"
    "buffer_load_dwordx4 v[124:127], v252, s[44:47], s82 offen sc0 sc1\n"
    "s_add_u32 s82, s54, 0x1108000\n"
    "buffer_load_dwordx4 v[36:39], v252, s[44:47], s82 offen sc0 sc1\n"
    "s_add_u32 s82, s54, 0x110c000\n"
    "buffer_load_dwordx4 v[144:147], v252, s[44:47], s82 offen sc0 sc1\n"
    "s_add_u32 s82, s54, 0x1106000\n"
    "buffer_load_dwordx4 v[152:155], v252, s[44:47], s82 offen sc0 sc1\n"
    "s_add_u32 s82, s54, 0x110a000\n"
    "buffer_load_dwordx4 v[160:163], v252, s[44:47], s82 offen sc0 sc1\n"
    "s_add_u32 s82, s54, 0x110e000\n"
    "buffer_load_dwordx4 v[168:171], v252, s[44:47], s82 offen sc0 sc1\n"
    "s_cmp_lg_u32 s58, 0\n"
    "s_cbranch_scc1 .La167_%=\n"
    "s_and_b32 s82, s91, 31\n"
    "s_cmp_lg_u32 s82, 31\n"
    "s_cbranch_scc1 .La167_%=\n"
    "v_add_u32 v128, 1, v128\n"
    "s_mov_b64 exec, 1\n"
    "buffer_atomic_umax v128, off, s[44:47], s55 offset:128 sc1\n"
    "s_mov_b64 exec, -1\n"
    ".La167_%=:\n"
    "s_waitcnt vmcnt(5)\n"
    "v_mfma_f32_16x16x32_bf16 v[116:119], v[2:5], v[124:127], v[116:119]\n"
    "v_mfma_f32_16x16x32_bf16 v[52:55], v[0:3], v[124:127], v[52:55]\n"
    "s_waitcnt vmcnt(4)\n"
    "v_mfma_f32_16x16x32_bf16 v[116:119], v[2:5], v[36:39], v[116:119]\n"
    "v_mfma_f32_16x16x32_bf16 v[52:55], v[0:3], v[36:39], v[52:55]\n"
    "s_waitcnt vmcnt(3)\n"
    "v_mfma_f32_16x16x32_bf16 v[116:119], v[2:5], v[144:147], v[116:119]\n"
    "v_mfma_f32_16x16x32_bf16 v[52:55], v[0:3], v[144:147], v[52:55]\n"
    "s_nop 6\n"
    "v_med3_f32 v144, v116, -s62, s62\n"
    "v_med3_f32 v145, v117, -s62, s62\n"
    "v_med3_f32 v146, v118, -s62, s62\n"
    "v_med3_f32 v147, v119, -s62, s62\n"
    "v_mul_f32 v144, s63, v144\n"
    "v_mul_f32 v145, s63, v145\n"
    "v_mul_f32 v146, s63, v146\n"
    "v_mul_f32 v147, s63, v147\n"
    "v_exp_f32 v144, v144\n"
    "v_exp_f32 v145, v145\n"
    "v_exp_f32 v146, v146\n"
    "v_exp_f32 v147, v147\n"
    "v_add_f32 v144, 1.0, v144\n"
    "v_add_f32 v145, 1.0, v145\n"
    "v_add_f32 v146, 1.0, v146\n"
    "v_add_f32 v147, 1.0, v147\n"
    "v_rcp_f32 v144, v144\n"
    "v_rcp_f32 v145, v145\n"
    "v_rcp_f32 v146, v146\n"
    "v_rcp_f32 v147, v147\n"
    "v_mul_f32 v116, v116, v144\n"
    "v_mul_f32 v117, v117, v145\n"
    "v_mul_f32 v118, v118, v146\n"
    "v_mul_f32 v119, v119, v147\n"
    "v_med3_f32 v144, v52, -s62, s62\n"
    "v_med3_f32 v145, v53, -s62, s62\n"
    "v_med3_f32 v146, v54, -s62, s62\n"
    "v_med3_f32 v147, v55, -s62, s62\n"
    "v_mul_f32 v144, s63, v144\n"
    "v_mul_f32 v145, s63, v145\n"
    "v_mul_f32 v146, s63, v146\n"
    "v_mul_f32 v147, s63, v147\n"
    "v_exp_f32 v144, v144\n"
    "v_exp_f32 v145, v145\n"
    "v_exp_f32 v146, v146\n"
    "v_exp_f32 v147, v147\n"
    "v_add_f32 v144, 1.0, v144\n"
    "v_add_f32 v145, 1.0, v145\n"
    "v_add_f32 v146, 1.0, v146\n"
    "v_add_f32 v147, 1.0, v147\n"
    "v_rcp_f32 v144, v144\n"
    "v_rcp_f32 v145, v145\n"
    "v_rcp_f32 v146, v146\n"
    "v_rcp_f32 v147, v147\n"
    "v_mul_f32 v52, v52, v144\n"
    "v_mul_f32 v53, v53, v145\n"
    "v_mul_f32 v54, v54, v146\n"
    "v_mul_f32 v55, v55, v147\n"
    "v_cvt_pk_bf16_f32 v144, v116, v117\n"
    "v_cvt_pk_bf16_f32 v145, v118, v119\n"
    "v_cvt_pk_bf16_f32 v146, v52, v53\n"
    "v_cvt_pk_bf16_f32 v147, v54, v55\n"
    "s_nop 0\n"
    "v_permlane16_swap_b32 v144, v146\n"
    "v_permlane16_swap_b32 v145, v147\n"
    "s_waitcnt vmcnt(2)\n"
    "v_mfma_f32_16x16x32_bf16 v[100:103], v[2:5], v[152:155], v[100:103]\n"
    "v_mfma_f32_16x16x32_bf16 v[108:111], v[0:3], v[152:155], v[108:111]\n"
    "s_waitcnt vmcnt(1)\n"
    "v_mfma_f32_16x16x32_bf16 v[100:103], v[2:5], v[160:163], v[100:103]\n"
    "v_mfma_f32_16x16x32_bf16 v[108:111], v[0:3], v[160:163], v[108:111]\n"
    "s_waitcnt vmcnt(0)\n"
    "v_mfma_f32_16x16x32_bf16 v[100:103], v[2:5], v[168:171], v[100:103]\n"
    "v_mfma_f32_16x16x32_bf16 v[108:111], v[0:3], v[168:171], v[108:111]\n"
    "s_nop 6\n"
    "v_med3_f32 v168, v100, -s62, s62\n"
    "v_med3_f32 v169, v101, -s62, s62\n"
    "v_med3_f32 v170, v102, -s62, s62\n"
    "v_med3_f32 v171, v103, -s62, s62\n"
    "v_mul_f32 v168, s63, v168\n"
    "v_mul_f32 v169, s63, v169\n"
    "v_mul_f32 v170, s63, v170\n"
    "v_mul_f32 v171, s63, v171\n"
    "v_exp_f32 v168, v168\n"
    "v_exp_f32 v169, v169\n"
    "v_exp_f32 v170, v170\n"
    "v_exp_f32 v171, v171\n"
    "v_add_f32 v168, 1.0, v168\n"
    "v_add_f32 v169, 1.0, v169\n"
    "v_add_f32 v170, 1.0, v170\n"
    "v_add_f32 v171, 1.0, v171\n"
    "v_rcp_f32 v168, v168\n"
    "v_rcp_f32 v169, v169\n"
    "v_rcp_f32 v170, v170\n"
    "v_rcp_f32 v171, v171\n"
    "v_mul_f32 v100, v100, v168\n"
    "v_mul_f32 v101, v101, v169\n"
    "v_mul_f32 v102, v102, v170\n"
    "v_mul_f32 v103, v103, v171\n"
    "v_med3_f32 v168, v108, -s62, s62\n"
    "v_med3_f32 v169, v109, -s62, s62\n"
    "v_med3_f32 v170, v110, -s62, s62\n"
    "v_med3_f32 v171, v111, -s62, s62\n"
    "v_mul_f32 v168, s63, v168\n"
    "v_mul_f32 v169, s63, v169\n"
    "v_mul_f32 v170, s63, v170\n"
    "v_mul_f32 v171, s63, v171\n"
    "v_exp_f32 v168, v168\n"
    "v_exp_f32 v169, v169\n"
    "v_exp_f32 v170, v170\n"
    "v_exp_f32 v171, v171\n"
    "v_add_f32 v168, 1.0, v168\n"
    "v_add_f32 v169, 1.0, v169\n"
    "v_add_f32 v170, 1.0, v170\n"
    "v_add_f32 v171, 1.0, v171\n"
    "v_rcp_f32 v168, v168\n"
    "v_rcp_f32 v169, v169\n"
    "v_rcp_f32 v170, v170\n"
    "v_rcp_f32 v171, v171\n"
    "v_mul_f32 v108, v108, v168\n"
    "v_mul_f32 v109, v109, v169\n"
    "v_mul_f32 v110, v110, v170\n"
    "v_mul_f32 v111, v111, v171\n"
    "v_cvt_pk_bf16_f32 v168, v100, v101\n"
    "v_cvt_pk_bf16_f32 v169, v102, v103\n"
    "v_cvt_pk_bf16_f32 v170, v108, v109\n"
    "v_cvt_pk_bf16_f32 v171, v110, v111\n"
    "s_nop 0\n"
    "v_permlane16_swap_b32 v168, v170\n"
    "v_permlane16_swap_b32 v169, v171\n"
    "v_mov_b32 v100, v144\n"
    "v_mov_b32 v101, v145\n"
    "v_mov_b32 v102, v146\n"
    "v_mov_b32 v103, v147\n"
    "v_mov_b32_dpp v144, v168 row_ror:8 row_mask:0xf bank_mask:0xc\n"
    "v_mov_b32_dpp v145, v169 row_ror:8 row_mask:0xf bank_mask:0xc\n"
    "v_mov_b32_dpp v146, v170 row_ror:8 row_mask:0xf bank_mask:0xc\n"
    "v_mov_b32_dpp v147, v171 row_ror:8 row_mask:0xf bank_mask:0xc\n"
    "v_mov_b32_dpp v168, v100 row_ror:8 row_mask:0xf bank_mask:0x3\n"
    "v_mov_b32_dpp v169, v101 row_ror:8 row_mask:0xf bank_mask:0x3\n"
    "v_mov_b32_dpp v170, v102 row_ror:8 row_mask:0xf bank_mask:0x3\n"
    "v_mov_b32_dpp v171, v103 row_ror:8 row_mask:0xf bank_mask:0x3\n"
    "s_cmp_eq_u64 s[22:23], s[44:45]\n"
    "s_cbranch_scc1 .La169_%=\n"
    "s_mov_b32 s83, s52\n"
    ".La168_%=:\n"
    "s_cmp_eq_u32 s83, 0\n"
    "s_cbranch_scc1 .La169_%=\n"
    "s_sleep 10\n"
    "s_sub_u32 s83, s83, 1\n"
    "s_branch .La168_%=\n"
    ".La169_%=:\n"
    "buffer_store_dwordx4 v[144:147], v255, s[72:75], s80 offen sc0 sc1\n"
    "s_cmp_eq_u64 s[22:23], s[44:45]\n"
    "s_cbranch_scc1 .La171_%=\n"
    "s_sleep 80\n"
    ".La171_%=:\n"
    "s_xor_b32 s82, s80, 0x4000\n"
    "buffer_store_dwordx4 v[168:171], v255, s[72:75], s82 offen sc0 sc1\n"
    "buffer_load_dword v6, off, s[44:47], s55 offset:128 sc0 sc1\n"
    "s_waitcnt vmcnt(0) lgkmcnt(0)\n"
    "s_barrier\n"
    "s_cmp_lg_u32 s58, 0\n"
    "s_cbranch_scc1 .La172_%=\n"
    "s_and_b32 s82, s91, 31\n"
    "s_cmp_eq_u32 s82, 31\n"
    "s_cbranch_scc1 .La173_%=\n"
    "v_cmp_lt_u32 vcc, s91, v6\n"
    "s_cbranch_vccnz .La173_%=\n"
    "s_memrealtime s[86:87]\n"
    ".La174_%=:\n"
    "s_sleep 4\n"
    "buffer_load_dword v6, off, s[44:47], s55 offset:128 sc0 sc1\n"
    "s_memrealtime s[88:89]\n"
    "s_waitcnt vmcnt(0) lgkmcnt(0)\n"
    "v_cmp_lt_u32 vcc, s91, v6\n"
    "s_cbranch_vccnz .La173_%=\n"
    "s_sub_u32 s83, s88, s86\n"
    "s_cmp_lt_u32 s83, 2500\n"
    "s_cbranch_scc1 .La174_%=\n"
    "buffer_wbl2 sc0 sc1\n"
    "s_waitcnt vmcnt(0)\n"
    ".La173_%=:\n"
    "s_mov_b64 exec, 1\n"
    "v_mov_b32 v6, s71\n"
    "s_cmp_eq_u64 s[22:23], s[44:45]\n"
    "s_cbranch_scc1 .La175_%=\n"
    "v_mov_b32 v3, 2\n"
    "global_atomic_add v6, v3, s[22:23] sc1\n"
    ".La175_%=:\n"
    "s_cmp_eq_u64 s[20:21], s[44:45]\n"
    "s_cbranch_scc1 .La176_%=\n"
    "v_mov_b32 v2, 2\n"
    "global_atomic_add v6, v2, s[20:21] sc1\n"
    ".La176_%=:\n"
    "s_cmp_eq_u64 s[18:19], s[44:45]\n"
    "s_cbranch_scc1 .La177_%=\n"
    "v_mov_b32 v1, 2\n"
    "global_atomic_add v6, v1, s[18:19] sc1\n"
    ".La177_%=:\n"
    "s_cmp_eq_u64 s[16:17], s[44:45]\n"
    "s_cbranch_scc1 .La178_%=\n"
    "v_mov_b32 v0, 2\n"
    "global_atomic_add v6, v0, s[16:17] sc1\n"
    ".La178_%=:\n"
    "s_mov_b64 exec, -1\n"
    ".La172_%=:\n"
    : "={s90}"(call), "+{s[44:45]}"(pS), "+{s30}"(rk), "+{s[28:29]}"(tabw), "+{s[48:63]}"(sp)
    : "{s[36:37]}"(pX), "{s[40:41]}"(pW), "{s31}"(x), "{v[244:247]}"(v4), "{v[248:255]}"(v8)
    : "memory", "scc", "vcc", "v0", "v1", "v2", "v3", "v4", "v5", "v6", "v7", "v8", "v9", "v10", "v11", "v12", "v13", "v14", "v15", "v16", "v17", "v18", "v19", "v20", "v21", "v22", "v23", "v24", "v25", "v26", "v27", "v28", "v29", "v30", "v31", "v32", "v33", "v34", "v35", "v36", "v37", "v38", "v39", "v40", "v41", "v42", "v43", "v44", "v45", "v46", "v47", "v48", "v49", "v50", "v51", "v52", "v53", "v54", "v55", "v56", "v57", "v58", "v59", "v60", "v61", "v62", "v63", "v64", "v65", "v66", "v67", "v68", "v69", "v70", "v71", "v72", "v73", "v74", "v75", "v76", "v77", "v78", "v79", "v80", "v81", "v82", "v83", "v84", "v85", "v86", "v87", "v88", "v89", "v90", "v91", "v92", "v93", "v94", "v95", "v96", "v97", "v98", "v99", "v100", "v101", "v102", "v103", "v104", "v105", "v106", "v107", "v108", "v109", "v110", "v111", "v112", "v113", "v114", "v115", "v116", "v117", "v118", "v119", "v120", "v121", "v122", "v123", "v124", "v125", "v126", "v127", "v128", "v129", "v130", "v131", "v132", "v133", "v134", "v135", "v136", "v137", "v138", "v139", "v140", "v141", "v142", "v143", "v144", "v145", "v146", "v147", "v148", "v149", "v150", "v151", "v152", "v153", "v154", "v155", "v156", "v157", "v158", "v159", "v160", "v161", "v162", "v163", "v164", "v165", "v166", "v167", "v168", "v169", "v170", "v171", "v172", "v173", "v174", "v175", "v176", "v177", "v178", "v179", "v180", "v181", "v182", "v183", "v184", "v185", "v186", "v187", "v188", "v189", "v190", "v191", "v192", "v193", "v194", "v195", "v196", "v197", "v198", "v199", "v200", "v201", "v202", "v203", "v204", "v205", "v206", "v207", "v208", "v209", "v210", "v211", "v212", "v213", "v214", "v215", "v216", "v217", "v218", "v219", "v220", "v221", "v222", "v223", "v224", "v225", "v226", "v227", "v228", "v229", "v230", "v231", "v232", "v233", "v234", "v235", "v236", "v237", "v238", "v239", "v240", "v241", "v242", "v243", "s16", "s17", "s18", "s19", "s20", "s21", "s22", "s23", "s38", "s39", "s42", "s43", "s46", "s47", "s64", "s65", "s66", "s67", "s68", "s71", "s72", "s73", "s74", "s75", "s76", "s77", "s78", "s79", "s80", "s81", "s82", "s83", "s84", "s85", "s86", "s87", "s88", "s89", "s91");
  const int rank = rk & (NR - 1);
  char* const wsl = (char*)(GAS char*)pS;
  asm volatile("v_mbcnt_lo_u32_b32 %0, -1, 0\n\tv_mbcnt_hi_u32_b32 %0, -1, %0" : "=v"(lane));
  if (wid == 0) {
    if (lane == 0) *(unsigned*)(wsl + WS_EPOCH + bid * 4) = call;
    if (bid == 1) {
      const unsigned want = call * ARRIVALS;
      const unsigned long long t0 = __builtin_amdgcn_s_memrealtime();
      for (;;) {
        unsigned got = __hip_atomic_load((unsigned*)(wsl + WS_ARR + lane * 128), __ATOMIC_RELAXED, __HIP_MEMORY_SCOPE_SYSTEM);
        got += __builtin_amdgcn_update_dpp(0u, got, 0xB1, 0xf, 0xf, false);
        got += __builtin_amdgcn_update_dpp(0u, got, 0x4E, 0xf, 0xf, false);
        got += __builtin_amdgcn_update_dpp(0u, got, 0x141, 0xf, 0xf, false);
        if (__all((lane >> 3) == rank || (int)(got - want) >= 0)) break;
        const unsigned long long dt = __builtin_amdgcn_s_memrealtime() - t0;
        if (dt > 10000ull && (dt > DEADLINE_TICKS || __hip_atomic_load((unsigned*)(wsl + WS_TMO), __ATOMIC_RELAXED, __HIP_MEMORY_SCOPE_AGENT) == call)) {
          if (lane == 0) { __hip_atomic_store((unsigned*)(wsl + WS_TMO), call, __ATOMIC_RELAXED, __HIP_MEMORY_SCOPE_AGENT);
            *(bf16_t*)((char*)a.out[rank] + (unsigned)(rank * M) * (NS * 2) + soffO + ((unsigned)(tabw >> (8 * (NSTAGE - 1))) & 7u) * 256u) = (bf16_t)0x7fc0; }
          break;
        }
        if (dt > 10000ull) __builtin_amdgcn_s_sleep(32);
      }
    }
  }
}

extern "C" void kernel_launch(int rank, int nranks, ncclComm_t comm, hipStream_t stream,
                              void* const* d_in, const int* in_sizes, int n_in,
                              void* d_out, int out_size, const int* in_dtypes, int out_dtype,
                              void* const* d_out_all, void* const* d_ws_all, size_t ws_bytes) {
  if (nranks != NR || (unsigned)rank >= (unsigned)NR || n_in < 2 || in_sizes[0] != M * K || in_sizes[1] != K * N || out_size != NR * M * NS || ws_bytes < WS_END) {
    fprintf(stderr, "kernel_launch: unexpected problem shape\n"); return; }
  Args a;
  a.X = (const bf16_t*)d_in[0]; a.W = (const bf16_t*)d_in[1]; a.rank = rank;
  static const unsigned tab[64] = {0x09090909u, 0x11111111u, 0x19191919u, 0x21212121u, 0x29292929u, 0x31313131u, 0x39393939u, 0x3f2f1f0fu, 0x0d0d0d0du, 0x15151515u, 0x1d1d1d1du, 0x25252525u, 0x2d2d2d2du, 0x35353535u, 0x3d3d3d3du, 0x0f372717u, 0x08080808u, 0x10101010u, 0x18181818u, 0x20202020u, 0x28282828u, 0x30303030u, 0x38383838u, 0x1f0f3727u, 0x0a0a0a0au, 0x12121212u, 0x1a1a1a1au, 0x22222222u, 0x2a2a2a2au, 0x32323232u, 0x3a3a3a3au, 0x173f2f1fu, 0x0b0b0b0bu, 0x13131313u, 0x1b1b1b1bu, 0x23232323u, 0x2b2b2b2bu, 0x33333333u, 0x3b3b3b3bu, 0x27173f2fu, 0x0c0c0c0cu, 0x14141414u, 0x1c1c1c1cu, 0x24242424u, 0x2c2c2c2cu, 0x34343434u, 0x3c3c3c3cu, 0x2f1f0f37u, 0x0e0e0e0eu, 0x16161616u, 0x1e1e1e1eu, 0x26262626u, 0x2e2e2e2eu, 0x36363636u, 0x3e3e3e3eu, 0x3727173fu, 0x07070707u, 0x06060606u, 0x05050505u, 0x04040404u, 0x03030303u, 0x02020202u, 0x01010101u, 0x00000000u};
  const int kSplit[NSTAGE] = K_SPLIT, kBase[NSTAGE] = K_BASE;
  for (int i = 0; i < 64; ++i) {
    const int h = i >> 5, j = i & 31, pm = j & 3; unsigned long long w = 0;
    for (int st = 0; st < NSTAGE; ++st) { const int c = (((j >> 2) + 0 * pm) & 7) + (kSplit[st] == 4 ? 8 * h : 0);
      w |= (unsigned long long)(((tab[kBase[st] + c] >> (8 * pm)) + rank * 8) & 63u) << (8 * st); }
    a.tab[i] = w;
  }
  for (int i = 0; i < 64; ++i) a.tab[i] |= ((a.tab[i ^ 32] >> (8 * (NSTAGE - 1))) & 63u) << (8 * NSTAGE);
  a.wsl = (char*)d_ws_all[rank]; a.pad = 0;
  for (int p = 0; p < NR; ++p) { a.out[p] = (bf16_t*)d_out_all[p]; a.ws[p] = (char*)d_ws_all[p]; }
  HIP_CHECK(hipFuncSetAttribute((const void*)gemm_a2a, hipFuncAttributeMaxDynamicSharedMemorySize, SLOTS * HTB));
  gemm_a2a<<<GRID, 512, SLOTS * HTB, stream>>>(a);
  HIP_CHECK(hipGetLastError());
}
```
